```python
import numpy as np

kernel_name = "gpu_a2a_gemm_m4096_k4096_n16384_e4m3e4m3_dequant"


ABSMAX_RTOL = 0.03

COMPUTE_DTYPE = "float32"
EPILOGUE = "dequant"
IN_QUANT = None
OUT_QUANT = None
IN_FP8 = "e4m3e4m3"
IN_DTYPES = {"x": "e4m3", "w_mat": "e4m3",
             "scale_x": "float32", "scale_w": "float32"}
WORLD_SIZE = 8

def _snap_fp8(x, mant_bits, max_abs, min_normal):
    a = np.abs(x.astype(np.float64))
    _m, _e = np.frexp(a)
    q = np.where(a >= min_normal, np.ldexp(1.0, _e - 1 - mant_bits),
                 min_normal * 2.0 ** -mant_bits)
    snapped = np.minimum(np.round(a / q) * q, max_abs)
    return (np.sign(x) * snapped).astype(np.float32)

def setup_inputs():

    x = _snap_fp8(np.random.randn(4096, 4096).astype(np.float32), 3, 448.0, 2.0 ** -6)
    w_mat = _snap_fp8(np.random.randn(4096, 16384).astype(np.float32), 3, 448.0, 2.0 ** -6)
    scale_x = (0.02 * (1.0 + np.random.rand(1))).astype(np.float32)
    scale_w = (0.02 * (1.0 + np.random.rand(1))).astype(np.float32)
    return {"x": x, "w_mat": w_mat, "scale_x": scale_x, "scale_w": scale_w}

def reference(x, w_mat, scale_x, scale_w):
    acc = (x @ w_mat).astype(np.float32)
    y = (acc * (scale_x * scale_w)).astype(np.float32)
    return y
```

```cpp
#include <hip/hip_runtime.h>
#include <rccl/rccl.h>
#include <cstdio>
#define GEMM_LDS 163840
#define GEMM_STAMP 0
#define GEMM_PW 1
#define GEMM_SP 0
#define GEMM_DW 0
#define GEMM_NVAR 4
#define GEMM_ASM \
  "v_and_b32 v1, 63, %[tid]\n\t" \
  "v_readfirstlane_b32 s4, %[tid]\n\t" \
  "s_load_dwordx8 s[40:47], %[ka], 0x0\n\t" \
  "s_load_dwordx4 s[48:51], %[ka], 0x20\n\t" \
  "s_load_dword s25, %[ka], 0x70\n\t" \
  "s_and_b32 s5, %[bid], 255\n\t" \
  "s_lshr_b32 s4, s4, 6\n\t" \
  "s_lshl_b32 s6, s5, 3\n\t" \
  "s_lshl_b32 s7, s4, 1\n\t" \
  "s_add_u32 s6, s6, s7\n\t" \
  "s_and_b32 s7, s6, 0xfffffffc\n\t" \
  "s_load_dword s27, %[ka], s7 offset:0x78\n\t" \
  "s_lshr_b32 s52, s5, 3\n\t" \
  "s_and_b32 s53, s52, 1\n\t" \
  "s_lshr_b32 s54, s52, 1\n\t" \
  "s_and_b32 s55, s5, 7\n\t" \
  "s_lshl_b32 s55, s55, 4\n\t" \
  "s_add_u32 s55, s55, s54\n\t" \
  "s_lshl_b32 s55, s55, 7\n\t" \
  "s_lshl_b32 s24, s4, 11\n\t" \
  "v_mov_b32 v65, 0\n\t" \
  "s_lshl_b32 s56, s4, 4\n\t" \
  "s_lshl_b32 s57, s53, 6\n\t" \
  "s_add_u32 s56, s56, s57\n\t" \
  "v_lshrrev_b32 v17, 3, v1\n\t" \
  "v_add_u32 v17, s56, v17\n\t" \
  "v_lshlrev_b32 v17, 14, v17\n\t" \
  "s_and_b32 s58, s4, 1\n\t" \
  "s_lshl_b32 s58, s58, 2\n\t" \
  "v_bfe_u32 v18, v1, 4, 2\n\t" \
  "v_or_b32 v18, s58, v18\n\t" \
  "v_and_b32 v19, 7, v1\n\t" \
  "v_xor_b32 v18, v18, v19\n\t" \
  "v_lshl_or_b32 v2, v18, 4, v17\n\t" \
  "v_xor_b32 v3, 0x100000, v2\n\t" \
  "v_and_b32 v17, 15, v1\n\t" \
  "v_lshrrev_b32 v18, 4, v1\n\t" \
  "v_lshlrev_b32 v18, 4, v18\n\t" \
  "v_lshl_or_b32 v7, v17, 9, v18\n\t" \
  "s_mov_b32 s22, 0x200000\n\t" \
  "s_mov_b32 s23, 0x20000\n\t" \
  "s_mov_b32 s11, 0x20000\n\t" \
  "s_sub_u32 s10, 0x4000000, s55\n\t" \
  "s_lshl_b32 s59, s5, 7\n\t" \
  "s_add_u32 s59, s59, 0x401080\n\t" \
  "s_and_b32 s6, s6, 2\n\t" \
  "s_lshl_b32 s6, s6, 3\n\t" \
  "s_waitcnt lgkmcnt(0)\n\t" \
  "s_add_u32 s34, s50, s59\n\t" \
  "s_addc_u32 s35, s51, 0\n\t" \
  "global_load_dword v64, v65, s[34:35] sc1\n\t" \
  "s_lshr_b32 s27, s27, s6\n\t" \
  "s_mov_b32 s20, s40\n\t" \
  "s_and_b32 s21, s41, 0xffff\n\t" \
  "s_and_b32 s60, s27, 7\n\t" \
  "s_add_u32 s61, s60, s25\n\t" \
  "s_add_u32 s61, s61, 1\n\t" \
  "s_and_b32 s61, s61, 7\n\t" \
  "s_lshl_b32 s61, s61, 18\n\t" \
  "s_bfe_u32 s62, s27, 0x50003\n\t" \
  "s_bfe_u32 s63, s27, 0x20008\n\t" \
  "s_lshl_b32 s64, s62, 13\n\t" \
  "s_add_u32 s61, s61, s64\n\t" \
  "s_lshl_b32 s64, s63, 7\n\t" \
  "s_add_u32 s61, s61, s64\n\t" \
  "s_lshr_b32 s64, s62, 4\n\t" \
  "s_lshl_b32 s64, s64, 6\n\t" \
  "s_add_u32 s61, s61, s64\n\t" \
  "v_add_u32 v7, s61, v7\n\t" \
  "v_xor_b32 v8, 64, v7\n\t" \
  "s_add_u32 s8, s42, s55\n\t" \
  "s_addc_u32 s9, s43, 0\n\t" \
  "s_and_b32 s9, s9, 0xffff\n\t" \
  "buffer_load_dwordx4 v[72:75], v7, s[20:23], 0 offen\n\t" \
  "buffer_load_dwordx4 v[76:79], v8, s[20:23], 0 offen\n\t" \
  "s_mov_b64 s[36:37], s[44:45]\n\t" \
  "s_mov_b64 s[38:39], s[46:47]\n\t" \
  "s_mov_b64 s[6:7], s[50:51]\n\t" \
  "s_mov_b64 s[16:17], s[48:49]\n\t" \
  "s_lshl_b32 s47, s25, 23\n\t" \
  "s_mov_b32 s72, s53\n\t" \
  "s_mov_b32 s73, s54\n\t" \
  "s_mov_b32 s74, s55\n\t" \
  "s_mov_b32 s75, s60\n\t" \
  "s_mov_b32 s76, s62\n\t" \
  "s_mov_b32 s77, s63\n\t" \
  "s_add_u32 m0, s24, 0x0\n\t" \
  "s_add_u32 s48, s47, 0x0\n\t" \
  "buffer_load_dwordx4 v2, s[8:11], s48 offen lds\n\t" \
  "s_add_u32 m0, s24, 0x400\n\t" \
  "s_add_u32 s49, s47, 0x20000\n\t" \
  "buffer_load_dwordx4 v2, s[8:11], s49 offen lds\n\t" \
  "s_add_u32 m0, s24, 0x2000\n\t" \
  "s_add_u32 s50, s47, 0x200000\n\t" \
  "buffer_load_dwordx4 v2, s[8:11], s50 offen lds\n\t" \
  "s_add_u32 m0, s24, 0x2400\n\t" \
  "s_add_u32 s51, s47, 0x220000\n\t" \
  "buffer_load_dwordx4 v2, s[8:11], s51 offen lds\n\t" \
  "s_lshl_b32 s78, s25, 18\n\t" \
  "s_lshl_b32 s64, s72, 17\n\t" \
  "s_add_u32 s78, s78, s64\n\t" \
  "s_lshl_b32 s64, s73, 13\n\t" \
  "s_add_u32 s64, s64, s78\n\t" \
  "s_add_u32 s64, s64, s24\n\t" \
  "v_lshl_add_u32 v11, v1, 5, s64\n\t" \
  "buffer_load_dword v62, v11, s[20:23], 0 offen\n\t" \
  "s_and_b32 s79, s73, 3\n\t" \
  "s_getpc_b64 s[80:81]\n\t" \
  "Lpc_%=:\n\t" \
  "s_mov_b32 s64, Lvar_0_%=-Lpc_%=\n\t" \
  "s_cmp_eq_u32 s79, 1\n\t" \
  "s_cselect_b32 s64, Lvar_1_%=-Lpc_%=, s64\n\t" \
  "s_cmp_eq_u32 s79, 2\n\t" \
  "s_cselect_b32 s64, Lvar_2_%=-Lpc_%=, s64\n\t" \
  "s_cmp_eq_u32 s79, 3\n\t" \
  "s_cselect_b32 s64, Lvar_3_%=-Lpc_%=, s64\n\t" \
  "s_add_u32 s80, s80, s64\n\t" \
  "s_addc_u32 s81, s81, 0\n\t" \
  "s_sub_u32 s65, Lfin_%=-Lpc_%=-4, s64\n\t" \
  "s_lshr_b32 s64, s73, 2\n\t" \
  "s_lshl_b32 s64, s64, 1\n\t" \
  "s_add_u32 s64, s64, s72\n\t" \
  "s_lshl_b32 s64, s64, 2\n\t" \
  "s_add_u32 s64, s64, s4\n\t" \
  "s_lshl_b32 s64, s64, 10\n\t" \
  "v_lshl_add_u32 v12, v1, 4, s64\n\t" \
  "v_min_u32 v12, s65, v12\n\t" \
  "global_load_dword v62, v12, s[80:81]\n\t" \
  "s_lshl_b32 s64, s75, 3\n\t" \
  "s_load_dwordx2 s[28:29], %[ka], s64 offset:0x30\n\t" \
  "s_load_dword s66, s[36:37], 0\n\t" \
  "s_load_dword s67, s[38:39], 0\n\t" \
  "s_memrealtime s[58:59]\n\t" \
  "s_setpc_b64 s[80:81]\n\t" \
  "Lvar_0_%=:\n\t" \
  "s_lshl_b32 s64, s4, 15\n\t" \
  "s_add_u32 s64, s64, s78\n\t" \
  "s_lshl_b32 s84, s4, 13\n\t" \
  "v_lshrrev_b32 v17, 3, v1\n\t" \
  "v_lshrrev_b32 v18, 4, v1\n\t" \
  "v_and_b32 v19, 7, v1\n\t" \
  "v_xor_b32 v18, v18, v19\n\t" \
  "v_lshlrev_b32 v18, 4, v18\n\t" \
  "v_lshl_or_b32 v5, v17, 9, v18\n\t" \
  "v_add_u32 v5, s64, v5\n\t" \
  "v_xor_b32 v6, 64, v5\n\t" \
  "v_lshlrev_b32 v4, 4, v1\n\t" \
  "s_lshl_b32 s64, s72, 15\n\t" \
  "s_lshl_b32 s65, s4, 13\n\t" \
  "s_add_u32 s64, s64, s65\n\t" \
  "s_add_u32 s12, s6, s64\n\t" \
  "s_addc_u32 s13, s7, 0\n\t" \
  "s_and_b32 s13, s13, 0xffff\n\t" \
  "s_sub_u32 s14, 0x400000, s64\n\t" \
  "s_mov_b32 s15, 0x20000\n\t" \
  "s_lshl_b32 s64, s72, 24\n\t" \
  "s_lshl_b32 s65, s4, 22\n\t" \
  "s_add_u32 s64, s64, s65\n\t" \
  "s_lshl_b32 s65, s74, 2\n\t" \
  "s_add_u32 s64, s64, s65\n\t" \
  "s_add_u32 s16, s16, s64\n\t" \
  "s_addc_u32 s17, s17, 0\n\t" \
  "s_and_b32 s17, s17, 0xffff\n\t" \
  "s_sub_u32 s18, 0x400000, s65\n\t" \
  "s_mov_b32 s19, 0x20000\n\t" \
  "s_bfe_u32 s26, s27, 0x4000a\n\t" \
  "s_sub_u32 s64, 6, s75\n\t" \
  "s_and_b32 s64, s64, 7\n\t" \
  "s_mul_i32 s65, s77, 7\n\t" \
  "s_add_u32 s65, s65, s64\n\t" \
  "s_add_u32 s65, s65, 4\n\t" \
  "s_cmp_eq_u32 s64, 7\n\t" \
  "s_cselect_b32 s65, s77, s65\n\t" \
  "s_lshl_b32 s65, s65, 16\n\t" \
  "s_lshl_b32 s82, s76, 11\n\t" \
  "s_add_u32 s65, s65, s82\n\t" \
  "v_add_u32 v9, s65, v4\n\t" \
  "s_lshr_b32 s82, s76, 2\n\t" \
  "s_lshl_b32 s82, s82, 9\n\t" \
  "s_lshl_b32 s83, s77, 7\n\t" \
  "s_add_u32 s82, s82, s83\n\t" \
  "s_lshl_b32 s83, s64, 4\n\t" \
  "s_add_u32 s82, s82, s83\n\t" \
  "s_and_b32 s83, s76, 3\n\t" \
  "s_lshl_b32 s83, s83, 2\n\t" \
  "s_add_u32 s82, s82, s83\n\t" \
  "s_add_u32 s82, s82, 0x400000\n\t" \
  "s_lshl_b32 s64, s72, 2\n\t" \
  "s_add_u32 s64, s64, s4\n\t" \
  "s_lshl_b32 s64, s64, 9\n\t" \
  "s_add_u32 s64, s64, 0x400000\n\t" \
  "v_lshl_add_u32 v10, v1, 4, s64\n\t" \
  "v_cmp_lt_u32 vcc, 31, v1\n\t" \
  "v_mov_b32 v18, 0x401000\n\t" \
  "s_nop 0\n\t" \
  "v_cndmask_b32 v10, v10, v18, vcc\n\t" \
  "v_and_b32 v17, 15, v1\n\t" \
  "v_lshrrev_b32 v18, 4, v1\n\t" \
  "v_lshlrev_b32 v19, 4, v18\n\t" \
  "v_lshl_or_b32 v13, v17, 16, v19\n\t" \
  "v_lshl_add_u32 v15, v17, 2, v18\n\t" \
  "v_lshlrev_b32 v15, 2, v15\n\t" \
  "v_lshrrev_b32 v19, 1, v17\n\t" \
  "v_lshl_add_u32 v19, v18, 4, v19\n\t" \
  "v_lshlrev_b32 v19, 7, v19\n\t" \
  "v_bfe_u32 v20, v1, 2, 3\n\t" \
  "v_lshl_or_b32 v19, v20, 4, v19\n\t" \
  "v_and_b32 v20, 1, v1\n\t" \
  "v_lshl_or_b32 v14, v20, 3, v19\n\t" \
  "v_lshrrev_b32 v17, 2, v1\n\t" \
  "v_min_u32 v17, 6, v17\n\t" \
  "v_lshlrev_b32 v17, 16, v17\n\t" \
  "v_and_b32 v18, 3, v1\n\t" \
  "v_lshl_or_b32 v17, v18, 7, v17\n\t" \
  "s_lshl_b32 s64, s73, 9\n\t" \
  "v_add_u32 v16, s64, v17\n\t" \
  "v_and_b32 v17, 15, v1\n\t" \
  "v_lshrrev_b32 v18, 4, v1\n\t" \
  "s_lshl_b32 s64, s72, 2\n\t" \
  "v_add_u32 v18, s64, v18\n\t" \
  "v_lshrrev_b32 v19, 1, v17\n\t" \
  "v_xor_b32 v18, v18, v19\n\t" \
  "v_lshlrev_b32 v18, 4, v18\n\t" \
  "v_lshl_or_b32 v17, v17, 7, v18\n\t" \
  "s_add_u32 s64, s84, 0x10000\n\t" \
  "v_add_u32 v20, s64, v17\n\t" \
  "v_xor_b32 v21, 64, v20\n\t" \
  "s_add_u32 s64, s84, 0x20000\n\t" \
  "v_add_u32 v22, s64, v17\n\t" \
  "v_xor_b32 v23, 64, v22\n\t" \
  "s_add_u32 s40, s25, 1\n\t" \
  "s_and_b32 s40, s40, 7\n\t" \
  "s_lshl_b32 s40, s40, 23\n\t" \
  "s_add_u32 s41, s25, 2\n\t" \
  "s_and_b32 s41, s41, 7\n\t" \
  "s_lshl_b32 s41, s41, 23\n\t" \
  "s_add_u32 s42, s25, 3\n\t" \
  "s_and_b32 s42, s42, 7\n\t" \
  "s_lshl_b32 s42, s42, 23\n\t" \
  "s_add_u32 s43, s25, 4\n\t" \
  "s_and_b32 s43, s43, 7\n\t" \
  "s_lshl_b32 s43, s43, 23\n\t" \
  "s_add_u32 s44, s25, 5\n\t" \
  "s_and_b32 s44, s44, 7\n\t" \
  "s_lshl_b32 s44, s44, 23\n\t" \
  "s_add_u32 s45, s25, 6\n\t" \
  "s_and_b32 s45, s45, 7\n\t" \
  "s_lshl_b32 s45, s45, 23\n\t" \
  "s_add_u32 s46, s25, 7\n\t" \
  "s_and_b32 s46, s46, 7\n\t" \
  "s_lshl_b32 s46, s46, 23\n\t" \
  "v_xor_b32 v80, 0x0, v14\n\t" \
  "v_xor_b32 v81, 0x10, v14\n\t" \
  "v_xor_b32 v82, 0x20, v14\n\t" \
  "v_xor_b32 v83, 0x30, v14\n\t" \
  "v_xor_b32 v84, 0x40, v14\n\t" \
  "v_xor_b32 v85, 0x50, v14\n\t" \
  "v_xor_b32 v86, 0x60, v14\n\t" \
  "v_xor_b32 v87, 0x70, v14\n\t" \
  "v_xor_b32 v88, 0x10000, v14\n\t" \
  "v_xor_b32 v89, 0x10010, v14\n\t" \
  "v_xor_b32 v90, 0x10020, v14\n\t" \
  "v_xor_b32 v91, 0x10030, v14\n\t" \
  "v_xor_b32 v92, 0x10040, v14\n\t" \
  "v_xor_b32 v93, 0x10050, v14\n\t" \
  "v_xor_b32 v94, 0x10060, v14\n\t" \
  "v_xor_b32 v95, 0x10070, v14\n\t" \
  "v_xor_b32 v96, 0x20000, v14\n\t" \
  "v_xor_b32 v97, 0x20010, v14\n\t" \
  "v_xor_b32 v98, 0x20020, v14\n\t" \
  "v_xor_b32 v99, 0x20030, v14\n\t" \
  "v_xor_b32 v100, 0x20040, v14\n\t" \
  "v_xor_b32 v101, 0x20050, v14\n\t" \
  "v_xor_b32 v102, 0x20060, v14\n\t" \
  "v_xor_b32 v103, 0x20070, v14\n\t" \
  "s_mov_b32 s56, 0\n\t" \
  "s_mov_b32 s63, 0\n\t" \
  "s_waitcnt lgkmcnt(0)\n\t" \
  "s_add_u32 s30, s28, s82\n\t" \
  "s_addc_u32 s31, s29, 0\n\t" \
  "s_waitcnt vmcnt(8)\n\t" \
  "v_add_u32 v64, 1, v64\n\t" \
  "s_nop 0\n\t" \
  "v_readfirstlane_b32 s57, v64\n\t" \
  "s_and_b32 s68, s57, 1\n\t" \
  "s_lshl_b32 s68, s68, 21\n\t" \
  "v_add_u32 v63, s68, v9\n\t" \
  "s_waitcnt lgkmcnt(0)\n\t" \
  "s_cmp_lg_u32 s26, 0\n\t" \
  "s_cbranch_scc1 Lns0_0_%=\n\t" \
  "s_waitcnt vmcnt(6)\n\t" \
  "global_store_dwordx4 v63, v[72:75], s[28:29] sc0 sc1\n\t" \
  "global_store_dwordx4 v63, v[76:79], s[28:29] offset:1024 sc0 sc1\n\t" \
  "s_branch Lms0_0_%=\n\t" \
  "Lns0_0_%=:\n\t" \
  "buffer_load_dword v62, v65, s[20:23], s22 offen\n\t" \
  "buffer_load_dword v62, v65, s[20:23], s22 offen\n\t" \
  "Lms0_0_%=:\n\t" \
  "s_add_u32 m0, s24, 0x4000\n\t" \
  "s_add_u32 s52, s47, 0x400000\n\t" \
  "buffer_load_dwordx4 v2, s[8:11], s52 offen lds\n\t" \
  "s_add_u32 m0, s24, 0x4400\n\t" \
  "s_add_u32 s53, s47, 0x420000\n\t" \
  "buffer_load_dwordx4 v2, s[8:11], s53 offen lds\n\t" \
  "s_add_u32 m0, s24, 0x6000\n\t" \
  "s_add_u32 s54, s47, 0x600000\n\t" \
  "buffer_load_dwordx4 v2, s[8:11], s54 offen lds\n\t" \
  "s_add_u32 m0, s24, 0x6400\n\t" \
  "s_add_u32 s55, s47, 0x620000\n\t" \
  "buffer_load_dwordx4 v2, s[8:11], s55 offen lds\n\t" \
  "s_add_u32 m0, s24, 0x8000\n\t" \
  "s_add_u32 s48, s40, 0x0\n\t" \
  "buffer_load_dwordx4 v2, s[8:11], s48 offen lds\n\t" \
  "s_add_u32 m0, s24, 0x8400\n\t" \
  "s_add_u32 s49, s40, 0x20000\n\t" \
  "buffer_load_dwordx4 v2, s[8:11], s49 offen lds\n\t" \
  "s_add_u32 m0, s24, 0xa000\n\t" \
  "s_add_u32 s50, s41, 0x0\n\t" \
  "buffer_load_dwordx4 v2, s[8:11], s50 offen lds\n\t" \
  "s_add_u32 m0, s24, 0xa400\n\t" \
  "s_add_u32 s51, s41, 0x20000\n\t" \
  "buffer_load_dwordx4 v2, s[8:11], s51 offen lds\n\t" \
  "s_add_u32 m0, s24, 0x18000\n\t" \
  "s_add_u32 s52, s47, 0x0\n\t" \
  "buffer_load_dwordx4 v3, s[8:11], s52 offen lds\n\t" \
  "s_add_u32 m0, s24, 0x18400\n\t" \
  "s_add_u32 s53, s47, 0x20000\n\t" \
  "buffer_load_dwordx4 v3, s[8:11], s53 offen lds\n\t" \
  "s_add_u32 m0, s84, 0x10000\n\t" \
  "s_mov_b32 s54, 0x0\n\t" \
  "buffer_load_dwordx4 v5, s[20:23], s54 offen lds\n\t" \
  "s_add_u32 m0, s84, 0x10400\n\t" \
  "s_mov_b32 s55, 0x1000\n\t" \
  "buffer_load_dwordx4 v6, s[20:23], s55 offen lds\n\t" \
  "s_add_u32 m0, s84, 0x10800\n\t" \
  "s_mov_b32 s48, 0x2000\n\t" \
  "buffer_load_dwordx4 v5, s[20:23], s48 offen lds\n\t" \
  "s_add_u32 m0, s84, 0x10c00\n\t" \
  "s_mov_b32 s49, 0x3000\n\t" \
  "buffer_load_dwordx4 v6, s[20:23], s49 offen lds\n\t" \
  "s_add_u32 m0, s84, 0x11000\n\t" \
  "s_mov_b32 s50, 0x4000\n\t" \
  "buffer_load_dwordx4 v5, s[20:23], s50 offen lds\n\t" \
  "s_add_u32 m0, s84, 0x11400\n\t" \
  "s_mov_b32 s51, 0x5000\n\t" \
  "buffer_load_dwordx4 v6, s[20:23], s51 offen lds\n\t" \
  "s_add_u32 m0, s84, 0x11800\n\t" \
  "s_mov_b32 s52, 0x6000\n\t" \
  "buffer_load_dwordx4 v5, s[20:23], s52 offen lds\n\t" \
  "s_add_u32 m0, s84, 0x11c00\n\t" \
  "s_mov_b32 s53, 0x7000\n\t" \
  "buffer_load_dwordx4 v6, s[20:23], s53 offen lds\n\t" \
  "s_cmp_lg_u32 s26, 0\n\t" \
  "s_cbranch_scc1 Lnf0_0_%=\n\t" \
  "s_waitcnt vmcnt(18)\n\t" \
  "s_mov_b64 exec, 1\n\t" \
  "global_store_dword v65, v64, s[30:31] sc0 sc1\n\t" \
  "s_mov_b64 exec, -1\n\t" \
  "s_branch Lmf0_0_%=\n\t" \
  "Lnf0_0_%=:\n\t" \
  "buffer_load_dword v62, v65, s[20:23], s22 offen\n\t" \
  "Lmf0_0_%=:\n\t" \
  "s_add_u32 m0, s24, 0x1a000\n\t" \
  "s_add_u32 s54, s47, 0x200000\n\t" \
  "buffer_load_dwordx4 v3, s[8:11], s54 offen lds\n\t" \
  "s_add_u32 m0, s24, 0x1a400\n\t" \
  "s_add_u32 s55, s47, 0x220000\n\t" \
  "buffer_load_dwordx4 v3, s[8:11], s55 offen lds\n\t" \
  "s_add_u32 m0, s84, 0x20000\n\t" \
  "s_mov_b32 s48, 0x80\n\t" \
  "buffer_load_dwordx4 v5, s[20:23], s48 offen lds\n\t" \
  "s_add_u32 m0, s84, 0x20400\n\t" \
  "s_mov_b32 s49, 0x1080\n\t" \
  "buffer_load_dwordx4 v6, s[20:23], s49 offen lds\n\t" \
  "s_add_u32 m0, s84, 0x20800\n\t" \
  "s_mov_b32 s50, 0x2080\n\t" \
  "buffer_load_dwordx4 v5, s[20:23], s50 offen lds\n\t" \
  "s_add_u32 m0, s84, 0x20c00\n\t" \
  "s_mov_b32 s51, 0x3080\n\t" \
  "buffer_load_dwordx4 v6, s[20:23], s51 offen lds\n\t" \
  "s_add_u32 m0, s84, 0x21000\n\t" \
  "s_mov_b32 s52, 0x4080\n\t" \
  "buffer_load_dwordx4 v5, s[20:23], s52 offen lds\n\t" \
  "s_add_u32 m0, s84, 0x21400\n\t" \
  "s_mov_b32 s53, 0x5080\n\t" \
  "buffer_load_dwordx4 v6, s[20:23], s53 offen lds\n\t" \
  "s_add_u32 m0, s84, 0x21800\n\t" \
  "s_mov_b32 s54, 0x6080\n\t" \
  "buffer_load_dwordx4 v5, s[20:23], s54 offen lds\n\t" \
  "s_add_u32 m0, s84, 0x21c00\n\t" \
  "s_mov_b32 s55, 0x7080\n\t" \
  "buffer_load_dwordx4 v6, s[20:23], s55 offen lds\n\t" \
  "s_waitcnt vmcnt(11)\n\t" \
  "ds_read_b128 a[0:3], v20 offset:0\n\t" \
  "ds_read_b128 a[4:7], v21 offset:0\n\t" \
  "ds_read_b128 a[8:11], v20 offset:2048\n\t" \
  "ds_read_b128 a[12:15], v21 offset:2048\n\t" \
  "ds_read_b128 a[16:19], v20 offset:4096\n\t" \
  "ds_read_b128 a[20:23], v21 offset:4096\n\t" \
  "ds_read_b128 a[24:27], v20 offset:6144\n\t" \
  "ds_read_b128 a[28:31], v21 offset:6144\n\t" \
  "s_waitcnt lgkmcnt(0)\n\t" \
  "s_waitcnt lgkmcnt(0)\n\t" \
  "s_barrier\n\t" \
  "s_mov_b64 exec, 1\n\t" \
  "global_store_dword v65, v64, s[34:35] sc1\n\t" \
  "s_mov_b64 exec, -1\n\t" \
  "ds_read_b64_tr_b8 v[104:105], v80 offset:0\n\t" \
  "ds_read_b64_tr_b8 v[106:107], v80 offset:1024\n\t" \
  "ds_read_b64_tr_b8 v[108:109], v88 offset:32768\n\t" \
  "ds_read_b64_tr_b8 v[110:111], v88 offset:33792\n\t" \
  "ds_read_b64_tr_b8 v[112:113], v81 offset:0\n\t" \
  "ds_read_b64_tr_b8 v[114:115], v81 offset:1024\n\t" \
  "ds_read_b64_tr_b8 v[116:117], v89 offset:32768\n\t" \
  "ds_read_b64_tr_b8 v[118:119], v89 offset:33792\n\t" \
  "s_waitcnt lgkmcnt(4)\n\t" \
  "v_mfma_f32_16x16x128_f8f6f4 v[128:131], v[104:111], a[0:7], 0\n\t" \
  "ds_read_b64_tr_b8 v[120:121], v82 offset:0\n\t" \
  "s_add_u32 m0, s24, 0x1c000\n\t" \
  "s_add_u32 s48, s47, 0x400000\n\t" \
  "buffer_load_dwordx4 v3, s[8:11], s48 offen lds\n\t" \
  "s_mov_b32 exec_lo, 0x7f\n\t" \
  "s_mov_b32 exec_hi, 0\n\t" \
  "global_load_dwordx4 v[68:71], v10, s[6:7] sc0 sc1\n\t" \
  "s_mov_b64 exec, -1\n\t" \
  "v_mfma_f32_16x16x128_f8f6f4 v[132:135], v[104:111], a[8:15], 0\n\t" \
  "ds_read_b64_tr_b8 v[122:123], v82 offset:1024\n\t" \
  "v_mfma_f32_16x16x128_f8f6f4 v[136:139], v[104:111], a[16:23], 0\n\t" \
  "ds_read_b64_tr_b8 v[124:125], v90 offset:32768\n\t" \
  "s_add_u32 m0, s24, 0x1c400\n\t" \
  "s_add_u32 s49, s47, 0x420000\n\t" \
  "buffer_load_dwordx4 v3, s[8:11], s49 offen lds\n\t" \
  "v_mfma_f32_16x16x128_f8f6f4 v[140:143], v[104:111], a[24:31], 0\n\t" \
  "ds_read_b64_tr_b8 v[126:127], v90 offset:33792\n\t" \
  "s_waitcnt lgkmcnt(4)\n\t" \
  "v_mfma_f32_16x16x128_f8f6f4 v[156:159], v[112:119], a[24:31], 0\n\t" \
  "ds_read_b64_tr_b8 v[104:105], v83 offset:0\n\t" \
  "s_add_u32 m0, s84, 0x10000\n\t" \
  "s_mov_b32 s50, 0x100\n\t" \
  "buffer_load_dwordx4 v5, s[20:23], s50 offen lds\n\t" \
  "v_mfma_f32_16x16x128_f8f6f4 v[152:155], v[112:119], a[16:23], 0\n\t" \
  "ds_read_b64_tr_b8 v[106:107], v83 offset:1024\n\t" \
  "v_mfma_f32_16x16x128_f8f6f4 v[148:151], v[112:119], a[8:15], 0\n\t" \
  "ds_read_b64_tr_b8 v[108:109], v91 offset:32768\n\t" \
  "s_add_u32 m0, s84, 0x10400\n\t" \
  "s_mov_b32 s51, 0x1100\n\t" \
  "buffer_load_dwordx4 v6, s[20:23], s51 offen lds\n\t" \
  "v_mfma_f32_16x16x128_f8f6f4 v[144:147], v[112:119], a[0:7], 0\n\t" \
  "ds_read_b64_tr_b8 v[110:111], v91 offset:33792\n\t" \
  "s_waitcnt lgkmcnt(4)\n\t" \
  "v_mfma_f32_16x16x128_f8f6f4 v[160:163], v[120:127], a[0:7], 0\n\t" \
  "ds_read_b64_tr_b8 v[112:113], v84 offset:0\n\t" \
  "s_add_u32 m0, s84, 0x10800\n\t" \
  "s_mov_b32 s52, 0x2100\n\t" \
  "buffer_load_dwordx4 v5, s[20:23], s52 offen lds\n\t" \
  "s_waitcnt vmcnt(7)\n\t" \
  "ds_read_b128 a[32:35], v22 offset:0\n\t" \
  "v_mfma_f32_16x16x128_f8f6f4 v[164:167], v[120:127], a[8:15], 0\n\t" \
  "ds_read_b64_tr_b8 v[114:115], v84 offset:1024\n\t" \
  "ds_read_b128 a[36:39], v23 offset:0\n\t" \
  "v_mfma_f32_16x16x128_f8f6f4 v[168:171], v[120:127], a[16:23], 0\n\t" \
  "ds_read_b64_tr_b8 v[116:117], v92 offset:32768\n\t" \
  "s_add_u32 m0, s84, 0x10c00\n\t" \
  "s_mov_b32 s53, 0x3100\n\t" \
  "buffer_load_dwordx4 v6, s[20:23], s53 offen lds\n\t" \
  "ds_read_b128 a[40:43], v22 offset:2048\n\t" \
  "v_mfma_f32_16x16x128_f8f6f4 v[172:175], v[120:127], a[24:31], 0\n\t" \
  "ds_read_b64_tr_b8 v[118:119], v92 offset:33792\n\t" \
  "ds_read_b128 a[44:47], v23 offset:2048\n\t" \
  "s_waitcnt lgkmcnt(8)\n\t" \
  "v_mfma_f32_16x16x128_f8f6f4 v[188:191], v[104:111], a[24:31], 0\n\t" \
  "ds_read_b64_tr_b8 v[120:121], v85 offset:0\n\t" \
  "s_add_u32 m0, s84, 0x11000\n\t" \
  "s_mov_b32 s54, 0x4100\n\t" \
  "buffer_load_dwordx4 v5, s[20:23], s54 offen lds\n\t" \
  "ds_read_b128 a[48:51], v22 offset:4096\n\t" \
  "v_mfma_f32_16x16x128_f8f6f4 v[184:187], v[104:111], a[16:23], 0\n\t" \
  "ds_read_b64_tr_b8 v[122:123], v85 offset:1024\n\t" \
  "ds_read_b128 a[52:55], v23 offset:4096\n\t" \
  "v_mfma_f32_16x16x128_f8f6f4 v[180:183], v[104:111], a[8:15], 0\n\t" \
  "ds_read_b64_tr_b8 v[124:125], v93 offset:32768\n\t" \
  "s_add_u32 m0, s84, 0x11400\n\t" \
  "s_mov_b32 s55, 0x5100\n\t" \
  "buffer_load_dwordx4 v6, s[20:23], s55 offen lds\n\t" \
  "ds_read_b128 a[56:59], v22 offset:6144\n\t" \
  "v_mfma_f32_16x16x128_f8f6f4 v[176:179], v[104:111], a[0:7], 0\n\t" \
  "ds_read_b64_tr_b8 v[126:127], v93 offset:33792\n\t" \
  "s_waitcnt lgkmcnt(14)\n\t" \
  "ds_read_b128 a[60:63], v23 offset:6144\n\t" \
  "s_waitcnt lgkmcnt(9)\n\t" \
  "v_mfma_f32_16x16x128_f8f6f4 v[192:195], v[112:119], a[0:7], 0\n\t" \
  "ds_read_b64_tr_b8 v[104:105], v86 offset:0\n\t" \
  "s_add_u32 m0, s84, 0x11800\n\t" \
  "s_mov_b32 s48, 0x6100\n\t" \
  "buffer_load_dwordx4 v5, s[20:23], s48 offen lds\n\t" \
  "v_mfma_f32_16x16x128_f8f6f4 v[196:199], v[112:119], a[8:15], 0\n\t" \
  "ds_read_b64_tr_b8 v[106:107], v86 offset:1024\n\t" \
  "v_mfma_f32_16x16x128_f8f6f4 v[200:203], v[112:119], a[16:23], 0\n\t" \
  "ds_read_b64_tr_b8 v[108:109], v94 offset:32768\n\t" \
  "s_add_u32 m0, s84, 0x11c00\n\t" \
  "s_mov_b32 s49, 0x7100\n\t" \
  "buffer_load_dwordx4 v6, s[20:23], s49 offen lds\n\t" \
  "v_mfma_f32_16x16x128_f8f6f4 v[204:207], v[112:119], a[24:31], 0\n\t" \
  "ds_read_b64_tr_b8 v[110:111], v94 offset:33792\n\t" \
  "s_waitcnt lgkmcnt(5)\n\t" \
  "v_mfma_f32_16x16x128_f8f6f4 v[220:223], v[120:127], a[24:31], 0\n\t" \
  "ds_read_b64_tr_b8 v[112:113], v87 offset:0\n\t" \
  "s_add_u32 m0, s24, 0x1e000\n\t" \
  "s_add_u32 s50, s47, 0x600000\n\t" \
  "buffer_load_dwordx4 v3, s[8:11], s50 offen lds\n\t" \
  "v_mfma_f32_16x16x128_f8f6f4 v[216:219], v[120:127], a[16:23], 0\n\t" \
  "ds_read_b64_tr_b8 v[114:115], v87 offset:1024\n\t" \
  "v_mfma_f32_16x16x128_f8f6f4 v[212:215], v[120:127], a[8:15], 0\n\t" \
  "ds_read_b64_tr_b8 v[116:117], v95 offset:32768\n\t" \
  "s_add_u32 m0, s24, 0x1e400\n\t" \
  "s_add_u32 s51, s47, 0x620000\n\t" \
  "buffer_load_dwordx4 v3, s[8:11], s51 offen lds\n\t" \
  "v_mfma_f32_16x16x128_f8f6f4 v[208:211], v[120:127], a[0:7], 0\n\t" \
  "ds_read_b64_tr_b8 v[118:119], v95 offset:33792\n\t" \
  "s_waitcnt lgkmcnt(0)\n\t" \
  "s_barrier\n\t" \
  "v_mfma_f32_16x16x128_f8f6f4 v[224:227], v[104:111], a[0:7], 0\n\t" \
  "ds_read_b64_tr_b8 v[120:121], v80 offset:8192\n\t" \
  "s_add_u32 m0, s84, 0x20000\n\t" \
  "s_mov_b32 s52, 0x180\n\t" \
  "buffer_load_dwordx4 v5, s[20:23], s52 offen lds\n\t" \
  "v_mfma_f32_16x16x128_f8f6f4 v[228:231], v[104:111], a[8:15], 0\n\t" \
  "ds_read_b64_tr_b8 v[122:123], v80 offset:9216\n\t" \
  "s_add_u32 m0, s84, 0x20400\n\t" \
  "s_mov_b32 s53, 0x1180\n\t" \
  "buffer_load_dwordx4 v6, s[20:23], s53 offen lds\n\t" \
  "v_mfma_f32_16x16x128_f8f6f4 v[232:235], v[104:111], a[16:23], 0\n\t" \
  "ds_read_b64_tr_b8 v[124:125], v88 offset:40960\n\t" \
  "s_add_u32 m0, s84, 0x20800\n\t" \
  "s_mov_b32 s54, 0x2180\n\t" \
  "buffer_load_dwordx4 v5, s[20:23], s54 offen lds\n\t" \
  "v_mfma_f32_16x16x128_f8f6f4 v[236:239], v[104:111], a[24:31], 0\n\t" \
  "ds_read_b64_tr_b8 v[126:127], v88 offset:41984\n\t" \
  "s_add_u32 m0, s84, 0x20c00\n\t" \
  "s_mov_b32 s55, 0x3180\n\t" \
  "buffer_load_dwordx4 v6, s[20:23], s55 offen lds\n\t" \
  "v_mfma_f32_16x16x128_f8f6f4 v[252:255], v[112:119], a[24:31], 0\n\t" \
  "ds_read_b64_tr_b8 v[104:105], v81 offset:8192\n\t" \
  "s_add_u32 m0, s84, 0x21000\n\t" \
  "s_mov_b32 s48, 0x4180\n\t" \
  "buffer_load_dwordx4 v5, s[20:23], s48 offen lds\n\t" \
  "v_mfma_f32_16x16x128_f8f6f4 v[248:251], v[112:119], a[16:23], 0\n\t" \
  "ds_read_b64_tr_b8 v[106:107], v81 offset:9216\n\t" \
  "s_add_u32 m0, s84, 0x21400\n\t" \
  "s_mov_b32 s49, 0x5180\n\t" \
  "buffer_load_dwordx4 v6, s[20:23], s49 offen lds\n\t" \
  "v_mfma_f32_16x16x128_f8f6f4 v[244:247], v[112:119], a[8:15], 0\n\t" \
  "ds_read_b64_tr_b8 v[108:109], v89 offset:40960\n\t" \
  "s_add_u32 m0, s84, 0x21800\n\t" \
  "s_mov_b32 s50, 0x6180\n\t" \
  "buffer_load_dwordx4 v5, s[20:23], s50 offen lds\n\t" \
  "v_mfma_f32_16x16x128_f8f6f4 v[240:243], v[112:119], a[0:7], 0\n\t" \
  "ds_read_b64_tr_b8 v[110:111], v89 offset:41984\n\t" \
  "s_add_u32 m0, s84, 0x21c00\n\t" \
  "s_mov_b32 s51, 0x7180\n\t" \
  "buffer_load_dwordx4 v6, s[20:23], s51 offen lds\n\t" \
  "s_waitcnt lgkmcnt(4)\n\t" \
  "v_mfma_f32_16x16x128_f8f6f4 v[128:131], v[120:127], a[32:39], v[128:131]\n\t" \
  "ds_read_b64_tr_b8 v[112:113], v82 offset:8192\n\t" \
  "v_mfma_f32_16x16x128_f8f6f4 v[132:135], v[120:127], a[40:47], v[132:135]\n\t" \
  "ds_read_b64_tr_b8 v[114:115], v82 offset:9216\n\t" \
  "v_mfma_f32_16x16x128_f8f6f4 v[136:139], v[120:127], a[48:55], v[136:139]\n\t" \
  "ds_read_b64_tr_b8 v[116:117], v90 offset:40960\n\t" \
  "v_mfma_f32_16x16x128_f8f6f4 v[140:143], v[120:127], a[56:63], v[140:143]\n\t" \
  "ds_read_b64_tr_b8 v[118:119], v90 offset:41984\n\t" \
  "s_waitcnt lgkmcnt(4)\n\t" \
  "v_mfma_f32_16x16x128_f8f6f4 v[156:159], v[104:111], a[56:63], v[156:159]\n\t" \
  "ds_read_b64_tr_b8 v[120:121], v83 offset:8192\n\t" \
  "s_waitcnt vmcnt(10)\n\t" \
  "ds_read_b128 a[64:67], v20 offset:0\n\t" \
  "v_mfma_f32_16x16x128_f8f6f4 v[152:155], v[104:111], a[48:55], v[152:155]\n\t" \
  "ds_read_b64_tr_b8 v[122:123], v83 offset:9216\n\t" \
  "ds_read_b128 a[68:71], v21 offset:0\n\t" \
  "v_mfma_f32_16x16x128_f8f6f4 v[148:151], v[104:111], a[40:47], v[148:151]\n\t" \
  "ds_read_b64_tr_b8 v[124:125], v91 offset:40960\n\t" \
  "ds_read_b128 a[72:75], v20 offset:2048\n\t" \
  "v_mfma_f32_16x16x128_f8f6f4 v[144:147], v[104:111], a[32:39], v[144:147]\n\t" \
  "ds_read_b64_tr_b8 v[126:127], v91 offset:41984\n\t" \
  "ds_read_b128 a[76:79], v21 offset:2048\n\t" \
  "s_waitcnt lgkmcnt(8)\n\t" \
  "v_mfma_f32_16x16x128_f8f6f4 v[160:163], v[112:119], a[32:39], v[160:163]\n\t" \
  "ds_read_b64_tr_b8 v[104:105], v84 offset:8192\n\t" \
  "ds_read_b128 a[80:83], v20 offset:4096\n\t" \
  "v_min_u32 v68, v68, v69\n\t" \
  "v_min3_u32 v68, v68, v70, v71\n\t" \
  "v_cmp_le_u32 vcc, s57, v68\n\t" \
  "s_and_b32 s64, vcc_lo, 0x7f\n\t" \
  "s_or_b32 s56, s56, s64\n\t" \
  "s_and_b32 s64, s56, 0x1\n\t" \
  "s_cmp_eq_u32 s64, 0x1\n\t" \
  "s_cbranch_scc1 Lok0_0_%=\n\t" \
  "s_mov_b32 s62, 0x1\n\t" \
  "s_mov_b32 s69, 0x7f\n\t" \
  "s_call_b64 s[60:61], Lpoll_0_%=\n\t" \
  "Lok0_0_%=:\n\t" \
  "s_add_u32 s52, s68, 0x40000\n\t" \
  "buffer_load_dwordx4 a[128:131], v4, s[12:15], s52 offen offset:0 sc0 sc1\n\t" \
  "v_mfma_f32_16x16x128_f8f6f4 v[164:167], v[112:119], a[40:47], v[164:167]\n\t" \
  "ds_read_b64_tr_b8 v[106:107], v84 offset:9216\n\t" \
  "ds_read_b128 a[84:87], v21 offset:4096\n\t" \
  "v_mfma_f32_16x16x128_f8f6f4 v[168:171], v[112:119], a[48:55], v[168:171]\n\t" \
  "ds_read_b64_tr_b8 v[108:109], v92 offset:40960\n\t" \
  "ds_read_b128 a[88:91], v20 offset:6144\n\t" \
  "s_add_u32 s53, s68, 0x40000\n\t" \
  "buffer_load_dwordx4 a[132:135], v4, s[12:15], s53 offen offset:1024 sc0 sc1\n\t" \
  "v_mfma_f32_16x16x128_f8f6f4 v[172:175], v[112:119], a[56:63], v[172:175]\n\t" \
  "ds_read_b64_tr_b8 v[110:111], v92 offset:41984\n\t" \
  "s_waitcnt lgkmcnt(14)\n\t" \
  "ds_read_b128 a[92:95], v21 offset:6144\n\t" \
  "s_waitcnt lgkmcnt(9)\n\t" \
  "v_mfma_f32_16x16x128_f8f6f4 v[188:191], v[120:127], a[56:63], v[188:191]\n\t" \
  "ds_read_b64_tr_b8 v[112:113], v85 offset:8192\n\t" \
  "s_add_u32 s54, s68, 0x40000\n\t" \
  "buffer_load_dwordx4 a[136:139], v4, s[12:15], s54 offen offset:2048 sc0 sc1\n\t" \
  "v_mfma_f32_16x16x128_f8f6f4 v[184:187], v[120:127], a[48:55], v[184:187]\n\t" \
  "ds_read_b64_tr_b8 v[114:115], v85 offset:9216\n\t" \
  "v_mfma_f32_16x16x128_f8f6f4 v[180:183], v[120:127], a[40:47], v[180:183]\n\t" \
  "ds_read_b64_tr_b8 v[116:117], v93 offset:40960\n\t" \
  "s_waitcnt vmcnt(3)\n\t" \
  "ds_read_b128 a[96:99], v22 offset:0\n\t" \
  "s_add_u32 s55, s68, 0x40000\n\t" \
  "buffer_load_dwordx4 a[140:143], v4, s[12:15], s55 offen offset:3072 sc0 sc1\n\t" \
  "v_mfma_f32_16x16x128_f8f6f4 v[176:179], v[120:127], a[32:39], v[176:179]\n\t" \
  "ds_read_b64_tr_b8 v[118:119], v93 offset:41984\n\t" \
  "ds_read_b128 a[100:103], v23 offset:0\n\t" \
  "s_waitcnt lgkmcnt(7)\n\t" \
  "v_mfma_f32_16x16x128_f8f6f4 v[192:195], v[104:111], a[32:39], v[192:195]\n\t" \
  "ds_read_b64_tr_b8 v[120:121], v86 offset:8192\n\t" \
  "ds_read_b128 a[104:107], v22 offset:2048\n\t" \
  "s_add_u32 s48, s68, 0x41000\n\t" \
  "buffer_load_dwordx4 a[144:147], v4, s[12:15], s48 offen offset:0 sc0 sc1\n\t" \
  "v_mfma_f32_16x16x128_f8f6f4 v[196:199], v[104:111], a[40:47], v[196:199]\n\t" \
  "ds_read_b64_tr_b8 v[122:123], v86 offset:9216\n\t" \
  "ds_read_b128 a[108:111], v23 offset:2048\n\t" \
  "v_mfma_f32_16x16x128_f8f6f4 v[200:203], v[104:111], a[48:55], v[200:203]\n\t" \
  "ds_read_b64_tr_b8 v[124:125], v94 offset:40960\n\t" \
  "ds_read_b128 a[112:115], v22 offset:4096\n\t" \
  "s_add_u32 s49, s68, 0x41000\n\t" \
  "buffer_load_dwordx4 a[148:151], v4, s[12:15], s49 offen offset:1024 sc0 sc1\n\t" \
  "v_mfma_f32_16x16x128_f8f6f4 v[204:207], v[104:111], a[56:63], v[204:207]\n\t" \
  "ds_read_b64_tr_b8 v[126:127], v94 offset:41984\n\t" \
  "ds_read_b128 a[116:119], v23 offset:4096\n\t" \
  "s_waitcnt lgkmcnt(9)\n\t" \
  "v_mfma_f32_16x16x128_f8f6f4 v[220:223], v[112:119], a[56:63], v[220:223]\n\t" \
  "ds_read_b64_tr_b8 v[104:105], v87 offset:8192\n\t" \
  "ds_read_b128 a[120:123], v22 offset:6144\n\t" \
  "s_add_u32 s50, s68, 0x41000\n\t" \
  "buffer_load_dwordx4 a[152:155], v4, s[12:15], s50 offen offset:2048 sc0 sc1\n\t" \
  "v_mfma_f32_16x16x128_f8f6f4 v[216:219], v[112:119], a[48:55], v[216:219]\n\t" \
  "ds_read_b64_tr_b8 v[106:107], v87 offset:9216\n\t" \
  "ds_read_b128 a[124:127], v23 offset:6144\n\t" \
  "v_mfma_f32_16x16x128_f8f6f4 v[212:215], v[112:119], a[40:47], v[212:215]\n\t" \
  "ds_read_b64_tr_b8 v[108:109], v95 offset:40960\n\t" \
  "s_add_u32 s51, s68, 0x41000\n\t" \
  "buffer_load_dwordx4 a[156:159], v4, s[12:15], s51 offen offset:3072 sc0 sc1\n\t" \
  "v_mfma_f32_16x16x128_f8f6f4 v[208:211], v[112:119], a[32:39], v[208:211]\n\t" \
  "ds_read_b64_tr_b8 v[110:111], v95 offset:41984\n\t" \
  "s_waitcnt lgkmcnt(0)\n\t" \
  "s_barrier\n\t" \
  "v_mfma_f32_16x16x128_f8f6f4 v[224:227], v[120:127], a[32:39], v[224:227]\n\t" \
  "ds_read_b64_tr_b8 v[112:113], v80 offset:16384\n\t" \
  "s_add_u32 m0, s24, 0xc000\n\t" \
  "s_add_u32 s52, s42, 0x0\n\t" \
  "buffer_load_dwordx4 v2, s[8:11], s52 offen lds\n\t" \
  "v_mfma_f32_16x16x128_f8f6f4 v[228:231], v[120:127], a[40:47], v[228:231]\n\t" \
  "ds_read_b64_tr_b8 v[114:115], v80 offset:17408\n\t" \
  "v_mfma_f32_16x16x128_f8f6f4 v[232:235], v[120:127], a[48:55], v[232:235]\n\t" \
  "ds_read_b64_tr_b8 v[116:117], v88 offset:49152\n\t" \
  "s_add_u32 m0, s24, 0xc400\n\t" \
  "s_add_u32 s53, s42, 0x20000\n\t" \
  "buffer_load_dwordx4 v2, s[8:11], s53 offen lds\n\t" \
  "v_mfma_f32_16x16x128_f8f6f4 v[236:239], v[120:127], a[56:63], v[236:239]\n\t" \
  "ds_read_b64_tr_b8 v[118:119], v88 offset:50176\n\t" \
  "v_mfma_f32_16x16x128_f8f6f4 v[252:255], v[104:111], a[56:63], v[252:255]\n\t" \
  "ds_read_b64_tr_b8 v[120:121], v81 offset:16384\n\t" \
  "s_add_u32 m0, s24, 0xe000\n\t" \
  "s_add_u32 s54, s43, 0x0\n\t" \
  "buffer_load_dwordx4 v2, s[8:11], s54 offen lds\n\t" \
  "v_mfma_f32_16x16x128_f8f6f4 v[248:251], v[104:111], a[48:55], v[248:251]\n\t" \
  "ds_read_b64_tr_b8 v[122:123], v81 offset:17408\n\t" \
  "v_mfma_f32_16x16x128_f8f6f4 v[244:247], v[104:111], a[40:47], v[244:247]\n\t" \
  "ds_read_b64_tr_b8 v[124:125], v89 offset:49152\n\t" \
  "s_add_u32 m0, s24, 0xe400\n\t" \
  "s_add_u32 s55, s43, 0x20000\n\t" \
  "buffer_load_dwordx4 v2, s[8:11], s55 offen lds\n\t" \
  "v_mfma_f32_16x16x128_f8f6f4 v[240:243], v[104:111], a[32:39], v[240:243]\n\t" \
  "ds_read_b64_tr_b8 v[126:127], v89 offset:50176\n\t" \
  "s_waitcnt lgkmcnt(4)\n\t" \
  "v_mfma_f32_16x16x128_f8f6f4 v[128:131], v[112:119], a[64:71], v[128:131]\n\t" \
  "ds_read_b64_tr_b8 v[104:105], v82 offset:16384\n\t" \
  "s_add_u32 m0, s24, 0x10000\n\t" \
  "s_add_u32 s48, s44, 0x0\n\t" \
  "buffer_load_dwordx4 v2, s[8:11], s48 offen lds\n\t" \
  "v_mfma_f32_16x16x128_f8f6f4 v[132:135], v[112:119], a[72:79], v[132:135]\n\t" \
  "ds_read_b64_tr_b8 v[106:107], v82 offset:17408\n\t" \
  "v_mfma_f32_16x16x128_f8f6f4 v[136:139], v[112:119], a[80:87], v[136:139]\n\t" \
  "ds_read_b64_tr_b8 v[108:109], v90 offset:49152\n\t" \
  "s_add_u32 m0, s24, 0x10400\n\t" \
  "s_add_u32 s49, s44, 0x20000\n\t" \
  "buffer_load_dwordx4 v2, s[8:11], s49 offen lds\n\t" \
  "v_mfma_f32_16x16x128_f8f6f4 v[140:143], v[112:119], a[88:95], v[140:143]\n\t" \
  "ds_read_b64_tr_b8 v[110:111], v90 offset:50176\n\t" \
  "s_waitcnt lgkmcnt(4)\n\t" \
  "v_mfma_f32_16x16x128_f8f6f4 v[156:159], v[120:127], a[88:95], v[156:159]\n\t" \
  "ds_read_b64_tr_b8 v[112:113], v83 offset:16384\n\t" \
  "s_add_u32 m0, s24, 0x12000\n\t" \
  "s_add_u32 s50, s45, 0x0\n\t" \
  "buffer_load_dwordx4 v2, s[8:11], s50 offen lds\n\t" \
  "v_mfma_f32_16x16x128_f8f6f4 v[152:155], v[120:127], a[80:87], v[152:155]\n\t" \
  "ds_read_b64_tr_b8 v[114:115], v83 offset:17408\n\t" \
  "v_mfma_f32_16x16x128_f8f6f4 v[148:151], v[120:127], a[72:79], v[148:151]\n\t" \
  "ds_read_b64_tr_b8 v[116:117], v91 offset:49152\n\t" \
  "s_add_u32 m0, s24, 0x12400\n\t" \
  "s_add_u32 s51, s45, 0x20000\n\t" \
  "buffer_load_dwordx4 v2, s[8:11], s51 offen lds\n\t" \
  "v_mfma_f32_16x16x128_f8f6f4 v[144:147], v[120:127], a[64:71], v[144:147]\n\t" \
  "ds_read_b64_tr_b8 v[118:119], v91 offset:50176\n\t" \
  "s_waitcnt lgkmcnt(4)\n\t" \
  "v_mfma_f32_16x16x128_f8f6f4 v[160:163], v[104:111], a[64:71], v[160:163]\n\t" \
  "ds_read_b64_tr_b8 v[120:121], v84 offset:16384\n\t" \
  "v_mfma_f32_16x16x128_f8f6f4 v[164:167], v[104:111], a[72:79], v[164:167]\n\t" \
  "ds_read_b64_tr_b8 v[122:123], v84 offset:17408\n\t" \
  "v_mfma_f32_16x16x128_f8f6f4 v[168:171], v[104:111], a[80:87], v[168:171]\n\t" \
  "ds_read_b64_tr_b8 v[124:125], v92 offset:49152\n\t" \
  "v_mfma_f32_16x16x128_f8f6f4 v[172:175], v[104:111], a[88:95], v[172:175]\n\t" \
  "ds_read_b64_tr_b8 v[126:127], v92 offset:50176\n\t" \
  "s_waitcnt lgkmcnt(4)\n\t" \
  "v_mfma_f32_16x16x128_f8f6f4 v[188:191], v[112:119], a[88:95], v[188:191]\n\t" \
  "ds_read_b64_tr_b8 v[104:105], v85 offset:16384\n\t" \
  "v_mfma_f32_16x16x128_f8f6f4 v[184:187], v[112:119], a[80:87], v[184:187]\n\t" \
  "ds_read_b64_tr_b8 v[106:107], v85 offset:17408\n\t" \
  "v_mfma_f32_16x16x128_f8f6f4 v[180:183], v[112:119], a[72:79], v[180:183]\n\t" \
  "ds_read_b64_tr_b8 v[108:109], v93 offset:49152\n\t" \
  "v_mfma_f32_16x16x128_f8f6f4 v[176:179], v[112:119], a[64:71], v[176:179]\n\t" \
  "ds_read_b64_tr_b8 v[110:111], v93 offset:50176\n\t" \
  "s_waitcnt lgkmcnt(4)\n\t" \
  "v_mfma_f32_16x16x128_f8f6f4 v[192:195], v[120:127], a[64:71], v[192:195]\n\t" \
  "ds_read_b64_tr_b8 v[112:113], v86 offset:16384\n\t" \
  "v_mfma_f32_16x16x128_f8f6f4 v[196:199], v[120:127], a[72:79], v[196:199]\n\t" \
  "ds_read_b64_tr_b8 v[114:115], v86 offset:17408\n\t" \
  "v_mfma_f32_16x16x128_f8f6f4 v[200:203], v[120:127], a[80:87], v[200:203]\n\t" \
  "ds_read_b64_tr_b8 v[116:117], v94 offset:49152\n\t" \
  "v_mfma_f32_16x16x128_f8f6f4 v[204:207], v[120:127], a[88:95], v[204:207]\n\t" \
  "ds_read_b64_tr_b8 v[118:119], v94 offset:50176\n\t" \
  "s_waitcnt lgkmcnt(4)\n\t" \
  "v_mfma_f32_16x16x128_f8f6f4 v[220:223], v[104:111], a[88:95], v[220:223]\n\t" \
  "ds_read_b64_tr_b8 v[120:121], v87 offset:16384\n\t" \
  "v_mfma_f32_16x16x128_f8f6f4 v[216:219], v[104:111], a[80:87], v[216:219]\n\t" \
  "ds_read_b64_tr_b8 v[122:123], v87 offset:17408\n\t" \
  "v_mfma_f32_16x16x128_f8f6f4 v[212:215], v[104:111], a[72:79], v[212:215]\n\t" \
  "ds_read_b64_tr_b8 v[124:125], v95 offset:49152\n\t" \
  "s_add_u32 m0, s24, 0x20000\n\t" \
  "s_add_u32 s52, s40, 0x0\n\t" \
  "buffer_load_dwordx4 v3, s[8:11], s52 offen lds\n\t" \
  "v_mfma_f32_16x16x128_f8f6f4 v[208:211], v[104:111], a[64:71], v[208:211]\n\t" \
  "ds_read_b64_tr_b8 v[126:127], v95 offset:50176\n\t" \
  "s_waitcnt lgkmcnt(0)\n\t" \
  "s_barrier\n\t" \
  "v_mfma_f32_16x16x128_f8f6f4 v[224:227], v[112:119], a[64:71], v[224:227]\n\t" \
  "ds_read_b64_tr_b8 v[104:105], v80 offset:24576\n\t" \
  "s_add_u32 m0, s24, 0x20400\n\t" \
  "s_add_u32 s53, s40, 0x20000\n\t" \
  "buffer_load_dwordx4 v3, s[8:11], s53 offen lds\n\t" \
  "v_mfma_f32_16x16x128_f8f6f4 v[228:231], v[112:119], a[72:79], v[228:231]\n\t" \
  "ds_read_b64_tr_b8 v[106:107], v80 offset:25600\n\t" \
  "v_mfma_f32_16x16x128_f8f6f4 v[232:235], v[112:119], a[80:87], v[232:235]\n\t" \
  "ds_read_b64_tr_b8 v[108:109], v88 offset:57344\n\t" \
  "s_add_u32 m0, s24, 0x22000\n\t" \
  "s_add_u32 s54, s41, 0x0\n\t" \
  "buffer_load_dwordx4 v3, s[8:11], s54 offen lds\n\t" \
  "v_mfma_f32_16x16x128_f8f6f4 v[236:239], v[112:119], a[88:95], v[236:239]\n\t" \
  "ds_read_b64_tr_b8 v[110:111], v88 offset:58368\n\t" \
  "v_mfma_f32_16x16x128_f8f6f4 v[252:255], v[120:127], a[88:95], v[252:255]\n\t" \
  "ds_read_b64_tr_b8 v[112:113], v81 offset:24576\n\t" \
  "s_add_u32 m0, s24, 0x22400\n\t" \
  "s_add_u32 s55, s41, 0x20000\n\t" \
  "buffer_load_dwordx4 v3, s[8:11], s55 offen lds\n\t" \
  "v_mfma_f32_16x16x128_f8f6f4 v[248:251], v[120:127], a[80:87], v[248:251]\n\t" \
  "ds_read_b64_tr_b8 v[114:115], v81 offset:25600\n\t" \
  "v_mfma_f32_16x16x128_f8f6f4 v[244:247], v[120:127], a[72:79], v[244:247]\n\t" \
  "ds_read_b64_tr_b8 v[116:117], v89 offset:57344\n\t" \
  "s_and_b32 s64, s56, 0x2\n\t" \
  "s_cmp_eq_u32 s64, 0x2\n\t" \
  "s_cbranch_scc1 Lok1_0_%=\n\t" \
  "s_mov_b32 s62, 0x2\n\t" \
  "s_mov_b32 s69, 0x7f\n\t" \
  "s_call_b64 s[60:61], Lpoll_0_%=\n\t" \
  "Lok1_0_%=:\n\t" \
  "s_add_u32 s48, s68, 0x50000\n\t" \
  "buffer_load_dwordx4 a[160:163], v4, s[12:15], s48 offen offset:0 sc0 sc1\n\t" \
  "v_mfma_f32_16x16x128_f8f6f4 v[240:243], v[120:127], a[64:71], v[240:243]\n\t" \
  "ds_read_b64_tr_b8 v[118:119], v89 offset:58368\n\t" \
  "s_waitcnt lgkmcnt(4)\n\t" \
  "v_mfma_f32_16x16x128_f8f6f4 v[128:131], v[104:111], a[96:103], v[128:131]\n\t" \
  "ds_read_b64_tr_b8 v[120:121], v82 offset:24576\n\t" \
  "s_add_u32 s49, s68, 0x50000\n\t" \
  "buffer_load_dwordx4 a[164:167], v4, s[12:15], s49 offen offset:1024 sc0 sc1\n\t" \
  "v_mfma_f32_16x16x128_f8f6f4 v[132:135], v[104:111], a[104:111], v[132:135]\n\t" \
  "ds_read_b64_tr_b8 v[122:123], v82 offset:25600\n\t" \
  "v_mfma_f32_16x16x128_f8f6f4 v[136:139], v[104:111], a[112:119], v[136:139]\n\t" \
  "ds_read_b64_tr_b8 v[124:125], v90 offset:57344\n\t" \
  "s_add_u32 s50, s68, 0x50000\n\t" \
  "buffer_load_dwordx4 a[168:171], v4, s[12:15], s50 offen offset:2048 sc0 sc1\n\t" \
  "v_mfma_f32_16x16x128_f8f6f4 v[140:143], v[104:111], a[120:127], v[140:143]\n\t" \
  "ds_read_b64_tr_b8 v[126:127], v90 offset:58368\n\t" \
  "s_waitcnt lgkmcnt(4)\n\t" \
  "v_mfma_f32_16x16x128_f8f6f4 v[156:159], v[112:119], a[120:127], v[156:159]\n\t" \
  "ds_read_b64_tr_b8 v[104:105], v83 offset:24576\n\t" \
  "s_add_u32 s51, s68, 0x50000\n\t" \
  "buffer_load_dwordx4 a[172:175], v4, s[12:15], s51 offen offset:3072 sc0 sc1\n\t" \
  "v_mfma_f32_16x16x128_f8f6f4 v[152:155], v[112:119], a[112:119], v[152:155]\n\t" \
  "ds_read_b64_tr_b8 v[106:107], v83 offset:25600\n\t" \
  "v_mfma_f32_16x16x128_f8f6f4 v[148:151], v[112:119], a[104:111], v[148:151]\n\t" \
  "ds_read_b64_tr_b8 v[108:109], v91 offset:57344\n\t" \
  "s_add_u32 s52, s68, 0x51000\n\t" \
  "buffer_load_dwordx4 a[176:179], v4, s[12:15], s52 offen offset:0 sc0 sc1\n\t" \
  "v_mfma_f32_16x16x128_f8f6f4 v[144:147], v[112:119], a[96:103], v[144:147]\n\t" \
  "ds_read_b64_tr_b8 v[110:111], v91 offset:58368\n\t" \
  "s_waitcnt lgkmcnt(4)\n\t" \
  "v_mfma_f32_16x16x128_f8f6f4 v[160:163], v[120:127], a[96:103], v[160:163]\n\t" \
  "ds_read_b64_tr_b8 v[112:113], v84 offset:24576\n\t" \
  "s_add_u32 s53, s68, 0x51000\n\t" \
  "buffer_load_dwordx4 a[180:183], v4, s[12:15], s53 offen offset:1024 sc0 sc1\n\t" \
  "v_mfma_f32_16x16x128_f8f6f4 v[164:167], v[120:127], a[104:111], v[164:167]\n\t" \
  "ds_read_b64_tr_b8 v[114:115], v84 offset:25600\n\t" \
  "v_mfma_f32_16x16x128_f8f6f4 v[168:171], v[120:127], a[112:119], v[168:171]\n\t" \
  "ds_read_b64_tr_b8 v[116:117], v92 offset:57344\n\t" \
  "s_add_u32 s54, s68, 0x51000\n\t" \
  "buffer_load_dwordx4 a[184:187], v4, s[12:15], s54 offen offset:2048 sc0 sc1\n\t" \
  "v_mfma_f32_16x16x128_f8f6f4 v[172:175], v[120:127], a[120:127], v[172:175]\n\t" \
  "ds_read_b64_tr_b8 v[118:119], v92 offset:58368\n\t" \
  "s_waitcnt lgkmcnt(4)\n\t" \
  "v_mfma_f32_16x16x128_f8f6f4 v[188:191], v[104:111], a[120:127], v[188:191]\n\t" \
  "ds_read_b64_tr_b8 v[120:121], v85 offset:24576\n\t" \
  "s_add_u32 s55, s68, 0x51000\n\t" \
  "buffer_load_dwordx4 a[188:191], v4, s[12:15], s55 offen offset:3072 sc0 sc1\n\t" \
  "v_mfma_f32_16x16x128_f8f6f4 v[184:187], v[104:111], a[112:119], v[184:187]\n\t" \
  "ds_read_b64_tr_b8 v[122:123], v85 offset:25600\n\t" \
  "v_mfma_f32_16x16x128_f8f6f4 v[180:183], v[104:111], a[104:111], v[180:183]\n\t" \
  "ds_read_b64_tr_b8 v[124:125], v93 offset:57344\n\t" \
  "s_add_u32 m0, s24, 0x24000\n\t" \
  "s_add_u32 s48, s42, 0x0\n\t" \
  "buffer_load_dwordx4 v3, s[8:11], s48 offen lds\n\t" \
  "v_mfma_f32_16x16x128_f8f6f4 v[176:179], v[104:111], a[96:103], v[176:179]\n\t" \
  "ds_read_b64_tr_b8 v[126:127], v93 offset:58368\n\t" \
  "s_waitcnt lgkmcnt(4)\n\t" \
  "v_mfma_f32_16x16x128_f8f6f4 v[192:195], v[112:119], a[96:103], v[192:195]\n\t" \
  "ds_read_b64_tr_b8 v[104:105], v86 offset:24576\n\t" \
  "s_add_u32 m0, s24, 0x24400\n\t" \
  "s_add_u32 s49, s42, 0x20000\n\t" \
  "buffer_load_dwordx4 v3, s[8:11], s49 offen lds\n\t" \
  "v_mfma_f32_16x16x128_f8f6f4 v[196:199], v[112:119], a[104:111], v[196:199]\n\t" \
  "ds_read_b64_tr_b8 v[106:107], v86 offset:25600\n\t" \
  "v_mfma_f32_16x16x128_f8f6f4 v[200:203], v[112:119], a[112:119], v[200:203]\n\t" \
  "ds_read_b64_tr_b8 v[108:109], v94 offset:57344\n\t" \
  "s_and_b32 s64, s56, 0x4\n\t" \
  "s_cmp_eq_u32 s64, 0x4\n\t" \
  "s_cbranch_scc1 Lok2_0_%=\n\t" \
  "s_mov_b32 s62, 0x4\n\t" \
  "s_mov_b32 s69, 0x7f\n\t" \
  "s_call_b64 s[60:61], Lpoll_0_%=\n\t" \
  "Lok2_0_%=:\n\t" \
  "s_add_u32 s50, s68, 0x60000\n\t" \
  "buffer_load_dwordx4 a[192:195], v4, s[12:15], s50 offen offset:0 sc0 sc1\n\t" \
  "v_mfma_f32_16x16x128_f8f6f4 v[204:207], v[112:119], a[120:127], v[204:207]\n\t" \
  "ds_read_b64_tr_b8 v[110:111], v94 offset:58368\n\t" \
  "s_waitcnt lgkmcnt(4)\n\t" \
  "v_mfma_f32_16x16x128_f8f6f4 v[220:223], v[120:127], a[120:127], v[220:223]\n\t" \
  "ds_read_b64_tr_b8 v[112:113], v87 offset:24576\n\t" \
  "s_add_u32 s51, s68, 0x60000\n\t" \
  "buffer_load_dwordx4 a[196:199], v4, s[12:15], s51 offen offset:1024 sc0 sc1\n\t" \
  "v_mfma_f32_16x16x128_f8f6f4 v[216:219], v[120:127], a[112:119], v[216:219]\n\t" \
  "ds_read_b64_tr_b8 v[114:115], v87 offset:25600\n\t" \
  "v_mfma_f32_16x16x128_f8f6f4 v[212:215], v[120:127], a[104:111], v[212:215]\n\t" \
  "ds_read_b64_tr_b8 v[116:117], v95 offset:57344\n\t" \
  "s_add_u32 s52, s68, 0x60000\n\t" \
  "buffer_load_dwordx4 a[200:203], v4, s[12:15], s52 offen offset:2048 sc0 sc1\n\t" \
  "v_mfma_f32_16x16x128_f8f6f4 v[208:211], v[120:127], a[96:103], v[208:211]\n\t" \
  "ds_read_b64_tr_b8 v[118:119], v95 offset:58368\n\t" \
  "s_waitcnt vmcnt(15) lgkmcnt(0)\n\t" \
  "s_barrier\n\t" \
  "v_mfma_f32_16x16x128_f8f6f4 v[224:227], v[104:111], a[96:103], v[224:227]\n\t" \
  "ds_read_b64_tr_b8 v[120:121], v80 offset:32768\n\t" \
  "s_add_u32 s53, s68, 0x60000\n\t" \
  "buffer_load_dwordx4 a[204:207], v4, s[12:15], s53 offen offset:3072 sc0 sc1\n\t" \
  "v_mfma_f32_16x16x128_f8f6f4 v[228:231], v[104:111], a[104:111], v[228:231]\n\t" \
  "ds_read_b64_tr_b8 v[122:123], v80 offset:33792\n\t" \
  "v_mfma_f32_16x16x128_f8f6f4 v[232:235], v[104:111], a[112:119], v[232:235]\n\t" \
  "ds_read_b64_tr_b8 v[124:125], v96 offset:0\n\t" \
  "s_add_u32 s54, s68, 0x61000\n\t" \
  "buffer_load_dwordx4 a[208:211], v4, s[12:15], s54 offen offset:0 sc0 sc1\n\t" \
  "v_mfma_f32_16x16x128_f8f6f4 v[236:239], v[104:111], a[120:127], v[236:239]\n\t" \
  "ds_read_b64_tr_b8 v[126:127], v96 offset:1024\n\t" \
  "v_mfma_f32_16x16x128_f8f6f4 v[252:255], v[112:119], a[120:127], v[252:255]\n\t" \
  "ds_read_b64_tr_b8 v[104:105], v81 offset:32768\n\t" \
  "s_add_u32 s55, s68, 0x61000\n\t" \
  "buffer_load_dwordx4 a[212:215], v4, s[12:15], s55 offen offset:1024 sc0 sc1\n\t" \
  "v_mfma_f32_16x16x128_f8f6f4 v[248:251], v[112:119], a[112:119], v[248:251]\n\t" \
  "ds_read_b64_tr_b8 v[106:107], v81 offset:33792\n\t" \
  "v_mfma_f32_16x16x128_f8f6f4 v[244:247], v[112:119], a[104:111], v[244:247]\n\t" \
  "ds_read_b64_tr_b8 v[108:109], v97 offset:0\n\t" \
  "s_add_u32 s48, s68, 0x61000\n\t" \
  "buffer_load_dwordx4 a[216:219], v4, s[12:15], s48 offen offset:2048 sc0 sc1\n\t" \
  "v_mfma_f32_16x16x128_f8f6f4 v[240:243], v[112:119], a[96:103], v[240:243]\n\t" \
  "ds_read_b64_tr_b8 v[110:111], v97 offset:1024\n\t" \
  "s_waitcnt lgkmcnt(4)\n\t" \
  "v_mfma_f32_16x16x128_f8f6f4 v[128:131], v[120:127], a[128:135], v[128:131]\n\t" \
  "ds_read_b64_tr_b8 v[112:113], v82 offset:32768\n\t" \
  "s_add_u32 s49, s68, 0x61000\n\t" \
  "buffer_load_dwordx4 a[220:223], v4, s[12:15], s49 offen offset:3072 sc0 sc1\n\t" \
  "v_mfma_f32_16x16x128_f8f6f4 v[132:135], v[120:127], a[136:143], v[132:135]\n\t" \
  "ds_read_b64_tr_b8 v[114:115], v82 offset:33792\n\t" \
  "v_mfma_f32_16x16x128_f8f6f4 v[136:139], v[120:127], a[144:151], v[136:139]\n\t" \
  "ds_read_b64_tr_b8 v[116:117], v98 offset:0\n\t" \
  "s_add_u32 m0, s24, 0x26000\n\t" \
  "s_add_u32 s50, s43, 0x0\n\t" \
  "buffer_load_dwordx4 v3, s[8:11], s50 offen lds\n\t" \
  "v_mfma_f32_16x16x128_f8f6f4 v[140:143], v[120:127], a[152:159], v[140:143]\n\t" \
  "ds_read_b64_tr_b8 v[118:119], v98 offset:1024\n\t" \
  "s_waitcnt lgkmcnt(4)\n\t" \
  "v_mfma_f32_16x16x128_f8f6f4 v[156:159], v[104:111], a[152:159], v[156:159]\n\t" \
  "ds_read_b64_tr_b8 v[120:121], v83 offset:32768\n\t" \
  "s_add_u32 m0, s24, 0x26400\n\t" \
  "s_add_u32 s51, s43, 0x20000\n\t" \
  "buffer_load_dwordx4 v3, s[8:11], s51 offen lds\n\t" \
  "v_mfma_f32_16x16x128_f8f6f4 v[152:155], v[104:111], a[144:151], v[152:155]\n\t" \
  "ds_read_b64_tr_b8 v[122:123], v83 offset:33792\n\t" \
  "v_mfma_f32_16x16x128_f8f6f4 v[148:151], v[104:111], a[136:143], v[148:151]\n\t" \
  "ds_read_b64_tr_b8 v[124:125], v99 offset:0\n\t" \
  "s_and_b32 s64, s56, 0x8\n\t" \
  "s_cmp_eq_u32 s64, 0x8\n\t" \
  "s_cbranch_scc1 Lok3_0_%=\n\t" \
  "s_mov_b32 s62, 0x8\n\t" \
  "s_mov_b32 s69, 0x7f\n\t" \
  "s_call_b64 s[60:61], Lpoll_0_%=\n\t" \
  "Lok3_0_%=:\n\t" \
  "s_add_u32 s52, s68, 0x70000\n\t" \
  "buffer_load_dwordx4 a[224:227], v4, s[12:15], s52 offen offset:0 sc0 sc1\n\t" \
  "v_mfma_f32_16x16x128_f8f6f4 v[144:147], v[104:111], a[128:135], v[144:147]\n\t" \
  "ds_read_b64_tr_b8 v[126:127], v99 offset:1024\n\t" \
  "s_waitcnt lgkmcnt(4)\n\t" \
  "v_mfma_f32_16x16x128_f8f6f4 v[160:163], v[112:119], a[128:135], v[160:163]\n\t" \
  "ds_read_b64_tr_b8 v[104:105], v84 offset:32768\n\t" \
  "s_add_u32 s53, s68, 0x70000\n\t" \
  "buffer_load_dwordx4 a[228:231], v4, s[12:15], s53 offen offset:1024 sc0 sc1\n\t" \
  "v_mfma_f32_16x16x128_f8f6f4 v[164:167], v[112:119], a[136:143], v[164:167]\n\t" \
  "ds_read_b64_tr_b8 v[106:107], v84 offset:33792\n\t" \
  "v_mfma_f32_16x16x128_f8f6f4 v[168:171], v[112:119], a[144:151], v[168:171]\n\t" \
  "ds_read_b64_tr_b8 v[108:109], v100 offset:0\n\t" \
  "s_add_u32 s54, s68, 0x70000\n\t" \
  "buffer_load_dwordx4 a[232:235], v4, s[12:15], s54 offen offset:2048 sc0 sc1\n\t" \
  "v_mfma_f32_16x16x128_f8f6f4 v[172:175], v[112:119], a[152:159], v[172:175]\n\t" \
  "ds_read_b64_tr_b8 v[110:111], v100 offset:1024\n\t" \
  "s_waitcnt lgkmcnt(4)\n\t" \
  "v_mfma_f32_16x16x128_f8f6f4 v[188:191], v[120:127], a[152:159], v[188:191]\n\t" \
  "ds_read_b64_tr_b8 v[112:113], v85 offset:32768\n\t" \
  "s_add_u32 s55, s68, 0x70000\n\t" \
  "buffer_load_dwordx4 a[236:239], v4, s[12:15], s55 offen offset:3072 sc0 sc1\n\t" \
  "v_mfma_f32_16x16x128_f8f6f4 v[184:187], v[120:127], a[144:151], v[184:187]\n\t" \
  "ds_read_b64_tr_b8 v[114:115], v85 offset:33792\n\t" \
  "v_mfma_f32_16x16x128_f8f6f4 v[180:183], v[120:127], a[136:143], v[180:183]\n\t" \
  "ds_read_b64_tr_b8 v[116:117], v101 offset:0\n\t" \
  "s_add_u32 s48, s68, 0x71000\n\t" \
  "buffer_load_dwordx4 a[240:243], v4, s[12:15], s48 offen offset:0 sc0 sc1\n\t" \
  "v_mfma_f32_16x16x128_f8f6f4 v[176:179], v[120:127], a[128:135], v[176:179]\n\t" \
  "ds_read_b64_tr_b8 v[118:119], v101 offset:1024\n\t" \
  "s_waitcnt lgkmcnt(4)\n\t" \
  "v_mfma_f32_16x16x128_f8f6f4 v[192:195], v[104:111], a[128:135], v[192:195]\n\t" \
  "ds_read_b64_tr_b8 v[120:121], v86 offset:32768\n\t" \
  "s_add_u32 s49, s68, 0x71000\n\t" \
  "buffer_load_dwordx4 a[244:247], v4, s[12:15], s49 offen offset:1024 sc0 sc1\n\t" \
  "v_mfma_f32_16x16x128_f8f6f4 v[196:199], v[104:111], a[136:143], v[196:199]\n\t" \
  "ds_read_b64_tr_b8 v[122:123], v86 offset:33792\n\t" \
  "v_mfma_f32_16x16x128_f8f6f4 v[200:203], v[104:111], a[144:151], v[200:203]\n\t" \
  "ds_read_b64_tr_b8 v[124:125], v102 offset:0\n\t" \
  "s_add_u32 s50, s68, 0x71000\n\t" \
  "buffer_load_dwordx4 a[248:251], v4, s[12:15], s50 offen offset:2048 sc0 sc1\n\t" \
  "v_mfma_f32_16x16x128_f8f6f4 v[204:207], v[104:111], a[152:159], v[204:207]\n\t" \
  "ds_read_b64_tr_b8 v[126:127], v102 offset:1024\n\t" \
  "s_waitcnt lgkmcnt(4)\n\t" \
  "v_mfma_f32_16x16x128_f8f6f4 v[220:223], v[112:119], a[152:159], v[220:223]\n\t" \
  "ds_read_b64_tr_b8 v[104:105], v87 offset:32768\n\t" \
  "s_add_u32 s51, s68, 0x71000\n\t" \
  "buffer_load_dwordx4 a[252:255], v4, s[12:15], s51 offen offset:3072 sc0 sc1\n\t" \
  "v_mfma_f32_16x16x128_f8f6f4 v[216:219], v[112:119], a[144:151], v[216:219]\n\t" \
  "ds_read_b64_tr_b8 v[106:107], v87 offset:33792\n\t" \
  "v_mfma_f32_16x16x128_f8f6f4 v[212:215], v[112:119], a[136:143], v[212:215]\n\t" \
  "ds_read_b64_tr_b8 v[108:109], v103 offset:0\n\t" \
  "s_add_u32 m0, s24, 0x18000\n\t" \
  "s_add_u32 s52, s44, 0x0\n\t" \
  "buffer_load_dwordx4 v3, s[8:11], s52 offen lds\n\t" \
  "v_mfma_f32_16x16x128_f8f6f4 v[208:211], v[112:119], a[128:135], v[208:211]\n\t" \
  "ds_read_b64_tr_b8 v[110:111], v103 offset:1024\n\t" \
  "s_waitcnt vmcnt(29) lgkmcnt(0)\n\t" \
  "s_barrier\n\t" \
  "v_mfma_f32_16x16x128_f8f6f4 v[224:227], v[120:127], a[128:135], v[224:227]\n\t" \
  "ds_read_b64_tr_b8 v[112:113], v80 offset:40960\n\t" \
  "s_add_u32 m0, s24, 0x18400\n\t" \
  "s_add_u32 s53, s44, 0x20000\n\t" \
  "buffer_load_dwordx4 v3, s[8:11], s53 offen lds\n\t" \
  "v_mfma_f32_16x16x128_f8f6f4 v[228:231], v[120:127], a[136:143], v[228:231]\n\t" \
  "ds_read_b64_tr_b8 v[114:115], v80 offset:41984\n\t" \
  "v_mfma_f32_16x16x128_f8f6f4 v[232:235], v[120:127], a[144:151], v[232:235]\n\t" \
  "ds_read_b64_tr_b8 v[116:117], v96 offset:8192\n\t" \
  "s_and_b32 s64, s56, 0x10\n\t" \
  "s_cmp_eq_u32 s64, 0x10\n\t" \
  "s_cbranch_scc1 Lok4_0_%=\n\t" \
  "s_mov_b32 s62, 0x10\n\t" \
  "s_mov_b32 s69, 0x7f\n\t" \
  "s_call_b64 s[60:61], Lpoll_0_%=\n\t" \
  "Lok4_0_%=:\n\t" \
  "s_add_u32 s54, s68, 0x80000\n\t" \
  "buffer_load_dwordx4 a[0:3], v4, s[12:15], s54 offen offset:0 sc0 sc1\n\t" \
  "v_mfma_f32_16x16x128_f8f6f4 v[236:239], v[120:127], a[152:159], v[236:239]\n\t" \
  "ds_read_b64_tr_b8 v[118:119], v96 offset:9216\n\t" \
  "v_mfma_f32_16x16x128_f8f6f4 v[252:255], v[104:111], a[152:159], v[252:255]\n\t" \
  "ds_read_b64_tr_b8 v[120:121], v81 offset:40960\n\t" \
  "s_add_u32 s55, s68, 0x80000\n\t" \
  "buffer_load_dwordx4 a[4:7], v4, s[12:15], s55 offen offset:1024 sc0 sc1\n\t" \
  "v_mfma_f32_16x16x128_f8f6f4 v[248:251], v[104:111], a[144:151], v[248:251]\n\t" \
  "ds_read_b64_tr_b8 v[122:123], v81 offset:41984\n\t" \
  "v_mfma_f32_16x16x128_f8f6f4 v[244:247], v[104:111], a[136:143], v[244:247]\n\t" \
  "ds_read_b64_tr_b8 v[124:125], v97 offset:8192\n\t" \
  "s_add_u32 s48, s68, 0x80000\n\t" \
  "buffer_load_dwordx4 a[8:11], v4, s[12:15], s48 offen offset:2048 sc0 sc1\n\t" \
  "v_mfma_f32_16x16x128_f8f6f4 v[240:243], v[104:111], a[128:135], v[240:243]\n\t" \
  "ds_read_b64_tr_b8 v[126:127], v97 offset:9216\n\t" \
  "s_waitcnt vmcnt(25) lgkmcnt(4)\n\t" \
  "v_mfma_f32_16x16x128_f8f6f4 v[128:131], v[112:119], a[160:167], v[128:131]\n\t" \
  "ds_read_b64_tr_b8 v[104:105], v82 offset:40960\n\t" \
  "s_add_u32 s49, s68, 0x80000\n\t" \
  "buffer_load_dwordx4 a[12:15], v4, s[12:15], s49 offen offset:3072 sc0 sc1\n\t" \
  "v_mfma_f32_16x16x128_f8f6f4 v[132:135], v[112:119], a[168:175], v[132:135]\n\t" \
  "ds_read_b64_tr_b8 v[106:107], v82 offset:41984\n\t" \
  "v_mfma_f32_16x16x128_f8f6f4 v[136:139], v[112:119], a[176:183], v[136:139]\n\t" \
  "ds_read_b64_tr_b8 v[108:109], v98 offset:8192\n\t" \
  "s_add_u32 s50, s68, 0x81000\n\t" \
  "buffer_load_dwordx4 a[16:19], v4, s[12:15], s50 offen offset:0 sc0 sc1\n\t" \
  "v_mfma_f32_16x16x128_f8f6f4 v[140:143], v[112:119], a[184:191], v[140:143]\n\t" \
  "ds_read_b64_tr_b8 v[110:111], v98 offset:9216\n\t" \
  "s_waitcnt lgkmcnt(4)\n\t" \
  "v_mfma_f32_16x16x128_f8f6f4 v[156:159], v[120:127], a[184:191], v[156:159]\n\t" \
  "ds_read_b64_tr_b8 v[112:113], v83 offset:40960\n\t" \
  "s_add_u32 s51, s68, 0x81000\n\t" \
  "buffer_load_dwordx4 a[20:23], v4, s[12:15], s51 offen offset:1024 sc0 sc1\n\t" \
  "v_mfma_f32_16x16x128_f8f6f4 v[152:155], v[120:127], a[176:183], v[152:155]\n\t" \
  "ds_read_b64_tr_b8 v[114:115], v83 offset:41984\n\t" \
  "v_mfma_f32_16x16x128_f8f6f4 v[148:151], v[120:127], a[168:175], v[148:151]\n\t" \
  "ds_read_b64_tr_b8 v[116:117], v99 offset:8192\n\t" \
  "s_add_u32 s52, s68, 0x81000\n\t" \
  "buffer_load_dwordx4 a[24:27], v4, s[12:15], s52 offen offset:2048 sc0 sc1\n\t" \
  "v_mfma_f32_16x16x128_f8f6f4 v[144:147], v[120:127], a[160:167], v[144:147]\n\t" \
  "ds_read_b64_tr_b8 v[118:119], v99 offset:9216\n\t" \
  "s_waitcnt lgkmcnt(4)\n\t" \
  "v_mfma_f32_16x16x128_f8f6f4 v[160:163], v[104:111], a[160:167], v[160:163]\n\t" \
  "ds_read_b64_tr_b8 v[120:121], v84 offset:40960\n\t" \
  "s_add_u32 s53, s68, 0x81000\n\t" \
  "buffer_load_dwordx4 a[28:31], v4, s[12:15], s53 offen offset:3072 sc0 sc1\n\t" \
  "v_mfma_f32_16x16x128_f8f6f4 v[164:167], v[104:111], a[168:175], v[164:167]\n\t" \
  "ds_read_b64_tr_b8 v[122:123], v84 offset:41984\n\t" \
  "v_mfma_f32_16x16x128_f8f6f4 v[168:171], v[104:111], a[176:183], v[168:171]\n\t" \
  "ds_read_b64_tr_b8 v[124:125], v100 offset:8192\n\t" \
  "s_add_u32 m0, s24, 0x14000\n\t" \
  "s_add_u32 s54, s46, 0x0\n\t" \
  "buffer_load_dwordx4 v2, s[8:11], s54 offen lds\n\t" \
  "v_mfma_f32_16x16x128_f8f6f4 v[172:175], v[104:111], a[184:191], v[172:175]\n\t" \
  "ds_read_b64_tr_b8 v[126:127], v100 offset:9216\n\t" \
  "s_waitcnt lgkmcnt(4)\n\t" \
  "v_mfma_f32_16x16x128_f8f6f4 v[188:191], v[112:119], a[184:191], v[188:191]\n\t" \
  "ds_read_b64_tr_b8 v[104:105], v85 offset:40960\n\t" \
  "s_add_u32 m0, s24, 0x14400\n\t" \
  "s_add_u32 s55, s46, 0x20000\n\t" \
  "buffer_load_dwordx4 v2, s[8:11], s55 offen lds\n\t" \
  "v_mfma_f32_16x16x128_f8f6f4 v[184:187], v[112:119], a[176:183], v[184:187]\n\t" \
  "ds_read_b64_tr_b8 v[106:107], v85 offset:41984\n\t" \
  "v_mfma_f32_16x16x128_f8f6f4 v[180:183], v[112:119], a[168:175], v[180:183]\n\t" \
  "ds_read_b64_tr_b8 v[108:109], v101 offset:8192\n\t" \
  "s_add_u32 m0, s24, 0x16000\n\t" \
  "s_add_u32 s48, s40, 0x200000\n\t" \
  "buffer_load_dwordx4 v2, s[8:11], s48 offen lds\n\t" \
  "v_mfma_f32_16x16x128_f8f6f4 v[176:179], v[112:119], a[160:167], v[176:179]\n\t" \
  "ds_read_b64_tr_b8 v[110:111], v101 offset:9216\n\t" \
  "s_waitcnt lgkmcnt(4)\n\t" \
  "v_mfma_f32_16x16x128_f8f6f4 v[192:195], v[120:127], a[160:167], v[192:195]\n\t" \
  "ds_read_b64_tr_b8 v[112:113], v86 offset:40960\n\t" \
  "s_add_u32 m0, s24, 0x16400\n\t" \
  "s_add_u32 s49, s40, 0x220000\n\t" \
  "buffer_load_dwordx4 v2, s[8:11], s49 offen lds\n\t" \
  "v_mfma_f32_16x16x128_f8f6f4 v[196:199], v[120:127], a[168:175], v[196:199]\n\t" \
  "ds_read_b64_tr_b8 v[114:115], v86 offset:41984\n\t" \
  "v_mfma_f32_16x16x128_f8f6f4 v[200:203], v[120:127], a[176:183], v[200:203]\n\t" \
  "ds_read_b64_tr_b8 v[116:117], v102 offset:8192\n\t" \
  "s_add_u32 m0, s24, 0x0\n\t" \
  "s_add_u32 s50, s41, 0x200000\n\t" \
  "buffer_load_dwordx4 v2, s[8:11], s50 offen lds\n\t" \
  "v_mfma_f32_16x16x128_f8f6f4 v[204:207], v[120:127], a[184:191], v[204:207]\n\t" \
  "ds_read_b64_tr_b8 v[118:119], v102 offset:9216\n\t" \
  "s_waitcnt lgkmcnt(4)\n\t" \
  "v_mfma_f32_16x16x128_f8f6f4 v[220:223], v[104:111], a[184:191], v[220:223]\n\t" \
  "ds_read_b64_tr_b8 v[120:121], v87 offset:40960\n\t" \
  "s_add_u32 m0, s24, 0x400\n\t" \
  "s_add_u32 s51, s41, 0x220000\n\t" \
  "buffer_load_dwordx4 v2, s[8:11], s51 offen lds\n\t" \
  "v_mfma_f32_16x16x128_f8f6f4 v[216:219], v[104:111], a[176:183], v[216:219]\n\t" \
  "ds_read_b64_tr_b8 v[122:123], v87 offset:41984\n\t" \
  "v_mfma_f32_16x16x128_f8f6f4 v[212:215], v[104:111], a[168:175], v[212:215]\n\t" \
  "ds_read_b64_tr_b8 v[124:125], v103 offset:8192\n\t" \
  "s_add_u32 m0, s24, 0x2000\n\t" \
  "s_add_u32 s52, s42, 0x200000\n\t" \
  "buffer_load_dwordx4 v2, s[8:11], s52 offen lds\n\t" \
  "v_mfma_f32_16x16x128_f8f6f4 v[208:211], v[104:111], a[160:167], v[208:211]\n\t" \
  "ds_read_b64_tr_b8 v[126:127], v103 offset:9216\n\t" \
  "s_waitcnt vmcnt(35) lgkmcnt(0)\n\t" \
  "s_barrier\n\t" \
  "v_mfma_f32_16x16x128_f8f6f4 v[224:227], v[112:119], a[160:167], v[224:227]\n\t" \
  "ds_read_b64_tr_b8 v[104:105], v80 offset:49152\n\t" \
  "s_add_u32 m0, s24, 0x2400\n\t" \
  "s_add_u32 s53, s42, 0x220000\n\t" \
  "buffer_load_dwordx4 v2, s[8:11], s53 offen lds\n\t" \
  "v_mfma_f32_16x16x128_f8f6f4 v[228:231], v[112:119], a[168:175], v[228:231]\n\t" \
  "ds_read_b64_tr_b8 v[106:107], v80 offset:50176\n\t" \
  "v_mfma_f32_16x16x128_f8f6f4 v[232:235], v[112:119], a[176:183], v[232:235]\n\t" \
  "ds_read_b64_tr_b8 v[108:109], v96 offset:16384\n\t" \
  "v_mfma_f32_16x16x128_f8f6f4 v[236:239], v[112:119], a[184:191], v[236:239]\n\t" \
  "ds_read_b64_tr_b8 v[110:111], v96 offset:17408\n\t" \
  "v_mfma_f32_16x16x128_f8f6f4 v[252:255], v[120:127], a[184:191], v[252:255]\n\t" \
  "ds_read_b64_tr_b8 v[112:113], v81 offset:49152\n\t" \
  "v_mfma_f32_16x16x128_f8f6f4 v[248:251], v[120:127], a[176:183], v[248:251]\n\t" \
  "ds_read_b64_tr_b8 v[114:115], v81 offset:50176\n\t" \
  "v_mfma_f32_16x16x128_f8f6f4 v[244:247], v[120:127], a[168:175], v[244:247]\n\t" \
  "ds_read_b64_tr_b8 v[116:117], v97 offset:16384\n\t" \
  "v_mfma_f32_16x16x128_f8f6f4 v[240:243], v[120:127], a[160:167], v[240:243]\n\t" \
  "ds_read_b64_tr_b8 v[118:119], v97 offset:17408\n\t" \
  "s_waitcnt vmcnt(28) lgkmcnt(4)\n\t" \
  "v_mfma_f32_16x16x128_f8f6f4 v[128:131], v[104:111], a[192:199], v[128:131]\n\t" \
  "ds_read_b64_tr_b8 v[120:121], v82 offset:49152\n\t" \
  "s_mov_b32 exec_lo, 0x7f00\n\t" \
  "s_mov_b32 exec_hi, 0\n\t" \
  "global_load_dwordx4 v[68:71], v10, s[6:7] sc0 sc1\n\t" \
  "s_mov_b64 exec, -1\n\t" \
  "v_mfma_f32_16x16x128_f8f6f4 v[132:135], v[104:111], a[200:207], v[132:135]\n\t" \
  "ds_read_b64_tr_b8 v[122:123], v82 offset:50176\n\t" \
  "v_mfma_f32_16x16x128_f8f6f4 v[136:139], v[104:111], a[208:215], v[136:139]\n\t" \
  "ds_read_b64_tr_b8 v[124:125], v98 offset:16384\n\t" \
  "v_mfma_f32_16x16x128_f8f6f4 v[140:143], v[104:111], a[216:223], v[140:143]\n\t" \
  "ds_read_b64_tr_b8 v[126:127], v98 offset:17408\n\t" \
  "s_waitcnt lgkmcnt(4)\n\t" \
  "v_mfma_f32_16x16x128_f8f6f4 v[156:159], v[112:119], a[216:223], v[156:159]\n\t" \
  "ds_read_b64_tr_b8 v[104:105], v83 offset:49152\n\t" \
  "v_mfma_f32_16x16x128_f8f6f4 v[152:155], v[112:119], a[208:215], v[152:155]\n\t" \
  "ds_read_b64_tr_b8 v[106:107], v83 offset:50176\n\t" \
  "v_mfma_f32_16x16x128_f8f6f4 v[148:151], v[112:119], a[200:207], v[148:151]\n\t" \
  "ds_read_b64_tr_b8 v[108:109], v99 offset:16384\n\t" \
  "v_mfma_f32_16x16x128_f8f6f4 v[144:147], v[112:119], a[192:199], v[144:147]\n\t" \
  "ds_read_b64_tr_b8 v[110:111], v99 offset:17408\n\t" \
  "s_waitcnt lgkmcnt(4)\n\t" \
  "v_mfma_f32_16x16x128_f8f6f4 v[160:163], v[120:127], a[192:199], v[160:163]\n\t" \
  "ds_read_b64_tr_b8 v[112:113], v84 offset:49152\n\t" \
  "s_add_u32 m0, s24, 0x1a000\n\t" \
  "s_add_u32 s54, s45, 0x0\n\t" \
  "buffer_load_dwordx4 v3, s[8:11], s54 offen lds\n\t" \
  "v_mfma_f32_16x16x128_f8f6f4 v[164:167], v[120:127], a[200:207], v[164:167]\n\t" \
  "ds_read_b64_tr_b8 v[114:115], v84 offset:50176\n\t" \
  "v_mfma_f32_16x16x128_f8f6f4 v[168:171], v[120:127], a[208:215], v[168:171]\n\t" \
  "ds_read_b64_tr_b8 v[116:117], v100 offset:16384\n\t" \
  "s_add_u32 m0, s24, 0x1a400\n\t" \
  "s_add_u32 s55, s45, 0x20000\n\t" \
  "buffer_load_dwordx4 v3, s[8:11], s55 offen lds\n\t" \
  "v_mfma_f32_16x16x128_f8f6f4 v[172:175], v[120:127], a[216:223], v[172:175]\n\t" \
  "ds_read_b64_tr_b8 v[118:119], v100 offset:17408\n\t" \
  "s_waitcnt lgkmcnt(4)\n\t" \
  "v_mfma_f32_16x16x128_f8f6f4 v[188:191], v[104:111], a[216:223], v[188:191]\n\t" \
  "ds_read_b64_tr_b8 v[120:121], v85 offset:49152\n\t" \
  "s_and_b32 s64, s56, 0x20\n\t" \
  "s_cmp_eq_u32 s64, 0x20\n\t" \
  "s_cbranch_scc1 Lok5_0_%=\n\t" \
  "s_mov_b32 s62, 0x20\n\t" \
  "s_mov_b32 s69, 0x7f\n\t" \
  "s_call_b64 s[60:61], Lpoll_0_%=\n\t" \
  "Lok5_0_%=:\n\t" \
  "s_add_u32 s48, s68, 0x90000\n\t" \
  "buffer_load_dwordx4 a[32:35], v4, s[12:15], s48 offen offset:0 sc0 sc1\n\t" \
  "v_mfma_f32_16x16x128_f8f6f4 v[184:187], v[104:111], a[208:215], v[184:187]\n\t" \
  "ds_read_b64_tr_b8 v[122:123], v85 offset:50176\n\t" \
  "v_mfma_f32_16x16x128_f8f6f4 v[180:183], v[104:111], a[200:207], v[180:183]\n\t" \
  "ds_read_b64_tr_b8 v[124:125], v101 offset:16384\n\t" \
  "s_add_u32 s49, s68, 0x90000\n\t" \
  "buffer_load_dwordx4 a[36:39], v4, s[12:15], s49 offen offset:1024 sc0 sc1\n\t" \
  "v_mfma_f32_16x16x128_f8f6f4 v[176:179], v[104:111], a[192:199], v[176:179]\n\t" \
  "ds_read_b64_tr_b8 v[126:127], v101 offset:17408\n\t" \
  "s_waitcnt lgkmcnt(4)\n\t" \
  "v_mfma_f32_16x16x128_f8f6f4 v[192:195], v[112:119], a[192:199], v[192:195]\n\t" \
  "ds_read_b64_tr_b8 v[104:105], v86 offset:49152\n\t" \
  "s_add_u32 s50, s68, 0x90000\n\t" \
  "buffer_load_dwordx4 a[40:43], v4, s[12:15], s50 offen offset:2048 sc0 sc1\n\t" \
  "v_mfma_f32_16x16x128_f8f6f4 v[196:199], v[112:119], a[200:207], v[196:199]\n\t" \
  "ds_read_b64_tr_b8 v[106:107], v86 offset:50176\n\t" \
  "v_mfma_f32_16x16x128_f8f6f4 v[200:203], v[112:119], a[208:215], v[200:203]\n\t" \
  "ds_read_b64_tr_b8 v[108:109], v102 offset:16384\n\t" \
  "s_add_u32 s51, s68, 0x90000\n\t" \
  "buffer_load_dwordx4 a[44:47], v4, s[12:15], s51 offen offset:3072 sc0 sc1\n\t" \
  "v_mfma_f32_16x16x128_f8f6f4 v[204:207], v[112:119], a[216:223], v[204:207]\n\t" \
  "ds_read_b64_tr_b8 v[110:111], v102 offset:17408\n\t" \
  "s_waitcnt lgkmcnt(4)\n\t" \
  "v_mfma_f32_16x16x128_f8f6f4 v[220:223], v[120:127], a[216:223], v[220:223]\n\t" \
  "ds_read_b64_tr_b8 v[112:113], v87 offset:49152\n\t" \
  "s_add_u32 s52, s68, 0x91000\n\t" \
  "buffer_load_dwordx4 a[48:51], v4, s[12:15], s52 offen offset:0 sc0 sc1\n\t" \
  "v_mfma_f32_16x16x128_f8f6f4 v[216:219], v[120:127], a[208:215], v[216:219]\n\t" \
  "ds_read_b64_tr_b8 v[114:115], v87 offset:50176\n\t" \
  "v_mfma_f32_16x16x128_f8f6f4 v[212:215], v[120:127], a[200:207], v[212:215]\n\t" \
  "ds_read_b64_tr_b8 v[116:117], v103 offset:16384\n\t" \
  "s_add_u32 s53, s68, 0x91000\n\t" \
  "buffer_load_dwordx4 a[52:55], v4, s[12:15], s53 offen offset:1024 sc0 sc1\n\t" \
  "v_mfma_f32_16x16x128_f8f6f4 v[208:211], v[120:127], a[192:199], v[208:211]\n\t" \
  "ds_read_b64_tr_b8 v[118:119], v103 offset:17408\n\t" \
  "s_waitcnt vmcnt(35) lgkmcnt(0)\n\t" \
  "s_barrier\n\t" \
  "v_mfma_f32_16x16x128_f8f6f4 v[224:227], v[104:111], a[192:199], v[224:227]\n\t" \
  "ds_read_b64_tr_b8 v[120:121], v80 offset:57344\n\t" \
  "s_add_u32 s54, s68, 0x91000\n\t" \
  "buffer_load_dwordx4 a[56:59], v4, s[12:15], s54 offen offset:2048 sc0 sc1\n\t" \
  "v_mfma_f32_16x16x128_f8f6f4 v[228:231], v[104:111], a[200:207], v[228:231]\n\t" \
  "ds_read_b64_tr_b8 v[122:123], v80 offset:58368\n\t" \
  "v_mfma_f32_16x16x128_f8f6f4 v[232:235], v[104:111], a[208:215], v[232:235]\n\t" \
  "ds_read_b64_tr_b8 v[124:125], v96 offset:24576\n\t" \
  "s_add_u32 s55, s68, 0x91000\n\t" \
  "buffer_load_dwordx4 a[60:63], v4, s[12:15], s55 offen offset:3072 sc0 sc1\n\t" \
  "v_mfma_f32_16x16x128_f8f6f4 v[236:239], v[104:111], a[216:223], v[236:239]\n\t" \
  "ds_read_b64_tr_b8 v[126:127], v96 offset:25600\n\t" \
  "v_mfma_f32_16x16x128_f8f6f4 v[252:255], v[112:119], a[216:223], v[252:255]\n\t" \
  "ds_read_b64_tr_b8 v[104:105], v81 offset:57344\n\t" \
  "s_add_u32 m0, s24, 0x1c000\n\t" \
  "s_add_u32 s48, s46, 0x0\n\t" \
  "buffer_load_dwordx4 v3, s[8:11], s48 offen lds\n\t" \
  "v_mfma_f32_16x16x128_f8f6f4 v[248:251], v[112:119], a[208:215], v[248:251]\n\t" \
  "ds_read_b64_tr_b8 v[106:107], v81 offset:58368\n\t" \
  "v_mfma_f32_16x16x128_f8f6f4 v[244:247], v[112:119], a[200:207], v[244:247]\n\t" \
  "ds_read_b64_tr_b8 v[108:109], v97 offset:24576\n\t" \
  "s_add_u32 m0, s24, 0x1c400\n\t" \
  "s_add_u32 s49, s46, 0x20000\n\t" \
  "buffer_load_dwordx4 v3, s[8:11], s49 offen lds\n\t" \
  "v_mfma_f32_16x16x128_f8f6f4 v[240:243], v[112:119], a[192:199], v[240:243]\n\t" \
  "ds_read_b64_tr_b8 v[110:111], v97 offset:25600\n\t" \
  "s_waitcnt vmcnt(31) lgkmcnt(4)\n\t" \
  "v_mfma_f32_16x16x128_f8f6f4 v[128:131], v[120:127], a[224:231], v[128:131]\n\t" \
  "ds_read_b64_tr_b8 v[112:113], v82 offset:57344\n\t" \
  "s_and_b32 s64, s56, 0x40\n\t" \
  "s_cmp_eq_u32 s64, 0x40\n\t" \
  "s_cbranch_scc1 Lok6_0_%=\n\t" \
  "s_mov_b32 s62, 0x40\n\t" \
  "s_mov_b32 s69, 0x7f\n\t" \
  "s_call_b64 s[60:61], Lpoll_0_%=\n\t" \
  "Lok6_0_%=:\n\t" \
  "s_add_u32 s50, s68, 0xa0000\n\t" \
  "buffer_load_dwordx4 a[64:67], v4, s[12:15], s50 offen offset:0 sc0 sc1\n\t" \
  "v_mfma_f32_16x16x128_f8f6f4 v[132:135], v[120:127], a[232:239], v[132:135]\n\t" \
  "ds_read_b64_tr_b8 v[114:115], v82 offset:58368\n\t" \
  "v_mfma_f32_16x16x128_f8f6f4 v[136:139], v[120:127], a[240:247], v[136:139]\n\t" \
  "ds_read_b64_tr_b8 v[116:117], v98 offset:24576\n\t" \
  "s_add_u32 s51, s68, 0xa0000\n\t" \
  "buffer_load_dwordx4 a[68:71], v4, s[12:15], s51 offen offset:1024 sc0 sc1\n\t" \
  "v_mfma_f32_16x16x128_f8f6f4 v[140:143], v[120:127], a[248:255], v[140:143]\n\t" \
  "ds_read_b64_tr_b8 v[118:119], v98 offset:25600\n\t" \
  "s_waitcnt lgkmcnt(4)\n\t" \
  "v_mfma_f32_16x16x128_f8f6f4 v[156:159], v[104:111], a[248:255], v[156:159]\n\t" \
  "ds_read_b64_tr_b8 v[120:121], v83 offset:57344\n\t" \
  "s_add_u32 s52, s68, 0xa0000\n\t" \
  "buffer_load_dwordx4 a[72:75], v4, s[12:15], s52 offen offset:2048 sc0 sc1\n\t" \
  "v_mfma_f32_16x16x128_f8f6f4 v[152:155], v[104:111], a[240:247], v[152:155]\n\t" \
  "ds_read_b64_tr_b8 v[122:123], v83 offset:58368\n\t" \
  "v_mfma_f32_16x16x128_f8f6f4 v[148:151], v[104:111], a[232:239], v[148:151]\n\t" \
  "ds_read_b64_tr_b8 v[124:125], v99 offset:24576\n\t" \
  "s_add_u32 s53, s68, 0xa0000\n\t" \
  "buffer_load_dwordx4 a[76:79], v4, s[12:15], s53 offen offset:3072 sc0 sc1\n\t" \
  "v_mfma_f32_16x16x128_f8f6f4 v[144:147], v[104:111], a[224:231], v[144:147]\n\t" \
  "ds_read_b64_tr_b8 v[126:127], v99 offset:25600\n\t" \
  "s_waitcnt lgkmcnt(4)\n\t" \
  "v_mfma_f32_16x16x128_f8f6f4 v[160:163], v[112:119], a[224:231], v[160:163]\n\t" \
  "ds_read_b64_tr_b8 v[104:105], v84 offset:57344\n\t" \
  "s_add_u32 s54, s68, 0xa1000\n\t" \
  "buffer_load_dwordx4 a[80:83], v4, s[12:15], s54 offen offset:0 sc0 sc1\n\t" \
  "v_mfma_f32_16x16x128_f8f6f4 v[164:167], v[112:119], a[232:239], v[164:167]\n\t" \
  "ds_read_b64_tr_b8 v[106:107], v84 offset:58368\n\t" \
  "v_mfma_f32_16x16x128_f8f6f4 v[168:171], v[112:119], a[240:247], v[168:171]\n\t" \
  "ds_read_b64_tr_b8 v[108:109], v100 offset:24576\n\t" \
  "s_add_u32 s55, s68, 0xa1000\n\t" \
  "buffer_load_dwordx4 a[84:87], v4, s[12:15], s55 offen offset:1024 sc0 sc1\n\t" \
  "v_mfma_f32_16x16x128_f8f6f4 v[172:175], v[112:119], a[248:255], v[172:175]\n\t" \
  "ds_read_b64_tr_b8 v[110:111], v100 offset:25600\n\t" \
  "s_waitcnt lgkmcnt(4)\n\t" \
  "v_mfma_f32_16x16x128_f8f6f4 v[188:191], v[120:127], a[248:255], v[188:191]\n\t" \
  "ds_read_b64_tr_b8 v[112:113], v85 offset:57344\n\t" \
  "s_add_u32 s48, s68, 0xa1000\n\t" \
  "buffer_load_dwordx4 a[88:91], v4, s[12:15], s48 offen offset:2048 sc0 sc1\n\t" \
  "v_mfma_f32_16x16x128_f8f6f4 v[184:187], v[120:127], a[240:247], v[184:187]\n\t" \
  "ds_read_b64_tr_b8 v[114:115], v85 offset:58368\n\t" \
  "v_mfma_f32_16x16x128_f8f6f4 v[180:183], v[120:127], a[232:239], v[180:183]\n\t" \
  "ds_read_b64_tr_b8 v[116:117], v101 offset:24576\n\t" \
  "s_add_u32 s49, s68, 0xa1000\n\t" \
  "buffer_load_dwordx4 a[92:95], v4, s[12:15], s49 offen offset:3072 sc0 sc1\n\t" \
  "v_mfma_f32_16x16x128_f8f6f4 v[176:179], v[120:127], a[224:231], v[176:179]\n\t" \
  "ds_read_b64_tr_b8 v[118:119], v101 offset:25600\n\t" \
  "s_waitcnt lgkmcnt(4)\n\t" \
  "v_mfma_f32_16x16x128_f8f6f4 v[192:195], v[104:111], a[224:231], v[192:195]\n\t" \
  "ds_read_b64_tr_b8 v[120:121], v86 offset:57344\n\t" \
  "s_add_u32 m0, s24, 0x1e000\n\t" \
  "s_add_u32 s50, s40, 0x200000\n\t" \
  "buffer_load_dwordx4 v3, s[8:11], s50 offen lds\n\t" \
  "v_mfma_f32_16x16x128_f8f6f4 v[196:199], v[104:111], a[232:239], v[196:199]\n\t" \
  "ds_read_b64_tr_b8 v[122:123], v86 offset:58368\n\t" \
  "v_mfma_f32_16x16x128_f8f6f4 v[200:203], v[104:111], a[240:247], v[200:203]\n\t" \
  "ds_read_b64_tr_b8 v[124:125], v102 offset:24576\n\t" \
  "s_add_u32 m0, s24, 0x1e400\n\t" \
  "s_add_u32 s51, s40, 0x220000\n\t" \
  "buffer_load_dwordx4 v3, s[8:11], s51 offen lds\n\t" \
  "v_mfma_f32_16x16x128_f8f6f4 v[204:207], v[104:111], a[248:255], v[204:207]\n\t" \
  "ds_read_b64_tr_b8 v[126:127], v102 offset:25600\n\t" \
  "s_waitcnt lgkmcnt(4)\n\t" \
  "v_mfma_f32_16x16x128_f8f6f4 v[220:223], v[112:119], a[248:255], v[220:223]\n\t" \
  "ds_read_b64_tr_b8 v[104:105], v87 offset:57344\n\t" \
  "s_waitcnt vmcnt(22)\n\t" \
  "v_min_u32 v68, v68, v69\n\t" \
  "v_min3_u32 v68, v68, v70, v71\n\t" \
  "v_cmp_le_u32 vcc, s57, v68\n\t" \
  "s_and_b32 s64, vcc_lo, 0x7f00\n\t" \
  "s_or_b32 s56, s56, s64\n\t" \
  "s_and_b32 s64, s56, 0x7f00\n\t" \
  "s_cmp_eq_u32 s64, 0x7f00\n\t" \
  "s_cbranch_scc1 Lok7_0_%=\n\t" \
  "s_mov_b32 s62, 0x7f00\n\t" \
  "s_mov_b32 s69, 0x7f00\n\t" \
  "s_call_b64 s[60:61], Lpoll_0_%=\n\t" \
  "Lok7_0_%=:\n\t" \
  "s_add_u32 s52, s68, 0xb0000\n\t" \
  "buffer_load_dwordx4 a[96:99], v4, s[12:15], s52 offen offset:0 sc0 sc1\n\t" \
  "v_mfma_f32_16x16x128_f8f6f4 v[216:219], v[112:119], a[240:247], v[216:219]\n\t" \
  "ds_read_b64_tr_b8 v[106:107], v87 offset:58368\n\t" \
  "v_mfma_f32_16x16x128_f8f6f4 v[212:215], v[112:119], a[232:239], v[212:215]\n\t" \
  "ds_read_b64_tr_b8 v[108:109], v103 offset:24576\n\t" \
  "s_add_u32 s53, s68, 0xb0000\n\t" \
  "buffer_load_dwordx4 a[100:103], v4, s[12:15], s53 offen offset:1024 sc0 sc1\n\t" \
  "v_mfma_f32_16x16x128_f8f6f4 v[208:211], v[112:119], a[224:231], v[208:211]\n\t" \
  "ds_read_b64_tr_b8 v[110:111], v103 offset:25600\n\t" \
  "s_waitcnt lgkmcnt(0)\n\t" \
  "s_barrier\n\t" \
  "v_mfma_f32_16x16x128_f8f6f4 v[224:227], v[120:127], a[224:231], v[224:227]\n\t" \
  "ds_read_b64_tr_b8 v[112:113], v88 offset:0\n\t" \
  "s_add_u32 s54, s68, 0xb0000\n\t" \
  "buffer_load_dwordx4 a[104:107], v4, s[12:15], s54 offen offset:2048 sc0 sc1\n\t" \
  "v_mfma_f32_16x16x128_f8f6f4 v[228:231], v[120:127], a[232:239], v[228:231]\n\t" \
  "ds_read_b64_tr_b8 v[114:115], v88 offset:1024\n\t" \
  "v_mfma_f32_16x16x128_f8f6f4 v[232:235], v[120:127], a[240:247], v[232:235]\n\t" \
  "ds_read_b64_tr_b8 v[116:117], v88 offset:32768\n\t" \
  "s_add_u32 s55, s68, 0xb0000\n\t" \
  "buffer_load_dwordx4 a[108:111], v4, s[12:15], s55 offen offset:3072 sc0 sc1\n\t" \
  "v_mfma_f32_16x16x128_f8f6f4 v[236:239], v[120:127], a[248:255], v[236:239]\n\t" \
  "ds_read_b64_tr_b8 v[118:119], v88 offset:33792\n\t" \
  "v_mfma_f32_16x16x128_f8f6f4 v[252:255], v[104:111], a[248:255], v[252:255]\n\t" \
  "ds_read_b64_tr_b8 v[120:121], v89 offset:0\n\t" \
  "s_add_u32 s48, s68, 0xb1000\n\t" \
  "buffer_load_dwordx4 a[112:115], v4, s[12:15], s48 offen offset:0 sc0 sc1\n\t" \
  "v_mfma_f32_16x16x128_f8f6f4 v[248:251], v[104:111], a[240:247], v[248:251]\n\t" \
  "ds_read_b64_tr_b8 v[122:123], v89 offset:1024\n\t" \
  "v_mfma_f32_16x16x128_f8f6f4 v[244:247], v[104:111], a[232:239], v[244:247]\n\t" \
  "ds_read_b64_tr_b8 v[124:125], v89 offset:32768\n\t" \
  "s_add_u32 s49, s68, 0xb1000\n\t" \
  "buffer_load_dwordx4 a[116:119], v4, s[12:15], s49 offen offset:1024 sc0 sc1\n\t" \
  "v_mfma_f32_16x16x128_f8f6f4 v[240:243], v[104:111], a[224:231], v[240:243]\n\t" \
  "ds_read_b64_tr_b8 v[126:127], v89 offset:33792\n\t" \
  "s_waitcnt lgkmcnt(4)\n\t" \
  "v_mfma_f32_16x16x128_f8f6f4 v[128:131], v[112:119], a[0:7], v[128:131]\n\t" \
  "ds_read_b64_tr_b8 v[104:105], v90 offset:0\n\t" \
  "s_add_u32 s50, s68, 0xb1000\n\t" \
  "buffer_load_dwordx4 a[120:123], v4, s[12:15], s50 offen offset:2048 sc0 sc1\n\t" \
  "v_mfma_f32_16x16x128_f8f6f4 v[132:135], v[112:119], a[8:15], v[132:135]\n\t" \
  "ds_read_b64_tr_b8 v[106:107], v90 offset:1024\n\t" \
  "v_mfma_f32_16x16x128_f8f6f4 v[136:139], v[112:119], a[16:23], v[136:139]\n\t" \
  "ds_read_b64_tr_b8 v[108:109], v90 offset:32768\n\t" \
  "s_add_u32 s51, s68, 0xb1000\n\t" \
  "buffer_load_dwordx4 a[124:127], v4, s[12:15], s51 offen offset:3072 sc0 sc1\n\t" \
  "v_mfma_f32_16x16x128_f8f6f4 v[140:143], v[112:119], a[24:31], v[140:143]\n\t" \
  "ds_read_b64_tr_b8 v[110:111], v90 offset:33792\n\t" \
  "s_waitcnt lgkmcnt(4)\n\t" \
  "v_mfma_f32_16x16x128_f8f6f4 v[156:159], v[120:127], a[24:31], v[156:159]\n\t" \
  "ds_read_b64_tr_b8 v[112:113], v91 offset:0\n\t" \
  "s_add_u32 m0, s24, 0x20000\n\t" \
  "s_add_u32 s52, s41, 0x200000\n\t" \
  "buffer_load_dwordx4 v3, s[8:11], s52 offen lds\n\t" \
  "v_mfma_f32_16x16x128_f8f6f4 v[152:155], v[120:127], a[16:23], v[152:155]\n\t" \
  "ds_read_b64_tr_b8 v[114:115], v91 offset:1024\n\t" \
  "v_mfma_f32_16x16x128_f8f6f4 v[148:151], v[120:127], a[8:15], v[148:151]\n\t" \
  "ds_read_b64_tr_b8 v[116:117], v91 offset:32768\n\t" \
  "s_add_u32 m0, s24, 0x20400\n\t" \
  "s_add_u32 s53, s41, 0x220000\n\t" \
  "buffer_load_dwordx4 v3, s[8:11], s53 offen lds\n\t" \
  "v_mfma_f32_16x16x128_f8f6f4 v[144:147], v[120:127], a[0:7], v[144:147]\n\t" \
  "ds_read_b64_tr_b8 v[118:119], v91 offset:33792\n\t" \
  "s_waitcnt lgkmcnt(4)\n\t" \
  "v_mfma_f32_16x16x128_f8f6f4 v[160:163], v[104:111], a[0:7], v[160:163]\n\t" \
  "ds_read_b64_tr_b8 v[120:121], v92 offset:0\n\t" \
  "s_add_u32 m0, s24, 0x4000\n\t" \
  "s_add_u32 s54, s43, 0x200000\n\t" \
  "buffer_load_dwordx4 v2, s[8:11], s54 offen lds\n\t" \
  "v_mfma_f32_16x16x128_f8f6f4 v[164:167], v[104:111], a[8:15], v[164:167]\n\t" \
  "ds_read_b64_tr_b8 v[122:123], v92 offset:1024\n\t" \
  "v_mfma_f32_16x16x128_f8f6f4 v[168:171], v[104:111], a[16:23], v[168:171]\n\t" \
  "ds_read_b64_tr_b8 v[124:125], v92 offset:32768\n\t" \
  "s_add_u32 m0, s24, 0x4400\n\t" \
  "s_add_u32 s55, s43, 0x220000\n\t" \
  "buffer_load_dwordx4 v2, s[8:11], s55 offen lds\n\t" \
  "v_mfma_f32_16x16x128_f8f6f4 v[172:175], v[104:111], a[24:31], v[172:175]\n\t" \
  "ds_read_b64_tr_b8 v[126:127], v92 offset:33792\n\t" \
  "s_waitcnt lgkmcnt(4)\n\t" \
  "v_mfma_f32_16x16x128_f8f6f4 v[188:191], v[112:119], a[24:31], v[188:191]\n\t" \
  "ds_read_b64_tr_b8 v[104:105], v93 offset:0\n\t" \
  "s_add_u32 m0, s24, 0x6000\n\t" \
  "s_add_u32 s48, s44, 0x200000\n\t" \
  "buffer_load_dwordx4 v2, s[8:11], s48 offen lds\n\t" \
  "v_mfma_f32_16x16x128_f8f6f4 v[184:187], v[112:119], a[16:23], v[184:187]\n\t" \
  "ds_read_b64_tr_b8 v[106:107], v93 offset:1024\n\t" \
  "v_mfma_f32_16x16x128_f8f6f4 v[180:183], v[112:119], a[8:15], v[180:183]\n\t" \
  "ds_read_b64_tr_b8 v[108:109], v93 offset:32768\n\t" \
  "s_add_u32 m0, s24, 0x6400\n\t" \
  "s_add_u32 s49, s44, 0x220000\n\t" \
  "buffer_load_dwordx4 v2, s[8:11], s49 offen lds\n\t" \
  "v_mfma_f32_16x16x128_f8f6f4 v[176:179], v[112:119], a[0:7], v[176:179]\n\t" \
  "ds_read_b64_tr_b8 v[110:111], v93 offset:33792\n\t" \
  "s_waitcnt lgkmcnt(4)\n\t" \
  "v_mfma_f32_16x16x128_f8f6f4 v[192:195], v[120:127], a[0:7], v[192:195]\n\t" \
  "ds_read_b64_tr_b8 v[112:113], v94 offset:0\n\t" \
  "s_add_u32 m0, s24, 0x8000\n\t" \
  "s_add_u32 s50, s45, 0x200000\n\t" \
  "buffer_load_dwordx4 v2, s[8:11], s50 offen lds\n\t" \
  "v_mfma_f32_16x16x128_f8f6f4 v[196:199], v[120:127], a[8:15], v[196:199]\n\t" \
  "ds_read_b64_tr_b8 v[114:115], v94 offset:1024\n\t" \
  "v_mfma_f32_16x16x128_f8f6f4 v[200:203], v[120:127], a[16:23], v[200:203]\n\t" \
  "ds_read_b64_tr_b8 v[116:117], v94 offset:32768\n\t" \
  "s_add_u32 m0, s24, 0x8400\n\t" \
  "s_add_u32 s51, s45, 0x220000\n\t" \
  "buffer_load_dwordx4 v2, s[8:11], s51 offen lds\n\t" \
  "v_mfma_f32_16x16x128_f8f6f4 v[204:207], v[120:127], a[24:31], v[204:207]\n\t" \
  "ds_read_b64_tr_b8 v[118:119], v94 offset:33792\n\t" \
  "s_waitcnt lgkmcnt(4)\n\t" \
  "v_mfma_f32_16x16x128_f8f6f4 v[220:223], v[104:111], a[24:31], v[220:223]\n\t" \
  "ds_read_b64_tr_b8 v[120:121], v95 offset:0\n\t" \
  "s_add_u32 m0, s24, 0xa000\n\t" \
  "s_add_u32 s52, s46, 0x200000\n\t" \
  "buffer_load_dwordx4 v2, s[8:11], s52 offen lds\n\t" \
  "v_mfma_f32_16x16x128_f8f6f4 v[216:219], v[104:111], a[16:23], v[216:219]\n\t" \
  "ds_read_b64_tr_b8 v[122:123], v95 offset:1024\n\t" \
  "v_mfma_f32_16x16x128_f8f6f4 v[212:215], v[104:111], a[8:15], v[212:215]\n\t" \
  "ds_read_b64_tr_b8 v[124:125], v95 offset:32768\n\t" \
  "s_add_u32 m0, s24, 0xa400\n\t" \
  "s_add_u32 s53, s46, 0x220000\n\t" \
  "buffer_load_dwordx4 v2, s[8:11], s53 offen lds\n\t" \
  "v_mfma_f32_16x16x128_f8f6f4 v[208:211], v[104:111], a[0:7], v[208:211]\n\t" \
  "ds_read_b64_tr_b8 v[126:127], v95 offset:33792\n\t" \
  "s_waitcnt vmcnt(38) lgkmcnt(0)\n\t" \
  "s_barrier\n\t" \
  "v_mfma_f32_16x16x128_f8f6f4 v[224:227], v[112:119], a[0:7], v[224:227]\n\t" \
  "ds_read_b64_tr_b8 v[104:105], v88 offset:8192\n\t" \
  "v_mfma_f32_16x16x128_f8f6f4 v[228:231], v[112:119], a[8:15], v[228:231]\n\t" \
  "ds_read_b64_tr_b8 v[106:107], v88 offset:9216\n\t" \
  "v_mfma_f32_16x16x128_f8f6f4 v[232:235], v[112:119], a[16:23], v[232:235]\n\t" \
  "ds_read_b64_tr_b8 v[108:109], v88 offset:40960\n\t" \
  "v_mfma_f32_16x16x128_f8f6f4 v[236:239], v[112:119], a[24:31], v[236:239]\n\t" \
  "ds_read_b64_tr_b8 v[110:111], v88 offset:41984\n\t" \
  "v_mfma_f32_16x16x128_f8f6f4 v[252:255], v[120:127], a[24:31], v[252:255]\n\t" \
  "ds_read_b64_tr_b8 v[112:113], v89 offset:8192\n\t" \
  "v_mfma_f32_16x16x128_f8f6f4 v[248:251], v[120:127], a[16:23], v[248:251]\n\t" \
  "ds_read_b64_tr_b8 v[114:115], v89 offset:9216\n\t" \
  "v_mfma_f32_16x16x128_f8f6f4 v[244:247], v[120:127], a[8:15], v[244:247]\n\t" \
  "ds_read_b64_tr_b8 v[116:117], v89 offset:40960\n\t" \
  "v_mfma_f32_16x16x128_f8f6f4 v[240:243], v[120:127], a[0:7], v[240:243]\n\t" \
  "ds_read_b64_tr_b8 v[118:119], v89 offset:41984\n\t" \
  "s_waitcnt vmcnt(30) lgkmcnt(4)\n\t" \
  "v_mfma_f32_16x16x128_f8f6f4 v[128:131], v[104:111], a[32:39], v[128:131]\n\t" \
  "ds_read_b64_tr_b8 v[120:121], v90 offset:8192\n\t" \
  "v_mfma_f32_16x16x128_f8f6f4 v[132:135], v[104:111], a[40:47], v[132:135]\n\t" \
  "ds_read_b64_tr_b8 v[122:123], v90 offset:9216\n\t" \
  "v_mfma_f32_16x16x128_f8f6f4 v[136:139], v[104:111], a[48:55], v[136:139]\n\t" \
  "ds_read_b64_tr_b8 v[124:125], v90 offset:40960\n\t" \
  "v_mfma_f32_16x16x128_f8f6f4 v[140:143], v[104:111], a[56:63], v[140:143]\n\t" \
  "ds_read_b64_tr_b8 v[126:127], v90 offset:41984\n\t" \
  "s_waitcnt lgkmcnt(4)\n\t" \
  "v_mfma_f32_16x16x128_f8f6f4 v[156:159], v[112:119], a[56:63], v[156:159]\n\t" \
  "ds_read_b64_tr_b8 v[104:105], v91 offset:8192\n\t" \
  "v_mfma_f32_16x16x128_f8f6f4 v[152:155], v[112:119], a[48:55], v[152:155]\n\t" \
  "ds_read_b64_tr_b8 v[106:107], v91 offset:9216\n\t" \
  "v_mfma_f32_16x16x128_f8f6f4 v[148:151], v[112:119], a[40:47], v[148:151]\n\t" \
  "ds_read_b64_tr_b8 v[108:109], v91 offset:40960\n\t" \
  "s_add_u32 s54, s68, 0xc0000\n\t" \
  "buffer_load_dwordx4 a[128:131], v4, s[12:15], s54 offen offset:0 sc0 sc1\n\t" \
  "v_mfma_f32_16x16x128_f8f6f4 v[144:147], v[112:119], a[32:39], v[144:147]\n\t" \
  "ds_read_b64_tr_b8 v[110:111], v91 offset:41984\n\t" \
  "s_waitcnt lgkmcnt(4)\n\t" \
  "v_mfma_f32_16x16x128_f8f6f4 v[160:163], v[120:127], a[32:39], v[160:163]\n\t" \
  "ds_read_b64_tr_b8 v[112:113], v92 offset:8192\n\t" \
  "s_add_u32 s55, s68, 0xc0000\n\t" \
  "buffer_load_dwordx4 a[132:135], v4, s[12:15], s55 offen offset:1024 sc0 sc1\n\t" \
  "v_mfma_f32_16x16x128_f8f6f4 v[164:167], v[120:127], a[40:47], v[164:167]\n\t" \
  "ds_read_b64_tr_b8 v[114:115], v92 offset:9216\n\t" \
  "v_mfma_f32_16x16x128_f8f6f4 v[168:171], v[120:127], a[48:55], v[168:171]\n\t" \
  "ds_read_b64_tr_b8 v[116:117], v92 offset:40960\n\t" \
  "s_add_u32 s48, s68, 0xc0000\n\t" \
  "buffer_load_dwordx4 a[136:139], v4, s[12:15], s48 offen offset:2048 sc0 sc1\n\t" \
  "v_mfma_f32_16x16x128_f8f6f4 v[172:175], v[120:127], a[56:63], v[172:175]\n\t" \
  "ds_read_b64_tr_b8 v[118:119], v92 offset:41984\n\t" \
  "s_waitcnt lgkmcnt(4)\n\t" \
  "v_mfma_f32_16x16x128_f8f6f4 v[188:191], v[104:111], a[56:63], v[188:191]\n\t" \
  "ds_read_b64_tr_b8 v[120:121], v93 offset:8192\n\t" \
  "s_add_u32 s49, s68, 0xc0000\n\t" \
  "buffer_load_dwordx4 a[140:143], v4, s[12:15], s49 offen offset:3072 sc0 sc1\n\t" \
  "v_mfma_f32_16x16x128_f8f6f4 v[184:187], v[104:111], a[48:55], v[184:187]\n\t" \
  "ds_read_b64_tr_b8 v[122:123], v93 offset:9216\n\t" \
  "v_mfma_f32_16x16x128_f8f6f4 v[180:183], v[104:111], a[40:47], v[180:183]\n\t" \
  "ds_read_b64_tr_b8 v[124:125], v93 offset:40960\n\t" \
  "s_add_u32 s50, s68, 0xc1000\n\t" \
  "buffer_load_dwordx4 a[144:147], v4, s[12:15], s50 offen offset:0 sc0 sc1\n\t" \
  "v_mfma_f32_16x16x128_f8f6f4 v[176:179], v[104:111], a[32:39], v[176:179]\n\t" \
  "ds_read_b64_tr_b8 v[126:127], v93 offset:41984\n\t" \
  "s_waitcnt lgkmcnt(4)\n\t" \
  "v_mfma_f32_16x16x128_f8f6f4 v[192:195], v[112:119], a[32:39], v[192:195]\n\t" \
  "ds_read_b64_tr_b8 v[104:105], v94 offset:8192\n\t" \
  "s_add_u32 s51, s68, 0xc1000\n\t" \
  "buffer_load_dwordx4 a[148:151], v4, s[12:15], s51 offen offset:1024 sc0 sc1\n\t" \
  "v_mfma_f32_16x16x128_f8f6f4 v[196:199], v[112:119], a[40:47], v[196:199]\n\t" \
  "ds_read_b64_tr_b8 v[106:107], v94 offset:9216\n\t" \
  "v_mfma_f32_16x16x128_f8f6f4 v[200:203], v[112:119], a[48:55], v[200:203]\n\t" \
  "ds_read_b64_tr_b8 v[108:109], v94 offset:40960\n\t" \
  "s_add_u32 s52, s68, 0xc1000\n\t" \
  "buffer_load_dwordx4 a[152:155], v4, s[12:15], s52 offen offset:2048 sc0 sc1\n\t" \
  "v_mfma_f32_16x16x128_f8f6f4 v[204:207], v[112:119], a[56:63], v[204:207]\n\t" \
  "ds_read_b64_tr_b8 v[110:111], v94 offset:41984\n\t" \
  "s_waitcnt lgkmcnt(4)\n\t" \
  "v_mfma_f32_16x16x128_f8f6f4 v[220:223], v[120:127], a[56:63], v[220:223]\n\t" \
  "ds_read_b64_tr_b8 v[112:113], v95 offset:8192\n\t" \
  "s_add_u32 s53, s68, 0xc1000\n\t" \
  "buffer_load_dwordx4 a[156:159], v4, s[12:15], s53 offen offset:3072 sc0 sc1\n\t" \
  "v_mfma_f32_16x16x128_f8f6f4 v[216:219], v[120:127], a[48:55], v[216:219]\n\t" \
  "ds_read_b64_tr_b8 v[114:115], v95 offset:9216\n\t" \
  "v_mfma_f32_16x16x128_f8f6f4 v[212:215], v[120:127], a[40:47], v[212:215]\n\t" \
  "ds_read_b64_tr_b8 v[116:117], v95 offset:40960\n\t" \
  "s_add_u32 m0, s24, 0x22000\n\t" \
  "s_add_u32 s54, s42, 0x200000\n\t" \
  "buffer_load_dwordx4 v3, s[8:11], s54 offen lds\n\t" \
  "v_mfma_f32_16x16x128_f8f6f4 v[208:211], v[120:127], a[32:39], v[208:211]\n\t" \
  "ds_read_b64_tr_b8 v[118:119], v95 offset:41984\n\t" \
  "s_waitcnt vmcnt(37) lgkmcnt(0)\n\t" \
  "s_barrier\n\t" \
  "v_mfma_f32_16x16x128_f8f6f4 v[224:227], v[104:111], a[32:39], v[224:227]\n\t" \
  "ds_read_b64_tr_b8 v[120:121], v88 offset:16384\n\t" \
  "s_add_u32 m0, s24, 0x22400\n\t" \
  "s_add_u32 s55, s42, 0x220000\n\t" \
  "buffer_load_dwordx4 v3, s[8:11], s55 offen lds\n\t" \
  "v_mfma_f32_16x16x128_f8f6f4 v[228:231], v[104:111], a[40:47], v[228:231]\n\t" \
  "ds_read_b64_tr_b8 v[122:123], v88 offset:17408\n\t" \
  "v_mfma_f32_16x16x128_f8f6f4 v[232:235], v[104:111], a[48:55], v[232:235]\n\t" \
  "ds_read_b64_tr_b8 v[124:125], v88 offset:49152\n\t" \
  "s_add_u32 s48, s68, 0xd0000\n\t" \
  "buffer_load_dwordx4 a[160:163], v4, s[12:15], s48 offen offset:0 sc0 sc1\n\t" \
  "v_mfma_f32_16x16x128_f8f6f4 v[236:239], v[104:111], a[56:63], v[236:239]\n\t" \
  "ds_read_b64_tr_b8 v[126:127], v88 offset:50176\n\t" \
  "v_mfma_f32_16x16x128_f8f6f4 v[252:255], v[112:119], a[56:63], v[252:255]\n\t" \
  "ds_read_b64_tr_b8 v[104:105], v89 offset:16384\n\t" \
  "s_add_u32 s49, s68, 0xd0000\n\t" \
  "buffer_load_dwordx4 a[164:167], v4, s[12:15], s49 offen offset:1024 sc0 sc1\n\t" \
  "v_mfma_f32_16x16x128_f8f6f4 v[248:251], v[112:119], a[48:55], v[248:251]\n\t" \
  "ds_read_b64_tr_b8 v[106:107], v89 offset:17408\n\t" \
  "v_mfma_f32_16x16x128_f8f6f4 v[244:247], v[112:119], a[40:47], v[244:247]\n\t" \
  "ds_read_b64_tr_b8 v[108:109], v89 offset:49152\n\t" \
  "s_add_u32 s50, s68, 0xd0000\n\t" \
  "buffer_load_dwordx4 a[168:171], v4, s[12:15], s50 offen offset:2048 sc0 sc1\n\t" \
  "v_mfma_f32_16x16x128_f8f6f4 v[240:243], v[112:119], a[32:39], v[240:243]\n\t" \
  "ds_read_b64_tr_b8 v[110:111], v89 offset:50176\n\t" \
  "s_waitcnt vmcnt(33) lgkmcnt(4)\n\t" \
  "v_mfma_f32_16x16x128_f8f6f4 v[128:131], v[120:127], a[64:71], v[128:131]\n\t" \
  "ds_read_b64_tr_b8 v[112:113], v90 offset:16384\n\t" \
  "s_add_u32 s51, s68, 0xd0000\n\t" \
  "buffer_load_dwordx4 a[172:175], v4, s[12:15], s51 offen offset:3072 sc0 sc1\n\t" \
  "v_mfma_f32_16x16x128_f8f6f4 v[132:135], v[120:127], a[72:79], v[132:135]\n\t" \
  "ds_read_b64_tr_b8 v[114:115], v90 offset:17408\n\t" \
  "v_mfma_f32_16x16x128_f8f6f4 v[136:139], v[120:127], a[80:87], v[136:139]\n\t" \
  "ds_read_b64_tr_b8 v[116:117], v90 offset:49152\n\t" \
  "s_add_u32 s52, s68, 0xd1000\n\t" \
  "buffer_load_dwordx4 a[176:179], v4, s[12:15], s52 offen offset:0 sc0 sc1\n\t" \
  "v_mfma_f32_16x16x128_f8f6f4 v[140:143], v[120:127], a[88:95], v[140:143]\n\t" \
  "ds_read_b64_tr_b8 v[118:119], v90 offset:50176\n\t" \
  "s_waitcnt lgkmcnt(4)\n\t" \
  "v_mfma_f32_16x16x128_f8f6f4 v[156:159], v[104:111], a[88:95], v[156:159]\n\t" \
  "ds_read_b64_tr_b8 v[120:121], v91 offset:16384\n\t" \
  "s_add_u32 s53, s68, 0xd1000\n\t" \
  "buffer_load_dwordx4 a[180:183], v4, s[12:15], s53 offen offset:1024 sc0 sc1\n\t" \
  "v_mfma_f32_16x16x128_f8f6f4 v[152:155], v[104:111], a[80:87], v[152:155]\n\t" \
  "ds_read_b64_tr_b8 v[122:123], v91 offset:17408\n\t" \
  "v_mfma_f32_16x16x128_f8f6f4 v[148:151], v[104:111], a[72:79], v[148:151]\n\t" \
  "ds_read_b64_tr_b8 v[124:125], v91 offset:49152\n\t" \
  "s_add_u32 s54, s68, 0xd1000\n\t" \
  "buffer_load_dwordx4 a[184:187], v4, s[12:15], s54 offen offset:2048 sc0 sc1\n\t" \
  "v_mfma_f32_16x16x128_f8f6f4 v[144:147], v[104:111], a[64:71], v[144:147]\n\t" \
  "ds_read_b64_tr_b8 v[126:127], v91 offset:50176\n\t" \
  "s_waitcnt lgkmcnt(4)\n\t" \
  "v_mfma_f32_16x16x128_f8f6f4 v[160:163], v[112:119], a[64:71], v[160:163]\n\t" \
  "ds_read_b64_tr_b8 v[104:105], v92 offset:16384\n\t" \
  "s_add_u32 s55, s68, 0xd1000\n\t" \
  "buffer_load_dwordx4 a[188:191], v4, s[12:15], s55 offen offset:3072 sc0 sc1\n\t" \
  "v_mfma_f32_16x16x128_f8f6f4 v[164:167], v[112:119], a[72:79], v[164:167]\n\t" \
  "ds_read_b64_tr_b8 v[106:107], v92 offset:17408\n\t" \
  "v_mfma_f32_16x16x128_f8f6f4 v[168:171], v[112:119], a[80:87], v[168:171]\n\t" \
  "ds_read_b64_tr_b8 v[108:109], v92 offset:49152\n\t" \
  "s_add_u32 m0, s24, 0x24000\n\t" \
  "s_add_u32 s48, s43, 0x200000\n\t" \
  "buffer_load_dwordx4 v3, s[8:11], s48 offen lds\n\t" \
  "v_mfma_f32_16x16x128_f8f6f4 v[172:175], v[112:119], a[88:95], v[172:175]\n\t" \
  "ds_read_b64_tr_b8 v[110:111], v92 offset:50176\n\t" \
  "s_waitcnt lgkmcnt(4)\n\t" \
  "v_mfma_f32_16x16x128_f8f6f4 v[188:191], v[120:127], a[88:95], v[188:191]\n\t" \
  "ds_read_b64_tr_b8 v[112:113], v93 offset:16384\n\t" \
  "s_add_u32 m0, s24, 0x24400\n\t" \
  "s_add_u32 s49, s43, 0x220000\n\t" \
  "buffer_load_dwordx4 v3, s[8:11], s49 offen lds\n\t" \
  "v_mfma_f32_16x16x128_f8f6f4 v[184:187], v[120:127], a[80:87], v[184:187]\n\t" \
  "ds_read_b64_tr_b8 v[114:115], v93 offset:17408\n\t" \
  "v_mfma_f32_16x16x128_f8f6f4 v[180:183], v[120:127], a[72:79], v[180:183]\n\t" \
  "ds_read_b64_tr_b8 v[116:117], v93 offset:49152\n\t" \
  "s_add_u32 s50, s68, 0xe0000\n\t" \
  "buffer_load_dwordx4 a[192:195], v4, s[12:15], s50 offen offset:0 sc0 sc1\n\t" \
  "v_mfma_f32_16x16x128_f8f6f4 v[176:179], v[120:127], a[64:71], v[176:179]\n\t" \
  "ds_read_b64_tr_b8 v[118:119], v93 offset:50176\n\t" \
  "s_waitcnt lgkmcnt(4)\n\t" \
  "v_mfma_f32_16x16x128_f8f6f4 v[192:195], v[104:111], a[64:71], v[192:195]\n\t" \
  "ds_read_b64_tr_b8 v[120:121], v94 offset:16384\n\t" \
  "s_add_u32 s51, s68, 0xe0000\n\t" \
  "buffer_load_dwordx4 a[196:199], v4, s[12:15], s51 offen offset:1024 sc0 sc1\n\t" \
  "v_mfma_f32_16x16x128_f8f6f4 v[196:199], v[104:111], a[72:79], v[196:199]\n\t" \
  "ds_read_b64_tr_b8 v[122:123], v94 offset:17408\n\t" \
  "v_mfma_f32_16x16x128_f8f6f4 v[200:203], v[104:111], a[80:87], v[200:203]\n\t" \
  "ds_read_b64_tr_b8 v[124:125], v94 offset:49152\n\t" \
  "s_add_u32 s52, s68, 0xe0000\n\t" \
  "buffer_load_dwordx4 a[200:203], v4, s[12:15], s52 offen offset:2048 sc0 sc1\n\t" \
  "v_mfma_f32_16x16x128_f8f6f4 v[204:207], v[104:111], a[88:95], v[204:207]\n\t" \
  "ds_read_b64_tr_b8 v[126:127], v94 offset:50176\n\t" \
  "s_waitcnt lgkmcnt(4)\n\t" \
  "v_mfma_f32_16x16x128_f8f6f4 v[220:223], v[112:119], a[88:95], v[220:223]\n\t" \
  "ds_read_b64_tr_b8 v[104:105], v95 offset:16384\n\t" \
  "s_add_u32 s53, s68, 0xe0000\n\t" \
  "buffer_load_dwordx4 a[204:207], v4, s[12:15], s53 offen offset:3072 sc0 sc1\n\t" \
  "v_mfma_f32_16x16x128_f8f6f4 v[216:219], v[112:119], a[80:87], v[216:219]\n\t" \
  "ds_read_b64_tr_b8 v[106:107], v95 offset:17408\n\t" \
  "v_mfma_f32_16x16x128_f8f6f4 v[212:215], v[112:119], a[72:79], v[212:215]\n\t" \
  "ds_read_b64_tr_b8 v[108:109], v95 offset:49152\n\t" \
  "s_add_u32 s54, s68, 0xe1000\n\t" \
  "buffer_load_dwordx4 a[208:211], v4, s[12:15], s54 offen offset:0 sc0 sc1\n\t" \
  "v_mfma_f32_16x16x128_f8f6f4 v[208:211], v[112:119], a[64:71], v[208:211]\n\t" \
  "ds_read_b64_tr_b8 v[110:111], v95 offset:50176\n\t" \
  "s_waitcnt vmcnt(43) lgkmcnt(0)\n\t" \
  "s_barrier\n\t" \
  "v_mfma_f32_16x16x128_f8f6f4 v[224:227], v[120:127], a[64:71], v[224:227]\n\t" \
  "ds_read_b64_tr_b8 v[112:113], v88 offset:24576\n\t" \
  "s_add_u32 s55, s68, 0xe1000\n\t" \
  "buffer_load_dwordx4 a[212:215], v4, s[12:15], s55 offen offset:1024 sc0 sc1\n\t" \
  "v_mfma_f32_16x16x128_f8f6f4 v[228:231], v[120:127], a[72:79], v[228:231]\n\t" \
  "ds_read_b64_tr_b8 v[114:115], v88 offset:25600\n\t" \
  "v_mfma_f32_16x16x128_f8f6f4 v[232:235], v[120:127], a[80:87], v[232:235]\n\t" \
  "ds_read_b64_tr_b8 v[116:117], v88 offset:57344\n\t" \
  "s_add_u32 s48, s68, 0xe1000\n\t" \
  "buffer_load_dwordx4 a[216:219], v4, s[12:15], s48 offen offset:2048 sc0 sc1\n\t" \
  "v_mfma_f32_16x16x128_f8f6f4 v[236:239], v[120:127], a[88:95], v[236:239]\n\t" \
  "ds_read_b64_tr_b8 v[118:119], v88 offset:58368\n\t" \
  "v_mfma_f32_16x16x128_f8f6f4 v[252:255], v[104:111], a[88:95], v[252:255]\n\t" \
  "ds_read_b64_tr_b8 v[120:121], v89 offset:24576\n\t" \
  "s_add_u32 s49, s68, 0xe1000\n\t" \
  "buffer_load_dwordx4 a[220:223], v4, s[12:15], s49 offen offset:3072 sc0 sc1\n\t" \
  "v_mfma_f32_16x16x128_f8f6f4 v[248:251], v[104:111], a[80:87], v[248:251]\n\t" \
  "ds_read_b64_tr_b8 v[122:123], v89 offset:25600\n\t" \
  "v_mfma_f32_16x16x128_f8f6f4 v[244:247], v[104:111], a[72:79], v[244:247]\n\t" \
  "ds_read_b64_tr_b8 v[124:125], v89 offset:57344\n\t" \
  "s_add_u32 m0, s24, 0x26000\n\t" \
  "s_add_u32 s50, s44, 0x200000\n\t" \
  "buffer_load_dwordx4 v3, s[8:11], s50 offen lds\n\t" \
  "v_mfma_f32_16x16x128_f8f6f4 v[240:243], v[104:111], a[64:71], v[240:243]\n\t" \
  "ds_read_b64_tr_b8 v[126:127], v89 offset:58368\n\t" \
  "s_waitcnt vmcnt(39) lgkmcnt(4)\n\t" \
  "v_mfma_f32_16x16x128_f8f6f4 v[128:131], v[112:119], a[96:103], v[128:131]\n\t" \
  "ds_read_b64_tr_b8 v[104:105], v90 offset:24576\n\t" \
  "s_add_u32 m0, s24, 0x26400\n\t" \
  "s_add_u32 s51, s44, 0x220000\n\t" \
  "buffer_load_dwordx4 v3, s[8:11], s51 offen lds\n\t" \
  "v_mfma_f32_16x16x128_f8f6f4 v[132:135], v[112:119], a[104:111], v[132:135]\n\t" \
  "ds_read_b64_tr_b8 v[106:107], v90 offset:25600\n\t" \
  "v_mfma_f32_16x16x128_f8f6f4 v[136:139], v[112:119], a[112:119], v[136:139]\n\t" \
  "ds_read_b64_tr_b8 v[108:109], v90 offset:57344\n\t" \
  "s_add_u32 s52, s68, 0xf0000\n\t" \
  "buffer_load_dwordx4 a[224:227], v4, s[12:15], s52 offen offset:0 sc0 sc1\n\t" \
  "v_mfma_f32_16x16x128_f8f6f4 v[140:143], v[112:119], a[120:127], v[140:143]\n\t" \
  "ds_read_b64_tr_b8 v[110:111], v90 offset:58368\n\t" \
  "s_waitcnt lgkmcnt(4)\n\t" \
  "v_mfma_f32_16x16x128_f8f6f4 v[156:159], v[120:127], a[120:127], v[156:159]\n\t" \
  "ds_read_b64_tr_b8 v[112:113], v91 offset:24576\n\t" \
  "s_add_u32 s53, s68, 0xf0000\n\t" \
  "buffer_load_dwordx4 a[228:231], v4, s[12:15], s53 offen offset:1024 sc0 sc1\n\t" \
  "v_mfma_f32_16x16x128_f8f6f4 v[152:155], v[120:127], a[112:119], v[152:155]\n\t" \
  "ds_read_b64_tr_b8 v[114:115], v91 offset:25600\n\t" \
  "v_mfma_f32_16x16x128_f8f6f4 v[148:151], v[120:127], a[104:111], v[148:151]\n\t" \
  "ds_read_b64_tr_b8 v[116:117], v91 offset:57344\n\t" \
  "s_add_u32 s54, s68, 0xf0000\n\t" \
  "buffer_load_dwordx4 a[232:235], v4, s[12:15], s54 offen offset:2048 sc0 sc1\n\t" \
  "v_mfma_f32_16x16x128_f8f6f4 v[144:147], v[120:127], a[96:103], v[144:147]\n\t" \
  "ds_read_b64_tr_b8 v[118:119], v91 offset:58368\n\t" \
  "s_waitcnt lgkmcnt(4)\n\t" \
  "v_mfma_f32_16x16x128_f8f6f4 v[160:163], v[104:111], a[96:103], v[160:163]\n\t" \
  "ds_read_b64_tr_b8 v[120:121], v92 offset:24576\n\t" \
  "s_add_u32 s55, s68, 0xf0000\n\t" \
  "buffer_load_dwordx4 a[236:239], v4, s[12:15], s55 offen offset:3072 sc0 sc1\n\t" \
  "v_mfma_f32_16x16x128_f8f6f4 v[164:167], v[104:111], a[104:111], v[164:167]\n\t" \
  "ds_read_b64_tr_b8 v[122:123], v92 offset:25600\n\t" \
  "v_mfma_f32_16x16x128_f8f6f4 v[168:171], v[104:111], a[112:119], v[168:171]\n\t" \
  "ds_read_b64_tr_b8 v[124:125], v92 offset:57344\n\t" \
  "s_add_u32 s48, s68, 0xf1000\n\t" \
  "buffer_load_dwordx4 a[240:243], v4, s[12:15], s48 offen offset:0 sc0 sc1\n\t" \
  "v_mfma_f32_16x16x128_f8f6f4 v[172:175], v[104:111], a[120:127], v[172:175]\n\t" \
  "ds_read_b64_tr_b8 v[126:127], v92 offset:58368\n\t" \
  "s_waitcnt lgkmcnt(4)\n\t" \
  "v_mfma_f32_16x16x128_f8f6f4 v[188:191], v[112:119], a[120:127], v[188:191]\n\t" \
  "ds_read_b64_tr_b8 v[104:105], v93 offset:24576\n\t" \
  "s_add_u32 s49, s68, 0xf1000\n\t" \
  "buffer_load_dwordx4 a[244:247], v4, s[12:15], s49 offen offset:1024 sc0 sc1\n\t" \
  "v_mfma_f32_16x16x128_f8f6f4 v[184:187], v[112:119], a[112:119], v[184:187]\n\t" \
  "ds_read_b64_tr_b8 v[106:107], v93 offset:25600\n\t" \
  "v_mfma_f32_16x16x128_f8f6f4 v[180:183], v[112:119], a[104:111], v[180:183]\n\t" \
  "ds_read_b64_tr_b8 v[108:109], v93 offset:57344\n\t" \
  "s_add_u32 s50, s68, 0xf1000\n\t" \
  "buffer_load_dwordx4 a[248:251], v4, s[12:15], s50 offen offset:2048 sc0 sc1\n\t" \
  "v_mfma_f32_16x16x128_f8f6f4 v[176:179], v[112:119], a[96:103], v[176:179]\n\t" \
  "ds_read_b64_tr_b8 v[110:111], v93 offset:58368\n\t" \
  "s_waitcnt lgkmcnt(4)\n\t" \
  "v_mfma_f32_16x16x128_f8f6f4 v[192:195], v[120:127], a[96:103], v[192:195]\n\t" \
  "ds_read_b64_tr_b8 v[112:113], v94 offset:24576\n\t" \
  "s_add_u32 s51, s68, 0xf1000\n\t" \
  "buffer_load_dwordx4 a[252:255], v4, s[12:15], s51 offen offset:3072 sc0 sc1\n\t" \
  "v_mfma_f32_16x16x128_f8f6f4 v[196:199], v[120:127], a[104:111], v[196:199]\n\t" \
  "ds_read_b64_tr_b8 v[114:115], v94 offset:25600\n\t" \
  "v_mfma_f32_16x16x128_f8f6f4 v[200:203], v[120:127], a[112:119], v[200:203]\n\t" \
  "ds_read_b64_tr_b8 v[116:117], v94 offset:57344\n\t" \
  "s_add_u32 m0, s24, 0xc000\n\t" \
  "s_add_u32 s52, s40, 0x400000\n\t" \
  "buffer_load_dwordx4 v2, s[8:11], s52 offen lds\n\t" \
  "v_mfma_f32_16x16x128_f8f6f4 v[204:207], v[120:127], a[120:127], v[204:207]\n\t" \
  "ds_read_b64_tr_b8 v[118:119], v94 offset:58368\n\t" \
  "s_waitcnt lgkmcnt(4)\n\t" \
  "v_mfma_f32_16x16x128_f8f6f4 v[220:223], v[104:111], a[120:127], v[220:223]\n\t" \
  "ds_read_b64_tr_b8 v[120:121], v95 offset:24576\n\t" \
  "s_add_u32 m0, s24, 0xc400\n\t" \
  "s_add_u32 s53, s40, 0x420000\n\t" \
  "buffer_load_dwordx4 v2, s[8:11], s53 offen lds\n\t" \
  "v_mfma_f32_16x16x128_f8f6f4 v[216:219], v[104:111], a[112:119], v[216:219]\n\t" \
  "ds_read_b64_tr_b8 v[122:123], v95 offset:25600\n\t" \
  "v_mfma_f32_16x16x128_f8f6f4 v[212:215], v[104:111], a[104:111], v[212:215]\n\t" \
  "ds_read_b64_tr_b8 v[124:125], v95 offset:57344\n\t" \
  "s_add_u32 m0, s24, 0xe000\n\t" \
  "s_add_u32 s54, s41, 0x400000\n\t" \
  "buffer_load_dwordx4 v2, s[8:11], s54 offen lds\n\t" \
  "v_mfma_f32_16x16x128_f8f6f4 v[208:211], v[104:111], a[96:103], v[208:211]\n\t" \
  "ds_read_b64_tr_b8 v[126:127], v95 offset:58368\n\t" \
  "s_waitcnt vmcnt(49) lgkmcnt(0)\n\t" \
  "s_barrier\n\t" \
  "v_mfma_f32_16x16x128_f8f6f4 v[224:227], v[112:119], a[96:103], v[224:227]\n\t" \
  "ds_read_b64_tr_b8 v[104:105], v80 offset:0\n\t" \
  "s_add_u32 m0, s24, 0xe400\n\t" \
  "s_add_u32 s55, s41, 0x420000\n\t" \
  "buffer_load_dwordx4 v2, s[8:11], s55 offen lds\n\t" \
  "v_mfma_f32_16x16x128_f8f6f4 v[228:231], v[112:119], a[104:111], v[228:231]\n\t" \
  "ds_read_b64_tr_b8 v[106:107], v80 offset:1024\n\t" \
  "v_mfma_f32_16x16x128_f8f6f4 v[232:235], v[112:119], a[112:119], v[232:235]\n\t" \
  "ds_read_b64_tr_b8 v[108:109], v96 offset:0\n\t" \
  "s_add_u32 m0, s24, 0x10000\n\t" \
  "s_add_u32 s48, s42, 0x400000\n\t" \
  "buffer_load_dwordx4 v2, s[8:11], s48 offen lds\n\t" \
  "v_mfma_f32_16x16x128_f8f6f4 v[236:239], v[112:119], a[120:127], v[236:239]\n\t" \
  "ds_read_b64_tr_b8 v[110:111], v96 offset:1024\n\t" \
  "v_mfma_f32_16x16x128_f8f6f4 v[252:255], v[120:127], a[120:127], v[252:255]\n\t" \
  "ds_read_b64_tr_b8 v[112:113], v81 offset:0\n\t" \
  "s_add_u32 m0, s24, 0x10400\n\t" \
  "s_add_u32 s49, s42, 0x420000\n\t" \
  "buffer_load_dwordx4 v2, s[8:11], s49 offen lds\n\t" \
  "v_mfma_f32_16x16x128_f8f6f4 v[248:251], v[120:127], a[112:119], v[248:251]\n\t" \
  "ds_read_b64_tr_b8 v[114:115], v81 offset:1024\n\t" \
  "v_mfma_f32_16x16x128_f8f6f4 v[244:247], v[120:127], a[104:111], v[244:247]\n\t" \
  "ds_read_b64_tr_b8 v[116:117], v97 offset:0\n\t" \
  "s_add_u32 m0, s24, 0x12000\n\t" \
  "s_add_u32 s50, s43, 0x400000\n\t" \
  "buffer_load_dwordx4 v2, s[8:11], s50 offen lds\n\t" \
  "v_mfma_f32_16x16x128_f8f6f4 v[240:243], v[120:127], a[96:103], v[240:243]\n\t" \
  "ds_read_b64_tr_b8 v[118:119], v97 offset:1024\n\t" \
  "s_waitcnt vmcnt(37) lgkmcnt(4)\n\t" \
  "v_mfma_f32_16x16x128_f8f6f4 v[128:131], v[104:111], a[128:135], v[128:131]\n\t" \
  "ds_read_b64_tr_b8 v[120:121], v82 offset:0\n\t" \
  "s_add_u32 m0, s24, 0x12400\n\t" \
  "s_add_u32 s51, s43, 0x420000\n\t" \
  "buffer_load_dwordx4 v2, s[8:11], s51 offen lds\n\t" \
  "s_mov_b32 exec_lo, 0x7f0000\n\t" \
  "s_mov_b32 exec_hi, 0\n\t" \
  "global_load_dwordx4 v[68:71], v10, s[6:7] sc0 sc1\n\t" \
  "s_mov_b64 exec, -1\n\t" \
  "v_mfma_f32_16x16x128_f8f6f4 v[132:135], v[104:111], a[136:143], v[132:135]\n\t" \
  "ds_read_b64_tr_b8 v[122:123], v82 offset:1024\n\t" \
  "v_mfma_f32_16x16x128_f8f6f4 v[136:139], v[104:111], a[144:151], v[136:139]\n\t" \
  "ds_read_b64_tr_b8 v[124:125], v98 offset:0\n\t" \
  "v_mfma_f32_16x16x128_f8f6f4 v[140:143], v[104:111], a[152:159], v[140:143]\n\t" \
  "ds_read_b64_tr_b8 v[126:127], v98 offset:1024\n\t" \
  "s_waitcnt lgkmcnt(4)\n\t" \
  "v_mfma_f32_16x16x128_f8f6f4 v[156:159], v[112:119], a[152:159], v[156:159]\n\t" \
  "ds_read_b64_tr_b8 v[104:105], v83 offset:0\n\t" \
  "v_mfma_f32_16x16x128_f8f6f4 v[152:155], v[112:119], a[144:151], v[152:155]\n\t" \
  "ds_read_b64_tr_b8 v[106:107], v83 offset:1024\n\t" \
  "v_mfma_f32_16x16x128_f8f6f4 v[148:151], v[112:119], a[136:143], v[148:151]\n\t" \
  "ds_read_b64_tr_b8 v[108:109], v99 offset:0\n\t" \
  "v_mfma_f32_16x16x128_f8f6f4 v[144:147], v[112:119], a[128:135], v[144:147]\n\t" \
  "ds_read_b64_tr_b8 v[110:111], v99 offset:1024\n\t" \
  "s_waitcnt lgkmcnt(4)\n\t" \
  "v_mfma_f32_16x16x128_f8f6f4 v[160:163], v[120:127], a[128:135], v[160:163]\n\t" \
  "ds_read_b64_tr_b8 v[112:113], v84 offset:0\n\t" \
  "v_mfma_f32_16x16x128_f8f6f4 v[164:167], v[120:127], a[136:143], v[164:167]\n\t" \
  "ds_read_b64_tr_b8 v[114:115], v84 offset:1024\n\t" \
  "v_mfma_f32_16x16x128_f8f6f4 v[168:171], v[120:127], a[144:151], v[168:171]\n\t" \
  "ds_read_b64_tr_b8 v[116:117], v100 offset:0\n\t" \
  "v_mfma_f32_16x16x128_f8f6f4 v[172:175], v[120:127], a[152:159], v[172:175]\n\t" \
  "ds_read_b64_tr_b8 v[118:119], v100 offset:1024\n\t" \
  "s_waitcnt lgkmcnt(4)\n\t" \
  "v_mfma_f32_16x16x128_f8f6f4 v[188:191], v[104:111], a[152:159], v[188:191]\n\t" \
  "ds_read_b64_tr_b8 v[120:121], v85 offset:0\n\t" \
  "v_mfma_f32_16x16x128_f8f6f4 v[184:187], v[104:111], a[144:151], v[184:187]\n\t" \
  "ds_read_b64_tr_b8 v[122:123], v85 offset:1024\n\t" \
  "v_mfma_f32_16x16x128_f8f6f4 v[180:183], v[104:111], a[136:143], v[180:183]\n\t" \
  "ds_read_b64_tr_b8 v[124:125], v101 offset:0\n\t" \
  "v_mfma_f32_16x16x128_f8f6f4 v[176:179], v[104:111], a[128:135], v[176:179]\n\t" \
  "ds_read_b64_tr_b8 v[126:127], v101 offset:1024\n\t" \
  "s_waitcnt lgkmcnt(4)\n\t" \
  "v_mfma_f32_16x16x128_f8f6f4 v[192:195], v[112:119], a[128:135], v[192:195]\n\t" \
  "ds_read_b64_tr_b8 v[104:105], v86 offset:0\n\t" \
  "s_add_u32 m0, s24, 0x18000\n\t" \
  "s_add_u32 s52, s45, 0x200000\n\t" \
  "buffer_load_dwordx4 v3, s[8:11], s52 offen lds\n\t" \
  "v_mfma_f32_16x16x128_f8f6f4 v[196:199], v[112:119], a[136:143], v[196:199]\n\t" \
  "ds_read_b64_tr_b8 v[106:107], v86 offset:1024\n\t" \
  "v_mfma_f32_16x16x128_f8f6f4 v[200:203], v[112:119], a[144:151], v[200:203]\n\t" \
  "ds_read_b64_tr_b8 v[108:109], v102 offset:0\n\t" \
  "s_add_u32 m0, s24, 0x18400\n\t" \
  "s_add_u32 s53, s45, 0x220000\n\t" \
  "buffer_load_dwordx4 v3, s[8:11], s53 offen lds\n\t" \
  "v_mfma_f32_16x16x128_f8f6f4 v[204:207], v[112:119], a[152:159], v[204:207]\n\t" \
  "ds_read_b64_tr_b8 v[110:111], v102 offset:1024\n\t" \
  "s_waitcnt lgkmcnt(4)\n\t" \
  "v_mfma_f32_16x16x128_f8f6f4 v[220:223], v[120:127], a[152:159], v[220:223]\n\t" \
  "ds_read_b64_tr_b8 v[112:113], v87 offset:0\n\t" \
  "s_add_u32 s54, s68, 0x100000\n\t" \
  "buffer_load_dwordx4 a[0:3], v4, s[12:15], s54 offen offset:0 sc0 sc1\n\t" \
  "v_mfma_f32_16x16x128_f8f6f4 v[216:219], v[120:127], a[144:151], v[216:219]\n\t" \
  "ds_read_b64_tr_b8 v[114:115], v87 offset:1024\n\t" \
  "v_mfma_f32_16x16x128_f8f6f4 v[212:215], v[120:127], a[136:143], v[212:215]\n\t" \
  "ds_read_b64_tr_b8 v[116:117], v103 offset:0\n\t" \
  "s_add_u32 s55, s68, 0x100000\n\t" \
  "buffer_load_dwordx4 a[4:7], v4, s[12:15], s55 offen offset:1024 sc0 sc1\n\t" \
  "v_mfma_f32_16x16x128_f8f6f4 v[208:211], v[120:127], a[128:135], v[208:211]\n\t" \
  "ds_read_b64_tr_b8 v[118:119], v103 offset:1024\n\t" \
  "s_waitcnt vmcnt(41) lgkmcnt(0)\n\t" \
  "s_barrier\n\t" \
  "v_mfma_f32_16x16x128_f8f6f4 v[224:227], v[104:111], a[128:135], v[224:227]\n\t" \
  "ds_read_b64_tr_b8 v[120:121], v80 offset:8192\n\t" \
  "s_add_u32 s48, s68, 0x100000\n\t" \
  "buffer_load_dwordx4 a[8:11], v4, s[12:15], s48 offen offset:2048 sc0 sc1\n\t" \
  "v_mfma_f32_16x16x128_f8f6f4 v[228:231], v[104:111], a[136:143], v[228:231]\n\t" \
  "ds_read_b64_tr_b8 v[122:123], v80 offset:9216\n\t" \
  "v_mfma_f32_16x16x128_f8f6f4 v[232:235], v[104:111], a[144:151], v[232:235]\n\t" \
  "ds_read_b64_tr_b8 v[124:125], v96 offset:8192\n\t" \
  "s_add_u32 s49, s68, 0x100000\n\t" \
  "buffer_load_dwordx4 a[12:15], v4, s[12:15], s49 offen offset:3072 sc0 sc1\n\t" \
  "v_mfma_f32_16x16x128_f8f6f4 v[236:239], v[104:111], a[152:159], v[236:239]\n\t" \
  "ds_read_b64_tr_b8 v[126:127], v96 offset:9216\n\t" \
  "v_mfma_f32_16x16x128_f8f6f4 v[252:255], v[112:119], a[152:159], v[252:255]\n\t" \
  "ds_read_b64_tr_b8 v[104:105], v81 offset:8192\n\t" \
  "s_add_u32 s50, s68, 0x101000\n\t" \
  "buffer_load_dwordx4 a[16:19], v4, s[12:15], s50 offen offset:0 sc0 sc1\n\t" \
  "v_mfma_f32_16x16x128_f8f6f4 v[248:251], v[112:119], a[144:151], v[248:251]\n\t" \
  "ds_read_b64_tr_b8 v[106:107], v81 offset:9216\n\t" \
  "v_mfma_f32_16x16x128_f8f6f4 v[244:247], v[112:119], a[136:143], v[244:247]\n\t" \
  "ds_read_b64_tr_b8 v[108:109], v97 offset:8192\n\t" \
  "s_add_u32 s51, s68, 0x101000\n\t" \
  "buffer_load_dwordx4 a[20:23], v4, s[12:15], s51 offen offset:1024 sc0 sc1\n\t" \
  "v_mfma_f32_16x16x128_f8f6f4 v[240:243], v[112:119], a[128:135], v[240:243]\n\t" \
  "ds_read_b64_tr_b8 v[110:111], v97 offset:9216\n\t" \
  "s_waitcnt vmcnt(37) lgkmcnt(4)\n\t" \
  "v_mfma_f32_16x16x128_f8f6f4 v[128:131], v[120:127], a[160:167], v[128:131]\n\t" \
  "ds_read_b64_tr_b8 v[112:113], v82 offset:8192\n\t" \
  "s_add_u32 s52, s68, 0x101000\n\t" \
  "buffer_load_dwordx4 a[24:27], v4, s[12:15], s52 offen offset:2048 sc0 sc1\n\t" \
  "v_mfma_f32_16x16x128_f8f6f4 v[132:135], v[120:127], a[168:175], v[132:135]\n\t" \
  "ds_read_b64_tr_b8 v[114:115], v82 offset:9216\n\t" \
  "v_mfma_f32_16x16x128_f8f6f4 v[136:139], v[120:127], a[176:183], v[136:139]\n\t" \
  "ds_read_b64_tr_b8 v[116:117], v98 offset:8192\n\t" \
  "s_add_u32 s53, s68, 0x101000\n\t" \
  "buffer_load_dwordx4 a[28:31], v4, s[12:15], s53 offen offset:3072 sc0 sc1\n\t" \
  "v_mfma_f32_16x16x128_f8f6f4 v[140:143], v[120:127], a[184:191], v[140:143]\n\t" \
  "ds_read_b64_tr_b8 v[118:119], v98 offset:9216\n\t" \
  "s_waitcnt lgkmcnt(4)\n\t" \
  "v_mfma_f32_16x16x128_f8f6f4 v[156:159], v[104:111], a[184:191], v[156:159]\n\t" \
  "ds_read_b64_tr_b8 v[120:121], v83 offset:8192\n\t" \
  "s_add_u32 m0, s24, 0x1a000\n\t" \
  "s_add_u32 s54, s46, 0x200000\n\t" \
  "buffer_load_dwordx4 v3, s[8:11], s54 offen lds\n\t" \
  "v_mfma_f32_16x16x128_f8f6f4 v[152:155], v[104:111], a[176:183], v[152:155]\n\t" \
  "ds_read_b64_tr_b8 v[122:123], v83 offset:9216\n\t" \
  "v_mfma_f32_16x16x128_f8f6f4 v[148:151], v[104:111], a[168:175], v[148:151]\n\t" \
  "ds_read_b64_tr_b8 v[124:125], v99 offset:8192\n\t" \
  "s_add_u32 m0, s24, 0x1a400\n\t" \
  "s_add_u32 s55, s46, 0x220000\n\t" \
  "buffer_load_dwordx4 v3, s[8:11], s55 offen lds\n\t" \
  "v_mfma_f32_16x16x128_f8f6f4 v[144:147], v[104:111], a[160:167], v[144:147]\n\t" \
  "ds_read_b64_tr_b8 v[126:127], v99 offset:9216\n\t" \
  "s_waitcnt lgkmcnt(4)\n\t" \
  "v_mfma_f32_16x16x128_f8f6f4 v[160:163], v[112:119], a[160:167], v[160:163]\n\t" \
  "ds_read_b64_tr_b8 v[104:105], v84 offset:8192\n\t" \
  "s_add_u32 s48, s68, 0x110000\n\t" \
  "buffer_load_dwordx4 a[32:35], v4, s[12:15], s48 offen offset:0 sc0 sc1\n\t" \
  "v_mfma_f32_16x16x128_f8f6f4 v[164:167], v[112:119], a[168:175], v[164:167]\n\t" \
  "ds_read_b64_tr_b8 v[106:107], v84 offset:9216\n\t" \
  "v_mfma_f32_16x16x128_f8f6f4 v[168:171], v[112:119], a[176:183], v[168:171]\n\t" \
  "ds_read_b64_tr_b8 v[108:109], v100 offset:8192\n\t" \
  "s_add_u32 s49, s68, 0x110000\n\t" \
  "buffer_load_dwordx4 a[36:39], v4, s[12:15], s49 offen offset:1024 sc0 sc1\n\t" \
  "v_mfma_f32_16x16x128_f8f6f4 v[172:175], v[112:119], a[184:191], v[172:175]\n\t" \
  "ds_read_b64_tr_b8 v[110:111], v100 offset:9216\n\t" \
  "s_waitcnt lgkmcnt(4)\n\t" \
  "v_mfma_f32_16x16x128_f8f6f4 v[188:191], v[120:127], a[184:191], v[188:191]\n\t" \
  "ds_read_b64_tr_b8 v[112:113], v85 offset:8192\n\t" \
  "s_add_u32 s50, s68, 0x110000\n\t" \
  "buffer_load_dwordx4 a[40:43], v4, s[12:15], s50 offen offset:2048 sc0 sc1\n\t" \
  "v_mfma_f32_16x16x128_f8f6f4 v[184:187], v[120:127], a[176:183], v[184:187]\n\t" \
  "ds_read_b64_tr_b8 v[114:115], v85 offset:9216\n\t" \
  "v_mfma_f32_16x16x128_f8f6f4 v[180:183], v[120:127], a[168:175], v[180:183]\n\t" \
  "ds_read_b64_tr_b8 v[116:117], v101 offset:8192\n\t" \
  "s_add_u32 s51, s68, 0x110000\n\t" \
  "buffer_load_dwordx4 a[44:47], v4, s[12:15], s51 offen offset:3072 sc0 sc1\n\t" \
  "v_mfma_f32_16x16x128_f8f6f4 v[176:179], v[120:127], a[160:167], v[176:179]\n\t" \
  "ds_read_b64_tr_b8 v[118:119], v101 offset:9216\n\t" \
  "s_waitcnt lgkmcnt(4)\n\t" \
  "v_mfma_f32_16x16x128_f8f6f4 v[192:195], v[104:111], a[160:167], v[192:195]\n\t" \
  "ds_read_b64_tr_b8 v[120:121], v86 offset:8192\n\t" \
  "s_add_u32 s52, s68, 0x111000\n\t" \
  "buffer_load_dwordx4 a[48:51], v4, s[12:15], s52 offen offset:0 sc0 sc1\n\t" \
  "v_mfma_f32_16x16x128_f8f6f4 v[196:199], v[104:111], a[168:175], v[196:199]\n\t" \
  "ds_read_b64_tr_b8 v[122:123], v86 offset:9216\n\t" \
  "v_mfma_f32_16x16x128_f8f6f4 v[200:203], v[104:111], a[176:183], v[200:203]\n\t" \
  "ds_read_b64_tr_b8 v[124:125], v102 offset:8192\n\t" \
  "s_add_u32 s53, s68, 0x111000\n\t" \
  "buffer_load_dwordx4 a[52:55], v4, s[12:15], s53 offen offset:1024 sc0 sc1\n\t" \
  "v_mfma_f32_16x16x128_f8f6f4 v[204:207], v[104:111], a[184:191], v[204:207]\n\t" \
  "ds_read_b64_tr_b8 v[126:127], v102 offset:9216\n\t" \
  "s_waitcnt lgkmcnt(4)\n\t" \
  "v_mfma_f32_16x16x128_f8f6f4 v[220:223], v[112:119], a[184:191], v[220:223]\n\t" \
  "ds_read_b64_tr_b8 v[104:105], v87 offset:8192\n\t" \
  "s_add_u32 s54, s68, 0x111000\n\t" \
  "buffer_load_dwordx4 a[56:59], v4, s[12:15], s54 offen offset:2048 sc0 sc1\n\t" \
  "v_mfma_f32_16x16x128_f8f6f4 v[216:219], v[112:119], a[176:183], v[216:219]\n\t" \
  "ds_read_b64_tr_b8 v[106:107], v87 offset:9216\n\t" \
  "v_mfma_f32_16x16x128_f8f6f4 v[212:215], v[112:119], a[168:175], v[212:215]\n\t" \
  "ds_read_b64_tr_b8 v[108:109], v103 offset:8192\n\t" \
  "s_add_u32 s55, s68, 0x111000\n\t" \
  "buffer_load_dwordx4 a[60:63], v4, s[12:15], s55 offen offset:3072 sc0 sc1\n\t" \
  "v_mfma_f32_16x16x128_f8f6f4 v[208:211], v[112:119], a[160:167], v[208:211]\n\t" \
  "ds_read_b64_tr_b8 v[110:111], v103 offset:9216\n\t" \
  "s_waitcnt vmcnt(47) lgkmcnt(0)\n\t" \
  "s_barrier\n\t" \
  "v_mfma_f32_16x16x128_f8f6f4 v[224:227], v[120:127], a[160:167], v[224:227]\n\t" \
  "ds_read_b64_tr_b8 v[112:113], v80 offset:16384\n\t" \
  "s_add_u32 m0, s24, 0x1c000\n\t" \
  "s_add_u32 s48, s40, 0x400000\n\t" \
  "buffer_load_dwordx4 v3, s[8:11], s48 offen lds\n\t" \
  "v_mfma_f32_16x16x128_f8f6f4 v[228:231], v[120:127], a[168:175], v[228:231]\n\t" \
  "ds_read_b64_tr_b8 v[114:115], v80 offset:17408\n\t" \
  "v_mfma_f32_16x16x128_f8f6f4 v[232:235], v[120:127], a[176:183], v[232:235]\n\t" \
  "ds_read_b64_tr_b8 v[116:117], v96 offset:16384\n\t" \
  "s_add_u32 m0, s24, 0x1c400\n\t" \
  "s_add_u32 s49, s40, 0x420000\n\t" \
  "buffer_load_dwordx4 v3, s[8:11], s49 offen lds\n\t" \
  "v_mfma_f32_16x16x128_f8f6f4 v[236:239], v[120:127], a[184:191], v[236:239]\n\t" \
  "ds_read_b64_tr_b8 v[118:119], v96 offset:17408\n\t" \
  "v_mfma_f32_16x16x128_f8f6f4 v[252:255], v[104:111], a[184:191], v[252:255]\n\t" \
  "ds_read_b64_tr_b8 v[120:121], v81 offset:16384\n\t" \
  "s_waitcnt vmcnt(22)\n\t" \
  "v_min_u32 v68, v68, v69\n\t" \
  "v_min3_u32 v68, v68, v70, v71\n\t" \
  "v_cmp_le_u32 vcc, s57, v68\n\t" \
  "s_and_b32 s64, vcc_lo, 0x7f0000\n\t" \
  "s_or_b32 s56, s56, s64\n\t" \
  "s_and_b32 s64, s56, 0x7f0000\n\t" \
  "s_cmp_eq_u32 s64, 0x7f0000\n\t" \
  "s_cbranch_scc1 Lok8_0_%=\n\t" \
  "s_mov_b32 s62, 0x7f0000\n\t" \
  "s_mov_b32 s69, 0x7f0000\n\t" \
  "s_call_b64 s[60:61], Lpoll_0_%=\n\t" \
  "Lok8_0_%=:\n\t" \
  "s_add_u32 s50, s68, 0x120000\n\t" \
  "buffer_load_dwordx4 a[64:67], v4, s[12:15], s50 offen offset:0 sc0 sc1\n\t" \
  "v_mfma_f32_16x16x128_f8f6f4 v[248:251], v[104:111], a[176:183], v[248:251]\n\t" \
  "ds_read_b64_tr_b8 v[122:123], v81 offset:17408\n\t" \
  "v_mfma_f32_16x16x128_f8f6f4 v[244:247], v[104:111], a[168:175], v[244:247]\n\t" \
  "ds_read_b64_tr_b8 v[124:125], v97 offset:16384\n\t" \
  "s_add_u32 s51, s68, 0x120000\n\t" \
  "buffer_load_dwordx4 a[68:71], v4, s[12:15], s51 offen offset:1024 sc0 sc1\n\t" \
  "v_mfma_f32_16x16x128_f8f6f4 v[240:243], v[104:111], a[160:167], v[240:243]\n\t" \
  "ds_read_b64_tr_b8 v[126:127], v97 offset:17408\n\t" \
  "s_waitcnt lgkmcnt(4)\n\t" \
  "v_mfma_f32_16x16x128_f8f6f4 v[128:131], v[112:119], a[192:199], v[128:131]\n\t" \
  "ds_read_b64_tr_b8 v[104:105], v82 offset:16384\n\t" \
  "s_add_u32 s52, s68, 0x120000\n\t" \
  "buffer_load_dwordx4 a[72:75], v4, s[12:15], s52 offen offset:2048 sc0 sc1\n\t" \
  "v_mfma_f32_16x16x128_f8f6f4 v[132:135], v[112:119], a[200:207], v[132:135]\n\t" \
  "ds_read_b64_tr_b8 v[106:107], v82 offset:17408\n\t" \
  "v_mfma_f32_16x16x128_f8f6f4 v[136:139], v[112:119], a[208:215], v[136:139]\n\t" \
  "ds_read_b64_tr_b8 v[108:109], v98 offset:16384\n\t" \
  "s_add_u32 s53, s68, 0x120000\n\t" \
  "buffer_load_dwordx4 a[76:79], v4, s[12:15], s53 offen offset:3072 sc0 sc1\n\t" \
  "v_mfma_f32_16x16x128_f8f6f4 v[140:143], v[112:119], a[216:223], v[140:143]\n\t" \
  "ds_read_b64_tr_b8 v[110:111], v98 offset:17408\n\t" \
  "s_waitcnt lgkmcnt(4)\n\t" \
  "v_mfma_f32_16x16x128_f8f6f4 v[156:159], v[120:127], a[216:223], v[156:159]\n\t" \
  "ds_read_b64_tr_b8 v[112:113], v83 offset:16384\n\t" \
  "s_add_u32 s54, s68, 0x121000\n\t" \
  "buffer_load_dwordx4 a[80:83], v4, s[12:15], s54 offen offset:0 sc0 sc1\n\t" \
  "v_mfma_f32_16x16x128_f8f6f4 v[152:155], v[120:127], a[208:215], v[152:155]\n\t" \
  "ds_read_b64_tr_b8 v[114:115], v83 offset:17408\n\t" \
  "v_mfma_f32_16x16x128_f8f6f4 v[148:151], v[120:127], a[200:207], v[148:151]\n\t" \
  "ds_read_b64_tr_b8 v[116:117], v99 offset:16384\n\t" \
  "s_add_u32 s55, s68, 0x121000\n\t" \
  "buffer_load_dwordx4 a[84:87], v4, s[12:15], s55 offen offset:1024 sc0 sc1\n\t" \
  "v_mfma_f32_16x16x128_f8f6f4 v[144:147], v[120:127], a[192:199], v[144:147]\n\t" \
  "ds_read_b64_tr_b8 v[118:119], v99 offset:17408\n\t" \
  "s_waitcnt lgkmcnt(4)\n\t" \
  "v_mfma_f32_16x16x128_f8f6f4 v[160:163], v[104:111], a[192:199], v[160:163]\n\t" \
  "ds_read_b64_tr_b8 v[120:121], v84 offset:16384\n\t" \
  "s_add_u32 s48, s68, 0x121000\n\t" \
  "buffer_load_dwordx4 a[88:91], v4, s[12:15], s48 offen offset:2048 sc0 sc1\n\t" \
  "v_mfma_f32_16x16x128_f8f6f4 v[164:167], v[104:111], a[200:207], v[164:167]\n\t" \
  "ds_read_b64_tr_b8 v[122:123], v84 offset:17408\n\t" \
  "v_mfma_f32_16x16x128_f8f6f4 v[168:171], v[104:111], a[208:215], v[168:171]\n\t" \
  "ds_read_b64_tr_b8 v[124:125], v100 offset:16384\n\t" \
  "s_add_u32 s49, s68, 0x121000\n\t" \
  "buffer_load_dwordx4 a[92:95], v4, s[12:15], s49 offen offset:3072 sc0 sc1\n\t" \
  "v_mfma_f32_16x16x128_f8f6f4 v[172:175], v[104:111], a[216:223], v[172:175]\n\t" \
  "ds_read_b64_tr_b8 v[126:127], v100 offset:17408\n\t" \
  "s_waitcnt lgkmcnt(4)\n\t" \
  "v_mfma_f32_16x16x128_f8f6f4 v[188:191], v[112:119], a[216:223], v[188:191]\n\t" \
  "ds_read_b64_tr_b8 v[104:105], v85 offset:16384\n\t" \
  "s_add_u32 m0, s24, 0x1e000\n\t" \
  "s_add_u32 s50, s41, 0x400000\n\t" \
  "buffer_load_dwordx4 v3, s[8:11], s50 offen lds\n\t" \
  "v_mfma_f32_16x16x128_f8f6f4 v[184:187], v[112:119], a[208:215], v[184:187]\n\t" \
  "ds_read_b64_tr_b8 v[106:107], v85 offset:17408\n\t" \
  "v_mfma_f32_16x16x128_f8f6f4 v[180:183], v[112:119], a[200:207], v[180:183]\n\t" \
  "ds_read_b64_tr_b8 v[108:109], v101 offset:16384\n\t" \
  "s_add_u32 m0, s24, 0x1e400\n\t" \
  "s_add_u32 s51, s41, 0x420000\n\t" \
  "buffer_load_dwordx4 v3, s[8:11], s51 offen lds\n\t" \
  "v_mfma_f32_16x16x128_f8f6f4 v[176:179], v[112:119], a[192:199], v[176:179]\n\t" \
  "ds_read_b64_tr_b8 v[110:111], v101 offset:17408\n\t" \
  "s_waitcnt lgkmcnt(4)\n\t" \
  "v_mfma_f32_16x16x128_f8f6f4 v[192:195], v[120:127], a[192:199], v[192:195]\n\t" \
  "ds_read_b64_tr_b8 v[112:113], v86 offset:16384\n\t" \
  "s_add_u32 s52, s68, 0x130000\n\t" \
  "buffer_load_dwordx4 a[96:99], v4, s[12:15], s52 offen offset:0 sc0 sc1\n\t" \
  "v_mfma_f32_16x16x128_f8f6f4 v[196:199], v[120:127], a[200:207], v[196:199]\n\t" \
  "ds_read_b64_tr_b8 v[114:115], v86 offset:17408\n\t" \
  "v_mfma_f32_16x16x128_f8f6f4 v[200:203], v[120:127], a[208:215], v[200:203]\n\t" \
  "ds_read_b64_tr_b8 v[116:117], v102 offset:16384\n\t" \
  "s_add_u32 s53, s68, 0x130000\n\t" \
  "buffer_load_dwordx4 a[100:103], v4, s[12:15], s53 offen offset:1024 sc0 sc1\n\t" \
  "v_mfma_f32_16x16x128_f8f6f4 v[204:207], v[120:127], a[216:223], v[204:207]\n\t" \
  "ds_read_b64_tr_b8 v[118:119], v102 offset:17408\n\t" \
  "s_waitcnt lgkmcnt(4)\n\t" \
  "v_mfma_f32_16x16x128_f8f6f4 v[220:223], v[104:111], a[216:223], v[220:223]\n\t" \
  "ds_read_b64_tr_b8 v[120:121], v87 offset:16384\n\t" \
  "s_add_u32 s54, s68, 0x130000\n\t" \
  "buffer_load_dwordx4 a[104:107], v4, s[12:15], s54 offen offset:2048 sc0 sc1\n\t" \
  "v_mfma_f32_16x16x128_f8f6f4 v[216:219], v[104:111], a[208:215], v[216:219]\n\t" \
  "ds_read_b64_tr_b8 v[122:123], v87 offset:17408\n\t" \
  "v_mfma_f32_16x16x128_f8f6f4 v[212:215], v[104:111], a[200:207], v[212:215]\n\t" \
  "ds_read_b64_tr_b8 v[124:125], v103 offset:16384\n\t" \
  "s_add_u32 s55, s68, 0x130000\n\t" \
  "buffer_load_dwordx4 a[108:111], v4, s[12:15], s55 offen offset:3072 sc0 sc1\n\t" \
  "v_mfma_f32_16x16x128_f8f6f4 v[208:211], v[104:111], a[192:199], v[208:211]\n\t" \
  "ds_read_b64_tr_b8 v[126:127], v103 offset:17408\n\t" \
  "s_waitcnt lgkmcnt(0)\n\t" \
  "s_barrier\n\t" \
  "v_mfma_f32_16x16x128_f8f6f4 v[224:227], v[112:119], a[192:199], v[224:227]\n\t" \
  "ds_read_b64_tr_b8 v[104:105], v80 offset:24576\n\t" \
  "s_add_u32 s48, s68, 0x131000\n\t" \
  "buffer_load_dwordx4 a[112:115], v4, s[12:15], s48 offen offset:0 sc0 sc1\n\t" \
  "v_mfma_f32_16x16x128_f8f6f4 v[228:231], v[112:119], a[200:207], v[228:231]\n\t" \
  "ds_read_b64_tr_b8 v[106:107], v80 offset:25600\n\t" \
  "v_mfma_f32_16x16x128_f8f6f4 v[232:235], v[112:119], a[208:215], v[232:235]\n\t" \
  "ds_read_b64_tr_b8 v[108:109], v96 offset:24576\n\t" \
  "s_add_u32 s49, s68, 0x131000\n\t" \
  "buffer_load_dwordx4 a[116:119], v4, s[12:15], s49 offen offset:1024 sc0 sc1\n\t" \
  "v_mfma_f32_16x16x128_f8f6f4 v[236:239], v[112:119], a[216:223], v[236:239]\n\t" \
  "ds_read_b64_tr_b8 v[110:111], v96 offset:25600\n\t" \
  "v_mfma_f32_16x16x128_f8f6f4 v[252:255], v[120:127], a[216:223], v[252:255]\n\t" \
  "ds_read_b64_tr_b8 v[112:113], v81 offset:24576\n\t" \
  "s_add_u32 s50, s68, 0x131000\n\t" \
  "buffer_load_dwordx4 a[120:123], v4, s[12:15], s50 offen offset:2048 sc0 sc1\n\t" \
  "v_mfma_f32_16x16x128_f8f6f4 v[248:251], v[120:127], a[208:215], v[248:251]\n\t" \
  "ds_read_b64_tr_b8 v[114:115], v81 offset:25600\n\t" \
  "v_mfma_f32_16x16x128_f8f6f4 v[244:247], v[120:127], a[200:207], v[244:247]\n\t" \
  "ds_read_b64_tr_b8 v[116:117], v97 offset:24576\n\t" \
  "s_add_u32 s51, s68, 0x131000\n\t" \
  "buffer_load_dwordx4 a[124:127], v4, s[12:15], s51 offen offset:3072 sc0 sc1\n\t" \
  "v_mfma_f32_16x16x128_f8f6f4 v[240:243], v[120:127], a[192:199], v[240:243]\n\t" \
  "ds_read_b64_tr_b8 v[118:119], v97 offset:25600\n\t" \
  "s_waitcnt lgkmcnt(4)\n\t" \
  "v_mfma_f32_16x16x128_f8f6f4 v[128:131], v[104:111], a[224:231], v[128:131]\n\t" \
  "ds_read_b64_tr_b8 v[120:121], v82 offset:24576\n\t" \
  "s_add_u32 m0, s24, 0x14000\n\t" \
  "s_add_u32 s52, s44, 0x400000\n\t" \
  "buffer_load_dwordx4 v2, s[8:11], s52 offen lds\n\t" \
  "v_mfma_f32_16x16x128_f8f6f4 v[132:135], v[104:111], a[232:239], v[132:135]\n\t" \
  "ds_read_b64_tr_b8 v[122:123], v82 offset:25600\n\t" \
  "v_mfma_f32_16x16x128_f8f6f4 v[136:139], v[104:111], a[240:247], v[136:139]\n\t" \
  "ds_read_b64_tr_b8 v[124:125], v98 offset:24576\n\t" \
  "s_add_u32 m0, s24, 0x14400\n\t" \
  "s_add_u32 s53, s44, 0x420000\n\t" \
  "buffer_load_dwordx4 v2, s[8:11], s53 offen lds\n\t" \
  "v_mfma_f32_16x16x128_f8f6f4 v[140:143], v[104:111], a[248:255], v[140:143]\n\t" \
  "ds_read_b64_tr_b8 v[126:127], v98 offset:25600\n\t" \
  "s_waitcnt lgkmcnt(4)\n\t" \
  "v_mfma_f32_16x16x128_f8f6f4 v[156:159], v[112:119], a[248:255], v[156:159]\n\t" \
  "ds_read_b64_tr_b8 v[104:105], v83 offset:24576\n\t" \
  "s_add_u32 m0, s24, 0x16000\n\t" \
  "s_add_u32 s54, s45, 0x400000\n\t" \
  "buffer_load_dwordx4 v2, s[8:11], s54 offen lds\n\t" \
  "v_mfma_f32_16x16x128_f8f6f4 v[152:155], v[112:119], a[240:247], v[152:155]\n\t" \
  "ds_read_b64_tr_b8 v[106:107], v83 offset:25600\n\t" \
  "v_mfma_f32_16x16x128_f8f6f4 v[148:151], v[112:119], a[232:239], v[148:151]\n\t" \
  "ds_read_b64_tr_b8 v[108:109], v99 offset:24576\n\t" \
  "s_add_u32 m0, s24, 0x16400\n\t" \
  "s_add_u32 s55, s45, 0x420000\n\t" \
  "buffer_load_dwordx4 v2, s[8:11], s55 offen lds\n\t" \
  "v_mfma_f32_16x16x128_f8f6f4 v[144:147], v[112:119], a[224:231], v[144:147]\n\t" \
  "ds_read_b64_tr_b8 v[110:111], v99 offset:25600\n\t" \
  "s_waitcnt lgkmcnt(4)\n\t" \
  "v_mfma_f32_16x16x128_f8f6f4 v[160:163], v[120:127], a[224:231], v[160:163]\n\t" \
  "ds_read_b64_tr_b8 v[112:113], v84 offset:24576\n\t" \
  "s_add_u32 m0, s24, 0x0\n\t" \
  "s_add_u32 s48, s46, 0x400000\n\t" \
  "buffer_load_dwordx4 v2, s[8:11], s48 offen lds\n\t" \
  "v_mfma_f32_16x16x128_f8f6f4 v[164:167], v[120:127], a[232:239], v[164:167]\n\t" \
  "ds_read_b64_tr_b8 v[114:115], v84 offset:25600\n\t" \
  "v_mfma_f32_16x16x128_f8f6f4 v[168:171], v[120:127], a[240:247], v[168:171]\n\t" \
  "ds_read_b64_tr_b8 v[116:117], v100 offset:24576\n\t" \
  "s_add_u32 m0, s24, 0x400\n\t" \
  "s_add_u32 s49, s46, 0x420000\n\t" \
  "buffer_load_dwordx4 v2, s[8:11], s49 offen lds\n\t" \
  "v_mfma_f32_16x16x128_f8f6f4 v[172:175], v[120:127], a[248:255], v[172:175]\n\t" \
  "ds_read_b64_tr_b8 v[118:119], v100 offset:25600\n\t" \
  "s_waitcnt lgkmcnt(4)\n\t" \
  "v_mfma_f32_16x16x128_f8f6f4 v[188:191], v[104:111], a[248:255], v[188:191]\n\t" \
  "ds_read_b64_tr_b8 v[120:121], v85 offset:24576\n\t" \
  "s_add_u32 m0, s24, 0x2000\n\t" \
  "s_add_u32 s50, s40, 0x600000\n\t" \
  "buffer_load_dwordx4 v2, s[8:11], s50 offen lds\n\t" \
  "v_mfma_f32_16x16x128_f8f6f4 v[184:187], v[104:111], a[240:247], v[184:187]\n\t" \
  "ds_read_b64_tr_b8 v[122:123], v85 offset:25600\n\t" \
  "v_mfma_f32_16x16x128_f8f6f4 v[180:183], v[104:111], a[232:239], v[180:183]\n\t" \
  "ds_read_b64_tr_b8 v[124:125], v101 offset:24576\n\t" \
  "s_add_u32 m0, s24, 0x2400\n\t" \
  "s_add_u32 s51, s40, 0x620000\n\t" \
  "buffer_load_dwordx4 v2, s[8:11], s51 offen lds\n\t" \
  "v_mfma_f32_16x16x128_f8f6f4 v[176:179], v[104:111], a[224:231], v[176:179]\n\t" \
  "ds_read_b64_tr_b8 v[126:127], v101 offset:25600\n\t" \
  "s_waitcnt lgkmcnt(4)\n\t" \
  "v_mfma_f32_16x16x128_f8f6f4 v[192:195], v[112:119], a[224:231], v[192:195]\n\t" \
  "ds_read_b64_tr_b8 v[104:105], v86 offset:24576\n\t" \
  "v_mfma_f32_16x16x128_f8f6f4 v[196:199], v[112:119], a[232:239], v[196:199]\n\t" \
  "ds_read_b64_tr_b8 v[106:107], v86 offset:25600\n\t" \
  "v_mfma_f32_16x16x128_f8f6f4 v[200:203], v[112:119], a[240:247], v[200:203]\n\t" \
  "ds_read_b64_tr_b8 v[108:109], v102 offset:24576\n\t" \
  "v_mfma_f32_16x16x128_f8f6f4 v[204:207], v[112:119], a[248:255], v[204:207]\n\t" \
  "ds_read_b64_tr_b8 v[110:111], v102 offset:25600\n\t" \
  "s_waitcnt lgkmcnt(4)\n\t" \
  "v_mfma_f32_16x16x128_f8f6f4 v[220:223], v[120:127], a[248:255], v[220:223]\n\t" \
  "ds_read_b64_tr_b8 v[112:113], v87 offset:24576\n\t" \
  "v_mfma_f32_16x16x128_f8f6f4 v[216:219], v[120:127], a[240:247], v[216:219]\n\t" \
  "ds_read_b64_tr_b8 v[114:115], v87 offset:25600\n\t" \
  "v_mfma_f32_16x16x128_f8f6f4 v[212:215], v[120:127], a[232:239], v[212:215]\n\t" \
  "ds_read_b64_tr_b8 v[116:117], v103 offset:24576\n\t" \
  "v_mfma_f32_16x16x128_f8f6f4 v[208:211], v[120:127], a[224:231], v[208:211]\n\t" \
  "ds_read_b64_tr_b8 v[118:119], v103 offset:25600\n\t" \
  "s_waitcnt vmcnt(46) lgkmcnt(0)\n\t" \
  "s_barrier\n\t" \
  "v_mfma_f32_16x16x128_f8f6f4 v[224:227], v[104:111], a[224:231], v[224:227]\n\t" \
  "ds_read_b64_tr_b8 v[120:121], v80 offset:32768\n\t" \
  "v_mfma_f32_16x16x128_f8f6f4 v[228:231], v[104:111], a[232:239], v[228:231]\n\t" \
  "ds_read_b64_tr_b8 v[122:123], v80 offset:33792\n\t" \
  "v_mfma_f32_16x16x128_f8f6f4 v[232:235], v[104:111], a[240:247], v[232:235]\n\t" \
  "ds_read_b64_tr_b8 v[124:125], v88 offset:32768\n\t" \
  "v_mfma_f32_16x16x128_f8f6f4 v[236:239], v[104:111], a[248:255], v[236:239]\n\t" \
  "ds_read_b64_tr_b8 v[126:127], v88 offset:33792\n\t" \
  "v_mfma_f32_16x16x128_f8f6f4 v[252:255], v[112:119], a[248:255], v[252:255]\n\t" \
  "ds_read_b64_tr_b8 v[104:105], v81 offset:32768\n\t" \
  "v_mfma_f32_16x16x128_f8f6f4 v[248:251], v[112:119], a[240:247], v[248:251]\n\t" \
  "ds_read_b64_tr_b8 v[106:107], v81 offset:33792\n\t" \
  "v_mfma_f32_16x16x128_f8f6f4 v[244:247], v[112:119], a[232:239], v[244:247]\n\t" \
  "ds_read_b64_tr_b8 v[108:109], v89 offset:32768\n\t" \
  "s_add_u32 m0, s24, 0x20000\n\t" \
  "s_add_u32 s52, s42, 0x400000\n\t" \
  "buffer_load_dwordx4 v3, s[8:11], s52 offen lds\n\t" \
  "v_mfma_f32_16x16x128_f8f6f4 v[240:243], v[112:119], a[224:231], v[240:243]\n\t" \
  "ds_read_b64_tr_b8 v[110:111], v89 offset:33792\n\t" \
  "s_waitcnt vmcnt(39) lgkmcnt(4)\n\t" \
  "v_mfma_f32_16x16x128_f8f6f4 v[128:131], v[120:127], a[0:7], v[128:131]\n\t" \
  "ds_read_b64_tr_b8 v[112:113], v82 offset:32768\n\t" \
  "s_add_u32 m0, s24, 0x20400\n\t" \
  "s_add_u32 s53, s42, 0x420000\n\t" \
  "buffer_load_dwordx4 v3, s[8:11], s53 offen lds\n\t" \
  "v_mfma_f32_16x16x128_f8f6f4 v[132:135], v[120:127], a[8:15], v[132:135]\n\t" \
  "ds_read_b64_tr_b8 v[114:115], v82 offset:33792\n\t" \
  "v_mfma_f32_16x16x128_f8f6f4 v[136:139], v[120:127], a[16:23], v[136:139]\n\t" \
  "ds_read_b64_tr_b8 v[116:117], v90 offset:32768\n\t" \
  "s_add_u32 s54, s68, 0x140000\n\t" \
  "buffer_load_dwordx4 a[128:131], v4, s[12:15], s54 offen offset:0 sc0 sc1\n\t" \
  "v_mfma_f32_16x16x128_f8f6f4 v[140:143], v[120:127], a[24:31], v[140:143]\n\t" \
  "ds_read_b64_tr_b8 v[118:119], v90 offset:33792\n\t" \
  "s_waitcnt lgkmcnt(4)\n\t" \
  "v_mfma_f32_16x16x128_f8f6f4 v[156:159], v[104:111], a[24:31], v[156:159]\n\t" \
  "ds_read_b64_tr_b8 v[120:121], v83 offset:32768\n\t" \
  "s_add_u32 s55, s68, 0x140000\n\t" \
  "buffer_load_dwordx4 a[132:135], v4, s[12:15], s55 offen offset:1024 sc0 sc1\n\t" \
  "v_mfma_f32_16x16x128_f8f6f4 v[152:155], v[104:111], a[16:23], v[152:155]\n\t" \
  "ds_read_b64_tr_b8 v[122:123], v83 offset:33792\n\t" \
  "v_mfma_f32_16x16x128_f8f6f4 v[148:151], v[104:111], a[8:15], v[148:151]\n\t" \
  "ds_read_b64_tr_b8 v[124:125], v91 offset:32768\n\t" \
  "s_add_u32 s48, s68, 0x140000\n\t" \
  "buffer_load_dwordx4 a[136:139], v4, s[12:15], s48 offen offset:2048 sc0 sc1\n\t" \
  "v_mfma_f32_16x16x128_f8f6f4 v[144:147], v[104:111], a[0:7], v[144:147]\n\t" \
  "ds_read_b64_tr_b8 v[126:127], v91 offset:33792\n\t" \
  "s_waitcnt lgkmcnt(4)\n\t" \
  "v_mfma_f32_16x16x128_f8f6f4 v[160:163], v[112:119], a[0:7], v[160:163]\n\t" \
  "ds_read_b64_tr_b8 v[104:105], v84 offset:32768\n\t" \
  "s_add_u32 s49, s68, 0x140000\n\t" \
  "buffer_load_dwordx4 a[140:143], v4, s[12:15], s49 offen offset:3072 sc0 sc1\n\t" \
  "v_mfma_f32_16x16x128_f8f6f4 v[164:167], v[112:119], a[8:15], v[164:167]\n\t" \
  "ds_read_b64_tr_b8 v[106:107], v84 offset:33792\n\t" \
  "v_mfma_f32_16x16x128_f8f6f4 v[168:171], v[112:119], a[16:23], v[168:171]\n\t" \
  "ds_read_b64_tr_b8 v[108:109], v92 offset:32768\n\t" \
  "s_add_u32 s50, s68, 0x141000\n\t" \
  "buffer_load_dwordx4 a[144:147], v4, s[12:15], s50 offen offset:0 sc0 sc1\n\t" \
  "v_mfma_f32_16x16x128_f8f6f4 v[172:175], v[112:119], a[24:31], v[172:175]\n\t" \
  "ds_read_b64_tr_b8 v[110:111], v92 offset:33792\n\t" \
  "s_waitcnt lgkmcnt(4)\n\t" \
  "v_mfma_f32_16x16x128_f8f6f4 v[188:191], v[120:127], a[24:31], v[188:191]\n\t" \
  "ds_read_b64_tr_b8 v[112:113], v85 offset:32768\n\t" \
  "s_add_u32 s51, s68, 0x141000\n\t" \
  "buffer_load_dwordx4 a[148:151], v4, s[12:15], s51 offen offset:1024 sc0 sc1\n\t" \
  "v_mfma_f32_16x16x128_f8f6f4 v[184:187], v[120:127], a[16:23], v[184:187]\n\t" \
  "ds_read_b64_tr_b8 v[114:115], v85 offset:33792\n\t" \
  "v_mfma_f32_16x16x128_f8f6f4 v[180:183], v[120:127], a[8:15], v[180:183]\n\t" \
  "ds_read_b64_tr_b8 v[116:117], v93 offset:32768\n\t" \
  "s_add_u32 s52, s68, 0x141000\n\t" \
  "buffer_load_dwordx4 a[152:155], v4, s[12:15], s52 offen offset:2048 sc0 sc1\n\t" \
  "v_mfma_f32_16x16x128_f8f6f4 v[176:179], v[120:127], a[0:7], v[176:179]\n\t" \
  "ds_read_b64_tr_b8 v[118:119], v93 offset:33792\n\t" \
  "s_waitcnt lgkmcnt(4)\n\t" \
  "v_mfma_f32_16x16x128_f8f6f4 v[192:195], v[104:111], a[0:7], v[192:195]\n\t" \
  "ds_read_b64_tr_b8 v[120:121], v86 offset:32768\n\t" \
  "s_add_u32 s53, s68, 0x141000\n\t" \
  "buffer_load_dwordx4 a[156:159], v4, s[12:15], s53 offen offset:3072 sc0 sc1\n\t" \
  "v_mfma_f32_16x16x128_f8f6f4 v[196:199], v[104:111], a[8:15], v[196:199]\n\t" \
  "ds_read_b64_tr_b8 v[122:123], v86 offset:33792\n\t" \
  "v_mfma_f32_16x16x128_f8f6f4 v[200:203], v[104:111], a[16:23], v[200:203]\n\t" \
  "ds_read_b64_tr_b8 v[124:125], v94 offset:32768\n\t" \
  "s_add_u32 m0, s24, 0x22000\n\t" \
  "s_add_u32 s54, s43, 0x400000\n\t" \
  "buffer_load_dwordx4 v3, s[8:11], s54 offen lds\n\t" \
  "v_mfma_f32_16x16x128_f8f6f4 v[204:207], v[104:111], a[24:31], v[204:207]\n\t" \
  "ds_read_b64_tr_b8 v[126:127], v94 offset:33792\n\t" \
  "s_waitcnt lgkmcnt(4)\n\t" \
  "v_mfma_f32_16x16x128_f8f6f4 v[220:223], v[112:119], a[24:31], v[220:223]\n\t" \
  "ds_read_b64_tr_b8 v[104:105], v87 offset:32768\n\t" \
  "s_add_u32 m0, s24, 0x22400\n\t" \
  "s_add_u32 s55, s43, 0x420000\n\t" \
  "buffer_load_dwordx4 v3, s[8:11], s55 offen lds\n\t" \
  "v_mfma_f32_16x16x128_f8f6f4 v[216:219], v[112:119], a[16:23], v[216:219]\n\t" \
  "ds_read_b64_tr_b8 v[106:107], v87 offset:33792\n\t" \
  "v_mfma_f32_16x16x128_f8f6f4 v[212:215], v[112:119], a[8:15], v[212:215]\n\t" \
  "ds_read_b64_tr_b8 v[108:109], v95 offset:32768\n\t" \
  "s_add_u32 s48, s68, 0x150000\n\t" \
  "buffer_load_dwordx4 a[160:163], v4, s[12:15], s48 offen offset:0 sc0 sc1\n\t" \
  "v_mfma_f32_16x16x128_f8f6f4 v[208:211], v[112:119], a[0:7], v[208:211]\n\t" \
  "ds_read_b64_tr_b8 v[110:111], v95 offset:33792\n\t" \
  "s_waitcnt vmcnt(49) lgkmcnt(0)\n\t" \
  "s_barrier\n\t" \
  "v_mfma_f32_16x16x128_f8f6f4 v[224:227], v[120:127], a[0:7], v[224:227]\n\t" \
  "ds_read_b64_tr_b8 v[112:113], v80 offset:40960\n\t" \
  "s_add_u32 s49, s68, 0x150000\n\t" \
  "buffer_load_dwordx4 a[164:167], v4, s[12:15], s49 offen offset:1024 sc0 sc1\n\t" \
  "v_mfma_f32_16x16x128_f8f6f4 v[228:231], v[120:127], a[8:15], v[228:231]\n\t" \
  "ds_read_b64_tr_b8 v[114:115], v80 offset:41984\n\t" \
  "v_mfma_f32_16x16x128_f8f6f4 v[232:235], v[120:127], a[16:23], v[232:235]\n\t" \
  "ds_read_b64_tr_b8 v[116:117], v88 offset:40960\n\t" \
  "s_add_u32 s50, s68, 0x150000\n\t" \
  "buffer_load_dwordx4 a[168:171], v4, s[12:15], s50 offen offset:2048 sc0 sc1\n\t" \
  "v_mfma_f32_16x16x128_f8f6f4 v[236:239], v[120:127], a[24:31], v[236:239]\n\t" \
  "ds_read_b64_tr_b8 v[118:119], v88 offset:41984\n\t" \
  "v_mfma_f32_16x16x128_f8f6f4 v[252:255], v[104:111], a[24:31], v[252:255]\n\t" \
  "ds_read_b64_tr_b8 v[120:121], v81 offset:40960\n\t" \
  "s_add_u32 s51, s68, 0x150000\n\t" \
  "buffer_load_dwordx4 a[172:175], v4, s[12:15], s51 offen offset:3072 sc0 sc1\n\t" \
  "v_mfma_f32_16x16x128_f8f6f4 v[248:251], v[104:111], a[16:23], v[248:251]\n\t" \
  "ds_read_b64_tr_b8 v[122:123], v81 offset:41984\n\t" \
  "v_mfma_f32_16x16x128_f8f6f4 v[244:247], v[104:111], a[8:15], v[244:247]\n\t" \
  "ds_read_b64_tr_b8 v[124:125], v89 offset:40960\n\t" \
  "s_add_u32 s52, s68, 0x151000\n\t" \
  "buffer_load_dwordx4 a[176:179], v4, s[12:15], s52 offen offset:0 sc0 sc1\n\t" \
  "v_mfma_f32_16x16x128_f8f6f4 v[240:243], v[104:111], a[0:7], v[240:243]\n\t" \
  "ds_read_b64_tr_b8 v[126:127], v89 offset:41984\n\t" \
  "s_waitcnt vmcnt(45) lgkmcnt(4)\n\t" \
  "v_mfma_f32_16x16x128_f8f6f4 v[128:131], v[112:119], a[32:39], v[128:131]\n\t" \
  "ds_read_b64_tr_b8 v[104:105], v82 offset:40960\n\t" \
  "s_add_u32 s53, s68, 0x151000\n\t" \
  "buffer_load_dwordx4 a[180:183], v4, s[12:15], s53 offen offset:1024 sc0 sc1\n\t" \
  "v_mfma_f32_16x16x128_f8f6f4 v[132:135], v[112:119], a[40:47], v[132:135]\n\t" \
  "ds_read_b64_tr_b8 v[106:107], v82 offset:41984\n\t" \
  "v_mfma_f32_16x16x128_f8f6f4 v[136:139], v[112:119], a[48:55], v[136:139]\n\t" \
  "ds_read_b64_tr_b8 v[108:109], v90 offset:40960\n\t" \
  "s_add_u32 s54, s68, 0x151000\n\t" \
  "buffer_load_dwordx4 a[184:187], v4, s[12:15], s54 offen offset:2048 sc0 sc1\n\t" \
  "v_mfma_f32_16x16x128_f8f6f4 v[140:143], v[112:119], a[56:63], v[140:143]\n\t" \
  "ds_read_b64_tr_b8 v[110:111], v90 offset:41984\n\t" \
  "s_waitcnt lgkmcnt(4)\n\t" \
  "v_mfma_f32_16x16x128_f8f6f4 v[156:159], v[120:127], a[56:63], v[156:159]\n\t" \
  "ds_read_b64_tr_b8 v[112:113], v83 offset:40960\n\t" \
  "s_add_u32 s55, s68, 0x151000\n\t" \
  "buffer_load_dwordx4 a[188:191], v4, s[12:15], s55 offen offset:3072 sc0 sc1\n\t" \
  "v_mfma_f32_16x16x128_f8f6f4 v[152:155], v[120:127], a[48:55], v[152:155]\n\t" \
  "ds_read_b64_tr_b8 v[114:115], v83 offset:41984\n\t" \
  "v_mfma_f32_16x16x128_f8f6f4 v[148:151], v[120:127], a[40:47], v[148:151]\n\t" \
  "ds_read_b64_tr_b8 v[116:117], v91 offset:40960\n\t" \
  "s_add_u32 m0, s24, 0x24000\n\t" \
  "s_add_u32 s48, s44, 0x400000\n\t" \
  "buffer_load_dwordx4 v3, s[8:11], s48 offen lds\n\t" \
  "v_mfma_f32_16x16x128_f8f6f4 v[144:147], v[120:127], a[32:39], v[144:147]\n\t" \
  "ds_read_b64_tr_b8 v[118:119], v91 offset:41984\n\t" \
  "s_waitcnt lgkmcnt(4)\n\t" \
  "v_mfma_f32_16x16x128_f8f6f4 v[160:163], v[104:111], a[32:39], v[160:163]\n\t" \
  "ds_read_b64_tr_b8 v[120:121], v84 offset:40960\n\t" \
  "s_add_u32 m0, s24, 0x24400\n\t" \
  "s_add_u32 s49, s44, 0x420000\n\t" \
  "buffer_load_dwordx4 v3, s[8:11], s49 offen lds\n\t" \
  "v_mfma_f32_16x16x128_f8f6f4 v[164:167], v[104:111], a[40:47], v[164:167]\n\t" \
  "ds_read_b64_tr_b8 v[122:123], v84 offset:41984\n\t" \
  "v_mfma_f32_16x16x128_f8f6f4 v[168:171], v[104:111], a[48:55], v[168:171]\n\t" \
  "ds_read_b64_tr_b8 v[124:125], v92 offset:40960\n\t" \
  "s_add_u32 s50, s68, 0x160000\n\t" \
  "buffer_load_dwordx4 a[192:195], v4, s[12:15], s50 offen offset:0 sc0 sc1\n\t" \
  "v_mfma_f32_16x16x128_f8f6f4 v[172:175], v[104:111], a[56:63], v[172:175]\n\t" \
  "ds_read_b64_tr_b8 v[126:127], v92 offset:41984\n\t" \
  "s_waitcnt lgkmcnt(4)\n\t" \
  "v_mfma_f32_16x16x128_f8f6f4 v[188:191], v[112:119], a[56:63], v[188:191]\n\t" \
  "ds_read_b64_tr_b8 v[104:105], v85 offset:40960\n\t" \
  "s_add_u32 s51, s68, 0x160000\n\t" \
  "buffer_load_dwordx4 a[196:199], v4, s[12:15], s51 offen offset:1024 sc0 sc1\n\t" \
  "v_mfma_f32_16x16x128_f8f6f4 v[184:187], v[112:119], a[48:55], v[184:187]\n\t" \
  "ds_read_b64_tr_b8 v[106:107], v85 offset:41984\n\t" \
  "v_mfma_f32_16x16x128_f8f6f4 v[180:183], v[112:119], a[40:47], v[180:183]\n\t" \
  "ds_read_b64_tr_b8 v[108:109], v93 offset:40960\n\t" \
  "s_add_u32 s52, s68, 0x160000\n\t" \
  "buffer_load_dwordx4 a[200:203], v4, s[12:15], s52 offen offset:2048 sc0 sc1\n\t" \
  "v_mfma_f32_16x16x128_f8f6f4 v[176:179], v[112:119], a[32:39], v[176:179]\n\t" \
  "ds_read_b64_tr_b8 v[110:111], v93 offset:41984\n\t" \
  "s_waitcnt lgkmcnt(4)\n\t" \
  "v_mfma_f32_16x16x128_f8f6f4 v[192:195], v[120:127], a[32:39], v[192:195]\n\t" \
  "ds_read_b64_tr_b8 v[112:113], v86 offset:40960\n\t" \
  "s_add_u32 s53, s68, 0x160000\n\t" \
  "buffer_load_dwordx4 a[204:207], v4, s[12:15], s53 offen offset:3072 sc0 sc1\n\t" \
  "v_mfma_f32_16x16x128_f8f6f4 v[196:199], v[120:127], a[40:47], v[196:199]\n\t" \
  "ds_read_b64_tr_b8 v[114:115], v86 offset:41984\n\t" \
  "v_mfma_f32_16x16x128_f8f6f4 v[200:203], v[120:127], a[48:55], v[200:203]\n\t" \
  "ds_read_b64_tr_b8 v[116:117], v94 offset:40960\n\t" \
  "s_add_u32 s54, s68, 0x161000\n\t" \
  "buffer_load_dwordx4 a[208:211], v4, s[12:15], s54 offen offset:0 sc0 sc1\n\t" \
  "v_mfma_f32_16x16x128_f8f6f4 v[204:207], v[120:127], a[56:63], v[204:207]\n\t" \
  "ds_read_b64_tr_b8 v[118:119], v94 offset:41984\n\t" \
  "s_waitcnt lgkmcnt(4)\n\t" \
  "v_mfma_f32_16x16x128_f8f6f4 v[220:223], v[104:111], a[56:63], v[220:223]\n\t" \
  "ds_read_b64_tr_b8 v[120:121], v87 offset:40960\n\t" \
  "s_add_u32 s55, s68, 0x161000\n\t" \
  "buffer_load_dwordx4 a[212:215], v4, s[12:15], s55 offen offset:1024 sc0 sc1\n\t" \
  "v_mfma_f32_16x16x128_f8f6f4 v[216:219], v[104:111], a[48:55], v[216:219]\n\t" \
  "ds_read_b64_tr_b8 v[122:123], v87 offset:41984\n\t" \
  "v_mfma_f32_16x16x128_f8f6f4 v[212:215], v[104:111], a[40:47], v[212:215]\n\t" \
  "ds_read_b64_tr_b8 v[124:125], v95 offset:40960\n\t" \
  "s_add_u32 s48, s68, 0x161000\n\t" \
  "buffer_load_dwordx4 a[216:219], v4, s[12:15], s48 offen offset:2048 sc0 sc1\n\t" \
  "v_mfma_f32_16x16x128_f8f6f4 v[208:211], v[104:111], a[32:39], v[208:211]\n\t" \
  "ds_read_b64_tr_b8 v[126:127], v95 offset:41984\n\t" \
  "s_waitcnt vmcnt(55) lgkmcnt(0)\n\t" \
  "s_barrier\n\t" \
  "v_mfma_f32_16x16x128_f8f6f4 v[224:227], v[112:119], a[32:39], v[224:227]\n\t" \
  "ds_read_b64_tr_b8 v[104:105], v80 offset:49152\n\t" \
  "s_add_u32 s49, s68, 0x161000\n\t" \
  "buffer_load_dwordx4 a[220:223], v4, s[12:15], s49 offen offset:3072 sc0 sc1\n\t" \
  "v_mfma_f32_16x16x128_f8f6f4 v[228:231], v[112:119], a[40:47], v[228:231]\n\t" \
  "ds_read_b64_tr_b8 v[106:107], v80 offset:50176\n\t" \
  "v_mfma_f32_16x16x128_f8f6f4 v[232:235], v[112:119], a[48:55], v[232:235]\n\t" \
  "ds_read_b64_tr_b8 v[108:109], v88 offset:49152\n\t" \
  "s_add_u32 m0, s24, 0x4000\n\t" \
  "s_add_u32 s50, s41, 0x600000\n\t" \
  "buffer_load_dwordx4 v2, s[8:11], s50 offen lds\n\t" \
  "v_mfma_f32_16x16x128_f8f6f4 v[236:239], v[112:119], a[56:63], v[236:239]\n\t" \
  "ds_read_b64_tr_b8 v[110:111], v88 offset:50176\n\t" \
  "v_mfma_f32_16x16x128_f8f6f4 v[252:255], v[120:127], a[56:63], v[252:255]\n\t" \
  "ds_read_b64_tr_b8 v[112:113], v81 offset:49152\n\t" \
  "s_add_u32 m0, s24, 0x4400\n\t" \
  "s_add_u32 s51, s41, 0x620000\n\t" \
  "buffer_load_dwordx4 v2, s[8:11], s51 offen lds\n\t" \
  "v_mfma_f32_16x16x128_f8f6f4 v[248:251], v[120:127], a[48:55], v[248:251]\n\t" \
  "ds_read_b64_tr_b8 v[114:115], v81 offset:50176\n\t" \
  "v_mfma_f32_16x16x128_f8f6f4 v[244:247], v[120:127], a[40:47], v[244:247]\n\t" \
  "ds_read_b64_tr_b8 v[116:117], v89 offset:49152\n\t" \
  "s_add_u32 m0, s24, 0x6000\n\t" \
  "s_add_u32 s52, s42, 0x600000\n\t" \
  "buffer_load_dwordx4 v2, s[8:11], s52 offen lds\n\t" \
  "v_mfma_f32_16x16x128_f8f6f4 v[240:243], v[120:127], a[32:39], v[240:243]\n\t" \
  "ds_read_b64_tr_b8 v[118:119], v89 offset:50176\n\t" \
  "s_waitcnt vmcnt(51) lgkmcnt(4)\n\t" \
  "v_mfma_f32_16x16x128_f8f6f4 v[128:131], v[104:111], a[64:71], v[128:131]\n\t" \
  "ds_read_b64_tr_b8 v[120:121], v82 offset:49152\n\t" \
  "s_add_u32 m0, s24, 0x6400\n\t" \
  "s_add_u32 s53, s42, 0x620000\n\t" \
  "buffer_load_dwordx4 v2, s[8:11], s53 offen lds\n\t" \
  "v_mfma_f32_16x16x128_f8f6f4 v[132:135], v[104:111], a[72:79], v[132:135]\n\t" \
  "ds_read_b64_tr_b8 v[122:123], v82 offset:50176\n\t" \
  "v_mfma_f32_16x16x128_f8f6f4 v[136:139], v[104:111], a[80:87], v[136:139]\n\t" \
  "ds_read_b64_tr_b8 v[124:125], v90 offset:49152\n\t" \
  "s_add_u32 m0, s24, 0x8000\n\t" \
  "s_add_u32 s54, s43, 0x600000\n\t" \
  "buffer_load_dwordx4 v2, s[8:11], s54 offen lds\n\t" \
  "v_mfma_f32_16x16x128_f8f6f4 v[140:143], v[104:111], a[88:95], v[140:143]\n\t" \
  "ds_read_b64_tr_b8 v[126:127], v90 offset:50176\n\t" \
  "s_waitcnt lgkmcnt(4)\n\t" \
  "v_mfma_f32_16x16x128_f8f6f4 v[156:159], v[112:119], a[88:95], v[156:159]\n\t" \
  "ds_read_b64_tr_b8 v[104:105], v83 offset:49152\n\t" \
  "s_add_u32 m0, s24, 0x8400\n\t" \
  "s_add_u32 s55, s43, 0x620000\n\t" \
  "buffer_load_dwordx4 v2, s[8:11], s55 offen lds\n\t" \
  "v_mfma_f32_16x16x128_f8f6f4 v[152:155], v[112:119], a[80:87], v[152:155]\n\t" \
  "ds_read_b64_tr_b8 v[106:107], v83 offset:50176\n\t" \
  "v_mfma_f32_16x16x128_f8f6f4 v[148:151], v[112:119], a[72:79], v[148:151]\n\t" \
  "ds_read_b64_tr_b8 v[108:109], v91 offset:49152\n\t" \
  "s_add_u32 m0, s24, 0xa000\n\t" \
  "s_add_u32 s48, s44, 0x600000\n\t" \
  "buffer_load_dwordx4 v2, s[8:11], s48 offen lds\n\t" \
  "v_mfma_f32_16x16x128_f8f6f4 v[144:147], v[112:119], a[64:71], v[144:147]\n\t" \
  "ds_read_b64_tr_b8 v[110:111], v91 offset:50176\n\t" \
  "s_waitcnt lgkmcnt(4)\n\t" \
  "v_mfma_f32_16x16x128_f8f6f4 v[160:163], v[120:127], a[64:71], v[160:163]\n\t" \
  "ds_read_b64_tr_b8 v[112:113], v84 offset:49152\n\t" \
  "s_add_u32 m0, s24, 0xa400\n\t" \
  "s_add_u32 s49, s44, 0x620000\n\t" \
  "buffer_load_dwordx4 v2, s[8:11], s49 offen lds\n\t" \
  "v_mfma_f32_16x16x128_f8f6f4 v[164:167], v[120:127], a[72:79], v[164:167]\n\t" \
  "ds_read_b64_tr_b8 v[114:115], v84 offset:50176\n\t" \
  "v_mfma_f32_16x16x128_f8f6f4 v[168:171], v[120:127], a[80:87], v[168:171]\n\t" \
  "ds_read_b64_tr_b8 v[116:117], v92 offset:49152\n\t" \
  "v_mfma_f32_16x16x128_f8f6f4 v[172:175], v[120:127], a[88:95], v[172:175]\n\t" \
  "ds_read_b64_tr_b8 v[118:119], v92 offset:50176\n\t" \
  "s_waitcnt lgkmcnt(4)\n\t" \
  "v_mfma_f32_16x16x128_f8f6f4 v[188:191], v[104:111], a[88:95], v[188:191]\n\t" \
  "ds_read_b64_tr_b8 v[120:121], v85 offset:49152\n\t" \
  "v_mfma_f32_16x16x128_f8f6f4 v[184:187], v[104:111], a[80:87], v[184:187]\n\t" \
  "ds_read_b64_tr_b8 v[122:123], v85 offset:50176\n\t" \
  "v_mfma_f32_16x16x128_f8f6f4 v[180:183], v[104:111], a[72:79], v[180:183]\n\t" \
  "ds_read_b64_tr_b8 v[124:125], v93 offset:49152\n\t" \
  "v_mfma_f32_16x16x128_f8f6f4 v[176:179], v[104:111], a[64:71], v[176:179]\n\t" \
  "ds_read_b64_tr_b8 v[126:127], v93 offset:50176\n\t" \
  "s_waitcnt lgkmcnt(4)\n\t" \
  "v_mfma_f32_16x16x128_f8f6f4 v[192:195], v[112:119], a[64:71], v[192:195]\n\t" \
  "ds_read_b64_tr_b8 v[104:105], v86 offset:49152\n\t" \
  "v_mfma_f32_16x16x128_f8f6f4 v[196:199], v[112:119], a[72:79], v[196:199]\n\t" \
  "ds_read_b64_tr_b8 v[106:107], v86 offset:50176\n\t" \
  "v_mfma_f32_16x16x128_f8f6f4 v[200:203], v[112:119], a[80:87], v[200:203]\n\t" \
  "ds_read_b64_tr_b8 v[108:109], v94 offset:49152\n\t" \
  "v_mfma_f32_16x16x128_f8f6f4 v[204:207], v[112:119], a[88:95], v[204:207]\n\t" \
  "ds_read_b64_tr_b8 v[110:111], v94 offset:50176\n\t" \
  "s_waitcnt lgkmcnt(4)\n\t" \
  "v_mfma_f32_16x16x128_f8f6f4 v[220:223], v[120:127], a[88:95], v[220:223]\n\t" \
  "ds_read_b64_tr_b8 v[112:113], v87 offset:49152\n\t" \
  "v_mfma_f32_16x16x128_f8f6f4 v[216:219], v[120:127], a[80:87], v[216:219]\n\t" \
  "ds_read_b64_tr_b8 v[114:115], v87 offset:50176\n\t" \
  "v_mfma_f32_16x16x128_f8f6f4 v[212:215], v[120:127], a[72:79], v[212:215]\n\t" \
  "ds_read_b64_tr_b8 v[116:117], v95 offset:49152\n\t" \
  "v_mfma_f32_16x16x128_f8f6f4 v[208:211], v[120:127], a[64:71], v[208:211]\n\t" \
  "ds_read_b64_tr_b8 v[118:119], v95 offset:50176\n\t" \
  "s_waitcnt vmcnt(54) lgkmcnt(0)\n\t" \
  "s_barrier\n\t" \
  "v_mfma_f32_16x16x128_f8f6f4 v[224:227], v[104:111], a[64:71], v[224:227]\n\t" \
  "ds_read_b64_tr_b8 v[120:121], v80 offset:57344\n\t" \
  "s_add_u32 m0, s24, 0x26000\n\t" \
  "s_add_u32 s50, s45, 0x400000\n\t" \
  "buffer_load_dwordx4 v3, s[8:11], s50 offen lds\n\t" \
  "v_mfma_f32_16x16x128_f8f6f4 v[228:231], v[104:111], a[72:79], v[228:231]\n\t" \
  "ds_read_b64_tr_b8 v[122:123], v80 offset:58368\n\t" \
  "v_mfma_f32_16x16x128_f8f6f4 v[232:235], v[104:111], a[80:87], v[232:235]\n\t" \
  "ds_read_b64_tr_b8 v[124:125], v88 offset:57344\n\t" \
  "s_add_u32 m0, s24, 0x26400\n\t" \
  "s_add_u32 s51, s45, 0x420000\n\t" \
  "buffer_load_dwordx4 v3, s[8:11], s51 offen lds\n\t" \
  "v_mfma_f32_16x16x128_f8f6f4 v[236:239], v[104:111], a[88:95], v[236:239]\n\t" \
  "ds_read_b64_tr_b8 v[126:127], v88 offset:58368\n\t" \
  "v_mfma_f32_16x16x128_f8f6f4 v[252:255], v[112:119], a[88:95], v[252:255]\n\t" \
  "ds_read_b64_tr_b8 v[104:105], v81 offset:57344\n\t" \
  "s_add_u32 s52, s68, 0x170000\n\t" \
  "buffer_load_dwordx4 a[224:227], v4, s[12:15], s52 offen offset:0 sc0 sc1\n\t" \
  "v_mfma_f32_16x16x128_f8f6f4 v[248:251], v[112:119], a[80:87], v[248:251]\n\t" \
  "ds_read_b64_tr_b8 v[106:107], v81 offset:58368\n\t" \
  "v_mfma_f32_16x16x128_f8f6f4 v[244:247], v[112:119], a[72:79], v[244:247]\n\t" \
  "ds_read_b64_tr_b8 v[108:109], v89 offset:57344\n\t" \
  "s_add_u32 s53, s68, 0x170000\n\t" \
  "buffer_load_dwordx4 a[228:231], v4, s[12:15], s53 offen offset:1024 sc0 sc1\n\t" \
  "v_mfma_f32_16x16x128_f8f6f4 v[240:243], v[112:119], a[64:71], v[240:243]\n\t" \
  "ds_read_b64_tr_b8 v[110:111], v89 offset:58368\n\t" \
  "s_waitcnt vmcnt(50) lgkmcnt(4)\n\t" \
  "v_mfma_f32_16x16x128_f8f6f4 v[128:131], v[120:127], a[96:103], v[128:131]\n\t" \
  "ds_read_b64_tr_b8 v[112:113], v82 offset:57344\n\t" \
  "s_add_u32 s54, s68, 0x170000\n\t" \
  "buffer_load_dwordx4 a[232:235], v4, s[12:15], s54 offen offset:2048 sc0 sc1\n\t" \
  "s_mov_b32 exec_lo, 0x7f000000\n\t" \
  "s_mov_b32 exec_hi, 0\n\t" \
  "global_load_dwordx4 v[68:71], v10, s[6:7] sc0 sc1\n\t" \
  "s_mov_b64 exec, -1\n\t" \
  "v_mfma_f32_16x16x128_f8f6f4 v[132:135], v[120:127], a[104:111], v[132:135]\n\t" \
  "ds_read_b64_tr_b8 v[114:115], v82 offset:58368\n\t" \
  "v_mfma_f32_16x16x128_f8f6f4 v[136:139], v[120:127], a[112:119], v[136:139]\n\t" \
  "ds_read_b64_tr_b8 v[116:117], v90 offset:57344\n\t" \
  "s_add_u32 s55, s68, 0x170000\n\t" \
  "buffer_load_dwordx4 a[236:239], v4, s[12:15], s55 offen offset:3072 sc0 sc1\n\t" \
  "v_mfma_f32_16x16x128_f8f6f4 v[140:143], v[120:127], a[120:127], v[140:143]\n\t" \
  "ds_read_b64_tr_b8 v[118:119], v90 offset:58368\n\t" \
  "s_waitcnt lgkmcnt(4)\n\t" \
  "v_mfma_f32_16x16x128_f8f6f4 v[156:159], v[104:111], a[120:127], v[156:159]\n\t" \
  "ds_read_b64_tr_b8 v[120:121], v83 offset:57344\n\t" \
  "s_add_u32 s48, s68, 0x171000\n\t" \
  "buffer_load_dwordx4 a[240:243], v4, s[12:15], s48 offen offset:0 sc0 sc1\n\t" \
  "v_mfma_f32_16x16x128_f8f6f4 v[152:155], v[104:111], a[112:119], v[152:155]\n\t" \
  "ds_read_b64_tr_b8 v[122:123], v83 offset:58368\n\t" \
  "v_mfma_f32_16x16x128_f8f6f4 v[148:151], v[104:111], a[104:111], v[148:151]\n\t" \
  "ds_read_b64_tr_b8 v[124:125], v91 offset:57344\n\t" \
  "s_add_u32 s49, s68, 0x171000\n\t" \
  "buffer_load_dwordx4 a[244:247], v4, s[12:15], s49 offen offset:1024 sc0 sc1\n\t" \
  "v_mfma_f32_16x16x128_f8f6f4 v[144:147], v[104:111], a[96:103], v[144:147]\n\t" \
  "ds_read_b64_tr_b8 v[126:127], v91 offset:58368\n\t" \
  "s_waitcnt lgkmcnt(4)\n\t" \
  "v_mfma_f32_16x16x128_f8f6f4 v[160:163], v[112:119], a[96:103], v[160:163]\n\t" \
  "ds_read_b64_tr_b8 v[104:105], v84 offset:57344\n\t" \
  "s_add_u32 s50, s68, 0x171000\n\t" \
  "buffer_load_dwordx4 a[248:251], v4, s[12:15], s50 offen offset:2048 sc0 sc1\n\t" \
  "v_mfma_f32_16x16x128_f8f6f4 v[164:167], v[112:119], a[104:111], v[164:167]\n\t" \
  "ds_read_b64_tr_b8 v[106:107], v84 offset:58368\n\t" \
  "v_mfma_f32_16x16x128_f8f6f4 v[168:171], v[112:119], a[112:119], v[168:171]\n\t" \
  "ds_read_b64_tr_b8 v[108:109], v92 offset:57344\n\t" \
  "s_add_u32 s51, s68, 0x171000\n\t" \
  "buffer_load_dwordx4 a[252:255], v4, s[12:15], s51 offen offset:3072 sc0 sc1\n\t" \
  "v_mfma_f32_16x16x128_f8f6f4 v[172:175], v[112:119], a[120:127], v[172:175]\n\t" \
  "ds_read_b64_tr_b8 v[110:111], v92 offset:58368\n\t" \
  "s_waitcnt lgkmcnt(4)\n\t" \
  "v_mfma_f32_16x16x128_f8f6f4 v[188:191], v[120:127], a[120:127], v[188:191]\n\t" \
  "ds_read_b64_tr_b8 v[112:113], v85 offset:57344\n\t" \
  "s_add_u32 m0, s24, 0x18000\n\t" \
  "s_add_u32 s52, s46, 0x400000\n\t" \
  "buffer_load_dwordx4 v3, s[8:11], s52 offen lds\n\t" \
  "v_mfma_f32_16x16x128_f8f6f4 v[184:187], v[120:127], a[112:119], v[184:187]\n\t" \
  "ds_read_b64_tr_b8 v[114:115], v85 offset:58368\n\t" \
  "v_mfma_f32_16x16x128_f8f6f4 v[180:183], v[120:127], a[104:111], v[180:183]\n\t" \
  "ds_read_b64_tr_b8 v[116:117], v93 offset:57344\n\t" \
  "s_add_u32 m0, s24, 0x18400\n\t" \
  "s_add_u32 s53, s46, 0x420000\n\t" \
  "buffer_load_dwordx4 v3, s[8:11], s53 offen lds\n\t" \
  "v_mfma_f32_16x16x128_f8f6f4 v[176:179], v[120:127], a[96:103], v[176:179]\n\t" \
  "ds_read_b64_tr_b8 v[118:119], v93 offset:58368\n\t" \
  "s_waitcnt lgkmcnt(4)\n\t" \
  "v_mfma_f32_16x16x128_f8f6f4 v[192:195], v[104:111], a[96:103], v[192:195]\n\t" \
  "ds_read_b64_tr_b8 v[120:121], v86 offset:57344\n\t" \
  "s_add_u32 s54, s68, 0x180000\n\t" \
  "buffer_load_dwordx4 a[0:3], v4, s[12:15], s54 offen offset:0 sc0 sc1\n\t" \
  "v_mfma_f32_16x16x128_f8f6f4 v[196:199], v[104:111], a[104:111], v[196:199]\n\t" \
  "ds_read_b64_tr_b8 v[122:123], v86 offset:58368\n\t" \
  "v_mfma_f32_16x16x128_f8f6f4 v[200:203], v[104:111], a[112:119], v[200:203]\n\t" \
  "ds_read_b64_tr_b8 v[124:125], v94 offset:57344\n\t" \
  "s_add_u32 s55, s68, 0x180000\n\t" \
  "buffer_load_dwordx4 a[4:7], v4, s[12:15], s55 offen offset:1024 sc0 sc1\n\t" \
  "v_mfma_f32_16x16x128_f8f6f4 v[204:207], v[104:111], a[120:127], v[204:207]\n\t" \
  "ds_read_b64_tr_b8 v[126:127], v94 offset:58368\n\t" \
  "s_waitcnt lgkmcnt(4)\n\t" \
  "v_mfma_f32_16x16x128_f8f6f4 v[220:223], v[112:119], a[120:127], v[220:223]\n\t" \
  "ds_read_b64_tr_b8 v[104:105], v87 offset:57344\n\t" \
  "s_add_u32 s48, s68, 0x180000\n\t" \
  "buffer_load_dwordx4 a[8:11], v4, s[12:15], s48 offen offset:2048 sc0 sc1\n\t" \
  "v_mfma_f32_16x16x128_f8f6f4 v[216:219], v[112:119], a[112:119], v[216:219]\n\t" \
  "ds_read_b64_tr_b8 v[106:107], v87 offset:58368\n\t" \
  "v_mfma_f32_16x16x128_f8f6f4 v[212:215], v[112:119], a[104:111], v[212:215]\n\t" \
  "ds_read_b64_tr_b8 v[108:109], v95 offset:57344\n\t" \
  "s_add_u32 s49, s68, 0x180000\n\t" \
  "s_waitcnt vmcnt(60)\n\t" \
  "buffer_load_dwordx4 a[12:15], v4, s[12:15], s49 offen offset:3072 sc0 sc1\n\t" \
  "v_mfma_f32_16x16x128_f8f6f4 v[208:211], v[112:119], a[96:103], v[208:211]\n\t" \
  "ds_read_b64_tr_b8 v[110:111], v95 offset:58368\n\t" \
  "s_waitcnt vmcnt(53) lgkmcnt(0)\n\t" \
  "s_barrier\n\t" \
  "v_mfma_f32_16x16x128_f8f6f4 v[224:227], v[120:127], a[96:103], v[224:227]\n\t" \
  "ds_read_b64_tr_b8 v[112:113], v88 offset:0\n\t" \
  "s_add_u32 s50, s68, 0x181000\n\t" \
  "buffer_load_dwordx4 a[16:19], v4, s[12:15], s50 offen offset:0 sc0 sc1\n\t" \
  "v_mfma_f32_16x16x128_f8f6f4 v[228:231], v[120:127], a[104:111], v[228:231]\n\t" \
  "ds_read_b64_tr_b8 v[114:115], v88 offset:1024\n\t" \
  "v_mfma_f32_16x16x128_f8f6f4 v[232:235], v[120:127], a[112:119], v[232:235]\n\t" \
  "ds_read_b64_tr_b8 v[116:117], v96 offset:0\n\t" \
  "s_add_u32 s51, s68, 0x181000\n\t" \
  "buffer_load_dwordx4 a[20:23], v4, s[12:15], s51 offen offset:1024 sc0 sc1\n\t" \
  "v_mfma_f32_16x16x128_f8f6f4 v[236:239], v[120:127], a[120:127], v[236:239]\n\t" \
  "ds_read_b64_tr_b8 v[118:119], v96 offset:1024\n\t" \
  "v_mfma_f32_16x16x128_f8f6f4 v[252:255], v[104:111], a[120:127], v[252:255]\n\t" \
  "ds_read_b64_tr_b8 v[120:121], v89 offset:0\n\t" \
  "s_add_u32 s52, s68, 0x181000\n\t" \
  "buffer_load_dwordx4 a[24:27], v4, s[12:15], s52 offen offset:2048 sc0 sc1\n\t" \
  "v_mfma_f32_16x16x128_f8f6f4 v[248:251], v[104:111], a[112:119], v[248:251]\n\t" \
  "ds_read_b64_tr_b8 v[122:123], v89 offset:1024\n\t" \
  "v_mfma_f32_16x16x128_f8f6f4 v[244:247], v[104:111], a[104:111], v[244:247]\n\t" \
  "ds_read_b64_tr_b8 v[124:125], v97 offset:0\n\t" \
  "s_add_u32 s53, s68, 0x181000\n\t" \
  "buffer_load_dwordx4 a[28:31], v4, s[12:15], s53 offen offset:3072 sc0 sc1\n\t" \
  "v_mfma_f32_16x16x128_f8f6f4 v[240:243], v[104:111], a[96:103], v[240:243]\n\t" \
  "ds_read_b64_tr_b8 v[126:127], v97 offset:1024\n\t" \
  "s_waitcnt vmcnt(49) lgkmcnt(4)\n\t" \
  "v_mfma_f32_16x16x128_f8f6f4 v[128:131], v[112:119], a[128:135], v[128:131]\n\t" \
  "ds_read_b64_tr_b8 v[104:105], v90 offset:0\n\t" \
  "s_add_u32 m0, s24, 0x1a000\n\t" \
  "s_add_u32 s54, s40, 0x600000\n\t" \
  "buffer_load_dwordx4 v3, s[8:11], s54 offen lds\n\t" \
  "v_mfma_f32_16x16x128_f8f6f4 v[132:135], v[112:119], a[136:143], v[132:135]\n\t" \
  "ds_read_b64_tr_b8 v[106:107], v90 offset:1024\n\t" \
  "v_mfma_f32_16x16x128_f8f6f4 v[136:139], v[112:119], a[144:151], v[136:139]\n\t" \
  "ds_read_b64_tr_b8 v[108:109], v98 offset:0\n\t" \
  "s_add_u32 m0, s24, 0x1a400\n\t" \
  "s_add_u32 s55, s40, 0x620000\n\t" \
  "buffer_load_dwordx4 v3, s[8:11], s55 offen lds\n\t" \
  "v_mfma_f32_16x16x128_f8f6f4 v[140:143], v[112:119], a[152:159], v[140:143]\n\t" \
  "ds_read_b64_tr_b8 v[110:111], v98 offset:1024\n\t" \
  "s_waitcnt lgkmcnt(4)\n\t" \
  "v_mfma_f32_16x16x128_f8f6f4 v[156:159], v[120:127], a[152:159], v[156:159]\n\t" \
  "ds_read_b64_tr_b8 v[112:113], v91 offset:0\n\t" \
  "s_waitcnt vmcnt(17)\n\t" \
  "v_min_u32 v68, v68, v69\n\t" \
  "v_min3_u32 v68, v68, v70, v71\n\t" \
  "v_cmp_le_u32 vcc, s57, v68\n\t" \
  "s_and_b32 s64, vcc_lo, 0x7f000000\n\t" \
  "s_or_b32 s56, s56, s64\n\t" \
  "s_and_b32 s64, s56, 0x7f000000\n\t" \
  "s_cmp_eq_u32 s64, 0x7f000000\n\t" \
  "s_cbranch_scc1 Lok9_0_%=\n\t" \
  "s_mov_b32 s62, 0x7f000000\n\t" \
  "s_mov_b32 s69, 0x7f000000\n\t" \
  "s_call_b64 s[60:61], Lpoll_0_%=\n\t" \
  "Lok9_0_%=:\n\t" \
  "s_add_u32 s48, s68, 0x190000\n\t" \
  "buffer_load_dwordx4 a[32:35], v4, s[12:15], s48 offen offset:0 sc0 sc1\n\t" \
  "v_mfma_f32_16x16x128_f8f6f4 v[152:155], v[120:127], a[144:151], v[152:155]\n\t" \
  "ds_read_b64_tr_b8 v[114:115], v91 offset:1024\n\t" \
  "v_mfma_f32_16x16x128_f8f6f4 v[148:151], v[120:127], a[136:143], v[148:151]\n\t" \
  "ds_read_b64_tr_b8 v[116:117], v99 offset:0\n\t" \
  "s_add_u32 s49, s68, 0x190000\n\t" \
  "buffer_load_dwordx4 a[36:39], v4, s[12:15], s49 offen offset:1024 sc0 sc1\n\t" \
  "v_mfma_f32_16x16x128_f8f6f4 v[144:147], v[120:127], a[128:135], v[144:147]\n\t" \
  "ds_read_b64_tr_b8 v[118:119], v99 offset:1024\n\t" \
  "s_waitcnt lgkmcnt(4)\n\t" \
  "v_mfma_f32_16x16x128_f8f6f4 v[160:163], v[104:111], a[128:135], v[160:163]\n\t" \
  "ds_read_b64_tr_b8 v[120:121], v92 offset:0\n\t" \
  "s_add_u32 s50, s68, 0x190000\n\t" \
  "buffer_load_dwordx4 a[40:43], v4, s[12:15], s50 offen offset:2048 sc0 sc1\n\t" \
  "v_mfma_f32_16x16x128_f8f6f4 v[164:167], v[104:111], a[136:143], v[164:167]\n\t" \
  "ds_read_b64_tr_b8 v[122:123], v92 offset:1024\n\t" \
  "v_mfma_f32_16x16x128_f8f6f4 v[168:171], v[104:111], a[144:151], v[168:171]\n\t" \
  "ds_read_b64_tr_b8 v[124:125], v100 offset:0\n\t" \
  "s_add_u32 s51, s68, 0x190000\n\t" \
  "buffer_load_dwordx4 a[44:47], v4, s[12:15], s51 offen offset:3072 sc0 sc1\n\t" \
  "v_mfma_f32_16x16x128_f8f6f4 v[172:175], v[104:111], a[152:159], v[172:175]\n\t" \
  "ds_read_b64_tr_b8 v[126:127], v100 offset:1024\n\t" \
  "s_waitcnt lgkmcnt(4)\n\t" \
  "v_mfma_f32_16x16x128_f8f6f4 v[188:191], v[112:119], a[152:159], v[188:191]\n\t" \
  "ds_read_b64_tr_b8 v[104:105], v93 offset:0\n\t" \
  "s_add_u32 s52, s68, 0x191000\n\t" \
  "buffer_load_dwordx4 a[48:51], v4, s[12:15], s52 offen offset:0 sc0 sc1\n\t" \
  "v_mfma_f32_16x16x128_f8f6f4 v[184:187], v[112:119], a[144:151], v[184:187]\n\t" \
  "ds_read_b64_tr_b8 v[106:107], v93 offset:1024\n\t" \
  "v_mfma_f32_16x16x128_f8f6f4 v[180:183], v[112:119], a[136:143], v[180:183]\n\t" \
  "ds_read_b64_tr_b8 v[108:109], v101 offset:0\n\t" \
  "s_add_u32 s53, s68, 0x191000\n\t" \
  "buffer_load_dwordx4 a[52:55], v4, s[12:15], s53 offen offset:1024 sc0 sc1\n\t" \
  "v_mfma_f32_16x16x128_f8f6f4 v[176:179], v[112:119], a[128:135], v[176:179]\n\t" \
  "ds_read_b64_tr_b8 v[110:111], v101 offset:1024\n\t" \
  "s_waitcnt lgkmcnt(4)\n\t" \
  "v_mfma_f32_16x16x128_f8f6f4 v[192:195], v[120:127], a[128:135], v[192:195]\n\t" \
  "ds_read_b64_tr_b8 v[112:113], v94 offset:0\n\t" \
  "s_add_u32 s54, s68, 0x191000\n\t" \
  "buffer_load_dwordx4 a[56:59], v4, s[12:15], s54 offen offset:2048 sc0 sc1\n\t" \
  "v_mfma_f32_16x16x128_f8f6f4 v[196:199], v[120:127], a[136:143], v[196:199]\n\t" \
  "ds_read_b64_tr_b8 v[114:115], v94 offset:1024\n\t" \
  "v_mfma_f32_16x16x128_f8f6f4 v[200:203], v[120:127], a[144:151], v[200:203]\n\t" \
  "ds_read_b64_tr_b8 v[116:117], v102 offset:0\n\t" \
  "s_add_u32 s55, s68, 0x191000\n\t" \
  "buffer_load_dwordx4 a[60:63], v4, s[12:15], s55 offen offset:3072 sc0 sc1\n\t" \
  "v_mfma_f32_16x16x128_f8f6f4 v[204:207], v[120:127], a[152:159], v[204:207]\n\t" \
  "ds_read_b64_tr_b8 v[118:119], v102 offset:1024\n\t" \
  "s_waitcnt lgkmcnt(4)\n\t" \
  "v_mfma_f32_16x16x128_f8f6f4 v[220:223], v[104:111], a[152:159], v[220:223]\n\t" \
  "ds_read_b64_tr_b8 v[120:121], v95 offset:0\n\t" \
  "s_add_u32 m0, s24, 0x1c000\n\t" \
  "s_add_u32 s48, s41, 0x600000\n\t" \
  "buffer_load_dwordx4 v3, s[8:11], s48 offen lds\n\t" \
  "v_mfma_f32_16x16x128_f8f6f4 v[216:219], v[104:111], a[144:151], v[216:219]\n\t" \
  "ds_read_b64_tr_b8 v[122:123], v95 offset:1024\n\t" \
  "v_mfma_f32_16x16x128_f8f6f4 v[212:215], v[104:111], a[136:143], v[212:215]\n\t" \
  "ds_read_b64_tr_b8 v[124:125], v103 offset:0\n\t" \
  "s_add_u32 m0, s24, 0x1c400\n\t" \
  "s_add_u32 s49, s41, 0x620000\n\t" \
  "buffer_load_dwordx4 v3, s[8:11], s49 offen lds\n\t" \
  "v_mfma_f32_16x16x128_f8f6f4 v[208:211], v[104:111], a[128:135], v[208:211]\n\t" \
  "ds_read_b64_tr_b8 v[126:127], v103 offset:1024\n\t" \
  "s_waitcnt lgkmcnt(0)\n\t" \
  "s_barrier\n\t" \
  "v_mfma_f32_16x16x128_f8f6f4 v[224:227], v[112:119], a[128:135], v[224:227]\n\t" \
  "ds_read_b64_tr_b8 v[104:105], v88 offset:8192\n\t" \
  "s_add_u32 s50, s68, 0x1a0000\n\t" \
  "buffer_load_dwordx4 a[64:67], v4, s[12:15], s50 offen offset:0 sc0 sc1\n\t" \
  "v_mfma_f32_16x16x128_f8f6f4 v[228:231], v[112:119], a[136:143], v[228:231]\n\t" \
  "ds_read_b64_tr_b8 v[106:107], v88 offset:9216\n\t" \
  "v_mfma_f32_16x16x128_f8f6f4 v[232:235], v[112:119], a[144:151], v[232:235]\n\t" \
  "ds_read_b64_tr_b8 v[108:109], v96 offset:8192\n\t" \
  "s_add_u32 s51, s68, 0x1a0000\n\t" \
  "buffer_load_dwordx4 a[68:71], v4, s[12:15], s51 offen offset:1024 sc0 sc1\n\t" \
  "v_mfma_f32_16x16x128_f8f6f4 v[236:239], v[112:119], a[152:159], v[236:239]\n\t" \
  "ds_read_b64_tr_b8 v[110:111], v96 offset:9216\n\t" \
  "v_mfma_f32_16x16x128_f8f6f4 v[252:255], v[120:127], a[152:159], v[252:255]\n\t" \
  "ds_read_b64_tr_b8 v[112:113], v89 offset:8192\n\t" \
  "s_add_u32 s52, s68, 0x1a0000\n\t" \
  "buffer_load_dwordx4 a[72:75], v4, s[12:15], s52 offen offset:2048 sc0 sc1\n\t" \
  "v_mfma_f32_16x16x128_f8f6f4 v[248:251], v[120:127], a[144:151], v[248:251]\n\t" \
  "ds_read_b64_tr_b8 v[114:115], v89 offset:9216\n\t" \
  "v_mfma_f32_16x16x128_f8f6f4 v[244:247], v[120:127], a[136:143], v[244:247]\n\t" \
  "ds_read_b64_tr_b8 v[116:117], v97 offset:8192\n\t" \
  "s_add_u32 s53, s68, 0x1a0000\n\t" \
  "buffer_load_dwordx4 a[76:79], v4, s[12:15], s53 offen offset:3072 sc0 sc1\n\t" \
  "v_mfma_f32_16x16x128_f8f6f4 v[240:243], v[120:127], a[128:135], v[240:243]\n\t" \
  "ds_read_b64_tr_b8 v[118:119], v97 offset:9216\n\t" \
  "s_waitcnt lgkmcnt(4)\n\t" \
  "v_mfma_f32_16x16x128_f8f6f4 v[128:131], v[104:111], a[160:167], v[128:131]\n\t" \
  "ds_read_b64_tr_b8 v[120:121], v90 offset:8192\n\t" \
  "s_add_u32 s54, s68, 0x1a1000\n\t" \
  "buffer_load_dwordx4 a[80:83], v4, s[12:15], s54 offen offset:0 sc0 sc1\n\t" \
  "v_mfma_f32_16x16x128_f8f6f4 v[132:135], v[104:111], a[168:175], v[132:135]\n\t" \
  "ds_read_b64_tr_b8 v[122:123], v90 offset:9216\n\t" \
  "v_mfma_f32_16x16x128_f8f6f4 v[136:139], v[104:111], a[176:183], v[136:139]\n\t" \
  "ds_read_b64_tr_b8 v[124:125], v98 offset:8192\n\t" \
  "s_add_u32 s55, s68, 0x1a1000\n\t" \
  "buffer_load_dwordx4 a[84:87], v4, s[12:15], s55 offen offset:1024 sc0 sc1\n\t" \
  "v_mfma_f32_16x16x128_f8f6f4 v[140:143], v[104:111], a[184:191], v[140:143]\n\t" \
  "ds_read_b64_tr_b8 v[126:127], v98 offset:9216\n\t" \
  "s_waitcnt lgkmcnt(4)\n\t" \
  "v_mfma_f32_16x16x128_f8f6f4 v[156:159], v[112:119], a[184:191], v[156:159]\n\t" \
  "ds_read_b64_tr_b8 v[104:105], v91 offset:8192\n\t" \
  "s_add_u32 s48, s68, 0x1a1000\n\t" \
  "buffer_load_dwordx4 a[88:91], v4, s[12:15], s48 offen offset:2048 sc0 sc1\n\t" \
  "v_mfma_f32_16x16x128_f8f6f4 v[152:155], v[112:119], a[176:183], v[152:155]\n\t" \
  "ds_read_b64_tr_b8 v[106:107], v91 offset:9216\n\t" \
  "v_mfma_f32_16x16x128_f8f6f4 v[148:151], v[112:119], a[168:175], v[148:151]\n\t" \
  "ds_read_b64_tr_b8 v[108:109], v99 offset:8192\n\t" \
  "s_add_u32 s49, s68, 0x1a1000\n\t" \
  "buffer_load_dwordx4 a[92:95], v4, s[12:15], s49 offen offset:3072 sc0 sc1\n\t" \
  "v_mfma_f32_16x16x128_f8f6f4 v[144:147], v[112:119], a[160:167], v[144:147]\n\t" \
  "ds_read_b64_tr_b8 v[110:111], v99 offset:9216\n\t" \
  "s_waitcnt lgkmcnt(4)\n\t" \
  "v_mfma_f32_16x16x128_f8f6f4 v[160:163], v[120:127], a[160:167], v[160:163]\n\t" \
  "ds_read_b64_tr_b8 v[112:113], v92 offset:8192\n\t" \
  "s_add_u32 m0, s24, 0xc000\n\t" \
  "s_add_u32 s50, s45, 0x600000\n\t" \
  "buffer_load_dwordx4 v2, s[8:11], s50 offen lds\n\t" \
  "v_mfma_f32_16x16x128_f8f6f4 v[164:167], v[120:127], a[168:175], v[164:167]\n\t" \
  "ds_read_b64_tr_b8 v[114:115], v92 offset:9216\n\t" \
  "v_mfma_f32_16x16x128_f8f6f4 v[168:171], v[120:127], a[176:183], v[168:171]\n\t" \
  "ds_read_b64_tr_b8 v[116:117], v100 offset:8192\n\t" \
  "s_add_u32 m0, s24, 0xc400\n\t" \
  "s_add_u32 s51, s45, 0x620000\n\t" \
  "buffer_load_dwordx4 v2, s[8:11], s51 offen lds\n\t" \
  "v_mfma_f32_16x16x128_f8f6f4 v[172:175], v[120:127], a[184:191], v[172:175]\n\t" \
  "ds_read_b64_tr_b8 v[118:119], v100 offset:9216\n\t" \
  "s_waitcnt lgkmcnt(4)\n\t" \
  "v_mfma_f32_16x16x128_f8f6f4 v[188:191], v[104:111], a[184:191], v[188:191]\n\t" \
  "ds_read_b64_tr_b8 v[120:121], v93 offset:8192\n\t" \
  "s_add_u32 m0, s24, 0xe000\n\t" \
  "s_add_u32 s52, s46, 0x600000\n\t" \
  "buffer_load_dwordx4 v2, s[8:11], s52 offen lds\n\t" \
  "v_mfma_f32_16x16x128_f8f6f4 v[184:187], v[104:111], a[176:183], v[184:187]\n\t" \
  "ds_read_b64_tr_b8 v[122:123], v93 offset:9216\n\t" \
  "v_mfma_f32_16x16x128_f8f6f4 v[180:183], v[104:111], a[168:175], v[180:183]\n\t" \
  "ds_read_b64_tr_b8 v[124:125], v101 offset:8192\n\t" \
  "s_add_u32 m0, s24, 0xe400\n\t" \
  "s_add_u32 s53, s46, 0x620000\n\t" \
  "buffer_load_dwordx4 v2, s[8:11], s53 offen lds\n\t" \
  "v_mfma_f32_16x16x128_f8f6f4 v[176:179], v[104:111], a[160:167], v[176:179]\n\t" \
  "ds_read_b64_tr_b8 v[126:127], v101 offset:9216\n\t" \
  "s_waitcnt lgkmcnt(4)\n\t" \
  "v_mfma_f32_16x16x128_f8f6f4 v[192:195], v[112:119], a[160:167], v[192:195]\n\t" \
  "ds_read_b64_tr_b8 v[104:105], v94 offset:8192\n\t" \
  "v_mfma_f32_16x16x128_f8f6f4 v[196:199], v[112:119], a[168:175], v[196:199]\n\t" \
  "ds_read_b64_tr_b8 v[106:107], v94 offset:9216\n\t" \
  "v_mfma_f32_16x16x128_f8f6f4 v[200:203], v[112:119], a[176:183], v[200:203]\n\t" \
  "ds_read_b64_tr_b8 v[108:109], v102 offset:8192\n\t" \
  "v_mfma_f32_16x16x128_f8f6f4 v[204:207], v[112:119], a[184:191], v[204:207]\n\t" \
  "ds_read_b64_tr_b8 v[110:111], v102 offset:9216\n\t" \
  "s_waitcnt lgkmcnt(4)\n\t" \
  "v_mfma_f32_16x16x128_f8f6f4 v[220:223], v[120:127], a[184:191], v[220:223]\n\t" \
  "ds_read_b64_tr_b8 v[112:113], v95 offset:8192\n\t" \
  "v_mfma_f32_16x16x128_f8f6f4 v[216:219], v[120:127], a[176:183], v[216:219]\n\t" \
  "ds_read_b64_tr_b8 v[114:115], v95 offset:9216\n\t" \
  "v_mfma_f32_16x16x128_f8f6f4 v[212:215], v[120:127], a[168:175], v[212:215]\n\t" \
  "ds_read_b64_tr_b8 v[116:117], v103 offset:8192\n\t" \
  "v_mfma_f32_16x16x128_f8f6f4 v[208:211], v[120:127], a[160:167], v[208:211]\n\t" \
  "ds_read_b64_tr_b8 v[118:119], v103 offset:9216\n\t" \
  "s_waitcnt lgkmcnt(0)\n\t" \
  "s_barrier\n\t" \
  "v_mfma_f32_16x16x128_f8f6f4 v[224:227], v[104:111], a[160:167], v[224:227]\n\t" \
  "ds_read_b64_tr_b8 v[120:121], v88 offset:16384\n\t" \
  "v_mfma_f32_16x16x128_f8f6f4 v[228:231], v[104:111], a[168:175], v[228:231]\n\t" \
  "ds_read_b64_tr_b8 v[122:123], v88 offset:17408\n\t" \
  "v_mfma_f32_16x16x128_f8f6f4 v[232:235], v[104:111], a[176:183], v[232:235]\n\t" \
  "ds_read_b64_tr_b8 v[124:125], v96 offset:16384\n\t" \
  "v_mfma_f32_16x16x128_f8f6f4 v[236:239], v[104:111], a[184:191], v[236:239]\n\t" \
  "ds_read_b64_tr_b8 v[126:127], v96 offset:17408\n\t" \
  "v_mfma_f32_16x16x128_f8f6f4 v[252:255], v[112:119], a[184:191], v[252:255]\n\t" \
  "ds_read_b64_tr_b8 v[104:105], v89 offset:16384\n\t" \
  "v_mfma_f32_16x16x128_f8f6f4 v[248:251], v[112:119], a[176:183], v[248:251]\n\t" \
  "ds_read_b64_tr_b8 v[106:107], v89 offset:17408\n\t" \
  "v_mfma_f32_16x16x128_f8f6f4 v[244:247], v[112:119], a[168:175], v[244:247]\n\t" \
  "ds_read_b64_tr_b8 v[108:109], v97 offset:16384\n\t" \
  "s_add_u32 m0, s24, 0x1e000\n\t" \
  "s_add_u32 s54, s42, 0x600000\n\t" \
  "buffer_load_dwordx4 v3, s[8:11], s54 offen lds\n\t" \
  "v_mfma_f32_16x16x128_f8f6f4 v[240:243], v[112:119], a[160:167], v[240:243]\n\t" \
  "ds_read_b64_tr_b8 v[110:111], v97 offset:17408\n\t" \
  "s_waitcnt lgkmcnt(4)\n\t" \
  "v_mfma_f32_16x16x128_f8f6f4 v[128:131], v[120:127], a[192:199], v[128:131]\n\t" \
  "ds_read_b64_tr_b8 v[112:113], v90 offset:16384\n\t" \
  "s_add_u32 m0, s24, 0x1e400\n\t" \
  "s_add_u32 s55, s42, 0x620000\n\t" \
  "buffer_load_dwordx4 v3, s[8:11], s55 offen lds\n\t" \
  "v_mfma_f32_16x16x128_f8f6f4 v[132:135], v[120:127], a[200:207], v[132:135]\n\t" \
  "ds_read_b64_tr_b8 v[114:115], v90 offset:17408\n\t" \
  "v_mfma_f32_16x16x128_f8f6f4 v[136:139], v[120:127], a[208:215], v[136:139]\n\t" \
  "ds_read_b64_tr_b8 v[116:117], v98 offset:16384\n\t" \
  "s_add_u32 m0, s24, 0x20000\n\t" \
  "s_add_u32 s48, s43, 0x600000\n\t" \
  "buffer_load_dwordx4 v3, s[8:11], s48 offen lds\n\t" \
  "v_mfma_f32_16x16x128_f8f6f4 v[140:143], v[120:127], a[216:223], v[140:143]\n\t" \
  "ds_read_b64_tr_b8 v[118:119], v98 offset:17408\n\t" \
  "s_waitcnt lgkmcnt(4)\n\t" \
  "v_mfma_f32_16x16x128_f8f6f4 v[156:159], v[104:111], a[216:223], v[156:159]\n\t" \
  "ds_read_b64_tr_b8 v[120:121], v91 offset:16384\n\t" \
  "s_add_u32 m0, s24, 0x20400\n\t" \
  "s_add_u32 s49, s43, 0x620000\n\t" \
  "buffer_load_dwordx4 v3, s[8:11], s49 offen lds\n\t" \
  "v_mfma_f32_16x16x128_f8f6f4 v[152:155], v[104:111], a[208:215], v[152:155]\n\t" \
  "ds_read_b64_tr_b8 v[122:123], v91 offset:17408\n\t" \
  "v_mfma_f32_16x16x128_f8f6f4 v[148:151], v[104:111], a[200:207], v[148:151]\n\t" \
  "ds_read_b64_tr_b8 v[124:125], v99 offset:16384\n\t" \
  "s_add_u32 m0, s24, 0x22000\n\t" \
  "s_add_u32 s50, s44, 0x600000\n\t" \
  "buffer_load_dwordx4 v3, s[8:11], s50 offen lds\n\t" \
  "v_mfma_f32_16x16x128_f8f6f4 v[144:147], v[104:111], a[192:199], v[144:147]\n\t" \
  "ds_read_b64_tr_b8 v[126:127], v99 offset:17408\n\t" \
  "s_waitcnt lgkmcnt(4)\n\t" \
  "v_mfma_f32_16x16x128_f8f6f4 v[160:163], v[112:119], a[192:199], v[160:163]\n\t" \
  "ds_read_b64_tr_b8 v[104:105], v92 offset:16384\n\t" \
  "s_add_u32 m0, s24, 0x22400\n\t" \
  "s_add_u32 s51, s44, 0x620000\n\t" \
  "buffer_load_dwordx4 v3, s[8:11], s51 offen lds\n\t" \
  "v_mfma_f32_16x16x128_f8f6f4 v[164:167], v[112:119], a[200:207], v[164:167]\n\t" \
  "ds_read_b64_tr_b8 v[106:107], v92 offset:17408\n\t" \
  "v_mfma_f32_16x16x128_f8f6f4 v[168:171], v[112:119], a[208:215], v[168:171]\n\t" \
  "ds_read_b64_tr_b8 v[108:109], v100 offset:16384\n\t" \
  "s_add_u32 s52, s68, 0x1b0000\n\t" \
  "buffer_load_dwordx4 a[96:99], v4, s[12:15], s52 offen offset:0 sc0 sc1\n\t" \
  "v_mfma_f32_16x16x128_f8f6f4 v[172:175], v[112:119], a[216:223], v[172:175]\n\t" \
  "ds_read_b64_tr_b8 v[110:111], v100 offset:17408\n\t" \
  "s_waitcnt lgkmcnt(4)\n\t" \
  "v_mfma_f32_16x16x128_f8f6f4 v[188:191], v[120:127], a[216:223], v[188:191]\n\t" \
  "ds_read_b64_tr_b8 v[112:113], v93 offset:16384\n\t" \
  "s_add_u32 s53, s68, 0x1b0000\n\t" \
  "buffer_load_dwordx4 a[100:103], v4, s[12:15], s53 offen offset:1024 sc0 sc1\n\t" \
  "v_mfma_f32_16x16x128_f8f6f4 v[184:187], v[120:127], a[208:215], v[184:187]\n\t" \
  "ds_read_b64_tr_b8 v[114:115], v93 offset:17408\n\t" \
  "v_mfma_f32_16x16x128_f8f6f4 v[180:183], v[120:127], a[200:207], v[180:183]\n\t" \
  "ds_read_b64_tr_b8 v[116:117], v101 offset:16384\n\t" \
  "s_add_u32 s54, s68, 0x1b0000\n\t" \
  "buffer_load_dwordx4 a[104:107], v4, s[12:15], s54 offen offset:2048 sc0 sc1\n\t" \
  "v_mfma_f32_16x16x128_f8f6f4 v[176:179], v[120:127], a[192:199], v[176:179]\n\t" \
  "ds_read_b64_tr_b8 v[118:119], v101 offset:17408\n\t" \
  "s_waitcnt lgkmcnt(4)\n\t" \
  "v_mfma_f32_16x16x128_f8f6f4 v[192:195], v[104:111], a[192:199], v[192:195]\n\t" \
  "ds_read_b64_tr_b8 v[120:121], v94 offset:16384\n\t" \
  "s_add_u32 s55, s68, 0x1b0000\n\t" \
  "buffer_load_dwordx4 a[108:111], v4, s[12:15], s55 offen offset:3072 sc0 sc1\n\t" \
  "v_mfma_f32_16x16x128_f8f6f4 v[196:199], v[104:111], a[200:207], v[196:199]\n\t" \
  "ds_read_b64_tr_b8 v[122:123], v94 offset:17408\n\t" \
  "v_mfma_f32_16x16x128_f8f6f4 v[200:203], v[104:111], a[208:215], v[200:203]\n\t" \
  "ds_read_b64_tr_b8 v[124:125], v102 offset:16384\n\t" \
  "s_add_u32 s48, s68, 0x1b1000\n\t" \
  "buffer_load_dwordx4 a[112:115], v4, s[12:15], s48 offen offset:0 sc0 sc1\n\t" \
  "v_mfma_f32_16x16x128_f8f6f4 v[204:207], v[104:111], a[216:223], v[204:207]\n\t" \
  "ds_read_b64_tr_b8 v[126:127], v102 offset:17408\n\t" \
  "s_waitcnt lgkmcnt(4)\n\t" \
  "v_mfma_f32_16x16x128_f8f6f4 v[220:223], v[112:119], a[216:223], v[220:223]\n\t" \
  "ds_read_b64_tr_b8 v[104:105], v95 offset:16384\n\t" \
  "s_add_u32 s49, s68, 0x1b1000\n\t" \
  "buffer_load_dwordx4 a[116:119], v4, s[12:15], s49 offen offset:1024 sc0 sc1\n\t" \
  "v_mfma_f32_16x16x128_f8f6f4 v[216:219], v[112:119], a[208:215], v[216:219]\n\t" \
  "ds_read_b64_tr_b8 v[106:107], v95 offset:17408\n\t" \
  "v_mfma_f32_16x16x128_f8f6f4 v[212:215], v[112:119], a[200:207], v[212:215]\n\t" \
  "ds_read_b64_tr_b8 v[108:109], v103 offset:16384\n\t" \
  "s_add_u32 s50, s68, 0x1b1000\n\t" \
  "buffer_load_dwordx4 a[120:123], v4, s[12:15], s50 offen offset:2048 sc0 sc1\n\t" \
  "v_mfma_f32_16x16x128_f8f6f4 v[208:211], v[112:119], a[192:199], v[208:211]\n\t" \
  "ds_read_b64_tr_b8 v[110:111], v103 offset:17408\n\t" \
  "s_waitcnt lgkmcnt(0)\n\t" \
  "s_barrier\n\t" \
  "v_mfma_f32_16x16x128_f8f6f4 v[224:227], v[120:127], a[192:199], v[224:227]\n\t" \
  "ds_read_b64_tr_b8 v[112:113], v88 offset:24576\n\t" \
  "s_add_u32 s51, s68, 0x1b1000\n\t" \
  "buffer_load_dwordx4 a[124:127], v4, s[12:15], s51 offen offset:3072 sc0 sc1\n\t" \
  "v_mfma_f32_16x16x128_f8f6f4 v[228:231], v[120:127], a[200:207], v[228:231]\n\t" \
  "ds_read_b64_tr_b8 v[114:115], v88 offset:25600\n\t" \
  "v_mfma_f32_16x16x128_f8f6f4 v[232:235], v[120:127], a[208:215], v[232:235]\n\t" \
  "ds_read_b64_tr_b8 v[116:117], v96 offset:24576\n\t" \
  "s_add_u32 m0, s24, 0x24000\n\t" \
  "s_add_u32 s52, s45, 0x600000\n\t" \
  "buffer_load_dwordx4 v3, s[8:11], s52 offen lds\n\t" \
  "v_mfma_f32_16x16x128_f8f6f4 v[236:239], v[120:127], a[216:223], v[236:239]\n\t" \
  "ds_read_b64_tr_b8 v[118:119], v96 offset:25600\n\t" \
  "v_mfma_f32_16x16x128_f8f6f4 v[252:255], v[104:111], a[216:223], v[252:255]\n\t" \
  "ds_read_b64_tr_b8 v[120:121], v89 offset:24576\n\t" \
  "s_add_u32 m0, s24, 0x24400\n\t" \
  "s_add_u32 s53, s45, 0x620000\n\t" \
  "buffer_load_dwordx4 v3, s[8:11], s53 offen lds\n\t" \
  "v_mfma_f32_16x16x128_f8f6f4 v[248:251], v[104:111], a[208:215], v[248:251]\n\t" \
  "ds_read_b64_tr_b8 v[122:123], v89 offset:25600\n\t" \
  "v_mfma_f32_16x16x128_f8f6f4 v[244:247], v[104:111], a[200:207], v[244:247]\n\t" \
  "ds_read_b64_tr_b8 v[124:125], v97 offset:24576\n\t" \
  "s_add_u32 s54, s68, 0x1c0000\n\t" \
  "buffer_load_dwordx4 a[128:131], v4, s[12:15], s54 offen offset:0 sc0 sc1\n\t" \
  "v_mfma_f32_16x16x128_f8f6f4 v[240:243], v[104:111], a[192:199], v[240:243]\n\t" \
  "ds_read_b64_tr_b8 v[126:127], v97 offset:25600\n\t" \
  "s_waitcnt vmcnt(51) lgkmcnt(4)\n\t" \
  "v_mfma_f32_16x16x128_f8f6f4 v[128:131], v[112:119], a[224:231], v[128:131]\n\t" \
  "ds_read_b64_tr_b8 v[104:105], v90 offset:24576\n\t" \
  "s_add_u32 s55, s68, 0x1c0000\n\t" \
  "buffer_load_dwordx4 a[132:135], v4, s[12:15], s55 offen offset:1024 sc0 sc1\n\t" \
  "v_mfma_f32_16x16x128_f8f6f4 v[132:135], v[112:119], a[232:239], v[132:135]\n\t" \
  "ds_read_b64_tr_b8 v[106:107], v90 offset:25600\n\t" \
  "v_mfma_f32_16x16x128_f8f6f4 v[136:139], v[112:119], a[240:247], v[136:139]\n\t" \
  "ds_read_b64_tr_b8 v[108:109], v98 offset:24576\n\t" \
  "s_add_u32 s48, s68, 0x1c0000\n\t" \
  "buffer_load_dwordx4 a[136:139], v4, s[12:15], s48 offen offset:2048 sc0 sc1\n\t" \
  "v_mfma_f32_16x16x128_f8f6f4 v[140:143], v[112:119], a[248:255], v[140:143]\n\t" \
  "ds_read_b64_tr_b8 v[110:111], v98 offset:25600\n\t" \
  "s_waitcnt lgkmcnt(4)\n\t" \
  "v_mfma_f32_16x16x128_f8f6f4 v[156:159], v[120:127], a[248:255], v[156:159]\n\t" \
  "ds_read_b64_tr_b8 v[112:113], v91 offset:24576\n\t" \
  "s_add_u32 s49, s68, 0x1c0000\n\t" \
  "buffer_load_dwordx4 a[140:143], v4, s[12:15], s49 offen offset:3072 sc0 sc1\n\t" \
  "v_mfma_f32_16x16x128_f8f6f4 v[152:155], v[120:127], a[240:247], v[152:155]\n\t" \
  "ds_read_b64_tr_b8 v[114:115], v91 offset:25600\n\t" \
  "v_mfma_f32_16x16x128_f8f6f4 v[148:151], v[120:127], a[232:239], v[148:151]\n\t" \
  "ds_read_b64_tr_b8 v[116:117], v99 offset:24576\n\t" \
  "s_add_u32 s50, s68, 0x1c1000\n\t" \
  "buffer_load_dwordx4 a[144:147], v4, s[12:15], s50 offen offset:0 sc0 sc1\n\t" \
  "v_mfma_f32_16x16x128_f8f6f4 v[144:147], v[120:127], a[224:231], v[144:147]\n\t" \
  "ds_read_b64_tr_b8 v[118:119], v99 offset:25600\n\t" \
  "s_waitcnt lgkmcnt(4)\n\t" \
  "v_mfma_f32_16x16x128_f8f6f4 v[160:163], v[104:111], a[224:231], v[160:163]\n\t" \
  "ds_read_b64_tr_b8 v[120:121], v92 offset:24576\n\t" \
  "s_add_u32 s51, s68, 0x1c1000\n\t" \
  "buffer_load_dwordx4 a[148:151], v4, s[12:15], s51 offen offset:1024 sc0 sc1\n\t" \
  "v_mfma_f32_16x16x128_f8f6f4 v[164:167], v[104:111], a[232:239], v[164:167]\n\t" \
  "ds_read_b64_tr_b8 v[122:123], v92 offset:25600\n\t" \
  "v_mfma_f32_16x16x128_f8f6f4 v[168:171], v[104:111], a[240:247], v[168:171]\n\t" \
  "ds_read_b64_tr_b8 v[124:125], v100 offset:24576\n\t" \
  "s_add_u32 s52, s68, 0x1c1000\n\t" \
  "buffer_load_dwordx4 a[152:155], v4, s[12:15], s52 offen offset:2048 sc0 sc1\n\t" \
  "v_mfma_f32_16x16x128_f8f6f4 v[172:175], v[104:111], a[248:255], v[172:175]\n\t" \
  "ds_read_b64_tr_b8 v[126:127], v100 offset:25600\n\t" \
  "s_waitcnt lgkmcnt(4)\n\t" \
  "v_mfma_f32_16x16x128_f8f6f4 v[188:191], v[112:119], a[248:255], v[188:191]\n\t" \
  "ds_read_b64_tr_b8 v[104:105], v93 offset:24576\n\t" \
  "s_add_u32 s53, s68, 0x1c1000\n\t" \
  "buffer_load_dwordx4 a[156:159], v4, s[12:15], s53 offen offset:3072 sc0 sc1\n\t" \
  "v_mfma_f32_16x16x128_f8f6f4 v[184:187], v[112:119], a[240:247], v[184:187]\n\t" \
  "ds_read_b64_tr_b8 v[106:107], v93 offset:25600\n\t" \
  "v_mfma_f32_16x16x128_f8f6f4 v[180:183], v[112:119], a[232:239], v[180:183]\n\t" \
  "ds_read_b64_tr_b8 v[108:109], v101 offset:24576\n\t" \
  "s_add_u32 s54, s68, 0x1d0000\n\t" \
  "buffer_load_dwordx4 a[160:163], v4, s[12:15], s54 offen offset:0 sc0 sc1\n\t" \
  "v_mfma_f32_16x16x128_f8f6f4 v[176:179], v[112:119], a[224:231], v[176:179]\n\t" \
  "ds_read_b64_tr_b8 v[110:111], v101 offset:25600\n\t" \
  "s_waitcnt lgkmcnt(4)\n\t" \
  "v_mfma_f32_16x16x128_f8f6f4 v[192:195], v[120:127], a[224:231], v[192:195]\n\t" \
  "ds_read_b64_tr_b8 v[112:113], v94 offset:24576\n\t" \
  "s_add_u32 s55, s68, 0x1d0000\n\t" \
  "buffer_load_dwordx4 a[164:167], v4, s[12:15], s55 offen offset:1024 sc0 sc1\n\t" \
  "v_mfma_f32_16x16x128_f8f6f4 v[196:199], v[120:127], a[232:239], v[196:199]\n\t" \
  "ds_read_b64_tr_b8 v[114:115], v94 offset:25600\n\t" \
  "v_mfma_f32_16x16x128_f8f6f4 v[200:203], v[120:127], a[240:247], v[200:203]\n\t" \
  "ds_read_b64_tr_b8 v[116:117], v102 offset:24576\n\t" \
  "s_add_u32 s48, s68, 0x1d0000\n\t" \
  "buffer_load_dwordx4 a[168:171], v4, s[12:15], s48 offen offset:2048 sc0 sc1\n\t" \
  "v_mfma_f32_16x16x128_f8f6f4 v[204:207], v[120:127], a[248:255], v[204:207]\n\t" \
  "ds_read_b64_tr_b8 v[118:119], v102 offset:25600\n\t" \
  "s_waitcnt lgkmcnt(4)\n\t" \
  "v_mfma_f32_16x16x128_f8f6f4 v[220:223], v[104:111], a[248:255], v[220:223]\n\t" \
  "ds_read_b64_tr_b8 v[120:121], v95 offset:24576\n\t" \
  "s_add_u32 s49, s68, 0x1d0000\n\t" \
  "buffer_load_dwordx4 a[172:175], v4, s[12:15], s49 offen offset:3072 sc0 sc1\n\t" \
  "v_mfma_f32_16x16x128_f8f6f4 v[216:219], v[104:111], a[240:247], v[216:219]\n\t" \
  "ds_read_b64_tr_b8 v[122:123], v95 offset:25600\n\t" \
  "v_mfma_f32_16x16x128_f8f6f4 v[212:215], v[104:111], a[232:239], v[212:215]\n\t" \
  "ds_read_b64_tr_b8 v[124:125], v103 offset:24576\n\t" \
  "s_add_u32 s50, s68, 0x1d1000\n\t" \
  "s_waitcnt vmcnt(60)\n\t" \
  "buffer_load_dwordx4 a[176:179], v4, s[12:15], s50 offen offset:0 sc0 sc1\n\t" \
  "v_mfma_f32_16x16x128_f8f6f4 v[208:211], v[104:111], a[224:231], v[208:211]\n\t" \
  "ds_read_b64_tr_b8 v[126:127], v103 offset:25600\n\t" \
  "s_waitcnt lgkmcnt(0)\n\t" \
  "s_barrier\n\t" \
  "v_mfma_f32_16x16x128_f8f6f4 v[224:227], v[112:119], a[224:231], v[224:227]\n\t" \
  "ds_read_b64_tr_b8 v[104:105], v80 offset:0\n\t" \
  "s_add_u32 s51, s68, 0x1d1000\n\t" \
  "buffer_load_dwordx4 a[180:183], v4, s[12:15], s51 offen offset:1024 sc0 sc1\n\t" \
  "v_mfma_f32_16x16x128_f8f6f4 v[228:231], v[112:119], a[232:239], v[228:231]\n\t" \
  "ds_read_b64_tr_b8 v[106:107], v80 offset:1024\n\t" \
  "v_mfma_f32_16x16x128_f8f6f4 v[232:235], v[112:119], a[240:247], v[232:235]\n\t" \
  "ds_read_b64_tr_b8 v[108:109], v88 offset:32768\n\t" \
  "s_add_u32 s52, s68, 0x1d1000\n\t" \
  "s_waitcnt vmcnt(60)\n\t" \
  "buffer_load_dwordx4 a[184:187], v4, s[12:15], s52 offen offset:2048 sc0 sc1\n\t" \
  "v_mfma_f32_16x16x128_f8f6f4 v[236:239], v[112:119], a[248:255], v[236:239]\n\t" \
  "ds_read_b64_tr_b8 v[110:111], v88 offset:33792\n\t" \
  "v_mfma_f32_16x16x128_f8f6f4 v[252:255], v[120:127], a[248:255], v[252:255]\n\t" \
  "ds_read_b64_tr_b8 v[112:113], v81 offset:0\n\t" \
  "s_add_u32 s53, s68, 0x1d1000\n\t" \
  "buffer_load_dwordx4 a[188:191], v4, s[12:15], s53 offen offset:3072 sc0 sc1\n\t" \
  "v_mfma_f32_16x16x128_f8f6f4 v[248:251], v[120:127], a[240:247], v[248:251]\n\t" \
  "ds_read_b64_tr_b8 v[114:115], v81 offset:1024\n\t" \
  "v_mfma_f32_16x16x128_f8f6f4 v[244:247], v[120:127], a[232:239], v[244:247]\n\t" \
  "ds_read_b64_tr_b8 v[116:117], v89 offset:32768\n\t" \
  "s_add_u32 m0, s24, 0x26000\n\t" \
  "s_add_u32 s54, s46, 0x600000\n\t" \
  "s_waitcnt vmcnt(60)\n\t" \
  "buffer_load_dwordx4 v3, s[8:11], s54 offen lds\n\t" \
  "v_mfma_f32_16x16x128_f8f6f4 v[240:243], v[120:127], a[224:231], v[240:243]\n\t" \
  "ds_read_b64_tr_b8 v[118:119], v89 offset:33792\n\t" \
  "s_waitcnt vmcnt(57) lgkmcnt(4)\n\t" \
  "v_mfma_f32_16x16x128_f8f6f4 v[128:131], v[104:111], a[0:7], v[128:131]\n\t" \
  "ds_read_b64_tr_b8 v[120:121], v82 offset:0\n\t" \
  "s_add_u32 m0, s24, 0x26400\n\t" \
  "s_add_u32 s55, s46, 0x620000\n\t" \
  "buffer_load_dwordx4 v3, s[8:11], s55 offen lds\n\t" \
  "v_mfma_f32_16x16x128_f8f6f4 v[132:135], v[104:111], a[8:15], v[132:135]\n\t" \
  "ds_read_b64_tr_b8 v[122:123], v82 offset:1024\n\t" \
  "v_mfma_f32_16x16x128_f8f6f4 v[136:139], v[104:111], a[16:23], v[136:139]\n\t" \
  "ds_read_b64_tr_b8 v[124:125], v90 offset:32768\n\t" \
  "s_add_u32 s48, s68, 0x1e0000\n\t" \
  "buffer_load_dwordx4 a[192:195], v4, s[12:15], s48 offen offset:0 sc0 sc1\n\t" \
  "v_mfma_f32_16x16x128_f8f6f4 v[140:143], v[104:111], a[24:31], v[140:143]\n\t" \
  "ds_read_b64_tr_b8 v[126:127], v90 offset:33792\n\t" \
  "s_waitcnt lgkmcnt(4)\n\t" \
  "v_mfma_f32_16x16x128_f8f6f4 v[156:159], v[112:119], a[24:31], v[156:159]\n\t" \
  "ds_read_b64_tr_b8 v[104:105], v83 offset:0\n\t" \
  "s_add_u32 s49, s68, 0x1e0000\n\t" \
  "buffer_load_dwordx4 a[196:199], v4, s[12:15], s49 offen offset:1024 sc0 sc1\n\t" \
  "v_mfma_f32_16x16x128_f8f6f4 v[152:155], v[112:119], a[16:23], v[152:155]\n\t" \
  "ds_read_b64_tr_b8 v[106:107], v83 offset:1024\n\t" \
  "v_mfma_f32_16x16x128_f8f6f4 v[148:151], v[112:119], a[8:15], v[148:151]\n\t" \
  "ds_read_b64_tr_b8 v[108:109], v91 offset:32768\n\t" \
  "s_add_u32 s50, s68, 0x1e0000\n\t" \
  "buffer_load_dwordx4 a[200:203], v4, s[12:15], s50 offen offset:2048 sc0 sc1\n\t" \
  "v_mfma_f32_16x16x128_f8f6f4 v[144:147], v[112:119], a[0:7], v[144:147]\n\t" \
  "ds_read_b64_tr_b8 v[110:111], v91 offset:33792\n\t" \
  "s_waitcnt lgkmcnt(4)\n\t" \
  "v_mfma_f32_16x16x128_f8f6f4 v[160:163], v[120:127], a[0:7], v[160:163]\n\t" \
  "ds_read_b64_tr_b8 v[112:113], v84 offset:0\n\t" \
  "s_add_u32 s51, s68, 0x1e0000\n\t" \
  "buffer_load_dwordx4 a[204:207], v4, s[12:15], s51 offen offset:3072 sc0 sc1\n\t" \
  "v_mfma_f32_16x16x128_f8f6f4 v[164:167], v[120:127], a[8:15], v[164:167]\n\t" \
  "ds_read_b64_tr_b8 v[114:115], v84 offset:1024\n\t" \
  "v_mfma_f32_16x16x128_f8f6f4 v[168:171], v[120:127], a[16:23], v[168:171]\n\t" \
  "ds_read_b64_tr_b8 v[116:117], v92 offset:32768\n\t" \
  "s_add_u32 s52, s68, 0x1e1000\n\t" \
  "s_waitcnt vmcnt(60)\n\t" \
  "buffer_load_dwordx4 a[208:211], v4, s[12:15], s52 offen offset:0 sc0 sc1\n\t" \
  "v_mfma_f32_16x16x128_f8f6f4 v[172:175], v[120:127], a[24:31], v[172:175]\n\t" \
  "ds_read_b64_tr_b8 v[118:119], v92 offset:33792\n\t" \
  "s_waitcnt lgkmcnt(4)\n\t" \
  "v_mfma_f32_16x16x128_f8f6f4 v[188:191], v[104:111], a[24:31], v[188:191]\n\t" \
  "ds_read_b64_tr_b8 v[120:121], v85 offset:0\n\t" \
  "s_add_u32 s53, s68, 0x1e1000\n\t" \
  "buffer_load_dwordx4 a[212:215], v4, s[12:15], s53 offen offset:1024 sc0 sc1\n\t" \
  "v_mfma_f32_16x16x128_f8f6f4 v[184:187], v[104:111], a[16:23], v[184:187]\n\t" \
  "ds_read_b64_tr_b8 v[122:123], v85 offset:1024\n\t" \
  "v_mfma_f32_16x16x128_f8f6f4 v[180:183], v[104:111], a[8:15], v[180:183]\n\t" \
  "ds_read_b64_tr_b8 v[124:125], v93 offset:32768\n\t" \
  "s_add_u32 s54, s68, 0x1e1000\n\t" \
  "s_waitcnt vmcnt(60)\n\t" \
  "buffer_load_dwordx4 a[216:219], v4, s[12:15], s54 offen offset:2048 sc0 sc1\n\t" \
  "v_mfma_f32_16x16x128_f8f6f4 v[176:179], v[104:111], a[0:7], v[176:179]\n\t" \
  "ds_read_b64_tr_b8 v[126:127], v93 offset:33792\n\t" \
  "s_waitcnt lgkmcnt(4)\n\t" \
  "v_mfma_f32_16x16x128_f8f6f4 v[192:195], v[112:119], a[0:7], v[192:195]\n\t" \
  "ds_read_b64_tr_b8 v[104:105], v86 offset:0\n\t" \
  "s_add_u32 s55, s68, 0x1e1000\n\t" \
  "buffer_load_dwordx4 a[220:223], v4, s[12:15], s55 offen offset:3072 sc0 sc1\n\t" \
  "v_mfma_f32_16x16x128_f8f6f4 v[196:199], v[112:119], a[8:15], v[196:199]\n\t" \
  "ds_read_b64_tr_b8 v[106:107], v86 offset:1024\n\t" \
  "v_mfma_f32_16x16x128_f8f6f4 v[200:203], v[112:119], a[16:23], v[200:203]\n\t" \
  "ds_read_b64_tr_b8 v[108:109], v94 offset:32768\n\t" \
  "s_add_u32 s48, s68, 0x1f0000\n\t" \
  "s_waitcnt vmcnt(60)\n\t" \
  "buffer_load_dwordx4 a[224:227], v4, s[12:15], s48 offen offset:0 sc0 sc1\n\t" \
  "v_mfma_f32_16x16x128_f8f6f4 v[204:207], v[112:119], a[24:31], v[204:207]\n\t" \
  "ds_read_b64_tr_b8 v[110:111], v94 offset:33792\n\t" \
  "s_waitcnt lgkmcnt(4)\n\t" \
  "v_mfma_f32_16x16x128_f8f6f4 v[220:223], v[120:127], a[24:31], v[220:223]\n\t" \
  "ds_read_b64_tr_b8 v[112:113], v87 offset:0\n\t" \
  "s_add_u32 s49, s68, 0x1f0000\n\t" \
  "buffer_load_dwordx4 a[228:231], v4, s[12:15], s49 offen offset:1024 sc0 sc1\n\t" \
  "v_mfma_f32_16x16x128_f8f6f4 v[216:219], v[120:127], a[16:23], v[216:219]\n\t" \
  "ds_read_b64_tr_b8 v[114:115], v87 offset:1024\n\t" \
  "v_mfma_f32_16x16x128_f8f6f4 v[212:215], v[120:127], a[8:15], v[212:215]\n\t" \
  "ds_read_b64_tr_b8 v[116:117], v95 offset:32768\n\t" \
  "s_add_u32 s50, s68, 0x1f0000\n\t" \
  "s_waitcnt vmcnt(60)\n\t" \
  "buffer_load_dwordx4 a[232:235], v4, s[12:15], s50 offen offset:2048 sc0 sc1\n\t" \
  "v_mfma_f32_16x16x128_f8f6f4 v[208:211], v[120:127], a[0:7], v[208:211]\n\t" \
  "ds_read_b64_tr_b8 v[118:119], v95 offset:33792\n\t" \
  "s_waitcnt lgkmcnt(0)\n\t" \
  "s_barrier\n\t" \
  "v_mfma_f32_16x16x128_f8f6f4 v[224:227], v[104:111], a[0:7], v[224:227]\n\t" \
  "ds_read_b64_tr_b8 v[120:121], v80 offset:8192\n\t" \
  "s_add_u32 s51, s68, 0x1f0000\n\t" \
  "buffer_load_dwordx4 a[236:239], v4, s[12:15], s51 offen offset:3072 sc0 sc1\n\t" \
  "v_mfma_f32_16x16x128_f8f6f4 v[228:231], v[104:111], a[8:15], v[228:231]\n\t" \
  "ds_read_b64_tr_b8 v[122:123], v80 offset:9216\n\t" \
  "v_mfma_f32_16x16x128_f8f6f4 v[232:235], v[104:111], a[16:23], v[232:235]\n\t" \
  "ds_read_b64_tr_b8 v[124:125], v88 offset:40960\n\t" \
  "s_add_u32 s52, s68, 0x1f1000\n\t" \
  "s_waitcnt vmcnt(60)\n\t" \
  "buffer_load_dwordx4 a[240:243], v4, s[12:15], s52 offen offset:0 sc0 sc1\n\t" \
  "v_mfma_f32_16x16x128_f8f6f4 v[236:239], v[104:111], a[24:31], v[236:239]\n\t" \
  "ds_read_b64_tr_b8 v[126:127], v88 offset:41984\n\t" \
  "v_mfma_f32_16x16x128_f8f6f4 v[252:255], v[112:119], a[24:31], v[252:255]\n\t" \
  "ds_read_b64_tr_b8 v[104:105], v81 offset:8192\n\t" \
  "s_add_u32 s53, s68, 0x1f1000\n\t" \
  "buffer_load_dwordx4 a[244:247], v4, s[12:15], s53 offen offset:1024 sc0 sc1\n\t" \
  "v_mfma_f32_16x16x128_f8f6f4 v[248:251], v[112:119], a[16:23], v[248:251]\n\t" \
  "ds_read_b64_tr_b8 v[106:107], v81 offset:9216\n\t" \
  "v_mfma_f32_16x16x128_f8f6f4 v[244:247], v[112:119], a[8:15], v[244:247]\n\t" \
  "ds_read_b64_tr_b8 v[108:109], v89 offset:40960\n\t" \
  "s_add_u32 s54, s68, 0x1f1000\n\t" \
  "s_waitcnt vmcnt(60)\n\t" \
  "buffer_load_dwordx4 a[248:251], v4, s[12:15], s54 offen offset:2048 sc0 sc1\n\t" \
  "v_mfma_f32_16x16x128_f8f6f4 v[240:243], v[112:119], a[0:7], v[240:243]\n\t" \
  "ds_read_b64_tr_b8 v[110:111], v89 offset:41984\n\t" \
  "s_waitcnt lgkmcnt(4)\n\t" \
  "v_mfma_f32_16x16x128_f8f6f4 v[128:131], v[120:127], a[32:39], v[128:131]\n\t" \
  "ds_read_b64_tr_b8 v[112:113], v82 offset:8192\n\t" \
  "s_add_u32 s55, s68, 0x1f1000\n\t" \
  "buffer_load_dwordx4 a[252:255], v4, s[12:15], s55 offen offset:3072 sc0 sc1\n\t" \
  "v_mfma_f32_16x16x128_f8f6f4 v[132:135], v[120:127], a[40:47], v[132:135]\n\t" \
  "ds_read_b64_tr_b8 v[114:115], v82 offset:9216\n\t" \
  "v_mfma_f32_16x16x128_f8f6f4 v[136:139], v[120:127], a[48:55], v[136:139]\n\t" \
  "ds_read_b64_tr_b8 v[116:117], v90 offset:40960\n\t" \
  "v_mfma_f32_16x16x128_f8f6f4 v[140:143], v[120:127], a[56:63], v[140:143]\n\t" \
  "ds_read_b64_tr_b8 v[118:119], v90 offset:41984\n\t" \
  "s_waitcnt lgkmcnt(4)\n\t" \
  "v_mfma_f32_16x16x128_f8f6f4 v[156:159], v[104:111], a[56:63], v[156:159]\n\t" \
  "ds_read_b64_tr_b8 v[120:121], v83 offset:8192\n\t" \
  "v_mfma_f32_16x16x128_f8f6f4 v[152:155], v[104:111], a[48:55], v[152:155]\n\t" \
  "ds_read_b64_tr_b8 v[122:123], v83 offset:9216\n\t" \
  "v_mfma_f32_16x16x128_f8f6f4 v[148:151], v[104:111], a[40:47], v[148:151]\n\t" \
  "ds_read_b64_tr_b8 v[124:125], v91 offset:40960\n\t" \
  "v_mfma_f32_16x16x128_f8f6f4 v[144:147], v[104:111], a[32:39], v[144:147]\n\t" \
  "ds_read_b64_tr_b8 v[126:127], v91 offset:41984\n\t" \
  "s_waitcnt lgkmcnt(4)\n\t" \
  "v_mfma_f32_16x16x128_f8f6f4 v[160:163], v[112:119], a[32:39], v[160:163]\n\t" \
  "ds_read_b64_tr_b8 v[104:105], v84 offset:8192\n\t" \
  "v_mfma_f32_16x16x128_f8f6f4 v[164:167], v[112:119], a[40:47], v[164:167]\n\t" \
  "ds_read_b64_tr_b8 v[106:107], v84 offset:9216\n\t" \
  "v_mfma_f32_16x16x128_f8f6f4 v[168:171], v[112:119], a[48:55], v[168:171]\n\t" \
  "ds_read_b64_tr_b8 v[108:109], v92 offset:40960\n\t" \
  "v_mfma_f32_16x16x128_f8f6f4 v[172:175], v[112:119], a[56:63], v[172:175]\n\t" \
  "ds_read_b64_tr_b8 v[110:111], v92 offset:41984\n\t" \
  "s_waitcnt lgkmcnt(4)\n\t" \
  "v_mfma_f32_16x16x128_f8f6f4 v[188:191], v[120:127], a[56:63], v[188:191]\n\t" \
  "ds_read_b64_tr_b8 v[112:113], v85 offset:8192\n\t" \
  "v_mfma_f32_16x16x128_f8f6f4 v[184:187], v[120:127], a[48:55], v[184:187]\n\t" \
  "ds_read_b64_tr_b8 v[114:115], v85 offset:9216\n\t" \
  "v_mfma_f32_16x16x128_f8f6f4 v[180:183], v[120:127], a[40:47], v[180:183]\n\t" \
  "ds_read_b64_tr_b8 v[116:117], v93 offset:40960\n\t" \
  "v_mfma_f32_16x16x128_f8f6f4 v[176:179], v[120:127], a[32:39], v[176:179]\n\t" \
  "ds_read_b64_tr_b8 v[118:119], v93 offset:41984\n\t" \
  "s_waitcnt lgkmcnt(4)\n\t" \
  "v_mfma_f32_16x16x128_f8f6f4 v[192:195], v[104:111], a[32:39], v[192:195]\n\t" \
  "ds_read_b64_tr_b8 v[120:121], v86 offset:8192\n\t" \
  "v_mfma_f32_16x16x128_f8f6f4 v[196:199], v[104:111], a[40:47], v[196:199]\n\t" \
  "ds_read_b64_tr_b8 v[122:123], v86 offset:9216\n\t" \
  "v_mfma_f32_16x16x128_f8f6f4 v[200:203], v[104:111], a[48:55], v[200:203]\n\t" \
  "ds_read_b64_tr_b8 v[124:125], v94 offset:40960\n\t" \
  "v_mfma_f32_16x16x128_f8f6f4 v[204:207], v[104:111], a[56:63], v[204:207]\n\t" \
  "ds_read_b64_tr_b8 v[126:127], v94 offset:41984\n\t" \
  "s_waitcnt lgkmcnt(4)\n\t" \
  "v_mfma_f32_16x16x128_f8f6f4 v[220:223], v[112:119], a[56:63], v[220:223]\n\t" \
  "ds_read_b64_tr_b8 v[104:105], v87 offset:8192\n\t" \
  "v_mfma_f32_16x16x128_f8f6f4 v[216:219], v[112:119], a[48:55], v[216:219]\n\t" \
  "ds_read_b64_tr_b8 v[106:107], v87 offset:9216\n\t" \
  "v_mfma_f32_16x16x128_f8f6f4 v[212:215], v[112:119], a[40:47], v[212:215]\n\t" \
  "ds_read_b64_tr_b8 v[108:109], v95 offset:40960\n\t" \
  "v_mfma_f32_16x16x128_f8f6f4 v[208:211], v[112:119], a[32:39], v[208:211]\n\t" \
  "ds_read_b64_tr_b8 v[110:111], v95 offset:41984\n\t" \
  "s_waitcnt lgkmcnt(0)\n\t" \
  "s_barrier\n\t" \
  "v_mfma_f32_16x16x128_f8f6f4 v[224:227], v[120:127], a[32:39], v[224:227]\n\t" \
  "ds_read_b64_tr_b8 v[112:113], v80 offset:16384\n\t" \
  "v_mfma_f32_16x16x128_f8f6f4 v[228:231], v[120:127], a[40:47], v[228:231]\n\t" \
  "ds_read_b64_tr_b8 v[114:115], v80 offset:17408\n\t" \
  "v_mfma_f32_16x16x128_f8f6f4 v[232:235], v[120:127], a[48:55], v[232:235]\n\t" \
  "ds_read_b64_tr_b8 v[116:117], v88 offset:49152\n\t" \
  "v_mfma_f32_16x16x128_f8f6f4 v[236:239], v[120:127], a[56:63], v[236:239]\n\t" \
  "ds_read_b64_tr_b8 v[118:119], v88 offset:50176\n\t" \
  "v_mfma_f32_16x16x128_f8f6f4 v[252:255], v[104:111], a[56:63], v[252:255]\n\t" \
  "ds_read_b64_tr_b8 v[120:121], v81 offset:16384\n\t" \
  "v_mfma_f32_16x16x128_f8f6f4 v[248:251], v[104:111], a[48:55], v[248:251]\n\t" \
  "ds_read_b64_tr_b8 v[122:123], v81 offset:17408\n\t" \
  "v_mfma_f32_16x16x128_f8f6f4 v[244:247], v[104:111], a[40:47], v[244:247]\n\t" \
  "ds_read_b64_tr_b8 v[124:125], v89 offset:49152\n\t" \
  "v_mfma_f32_16x16x128_f8f6f4 v[240:243], v[104:111], a[32:39], v[240:243]\n\t" \
  "ds_read_b64_tr_b8 v[126:127], v89 offset:50176\n\t" \
  "s_waitcnt vmcnt(54) lgkmcnt(4)\n\t" \
  "v_mfma_f32_16x16x128_f8f6f4 v[128:131], v[112:119], a[64:71], v[128:131]\n\t" \
  "ds_read_b64_tr_b8 v[104:105], v82 offset:16384\n\t" \
  "v_mfma_f32_16x16x128_f8f6f4 v[132:135], v[112:119], a[72:79], v[132:135]\n\t" \
  "ds_read_b64_tr_b8 v[106:107], v82 offset:17408\n\t" \
  "v_mfma_f32_16x16x128_f8f6f4 v[136:139], v[112:119], a[80:87], v[136:139]\n\t" \
  "ds_read_b64_tr_b8 v[108:109], v90 offset:49152\n\t" \
  "v_mfma_f32_16x16x128_f8f6f4 v[140:143], v[112:119], a[88:95], v[140:143]\n\t" \
  "ds_read_b64_tr_b8 v[110:111], v90 offset:50176\n\t" \
  "s_waitcnt lgkmcnt(4)\n\t" \
  "v_mfma_f32_16x16x128_f8f6f4 v[156:159], v[120:127], a[88:95], v[156:159]\n\t" \
  "ds_read_b64_tr_b8 v[112:113], v83 offset:16384\n\t" \
  "v_mfma_f32_16x16x128_f8f6f4 v[152:155], v[120:127], a[80:87], v[152:155]\n\t" \
  "ds_read_b64_tr_b8 v[114:115], v83 offset:17408\n\t" \
  "v_mfma_f32_16x16x128_f8f6f4 v[148:151], v[120:127], a[72:79], v[148:151]\n\t" \
  "ds_read_b64_tr_b8 v[116:117], v91 offset:49152\n\t" \
  "v_mfma_f32_16x16x128_f8f6f4 v[144:147], v[120:127], a[64:71], v[144:147]\n\t" \
  "ds_read_b64_tr_b8 v[118:119], v91 offset:50176\n\t" \
  "s_waitcnt lgkmcnt(4)\n\t" \
  "v_mfma_f32_16x16x128_f8f6f4 v[160:163], v[104:111], a[64:71], v[160:163]\n\t" \
  "ds_read_b64_tr_b8 v[120:121], v84 offset:16384\n\t" \
  "v_mfma_f32_16x16x128_f8f6f4 v[164:167], v[104:111], a[72:79], v[164:167]\n\t" \
  "ds_read_b64_tr_b8 v[122:123], v84 offset:17408\n\t" \
  "v_mfma_f32_16x16x128_f8f6f4 v[168:171], v[104:111], a[80:87], v[168:171]\n\t" \
  "ds_read_b64_tr_b8 v[124:125], v92 offset:49152\n\t" \
  "v_mfma_f32_16x16x128_f8f6f4 v[172:175], v[104:111], a[88:95], v[172:175]\n\t" \
  "ds_read_b64_tr_b8 v[126:127], v92 offset:50176\n\t" \
  "s_waitcnt lgkmcnt(4)\n\t" \
  "v_mfma_f32_16x16x128_f8f6f4 v[188:191], v[112:119], a[88:95], v[188:191]\n\t" \
  "ds_read_b64_tr_b8 v[104:105], v85 offset:16384\n\t" \
  "v_mfma_f32_16x16x128_f8f6f4 v[184:187], v[112:119], a[80:87], v[184:187]\n\t" \
  "ds_read_b64_tr_b8 v[106:107], v85 offset:17408\n\t" \
  "v_mfma_f32_16x16x128_f8f6f4 v[180:183], v[112:119], a[72:79], v[180:183]\n\t" \
  "ds_read_b64_tr_b8 v[108:109], v93 offset:49152\n\t" \
  "v_mfma_f32_16x16x128_f8f6f4 v[176:179], v[112:119], a[64:71], v[176:179]\n\t" \
  "ds_read_b64_tr_b8 v[110:111], v93 offset:50176\n\t" \
  "s_waitcnt lgkmcnt(4)\n\t" \
  "v_mfma_f32_16x16x128_f8f6f4 v[192:195], v[120:127], a[64:71], v[192:195]\n\t" \
  "ds_read_b64_tr_b8 v[112:113], v86 offset:16384\n\t" \
  "v_mfma_f32_16x16x128_f8f6f4 v[196:199], v[120:127], a[72:79], v[196:199]\n\t" \
  "ds_read_b64_tr_b8 v[114:115], v86 offset:17408\n\t" \
  "v_mfma_f32_16x16x128_f8f6f4 v[200:203], v[120:127], a[80:87], v[200:203]\n\t" \
  "ds_read_b64_tr_b8 v[116:117], v94 offset:49152\n\t" \
  "v_mfma_f32_16x16x128_f8f6f4 v[204:207], v[120:127], a[88:95], v[204:207]\n\t" \
  "ds_read_b64_tr_b8 v[118:119], v94 offset:50176\n\t" \
  "s_waitcnt lgkmcnt(4)\n\t" \
  "v_mfma_f32_16x16x128_f8f6f4 v[220:223], v[104:111], a[88:95], v[220:223]\n\t" \
  "ds_read_b64_tr_b8 v[120:121], v87 offset:16384\n\t" \
  "v_mfma_f32_16x16x128_f8f6f4 v[216:219], v[104:111], a[80:87], v[216:219]\n\t" \
  "ds_read_b64_tr_b8 v[122:123], v87 offset:17408\n\t" \
  "v_mfma_f32_16x16x128_f8f6f4 v[212:215], v[104:111], a[72:79], v[212:215]\n\t" \
  "ds_read_b64_tr_b8 v[124:125], v95 offset:49152\n\t" \
  "v_mfma_f32_16x16x128_f8f6f4 v[208:211], v[104:111], a[64:71], v[208:211]\n\t" \
  "ds_read_b64_tr_b8 v[126:127], v95 offset:50176\n\t" \
  "s_waitcnt vmcnt(16) lgkmcnt(0)\n\t" \
  "s_barrier\n\t" \
  "v_mfma_f32_16x16x128_f8f6f4 v[224:227], v[112:119], a[64:71], v[224:227]\n\t" \
  "ds_read_b64_tr_b8 v[104:105], v80 offset:24576\n\t" \
  "v_mov_b32 v66, s66\n\t" \
  "v_mul_f32 v66, s67, v66\n\t" \
  "v_or_b32 v66, s63, v66\n\t" \
  "v_mov_b32 v67, v66\n\t" \
  "v_mfma_f32_16x16x128_f8f6f4 v[228:231], v[112:119], a[72:79], v[228:231]\n\t" \
  "ds_read_b64_tr_b8 v[106:107], v80 offset:25600\n\t" \
  "v_mfma_f32_16x16x128_f8f6f4 v[232:235], v[112:119], a[80:87], v[232:235]\n\t" \
  "ds_read_b64_tr_b8 v[108:109], v88 offset:57344\n\t" \
  "v_mfma_f32_16x16x128_f8f6f4 v[236:239], v[112:119], a[88:95], v[236:239]\n\t" \
  "ds_read_b64_tr_b8 v[110:111], v88 offset:58368\n\t" \
  "v_mfma_f32_16x16x128_f8f6f4 v[252:255], v[120:127], a[88:95], v[252:255]\n\t" \
  "ds_read_b64_tr_b8 v[112:113], v80 offset:32768\n\t" \
  "v_mfma_f32_16x16x128_f8f6f4 v[248:251], v[120:127], a[80:87], v[248:251]\n\t" \
  "ds_read_b64_tr_b8 v[114:115], v80 offset:33792\n\t" \
  "v_mfma_f32_16x16x128_f8f6f4 v[244:247], v[120:127], a[72:79], v[244:247]\n\t" \
  "ds_read_b64_tr_b8 v[116:117], v96 offset:0\n\t" \
  "v_mfma_f32_16x16x128_f8f6f4 v[240:243], v[120:127], a[64:71], v[240:243]\n\t" \
  "ds_read_b64_tr_b8 v[118:119], v96 offset:1024\n\t" \
  "s_waitcnt lgkmcnt(4)\n\t" \
  "v_mfma_f32_16x16x128_f8f6f4 v[128:131], v[104:111], a[96:103], v[128:131]\n\t" \
  "ds_read_b64_tr_b8 v[120:121], v80 offset:40960\n\t" \
  "v_mfma_f32_16x16x128_f8f6f4 v[132:135], v[104:111], a[104:111], v[132:135]\n\t" \
  "ds_read_b64_tr_b8 v[122:123], v80 offset:41984\n\t" \
  "v_mfma_f32_16x16x128_f8f6f4 v[136:139], v[104:111], a[112:119], v[136:139]\n\t" \
  "ds_read_b64_tr_b8 v[124:125], v96 offset:8192\n\t" \
  "v_mfma_f32_16x16x128_f8f6f4 v[140:143], v[104:111], a[120:127], v[140:143]\n\t" \
  "ds_read_b64_tr_b8 v[126:127], v96 offset:9216\n\t" \
  "s_waitcnt lgkmcnt(4)\n\t" \
  "v_mfma_f32_16x16x128_f8f6f4 v[140:143], v[112:119], a[152:159], v[140:143]\n\t" \
  "ds_read_b64_tr_b8 v[104:105], v80 offset:49152\n\t" \
  "v_mfma_f32_16x16x128_f8f6f4 v[136:139], v[112:119], a[144:151], v[136:139]\n\t" \
  "ds_read_b64_tr_b8 v[106:107], v80 offset:50176\n\t" \
  "v_mfma_f32_16x16x128_f8f6f4 v[132:135], v[112:119], a[136:143], v[132:135]\n\t" \
  "ds_read_b64_tr_b8 v[108:109], v96 offset:16384\n\t" \
  "v_mfma_f32_16x16x128_f8f6f4 v[128:131], v[112:119], a[128:135], v[128:131]\n\t" \
  "ds_read_b64_tr_b8 v[110:111], v96 offset:17408\n\t" \
  "s_waitcnt lgkmcnt(4)\n\t" \
  "v_mfma_f32_16x16x128_f8f6f4 v[128:131], v[120:127], a[160:167], v[128:131]\n\t" \
  "ds_read_b64_tr_b8 v[112:113], v80 offset:57344\n\t" \
  "v_mfma_f32_16x16x128_f8f6f4 v[132:135], v[120:127], a[168:175], v[132:135]\n\t" \
  "ds_read_b64_tr_b8 v[114:115], v80 offset:58368\n\t" \
  "v_mfma_f32_16x16x128_f8f6f4 v[136:139], v[120:127], a[176:183], v[136:139]\n\t" \
  "ds_read_b64_tr_b8 v[116:117], v96 offset:24576\n\t" \
  "v_mfma_f32_16x16x128_f8f6f4 v[140:143], v[120:127], a[184:191], v[140:143]\n\t" \
  "ds_read_b64_tr_b8 v[118:119], v96 offset:25600\n\t" \
  "s_waitcnt vmcnt(8) lgkmcnt(4)\n\t" \
  "v_mfma_f32_16x16x128_f8f6f4 v[140:143], v[104:111], a[216:223], v[140:143]\n\t" \
  "ds_read_b64_tr_b8 v[120:121], v81 offset:24576\n\t" \
  "v_mfma_f32_16x16x128_f8f6f4 v[136:139], v[104:111], a[208:215], v[136:139]\n\t" \
  "ds_read_b64_tr_b8 v[122:123], v81 offset:25600\n\t" \
  "v_mfma_f32_16x16x128_f8f6f4 v[132:135], v[104:111], a[200:207], v[132:135]\n\t" \
  "ds_read_b64_tr_b8 v[124:125], v89 offset:57344\n\t" \
  "v_mfma_f32_16x16x128_f8f6f4 v[128:131], v[104:111], a[192:199], v[128:131]\n\t" \
  "ds_read_b64_tr_b8 v[126:127], v89 offset:58368\n\t" \
  "s_waitcnt vmcnt(0) lgkmcnt(4)\n\t" \
  "v_mfma_f32_16x16x128_f8f6f4 v[128:131], v[112:119], a[224:231], v[128:131]\n\t" \
  "ds_read_b64_tr_b8 v[104:105], v81 offset:32768\n\t" \
  "v_mfma_f32_16x16x128_f8f6f4 v[132:135], v[112:119], a[232:239], v[132:135]\n\t" \
  "ds_read_b64_tr_b8 v[106:107], v81 offset:33792\n\t" \
  "v_mfma_f32_16x16x128_f8f6f4 v[136:139], v[112:119], a[240:247], v[136:139]\n\t" \
  "ds_read_b64_tr_b8 v[108:109], v97 offset:0\n\t" \
  "v_mfma_f32_16x16x128_f8f6f4 v[140:143], v[112:119], a[248:255], v[140:143]\n\t" \
  "ds_read_b64_tr_b8 v[110:111], v97 offset:1024\n\t" \
  "s_waitcnt lgkmcnt(4)\n\t" \
  "v_mfma_f32_16x16x128_f8f6f4 v[156:159], v[120:127], a[120:127], v[156:159]\n\t" \
  "ds_read_b64_tr_b8 v[112:113], v81 offset:40960\n\t" \
  "v_mfma_f32_16x16x128_f8f6f4 v[152:155], v[120:127], a[112:119], v[152:155]\n\t" \
  "ds_read_b64_tr_b8 v[114:115], v81 offset:41984\n\t" \
  "v_mfma_f32_16x16x128_f8f6f4 v[148:151], v[120:127], a[104:111], v[148:151]\n\t" \
  "ds_read_b64_tr_b8 v[116:117], v97 offset:8192\n\t" \
  "v_mfma_f32_16x16x128_f8f6f4 v[144:147], v[120:127], a[96:103], v[144:147]\n\t" \
  "ds_read_b64_tr_b8 v[118:119], v97 offset:9216\n\t" \
  "s_waitcnt lgkmcnt(4)\n\t" \
  "v_mfma_f32_16x16x128_f8f6f4 v[144:147], v[104:111], a[128:135], v[144:147]\n\t" \
  "ds_read_b64_tr_b8 v[120:121], v81 offset:49152\n\t" \
  "v_pk_mul_f32 v[128:129], v[128:129], v[66:67]\n\t" \
  "v_pk_mul_f32 v[130:131], v[130:131], v[66:67]\n\t" \
  "s_mov_b32 s48, 0x0\n\t" \
  "buffer_store_dwordx4 v[128:131], v13, s[16:19], s48 offen offset:0 sc1\n\t" \
  "v_mfma_f32_16x16x128_f8f6f4 v[148:151], v[104:111], a[136:143], v[148:151]\n\t" \
  "ds_read_b64_tr_b8 v[122:123], v81 offset:50176\n\t" \
  "v_mfma_f32_16x16x128_f8f6f4 v[152:155], v[104:111], a[144:151], v[152:155]\n\t" \
  "ds_read_b64_tr_b8 v[124:125], v97 offset:16384\n\t" \
  "v_mfma_f32_16x16x128_f8f6f4 v[156:159], v[104:111], a[152:159], v[156:159]\n\t" \
  "ds_read_b64_tr_b8 v[126:127], v97 offset:17408\n\t" \
  "s_waitcnt lgkmcnt(4)\n\t" \
  "v_mfma_f32_16x16x128_f8f6f4 v[156:159], v[112:119], a[184:191], v[156:159]\n\t" \
  "ds_read_b64_tr_b8 v[104:105], v81 offset:57344\n\t" \
  "v_pk_mul_f32 v[132:133], v[132:133], v[66:67]\n\t" \
  "v_pk_mul_f32 v[134:135], v[134:135], v[66:67]\n\t" \
  "s_mov_b32 s49, 0x100000\n\t" \
  "buffer_store_dwordx4 v[132:135], v13, s[16:19], s49 offen offset:0 sc1\n\t" \
  "v_mfma_f32_16x16x128_f8f6f4 v[152:155], v[112:119], a[176:183], v[152:155]\n\t" \
  "ds_read_b64_tr_b8 v[106:107], v81 offset:58368\n\t" \
  "v_mfma_f32_16x16x128_f8f6f4 v[148:151], v[112:119], a[168:175], v[148:151]\n\t" \
  "ds_read_b64_tr_b8 v[108:109], v97 offset:24576\n\t" \
  "v_mfma_f32_16x16x128_f8f6f4 v[144:147], v[112:119], a[160:167], v[144:147]\n\t" \
  "ds_read_b64_tr_b8 v[110:111], v97 offset:25600\n\t" \
  "s_waitcnt lgkmcnt(4)\n\t" \
  "v_mfma_f32_16x16x128_f8f6f4 v[144:147], v[120:127], a[192:199], v[144:147]\n\t" \
  "ds_read_b64_tr_b8 v[112:113], v82 offset:24576\n\t" \
  "v_pk_mul_f32 v[136:137], v[136:137], v[66:67]\n\t" \
  "v_pk_mul_f32 v[138:139], v[138:139], v[66:67]\n\t" \
  "s_mov_b32 s50, 0x200000\n\t" \
  "buffer_store_dwordx4 v[136:139], v13, s[16:19], s50 offen offset:0 sc1\n\t" \
  "v_mfma_f32_16x16x128_f8f6f4 v[148:151], v[120:127], a[200:207], v[148:151]\n\t" \
  "ds_read_b64_tr_b8 v[114:115], v82 offset:25600\n\t" \
  "v_mfma_f32_16x16x128_f8f6f4 v[152:155], v[120:127], a[208:215], v[152:155]\n\t" \
  "ds_read_b64_tr_b8 v[116:117], v90 offset:57344\n\t" \
  "v_mfma_f32_16x16x128_f8f6f4 v[156:159], v[120:127], a[216:223], v[156:159]\n\t" \
  "ds_read_b64_tr_b8 v[118:119], v90 offset:58368\n\t" \
  "s_waitcnt lgkmcnt(4)\n\t" \
  "v_mfma_f32_16x16x128_f8f6f4 v[156:159], v[104:111], a[248:255], v[156:159]\n\t" \
  "ds_read_b64_tr_b8 v[120:121], v82 offset:32768\n\t" \
  "v_pk_mul_f32 v[140:141], v[140:141], v[66:67]\n\t" \
  "v_pk_mul_f32 v[142:143], v[142:143], v[66:67]\n\t" \
  "s_mov_b32 s51, 0x300000\n\t" \
  "buffer_store_dwordx4 v[140:143], v13, s[16:19], s51 offen offset:0 sc1\n\t" \
  "v_mfma_f32_16x16x128_f8f6f4 v[152:155], v[104:111], a[240:247], v[152:155]\n\t" \
  "ds_read_b64_tr_b8 v[122:123], v82 offset:33792\n\t" \
  "v_mfma_f32_16x16x128_f8f6f4 v[148:151], v[104:111], a[232:239], v[148:151]\n\t" \
  "ds_read_b64_tr_b8 v[124:125], v98 offset:0\n\t" \
  "v_mfma_f32_16x16x128_f8f6f4 v[144:147], v[104:111], a[224:231], v[144:147]\n\t" \
  "ds_read_b64_tr_b8 v[126:127], v98 offset:1024\n\t" \
  "s_waitcnt lgkmcnt(4)\n\t" \
  "v_mfma_f32_16x16x128_f8f6f4 v[160:163], v[112:119], a[96:103], v[160:163]\n\t" \
  "ds_read_b64_tr_b8 v[104:105], v82 offset:40960\n\t" \
  "v_mfma_f32_16x16x128_f8f6f4 v[164:167], v[112:119], a[104:111], v[164:167]\n\t" \
  "ds_read_b64_tr_b8 v[106:107], v82 offset:41984\n\t" \
  "v_mfma_f32_16x16x128_f8f6f4 v[168:171], v[112:119], a[112:119], v[168:171]\n\t" \
  "ds_read_b64_tr_b8 v[108:109], v98 offset:8192\n\t" \
  "v_mfma_f32_16x16x128_f8f6f4 v[172:175], v[112:119], a[120:127], v[172:175]\n\t" \
  "ds_read_b64_tr_b8 v[110:111], v98 offset:9216\n\t" \
  "s_waitcnt lgkmcnt(4)\n\t" \
  "v_mfma_f32_16x16x128_f8f6f4 v[172:175], v[120:127], a[152:159], v[172:175]\n\t" \
  "ds_read_b64_tr_b8 v[112:113], v82 offset:49152\n\t" \
  "v_pk_mul_f32 v[144:145], v[144:145], v[66:67]\n\t" \
  "v_pk_mul_f32 v[146:147], v[146:147], v[66:67]\n\t" \
  "s_mov_b32 s52, 0x0\n\t" \
  "buffer_store_dwordx4 v[144:147], v13, s[16:19], s52 offen offset:64 sc1\n\t" \
  "v_mfma_f32_16x16x128_f8f6f4 v[168:171], v[120:127], a[144:151], v[168:171]\n\t" \
  "ds_read_b64_tr_b8 v[114:115], v82 offset:50176\n\t" \
  "v_mfma_f32_16x16x128_f8f6f4 v[164:167], v[120:127], a[136:143], v[164:167]\n\t" \
  "ds_read_b64_tr_b8 v[116:117], v98 offset:16384\n\t" \
  "v_mfma_f32_16x16x128_f8f6f4 v[160:163], v[120:127], a[128:135], v[160:163]\n\t" \
  "ds_read_b64_tr_b8 v[118:119], v98 offset:17408\n\t" \
  "s_waitcnt lgkmcnt(4)\n\t" \
  "v_mfma_f32_16x16x128_f8f6f4 v[160:163], v[104:111], a[160:167], v[160:163]\n\t" \
  "ds_read_b64_tr_b8 v[120:121], v82 offset:57344\n\t" \
  "v_pk_mul_f32 v[148:149], v[148:149], v[66:67]\n\t" \
  "v_pk_mul_f32 v[150:151], v[150:151], v[66:67]\n\t" \
  "s_mov_b32 s53, 0x100000\n\t" \
  "buffer_store_dwordx4 v[148:151], v13, s[16:19], s53 offen offset:64 sc1\n\t" \
  "v_mfma_f32_16x16x128_f8f6f4 v[164:167], v[104:111], a[168:175], v[164:167]\n\t" \
  "ds_read_b64_tr_b8 v[122:123], v82 offset:58368\n\t" \
  "v_mfma_f32_16x16x128_f8f6f4 v[168:171], v[104:111], a[176:183], v[168:171]\n\t" \
  "ds_read_b64_tr_b8 v[124:125], v98 offset:24576\n\t" \
  "v_mfma_f32_16x16x128_f8f6f4 v[172:175], v[104:111], a[184:191], v[172:175]\n\t" \
  "ds_read_b64_tr_b8 v[126:127], v98 offset:25600\n\t" \
  "s_waitcnt lgkmcnt(4)\n\t" \
  "v_mfma_f32_16x16x128_f8f6f4 v[172:175], v[112:119], a[216:223], v[172:175]\n\t" \
  "ds_read_b64_tr_b8 v[104:105], v83 offset:24576\n\t" \
  "v_pk_mul_f32 v[152:153], v[152:153], v[66:67]\n\t" \
  "v_pk_mul_f32 v[154:155], v[154:155], v[66:67]\n\t" \
  "s_mov_b32 s54, 0x200000\n\t" \
  "buffer_store_dwordx4 v[152:155], v13, s[16:19], s54 offen offset:64 sc1\n\t" \
  "v_mfma_f32_16x16x128_f8f6f4 v[168:171], v[112:119], a[208:215], v[168:171]\n\t" \
  "ds_read_b64_tr_b8 v[106:107], v83 offset:25600\n\t" \
  "v_mfma_f32_16x16x128_f8f6f4 v[164:167], v[112:119], a[200:207], v[164:167]\n\t" \
  "ds_read_b64_tr_b8 v[108:109], v91 offset:57344\n\t" \
  "v_mfma_f32_16x16x128_f8f6f4 v[160:163], v[112:119], a[192:199], v[160:163]\n\t" \
  "ds_read_b64_tr_b8 v[110:111], v91 offset:58368\n\t" \
  "s_waitcnt lgkmcnt(4)\n\t" \
  "v_mfma_f32_16x16x128_f8f6f4 v[160:163], v[120:127], a[224:231], v[160:163]\n\t" \
  "ds_read_b64_tr_b8 v[112:113], v83 offset:32768\n\t" \
  "v_pk_mul_f32 v[156:157], v[156:157], v[66:67]\n\t" \
  "v_pk_mul_f32 v[158:159], v[158:159], v[66:67]\n\t" \
  "s_mov_b32 s55, 0x300000\n\t" \
  "buffer_store_dwordx4 v[156:159], v13, s[16:19], s55 offen offset:64 sc1\n\t" \
  "v_mfma_f32_16x16x128_f8f6f4 v[164:167], v[120:127], a[232:239], v[164:167]\n\t" \
  "ds_read_b64_tr_b8 v[114:115], v83 offset:33792\n\t" \
  "v_mfma_f32_16x16x128_f8f6f4 v[168:171], v[120:127], a[240:247], v[168:171]\n\t" \
  "ds_read_b64_tr_b8 v[116:117], v99 offset:0\n\t" \
  "v_mfma_f32_16x16x128_f8f6f4 v[172:175], v[120:127], a[248:255], v[172:175]\n\t" \
  "ds_read_b64_tr_b8 v[118:119], v99 offset:1024\n\t" \
  "s_waitcnt lgkmcnt(4)\n\t" \
  "v_mfma_f32_16x16x128_f8f6f4 v[188:191], v[104:111], a[120:127], v[188:191]\n\t" \
  "ds_read_b64_tr_b8 v[120:121], v83 offset:40960\n\t" \
  "v_mfma_f32_16x16x128_f8f6f4 v[184:187], v[104:111], a[112:119], v[184:187]\n\t" \
  "ds_read_b64_tr_b8 v[122:123], v83 offset:41984\n\t" \
  "v_mfma_f32_16x16x128_f8f6f4 v[180:183], v[104:111], a[104:111], v[180:183]\n\t" \
  "ds_read_b64_tr_b8 v[124:125], v99 offset:8192\n\t" \
  "v_mfma_f32_16x16x128_f8f6f4 v[176:179], v[104:111], a[96:103], v[176:179]\n\t" \
  "ds_read_b64_tr_b8 v[126:127], v99 offset:9216\n\t" \
  "s_waitcnt lgkmcnt(4)\n\t" \
  "v_mfma_f32_16x16x128_f8f6f4 v[176:179], v[112:119], a[128:135], v[176:179]\n\t" \
  "ds_read_b64_tr_b8 v[104:105], v83 offset:49152\n\t" \
  "v_pk_mul_f32 v[160:161], v[160:161], v[66:67]\n\t" \
  "v_pk_mul_f32 v[162:163], v[162:163], v[66:67]\n\t" \
  "s_mov_b32 s48, 0x0\n\t" \
  "buffer_store_dwordx4 v[160:163], v13, s[16:19], s48 offen offset:128 sc1\n\t" \
  "v_mfma_f32_16x16x128_f8f6f4 v[180:183], v[112:119], a[136:143], v[180:183]\n\t" \
  "ds_read_b64_tr_b8 v[106:107], v83 offset:50176\n\t" \
  "v_mfma_f32_16x16x128_f8f6f4 v[184:187], v[112:119], a[144:151], v[184:187]\n\t" \
  "ds_read_b64_tr_b8 v[108:109], v99 offset:16384\n\t" \
  "v_mfma_f32_16x16x128_f8f6f4 v[188:191], v[112:119], a[152:159], v[188:191]\n\t" \
  "ds_read_b64_tr_b8 v[110:111], v99 offset:17408\n\t" \
  "s_waitcnt lgkmcnt(4)\n\t" \
  "v_mfma_f32_16x16x128_f8f6f4 v[188:191], v[120:127], a[184:191], v[188:191]\n\t" \
  "ds_read_b64_tr_b8 v[112:113], v83 offset:57344\n\t" \
  "v_pk_mul_f32 v[164:165], v[164:165], v[66:67]\n\t" \
  "v_pk_mul_f32 v[166:167], v[166:167], v[66:67]\n\t" \
  "s_mov_b32 s49, 0x100000\n\t" \
  "buffer_store_dwordx4 v[164:167], v13, s[16:19], s49 offen offset:128 sc1\n\t" \
  "v_mfma_f32_16x16x128_f8f6f4 v[184:187], v[120:127], a[176:183], v[184:187]\n\t" \
  "ds_read_b64_tr_b8 v[114:115], v83 offset:58368\n\t" \
  "v_mfma_f32_16x16x128_f8f6f4 v[180:183], v[120:127], a[168:175], v[180:183]\n\t" \
  "ds_read_b64_tr_b8 v[116:117], v99 offset:24576\n\t" \
  "v_mfma_f32_16x16x128_f8f6f4 v[176:179], v[120:127], a[160:167], v[176:179]\n\t" \
  "ds_read_b64_tr_b8 v[118:119], v99 offset:25600\n\t" \
  "s_waitcnt lgkmcnt(4)\n\t" \
  "v_mfma_f32_16x16x128_f8f6f4 v[176:179], v[104:111], a[192:199], v[176:179]\n\t" \
  "ds_read_b64_tr_b8 v[120:121], v84 offset:24576\n\t" \
  "v_pk_mul_f32 v[168:169], v[168:169], v[66:67]\n\t" \
  "v_pk_mul_f32 v[170:171], v[170:171], v[66:67]\n\t" \
  "s_mov_b32 s50, 0x200000\n\t" \
  "buffer_store_dwordx4 v[168:171], v13, s[16:19], s50 offen offset:128 sc1\n\t" \
  "v_mfma_f32_16x16x128_f8f6f4 v[180:183], v[104:111], a[200:207], v[180:183]\n\t" \
  "ds_read_b64_tr_b8 v[122:123], v84 offset:25600\n\t" \
  "v_mfma_f32_16x16x128_f8f6f4 v[184:187], v[104:111], a[208:215], v[184:187]\n\t" \
  "ds_read_b64_tr_b8 v[124:125], v92 offset:57344\n\t" \
  "v_mfma_f32_16x16x128_f8f6f4 v[188:191], v[104:111], a[216:223], v[188:191]\n\t" \
  "ds_read_b64_tr_b8 v[126:127], v92 offset:58368\n\t" \
  "s_waitcnt lgkmcnt(4)\n\t" \
  "v_mfma_f32_16x16x128_f8f6f4 v[188:191], v[112:119], a[248:255], v[188:191]\n\t" \
  "ds_read_b64_tr_b8 v[104:105], v84 offset:32768\n\t" \
  "v_pk_mul_f32 v[172:173], v[172:173], v[66:67]\n\t" \
  "v_pk_mul_f32 v[174:175], v[174:175], v[66:67]\n\t" \
  "s_mov_b32 s51, 0x300000\n\t" \
  "buffer_store_dwordx4 v[172:175], v13, s[16:19], s51 offen offset:128 sc1\n\t" \
  "v_mfma_f32_16x16x128_f8f6f4 v[184:187], v[112:119], a[240:247], v[184:187]\n\t" \
  "ds_read_b64_tr_b8 v[106:107], v84 offset:33792\n\t" \
  "v_mfma_f32_16x16x128_f8f6f4 v[180:183], v[112:119], a[232:239], v[180:183]\n\t" \
  "ds_read_b64_tr_b8 v[108:109], v100 offset:0\n\t" \
  "v_mfma_f32_16x16x128_f8f6f4 v[176:179], v[112:119], a[224:231], v[176:179]\n\t" \
  "ds_read_b64_tr_b8 v[110:111], v100 offset:1024\n\t" \
  "s_waitcnt lgkmcnt(4)\n\t" \
  "v_mfma_f32_16x16x128_f8f6f4 v[192:195], v[120:127], a[96:103], v[192:195]\n\t" \
  "ds_read_b64_tr_b8 v[112:113], v84 offset:40960\n\t" \
  "v_mfma_f32_16x16x128_f8f6f4 v[196:199], v[120:127], a[104:111], v[196:199]\n\t" \
  "ds_read_b64_tr_b8 v[114:115], v84 offset:41984\n\t" \
  "v_mfma_f32_16x16x128_f8f6f4 v[200:203], v[120:127], a[112:119], v[200:203]\n\t" \
  "ds_read_b64_tr_b8 v[116:117], v100 offset:8192\n\t" \
  "v_mfma_f32_16x16x128_f8f6f4 v[204:207], v[120:127], a[120:127], v[204:207]\n\t" \
  "ds_read_b64_tr_b8 v[118:119], v100 offset:9216\n\t" \
  "s_waitcnt lgkmcnt(4)\n\t" \
  "v_mfma_f32_16x16x128_f8f6f4 v[204:207], v[104:111], a[152:159], v[204:207]\n\t" \
  "ds_read_b64_tr_b8 v[120:121], v84 offset:49152\n\t" \
  "v_pk_mul_f32 v[176:177], v[176:177], v[66:67]\n\t" \
  "v_pk_mul_f32 v[178:179], v[178:179], v[66:67]\n\t" \
  "s_mov_b32 s52, 0x0\n\t" \
  "buffer_store_dwordx4 v[176:179], v13, s[16:19], s52 offen offset:192 sc1\n\t" \
  "v_mfma_f32_16x16x128_f8f6f4 v[200:203], v[104:111], a[144:151], v[200:203]\n\t" \
  "ds_read_b64_tr_b8 v[122:123], v84 offset:50176\n\t" \
  "v_mfma_f32_16x16x128_f8f6f4 v[196:199], v[104:111], a[136:143], v[196:199]\n\t" \
  "ds_read_b64_tr_b8 v[124:125], v100 offset:16384\n\t" \
  "v_mfma_f32_16x16x128_f8f6f4 v[192:195], v[104:111], a[128:135], v[192:195]\n\t" \
  "ds_read_b64_tr_b8 v[126:127], v100 offset:17408\n\t" \
  "s_waitcnt lgkmcnt(4)\n\t" \
  "v_mfma_f32_16x16x128_f8f6f4 v[192:195], v[112:119], a[160:167], v[192:195]\n\t" \
  "ds_read_b64_tr_b8 v[104:105], v84 offset:57344\n\t" \
  "v_pk_mul_f32 v[180:181], v[180:181], v[66:67]\n\t" \
  "v_pk_mul_f32 v[182:183], v[182:183], v[66:67]\n\t" \
  "s_mov_b32 s53, 0x100000\n\t" \
  "buffer_store_dwordx4 v[180:183], v13, s[16:19], s53 offen offset:192 sc1\n\t" \
  "v_mfma_f32_16x16x128_f8f6f4 v[196:199], v[112:119], a[168:175], v[196:199]\n\t" \
  "ds_read_b64_tr_b8 v[106:107], v84 offset:58368\n\t" \
  "v_mfma_f32_16x16x128_f8f6f4 v[200:203], v[112:119], a[176:183], v[200:203]\n\t" \
  "ds_read_b64_tr_b8 v[108:109], v100 offset:24576\n\t" \
  "v_mfma_f32_16x16x128_f8f6f4 v[204:207], v[112:119], a[184:191], v[204:207]\n\t" \
  "ds_read_b64_tr_b8 v[110:111], v100 offset:25600\n\t" \
  "s_waitcnt lgkmcnt(4)\n\t" \
  "v_mfma_f32_16x16x128_f8f6f4 v[204:207], v[120:127], a[216:223], v[204:207]\n\t" \
  "ds_read_b64_tr_b8 v[112:113], v85 offset:24576\n\t" \
  "v_pk_mul_f32 v[184:185], v[184:185], v[66:67]\n\t" \
  "v_pk_mul_f32 v[186:187], v[186:187], v[66:67]\n\t" \
  "s_mov_b32 s54, 0x200000\n\t" \
  "buffer_store_dwordx4 v[184:187], v13, s[16:19], s54 offen offset:192 sc1\n\t" \
  "v_mfma_f32_16x16x128_f8f6f4 v[200:203], v[120:127], a[208:215], v[200:203]\n\t" \
  "ds_read_b64_tr_b8 v[114:115], v85 offset:25600\n\t" \
  "v_mfma_f32_16x16x128_f8f6f4 v[196:199], v[120:127], a[200:207], v[196:199]\n\t" \
  "ds_read_b64_tr_b8 v[116:117], v93 offset:57344\n\t" \
  "v_mfma_f32_16x16x128_f8f6f4 v[192:195], v[120:127], a[192:199], v[192:195]\n\t" \
  "ds_read_b64_tr_b8 v[118:119], v93 offset:58368\n\t" \
  "s_waitcnt lgkmcnt(4)\n\t" \
  "v_mfma_f32_16x16x128_f8f6f4 v[192:195], v[104:111], a[224:231], v[192:195]\n\t" \
  "ds_read_b64_tr_b8 v[120:121], v85 offset:32768\n\t" \
  "v_pk_mul_f32 v[188:189], v[188:189], v[66:67]\n\t" \
  "v_pk_mul_f32 v[190:191], v[190:191], v[66:67]\n\t" \
  "s_mov_b32 s55, 0x300000\n\t" \
  "buffer_store_dwordx4 v[188:191], v13, s[16:19], s55 offen offset:192 sc1\n\t" \
  "v_mfma_f32_16x16x128_f8f6f4 v[196:199], v[104:111], a[232:239], v[196:199]\n\t" \
  "ds_read_b64_tr_b8 v[122:123], v85 offset:33792\n\t" \
  "v_mfma_f32_16x16x128_f8f6f4 v[200:203], v[104:111], a[240:247], v[200:203]\n\t" \
  "ds_read_b64_tr_b8 v[124:125], v101 offset:0\n\t" \
  "v_mfma_f32_16x16x128_f8f6f4 v[204:207], v[104:111], a[248:255], v[204:207]\n\t" \
  "ds_read_b64_tr_b8 v[126:127], v101 offset:1024\n\t" \
  "s_waitcnt lgkmcnt(4)\n\t" \
  "v_mfma_f32_16x16x128_f8f6f4 v[220:223], v[112:119], a[120:127], v[220:223]\n\t" \
  "ds_read_b64_tr_b8 v[104:105], v85 offset:40960\n\t" \
  "v_mfma_f32_16x16x128_f8f6f4 v[216:219], v[112:119], a[112:119], v[216:219]\n\t" \
  "ds_read_b64_tr_b8 v[106:107], v85 offset:41984\n\t" \
  "v_mfma_f32_16x16x128_f8f6f4 v[212:215], v[112:119], a[104:111], v[212:215]\n\t" \
  "ds_read_b64_tr_b8 v[108:109], v101 offset:8192\n\t" \
  "v_mfma_f32_16x16x128_f8f6f4 v[208:211], v[112:119], a[96:103], v[208:211]\n\t" \
  "ds_read_b64_tr_b8 v[110:111], v101 offset:9216\n\t" \
  "s_waitcnt lgkmcnt(4)\n\t" \
  "v_mfma_f32_16x16x128_f8f6f4 v[208:211], v[120:127], a[128:135], v[208:211]\n\t" \
  "ds_read_b64_tr_b8 v[112:113], v85 offset:49152\n\t" \
  "v_pk_mul_f32 v[192:193], v[192:193], v[66:67]\n\t" \
  "v_pk_mul_f32 v[194:195], v[194:195], v[66:67]\n\t" \
  "s_mov_b32 s48, 0x0\n\t" \
  "buffer_store_dwordx4 v[192:195], v13, s[16:19], s48 offen offset:256 sc1\n\t" \
  "v_mfma_f32_16x16x128_f8f6f4 v[212:215], v[120:127], a[136:143], v[212:215]\n\t" \
  "ds_read_b64_tr_b8 v[114:115], v85 offset:50176\n\t" \
  "v_mfma_f32_16x16x128_f8f6f4 v[216:219], v[120:127], a[144:151], v[216:219]\n\t" \
  "ds_read_b64_tr_b8 v[116:117], v101 offset:16384\n\t" \
  "v_mfma_f32_16x16x128_f8f6f4 v[220:223], v[120:127], a[152:159], v[220:223]\n\t" \
  "ds_read_b64_tr_b8 v[118:119], v101 offset:17408\n\t" \
  "s_waitcnt lgkmcnt(4)\n\t" \
  "v_mfma_f32_16x16x128_f8f6f4 v[220:223], v[104:111], a[184:191], v[220:223]\n\t" \
  "ds_read_b64_tr_b8 v[120:121], v85 offset:57344\n\t" \
  "v_pk_mul_f32 v[196:197], v[196:197], v[66:67]\n\t" \
  "v_pk_mul_f32 v[198:199], v[198:199], v[66:67]\n\t" \
  "s_mov_b32 s49, 0x100000\n\t" \
  "buffer_store_dwordx4 v[196:199], v13, s[16:19], s49 offen offset:256 sc1\n\t" \
  "v_mfma_f32_16x16x128_f8f6f4 v[216:219], v[104:111], a[176:183], v[216:219]\n\t" \
  "ds_read_b64_tr_b8 v[122:123], v85 offset:58368\n\t" \
  "v_mfma_f32_16x16x128_f8f6f4 v[212:215], v[104:111], a[168:175], v[212:215]\n\t" \
  "ds_read_b64_tr_b8 v[124:125], v101 offset:24576\n\t" \
  "v_mfma_f32_16x16x128_f8f6f4 v[208:211], v[104:111], a[160:167], v[208:211]\n\t" \
  "ds_read_b64_tr_b8 v[126:127], v101 offset:25600\n\t" \
  "s_waitcnt lgkmcnt(4)\n\t" \
  "v_mfma_f32_16x16x128_f8f6f4 v[208:211], v[112:119], a[192:199], v[208:211]\n\t" \
  "ds_read_b64_tr_b8 v[104:105], v86 offset:24576\n\t" \
  "v_pk_mul_f32 v[200:201], v[200:201], v[66:67]\n\t" \
  "v_pk_mul_f32 v[202:203], v[202:203], v[66:67]\n\t" \
  "s_mov_b32 s50, 0x200000\n\t" \
  "buffer_store_dwordx4 v[200:203], v13, s[16:19], s50 offen offset:256 sc1\n\t" \
  "v_mfma_f32_16x16x128_f8f6f4 v[212:215], v[112:119], a[200:207], v[212:215]\n\t" \
  "ds_read_b64_tr_b8 v[106:107], v86 offset:25600\n\t" \
  "v_mfma_f32_16x16x128_f8f6f4 v[216:219], v[112:119], a[208:215], v[216:219]\n\t" \
  "ds_read_b64_tr_b8 v[108:109], v94 offset:57344\n\t" \
  "v_mfma_f32_16x16x128_f8f6f4 v[220:223], v[112:119], a[216:223], v[220:223]\n\t" \
  "ds_read_b64_tr_b8 v[110:111], v94 offset:58368\n\t" \
  "s_waitcnt lgkmcnt(4)\n\t" \
  "v_mfma_f32_16x16x128_f8f6f4 v[220:223], v[120:127], a[248:255], v[220:223]\n\t" \
  "ds_read_b64_tr_b8 v[112:113], v86 offset:32768\n\t" \
  "v_pk_mul_f32 v[204:205], v[204:205], v[66:67]\n\t" \
  "v_pk_mul_f32 v[206:207], v[206:207], v[66:67]\n\t" \
  "s_mov_b32 s51, 0x300000\n\t" \
  "buffer_store_dwordx4 v[204:207], v13, s[16:19], s51 offen offset:256 sc1\n\t" \
  "v_mfma_f32_16x16x128_f8f6f4 v[216:219], v[120:127], a[240:247], v[216:219]\n\t" \
  "ds_read_b64_tr_b8 v[114:115], v86 offset:33792\n\t" \
  "v_mfma_f32_16x16x128_f8f6f4 v[212:215], v[120:127], a[232:239], v[212:215]\n\t" \
  "ds_read_b64_tr_b8 v[116:117], v102 offset:0\n\t" \
  "v_mfma_f32_16x16x128_f8f6f4 v[208:211], v[120:127], a[224:231], v[208:211]\n\t" \
  "ds_read_b64_tr_b8 v[118:119], v102 offset:1024\n\t" \
  "s_waitcnt lgkmcnt(4)\n\t" \
  "v_mfma_f32_16x16x128_f8f6f4 v[224:227], v[104:111], a[96:103], v[224:227]\n\t" \
  "ds_read_b64_tr_b8 v[120:121], v86 offset:40960\n\t" \
  "v_mfma_f32_16x16x128_f8f6f4 v[228:231], v[104:111], a[104:111], v[228:231]\n\t" \
  "ds_read_b64_tr_b8 v[122:123], v86 offset:41984\n\t" \
  "v_mfma_f32_16x16x128_f8f6f4 v[232:235], v[104:111], a[112:119], v[232:235]\n\t" \
  "ds_read_b64_tr_b8 v[124:125], v102 offset:8192\n\t" \
  "v_mfma_f32_16x16x128_f8f6f4 v[236:239], v[104:111], a[120:127], v[236:239]\n\t" \
  "ds_read_b64_tr_b8 v[126:127], v102 offset:9216\n\t" \
  "s_waitcnt lgkmcnt(4)\n\t" \
  "v_mfma_f32_16x16x128_f8f6f4 v[236:239], v[112:119], a[152:159], v[236:239]\n\t" \
  "ds_read_b64_tr_b8 v[104:105], v86 offset:49152\n\t" \
  "v_pk_mul_f32 v[208:209], v[208:209], v[66:67]\n\t" \
  "v_pk_mul_f32 v[210:211], v[210:211], v[66:67]\n\t" \
  "s_mov_b32 s52, 0x0\n\t" \
  "buffer_store_dwordx4 v[208:211], v13, s[16:19], s52 offen offset:320 sc1\n\t" \
  "v_mfma_f32_16x16x128_f8f6f4 v[232:235], v[112:119], a[144:151], v[232:235]\n\t" \
  "ds_read_b64_tr_b8 v[106:107], v86 offset:50176\n\t" \
  "v_mfma_f32_16x16x128_f8f6f4 v[228:231], v[112:119], a[136:143], v[228:231]\n\t" \
  "ds_read_b64_tr_b8 v[108:109], v102 offset:16384\n\t" \
  "v_mfma_f32_16x16x128_f8f6f4 v[224:227], v[112:119], a[128:135], v[224:227]\n\t" \
  "ds_read_b64_tr_b8 v[110:111], v102 offset:17408\n\t" \
  "s_waitcnt lgkmcnt(4)\n\t" \
  "v_mfma_f32_16x16x128_f8f6f4 v[224:227], v[120:127], a[160:167], v[224:227]\n\t" \
  "ds_read_b64_tr_b8 v[112:113], v86 offset:57344\n\t" \
  "v_pk_mul_f32 v[212:213], v[212:213], v[66:67]\n\t" \
  "v_pk_mul_f32 v[214:215], v[214:215], v[66:67]\n\t" \
  "s_mov_b32 s53, 0x100000\n\t" \
  "buffer_store_dwordx4 v[212:215], v13, s[16:19], s53 offen offset:320 sc1\n\t" \
  "v_mfma_f32_16x16x128_f8f6f4 v[228:231], v[120:127], a[168:175], v[228:231]\n\t" \
  "ds_read_b64_tr_b8 v[114:115], v86 offset:58368\n\t" \
  "v_mfma_f32_16x16x128_f8f6f4 v[232:235], v[120:127], a[176:183], v[232:235]\n\t" \
  "ds_read_b64_tr_b8 v[116:117], v102 offset:24576\n\t" \
  "v_mfma_f32_16x16x128_f8f6f4 v[236:239], v[120:127], a[184:191], v[236:239]\n\t" \
  "ds_read_b64_tr_b8 v[118:119], v102 offset:25600\n\t" \
  "s_waitcnt lgkmcnt(4)\n\t" \
  "v_mfma_f32_16x16x128_f8f6f4 v[236:239], v[104:111], a[216:223], v[236:239]\n\t" \
  "ds_read_b64_tr_b8 v[120:121], v87 offset:24576\n\t" \
  "v_pk_mul_f32 v[216:217], v[216:217], v[66:67]\n\t" \
  "v_pk_mul_f32 v[218:219], v[218:219], v[66:67]\n\t" \
  "s_mov_b32 s54, 0x200000\n\t" \
  "buffer_store_dwordx4 v[216:219], v13, s[16:19], s54 offen offset:320 sc1\n\t" \
  "v_mfma_f32_16x16x128_f8f6f4 v[232:235], v[104:111], a[208:215], v[232:235]\n\t" \
  "ds_read_b64_tr_b8 v[122:123], v87 offset:25600\n\t" \
  "v_mfma_f32_16x16x128_f8f6f4 v[228:231], v[104:111], a[200:207], v[228:231]\n\t" \
  "ds_read_b64_tr_b8 v[124:125], v95 offset:57344\n\t" \
  "v_mfma_f32_16x16x128_f8f6f4 v[224:227], v[104:111], a[192:199], v[224:227]\n\t" \
  "ds_read_b64_tr_b8 v[126:127], v95 offset:58368\n\t" \
  "s_waitcnt lgkmcnt(4)\n\t" \
  "v_mfma_f32_16x16x128_f8f6f4 v[224:227], v[112:119], a[224:231], v[224:227]\n\t" \
  "ds_read_b64_tr_b8 v[104:105], v87 offset:32768\n\t" \
  "v_pk_mul_f32 v[220:221], v[220:221], v[66:67]\n\t" \
  "v_pk_mul_f32 v[222:223], v[222:223], v[66:67]\n\t" \
  "s_mov_b32 s55, 0x300000\n\t" \
  "buffer_store_dwordx4 v[220:223], v13, s[16:19], s55 offen offset:320 sc1\n\t" \
  "v_mfma_f32_16x16x128_f8f6f4 v[228:231], v[112:119], a[232:239], v[228:231]\n\t" \
  "ds_read_b64_tr_b8 v[106:107], v87 offset:33792\n\t" \
  "v_mfma_f32_16x16x128_f8f6f4 v[232:235], v[112:119], a[240:247], v[232:235]\n\t" \
  "ds_read_b64_tr_b8 v[108:109], v103 offset:0\n\t" \
  "v_mfma_f32_16x16x128_f8f6f4 v[236:239], v[112:119], a[248:255], v[236:239]\n\t" \
  "ds_read_b64_tr_b8 v[110:111], v103 offset:1024\n\t" \
  "s_waitcnt lgkmcnt(4)\n\t" \
  "v_mfma_f32_16x16x128_f8f6f4 v[252:255], v[120:127], a[120:127], v[252:255]\n\t" \
  "ds_read_b64_tr_b8 v[112:113], v87 offset:40960\n\t" \
  "v_mfma_f32_16x16x128_f8f6f4 v[248:251], v[120:127], a[112:119], v[248:251]\n\t" \
  "ds_read_b64_tr_b8 v[114:115], v87 offset:41984\n\t" \
  "v_mfma_f32_16x16x128_f8f6f4 v[244:247], v[120:127], a[104:111], v[244:247]\n\t" \
  "ds_read_b64_tr_b8 v[116:117], v103 offset:8192\n\t" \
  "v_mfma_f32_16x16x128_f8f6f4 v[240:243], v[120:127], a[96:103], v[240:243]\n\t" \
  "ds_read_b64_tr_b8 v[118:119], v103 offset:9216\n\t" \
  "s_waitcnt lgkmcnt(4)\n\t" \
  "v_mfma_f32_16x16x128_f8f6f4 v[240:243], v[104:111], a[128:135], v[240:243]\n\t" \
  "ds_read_b64_tr_b8 v[120:121], v87 offset:49152\n\t" \
  "v_pk_mul_f32 v[224:225], v[224:225], v[66:67]\n\t" \
  "v_pk_mul_f32 v[226:227], v[226:227], v[66:67]\n\t" \
  "s_mov_b32 s48, 0x0\n\t" \
  "buffer_store_dwordx4 v[224:227], v13, s[16:19], s48 offen offset:384 sc1\n\t" \
  "v_mfma_f32_16x16x128_f8f6f4 v[244:247], v[104:111], a[136:143], v[244:247]\n\t" \
  "ds_read_b64_tr_b8 v[122:123], v87 offset:50176\n\t" \
  "v_mfma_f32_16x16x128_f8f6f4 v[248:251], v[104:111], a[144:151], v[248:251]\n\t" \
  "ds_read_b64_tr_b8 v[124:125], v103 offset:16384\n\t" \
  "v_mfma_f32_16x16x128_f8f6f4 v[252:255], v[104:111], a[152:159], v[252:255]\n\t" \
  "ds_read_b64_tr_b8 v[126:127], v103 offset:17408\n\t" \
  "s_waitcnt lgkmcnt(4)\n\t" \
  "v_mfma_f32_16x16x128_f8f6f4 v[252:255], v[112:119], a[184:191], v[252:255]\n\t" \
  "ds_read_b64_tr_b8 v[104:105], v87 offset:57344\n\t" \
  "v_pk_mul_f32 v[228:229], v[228:229], v[66:67]\n\t" \
  "v_pk_mul_f32 v[230:231], v[230:231], v[66:67]\n\t" \
  "s_mov_b32 s49, 0x100000\n\t" \
  "buffer_store_dwordx4 v[228:231], v13, s[16:19], s49 offen offset:384 sc1\n\t" \
  "v_mfma_f32_16x16x128_f8f6f4 v[248:251], v[112:119], a[176:183], v[248:251]\n\t" \
  "ds_read_b64_tr_b8 v[106:107], v87 offset:58368\n\t" \
  "v_mfma_f32_16x16x128_f8f6f4 v[244:247], v[112:119], a[168:175], v[244:247]\n\t" \
  "ds_read_b64_tr_b8 v[108:109], v103 offset:24576\n\t" \
  "v_mfma_f32_16x16x128_f8f6f4 v[240:243], v[112:119], a[160:167], v[240:243]\n\t" \
  "ds_read_b64_tr_b8 v[110:111], v103 offset:25600\n\t" \
  "s_waitcnt lgkmcnt(4)\n\t" \
  "v_mfma_f32_16x16x128_f8f6f4 v[240:243], v[120:127], a[192:199], v[240:243]\n\t" \
  "v_pk_mul_f32 v[232:233], v[232:233], v[66:67]\n\t" \
  "v_pk_mul_f32 v[234:235], v[234:235], v[66:67]\n\t" \
  "s_mov_b32 s50, 0x200000\n\t" \
  "buffer_store_dwordx4 v[232:235], v13, s[16:19], s50 offen offset:384 sc1\n\t" \
  "v_mfma_f32_16x16x128_f8f6f4 v[244:247], v[120:127], a[200:207], v[244:247]\n\t" \
  "v_mfma_f32_16x16x128_f8f6f4 v[248:251], v[120:127], a[208:215], v[248:251]\n\t" \
  "v_mfma_f32_16x16x128_f8f6f4 v[252:255], v[120:127], a[216:223], v[252:255]\n\t" \
  "s_waitcnt lgkmcnt(0)\n\t" \
  "v_mfma_f32_16x16x128_f8f6f4 v[252:255], v[104:111], a[248:255], v[252:255]\n\t" \
  "v_pk_mul_f32 v[236:237], v[236:237], v[66:67]\n\t" \
  "v_pk_mul_f32 v[238:239], v[238:239], v[66:67]\n\t" \
  "s_mov_b32 s51, 0x300000\n\t" \
  "buffer_store_dwordx4 v[236:239], v13, s[16:19], s51 offen offset:384 sc1\n\t" \
  "v_mfma_f32_16x16x128_f8f6f4 v[248:251], v[104:111], a[240:247], v[248:251]\n\t" \
  "v_mfma_f32_16x16x128_f8f6f4 v[244:247], v[104:111], a[232:239], v[244:247]\n\t" \
  "v_mfma_f32_16x16x128_f8f6f4 v[240:243], v[104:111], a[224:231], v[240:243]\n\t" \
  "s_nop 15\n\t" \
  "s_nop 7\n\t" \
  "v_pk_mul_f32 v[240:241], v[240:241], v[66:67]\n\t" \
  "v_pk_mul_f32 v[242:243], v[242:243], v[66:67]\n\t" \
  "s_mov_b32 s52, 0x0\n\t" \
  "buffer_store_dwordx4 v[240:243], v13, s[16:19], s52 offen offset:448 sc1\n\t" \
  "v_pk_mul_f32 v[244:245], v[244:245], v[66:67]\n\t" \
  "v_pk_mul_f32 v[246:247], v[246:247], v[66:67]\n\t" \
  "s_mov_b32 s53, 0x100000\n\t" \
  "buffer_store_dwordx4 v[244:247], v13, s[16:19], s53 offen offset:448 sc1\n\t" \
  "v_pk_mul_f32 v[248:249], v[248:249], v[66:67]\n\t" \
  "v_pk_mul_f32 v[250:251], v[250:251], v[66:67]\n\t" \
  "s_mov_b32 s54, 0x200000\n\t" \
  "buffer_store_dwordx4 v[248:251], v13, s[16:19], s54 offen offset:448 sc1\n\t" \
  "v_pk_mul_f32 v[252:253], v[252:253], v[66:67]\n\t" \
  "v_pk_mul_f32 v[254:255], v[254:255], v[66:67]\n\t" \
  "s_mov_b32 s55, 0x300000\n\t" \
  "buffer_store_dwordx4 v[252:255], v13, s[16:19], s55 offen offset:448 sc1\n\t" \
  "s_endpgm\n\t" \
  "Lpoll_0_%=:\n\t" \
  "s_mov_b32 exec_lo, s69\n\t" \
  "s_mov_b32 exec_hi, 1\n\t" \
  "global_load_dwordx4 v[68:71], v10, s[6:7] sc0 sc1\n\t" \
  "s_mov_b64 exec, -1\n\t" \
  "s_waitcnt vmcnt(0)\n\t" \
  "v_min_u32 v68, v68, v69\n\t" \
  "v_min3_u32 v68, v68, v70, v71\n\t" \
  "v_cmp_le_u32 vcc, s57, v68\n\t" \
  "s_and_b32 s64, vcc_lo, s69\n\t" \
  "s_or_b32 s56, s56, s64\n\t" \
  "s_and_b32 s64, s56, s62\n\t" \
  "s_cmp_eq_u32 s64, s62\n\t" \
  "s_cbranch_scc1 Lret_0_%=\n\t" \
  "s_bitcmp1_b32 vcc_hi, 0\n\t" \
  "s_cbranch_scc1 Lfail_0_%=\n\t" \
  "s_memrealtime s[64:65]\n\t" \
  "s_waitcnt lgkmcnt(0)\n\t" \
  "s_sub_u32 s64, s64, s58\n\t" \
  "s_subb_u32 s65, s65, s59\n\t" \
  "s_cmp_lg_u32 s65, 0\n\t" \
  "s_cbranch_scc1 Lfail_0_%=\n\t" \
  "s_cmp_gt_u32 s64, 50000000\n\t" \
  "s_cbranch_scc1 Lfail_0_%=\n\t" \
  "s_sleep 4\n\t" \
  "s_branch Lpoll_0_%=\n\t" \
  "Lfail_0_%=:\n\t" \
  "v_mov_b32 v68, s57\n\t" \
  "v_mov_b32 v69, s57\n\t" \
  "v_mov_b32 v70, s57\n\t" \
  "v_mov_b32 v71, s57\n\t" \
  "s_mov_b32 exec_lo, 0\n\t" \
  "s_mov_b32 exec_hi, 1\n\t" \
  "global_store_dwordx4 v10, v[68:71], s[6:7] sc0 sc1\n\t" \
  "s_mov_b64 exec, -1\n\t" \
  "s_waitcnt vmcnt(0)\n\t" \
  "s_mov_b32 s63, 0x7fc00000\n\t" \
  "s_mov_b32 s56, -1\n\t" \
  "Lret_0_%=:\n\t" \
  "s_setpc_b64 s[60:61]\n\t" \
  "Lend_0_%=:\n\t" \
  "Lvar_1_%=:\n\t" \
  "s_lshl_b32 s64, s4, 15\n\t" \
  "s_add_u32 s64, s64, s78\n\t" \
  "s_lshl_b32 s84, s4, 13\n\t" \
  "v_lshrrev_b32 v17, 3, v1\n\t" \
  "v_lshrrev_b32 v18, 4, v1\n\t" \
  "v_and_b32 v19, 7, v1\n\t" \
  "v_xor_b32 v18, v18, v19\n\t" \
  "v_lshlrev_b32 v18, 4, v18\n\t" \
  "v_lshl_or_b32 v5, v17, 9, v18\n\t" \
  "v_add_u32 v5, s64, v5\n\t" \
  "v_xor_b32 v6, 64, v5\n\t" \
  "v_lshlrev_b32 v4, 4, v1\n\t" \
  "s_lshl_b32 s64, s72, 15\n\t" \
  "s_lshl_b32 s65, s4, 13\n\t" \
  "s_add_u32 s64, s64, s65\n\t" \
  "s_add_u32 s12, s6, s64\n\t" \
  "s_addc_u32 s13, s7, 0\n\t" \
  "s_and_b32 s13, s13, 0xffff\n\t" \
  "s_sub_u32 s14, 0x400000, s64\n\t" \
  "s_mov_b32 s15, 0x20000\n\t" \
  "s_lshl_b32 s64, s72, 24\n\t" \
  "s_lshl_b32 s65, s4, 22\n\t" \
  "s_add_u32 s64, s64, s65\n\t" \
  "s_lshl_b32 s65, s74, 2\n\t" \
  "s_add_u32 s64, s64, s65\n\t" \
  "s_add_u32 s16, s16, s64\n\t" \
  "s_addc_u32 s17, s17, 0\n\t" \
  "s_and_b32 s17, s17, 0xffff\n\t" \
  "s_sub_u32 s18, 0x400000, s65\n\t" \
  "s_mov_b32 s19, 0x20000\n\t" \
  "s_bfe_u32 s26, s27, 0x4000a\n\t" \
  "s_sub_u32 s64, 6, s75\n\t" \
  "s_and_b32 s64, s64, 7\n\t" \
  "s_mul_i32 s65, s77, 7\n\t" \
  "s_add_u32 s65, s65, s64\n\t" \
  "s_add_u32 s65, s65, 4\n\t" \
  "s_cmp_eq_u32 s64, 7\n\t" \
  "s_cselect_b32 s65, s77, s65\n\t" \
  "s_lshl_b32 s65, s65, 16\n\t" \
  "s_lshl_b32 s82, s76, 11\n\t" \
  "s_add_u32 s65, s65, s82\n\t" \
  "v_add_u32 v9, s65, v4\n\t" \
  "s_lshr_b32 s82, s76, 2\n\t" \
  "s_lshl_b32 s82, s82, 9\n\t" \
  "s_lshl_b32 s83, s77, 7\n\t" \
  "s_add_u32 s82, s82, s83\n\t" \
  "s_lshl_b32 s83, s64, 4\n\t" \
  "s_add_u32 s82, s82, s83\n\t" \
  "s_and_b32 s83, s76, 3\n\t" \
  "s_lshl_b32 s83, s83, 2\n\t" \
  "s_add_u32 s82, s82, s83\n\t" \
  "s_add_u32 s82, s82, 0x400000\n\t" \
  "s_lshl_b32 s64, s72, 2\n\t" \
  "s_add_u32 s64, s64, s4\n\t" \
  "s_lshl_b32 s64, s64, 9\n\t" \
  "s_add_u32 s64, s64, 0x400000\n\t" \
  "v_lshl_add_u32 v10, v1, 4, s64\n\t" \
  "v_cmp_lt_u32 vcc, 31, v1\n\t" \
  "v_mov_b32 v18, 0x401000\n\t" \
  "s_nop 0\n\t" \
  "v_cndmask_b32 v10, v10, v18, vcc\n\t" \
  "v_and_b32 v17, 15, v1\n\t" \
  "v_lshrrev_b32 v18, 4, v1\n\t" \
  "v_lshlrev_b32 v19, 4, v18\n\t" \
  "v_lshl_or_b32 v13, v17, 16, v19\n\t" \
  "v_lshl_add_u32 v15, v17, 2, v18\n\t" \
  "v_lshlrev_b32 v15, 2, v15\n\t" \
  "v_lshrrev_b32 v19, 1, v17\n\t" \
  "v_lshl_add_u32 v19, v18, 4, v19\n\t" \
  "v_lshlrev_b32 v19, 7, v19\n\t" \
  "v_bfe_u32 v20, v1, 2, 3\n\t" \
  "v_lshl_or_b32 v19, v20, 4, v19\n\t" \
  "v_and_b32 v20, 1, v1\n\t" \
  "v_lshl_or_b32 v14, v20, 3, v19\n\t" \
  "v_lshrrev_b32 v17, 2, v1\n\t" \
  "v_min_u32 v17, 6, v17\n\t" \
  "v_lshlrev_b32 v17, 16, v17\n\t" \
  "v_and_b32 v18, 3, v1\n\t" \
  "v_lshl_or_b32 v17, v18, 7, v17\n\t" \
  "s_lshl_b32 s64, s73, 9\n\t" \
  "v_add_u32 v16, s64, v17\n\t" \
  "v_and_b32 v17, 15, v1\n\t" \
  "v_lshrrev_b32 v18, 4, v1\n\t" \
  "s_lshl_b32 s64, s72, 2\n\t" \
  "v_add_u32 v18, s64, v18\n\t" \
  "v_lshrrev_b32 v19, 1, v17\n\t" \
  "v_xor_b32 v18, v18, v19\n\t" \
  "v_lshlrev_b32 v18, 4, v18\n\t" \
  "v_lshl_or_b32 v17, v17, 7, v18\n\t" \
  "s_add_u32 s64, s84, 0x10000\n\t" \
  "v_add_u32 v20, s64, v17\n\t" \
  "v_xor_b32 v21, 64, v20\n\t" \
  "s_add_u32 s64, s84, 0x20000\n\t" \
  "v_add_u32 v22, s64, v17\n\t" \
  "v_xor_b32 v23, 64, v22\n\t" \
  "s_add_u32 s40, s25, 1\n\t" \
  "s_and_b32 s40, s40, 7\n\t" \
  "s_lshl_b32 s40, s40, 23\n\t" \
  "s_add_u32 s41, s25, 2\n\t" \
  "s_and_b32 s41, s41, 7\n\t" \
  "s_lshl_b32 s41, s41, 23\n\t" \
  "s_add_u32 s42, s25, 3\n\t" \
  "s_and_b32 s42, s42, 7\n\t" \
  "s_lshl_b32 s42, s42, 23\n\t" \
  "s_add_u32 s43, s25, 4\n\t" \
  "s_and_b32 s43, s43, 7\n\t" \
  "s_lshl_b32 s43, s43, 23\n\t" \
  "s_add_u32 s44, s25, 5\n\t" \
  "s_and_b32 s44, s44, 7\n\t" \
  "s_lshl_b32 s44, s44, 23\n\t" \
  "s_add_u32 s45, s25, 6\n\t" \
  "s_and_b32 s45, s45, 7\n\t" \
  "s_lshl_b32 s45, s45, 23\n\t" \
  "s_add_u32 s46, s25, 7\n\t" \
  "s_and_b32 s46, s46, 7\n\t" \
  "s_lshl_b32 s46, s46, 23\n\t" \
  "v_xor_b32 v80, 0x0, v14\n\t" \
  "v_xor_b32 v81, 0x10, v14\n\t" \
  "v_xor_b32 v82, 0x20, v14\n\t" \
  "v_xor_b32 v83, 0x30, v14\n\t" \
  "v_xor_b32 v84, 0x40, v14\n\t" \
  "v_xor_b32 v85, 0x50, v14\n\t" \
  "v_xor_b32 v86, 0x60, v14\n\t" \
  "v_xor_b32 v87, 0x70, v14\n\t" \
  "v_xor_b32 v88, 0x10000, v14\n\t" \
  "v_xor_b32 v89, 0x10010, v14\n\t" \
  "v_xor_b32 v90, 0x10020, v14\n\t" \
  "v_xor_b32 v91, 0x10030, v14\n\t" \
  "v_xor_b32 v92, 0x10040, v14\n\t" \
  "v_xor_b32 v93, 0x10050, v14\n\t" \
  "v_xor_b32 v94, 0x10060, v14\n\t" \
  "v_xor_b32 v95, 0x10070, v14\n\t" \
  "v_xor_b32 v96, 0x20000, v14\n\t" \
  "v_xor_b32 v97, 0x20010, v14\n\t" \
  "v_xor_b32 v98, 0x20020, v14\n\t" \
  "v_xor_b32 v99, 0x20030, v14\n\t" \
  "v_xor_b32 v100, 0x20040, v14\n\t" \
  "v_xor_b32 v101, 0x20050, v14\n\t" \
  "v_xor_b32 v102, 0x20060, v14\n\t" \
  "v_xor_b32 v103, 0x20070, v14\n\t" \
  "s_mov_b32 s56, 0\n\t" \
  "s_mov_b32 s63, 0\n\t" \
  "s_waitcnt lgkmcnt(0)\n\t" \
  "s_add_u32 s30, s28, s82\n\t" \
  "s_addc_u32 s31, s29, 0\n\t" \
  "s_add_u32 m0, s24, 0x18000\n\t" \
  "s_add_u32 s52, s47, 0x0\n\t" \
  "buffer_load_dwordx4 v3, s[8:11], s52 offen lds\n\t" \
  "s_add_u32 m0, s24, 0x18400\n\t" \
  "s_add_u32 s53, s47, 0x20000\n\t" \
  "buffer_load_dwordx4 v3, s[8:11], s53 offen lds\n\t" \
  "s_add_u32 m0, s84, 0x10000\n\t" \
  "s_mov_b32 s54, 0x0\n\t" \
  "buffer_load_dwordx4 v5, s[20:23], s54 offen lds\n\t" \
  "s_add_u32 m0, s84, 0x10400\n\t" \
  "s_mov_b32 s55, 0x1000\n\t" \
  "buffer_load_dwordx4 v6, s[20:23], s55 offen lds\n\t" \
  "s_add_u32 m0, s84, 0x10800\n\t" \
  "s_mov_b32 s48, 0x2000\n\t" \
  "buffer_load_dwordx4 v5, s[20:23], s48 offen lds\n\t" \
  "s_add_u32 m0, s84, 0x10c00\n\t" \
  "s_mov_b32 s49, 0x3000\n\t" \
  "buffer_load_dwordx4 v6, s[20:23], s49 offen lds\n\t" \
  "s_add_u32 m0, s84, 0x11000\n\t" \
  "s_mov_b32 s50, 0x4000\n\t" \
  "buffer_load_dwordx4 v5, s[20:23], s50 offen lds\n\t" \
  "s_add_u32 m0, s84, 0x11400\n\t" \
  "s_mov_b32 s51, 0x5000\n\t" \
  "buffer_load_dwordx4 v6, s[20:23], s51 offen lds\n\t" \
  "s_add_u32 m0, s84, 0x11800\n\t" \
  "s_mov_b32 s52, 0x6000\n\t" \
  "buffer_load_dwordx4 v5, s[20:23], s52 offen lds\n\t" \
  "s_add_u32 m0, s84, 0x11c00\n\t" \
  "s_mov_b32 s53, 0x7000\n\t" \
  "buffer_load_dwordx4 v6, s[20:23], s53 offen lds\n\t" \
  "s_waitcnt vmcnt(18)\n\t" \
  "v_add_u32 v64, 1, v64\n\t" \
  "s_nop 0\n\t" \
  "v_readfirstlane_b32 s57, v64\n\t" \
  "s_and_b32 s68, s57, 1\n\t" \
  "s_lshl_b32 s68, s68, 21\n\t" \
  "v_add_u32 v63, s68, v9\n\t" \
  "s_waitcnt lgkmcnt(0)\n\t" \
  "s_add_u32 m0, s24, 0x1a000\n\t" \
  "s_add_u32 s54, s47, 0x200000\n\t" \
  "buffer_load_dwordx4 v3, s[8:11], s54 offen lds\n\t" \
  "s_add_u32 m0, s24, 0x1a400\n\t" \
  "s_add_u32 s55, s47, 0x220000\n\t" \
  "buffer_load_dwordx4 v3, s[8:11], s55 offen lds\n\t" \
  "s_waitcnt vmcnt(2)\n\t" \
  "ds_read_b128 a[0:3], v20 offset:0\n\t" \
  "ds_read_b128 a[4:7], v21 offset:0\n\t" \
  "ds_read_b128 a[8:11], v20 offset:2048\n\t" \
  "ds_read_b128 a[12:15], v21 offset:2048\n\t" \
  "ds_read_b128 a[16:19], v20 offset:4096\n\t" \
  "ds_read_b128 a[20:23], v21 offset:4096\n\t" \
  "ds_read_b128 a[24:27], v20 offset:6144\n\t" \
  "ds_read_b128 a[28:31], v21 offset:6144\n\t" \
  "s_waitcnt lgkmcnt(0)\n\t" \
  "s_waitcnt lgkmcnt(0)\n\t" \
  "s_barrier\n\t" \
  "s_mov_b64 exec, 1\n\t" \
  "global_store_dword v65, v64, s[34:35] sc1\n\t" \
  "s_mov_b64 exec, -1\n\t" \
  "ds_read_b64_tr_b8 v[104:105], v80 offset:0\n\t" \
  "ds_read_b64_tr_b8 v[106:107], v80 offset:1024\n\t" \
  "ds_read_b64_tr_b8 v[108:109], v88 offset:32768\n\t" \
  "ds_read_b64_tr_b8 v[110:111], v88 offset:33792\n\t" \
  "ds_read_b64_tr_b8 v[112:113], v81 offset:0\n\t" \
  "ds_read_b64_tr_b8 v[114:115], v81 offset:1024\n\t" \
  "ds_read_b64_tr_b8 v[116:117], v89 offset:32768\n\t" \
  "ds_read_b64_tr_b8 v[118:119], v89 offset:33792\n\t" \
  "s_waitcnt lgkmcnt(4)\n\t" \
  "v_mfma_f32_16x16x128_f8f6f4 v[128:131], v[104:111], a[0:7], 0\n\t" \
  "ds_read_b64_tr_b8 v[120:121], v82 offset:0\n\t" \
  "s_add_u32 m0, s84, 0x20000\n\t" \
  "s_mov_b32 s48, 0x80\n\t" \
  "buffer_load_dwordx4 v5, s[20:23], s48 offen lds\n\t" \
  "v_mfma_f32_16x16x128_f8f6f4 v[132:135], v[104:111], a[8:15], 0\n\t" \
  "ds_read_b64_tr_b8 v[122:123], v82 offset:1024\n\t" \
  "s_add_u32 m0, s84, 0x20400\n\t" \
  "s_mov_b32 s49, 0x1080\n\t" \
  "buffer_load_dwordx4 v6, s[20:23], s49 offen lds\n\t" \
  "v_mfma_f32_16x16x128_f8f6f4 v[136:139], v[104:111], a[16:23], 0\n\t" \
  "ds_read_b64_tr_b8 v[124:125], v90 offset:32768\n\t" \
  "s_add_u32 m0, s84, 0x20800\n\t" \
  "s_mov_b32 s50, 0x2080\n\t" \
  "buffer_load_dwordx4 v5, s[20:23], s50 offen lds\n\t" \
  "v_mfma_f32_16x16x128_f8f6f4 v[140:143], v[104:111], a[24:31], 0\n\t" \
  "ds_read_b64_tr_b8 v[126:127], v90 offset:33792\n\t" \
  "s_add_u32 m0, s84, 0x20c00\n\t" \
  "s_mov_b32 s51, 0x3080\n\t" \
  "buffer_load_dwordx4 v6, s[20:23], s51 offen lds\n\t" \
  "s_waitcnt lgkmcnt(4)\n\t" \
  "v_mfma_f32_16x16x128_f8f6f4 v[156:159], v[112:119], a[24:31], 0\n\t" \
  "ds_read_b64_tr_b8 v[104:105], v83 offset:0\n\t" \
  "s_add_u32 m0, s84, 0x21000\n\t" \
  "s_mov_b32 s52, 0x4080\n\t" \
  "buffer_load_dwordx4 v5, s[20:23], s52 offen lds\n\t" \
  "v_mfma_f32_16x16x128_f8f6f4 v[152:155], v[112:119], a[16:23], 0\n\t" \
  "ds_read_b64_tr_b8 v[106:107], v83 offset:1024\n\t" \
  "s_add_u32 m0, s84, 0x21400\n\t" \
  "s_mov_b32 s53, 0x5080\n\t" \
  "buffer_load_dwordx4 v6, s[20:23], s53 offen lds\n\t" \
  "v_mfma_f32_16x16x128_f8f6f4 v[148:151], v[112:119], a[8:15], 0\n\t" \
  "ds_read_b64_tr_b8 v[108:109], v91 offset:32768\n\t" \
  "s_add_u32 m0, s84, 0x21800\n\t" \
  "s_mov_b32 s54, 0x6080\n\t" \
  "buffer_load_dwordx4 v5, s[20:23], s54 offen lds\n\t" \
  "v_mfma_f32_16x16x128_f8f6f4 v[144:147], v[112:119], a[0:7], 0\n\t" \
  "ds_read_b64_tr_b8 v[110:111], v91 offset:33792\n\t" \
  "s_add_u32 m0, s84, 0x21c00\n\t" \
  "s_mov_b32 s55, 0x7080\n\t" \
  "buffer_load_dwordx4 v6, s[20:23], s55 offen lds\n\t" \
  "s_waitcnt lgkmcnt(4)\n\t" \
  "v_mfma_f32_16x16x128_f8f6f4 v[160:163], v[120:127], a[0:7], 0\n\t" \
  "ds_read_b64_tr_b8 v[112:113], v84 offset:0\n\t" \
  "s_add_u32 m0, s24, 0x4000\n\t" \
  "s_add_u32 s48, s47, 0x400000\n\t" \
  "buffer_load_dwordx4 v2, s[8:11], s48 offen lds\n\t" \
  "v_mfma_f32_16x16x128_f8f6f4 v[164:167], v[120:127], a[8:15], 0\n\t" \
  "ds_read_b64_tr_b8 v[114:115], v84 offset:1024\n\t" \
  "s_add_u32 m0, s24, 0x4400\n\t" \
  "s_add_u32 s49, s47, 0x420000\n\t" \
  "buffer_load_dwordx4 v2, s[8:11], s49 offen lds\n\t" \
  "v_mfma_f32_16x16x128_f8f6f4 v[168:171], v[120:127], a[16:23], 0\n\t" \
  "ds_read_b64_tr_b8 v[116:117], v92 offset:32768\n\t" \
  "s_add_u32 m0, s24, 0x6000\n\t" \
  "s_add_u32 s50, s47, 0x600000\n\t" \
  "buffer_load_dwordx4 v2, s[8:11], s50 offen lds\n\t" \
  "v_mfma_f32_16x16x128_f8f6f4 v[172:175], v[120:127], a[24:31], 0\n\t" \
  "ds_read_b64_tr_b8 v[118:119], v92 offset:33792\n\t" \
  "s_add_u32 m0, s24, 0x6400\n\t" \
  "s_add_u32 s51, s47, 0x620000\n\t" \
  "buffer_load_dwordx4 v2, s[8:11], s51 offen lds\n\t" \
  "s_waitcnt lgkmcnt(4)\n\t" \
  "v_mfma_f32_16x16x128_f8f6f4 v[188:191], v[104:111], a[24:31], 0\n\t" \
  "ds_read_b64_tr_b8 v[120:121], v85 offset:0\n\t" \
  "v_mfma_f32_16x16x128_f8f6f4 v[184:187], v[104:111], a[16:23], 0\n\t" \
  "ds_read_b64_tr_b8 v[122:123], v85 offset:1024\n\t" \
  "v_mfma_f32_16x16x128_f8f6f4 v[180:183], v[104:111], a[8:15], 0\n\t" \
  "ds_read_b64_tr_b8 v[124:125], v93 offset:32768\n\t" \
  "v_mfma_f32_16x16x128_f8f6f4 v[176:179], v[104:111], a[0:7], 0\n\t" \
  "ds_read_b64_tr_b8 v[126:127], v93 offset:33792\n\t" \
  "s_waitcnt lgkmcnt(4)\n\t" \
  "v_mfma_f32_16x16x128_f8f6f4 v[192:195], v[112:119], a[0:7], 0\n\t" \
  "ds_read_b64_tr_b8 v[104:105], v86 offset:0\n\t" \
  "v_mfma_f32_16x16x128_f8f6f4 v[196:199], v[112:119], a[8:15], 0\n\t" \
  "ds_read_b64_tr_b8 v[106:107], v86 offset:1024\n\t" \
  "v_mfma_f32_16x16x128_f8f6f4 v[200:203], v[112:119], a[16:23], 0\n\t" \
  "ds_read_b64_tr_b8 v[108:109], v94 offset:32768\n\t" \
  "v_mfma_f32_16x16x128_f8f6f4 v[204:207], v[112:119], a[24:31], 0\n\t" \
  "ds_read_b64_tr_b8 v[110:111], v94 offset:33792\n\t" \
  "s_waitcnt lgkmcnt(4)\n\t" \
  "v_mfma_f32_16x16x128_f8f6f4 v[220:223], v[120:127], a[24:31], 0\n\t" \
  "ds_read_b64_tr_b8 v[112:113], v87 offset:0\n\t" \
  "v_mfma_f32_16x16x128_f8f6f4 v[216:219], v[120:127], a[16:23], 0\n\t" \
  "ds_read_b64_tr_b8 v[114:115], v87 offset:1024\n\t" \
  "v_mfma_f32_16x16x128_f8f6f4 v[212:215], v[120:127], a[8:15], 0\n\t" \
  "ds_read_b64_tr_b8 v[116:117], v95 offset:32768\n\t" \
  "s_waitcnt vmcnt(4)\n\t" \
  "ds_read_b128 a[32:35], v22 offset:0\n\t" \
  "v_mfma_f32_16x16x128_f8f6f4 v[208:211], v[120:127], a[0:7], 0\n\t" \
  "ds_read_b64_tr_b8 v[118:119], v95 offset:33792\n\t" \
  "ds_read_b128 a[36:39], v23 offset:0\n\t" \
  "s_waitcnt lgkmcnt(0)\n\t" \
  "s_barrier\n\t" \
  "v_mfma_f32_16x16x128_f8f6f4 v[224:227], v[104:111], a[0:7], 0\n\t" \
  "ds_read_b64_tr_b8 v[120:121], v80 offset:8192\n\t" \
  "ds_read_b128 a[40:43], v22 offset:2048\n\t" \
  "v_mfma_f32_16x16x128_f8f6f4 v[228:231], v[104:111], a[8:15], 0\n\t" \
  "ds_read_b64_tr_b8 v[122:123], v80 offset:9216\n\t" \
  "ds_read_b128 a[44:47], v23 offset:2048\n\t" \
  "v_mfma_f32_16x16x128_f8f6f4 v[232:235], v[104:111], a[16:23], 0\n\t" \
  "ds_read_b64_tr_b8 v[124:125], v88 offset:40960\n\t" \
  "ds_read_b128 a[48:51], v22 offset:4096\n\t" \
  "v_mfma_f32_16x16x128_f8f6f4 v[236:239], v[104:111], a[24:31], 0\n\t" \
  "ds_read_b64_tr_b8 v[126:127], v88 offset:41984\n\t" \
  "ds_read_b128 a[52:55], v23 offset:4096\n\t" \
  "v_mfma_f32_16x16x128_f8f6f4 v[252:255], v[112:119], a[24:31], 0\n\t" \
  "ds_read_b64_tr_b8 v[104:105], v81 offset:8192\n\t" \
  "ds_read_b128 a[56:59], v22 offset:6144\n\t" \
  "v_mfma_f32_16x16x128_f8f6f4 v[248:251], v[112:119], a[16:23], 0\n\t" \
  "ds_read_b64_tr_b8 v[106:107], v81 offset:9216\n\t" \
  "ds_read_b128 a[60:63], v23 offset:6144\n\t" \
  "v_mfma_f32_16x16x128_f8f6f4 v[244:247], v[112:119], a[8:15], 0\n\t" \
  "ds_read_b64_tr_b8 v[108:109], v89 offset:40960\n\t" \
  "v_mfma_f32_16x16x128_f8f6f4 v[240:243], v[112:119], a[0:7], 0\n\t" \
  "ds_read_b64_tr_b8 v[110:111], v89 offset:41984\n\t" \
  "s_waitcnt lgkmcnt(2)\n\t" \
  "v_mfma_f32_16x16x128_f8f6f4 v[128:131], v[120:127], a[32:39], v[128:131]\n\t" \
  "ds_read_b64_tr_b8 v[112:113], v82 offset:8192\n\t" \
  "s_add_u32 m0, s24, 0x1c000\n\t" \
  "s_add_u32 s52, s47, 0x400000\n\t" \
  "buffer_load_dwordx4 v3, s[8:11], s52 offen lds\n\t" \
  "v_mfma_f32_16x16x128_f8f6f4 v[132:135], v[120:127], a[40:47], v[132:135]\n\t" \
  "ds_read_b64_tr_b8 v[114:115], v82 offset:9216\n\t" \
  "s_add_u32 m0, s24, 0x1c400\n\t" \
  "s_add_u32 s53, s47, 0x420000\n\t" \
  "buffer_load_dwordx4 v3, s[8:11], s53 offen lds\n\t" \
  "v_mfma_f32_16x16x128_f8f6f4 v[136:139], v[120:127], a[48:55], v[136:139]\n\t" \
  "ds_read_b64_tr_b8 v[116:117], v90 offset:40960\n\t" \
  "s_add_u32 m0, s84, 0x10000\n\t" \
  "s_mov_b32 s54, 0x100\n\t" \
  "buffer_load_dwordx4 v5, s[20:23], s54 offen lds\n\t" \
  "v_mfma_f32_16x16x128_f8f6f4 v[140:143], v[120:127], a[56:63], v[140:143]\n\t" \
  "ds_read_b64_tr_b8 v[118:119], v90 offset:41984\n\t" \
  "s_add_u32 m0, s84, 0x10400\n\t" \
  "s_mov_b32 s55, 0x1100\n\t" \
  "buffer_load_dwordx4 v6, s[20:23], s55 offen lds\n\t" \
  "s_waitcnt lgkmcnt(4)\n\t" \
  "v_mfma_f32_16x16x128_f8f6f4 v[156:159], v[104:111], a[56:63], v[156:159]\n\t" \
  "ds_read_b64_tr_b8 v[120:121], v83 offset:8192\n\t" \
  "s_add_u32 m0, s84, 0x10800\n\t" \
  "s_mov_b32 s48, 0x2100\n\t" \
  "buffer_load_dwordx4 v5, s[20:23], s48 offen lds\n\t" \
  "v_mfma_f32_16x16x128_f8f6f4 v[152:155], v[104:111], a[48:55], v[152:155]\n\t" \
  "ds_read_b64_tr_b8 v[122:123], v83 offset:9216\n\t" \
  "s_add_u32 m0, s84, 0x10c00\n\t" \
  "s_mov_b32 s49, 0x3100\n\t" \
  "buffer_load_dwordx4 v6, s[20:23], s49 offen lds\n\t" \
  "v_mfma_f32_16x16x128_f8f6f4 v[148:151], v[104:111], a[40:47], v[148:151]\n\t" \
  "ds_read_b64_tr_b8 v[124:125], v91 offset:40960\n\t" \
  "s_add_u32 m0, s84, 0x11000\n\t" \
  "s_mov_b32 s50, 0x4100\n\t" \
  "buffer_load_dwordx4 v5, s[20:23], s50 offen lds\n\t" \
  "v_mfma_f32_16x16x128_f8f6f4 v[144:147], v[104:111], a[32:39], v[144:147]\n\t" \
  "ds_read_b64_tr_b8 v[126:127], v91 offset:41984\n\t" \
  "s_add_u32 m0, s84, 0x11400\n\t" \
  "s_mov_b32 s51, 0x5100\n\t" \
  "buffer_load_dwordx4 v6, s[20:23], s51 offen lds\n\t" \
  "s_waitcnt lgkmcnt(4)\n\t" \
  "v_mfma_f32_16x16x128_f8f6f4 v[160:163], v[112:119], a[32:39], v[160:163]\n\t" \
  "ds_read_b64_tr_b8 v[104:105], v84 offset:8192\n\t" \
  "s_add_u32 m0, s84, 0x11800\n\t" \
  "s_mov_b32 s52, 0x6100\n\t" \
  "buffer_load_dwordx4 v5, s[20:23], s52 offen lds\n\t" \
  "v_mfma_f32_16x16x128_f8f6f4 v[164:167], v[112:119], a[40:47], v[164:167]\n\t" \
  "ds_read_b64_tr_b8 v[106:107], v84 offset:9216\n\t" \
  "s_add_u32 m0, s84, 0x11c00\n\t" \
  "s_mov_b32 s53, 0x7100\n\t" \
  "buffer_load_dwordx4 v6, s[20:23], s53 offen lds\n\t" \
  "v_mfma_f32_16x16x128_f8f6f4 v[168:171], v[112:119], a[48:55], v[168:171]\n\t" \
  "ds_read_b64_tr_b8 v[108:109], v92 offset:40960\n\t" \
  "s_add_u32 m0, s24, 0x1e000\n\t" \
  "s_add_u32 s54, s47, 0x600000\n\t" \
  "buffer_load_dwordx4 v3, s[8:11], s54 offen lds\n\t" \
  "v_mfma_f32_16x16x128_f8f6f4 v[172:175], v[112:119], a[56:63], v[172:175]\n\t" \
  "ds_read_b64_tr_b8 v[110:111], v92 offset:41984\n\t" \
  "s_add_u32 m0, s24, 0x1e400\n\t" \
  "s_add_u32 s55, s47, 0x620000\n\t" \
  "buffer_load_dwordx4 v3, s[8:11], s55 offen lds\n\t" \
  "s_waitcnt lgkmcnt(4)\n\t" \
  "v_mfma_f32_16x16x128_f8f6f4 v[188:191], v[120:127], a[56:63], v[188:191]\n\t" \
  "ds_read_b64_tr_b8 v[112:113], v85 offset:8192\n\t" \
  "s_add_u32 m0, s84, 0x20000\n\t" \
  "s_mov_b32 s48, 0x180\n\t" \
  "buffer_load_dwordx4 v5, s[20:23], s48 offen lds\n\t" \
  "v_mfma_f32_16x16x128_f8f6f4 v[184:187], v[120:127], a[48:55], v[184:187]\n\t" \
  "ds_read_b64_tr_b8 v[114:115], v85 offset:9216\n\t" \
  "s_add_u32 m0, s84, 0x20400\n\t" \
  "s_mov_b32 s49, 0x1180\n\t" \
  "buffer_load_dwordx4 v6, s[20:23], s49 offen lds\n\t" \
  "v_mfma_f32_16x16x128_f8f6f4 v[180:183], v[120:127], a[40:47], v[180:183]\n\t" \
  "ds_read_b64_tr_b8 v[116:117], v93 offset:40960\n\t" \
  "s_add_u32 m0, s84, 0x20800\n\t" \
  "s_mov_b32 s50, 0x2180\n\t" \
  "buffer_load_dwordx4 v5, s[20:23], s50 offen lds\n\t" \
  "v_mfma_f32_16x16x128_f8f6f4 v[176:179], v[120:127], a[32:39], v[176:179]\n\t" \
  "ds_read_b64_tr_b8 v[118:119], v93 offset:41984\n\t" \
  "s_add_u32 m0, s84, 0x20c00\n\t" \
  "s_mov_b32 s51, 0x3180\n\t" \
  "buffer_load_dwordx4 v6, s[20:23], s51 offen lds\n\t" \
  "s_waitcnt lgkmcnt(4)\n\t" \
  "v_mfma_f32_16x16x128_f8f6f4 v[192:195], v[104:111], a[32:39], v[192:195]\n\t" \
  "ds_read_b64_tr_b8 v[120:121], v86 offset:8192\n\t" \
  "s_add_u32 m0, s84, 0x21000\n\t" \
  "s_mov_b32 s52, 0x4180\n\t" \
  "buffer_load_dwordx4 v5, s[20:23], s52 offen lds\n\t" \
  "v_mfma_f32_16x16x128_f8f6f4 v[196:199], v[104:111], a[40:47], v[196:199]\n\t" \
  "ds_read_b64_tr_b8 v[122:123], v86 offset:9216\n\t" \
  "s_add_u32 m0, s84, 0x21400\n\t" \
  "s_mov_b32 s53, 0x5180\n\t" \
  "buffer_load_dwordx4 v6, s[20:23], s53 offen lds\n\t" \
  "v_mfma_f32_16x16x128_f8f6f4 v[200:203], v[104:111], a[48:55], v[200:203]\n\t" \
  "ds_read_b64_tr_b8 v[124:125], v94 offset:40960\n\t" \
  "s_add_u32 m0, s84, 0x21800\n\t" \
  "s_mov_b32 s54, 0x6180\n\t" \
  "buffer_load_dwordx4 v5, s[20:23], s54 offen lds\n\t" \
  "v_mfma_f32_16x16x128_f8f6f4 v[204:207], v[104:111], a[56:63], v[204:207]\n\t" \
  "ds_read_b64_tr_b8 v[126:127], v94 offset:41984\n\t" \
  "s_add_u32 m0, s84, 0x21c00\n\t" \
  "s_mov_b32 s55, 0x7180\n\t" \
  "buffer_load_dwordx4 v6, s[20:23], s55 offen lds\n\t" \
  "s_waitcnt lgkmcnt(4)\n\t" \
  "v_mfma_f32_16x16x128_f8f6f4 v[220:223], v[112:119], a[56:63], v[220:223]\n\t" \
  "ds_read_b64_tr_b8 v[104:105], v87 offset:8192\n\t" \
  "v_mfma_f32_16x16x128_f8f6f4 v[216:219], v[112:119], a[48:55], v[216:219]\n\t" \
  "ds_read_b64_tr_b8 v[106:107], v87 offset:9216\n\t" \
  "v_mfma_f32_16x16x128_f8f6f4 v[212:215], v[112:119], a[40:47], v[212:215]\n\t" \
  "ds_read_b64_tr_b8 v[108:109], v95 offset:40960\n\t" \
  "s_waitcnt vmcnt(10)\n\t" \
  "ds_read_b128 a[64:67], v20 offset:0\n\t" \
  "s_cmp_lg_u32 s26, 1\n\t" \
  "s_cbranch_scc1 Lns1_1_%=\n\t" \
  "global_store_dwordx4 v63, v[72:75], s[28:29] sc0 sc1\n\t" \
  "global_store_dwordx4 v63, v[76:79], s[28:29] offset:1024 sc0 sc1\n\t" \
  "s_branch Lms1_1_%=\n\t" \
  "Lns1_1_%=:\n\t" \
  "buffer_load_dword v62, v65, s[20:23], s22 offen\n\t" \
  "buffer_load_dword v62, v65, s[20:23], s22 offen\n\t" \
  "Lms1_1_%=:\n\t" \
  "s_add_u32 m0, s24, 0x8000\n\t" \
  "s_add_u32 s48, s40, 0x0\n\t" \
  "buffer_load_dwordx4 v2, s[8:11], s48 offen lds\n\t" \
  "v_mfma_f32_16x16x128_f8f6f4 v[208:211], v[112:119], a[32:39], v[208:211]\n\t" \
  "ds_read_b64_tr_b8 v[110:111], v95 offset:41984\n\t" \
  "ds_read_b128 a[68:71], v21 offset:0\n\t" \
  "s_waitcnt lgkmcnt(0)\n\t" \
  "s_barrier\n\t" \
  "v_mfma_f32_16x16x128_f8f6f4 v[224:227], v[120:127], a[32:39], v[224:227]\n\t" \
  "ds_read_b64_tr_b8 v[112:113], v80 offset:16384\n\t" \
  "ds_read_b128 a[72:75], v20 offset:2048\n\t" \
  "s_add_u32 m0, s24, 0x8400\n\t" \
  "s_add_u32 s49, s40, 0x20000\n\t" \
  "buffer_load_dwordx4 v2, s[8:11], s49 offen lds\n\t" \
  "v_mfma_f32_16x16x128_f8f6f4 v[228:231], v[120:127], a[40:47], v[228:231]\n\t" \
  "ds_read_b64_tr_b8 v[114:115], v80 offset:17408\n\t" \
  "ds_read_b128 a[76:79], v21 offset:2048\n\t" \
  "v_mfma_f32_16x16x128_f8f6f4 v[232:235], v[120:127], a[48:55], v[232:235]\n\t" \
  "ds_read_b64_tr_b8 v[116:117], v88 offset:49152\n\t" \
  "ds_read_b128 a[80:83], v20 offset:4096\n\t" \
  "s_add_u32 m0, s24, 0xa000\n\t" \
  "s_add_u32 s50, s41, 0x0\n\t" \
  "buffer_load_dwordx4 v2, s[8:11], s50 offen lds\n\t" \
  "v_mfma_f32_16x16x128_f8f6f4 v[236:239], v[120:127], a[56:63], v[236:239]\n\t" \
  "ds_read_b64_tr_b8 v[118:119], v88 offset:50176\n\t" \
  "ds_read_b128 a[84:87], v21 offset:4096\n\t" \
  "v_mfma_f32_16x16x128_f8f6f4 v[252:255], v[104:111], a[56:63], v[252:255]\n\t" \
  "ds_read_b64_tr_b8 v[120:121], v81 offset:16384\n\t" \
  "ds_read_b128 a[88:91], v20 offset:6144\n\t" \
  "s_add_u32 m0, s24, 0xa400\n\t" \
  "s_add_u32 s51, s41, 0x20000\n\t" \
  "buffer_load_dwordx4 v2, s[8:11], s51 offen lds\n\t" \
  "v_mfma_f32_16x16x128_f8f6f4 v[248:251], v[104:111], a[48:55], v[248:251]\n\t" \
  "ds_read_b64_tr_b8 v[122:123], v81 offset:17408\n\t" \
  "ds_read_b128 a[92:95], v21 offset:6144\n\t" \
  "v_mfma_f32_16x16x128_f8f6f4 v[244:247], v[104:111], a[40:47], v[244:247]\n\t" \
  "ds_read_b64_tr_b8 v[124:125], v89 offset:49152\n\t" \
  "s_add_u32 m0, s24, 0xc000\n\t" \
  "s_add_u32 s52, s42, 0x0\n\t" \
  "buffer_load_dwordx4 v2, s[8:11], s52 offen lds\n\t" \
  "v_mfma_f32_16x16x128_f8f6f4 v[240:243], v[104:111], a[32:39], v[240:243]\n\t" \
  "ds_read_b64_tr_b8 v[126:127], v89 offset:50176\n\t" \
  "s_waitcnt lgkmcnt(2)\n\t" \
  "v_mfma_f32_16x16x128_f8f6f4 v[128:131], v[112:119], a[64:71], v[128:131]\n\t" \
  "ds_read_b64_tr_b8 v[104:105], v82 offset:16384\n\t" \
  "s_waitcnt vmcnt(7)\n\t" \
  "ds_read_b128 a[96:99], v22 offset:0\n\t" \
  "s_add_u32 m0, s24, 0xc400\n\t" \
  "s_add_u32 s53, s42, 0x20000\n\t" \
  "buffer_load_dwordx4 v2, s[8:11], s53 offen lds\n\t" \
  "v_mfma_f32_16x16x128_f8f6f4 v[132:135], v[112:119], a[72:79], v[132:135]\n\t" \
  "ds_read_b64_tr_b8 v[106:107], v82 offset:17408\n\t" \
  "ds_read_b128 a[100:103], v23 offset:0\n\t" \
  "v_mfma_f32_16x16x128_f8f6f4 v[136:139], v[112:119], a[80:87], v[136:139]\n\t" \
  "ds_read_b64_tr_b8 v[108:109], v90 offset:49152\n\t" \
  "ds_read_b128 a[104:107], v22 offset:2048\n\t" \
  "s_add_u32 m0, s24, 0xe000\n\t" \
  "s_add_u32 s54, s43, 0x0\n\t" \
  "buffer_load_dwordx4 v2, s[8:11], s54 offen lds\n\t" \
  "v_mfma_f32_16x16x128_f8f6f4 v[140:143], v[112:119], a[88:95], v[140:143]\n\t" \
  "ds_read_b64_tr_b8 v[110:111], v90 offset:50176\n\t" \
  "ds_read_b128 a[108:111], v23 offset:2048\n\t" \
  "s_waitcnt lgkmcnt(8)\n\t" \
  "v_mfma_f32_16x16x128_f8f6f4 v[156:159], v[120:127], a[88:95], v[156:159]\n\t" \
  "ds_read_b64_tr_b8 v[112:113], v83 offset:16384\n\t" \
  "ds_read_b128 a[112:115], v22 offset:4096\n\t" \
  "s_add_u32 m0, s24, 0xe400\n\t" \
  "s_add_u32 s55, s43, 0x20000\n\t" \
  "buffer_load_dwordx4 v2, s[8:11], s55 offen lds\n\t" \
  "v_mfma_f32_16x16x128_f8f6f4 v[152:155], v[120:127], a[80:87], v[152:155]\n\t" \
  "ds_read_b64_tr_b8 v[114:115], v83 offset:17408\n\t" \
  "ds_read_b128 a[116:119], v23 offset:4096\n\t" \
  "v_mfma_f32_16x16x128_f8f6f4 v[148:151], v[120:127], a[72:79], v[148:151]\n\t" \
  "ds_read_b64_tr_b8 v[116:117], v91 offset:49152\n\t" \
  "ds_read_b128 a[120:123], v22 offset:6144\n\t" \
  "v_mfma_f32_16x16x128_f8f6f4 v[144:147], v[120:127], a[64:71], v[144:147]\n\t" \
  "ds_read_b64_tr_b8 v[118:119], v91 offset:50176\n\t" \
  "s_waitcnt lgkmcnt(14)\n\t" \
  "ds_read_b128 a[124:127], v23 offset:6144\n\t" \
  "s_waitcnt lgkmcnt(9)\n\t" \
  "v_mfma_f32_16x16x128_f8f6f4 v[160:163], v[104:111], a[64:71], v[160:163]\n\t" \
  "ds_read_b64_tr_b8 v[120:121], v84 offset:16384\n\t" \
  "v_mfma_f32_16x16x128_f8f6f4 v[164:167], v[104:111], a[72:79], v[164:167]\n\t" \
  "ds_read_b64_tr_b8 v[122:123], v84 offset:17408\n\t" \
  "v_mfma_f32_16x16x128_f8f6f4 v[168:171], v[104:111], a[80:87], v[168:171]\n\t" \
  "ds_read_b64_tr_b8 v[124:125], v92 offset:49152\n\t" \
  "v_mfma_f32_16x16x128_f8f6f4 v[172:175], v[104:111], a[88:95], v[172:175]\n\t" \
  "ds_read_b64_tr_b8 v[126:127], v92 offset:50176\n\t" \
  "s_waitcnt lgkmcnt(5)\n\t" \
  "v_mfma_f32_16x16x128_f8f6f4 v[188:191], v[112:119], a[88:95], v[188:191]\n\t" \
  "ds_read_b64_tr_b8 v[104:105], v85 offset:16384\n\t" \
  "v_mfma_f32_16x16x128_f8f6f4 v[184:187], v[112:119], a[80:87], v[184:187]\n\t" \
  "ds_read_b64_tr_b8 v[106:107], v85 offset:17408\n\t" \
  "v_mfma_f32_16x16x128_f8f6f4 v[180:183], v[112:119], a[72:79], v[180:183]\n\t" \
  "ds_read_b64_tr_b8 v[108:109], v93 offset:49152\n\t" \
  "v_mfma_f32_16x16x128_f8f6f4 v[176:179], v[112:119], a[64:71], v[176:179]\n\t" \
  "ds_read_b64_tr_b8 v[110:111], v93 offset:50176\n\t" \
  "s_waitcnt lgkmcnt(4)\n\t" \
  "v_mfma_f32_16x16x128_f8f6f4 v[192:195], v[120:127], a[64:71], v[192:195]\n\t" \
  "ds_read_b64_tr_b8 v[112:113], v86 offset:16384\n\t" \
  "v_mfma_f32_16x16x128_f8f6f4 v[196:199], v[120:127], a[72:79], v[196:199]\n\t" \
  "ds_read_b64_tr_b8 v[114:115], v86 offset:17408\n\t" \
  "v_mfma_f32_16x16x128_f8f6f4 v[200:203], v[120:127], a[80:87], v[200:203]\n\t" \
  "ds_read_b64_tr_b8 v[116:117], v94 offset:49152\n\t" \
  "v_mfma_f32_16x16x128_f8f6f4 v[204:207], v[120:127], a[88:95], v[204:207]\n\t" \
  "ds_read_b64_tr_b8 v[118:119], v94 offset:50176\n\t" \
  "s_waitcnt lgkmcnt(4)\n\t" \
  "v_mfma_f32_16x16x128_f8f6f4 v[220:223], v[104:111], a[88:95], v[220:223]\n\t" \
  "ds_read_b64_tr_b8 v[120:121], v87 offset:16384\n\t" \
  "v_mfma_f32_16x16x128_f8f6f4 v[216:219], v[104:111], a[80:87], v[216:219]\n\t" \
  "ds_read_b64_tr_b8 v[122:123], v87 offset:17408\n\t" \
  "v_mfma_f32_16x16x128_f8f6f4 v[212:215], v[104:111], a[72:79], v[212:215]\n\t" \
  "ds_read_b64_tr_b8 v[124:125], v95 offset:49152\n\t" \
  "v_mfma_f32_16x16x128_f8f6f4 v[208:211], v[104:111], a[64:71], v[208:211]\n\t" \
  "ds_read_b64_tr_b8 v[126:127], v95 offset:50176\n\t" \
  "s_waitcnt lgkmcnt(0)\n\t" \
  "s_barrier\n\t" \
  "v_mfma_f32_16x16x128_f8f6f4 v[224:227], v[112:119], a[64:71], v[224:227]\n\t" \
  "ds_read_b64_tr_b8 v[104:105], v80 offset:24576\n\t" \
  "s_mov_b32 exec_lo, 0x7f\n\t" \
  "s_mov_b32 exec_hi, 0\n\t" \
  "global_load_dwordx4 v[68:71], v10, s[6:7] sc0 sc1\n\t" \
  "s_mov_b64 exec, -1\n\t" \
  "v_mfma_f32_16x16x128_f8f6f4 v[228:231], v[112:119], a[72:79], v[228:231]\n\t" \
  "ds_read_b64_tr_b8 v[106:107], v80 offset:25600\n\t" \
  "v_mfma_f32_16x16x128_f8f6f4 v[232:235], v[112:119], a[80:87], v[232:235]\n\t" \
  "ds_read_b64_tr_b8 v[108:109], v88 offset:57344\n\t" \
  "s_add_u32 m0, s24, 0x20000\n\t" \
  "s_add_u32 s48, s40, 0x0\n\t" \
  "buffer_load_dwordx4 v3, s[8:11], s48 offen lds\n\t" \
  "v_mfma_f32_16x16x128_f8f6f4 v[236:239], v[112:119], a[88:95], v[236:239]\n\t" \
  "ds_read_b64_tr_b8 v[110:111], v88 offset:58368\n\t" \
  "v_mfma_f32_16x16x128_f8f6f4 v[252:255], v[120:127], a[88:95], v[252:255]\n\t" \
  "ds_read_b64_tr_b8 v[112:113], v81 offset:24576\n\t" \
  "s_add_u32 m0, s24, 0x20400\n\t" \
  "s_add_u32 s49, s40, 0x20000\n\t" \
  "buffer_load_dwordx4 v3, s[8:11], s49 offen lds\n\t" \
  "v_mfma_f32_16x16x128_f8f6f4 v[248:251], v[120:127], a[80:87], v[248:251]\n\t" \
  "ds_read_b64_tr_b8 v[114:115], v81 offset:25600\n\t" \
  "v_mfma_f32_16x16x128_f8f6f4 v[244:247], v[120:127], a[72:79], v[244:247]\n\t" \
  "ds_read_b64_tr_b8 v[116:117], v89 offset:57344\n\t" \
  "s_add_u32 m0, s24, 0x22000\n\t" \
  "s_add_u32 s50, s41, 0x0\n\t" \
  "buffer_load_dwordx4 v3, s[8:11], s50 offen lds\n\t" \
  "v_mfma_f32_16x16x128_f8f6f4 v[240:243], v[120:127], a[64:71], v[240:243]\n\t" \
  "ds_read_b64_tr_b8 v[118:119], v89 offset:58368\n\t" \
  "s_waitcnt lgkmcnt(4)\n\t" \
  "v_mfma_f32_16x16x128_f8f6f4 v[128:131], v[104:111], a[96:103], v[128:131]\n\t" \
  "ds_read_b64_tr_b8 v[120:121], v82 offset:24576\n\t" \
  "s_add_u32 m0, s24, 0x22400\n\t" \
  "s_add_u32 s51, s41, 0x20000\n\t" \
  "buffer_load_dwordx4 v3, s[8:11], s51 offen lds\n\t" \
  "v_mfma_f32_16x16x128_f8f6f4 v[132:135], v[104:111], a[104:111], v[132:135]\n\t" \
  "ds_read_b64_tr_b8 v[122:123], v82 offset:25600\n\t" \
  "v_mfma_f32_16x16x128_f8f6f4 v[136:139], v[104:111], a[112:119], v[136:139]\n\t" \
  "ds_read_b64_tr_b8 v[124:125], v90 offset:57344\n\t" \
  "s_waitcnt vmcnt(4)\n\t" \
  "v_min_u32 v68, v68, v69\n\t" \
  "v_min3_u32 v68, v68, v70, v71\n\t" \
  "v_cmp_le_u32 vcc, s57, v68\n\t" \
  "s_and_b32 s64, vcc_lo, 0x7f\n\t" \
  "s_or_b32 s56, s56, s64\n\t" \
  "s_and_b32 s64, s56, 0x1\n\t" \
  "s_cmp_eq_u32 s64, 0x1\n\t" \
  "s_cbranch_scc1 Lok0_1_%=\n\t" \
  "s_mov_b32 s62, 0x1\n\t" \
  "s_mov_b32 s69, 0x7f\n\t" \
  "s_call_b64 s[60:61], Lpoll_1_%=\n\t" \
  "Lok0_1_%=:\n\t" \
  "s_add_u32 s52, s68, 0x40000\n\t" \
  "buffer_load_dwordx4 a[128:131], v4, s[12:15], s52 offen offset:0 sc0 sc1\n\t" \
  "v_mfma_f32_16x16x128_f8f6f4 v[140:143], v[104:111], a[120:127], v[140:143]\n\t" \
  "ds_read_b64_tr_b8 v[126:127], v90 offset:58368\n\t" \
  "s_waitcnt lgkmcnt(4)\n\t" \
  "v_mfma_f32_16x16x128_f8f6f4 v[156:159], v[112:119], a[120:127], v[156:159]\n\t" \
  "ds_read_b64_tr_b8 v[104:105], v83 offset:24576\n\t" \
  "s_add_u32 s53, s68, 0x40000\n\t" \
  "buffer_load_dwordx4 a[132:135], v4, s[12:15], s53 offen offset:1024 sc0 sc1\n\t" \
  "v_mfma_f32_16x16x128_f8f6f4 v[152:155], v[112:119], a[112:119], v[152:155]\n\t" \
  "ds_read_b64_tr_b8 v[106:107], v83 offset:25600\n\t" \
  "v_mfma_f32_16x16x128_f8f6f4 v[148:151], v[112:119], a[104:111], v[148:151]\n\t" \
  "ds_read_b64_tr_b8 v[108:109], v91 offset:57344\n\t" \
  "s_add_u32 s54, s68, 0x40000\n\t" \
  "buffer_load_dwordx4 a[136:139], v4, s[12:15], s54 offen offset:2048 sc0 sc1\n\t" \
  "v_mfma_f32_16x16x128_f8f6f4 v[144:147], v[112:119], a[96:103], v[144:147]\n\t" \
  "ds_read_b64_tr_b8 v[110:111], v91 offset:58368\n\t" \
  "s_waitcnt lgkmcnt(4)\n\t" \
  "v_mfma_f32_16x16x128_f8f6f4 v[160:163], v[120:127], a[96:103], v[160:163]\n\t" \
  "ds_read_b64_tr_b8 v[112:113], v84 offset:24576\n\t" \
  "s_add_u32 s55, s68, 0x40000\n\t" \
  "buffer_load_dwordx4 a[140:143], v4, s[12:15], s55 offen offset:3072 sc0 sc1\n\t" \
  "v_mfma_f32_16x16x128_f8f6f4 v[164:167], v[120:127], a[104:111], v[164:167]\n\t" \
  "ds_read_b64_tr_b8 v[114:115], v84 offset:25600\n\t" \
  "v_mfma_f32_16x16x128_f8f6f4 v[168:171], v[120:127], a[112:119], v[168:171]\n\t" \
  "ds_read_b64_tr_b8 v[116:117], v92 offset:57344\n\t" \
  "s_add_u32 s48, s68, 0x41000\n\t" \
  "buffer_load_dwordx4 a[144:147], v4, s[12:15], s48 offen offset:0 sc0 sc1\n\t" \
  "v_mfma_f32_16x16x128_f8f6f4 v[172:175], v[120:127], a[120:127], v[172:175]\n\t" \
  "ds_read_b64_tr_b8 v[118:119], v92 offset:58368\n\t" \
  "s_waitcnt lgkmcnt(4)\n\t" \
  "v_mfma_f32_16x16x128_f8f6f4 v[188:191], v[104:111], a[120:127], v[188:191]\n\t" \
  "ds_read_b64_tr_b8 v[120:121], v85 offset:24576\n\t" \
  "s_add_u32 s49, s68, 0x41000\n\t" \
  "buffer_load_dwordx4 a[148:151], v4, s[12:15], s49 offen offset:1024 sc0 sc1\n\t" \
  "v_mfma_f32_16x16x128_f8f6f4 v[184:187], v[104:111], a[112:119], v[184:187]\n\t" \
  "ds_read_b64_tr_b8 v[122:123], v85 offset:25600\n\t" \
  "v_mfma_f32_16x16x128_f8f6f4 v[180:183], v[104:111], a[104:111], v[180:183]\n\t" \
  "ds_read_b64_tr_b8 v[124:125], v93 offset:57344\n\t" \
  "s_add_u32 s50, s68, 0x41000\n\t" \
  "buffer_load_dwordx4 a[152:155], v4, s[12:15], s50 offen offset:2048 sc0 sc1\n\t" \
  "v_mfma_f32_16x16x128_f8f6f4 v[176:179], v[104:111], a[96:103], v[176:179]\n\t" \
  "ds_read_b64_tr_b8 v[126:127], v93 offset:58368\n\t" \
  "s_waitcnt lgkmcnt(4)\n\t" \
  "v_mfma_f32_16x16x128_f8f6f4 v[192:195], v[112:119], a[96:103], v[192:195]\n\t" \
  "ds_read_b64_tr_b8 v[104:105], v86 offset:24576\n\t" \
  "s_add_u32 s51, s68, 0x41000\n\t" \
  "buffer_load_dwordx4 a[156:159], v4, s[12:15], s51 offen offset:3072 sc0 sc1\n\t" \
  "v_mfma_f32_16x16x128_f8f6f4 v[196:199], v[112:119], a[104:111], v[196:199]\n\t" \
  "ds_read_b64_tr_b8 v[106:107], v86 offset:25600\n\t" \
  "v_mfma_f32_16x16x128_f8f6f4 v[200:203], v[112:119], a[112:119], v[200:203]\n\t" \
  "ds_read_b64_tr_b8 v[108:109], v94 offset:57344\n\t" \
  "s_and_b32 s64, s56, 0x2\n\t" \
  "s_cmp_eq_u32 s64, 0x2\n\t" \
  "s_cbranch_scc1 Lok1_1_%=\n\t" \
  "s_mov_b32 s62, 0x2\n\t" \
  "s_mov_b32 s69, 0x7f\n\t" \
  "s_call_b64 s[60:61], Lpoll_1_%=\n\t" \
  "Lok1_1_%=:\n\t" \
  "s_add_u32 s52, s68, 0x50000\n\t" \
  "buffer_load_dwordx4 a[160:163], v4, s[12:15], s52 offen offset:0 sc0 sc1\n\t" \
  "v_mfma_f32_16x16x128_f8f6f4 v[204:207], v[112:119], a[120:127], v[204:207]\n\t" \
  "ds_read_b64_tr_b8 v[110:111], v94 offset:58368\n\t" \
  "s_waitcnt lgkmcnt(4)\n\t" \
  "v_mfma_f32_16x16x128_f8f6f4 v[220:223], v[120:127], a[120:127], v[220:223]\n\t" \
  "ds_read_b64_tr_b8 v[112:113], v87 offset:24576\n\t" \
  "s_add_u32 s53, s68, 0x50000\n\t" \
  "buffer_load_dwordx4 a[164:167], v4, s[12:15], s53 offen offset:1024 sc0 sc1\n\t" \
  "v_mfma_f32_16x16x128_f8f6f4 v[216:219], v[120:127], a[112:119], v[216:219]\n\t" \
  "ds_read_b64_tr_b8 v[114:115], v87 offset:25600\n\t" \
  "v_mfma_f32_16x16x128_f8f6f4 v[212:215], v[120:127], a[104:111], v[212:215]\n\t" \
  "ds_read_b64_tr_b8 v[116:117], v95 offset:57344\n\t" \
  "s_add_u32 s54, s68, 0x50000\n\t" \
  "buffer_load_dwordx4 a[168:171], v4, s[12:15], s54 offen offset:2048 sc0 sc1\n\t" \
  "v_mfma_f32_16x16x128_f8f6f4 v[208:211], v[120:127], a[96:103], v[208:211]\n\t" \
  "ds_read_b64_tr_b8 v[118:119], v95 offset:58368\n\t" \
  "s_waitcnt vmcnt(13) lgkmcnt(0)\n\t" \
  "s_barrier\n\t" \
  "v_mfma_f32_16x16x128_f8f6f4 v[224:227], v[104:111], a[96:103], v[224:227]\n\t" \
  "ds_read_b64_tr_b8 v[120:121], v80 offset:32768\n\t" \
  "s_add_u32 s55, s68, 0x50000\n\t" \
  "buffer_load_dwordx4 a[172:175], v4, s[12:15], s55 offen offset:3072 sc0 sc1\n\t" \
  "v_mfma_f32_16x16x128_f8f6f4 v[228:231], v[104:111], a[104:111], v[228:231]\n\t" \
  "ds_read_b64_tr_b8 v[122:123], v80 offset:33792\n\t" \
  "v_mfma_f32_16x16x128_f8f6f4 v[232:235], v[104:111], a[112:119], v[232:235]\n\t" \
  "ds_read_b64_tr_b8 v[124:125], v96 offset:0\n\t" \
  "s_add_u32 s48, s68, 0x51000\n\t" \
  "buffer_load_dwordx4 a[176:179], v4, s[12:15], s48 offen offset:0 sc0 sc1\n\t" \
  "v_mfma_f32_16x16x128_f8f6f4 v[236:239], v[104:111], a[120:127], v[236:239]\n\t" \
  "ds_read_b64_tr_b8 v[126:127], v96 offset:1024\n\t" \
  "v_mfma_f32_16x16x128_f8f6f4 v[252:255], v[112:119], a[120:127], v[252:255]\n\t" \
  "ds_read_b64_tr_b8 v[104:105], v81 offset:32768\n\t" \
  "s_add_u32 s49, s68, 0x51000\n\t" \
  "buffer_load_dwordx4 a[180:183], v4, s[12:15], s49 offen offset:1024 sc0 sc1\n\t" \
  "v_mfma_f32_16x16x128_f8f6f4 v[248:251], v[112:119], a[112:119], v[248:251]\n\t" \
  "ds_read_b64_tr_b8 v[106:107], v81 offset:33792\n\t" \
  "v_mfma_f32_16x16x128_f8f6f4 v[244:247], v[112:119], a[104:111], v[244:247]\n\t" \
  "ds_read_b64_tr_b8 v[108:109], v97 offset:0\n\t" \
  "s_add_u32 s50, s68, 0x51000\n\t" \
  "buffer_load_dwordx4 a[184:187], v4, s[12:15], s50 offen offset:2048 sc0 sc1\n\t" \
  "v_mfma_f32_16x16x128_f8f6f4 v[240:243], v[112:119], a[96:103], v[240:243]\n\t" \
  "ds_read_b64_tr_b8 v[110:111], v97 offset:1024\n\t" \
  "s_waitcnt vmcnt(7) lgkmcnt(4)\n\t" \
  "v_mfma_f32_16x16x128_f8f6f4 v[128:131], v[120:127], a[128:135], v[128:131]\n\t" \
  "ds_read_b64_tr_b8 v[112:113], v82 offset:32768\n\t" \
  "s_add_u32 s51, s68, 0x51000\n\t" \
  "buffer_load_dwordx4 a[188:191], v4, s[12:15], s51 offen offset:3072 sc0 sc1\n\t" \
  "v_mfma_f32_16x16x128_f8f6f4 v[132:135], v[120:127], a[136:143], v[132:135]\n\t" \
  "ds_read_b64_tr_b8 v[114:115], v82 offset:33792\n\t" \
  "v_mfma_f32_16x16x128_f8f6f4 v[136:139], v[120:127], a[144:151], v[136:139]\n\t" \
  "ds_read_b64_tr_b8 v[116:117], v98 offset:0\n\t" \
  "s_cmp_lg_u32 s26, 1\n\t" \
  "s_cbranch_scc1 Lnf1_1_%=\n\t" \
  "s_mov_b64 exec, 1\n\t" \
  "global_store_dword v65, v64, s[30:31] sc0 sc1\n\t" \
  "s_mov_b64 exec, -1\n\t" \
  "s_branch Lmf1_1_%=\n\t" \
  "Lnf1_1_%=:\n\t" \
  "buffer_load_dword v62, v65, s[20:23], s22 offen\n\t" \
  "Lmf1_1_%=:\n\t" \
  "s_cmp_lg_u32 s26, 2\n\t" \
  "s_cbranch_scc1 Lns2_1_%=\n\t" \
  "global_store_dwordx4 v63, v[72:75], s[28:29] sc0 sc1\n\t" \
  "global_store_dwordx4 v63, v[76:79], s[28:29] offset:1024 sc0 sc1\n\t" \
  "s_branch Lms2_1_%=\n\t" \
  "Lns2_1_%=:\n\t" \
  "buffer_load_dword v62, v65, s[20:23], s22 offen\n\t" \
  "buffer_load_dword v62, v65, s[20:23], s22 offen\n\t" \
  "Lms2_1_%=:\n\t" \
  "s_add_u32 m0, s24, 0x10000\n\t" \
  "s_add_u32 s52, s44, 0x0\n\t" \
  "buffer_load_dwordx4 v2, s[8:11], s52 offen lds\n\t" \
  "v_mfma_f32_16x16x128_f8f6f4 v[140:143], v[120:127], a[152:159], v[140:143]\n\t" \
  "ds_read_b64_tr_b8 v[118:119], v98 offset:1024\n\t" \
  "s_waitcnt lgkmcnt(4)\n\t" \
  "v_mfma_f32_16x16x128_f8f6f4 v[156:159], v[104:111], a[152:159], v[156:159]\n\t" \
  "ds_read_b64_tr_b8 v[120:121], v83 offset:32768\n\t" \
  "s_add_u32 m0, s24, 0x10400\n\t" \
  "s_add_u32 s53, s44, 0x20000\n\t" \
  "buffer_load_dwordx4 v2, s[8:11], s53 offen lds\n\t" \
  "v_mfma_f32_16x16x128_f8f6f4 v[152:155], v[104:111], a[144:151], v[152:155]\n\t" \
  "ds_read_b64_tr_b8 v[122:123], v83 offset:33792\n\t" \
  "v_mfma_f32_16x16x128_f8f6f4 v[148:151], v[104:111], a[136:143], v[148:151]\n\t" \
  "ds_read_b64_tr_b8 v[124:125], v99 offset:0\n\t" \
  "s_add_u32 m0, s24, 0x12000\n\t" \
  "s_add_u32 s54, s45, 0x0\n\t" \
  "buffer_load_dwordx4 v2, s[8:11], s54 offen lds\n\t" \
  "v_mfma_f32_16x16x128_f8f6f4 v[144:147], v[104:111], a[128:135], v[144:147]\n\t" \
  "ds_read_b64_tr_b8 v[126:127], v99 offset:1024\n\t" \
  "s_waitcnt lgkmcnt(4)\n\t" \
  "v_mfma_f32_16x16x128_f8f6f4 v[160:163], v[112:119], a[128:135], v[160:163]\n\t" \
  "ds_read_b64_tr_b8 v[104:105], v84 offset:32768\n\t" \
  "s_add_u32 m0, s24, 0x12400\n\t" \
  "s_add_u32 s55, s45, 0x20000\n\t" \
  "buffer_load_dwordx4 v2, s[8:11], s55 offen lds\n\t" \
  "v_mfma_f32_16x16x128_f8f6f4 v[164:167], v[112:119], a[136:143], v[164:167]\n\t" \
  "ds_read_b64_tr_b8 v[106:107], v84 offset:33792\n\t" \
  "v_mfma_f32_16x16x128_f8f6f4 v[168:171], v[112:119], a[144:151], v[168:171]\n\t" \
  "ds_read_b64_tr_b8 v[108:109], v100 offset:0\n\t" \
  "s_add_u32 m0, s24, 0x14000\n\t" \
  "s_add_u32 s48, s46, 0x0\n\t" \
  "buffer_load_dwordx4 v2, s[8:11], s48 offen lds\n\t" \
  "v_mfma_f32_16x16x128_f8f6f4 v[172:175], v[112:119], a[152:159], v[172:175]\n\t" \
  "ds_read_b64_tr_b8 v[110:111], v100 offset:1024\n\t" \
  "s_waitcnt lgkmcnt(4)\n\t" \
  "v_mfma_f32_16x16x128_f8f6f4 v[188:191], v[120:127], a[152:159], v[188:191]\n\t" \
  "ds_read_b64_tr_b8 v[112:113], v85 offset:32768\n\t" \
  "s_add_u32 m0, s24, 0x14400\n\t" \
  "s_add_u32 s49, s46, 0x20000\n\t" \
  "buffer_load_dwordx4 v2, s[8:11], s49 offen lds\n\t" \
  "v_mfma_f32_16x16x128_f8f6f4 v[184:187], v[120:127], a[144:151], v[184:187]\n\t" \
  "ds_read_b64_tr_b8 v[114:115], v85 offset:33792\n\t" \
  "v_mfma_f32_16x16x128_f8f6f4 v[180:183], v[120:127], a[136:143], v[180:183]\n\t" \
  "ds_read_b64_tr_b8 v[116:117], v101 offset:0\n\t" \
  "s_add_u32 m0, s24, 0x16000\n\t" \
  "s_add_u32 s50, s40, 0x200000\n\t" \
  "buffer_load_dwordx4 v2, s[8:11], s50 offen lds\n\t" \
  "v_mfma_f32_16x16x128_f8f6f4 v[176:179], v[120:127], a[128:135], v[176:179]\n\t" \
  "ds_read_b64_tr_b8 v[118:119], v101 offset:1024\n\t" \
  "s_waitcnt lgkmcnt(4)\n\t" \
  "v_mfma_f32_16x16x128_f8f6f4 v[192:195], v[104:111], a[128:135], v[192:195]\n\t" \
  "ds_read_b64_tr_b8 v[120:121], v86 offset:32768\n\t" \
  "s_add_u32 m0, s24, 0x16400\n\t" \
  "s_add_u32 s51, s40, 0x220000\n\t" \
  "buffer_load_dwordx4 v2, s[8:11], s51 offen lds\n\t" \
  "v_mfma_f32_16x16x128_f8f6f4 v[196:199], v[104:111], a[136:143], v[196:199]\n\t" \
  "ds_read_b64_tr_b8 v[122:123], v86 offset:33792\n\t" \
  "v_mfma_f32_16x16x128_f8f6f4 v[200:203], v[104:111], a[144:151], v[200:203]\n\t" \
  "ds_read_b64_tr_b8 v[124:125], v102 offset:0\n\t" \
  "v_mfma_f32_16x16x128_f8f6f4 v[204:207], v[104:111], a[152:159], v[204:207]\n\t" \
  "ds_read_b64_tr_b8 v[126:127], v102 offset:1024\n\t" \
  "s_waitcnt lgkmcnt(4)\n\t" \
  "v_mfma_f32_16x16x128_f8f6f4 v[220:223], v[112:119], a[152:159], v[220:223]\n\t" \
  "ds_read_b64_tr_b8 v[104:105], v87 offset:32768\n\t" \
  "v_mfma_f32_16x16x128_f8f6f4 v[216:219], v[112:119], a[144:151], v[216:219]\n\t" \
  "ds_read_b64_tr_b8 v[106:107], v87 offset:33792\n\t" \
  "v_mfma_f32_16x16x128_f8f6f4 v[212:215], v[112:119], a[136:143], v[212:215]\n\t" \
  "ds_read_b64_tr_b8 v[108:109], v103 offset:0\n\t" \
  "v_mfma_f32_16x16x128_f8f6f4 v[208:211], v[112:119], a[128:135], v[208:211]\n\t" \
  "ds_read_b64_tr_b8 v[110:111], v103 offset:1024\n\t" \
  "s_waitcnt lgkmcnt(0)\n\t" \
  "s_barrier\n\t" \
  "v_mfma_f32_16x16x128_f8f6f4 v[224:227], v[120:127], a[128:135], v[224:227]\n\t" \
  "ds_read_b64_tr_b8 v[112:113], v80 offset:40960\n\t" \
  "v_mfma_f32_16x16x128_f8f6f4 v[228:231], v[120:127], a[136:143], v[228:231]\n\t" \
  "ds_read_b64_tr_b8 v[114:115], v80 offset:41984\n\t" \
  "v_mfma_f32_16x16x128_f8f6f4 v[232:235], v[120:127], a[144:151], v[232:235]\n\t" \
  "ds_read_b64_tr_b8 v[116:117], v96 offset:8192\n\t" \
  "v_mfma_f32_16x16x128_f8f6f4 v[236:239], v[120:127], a[152:159], v[236:239]\n\t" \
  "ds_read_b64_tr_b8 v[118:119], v96 offset:9216\n\t" \
  "v_mfma_f32_16x16x128_f8f6f4 v[252:255], v[104:111], a[152:159], v[252:255]\n\t" \
  "ds_read_b64_tr_b8 v[120:121], v81 offset:40960\n\t" \
  "v_mfma_f32_16x16x128_f8f6f4 v[248:251], v[104:111], a[144:151], v[248:251]\n\t" \
  "ds_read_b64_tr_b8 v[122:123], v81 offset:41984\n\t" \
  "v_mfma_f32_16x16x128_f8f6f4 v[244:247], v[104:111], a[136:143], v[244:247]\n\t" \
  "ds_read_b64_tr_b8 v[124:125], v97 offset:8192\n\t" \
  "v_mfma_f32_16x16x128_f8f6f4 v[240:243], v[104:111], a[128:135], v[240:243]\n\t" \
  "ds_read_b64_tr_b8 v[126:127], v97 offset:9216\n\t" \
  "s_waitcnt vmcnt(11) lgkmcnt(4)\n\t" \
  "v_mfma_f32_16x16x128_f8f6f4 v[128:131], v[112:119], a[160:167], v[128:131]\n\t" \
  "ds_read_b64_tr_b8 v[104:105], v82 offset:40960\n\t" \
  "v_mfma_f32_16x16x128_f8f6f4 v[132:135], v[112:119], a[168:175], v[132:135]\n\t" \
  "ds_read_b64_tr_b8 v[106:107], v82 offset:41984\n\t" \
  "v_mfma_f32_16x16x128_f8f6f4 v[136:139], v[112:119], a[176:183], v[136:139]\n\t" \
  "ds_read_b64_tr_b8 v[108:109], v98 offset:8192\n\t" \
  "v_mfma_f32_16x16x128_f8f6f4 v[140:143], v[112:119], a[184:191], v[140:143]\n\t" \
  "ds_read_b64_tr_b8 v[110:111], v98 offset:9216\n\t" \
  "s_waitcnt lgkmcnt(4)\n\t" \
  "v_mfma_f32_16x16x128_f8f6f4 v[156:159], v[120:127], a[184:191], v[156:159]\n\t" \
  "ds_read_b64_tr_b8 v[112:113], v83 offset:40960\n\t" \
  "v_mfma_f32_16x16x128_f8f6f4 v[152:155], v[120:127], a[176:183], v[152:155]\n\t" \
  "ds_read_b64_tr_b8 v[114:115], v83 offset:41984\n\t" \
  "v_mfma_f32_16x16x128_f8f6f4 v[148:151], v[120:127], a[168:175], v[148:151]\n\t" \
  "ds_read_b64_tr_b8 v[116:117], v99 offset:8192\n\t" \
  "s_add_u32 m0, s24, 0x24000\n\t" \
  "s_add_u32 s52, s42, 0x0\n\t" \
  "buffer_load_dwordx4 v3, s[8:11], s52 offen lds\n\t" \
  "v_mfma_f32_16x16x128_f8f6f4 v[144:147], v[120:127], a[160:167], v[144:147]\n\t" \
  "ds_read_b64_tr_b8 v[118:119], v99 offset:9216\n\t" \
  "s_waitcnt lgkmcnt(4)\n\t" \
  "v_mfma_f32_16x16x128_f8f6f4 v[160:163], v[104:111], a[160:167], v[160:163]\n\t" \
  "ds_read_b64_tr_b8 v[120:121], v84 offset:40960\n\t" \
  "s_add_u32 m0, s24, 0x24400\n\t" \
  "s_add_u32 s53, s42, 0x20000\n\t" \
  "buffer_load_dwordx4 v3, s[8:11], s53 offen lds\n\t" \
  "v_mfma_f32_16x16x128_f8f6f4 v[164:167], v[104:111], a[168:175], v[164:167]\n\t" \
  "ds_read_b64_tr_b8 v[122:123], v84 offset:41984\n\t" \
  "v_mfma_f32_16x16x128_f8f6f4 v[168:171], v[104:111], a[176:183], v[168:171]\n\t" \
  "ds_read_b64_tr_b8 v[124:125], v100 offset:8192\n\t" \
  "s_and_b32 s64, s56, 0x4\n\t" \
  "s_cmp_eq_u32 s64, 0x4\n\t" \
  "s_cbranch_scc1 Lok2_1_%=\n\t" \
  "s_mov_b32 s62, 0x4\n\t" \
  "s_mov_b32 s69, 0x7f\n\t" \
  "s_call_b64 s[60:61], Lpoll_1_%=\n\t" \
  "Lok2_1_%=:\n\t" \
  "s_add_u32 s54, s68, 0x60000\n\t" \
  "buffer_load_dwordx4 a[192:195], v4, s[12:15], s54 offen offset:0 sc0 sc1\n\t" \
  "v_mfma_f32_16x16x128_f8f6f4 v[172:175], v[104:111], a[184:191], v[172:175]\n\t" \
  "ds_read_b64_tr_b8 v[126:127], v100 offset:9216\n\t" \
  "s_waitcnt lgkmcnt(4)\n\t" \
  "v_mfma_f32_16x16x128_f8f6f4 v[188:191], v[112:119], a[184:191], v[188:191]\n\t" \
  "ds_read_b64_tr_b8 v[104:105], v85 offset:40960\n\t" \
  "s_add_u32 s55, s68, 0x60000\n\t" \
  "buffer_load_dwordx4 a[196:199], v4, s[12:15], s55 offen offset:1024 sc0 sc1\n\t" \
  "v_mfma_f32_16x16x128_f8f6f4 v[184:187], v[112:119], a[176:183], v[184:187]\n\t" \
  "ds_read_b64_tr_b8 v[106:107], v85 offset:41984\n\t" \
  "v_mfma_f32_16x16x128_f8f6f4 v[180:183], v[112:119], a[168:175], v[180:183]\n\t" \
  "ds_read_b64_tr_b8 v[108:109], v101 offset:8192\n\t" \
  "s_add_u32 s48, s68, 0x60000\n\t" \
  "buffer_load_dwordx4 a[200:203], v4, s[12:15], s48 offen offset:2048 sc0 sc1\n\t" \
  "v_mfma_f32_16x16x128_f8f6f4 v[176:179], v[112:119], a[160:167], v[176:179]\n\t" \
  "ds_read_b64_tr_b8 v[110:111], v101 offset:9216\n\t" \
  "s_waitcnt lgkmcnt(4)\n\t" \
  "v_mfma_f32_16x16x128_f8f6f4 v[192:195], v[120:127], a[160:167], v[192:195]\n\t" \
  "ds_read_b64_tr_b8 v[112:113], v86 offset:40960\n\t" \
  "s_add_u32 s49, s68, 0x60000\n\t" \
  "buffer_load_dwordx4 a[204:207], v4, s[12:15], s49 offen offset:3072 sc0 sc1\n\t" \
  "v_mfma_f32_16x16x128_f8f6f4 v[196:199], v[120:127], a[168:175], v[196:199]\n\t" \
  "ds_read_b64_tr_b8 v[114:115], v86 offset:41984\n\t" \
  "v_mfma_f32_16x16x128_f8f6f4 v[200:203], v[120:127], a[176:183], v[200:203]\n\t" \
  "ds_read_b64_tr_b8 v[116:117], v102 offset:8192\n\t" \
  "s_add_u32 s50, s68, 0x61000\n\t" \
  "buffer_load_dwordx4 a[208:211], v4, s[12:15], s50 offen offset:0 sc0 sc1\n\t" \
  "v_mfma_f32_16x16x128_f8f6f4 v[204:207], v[120:127], a[184:191], v[204:207]\n\t" \
  "ds_read_b64_tr_b8 v[118:119], v102 offset:9216\n\t" \
  "s_waitcnt lgkmcnt(4)\n\t" \
  "v_mfma_f32_16x16x128_f8f6f4 v[220:223], v[104:111], a[184:191], v[220:223]\n\t" \
  "ds_read_b64_tr_b8 v[120:121], v87 offset:40960\n\t" \
  "s_add_u32 s51, s68, 0x61000\n\t" \
  "buffer_load_dwordx4 a[212:215], v4, s[12:15], s51 offen offset:1024 sc0 sc1\n\t" \
  "v_mfma_f32_16x16x128_f8f6f4 v[216:219], v[104:111], a[176:183], v[216:219]\n\t" \
  "ds_read_b64_tr_b8 v[122:123], v87 offset:41984\n\t" \
  "v_mfma_f32_16x16x128_f8f6f4 v[212:215], v[104:111], a[168:175], v[212:215]\n\t" \
  "ds_read_b64_tr_b8 v[124:125], v103 offset:8192\n\t" \
  "s_add_u32 s52, s68, 0x61000\n\t" \
  "buffer_load_dwordx4 a[216:219], v4, s[12:15], s52 offen offset:2048 sc0 sc1\n\t" \
  "v_mfma_f32_16x16x128_f8f6f4 v[208:211], v[104:111], a[160:167], v[208:211]\n\t" \
  "ds_read_b64_tr_b8 v[126:127], v103 offset:9216\n\t" \
  "s_waitcnt vmcnt(7) lgkmcnt(0)\n\t" \
  "s_barrier\n\t" \
  "v_mfma_f32_16x16x128_f8f6f4 v[224:227], v[112:119], a[160:167], v[224:227]\n\t" \
  "ds_read_b64_tr_b8 v[104:105], v80 offset:49152\n\t" \
  "s_add_u32 s53, s68, 0x61000\n\t" \
  "buffer_load_dwordx4 a[220:223], v4, s[12:15], s53 offen offset:3072 sc0 sc1\n\t" \
  "v_mfma_f32_16x16x128_f8f6f4 v[228:231], v[112:119], a[168:175], v[228:231]\n\t" \
  "ds_read_b64_tr_b8 v[106:107], v80 offset:50176\n\t" \
  "v_mfma_f32_16x16x128_f8f6f4 v[232:235], v[112:119], a[176:183], v[232:235]\n\t" \
  "ds_read_b64_tr_b8 v[108:109], v96 offset:16384\n\t" \
  "s_add_u32 m0, s24, 0x26000\n\t" \
  "s_add_u32 s54, s43, 0x0\n\t" \
  "buffer_load_dwordx4 v3, s[8:11], s54 offen lds\n\t" \
  "v_mfma_f32_16x16x128_f8f6f4 v[236:239], v[112:119], a[184:191], v[236:239]\n\t" \
  "ds_read_b64_tr_b8 v[110:111], v96 offset:17408\n\t" \
  "v_mfma_f32_16x16x128_f8f6f4 v[252:255], v[120:127], a[184:191], v[252:255]\n\t" \
  "ds_read_b64_tr_b8 v[112:113], v81 offset:49152\n\t" \
  "s_add_u32 m0, s24, 0x26400\n\t" \
  "s_add_u32 s55, s43, 0x20000\n\t" \
  "buffer_load_dwordx4 v3, s[8:11], s55 offen lds\n\t" \
  "v_mfma_f32_16x16x128_f8f6f4 v[248:251], v[120:127], a[176:183], v[248:251]\n\t" \
  "ds_read_b64_tr_b8 v[114:115], v81 offset:50176\n\t" \
  "v_mfma_f32_16x16x128_f8f6f4 v[244:247], v[120:127], a[168:175], v[244:247]\n\t" \
  "ds_read_b64_tr_b8 v[116:117], v97 offset:16384\n\t" \
  "s_and_b32 s64, s56, 0x8\n\t" \
  "s_cmp_eq_u32 s64, 0x8\n\t" \
  "s_cbranch_scc1 Lok3_1_%=\n\t" \
  "s_mov_b32 s62, 0x8\n\t" \
  "s_mov_b32 s69, 0x7f\n\t" \
  "s_call_b64 s[60:61], Lpoll_1_%=\n\t" \
  "Lok3_1_%=:\n\t" \
  "s_add_u32 s48, s68, 0x70000\n\t" \
  "buffer_load_dwordx4 a[224:227], v4, s[12:15], s48 offen offset:0 sc0 sc1\n\t" \
  "v_mfma_f32_16x16x128_f8f6f4 v[240:243], v[120:127], a[160:167], v[240:243]\n\t" \
  "ds_read_b64_tr_b8 v[118:119], v97 offset:17408\n\t" \
  "s_waitcnt vmcnt(3) lgkmcnt(4)\n\t" \
  "v_mfma_f32_16x16x128_f8f6f4 v[128:131], v[104:111], a[192:199], v[128:131]\n\t" \
  "ds_read_b64_tr_b8 v[120:121], v82 offset:49152\n\t" \
  "s_add_u32 s49, s68, 0x70000\n\t" \
  "buffer_load_dwordx4 a[228:231], v4, s[12:15], s49 offen offset:1024 sc0 sc1\n\t" \
  "v_mfma_f32_16x16x128_f8f6f4 v[132:135], v[104:111], a[200:207], v[132:135]\n\t" \
  "ds_read_b64_tr_b8 v[122:123], v82 offset:50176\n\t" \
  "v_mfma_f32_16x16x128_f8f6f4 v[136:139], v[104:111], a[208:215], v[136:139]\n\t" \
  "ds_read_b64_tr_b8 v[124:125], v98 offset:16384\n\t" \
  "s_add_u32 s50, s68, 0x70000\n\t" \
  "buffer_load_dwordx4 a[232:235], v4, s[12:15], s50 offen offset:2048 sc0 sc1\n\t" \
  "v_mfma_f32_16x16x128_f8f6f4 v[140:143], v[104:111], a[216:223], v[140:143]\n\t" \
  "ds_read_b64_tr_b8 v[126:127], v98 offset:17408\n\t" \
  "s_waitcnt lgkmcnt(4)\n\t" \
  "v_mfma_f32_16x16x128_f8f6f4 v[156:159], v[112:119], a[216:223], v[156:159]\n\t" \
  "ds_read_b64_tr_b8 v[104:105], v83 offset:49152\n\t" \
  "s_add_u32 s51, s68, 0x70000\n\t" \
  "buffer_load_dwordx4 a[236:239], v4, s[12:15], s51 offen offset:3072 sc0 sc1\n\t" \
  "v_mfma_f32_16x16x128_f8f6f4 v[152:155], v[112:119], a[208:215], v[152:155]\n\t" \
  "ds_read_b64_tr_b8 v[106:107], v83 offset:50176\n\t" \
  "v_mfma_f32_16x16x128_f8f6f4 v[148:151], v[112:119], a[200:207], v[148:151]\n\t" \
  "ds_read_b64_tr_b8 v[108:109], v99 offset:16384\n\t" \
  "s_add_u32 s52, s68, 0x71000\n\t" \
  "buffer_load_dwordx4 a[240:243], v4, s[12:15], s52 offen offset:0 sc0 sc1\n\t" \
  "v_mfma_f32_16x16x128_f8f6f4 v[144:147], v[112:119], a[192:199], v[144:147]\n\t" \
  "ds_read_b64_tr_b8 v[110:111], v99 offset:17408\n\t" \
  "s_waitcnt lgkmcnt(4)\n\t" \
  "v_mfma_f32_16x16x128_f8f6f4 v[160:163], v[120:127], a[192:199], v[160:163]\n\t" \
  "ds_read_b64_tr_b8 v[112:113], v84 offset:49152\n\t" \
  "s_add_u32 s53, s68, 0x71000\n\t" \
  "buffer_load_dwordx4 a[244:247], v4, s[12:15], s53 offen offset:1024 sc0 sc1\n\t" \
  "v_mfma_f32_16x16x128_f8f6f4 v[164:167], v[120:127], a[200:207], v[164:167]\n\t" \
  "ds_read_b64_tr_b8 v[114:115], v84 offset:50176\n\t" \
  "v_mfma_f32_16x16x128_f8f6f4 v[168:171], v[120:127], a[208:215], v[168:171]\n\t" \
  "ds_read_b64_tr_b8 v[116:117], v100 offset:16384\n\t" \
  "s_add_u32 s54, s68, 0x71000\n\t" \
  "buffer_load_dwordx4 a[248:251], v4, s[12:15], s54 offen offset:2048 sc0 sc1\n\t" \
  "v_mfma_f32_16x16x128_f8f6f4 v[172:175], v[120:127], a[216:223], v[172:175]\n\t" \
  "ds_read_b64_tr_b8 v[118:119], v100 offset:17408\n\t" \
  "s_waitcnt lgkmcnt(4)\n\t" \
  "v_mfma_f32_16x16x128_f8f6f4 v[188:191], v[104:111], a[216:223], v[188:191]\n\t" \
  "ds_read_b64_tr_b8 v[120:121], v85 offset:49152\n\t" \
  "s_add_u32 s55, s68, 0x71000\n\t" \
  "buffer_load_dwordx4 a[252:255], v4, s[12:15], s55 offen offset:3072 sc0 sc1\n\t" \
  "v_mfma_f32_16x16x128_f8f6f4 v[184:187], v[104:111], a[208:215], v[184:187]\n\t" \
  "ds_read_b64_tr_b8 v[122:123], v85 offset:50176\n\t" \
  "v_mfma_f32_16x16x128_f8f6f4 v[180:183], v[104:111], a[200:207], v[180:183]\n\t" \
  "ds_read_b64_tr_b8 v[124:125], v101 offset:16384\n\t" \
  "s_add_u32 m0, s24, 0x18000\n\t" \
  "s_add_u32 s48, s44, 0x0\n\t" \
  "buffer_load_dwordx4 v3, s[8:11], s48 offen lds\n\t" \
  "v_mfma_f32_16x16x128_f8f6f4 v[176:179], v[104:111], a[192:199], v[176:179]\n\t" \
  "ds_read_b64_tr_b8 v[126:127], v101 offset:17408\n\t" \
  "s_waitcnt lgkmcnt(4)\n\t" \
  "v_mfma_f32_16x16x128_f8f6f4 v[192:195], v[112:119], a[192:199], v[192:195]\n\t" \
  "ds_read_b64_tr_b8 v[104:105], v86 offset:49152\n\t" \
  "s_add_u32 m0, s24, 0x18400\n\t" \
  "s_add_u32 s49, s44, 0x20000\n\t" \
  "buffer_load_dwordx4 v3, s[8:11], s49 offen lds\n\t" \
  "v_mfma_f32_16x16x128_f8f6f4 v[196:199], v[112:119], a[200:207], v[196:199]\n\t" \
  "ds_read_b64_tr_b8 v[106:107], v86 offset:50176\n\t" \
  "v_mfma_f32_16x16x128_f8f6f4 v[200:203], v[112:119], a[208:215], v[200:203]\n\t" \
  "ds_read_b64_tr_b8 v[108:109], v102 offset:16384\n\t" \
  "s_and_b32 s64, s56, 0x10\n\t" \
  "s_cmp_eq_u32 s64, 0x10\n\t" \
  "s_cbranch_scc1 Lok4_1_%=\n\t" \
  "s_mov_b32 s62, 0x10\n\t" \
  "s_mov_b32 s69, 0x7f\n\t" \
  "s_call_b64 s[60:61], Lpoll_1_%=\n\t" \
  "Lok4_1_%=:\n\t" \
  "s_add_u32 s50, s68, 0x80000\n\t" \
  "buffer_load_dwordx4 a[0:3], v4, s[12:15], s50 offen offset:0 sc0 sc1\n\t" \
  "v_mfma_f32_16x16x128_f8f6f4 v[204:207], v[112:119], a[216:223], v[204:207]\n\t" \
  "ds_read_b64_tr_b8 v[110:111], v102 offset:17408\n\t" \
  "s_waitcnt lgkmcnt(4)\n\t" \
  "v_mfma_f32_16x16x128_f8f6f4 v[220:223], v[120:127], a[216:223], v[220:223]\n\t" \
  "ds_read_b64_tr_b8 v[112:113], v87 offset:49152\n\t" \
  "s_add_u32 s51, s68, 0x80000\n\t" \
  "buffer_load_dwordx4 a[4:7], v4, s[12:15], s51 offen offset:1024 sc0 sc1\n\t" \
  "v_mfma_f32_16x16x128_f8f6f4 v[216:219], v[120:127], a[208:215], v[216:219]\n\t" \
  "ds_read_b64_tr_b8 v[114:115], v87 offset:50176\n\t" \
  "v_mfma_f32_16x16x128_f8f6f4 v[212:215], v[120:127], a[200:207], v[212:215]\n\t" \
  "ds_read_b64_tr_b8 v[116:117], v103 offset:16384\n\t" \
  "s_add_u32 s52, s68, 0x80000\n\t" \
  "buffer_load_dwordx4 a[8:11], v4, s[12:15], s52 offen offset:2048 sc0 sc1\n\t" \
  "v_mfma_f32_16x16x128_f8f6f4 v[208:211], v[120:127], a[192:199], v[208:211]\n\t" \
  "ds_read_b64_tr_b8 v[118:119], v103 offset:17408\n\t" \
  "s_waitcnt vmcnt(13) lgkmcnt(0)\n\t" \
  "s_barrier\n\t" \
  "v_mfma_f32_16x16x128_f8f6f4 v[224:227], v[104:111], a[192:199], v[224:227]\n\t" \
  "ds_read_b64_tr_b8 v[120:121], v80 offset:57344\n\t" \
  "s_add_u32 s53, s68, 0x80000\n\t" \
  "buffer_load_dwordx4 a[12:15], v4, s[12:15], s53 offen offset:3072 sc0 sc1\n\t" \
  "v_mfma_f32_16x16x128_f8f6f4 v[228:231], v[104:111], a[200:207], v[228:231]\n\t" \
  "ds_read_b64_tr_b8 v[122:123], v80 offset:58368\n\t" \
  "v_mfma_f32_16x16x128_f8f6f4 v[232:235], v[104:111], a[208:215], v[232:235]\n\t" \
  "ds_read_b64_tr_b8 v[124:125], v96 offset:24576\n\t" \
  "s_add_u32 s54, s68, 0x81000\n\t" \
  "buffer_load_dwordx4 a[16:19], v4, s[12:15], s54 offen offset:0 sc0 sc1\n\t" \
  "v_mfma_f32_16x16x128_f8f6f4 v[236:239], v[104:111], a[216:223], v[236:239]\n\t" \
  "ds_read_b64_tr_b8 v[126:127], v96 offset:25600\n\t" \
  "v_mfma_f32_16x16x128_f8f6f4 v[252:255], v[112:119], a[216:223], v[252:255]\n\t" \
  "ds_read_b64_tr_b8 v[104:105], v81 offset:57344\n\t" \
  "s_add_u32 s55, s68, 0x81000\n\t" \
  "buffer_load_dwordx4 a[20:23], v4, s[12:15], s55 offen offset:1024 sc0 sc1\n\t" \
  "v_mfma_f32_16x16x128_f8f6f4 v[248:251], v[112:119], a[208:215], v[248:251]\n\t" \
  "ds_read_b64_tr_b8 v[106:107], v81 offset:58368\n\t" \
  "v_mfma_f32_16x16x128_f8f6f4 v[244:247], v[112:119], a[200:207], v[244:247]\n\t" \
  "ds_read_b64_tr_b8 v[108:109], v97 offset:24576\n\t" \
  "s_add_u32 s48, s68, 0x81000\n\t" \
  "buffer_load_dwordx4 a[24:27], v4, s[12:15], s48 offen offset:2048 sc0 sc1\n\t" \
  "v_mfma_f32_16x16x128_f8f6f4 v[240:243], v[112:119], a[192:199], v[240:243]\n\t" \
  "ds_read_b64_tr_b8 v[110:111], v97 offset:25600\n\t" \
  "s_waitcnt vmcnt(9) lgkmcnt(4)\n\t" \
  "v_mfma_f32_16x16x128_f8f6f4 v[128:131], v[120:127], a[224:231], v[128:131]\n\t" \
  "ds_read_b64_tr_b8 v[112:113], v82 offset:57344\n\t" \
  "s_add_u32 s49, s68, 0x81000\n\t" \
  "buffer_load_dwordx4 a[28:31], v4, s[12:15], s49 offen offset:3072 sc0 sc1\n\t" \
  "v_mfma_f32_16x16x128_f8f6f4 v[132:135], v[120:127], a[232:239], v[132:135]\n\t" \
  "ds_read_b64_tr_b8 v[114:115], v82 offset:58368\n\t" \
  "v_mfma_f32_16x16x128_f8f6f4 v[136:139], v[120:127], a[240:247], v[136:139]\n\t" \
  "ds_read_b64_tr_b8 v[116:117], v98 offset:24576\n\t" \
  "s_add_u32 m0, s24, 0x1a000\n\t" \
  "s_add_u32 s50, s45, 0x0\n\t" \
  "buffer_load_dwordx4 v3, s[8:11], s50 offen lds\n\t" \
  "v_mfma_f32_16x16x128_f8f6f4 v[140:143], v[120:127], a[248:255], v[140:143]\n\t" \
  "ds_read_b64_tr_b8 v[118:119], v98 offset:25600\n\t" \
  "s_waitcnt lgkmcnt(4)\n\t" \
  "v_mfma_f32_16x16x128_f8f6f4 v[156:159], v[104:111], a[248:255], v[156:159]\n\t" \
  "ds_read_b64_tr_b8 v[120:121], v83 offset:57344\n\t" \
  "s_add_u32 m0, s24, 0x1a400\n\t" \
  "s_add_u32 s51, s45, 0x20000\n\t" \
  "buffer_load_dwordx4 v3, s[8:11], s51 offen lds\n\t" \
  "v_mfma_f32_16x16x128_f8f6f4 v[152:155], v[104:111], a[240:247], v[152:155]\n\t" \
  "ds_read_b64_tr_b8 v[122:123], v83 offset:58368\n\t" \
  "v_mfma_f32_16x16x128_f8f6f4 v[148:151], v[104:111], a[232:239], v[148:151]\n\t" \
  "ds_read_b64_tr_b8 v[124:125], v99 offset:24576\n\t" \
  "s_and_b32 s64, s56, 0x20\n\t" \
  "s_cmp_eq_u32 s64, 0x20\n\t" \
  "s_cbranch_scc1 Lok5_1_%=\n\t" \
  "s_mov_b32 s62, 0x20\n\t" \
  "s_mov_b32 s69, 0x7f\n\t" \
  "s_call_b64 s[60:61], Lpoll_1_%=\n\t" \
  "Lok5_1_%=:\n\t" \
  "s_add_u32 s52, s68, 0x90000\n\t" \
  "buffer_load_dwordx4 a[32:35], v4, s[12:15], s52 offen offset:0 sc0 sc1\n\t" \
  "v_mfma_f32_16x16x128_f8f6f4 v[144:147], v[104:111], a[224:231], v[144:147]\n\t" \
  "ds_read_b64_tr_b8 v[126:127], v99 offset:25600\n\t" \
  "s_waitcnt lgkmcnt(4)\n\t" \
  "v_mfma_f32_16x16x128_f8f6f4 v[160:163], v[112:119], a[224:231], v[160:163]\n\t" \
  "ds_read_b64_tr_b8 v[104:105], v84 offset:57344\n\t" \
  "s_add_u32 s53, s68, 0x90000\n\t" \
  "buffer_load_dwordx4 a[36:39], v4, s[12:15], s53 offen offset:1024 sc0 sc1\n\t" \
  "v_mfma_f32_16x16x128_f8f6f4 v[164:167], v[112:119], a[232:239], v[164:167]\n\t" \
  "ds_read_b64_tr_b8 v[106:107], v84 offset:58368\n\t" \
  "v_mfma_f32_16x16x128_f8f6f4 v[168:171], v[112:119], a[240:247], v[168:171]\n\t" \
  "ds_read_b64_tr_b8 v[108:109], v100 offset:24576\n\t" \
  "s_add_u32 s54, s68, 0x90000\n\t" \
  "buffer_load_dwordx4 a[40:43], v4, s[12:15], s54 offen offset:2048 sc0 sc1\n\t" \
  "v_mfma_f32_16x16x128_f8f6f4 v[172:175], v[112:119], a[248:255], v[172:175]\n\t" \
  "ds_read_b64_tr_b8 v[110:111], v100 offset:25600\n\t" \
  "s_waitcnt lgkmcnt(4)\n\t" \
  "v_mfma_f32_16x16x128_f8f6f4 v[188:191], v[120:127], a[248:255], v[188:191]\n\t" \
  "ds_read_b64_tr_b8 v[112:113], v85 offset:57344\n\t" \
  "s_add_u32 s55, s68, 0x90000\n\t" \
  "buffer_load_dwordx4 a[44:47], v4, s[12:15], s55 offen offset:3072 sc0 sc1\n\t" \
  "v_mfma_f32_16x16x128_f8f6f4 v[184:187], v[120:127], a[240:247], v[184:187]\n\t" \
  "ds_read_b64_tr_b8 v[114:115], v85 offset:58368\n\t" \
  "v_mfma_f32_16x16x128_f8f6f4 v[180:183], v[120:127], a[232:239], v[180:183]\n\t" \
  "ds_read_b64_tr_b8 v[116:117], v101 offset:24576\n\t" \
  "s_add_u32 s48, s68, 0x91000\n\t" \
  "buffer_load_dwordx4 a[48:51], v4, s[12:15], s48 offen offset:0 sc0 sc1\n\t" \
  "v_mfma_f32_16x16x128_f8f6f4 v[176:179], v[120:127], a[224:231], v[176:179]\n\t" \
  "ds_read_b64_tr_b8 v[118:119], v101 offset:25600\n\t" \
  "s_waitcnt lgkmcnt(4)\n\t" \
  "v_mfma_f32_16x16x128_f8f6f4 v[192:195], v[104:111], a[224:231], v[192:195]\n\t" \
  "ds_read_b64_tr_b8 v[120:121], v86 offset:57344\n\t" \
  "s_add_u32 s49, s68, 0x91000\n\t" \
  "buffer_load_dwordx4 a[52:55], v4, s[12:15], s49 offen offset:1024 sc0 sc1\n\t" \
  "v_mfma_f32_16x16x128_f8f6f4 v[196:199], v[104:111], a[232:239], v[196:199]\n\t" \
  "ds_read_b64_tr_b8 v[122:123], v86 offset:58368\n\t" \
  "v_mfma_f32_16x16x128_f8f6f4 v[200:203], v[104:111], a[240:247], v[200:203]\n\t" \
  "ds_read_b64_tr_b8 v[124:125], v102 offset:24576\n\t" \
  "s_add_u32 s50, s68, 0x91000\n\t" \
  "buffer_load_dwordx4 a[56:59], v4, s[12:15], s50 offen offset:2048 sc0 sc1\n\t" \
  "v_mfma_f32_16x16x128_f8f6f4 v[204:207], v[104:111], a[248:255], v[204:207]\n\t" \
  "ds_read_b64_tr_b8 v[126:127], v102 offset:25600\n\t" \
  "s_waitcnt lgkmcnt(4)\n\t" \
  "v_mfma_f32_16x16x128_f8f6f4 v[220:223], v[112:119], a[248:255], v[220:223]\n\t" \
  "ds_read_b64_tr_b8 v[104:105], v87 offset:57344\n\t" \
  "s_add_u32 s51, s68, 0x91000\n\t" \
  "buffer_load_dwordx4 a[60:63], v4, s[12:15], s51 offen offset:3072 sc0 sc1\n\t" \
  "v_mfma_f32_16x16x128_f8f6f4 v[216:219], v[112:119], a[240:247], v[216:219]\n\t" \
  "ds_read_b64_tr_b8 v[106:107], v87 offset:58368\n\t" \
  "v_mfma_f32_16x16x128_f8f6f4 v[212:215], v[112:119], a[232:239], v[212:215]\n\t" \
  "ds_read_b64_tr_b8 v[108:109], v103 offset:24576\n\t" \
  "s_cmp_lg_u32 s26, 2\n\t" \
  "s_cbranch_scc1 Lnf2_1_%=\n\t" \
  "s_mov_b64 exec, 1\n\t" \
  "global_store_dword v65, v64, s[30:31] sc0 sc1\n\t" \
  "s_mov_b64 exec, -1\n\t" \
  "s_branch Lmf2_1_%=\n\t" \
  "Lnf2_1_%=:\n\t" \
  "buffer_load_dword v62, v65, s[20:23], s22 offen\n\t" \
  "Lmf2_1_%=:\n\t" \
  "s_cmp_lg_u32 s26, 3\n\t" \
  "s_cbranch_scc1 Lns3_1_%=\n\t" \
  "global_store_dwordx4 v63, v[72:75], s[28:29] sc0 sc1\n\t" \
  "global_store_dwordx4 v63, v[76:79], s[28:29] offset:1024 sc0 sc1\n\t" \
  "s_branch Lms3_1_%=\n\t" \
  "Lns3_1_%=:\n\t" \
  "buffer_load_dword v62, v65, s[20:23], s22 offen\n\t" \
  "buffer_load_dword v62, v65, s[20:23], s22 offen\n\t" \
  "Lms3_1_%=:\n\t" \
  "s_add_u32 m0, s24, 0x0\n\t" \
  "s_add_u32 s52, s41, 0x200000\n\t" \
  "buffer_load_dwordx4 v2, s[8:11], s52 offen lds\n\t" \
  "v_mfma_f32_16x16x128_f8f6f4 v[208:211], v[112:119], a[224:231], v[208:211]\n\t" \
  "ds_read_b64_tr_b8 v[110:111], v103 offset:25600\n\t" \
  "s_waitcnt vmcnt(22) lgkmcnt(0)\n\t" \
  "s_barrier\n\t" \
  "v_mfma_f32_16x16x128_f8f6f4 v[224:227], v[120:127], a[224:231], v[224:227]\n\t" \
  "ds_read_b64_tr_b8 v[112:113], v88 offset:0\n\t" \
  "s_add_u32 m0, s24, 0x400\n\t" \
  "s_add_u32 s53, s41, 0x220000\n\t" \
  "buffer_load_dwordx4 v2, s[8:11], s53 offen lds\n\t" \
  "v_mfma_f32_16x16x128_f8f6f4 v[228:231], v[120:127], a[232:239], v[228:231]\n\t" \
  "ds_read_b64_tr_b8 v[114:115], v88 offset:1024\n\t" \
  "v_mfma_f32_16x16x128_f8f6f4 v[232:235], v[120:127], a[240:247], v[232:235]\n\t" \
  "ds_read_b64_tr_b8 v[116:117], v88 offset:32768\n\t" \
  "s_add_u32 m0, s24, 0x2000\n\t" \
  "s_add_u32 s54, s42, 0x200000\n\t" \
  "buffer_load_dwordx4 v2, s[8:11], s54 offen lds\n\t" \
  "v_mfma_f32_16x16x128_f8f6f4 v[236:239], v[120:127], a[248:255], v[236:239]\n\t" \
  "ds_read_b64_tr_b8 v[118:119], v88 offset:33792\n\t" \
  "v_mfma_f32_16x16x128_f8f6f4 v[252:255], v[104:111], a[248:255], v[252:255]\n\t" \
  "ds_read_b64_tr_b8 v[120:121], v89 offset:0\n\t" \
  "s_add_u32 m0, s24, 0x2400\n\t" \
  "s_add_u32 s55, s42, 0x220000\n\t" \
  "buffer_load_dwordx4 v2, s[8:11], s55 offen lds\n\t" \
  "v_mfma_f32_16x16x128_f8f6f4 v[248:251], v[104:111], a[240:247], v[248:251]\n\t" \
  "ds_read_b64_tr_b8 v[122:123], v89 offset:1024\n\t" \
  "v_mfma_f32_16x16x128_f8f6f4 v[244:247], v[104:111], a[232:239], v[244:247]\n\t" \
  "ds_read_b64_tr_b8 v[124:125], v89 offset:32768\n\t" \
  "s_add_u32 m0, s24, 0x4000\n\t" \
  "s_add_u32 s48, s43, 0x200000\n\t" \
  "buffer_load_dwordx4 v2, s[8:11], s48 offen lds\n\t" \
  "v_mfma_f32_16x16x128_f8f6f4 v[240:243], v[104:111], a[224:231], v[240:243]\n\t" \
  "ds_read_b64_tr_b8 v[126:127], v89 offset:33792\n\t" \
  "s_waitcnt vmcnt(18) lgkmcnt(4)\n\t" \
  "v_mfma_f32_16x16x128_f8f6f4 v[128:131], v[112:119], a[0:7], v[128:131]\n\t" \
  "ds_read_b64_tr_b8 v[104:105], v90 offset:0\n\t" \
  "s_add_u32 m0, s24, 0x4400\n\t" \
  "s_add_u32 s49, s43, 0x220000\n\t" \
  "buffer_load_dwordx4 v2, s[8:11], s49 offen lds\n\t" \
  "s_mov_b32 exec_lo, 0x7f00\n\t" \
  "s_mov_b32 exec_hi, 0\n\t" \
  "global_load_dwordx4 v[68:71], v10, s[6:7] sc0 sc1\n\t" \
  "s_mov_b64 exec, -1\n\t" \
  "v_mfma_f32_16x16x128_f8f6f4 v[132:135], v[112:119], a[8:15], v[132:135]\n\t" \
  "ds_read_b64_tr_b8 v[106:107], v90 offset:1024\n\t" \
  "v_mfma_f32_16x16x128_f8f6f4 v[136:139], v[112:119], a[16:23], v[136:139]\n\t" \
  "ds_read_b64_tr_b8 v[108:109], v90 offset:32768\n\t" \
  "s_add_u32 m0, s24, 0x6000\n\t" \
  "s_add_u32 s50, s44, 0x200000\n\t" \
  "buffer_load_dwordx4 v2, s[8:11], s50 offen lds\n\t" \
  "v_mfma_f32_16x16x128_f8f6f4 v[140:143], v[112:119], a[24:31], v[140:143]\n\t" \
  "ds_read_b64_tr_b8 v[110:111], v90 offset:33792\n\t" \
  "s_waitcnt lgkmcnt(4)\n\t" \
  "v_mfma_f32_16x16x128_f8f6f4 v[156:159], v[120:127], a[24:31], v[156:159]\n\t" \
  "ds_read_b64_tr_b8 v[112:113], v91 offset:0\n\t" \
  "s_add_u32 m0, s24, 0x6400\n\t" \
  "s_add_u32 s51, s44, 0x220000\n\t" \
  "buffer_load_dwordx4 v2, s[8:11], s51 offen lds\n\t" \
  "v_mfma_f32_16x16x128_f8f6f4 v[152:155], v[120:127], a[16:23], v[152:155]\n\t" \
  "ds_read_b64_tr_b8 v[114:115], v91 offset:1024\n\t" \
  "v_mfma_f32_16x16x128_f8f6f4 v[148:151], v[120:127], a[8:15], v[148:151]\n\t" \
  "ds_read_b64_tr_b8 v[116:117], v91 offset:32768\n\t" \
  "v_mfma_f32_16x16x128_f8f6f4 v[144:147], v[120:127], a[0:7], v[144:147]\n\t" \
  "ds_read_b64_tr_b8 v[118:119], v91 offset:33792\n\t" \
  "s_waitcnt lgkmcnt(4)\n\t" \
  "v_mfma_f32_16x16x128_f8f6f4 v[160:163], v[104:111], a[0:7], v[160:163]\n\t" \
  "ds_read_b64_tr_b8 v[120:121], v92 offset:0\n\t" \
  "v_mfma_f32_16x16x128_f8f6f4 v[164:167], v[104:111], a[8:15], v[164:167]\n\t" \
  "ds_read_b64_tr_b8 v[122:123], v92 offset:1024\n\t" \
  "v_mfma_f32_16x16x128_f8f6f4 v[168:171], v[104:111], a[16:23], v[168:171]\n\t" \
  "ds_read_b64_tr_b8 v[124:125], v92 offset:32768\n\t" \
  "v_mfma_f32_16x16x128_f8f6f4 v[172:175], v[104:111], a[24:31], v[172:175]\n\t" \
  "ds_read_b64_tr_b8 v[126:127], v92 offset:33792\n\t" \
  "s_waitcnt lgkmcnt(4)\n\t" \
  "v_mfma_f32_16x16x128_f8f6f4 v[188:191], v[112:119], a[24:31], v[188:191]\n\t" \
  "ds_read_b64_tr_b8 v[104:105], v93 offset:0\n\t" \
  "v_mfma_f32_16x16x128_f8f6f4 v[184:187], v[112:119], a[16:23], v[184:187]\n\t" \
  "ds_read_b64_tr_b8 v[106:107], v93 offset:1024\n\t" \
  "v_mfma_f32_16x16x128_f8f6f4 v[180:183], v[112:119], a[8:15], v[180:183]\n\t" \
  "ds_read_b64_tr_b8 v[108:109], v93 offset:32768\n\t" \
  "v_mfma_f32_16x16x128_f8f6f4 v[176:179], v[112:119], a[0:7], v[176:179]\n\t" \
  "ds_read_b64_tr_b8 v[110:111], v93 offset:33792\n\t" \
  "s_waitcnt lgkmcnt(4)\n\t" \
  "v_mfma_f32_16x16x128_f8f6f4 v[192:195], v[120:127], a[0:7], v[192:195]\n\t" \
  "ds_read_b64_tr_b8 v[112:113], v94 offset:0\n\t" \
  "v_mfma_f32_16x16x128_f8f6f4 v[196:199], v[120:127], a[8:15], v[196:199]\n\t" \
  "ds_read_b64_tr_b8 v[114:115], v94 offset:1024\n\t" \
  "v_mfma_f32_16x16x128_f8f6f4 v[200:203], v[120:127], a[16:23], v[200:203]\n\t" \
  "ds_read_b64_tr_b8 v[116:117], v94 offset:32768\n\t" \
  "v_mfma_f32_16x16x128_f8f6f4 v[204:207], v[120:127], a[24:31], v[204:207]\n\t" \
  "ds_read_b64_tr_b8 v[118:119], v94 offset:33792\n\t" \
  "s_waitcnt lgkmcnt(4)\n\t" \
  "v_mfma_f32_16x16x128_f8f6f4 v[220:223], v[104:111], a[24:31], v[220:223]\n\t" \
  "ds_read_b64_tr_b8 v[120:121], v95 offset:0\n\t" \
  "v_mfma_f32_16x16x128_f8f6f4 v[216:219], v[104:111], a[16:23], v[216:219]\n\t" \
  "ds_read_b64_tr_b8 v[122:123], v95 offset:1024\n\t" \
  "v_mfma_f32_16x16x128_f8f6f4 v[212:215], v[104:111], a[8:15], v[212:215]\n\t" \
  "ds_read_b64_tr_b8 v[124:125], v95 offset:32768\n\t" \
  "v_mfma_f32_16x16x128_f8f6f4 v[208:211], v[104:111], a[0:7], v[208:211]\n\t" \
  "ds_read_b64_tr_b8 v[126:127], v95 offset:33792\n\t" \
  "s_waitcnt vmcnt(20) lgkmcnt(0)\n\t" \
  "s_barrier\n\t" \
  "v_mfma_f32_16x16x128_f8f6f4 v[224:227], v[112:119], a[0:7], v[224:227]\n\t" \
  "ds_read_b64_tr_b8 v[104:105], v88 offset:8192\n\t" \
  "v_mfma_f32_16x16x128_f8f6f4 v[228:231], v[112:119], a[8:15], v[228:231]\n\t" \
  "ds_read_b64_tr_b8 v[106:107], v88 offset:9216\n\t" \
  "v_mfma_f32_16x16x128_f8f6f4 v[232:235], v[112:119], a[16:23], v[232:235]\n\t" \
  "ds_read_b64_tr_b8 v[108:109], v88 offset:40960\n\t" \
  "s_add_u32 m0, s24, 0x1c000\n\t" \
  "s_add_u32 s52, s46, 0x0\n\t" \
  "buffer_load_dwordx4 v3, s[8:11], s52 offen lds\n\t" \
  "v_mfma_f32_16x16x128_f8f6f4 v[236:239], v[112:119], a[24:31], v[236:239]\n\t" \
  "ds_read_b64_tr_b8 v[110:111], v88 offset:41984\n\t" \
  "v_mfma_f32_16x16x128_f8f6f4 v[252:255], v[120:127], a[24:31], v[252:255]\n\t" \
  "ds_read_b64_tr_b8 v[112:113], v89 offset:8192\n\t" \
  "s_add_u32 m0, s24, 0x1c400\n\t" \
  "s_add_u32 s53, s46, 0x20000\n\t" \
  "buffer_load_dwordx4 v3, s[8:11], s53 offen lds\n\t" \
  "v_mfma_f32_16x16x128_f8f6f4 v[248:251], v[120:127], a[16:23], v[248:251]\n\t" \
  "ds_read_b64_tr_b8 v[114:115], v89 offset:9216\n\t" \
  "v_mfma_f32_16x16x128_f8f6f4 v[244:247], v[120:127], a[8:15], v[244:247]\n\t" \
  "ds_read_b64_tr_b8 v[116:117], v89 offset:40960\n\t" \
  "s_and_b32 s64, s56, 0x40\n\t" \
  "s_cmp_eq_u32 s64, 0x40\n\t" \
  "s_cbranch_scc1 Lok6_1_%=\n\t" \
  "s_mov_b32 s62, 0x40\n\t" \
  "s_mov_b32 s69, 0x7f\n\t" \
  "s_call_b64 s[60:61], Lpoll_1_%=\n\t" \
  "Lok6_1_%=:\n\t" \
  "s_add_u32 s54, s68, 0xa0000\n\t" \
  "buffer_load_dwordx4 a[64:67], v4, s[12:15], s54 offen offset:0 sc0 sc1\n\t" \
  "v_mfma_f32_16x16x128_f8f6f4 v[240:243], v[120:127], a[0:7], v[240:243]\n\t" \
  "ds_read_b64_tr_b8 v[118:119], v89 offset:41984\n\t" \
  "s_waitcnt vmcnt(15) lgkmcnt(4)\n\t" \
  "v_mfma_f32_16x16x128_f8f6f4 v[128:131], v[104:111], a[32:39], v[128:131]\n\t" \
  "ds_read_b64_tr_b8 v[120:121], v90 offset:8192\n\t" \
  "s_add_u32 s55, s68, 0xa0000\n\t" \
  "buffer_load_dwordx4 a[68:71], v4, s[12:15], s55 offen offset:1024 sc0 sc1\n\t" \
  "v_mfma_f32_16x16x128_f8f6f4 v[132:135], v[104:111], a[40:47], v[132:135]\n\t" \
  "ds_read_b64_tr_b8 v[122:123], v90 offset:9216\n\t" \
  "v_mfma_f32_16x16x128_f8f6f4 v[136:139], v[104:111], a[48:55], v[136:139]\n\t" \
  "ds_read_b64_tr_b8 v[124:125], v90 offset:40960\n\t" \
  "s_add_u32 s48, s68, 0xa0000\n\t" \
  "buffer_load_dwordx4 a[72:75], v4, s[12:15], s48 offen offset:2048 sc0 sc1\n\t" \
  "v_mfma_f32_16x16x128_f8f6f4 v[140:143], v[104:111], a[56:63], v[140:143]\n\t" \
  "ds_read_b64_tr_b8 v[126:127], v90 offset:41984\n\t" \
  "s_waitcnt lgkmcnt(4)\n\t" \
  "v_mfma_f32_16x16x128_f8f6f4 v[156:159], v[112:119], a[56:63], v[156:159]\n\t" \
  "ds_read_b64_tr_b8 v[104:105], v91 offset:8192\n\t" \
  "s_add_u32 s49, s68, 0xa0000\n\t" \
  "buffer_load_dwordx4 a[76:79], v4, s[12:15], s49 offen offset:3072 sc0 sc1\n\t" \
  "v_mfma_f32_16x16x128_f8f6f4 v[152:155], v[112:119], a[48:55], v[152:155]\n\t" \
  "ds_read_b64_tr_b8 v[106:107], v91 offset:9216\n\t" \
  "v_mfma_f32_16x16x128_f8f6f4 v[148:151], v[112:119], a[40:47], v[148:151]\n\t" \
  "ds_read_b64_tr_b8 v[108:109], v91 offset:40960\n\t" \
  "s_add_u32 s50, s68, 0xa1000\n\t" \
  "buffer_load_dwordx4 a[80:83], v4, s[12:15], s50 offen offset:0 sc0 sc1\n\t" \
  "v_mfma_f32_16x16x128_f8f6f4 v[144:147], v[112:119], a[32:39], v[144:147]\n\t" \
  "ds_read_b64_tr_b8 v[110:111], v91 offset:41984\n\t" \
  "s_waitcnt lgkmcnt(4)\n\t" \
  "v_mfma_f32_16x16x128_f8f6f4 v[160:163], v[120:127], a[32:39], v[160:163]\n\t" \
  "ds_read_b64_tr_b8 v[112:113], v92 offset:8192\n\t" \
  "s_add_u32 s51, s68, 0xa1000\n\t" \
  "buffer_load_dwordx4 a[84:87], v4, s[12:15], s51 offen offset:1024 sc0 sc1\n\t" \
  "v_mfma_f32_16x16x128_f8f6f4 v[164:167], v[120:127], a[40:47], v[164:167]\n\t" \
  "ds_read_b64_tr_b8 v[114:115], v92 offset:9216\n\t" \
  "v_mfma_f32_16x16x128_f8f6f4 v[168:171], v[120:127], a[48:55], v[168:171]\n\t" \
  "ds_read_b64_tr_b8 v[116:117], v92 offset:40960\n\t" \
  "s_add_u32 s52, s68, 0xa1000\n\t" \
  "buffer_load_dwordx4 a[88:91], v4, s[12:15], s52 offen offset:2048 sc0 sc1\n\t" \
  "v_mfma_f32_16x16x128_f8f6f4 v[172:175], v[120:127], a[56:63], v[172:175]\n\t" \
  "ds_read_b64_tr_b8 v[118:119], v92 offset:41984\n\t" \
  "s_waitcnt lgkmcnt(4)\n\t" \
  "v_mfma_f32_16x16x128_f8f6f4 v[188:191], v[104:111], a[56:63], v[188:191]\n\t" \
  "ds_read_b64_tr_b8 v[120:121], v93 offset:8192\n\t" \
  "s_add_u32 s53, s68, 0xa1000\n\t" \
  "buffer_load_dwordx4 a[92:95], v4, s[12:15], s53 offen offset:3072 sc0 sc1\n\t" \
  "v_mfma_f32_16x16x128_f8f6f4 v[184:187], v[104:111], a[48:55], v[184:187]\n\t" \
  "ds_read_b64_tr_b8 v[122:123], v93 offset:9216\n\t" \
  "v_mfma_f32_16x16x128_f8f6f4 v[180:183], v[104:111], a[40:47], v[180:183]\n\t" \
  "ds_read_b64_tr_b8 v[124:125], v93 offset:40960\n\t" \
  "s_add_u32 m0, s24, 0x1e000\n\t" \
  "s_add_u32 s54, s40, 0x200000\n\t" \
  "buffer_load_dwordx4 v3, s[8:11], s54 offen lds\n\t" \
  "v_mfma_f32_16x16x128_f8f6f4 v[176:179], v[104:111], a[32:39], v[176:179]\n\t" \
  "ds_read_b64_tr_b8 v[126:127], v93 offset:41984\n\t" \
  "s_waitcnt lgkmcnt(4)\n\t" \
  "v_mfma_f32_16x16x128_f8f6f4 v[192:195], v[112:119], a[32:39], v[192:195]\n\t" \
  "ds_read_b64_tr_b8 v[104:105], v94 offset:8192\n\t" \
  "s_add_u32 m0, s24, 0x1e400\n\t" \
  "s_add_u32 s55, s40, 0x220000\n\t" \
  "buffer_load_dwordx4 v3, s[8:11], s55 offen lds\n\t" \
  "v_mfma_f32_16x16x128_f8f6f4 v[196:199], v[112:119], a[40:47], v[196:199]\n\t" \
  "ds_read_b64_tr_b8 v[106:107], v94 offset:9216\n\t" \
  "v_mfma_f32_16x16x128_f8f6f4 v[200:203], v[112:119], a[48:55], v[200:203]\n\t" \
  "ds_read_b64_tr_b8 v[108:109], v94 offset:40960\n\t" \
  "s_waitcnt vmcnt(14)\n\t" \
  "v_min_u32 v68, v68, v69\n\t" \
  "v_min3_u32 v68, v68, v70, v71\n\t" \
  "v_cmp_le_u32 vcc, s57, v68\n\t" \
  "s_and_b32 s64, vcc_lo, 0x7f00\n\t" \
  "s_or_b32 s56, s56, s64\n\t" \
  "s_and_b32 s64, s56, 0x7f00\n\t" \
  "s_cmp_eq_u32 s64, 0x7f00\n\t" \
  "s_cbranch_scc1 Lok7_1_%=\n\t" \
  "s_mov_b32 s62, 0x7f00\n\t" \
  "s_mov_b32 s69, 0x7f00\n\t" \
  "s_call_b64 s[60:61], Lpoll_1_%=\n\t" \
  "Lok7_1_%=:\n\t" \
  "s_add_u32 s48, s68, 0xb0000\n\t" \
  "buffer_load_dwordx4 a[96:99], v4, s[12:15], s48 offen offset:0 sc0 sc1\n\t" \
  "v_mfma_f32_16x16x128_f8f6f4 v[204:207], v[112:119], a[56:63], v[204:207]\n\t" \
  "ds_read_b64_tr_b8 v[110:111], v94 offset:41984\n\t" \
  "s_waitcnt lgkmcnt(4)\n\t" \
  "v_mfma_f32_16x16x128_f8f6f4 v[220:223], v[120:127], a[56:63], v[220:223]\n\t" \
  "ds_read_b64_tr_b8 v[112:113], v95 offset:8192\n\t" \
  "s_add_u32 s49, s68, 0xb0000\n\t" \
  "buffer_load_dwordx4 a[100:103], v4, s[12:15], s49 offen offset:1024 sc0 sc1\n\t" \
  "v_mfma_f32_16x16x128_f8f6f4 v[216:219], v[120:127], a[48:55], v[216:219]\n\t" \
  "ds_read_b64_tr_b8 v[114:115], v95 offset:9216\n\t" \
  "v_mfma_f32_16x16x128_f8f6f4 v[212:215], v[120:127], a[40:47], v[212:215]\n\t" \
  "ds_read_b64_tr_b8 v[116:117], v95 offset:40960\n\t" \
  "s_add_u32 s50, s68, 0xb0000\n\t" \
  "buffer_load_dwordx4 a[104:107], v4, s[12:15], s50 offen offset:2048 sc0 sc1\n\t" \
  "v_mfma_f32_16x16x128_f8f6f4 v[208:211], v[120:127], a[32:39], v[208:211]\n\t" \
  "ds_read_b64_tr_b8 v[118:119], v95 offset:41984\n\t" \
  "s_waitcnt vmcnt(13) lgkmcnt(0)\n\t" \
  "s_barrier\n\t" \
  "v_mfma_f32_16x16x128_f8f6f4 v[224:227], v[104:111], a[32:39], v[224:227]\n\t" \
  "ds_read_b64_tr_b8 v[120:121], v88 offset:16384\n\t" \
  "s_add_u32 s51, s68, 0xb0000\n\t" \
  "buffer_load_dwordx4 a[108:111], v4, s[12:15], s51 offen offset:3072 sc0 sc1\n\t" \
  "v_mfma_f32_16x16x128_f8f6f4 v[228:231], v[104:111], a[40:47], v[228:231]\n\t" \
  "ds_read_b64_tr_b8 v[122:123], v88 offset:17408\n\t" \
  "v_mfma_f32_16x16x128_f8f6f4 v[232:235], v[104:111], a[48:55], v[232:235]\n\t" \
  "ds_read_b64_tr_b8 v[124:125], v88 offset:49152\n\t" \
  "s_add_u32 s52, s68, 0xb1000\n\t" \
  "buffer_load_dwordx4 a[112:115], v4, s[12:15], s52 offen offset:0 sc0 sc1\n\t" \
  "v_mfma_f32_16x16x128_f8f6f4 v[236:239], v[104:111], a[56:63], v[236:239]\n\t" \
  "ds_read_b64_tr_b8 v[126:127], v88 offset:50176\n\t" \
  "v_mfma_f32_16x16x128_f8f6f4 v[252:255], v[112:119], a[56:63], v[252:255]\n\t" \
  "ds_read_b64_tr_b8 v[104:105], v89 offset:16384\n\t" \
  "s_add_u32 s53, s68, 0xb1000\n\t" \
  "buffer_load_dwordx4 a[116:119], v4, s[12:15], s53 offen offset:1024 sc0 sc1\n\t" \
  "v_mfma_f32_16x16x128_f8f6f4 v[248:251], v[112:119], a[48:55], v[248:251]\n\t" \
  "ds_read_b64_tr_b8 v[106:107], v89 offset:17408\n\t" \
  "v_mfma_f32_16x16x128_f8f6f4 v[244:247], v[112:119], a[40:47], v[244:247]\n\t" \
  "ds_read_b64_tr_b8 v[108:109], v89 offset:49152\n\t" \
  "s_add_u32 s54, s68, 0xb1000\n\t" \
  "buffer_load_dwordx4 a[120:123], v4, s[12:15], s54 offen offset:2048 sc0 sc1\n\t" \
  "v_mfma_f32_16x16x128_f8f6f4 v[240:243], v[112:119], a[32:39], v[240:243]\n\t" \
  "ds_read_b64_tr_b8 v[110:111], v89 offset:50176\n\t" \
  "s_waitcnt vmcnt(9) lgkmcnt(4)\n\t" \
  "v_mfma_f32_16x16x128_f8f6f4 v[128:131], v[120:127], a[64:71], v[128:131]\n\t" \
  "ds_read_b64_tr_b8 v[112:113], v90 offset:16384\n\t" \
  "s_add_u32 s55, s68, 0xb1000\n\t" \
  "buffer_load_dwordx4 a[124:127], v4, s[12:15], s55 offen offset:3072 sc0 sc1\n\t" \
  "v_mfma_f32_16x16x128_f8f6f4 v[132:135], v[120:127], a[72:79], v[132:135]\n\t" \
  "ds_read_b64_tr_b8 v[114:115], v90 offset:17408\n\t" \
  "v_mfma_f32_16x16x128_f8f6f4 v[136:139], v[120:127], a[80:87], v[136:139]\n\t" \
  "ds_read_b64_tr_b8 v[116:117], v90 offset:49152\n\t" \
  "s_add_u32 m0, s24, 0x20000\n\t" \
  "s_add_u32 s48, s41, 0x200000\n\t" \
  "buffer_load_dwordx4 v3, s[8:11], s48 offen lds\n\t" \
  "v_mfma_f32_16x16x128_f8f6f4 v[140:143], v[120:127], a[88:95], v[140:143]\n\t" \
  "ds_read_b64_tr_b8 v[118:119], v90 offset:50176\n\t" \
  "s_waitcnt lgkmcnt(4)\n\t" \
  "v_mfma_f32_16x16x128_f8f6f4 v[156:159], v[104:111], a[88:95], v[156:159]\n\t" \
  "ds_read_b64_tr_b8 v[120:121], v91 offset:16384\n\t" \
  "s_add_u32 m0, s24, 0x20400\n\t" \
  "s_add_u32 s49, s41, 0x220000\n\t" \
  "buffer_load_dwordx4 v3, s[8:11], s49 offen lds\n\t" \
  "v_mfma_f32_16x16x128_f8f6f4 v[152:155], v[104:111], a[80:87], v[152:155]\n\t" \
  "ds_read_b64_tr_b8 v[122:123], v91 offset:17408\n\t" \
  "v_mfma_f32_16x16x128_f8f6f4 v[148:151], v[104:111], a[72:79], v[148:151]\n\t" \
  "ds_read_b64_tr_b8 v[124:125], v91 offset:49152\n\t" \
  "s_add_u32 s50, s68, 0xc0000\n\t" \
  "buffer_load_dwordx4 a[128:131], v4, s[12:15], s50 offen offset:0 sc0 sc1\n\t" \
  "v_mfma_f32_16x16x128_f8f6f4 v[144:147], v[104:111], a[64:71], v[144:147]\n\t" \
  "ds_read_b64_tr_b8 v[126:127], v91 offset:50176\n\t" \
  "s_waitcnt lgkmcnt(4)\n\t" \
  "v_mfma_f32_16x16x128_f8f6f4 v[160:163], v[112:119], a[64:71], v[160:163]\n\t" \
  "ds_read_b64_tr_b8 v[104:105], v92 offset:16384\n\t" \
  "s_add_u32 s51, s68, 0xc0000\n\t" \
  "buffer_load_dwordx4 a[132:135], v4, s[12:15], s51 offen offset:1024 sc0 sc1\n\t" \
  "v_mfma_f32_16x16x128_f8f6f4 v[164:167], v[112:119], a[72:79], v[164:167]\n\t" \
  "ds_read_b64_tr_b8 v[106:107], v92 offset:17408\n\t" \
  "v_mfma_f32_16x16x128_f8f6f4 v[168:171], v[112:119], a[80:87], v[168:171]\n\t" \
  "ds_read_b64_tr_b8 v[108:109], v92 offset:49152\n\t" \
  "s_add_u32 s52, s68, 0xc0000\n\t" \
  "buffer_load_dwordx4 a[136:139], v4, s[12:15], s52 offen offset:2048 sc0 sc1\n\t" \
  "v_mfma_f32_16x16x128_f8f6f4 v[172:175], v[112:119], a[88:95], v[172:175]\n\t" \
  "ds_read_b64_tr_b8 v[110:111], v92 offset:50176\n\t" \
  "s_waitcnt lgkmcnt(4)\n\t" \
  "v_mfma_f32_16x16x128_f8f6f4 v[188:191], v[120:127], a[88:95], v[188:191]\n\t" \
  "ds_read_b64_tr_b8 v[112:113], v93 offset:16384\n\t" \
  "s_add_u32 s53, s68, 0xc0000\n\t" \
  "buffer_load_dwordx4 a[140:143], v4, s[12:15], s53 offen offset:3072 sc0 sc1\n\t" \
  "v_mfma_f32_16x16x128_f8f6f4 v[184:187], v[120:127], a[80:87], v[184:187]\n\t" \
  "ds_read_b64_tr_b8 v[114:115], v93 offset:17408\n\t" \
  "v_mfma_f32_16x16x128_f8f6f4 v[180:183], v[120:127], a[72:79], v[180:183]\n\t" \
  "ds_read_b64_tr_b8 v[116:117], v93 offset:49152\n\t" \
  "s_add_u32 s54, s68, 0xc1000\n\t" \
  "buffer_load_dwordx4 a[144:147], v4, s[12:15], s54 offen offset:0 sc0 sc1\n\t" \
  "v_mfma_f32_16x16x128_f8f6f4 v[176:179], v[120:127], a[64:71], v[176:179]\n\t" \
  "ds_read_b64_tr_b8 v[118:119], v93 offset:50176\n\t" \
  "s_waitcnt lgkmcnt(4)\n\t" \
  "v_mfma_f32_16x16x128_f8f6f4 v[192:195], v[104:111], a[64:71], v[192:195]\n\t" \
  "ds_read_b64_tr_b8 v[120:121], v94 offset:16384\n\t" \
  "s_add_u32 s55, s68, 0xc1000\n\t" \
  "buffer_load_dwordx4 a[148:151], v4, s[12:15], s55 offen offset:1024 sc0 sc1\n\t" \
  "v_mfma_f32_16x16x128_f8f6f4 v[196:199], v[104:111], a[72:79], v[196:199]\n\t" \
  "ds_read_b64_tr_b8 v[122:123], v94 offset:17408\n\t" \
  "v_mfma_f32_16x16x128_f8f6f4 v[200:203], v[104:111], a[80:87], v[200:203]\n\t" \
  "ds_read_b64_tr_b8 v[124:125], v94 offset:49152\n\t" \
  "s_add_u32 s48, s68, 0xc1000\n\t" \
  "buffer_load_dwordx4 a[152:155], v4, s[12:15], s48 offen offset:2048 sc0 sc1\n\t" \
  "v_mfma_f32_16x16x128_f8f6f4 v[204:207], v[104:111], a[88:95], v[204:207]\n\t" \
  "ds_read_b64_tr_b8 v[126:127], v94 offset:50176\n\t" \
  "s_waitcnt lgkmcnt(4)\n\t" \
  "v_mfma_f32_16x16x128_f8f6f4 v[220:223], v[112:119], a[88:95], v[220:223]\n\t" \
  "ds_read_b64_tr_b8 v[104:105], v95 offset:16384\n\t" \
  "s_add_u32 s49, s68, 0xc1000\n\t" \
  "buffer_load_dwordx4 a[156:159], v4, s[12:15], s49 offen offset:3072 sc0 sc1\n\t" \
  "v_mfma_f32_16x16x128_f8f6f4 v[216:219], v[112:119], a[80:87], v[216:219]\n\t" \
  "ds_read_b64_tr_b8 v[106:107], v95 offset:17408\n\t" \
  "v_mfma_f32_16x16x128_f8f6f4 v[212:215], v[112:119], a[72:79], v[212:215]\n\t" \
  "ds_read_b64_tr_b8 v[108:109], v95 offset:49152\n\t" \
  "s_cmp_lg_u32 s26, 3\n\t" \
  "s_cbranch_scc1 Lnf3_1_%=\n\t" \
  "s_mov_b64 exec, 1\n\t" \
  "global_store_dword v65, v64, s[30:31] sc0 sc1\n\t" \
  "s_mov_b64 exec, -1\n\t" \
  "s_branch Lmf3_1_%=\n\t" \
  "Lnf3_1_%=:\n\t" \
  "buffer_load_dword v62, v65, s[20:23], s22 offen\n\t" \
  "Lmf3_1_%=:\n\t" \
  "s_add_u32 m0, s24, 0x8000\n\t" \
  "s_add_u32 s50, s45, 0x200000\n\t" \
  "buffer_load_dwordx4 v2, s[8:11], s50 offen lds\n\t" \
  "v_mfma_f32_16x16x128_f8f6f4 v[208:211], v[112:119], a[64:71], v[208:211]\n\t" \
  "ds_read_b64_tr_b8 v[110:111], v95 offset:50176\n\t" \
  "s_waitcnt vmcnt(20) lgkmcnt(0)\n\t" \
  "s_barrier\n\t" \
  "v_mfma_f32_16x16x128_f8f6f4 v[224:227], v[120:127], a[64:71], v[224:227]\n\t" \
  "ds_read_b64_tr_b8 v[112:113], v88 offset:24576\n\t" \
  "s_add_u32 m0, s24, 0x8400\n\t" \
  "s_add_u32 s51, s45, 0x220000\n\t" \
  "buffer_load_dwordx4 v2, s[8:11], s51 offen lds\n\t" \
  "v_mfma_f32_16x16x128_f8f6f4 v[228:231], v[120:127], a[72:79], v[228:231]\n\t" \
  "ds_read_b64_tr_b8 v[114:115], v88 offset:25600\n\t" \
  "v_mfma_f32_16x16x128_f8f6f4 v[232:235], v[120:127], a[80:87], v[232:235]\n\t" \
  "ds_read_b64_tr_b8 v[116:117], v88 offset:57344\n\t" \
  "s_add_u32 m0, s24, 0xa000\n\t" \
  "s_add_u32 s52, s46, 0x200000\n\t" \
  "buffer_load_dwordx4 v2, s[8:11], s52 offen lds\n\t" \
  "v_mfma_f32_16x16x128_f8f6f4 v[236:239], v[120:127], a[88:95], v[236:239]\n\t" \
  "ds_read_b64_tr_b8 v[118:119], v88 offset:58368\n\t" \
  "v_mfma_f32_16x16x128_f8f6f4 v[252:255], v[104:111], a[88:95], v[252:255]\n\t" \
  "ds_read_b64_tr_b8 v[120:121], v89 offset:24576\n\t" \
  "s_add_u32 m0, s24, 0xa400\n\t" \
  "s_add_u32 s53, s46, 0x220000\n\t" \
  "buffer_load_dwordx4 v2, s[8:11], s53 offen lds\n\t" \
  "v_mfma_f32_16x16x128_f8f6f4 v[248:251], v[104:111], a[80:87], v[248:251]\n\t" \
  "ds_read_b64_tr_b8 v[122:123], v89 offset:25600\n\t" \
  "v_mfma_f32_16x16x128_f8f6f4 v[244:247], v[104:111], a[72:79], v[244:247]\n\t" \
  "ds_read_b64_tr_b8 v[124:125], v89 offset:57344\n\t" \
  "s_add_u32 m0, s24, 0xc000\n\t" \
  "s_add_u32 s54, s40, 0x400000\n\t" \
  "buffer_load_dwordx4 v2, s[8:11], s54 offen lds\n\t" \
  "v_mfma_f32_16x16x128_f8f6f4 v[240:243], v[104:111], a[64:71], v[240:243]\n\t" \
  "ds_read_b64_tr_b8 v[126:127], v89 offset:58368\n\t" \
  "s_waitcnt vmcnt(16) lgkmcnt(4)\n\t" \
  "v_mfma_f32_16x16x128_f8f6f4 v[128:131], v[112:119], a[96:103], v[128:131]\n\t" \
  "ds_read_b64_tr_b8 v[104:105], v90 offset:24576\n\t" \
  "s_add_u32 m0, s24, 0xc400\n\t" \
  "s_add_u32 s55, s40, 0x420000\n\t" \
  "buffer_load_dwordx4 v2, s[8:11], s55 offen lds\n\t" \
  "v_mfma_f32_16x16x128_f8f6f4 v[132:135], v[112:119], a[104:111], v[132:135]\n\t" \
  "ds_read_b64_tr_b8 v[106:107], v90 offset:25600\n\t" \
  "v_mfma_f32_16x16x128_f8f6f4 v[136:139], v[112:119], a[112:119], v[136:139]\n\t" \
  "ds_read_b64_tr_b8 v[108:109], v90 offset:57344\n\t" \
  "s_add_u32 m0, s24, 0xe000\n\t" \
  "s_add_u32 s48, s41, 0x400000\n\t" \
  "buffer_load_dwordx4 v2, s[8:11], s48 offen lds\n\t" \
  "v_mfma_f32_16x16x128_f8f6f4 v[140:143], v[112:119], a[120:127], v[140:143]\n\t" \
  "ds_read_b64_tr_b8 v[110:111], v90 offset:58368\n\t" \
  "s_waitcnt lgkmcnt(4)\n\t" \
  "v_mfma_f32_16x16x128_f8f6f4 v[156:159], v[120:127], a[120:127], v[156:159]\n\t" \
  "ds_read_b64_tr_b8 v[112:113], v91 offset:24576\n\t" \
  "s_add_u32 m0, s24, 0xe400\n\t" \
  "s_add_u32 s49, s41, 0x420000\n\t" \
  "buffer_load_dwordx4 v2, s[8:11], s49 offen lds\n\t" \
  "v_mfma_f32_16x16x128_f8f6f4 v[152:155], v[120:127], a[112:119], v[152:155]\n\t" \
  "ds_read_b64_tr_b8 v[114:115], v91 offset:25600\n\t" \
  "v_mfma_f32_16x16x128_f8f6f4 v[148:151], v[120:127], a[104:111], v[148:151]\n\t" \
  "ds_read_b64_tr_b8 v[116:117], v91 offset:57344\n\t" \
  "v_mfma_f32_16x16x128_f8f6f4 v[144:147], v[120:127], a[96:103], v[144:147]\n\t" \
  "ds_read_b64_tr_b8 v[118:119], v91 offset:58368\n\t" \
  "s_waitcnt lgkmcnt(4)\n\t" \
  "v_mfma_f32_16x16x128_f8f6f4 v[160:163], v[104:111], a[96:103], v[160:163]\n\t" \
  "ds_read_b64_tr_b8 v[120:121], v92 offset:24576\n\t" \
  "v_mfma_f32_16x16x128_f8f6f4 v[164:167], v[104:111], a[104:111], v[164:167]\n\t" \
  "ds_read_b64_tr_b8 v[122:123], v92 offset:25600\n\t" \
  "v_mfma_f32_16x16x128_f8f6f4 v[168:171], v[104:111], a[112:119], v[168:171]\n\t" \
  "ds_read_b64_tr_b8 v[124:125], v92 offset:57344\n\t" \
  "v_mfma_f32_16x16x128_f8f6f4 v[172:175], v[104:111], a[120:127], v[172:175]\n\t" \
  "ds_read_b64_tr_b8 v[126:127], v92 offset:58368\n\t" \
  "s_waitcnt lgkmcnt(4)\n\t" \
  "v_mfma_f32_16x16x128_f8f6f4 v[188:191], v[112:119], a[120:127], v[188:191]\n\t" \
  "ds_read_b64_tr_b8 v[104:105], v93 offset:24576\n\t" \
  "v_mfma_f32_16x16x128_f8f6f4 v[184:187], v[112:119], a[112:119], v[184:187]\n\t" \
  "ds_read_b64_tr_b8 v[106:107], v93 offset:25600\n\t" \
  "v_mfma_f32_16x16x128_f8f6f4 v[180:183], v[112:119], a[104:111], v[180:183]\n\t" \
  "ds_read_b64_tr_b8 v[108:109], v93 offset:57344\n\t" \
  "v_mfma_f32_16x16x128_f8f6f4 v[176:179], v[112:119], a[96:103], v[176:179]\n\t" \
  "ds_read_b64_tr_b8 v[110:111], v93 offset:58368\n\t" \
  "s_waitcnt lgkmcnt(4)\n\t" \
  "v_mfma_f32_16x16x128_f8f6f4 v[192:195], v[120:127], a[96:103], v[192:195]\n\t" \
  "ds_read_b64_tr_b8 v[112:113], v94 offset:24576\n\t" \
  "v_mfma_f32_16x16x128_f8f6f4 v[196:199], v[120:127], a[104:111], v[196:199]\n\t" \
  "ds_read_b64_tr_b8 v[114:115], v94 offset:25600\n\t" \
  "v_mfma_f32_16x16x128_f8f6f4 v[200:203], v[120:127], a[112:119], v[200:203]\n\t" \
  "ds_read_b64_tr_b8 v[116:117], v94 offset:57344\n\t" \
  "v_mfma_f32_16x16x128_f8f6f4 v[204:207], v[120:127], a[120:127], v[204:207]\n\t" \
  "ds_read_b64_tr_b8 v[118:119], v94 offset:58368\n\t" \
  "s_waitcnt lgkmcnt(4)\n\t" \
  "v_mfma_f32_16x16x128_f8f6f4 v[220:223], v[104:111], a[120:127], v[220:223]\n\t" \
  "ds_read_b64_tr_b8 v[120:121], v95 offset:24576\n\t" \
  "s_add_u32 m0, s24, 0x22000\n\t" \
  "s_add_u32 s50, s42, 0x200000\n\t" \
  "buffer_load_dwordx4 v3, s[8:11], s50 offen lds\n\t" \
  "v_mfma_f32_16x16x128_f8f6f4 v[216:219], v[104:111], a[112:119], v[216:219]\n\t" \
  "ds_read_b64_tr_b8 v[122:123], v95 offset:25600\n\t" \
  "v_mfma_f32_16x16x128_f8f6f4 v[212:215], v[104:111], a[104:111], v[212:215]\n\t" \
  "ds_read_b64_tr_b8 v[124:125], v95 offset:57344\n\t" \
  "s_add_u32 m0, s24, 0x22400\n\t" \
  "s_add_u32 s51, s42, 0x220000\n\t" \
  "buffer_load_dwordx4 v3, s[8:11], s51 offen lds\n\t" \
  "v_mfma_f32_16x16x128_f8f6f4 v[208:211], v[104:111], a[96:103], v[208:211]\n\t" \
  "ds_read_b64_tr_b8 v[126:127], v95 offset:58368\n\t" \
  "s_waitcnt vmcnt(19) lgkmcnt(0)\n\t" \
  "s_barrier\n\t" \
  "v_mfma_f32_16x16x128_f8f6f4 v[224:227], v[112:119], a[96:103], v[224:227]\n\t" \
  "ds_read_b64_tr_b8 v[104:105], v80 offset:0\n\t" \
  "s_add_u32 s52, s68, 0xd0000\n\t" \
  "buffer_load_dwordx4 a[160:163], v4, s[12:15], s52 offen offset:0 sc0 sc1\n\t" \
  "v_mfma_f32_16x16x128_f8f6f4 v[228:231], v[112:119], a[104:111], v[228:231]\n\t" \
  "ds_read_b64_tr_b8 v[106:107], v80 offset:1024\n\t" \
  "v_mfma_f32_16x16x128_f8f6f4 v[232:235], v[112:119], a[112:119], v[232:235]\n\t" \
  "ds_read_b64_tr_b8 v[108:109], v96 offset:0\n\t" \
  "s_add_u32 s53, s68, 0xd0000\n\t" \
  "buffer_load_dwordx4 a[164:167], v4, s[12:15], s53 offen offset:1024 sc0 sc1\n\t" \
  "v_mfma_f32_16x16x128_f8f6f4 v[236:239], v[112:119], a[120:127], v[236:239]\n\t" \
  "ds_read_b64_tr_b8 v[110:111], v96 offset:1024\n\t" \
  "v_mfma_f32_16x16x128_f8f6f4 v[252:255], v[120:127], a[120:127], v[252:255]\n\t" \
  "ds_read_b64_tr_b8 v[112:113], v81 offset:0\n\t" \
  "s_add_u32 s54, s68, 0xd0000\n\t" \
  "buffer_load_dwordx4 a[168:171], v4, s[12:15], s54 offen offset:2048 sc0 sc1\n\t" \
  "v_mfma_f32_16x16x128_f8f6f4 v[248:251], v[120:127], a[112:119], v[248:251]\n\t" \
  "ds_read_b64_tr_b8 v[114:115], v81 offset:1024\n\t" \
  "v_mfma_f32_16x16x128_f8f6f4 v[244:247], v[120:127], a[104:111], v[244:247]\n\t" \
  "ds_read_b64_tr_b8 v[116:117], v97 offset:0\n\t" \
  "s_add_u32 s55, s68, 0xd0000\n\t" \
  "buffer_load_dwordx4 a[172:175], v4, s[12:15], s55 offen offset:3072 sc0 sc1\n\t" \
  "v_mfma_f32_16x16x128_f8f6f4 v[240:243], v[120:127], a[96:103], v[240:243]\n\t" \
  "ds_read_b64_tr_b8 v[118:119], v97 offset:1024\n\t" \
  "s_waitcnt vmcnt(15) lgkmcnt(4)\n\t" \
  "v_mfma_f32_16x16x128_f8f6f4 v[128:131], v[104:111], a[128:135], v[128:131]\n\t" \
  "ds_read_b64_tr_b8 v[120:121], v82 offset:0\n\t" \
  "s_add_u32 s48, s68, 0xd1000\n\t" \
  "buffer_load_dwordx4 a[176:179], v4, s[12:15], s48 offen offset:0 sc0 sc1\n\t" \
  "v_mfma_f32_16x16x128_f8f6f4 v[132:135], v[104:111], a[136:143], v[132:135]\n\t" \
  "ds_read_b64_tr_b8 v[122:123], v82 offset:1024\n\t" \
  "v_mfma_f32_16x16x128_f8f6f4 v[136:139], v[104:111], a[144:151], v[136:139]\n\t" \
  "ds_read_b64_tr_b8 v[124:125], v98 offset:0\n\t" \
  "s_add_u32 s49, s68, 0xd1000\n\t" \
  "buffer_load_dwordx4 a[180:183], v4, s[12:15], s49 offen offset:1024 sc0 sc1\n\t" \
  "v_mfma_f32_16x16x128_f8f6f4 v[140:143], v[104:111], a[152:159], v[140:143]\n\t" \
  "ds_read_b64_tr_b8 v[126:127], v98 offset:1024\n\t" \
  "s_waitcnt lgkmcnt(4)\n\t" \
  "v_mfma_f32_16x16x128_f8f6f4 v[156:159], v[112:119], a[152:159], v[156:159]\n\t" \
  "ds_read_b64_tr_b8 v[104:105], v83 offset:0\n\t" \
  "s_add_u32 s50, s68, 0xd1000\n\t" \
  "buffer_load_dwordx4 a[184:187], v4, s[12:15], s50 offen offset:2048 sc0 sc1\n\t" \
  "v_mfma_f32_16x16x128_f8f6f4 v[152:155], v[112:119], a[144:151], v[152:155]\n\t" \
  "ds_read_b64_tr_b8 v[106:107], v83 offset:1024\n\t" \
  "v_mfma_f32_16x16x128_f8f6f4 v[148:151], v[112:119], a[136:143], v[148:151]\n\t" \
  "ds_read_b64_tr_b8 v[108:109], v99 offset:0\n\t" \
  "s_add_u32 s51, s68, 0xd1000\n\t" \
  "buffer_load_dwordx4 a[188:191], v4, s[12:15], s51 offen offset:3072 sc0 sc1\n\t" \
  "v_mfma_f32_16x16x128_f8f6f4 v[144:147], v[112:119], a[128:135], v[144:147]\n\t" \
  "ds_read_b64_tr_b8 v[110:111], v99 offset:1024\n\t" \
  "s_waitcnt lgkmcnt(4)\n\t" \
  "v_mfma_f32_16x16x128_f8f6f4 v[160:163], v[120:127], a[128:135], v[160:163]\n\t" \
  "ds_read_b64_tr_b8 v[112:113], v84 offset:0\n\t" \
  "s_add_u32 m0, s24, 0x24000\n\t" \
  "s_add_u32 s52, s43, 0x200000\n\t" \
  "buffer_load_dwordx4 v3, s[8:11], s52 offen lds\n\t" \
  "v_mfma_f32_16x16x128_f8f6f4 v[164:167], v[120:127], a[136:143], v[164:167]\n\t" \
  "ds_read_b64_tr_b8 v[114:115], v84 offset:1024\n\t" \
  "v_mfma_f32_16x16x128_f8f6f4 v[168:171], v[120:127], a[144:151], v[168:171]\n\t" \
  "ds_read_b64_tr_b8 v[116:117], v100 offset:0\n\t" \
  "s_add_u32 m0, s24, 0x24400\n\t" \
  "s_add_u32 s53, s43, 0x220000\n\t" \
  "buffer_load_dwordx4 v3, s[8:11], s53 offen lds\n\t" \
  "v_mfma_f32_16x16x128_f8f6f4 v[172:175], v[120:127], a[152:159], v[172:175]\n\t" \
  "ds_read_b64_tr_b8 v[118:119], v100 offset:1024\n\t" \
  "s_waitcnt lgkmcnt(4)\n\t" \
  "v_mfma_f32_16x16x128_f8f6f4 v[188:191], v[104:111], a[152:159], v[188:191]\n\t" \
  "ds_read_b64_tr_b8 v[120:121], v85 offset:0\n\t" \
  "s_add_u32 s54, s68, 0xe0000\n\t" \
  "buffer_load_dwordx4 a[192:195], v4, s[12:15], s54 offen offset:0 sc0 sc1\n\t" \
  "v_mfma_f32_16x16x128_f8f6f4 v[184:187], v[104:111], a[144:151], v[184:187]\n\t" \
  "ds_read_b64_tr_b8 v[122:123], v85 offset:1024\n\t" \
  "v_mfma_f32_16x16x128_f8f6f4 v[180:183], v[104:111], a[136:143], v[180:183]\n\t" \
  "ds_read_b64_tr_b8 v[124:125], v101 offset:0\n\t" \
  "s_add_u32 s55, s68, 0xe0000\n\t" \
  "buffer_load_dwordx4 a[196:199], v4, s[12:15], s55 offen offset:1024 sc0 sc1\n\t" \
  "v_mfma_f32_16x16x128_f8f6f4 v[176:179], v[104:111], a[128:135], v[176:179]\n\t" \
  "ds_read_b64_tr_b8 v[126:127], v101 offset:1024\n\t" \
  "s_waitcnt lgkmcnt(4)\n\t" \
  "v_mfma_f32_16x16x128_f8f6f4 v[192:195], v[112:119], a[128:135], v[192:195]\n\t" \
  "ds_read_b64_tr_b8 v[104:105], v86 offset:0\n\t" \
  "s_add_u32 s48, s68, 0xe0000\n\t" \
  "buffer_load_dwordx4 a[200:203], v4, s[12:15], s48 offen offset:2048 sc0 sc1\n\t" \
  "v_mfma_f32_16x16x128_f8f6f4 v[196:199], v[112:119], a[136:143], v[196:199]\n\t" \
  "ds_read_b64_tr_b8 v[106:107], v86 offset:1024\n\t" \
  "v_mfma_f32_16x16x128_f8f6f4 v[200:203], v[112:119], a[144:151], v[200:203]\n\t" \
  "ds_read_b64_tr_b8 v[108:109], v102 offset:0\n\t" \
  "s_add_u32 s49, s68, 0xe0000\n\t" \
  "buffer_load_dwordx4 a[204:207], v4, s[12:15], s49 offen offset:3072 sc0 sc1\n\t" \
  "v_mfma_f32_16x16x128_f8f6f4 v[204:207], v[112:119], a[152:159], v[204:207]\n\t" \
  "ds_read_b64_tr_b8 v[110:111], v102 offset:1024\n\t" \
  "s_waitcnt lgkmcnt(4)\n\t" \
  "v_mfma_f32_16x16x128_f8f6f4 v[220:223], v[120:127], a[152:159], v[220:223]\n\t" \
  "ds_read_b64_tr_b8 v[112:113], v87 offset:0\n\t" \
  "s_add_u32 s50, s68, 0xe1000\n\t" \
  "buffer_load_dwordx4 a[208:211], v4, s[12:15], s50 offen offset:0 sc0 sc1\n\t" \
  "v_mfma_f32_16x16x128_f8f6f4 v[216:219], v[120:127], a[144:151], v[216:219]\n\t" \
  "ds_read_b64_tr_b8 v[114:115], v87 offset:1024\n\t" \
  "v_mfma_f32_16x16x128_f8f6f4 v[212:215], v[120:127], a[136:143], v[212:215]\n\t" \
  "ds_read_b64_tr_b8 v[116:117], v103 offset:0\n\t" \
  "s_add_u32 s51, s68, 0xe1000\n\t" \
  "buffer_load_dwordx4 a[212:215], v4, s[12:15], s51 offen offset:1024 sc0 sc1\n\t" \
  "v_mfma_f32_16x16x128_f8f6f4 v[208:211], v[120:127], a[128:135], v[208:211]\n\t" \
  "ds_read_b64_tr_b8 v[118:119], v103 offset:1024\n\t" \
  "s_waitcnt vmcnt(16) lgkmcnt(0)\n\t" \
  "s_barrier\n\t" \
  "v_mfma_f32_16x16x128_f8f6f4 v[224:227], v[104:111], a[128:135], v[224:227]\n\t" \
  "ds_read_b64_tr_b8 v[120:121], v80 offset:8192\n\t" \
  "s_add_u32 s52, s68, 0xe1000\n\t" \
  "buffer_load_dwordx4 a[216:219], v4, s[12:15], s52 offen offset:2048 sc0 sc1\n\t" \
  "v_mfma_f32_16x16x128_f8f6f4 v[228:231], v[104:111], a[136:143], v[228:231]\n\t" \
  "ds_read_b64_tr_b8 v[122:123], v80 offset:9216\n\t" \
  "v_mfma_f32_16x16x128_f8f6f4 v[232:235], v[104:111], a[144:151], v[232:235]\n\t" \
  "ds_read_b64_tr_b8 v[124:125], v96 offset:8192\n\t" \
  "s_add_u32 s53, s68, 0xe1000\n\t" \
  "buffer_load_dwordx4 a[220:223], v4, s[12:15], s53 offen offset:3072 sc0 sc1\n\t" \
  "v_mfma_f32_16x16x128_f8f6f4 v[236:239], v[104:111], a[152:159], v[236:239]\n\t" \
  "ds_read_b64_tr_b8 v[126:127], v96 offset:9216\n\t" \
  "v_mfma_f32_16x16x128_f8f6f4 v[252:255], v[112:119], a[152:159], v[252:255]\n\t" \
  "ds_read_b64_tr_b8 v[104:105], v81 offset:8192\n\t" \
  "s_add_u32 m0, s24, 0x26000\n\t" \
  "s_add_u32 s54, s44, 0x200000\n\t" \
  "buffer_load_dwordx4 v3, s[8:11], s54 offen lds\n\t" \
  "v_mfma_f32_16x16x128_f8f6f4 v[248:251], v[112:119], a[144:151], v[248:251]\n\t" \
  "ds_read_b64_tr_b8 v[106:107], v81 offset:9216\n\t" \
  "v_mfma_f32_16x16x128_f8f6f4 v[244:247], v[112:119], a[136:143], v[244:247]\n\t" \
  "ds_read_b64_tr_b8 v[108:109], v97 offset:8192\n\t" \
  "s_add_u32 m0, s24, 0x26400\n\t" \
  "s_add_u32 s55, s44, 0x220000\n\t" \
  "buffer_load_dwordx4 v3, s[8:11], s55 offen lds\n\t" \
  "v_mfma_f32_16x16x128_f8f6f4 v[240:243], v[112:119], a[128:135], v[240:243]\n\t" \
  "ds_read_b64_tr_b8 v[110:111], v97 offset:9216\n\t" \
  "s_waitcnt vmcnt(12) lgkmcnt(4)\n\t" \
  "v_mfma_f32_16x16x128_f8f6f4 v[128:131], v[120:127], a[160:167], v[128:131]\n\t" \
  "ds_read_b64_tr_b8 v[112:113], v82 offset:8192\n\t" \
  "s_add_u32 s48, s68, 0xf0000\n\t" \
  "buffer_load_dwordx4 a[224:227], v4, s[12:15], s48 offen offset:0 sc0 sc1\n\t" \
  "v_mfma_f32_16x16x128_f8f6f4 v[132:135], v[120:127], a[168:175], v[132:135]\n\t" \
  "ds_read_b64_tr_b8 v[114:115], v82 offset:9216\n\t" \
  "v_mfma_f32_16x16x128_f8f6f4 v[136:139], v[120:127], a[176:183], v[136:139]\n\t" \
  "ds_read_b64_tr_b8 v[116:117], v98 offset:8192\n\t" \
  "s_add_u32 s49, s68, 0xf0000\n\t" \
  "buffer_load_dwordx4 a[228:231], v4, s[12:15], s49 offen offset:1024 sc0 sc1\n\t" \
  "v_mfma_f32_16x16x128_f8f6f4 v[140:143], v[120:127], a[184:191], v[140:143]\n\t" \
  "ds_read_b64_tr_b8 v[118:119], v98 offset:9216\n\t" \
  "s_waitcnt lgkmcnt(4)\n\t" \
  "v_mfma_f32_16x16x128_f8f6f4 v[156:159], v[104:111], a[184:191], v[156:159]\n\t" \
  "ds_read_b64_tr_b8 v[120:121], v83 offset:8192\n\t" \
  "s_add_u32 s50, s68, 0xf0000\n\t" \
  "buffer_load_dwordx4 a[232:235], v4, s[12:15], s50 offen offset:2048 sc0 sc1\n\t" \
  "v_mfma_f32_16x16x128_f8f6f4 v[152:155], v[104:111], a[176:183], v[152:155]\n\t" \
  "ds_read_b64_tr_b8 v[122:123], v83 offset:9216\n\t" \
  "v_mfma_f32_16x16x128_f8f6f4 v[148:151], v[104:111], a[168:175], v[148:151]\n\t" \
  "ds_read_b64_tr_b8 v[124:125], v99 offset:8192\n\t" \
  "s_add_u32 s51, s68, 0xf0000\n\t" \
  "buffer_load_dwordx4 a[236:239], v4, s[12:15], s51 offen offset:3072 sc0 sc1\n\t" \
  "v_mfma_f32_16x16x128_f8f6f4 v[144:147], v[104:111], a[160:167], v[144:147]\n\t" \
  "ds_read_b64_tr_b8 v[126:127], v99 offset:9216\n\t" \
  "s_waitcnt lgkmcnt(4)\n\t" \
  "v_mfma_f32_16x16x128_f8f6f4 v[160:163], v[112:119], a[160:167], v[160:163]\n\t" \
  "ds_read_b64_tr_b8 v[104:105], v84 offset:8192\n\t" \
  "s_add_u32 s52, s68, 0xf1000\n\t" \
  "buffer_load_dwordx4 a[240:243], v4, s[12:15], s52 offen offset:0 sc0 sc1\n\t" \
  "v_mfma_f32_16x16x128_f8f6f4 v[164:167], v[112:119], a[168:175], v[164:167]\n\t" \
  "ds_read_b64_tr_b8 v[106:107], v84 offset:9216\n\t" \
  "v_mfma_f32_16x16x128_f8f6f4 v[168:171], v[112:119], a[176:183], v[168:171]\n\t" \
  "ds_read_b64_tr_b8 v[108:109], v100 offset:8192\n\t" \
  "s_add_u32 s53, s68, 0xf1000\n\t" \
  "buffer_load_dwordx4 a[244:247], v4, s[12:15], s53 offen offset:1024 sc0 sc1\n\t" \
  "v_mfma_f32_16x16x128_f8f6f4 v[172:175], v[112:119], a[184:191], v[172:175]\n\t" \
  "ds_read_b64_tr_b8 v[110:111], v100 offset:9216\n\t" \
  "s_waitcnt lgkmcnt(4)\n\t" \
  "v_mfma_f32_16x16x128_f8f6f4 v[188:191], v[120:127], a[184:191], v[188:191]\n\t" \
  "ds_read_b64_tr_b8 v[112:113], v85 offset:8192\n\t" \
  "s_add_u32 s54, s68, 0xf1000\n\t" \
  "buffer_load_dwordx4 a[248:251], v4, s[12:15], s54 offen offset:2048 sc0 sc1\n\t" \
  "v_mfma_f32_16x16x128_f8f6f4 v[184:187], v[120:127], a[176:183], v[184:187]\n\t" \
  "ds_read_b64_tr_b8 v[114:115], v85 offset:9216\n\t" \
  "v_mfma_f32_16x16x128_f8f6f4 v[180:183], v[120:127], a[168:175], v[180:183]\n\t" \
  "ds_read_b64_tr_b8 v[116:117], v101 offset:8192\n\t" \
  "s_add_u32 s55, s68, 0xf1000\n\t" \
  "buffer_load_dwordx4 a[252:255], v4, s[12:15], s55 offen offset:3072 sc0 sc1\n\t" \
  "v_mfma_f32_16x16x128_f8f6f4 v[176:179], v[120:127], a[160:167], v[176:179]\n\t" \
  "ds_read_b64_tr_b8 v[118:119], v101 offset:9216\n\t" \
  "s_waitcnt lgkmcnt(4)\n\t" \
  "v_mfma_f32_16x16x128_f8f6f4 v[192:195], v[104:111], a[160:167], v[192:195]\n\t" \
  "ds_read_b64_tr_b8 v[120:121], v86 offset:8192\n\t" \
  "s_add_u32 m0, s24, 0x18000\n\t" \
  "s_add_u32 s48, s45, 0x200000\n\t" \
  "buffer_load_dwordx4 v3, s[8:11], s48 offen lds\n\t" \
  "v_mfma_f32_16x16x128_f8f6f4 v[196:199], v[104:111], a[168:175], v[196:199]\n\t" \
  "ds_read_b64_tr_b8 v[122:123], v86 offset:9216\n\t" \
  "v_mfma_f32_16x16x128_f8f6f4 v[200:203], v[104:111], a[176:183], v[200:203]\n\t" \
  "ds_read_b64_tr_b8 v[124:125], v102 offset:8192\n\t" \
  "s_add_u32 m0, s24, 0x18400\n\t" \
  "s_add_u32 s49, s45, 0x220000\n\t" \
  "buffer_load_dwordx4 v3, s[8:11], s49 offen lds\n\t" \
  "v_mfma_f32_16x16x128_f8f6f4 v[204:207], v[104:111], a[184:191], v[204:207]\n\t" \
  "ds_read_b64_tr_b8 v[126:127], v102 offset:9216\n\t" \
  "s_waitcnt lgkmcnt(4)\n\t" \
  "v_mfma_f32_16x16x128_f8f6f4 v[220:223], v[112:119], a[184:191], v[220:223]\n\t" \
  "ds_read_b64_tr_b8 v[104:105], v87 offset:8192\n\t" \
  "s_add_u32 s50, s68, 0x100000\n\t" \
  "buffer_load_dwordx4 a[0:3], v4, s[12:15], s50 offen offset:0 sc0 sc1\n\t" \
  "v_mfma_f32_16x16x128_f8f6f4 v[216:219], v[112:119], a[176:183], v[216:219]\n\t" \
  "ds_read_b64_tr_b8 v[106:107], v87 offset:9216\n\t" \
  "v_mfma_f32_16x16x128_f8f6f4 v[212:215], v[112:119], a[168:175], v[212:215]\n\t" \
  "ds_read_b64_tr_b8 v[108:109], v103 offset:8192\n\t" \
  "s_add_u32 s51, s68, 0x100000\n\t" \
  "buffer_load_dwordx4 a[4:7], v4, s[12:15], s51 offen offset:1024 sc0 sc1\n\t" \
  "v_mfma_f32_16x16x128_f8f6f4 v[208:211], v[112:119], a[160:167], v[208:211]\n\t" \
  "ds_read_b64_tr_b8 v[110:111], v103 offset:9216\n\t" \
  "s_waitcnt vmcnt(22) lgkmcnt(0)\n\t" \
  "s_barrier\n\t" \
  "v_mfma_f32_16x16x128_f8f6f4 v[224:227], v[120:127], a[160:167], v[224:227]\n\t" \
  "ds_read_b64_tr_b8 v[112:113], v80 offset:16384\n\t" \
  "s_add_u32 s52, s68, 0x100000\n\t" \
  "buffer_load_dwordx4 a[8:11], v4, s[12:15], s52 offen offset:2048 sc0 sc1\n\t" \
  "v_mfma_f32_16x16x128_f8f6f4 v[228:231], v[120:127], a[168:175], v[228:231]\n\t" \
  "ds_read_b64_tr_b8 v[114:115], v80 offset:17408\n\t" \
  "v_mfma_f32_16x16x128_f8f6f4 v[232:235], v[120:127], a[176:183], v[232:235]\n\t" \
  "ds_read_b64_tr_b8 v[116:117], v96 offset:16384\n\t" \
  "s_add_u32 s53, s68, 0x100000\n\t" \
  "buffer_load_dwordx4 a[12:15], v4, s[12:15], s53 offen offset:3072 sc0 sc1\n\t" \
  "v_mfma_f32_16x16x128_f8f6f4 v[236:239], v[120:127], a[184:191], v[236:239]\n\t" \
  "ds_read_b64_tr_b8 v[118:119], v96 offset:17408\n\t" \
  "v_mfma_f32_16x16x128_f8f6f4 v[252:255], v[104:111], a[184:191], v[252:255]\n\t" \
  "ds_read_b64_tr_b8 v[120:121], v81 offset:16384\n\t" \
  "s_add_u32 s54, s68, 0x101000\n\t" \
  "buffer_load_dwordx4 a[16:19], v4, s[12:15], s54 offen offset:0 sc0 sc1\n\t" \
  "v_mfma_f32_16x16x128_f8f6f4 v[248:251], v[104:111], a[176:183], v[248:251]\n\t" \
  "ds_read_b64_tr_b8 v[122:123], v81 offset:17408\n\t" \
  "v_mfma_f32_16x16x128_f8f6f4 v[244:247], v[104:111], a[168:175], v[244:247]\n\t" \
  "ds_read_b64_tr_b8 v[124:125], v97 offset:16384\n\t" \
  "s_add_u32 s55, s68, 0x101000\n\t" \
  "buffer_load_dwordx4 a[20:23], v4, s[12:15], s55 offen offset:1024 sc0 sc1\n\t" \
  "v_mfma_f32_16x16x128_f8f6f4 v[240:243], v[104:111], a[160:167], v[240:243]\n\t" \
  "ds_read_b64_tr_b8 v[126:127], v97 offset:17408\n\t" \
  "s_waitcnt vmcnt(18) lgkmcnt(4)\n\t" \
  "v_mfma_f32_16x16x128_f8f6f4 v[128:131], v[112:119], a[192:199], v[128:131]\n\t" \
  "ds_read_b64_tr_b8 v[104:105], v82 offset:16384\n\t" \
  "s_add_u32 s48, s68, 0x101000\n\t" \
  "buffer_load_dwordx4 a[24:27], v4, s[12:15], s48 offen offset:2048 sc0 sc1\n\t" \
  "s_mov_b32 exec_lo, 0x7f0000\n\t" \
  "s_mov_b32 exec_hi, 0\n\t" \
  "global_load_dwordx4 v[68:71], v10, s[6:7] sc0 sc1\n\t" \
  "s_mov_b64 exec, -1\n\t" \
  "v_mfma_f32_16x16x128_f8f6f4 v[132:135], v[112:119], a[200:207], v[132:135]\n\t" \
  "ds_read_b64_tr_b8 v[106:107], v82 offset:17408\n\t" \
  "v_mfma_f32_16x16x128_f8f6f4 v[136:139], v[112:119], a[208:215], v[136:139]\n\t" \
  "ds_read_b64_tr_b8 v[108:109], v98 offset:16384\n\t" \
  "s_add_u32 s49, s68, 0x101000\n\t" \
  "buffer_load_dwordx4 a[28:31], v4, s[12:15], s49 offen offset:3072 sc0 sc1\n\t" \
  "v_mfma_f32_16x16x128_f8f6f4 v[140:143], v[112:119], a[216:223], v[140:143]\n\t" \
  "ds_read_b64_tr_b8 v[110:111], v98 offset:17408\n\t" \
  "s_waitcnt lgkmcnt(4)\n\t" \
  "v_mfma_f32_16x16x128_f8f6f4 v[156:159], v[120:127], a[216:223], v[156:159]\n\t" \
  "ds_read_b64_tr_b8 v[112:113], v83 offset:16384\n\t" \
  "s_add_u32 m0, s24, 0x10000\n\t" \
  "s_add_u32 s50, s42, 0x400000\n\t" \
  "buffer_load_dwordx4 v2, s[8:11], s50 offen lds\n\t" \
  "v_mfma_f32_16x16x128_f8f6f4 v[152:155], v[120:127], a[208:215], v[152:155]\n\t" \
  "ds_read_b64_tr_b8 v[114:115], v83 offset:17408\n\t" \
  "v_mfma_f32_16x16x128_f8f6f4 v[148:151], v[120:127], a[200:207], v[148:151]\n\t" \
  "ds_read_b64_tr_b8 v[116:117], v99 offset:16384\n\t" \
  "s_add_u32 m0, s24, 0x10400\n\t" \
  "s_add_u32 s51, s42, 0x420000\n\t" \
  "buffer_load_dwordx4 v2, s[8:11], s51 offen lds\n\t" \
  "v_mfma_f32_16x16x128_f8f6f4 v[144:147], v[120:127], a[192:199], v[144:147]\n\t" \
  "ds_read_b64_tr_b8 v[118:119], v99 offset:17408\n\t" \
  "s_waitcnt lgkmcnt(4)\n\t" \
  "v_mfma_f32_16x16x128_f8f6f4 v[160:163], v[104:111], a[192:199], v[160:163]\n\t" \
  "ds_read_b64_tr_b8 v[120:121], v84 offset:16384\n\t" \
  "s_add_u32 m0, s24, 0x12000\n\t" \
  "s_add_u32 s52, s43, 0x400000\n\t" \
  "buffer_load_dwordx4 v2, s[8:11], s52 offen lds\n\t" \
  "v_mfma_f32_16x16x128_f8f6f4 v[164:167], v[104:111], a[200:207], v[164:167]\n\t" \
  "ds_read_b64_tr_b8 v[122:123], v84 offset:17408\n\t" \
  "v_mfma_f32_16x16x128_f8f6f4 v[168:171], v[104:111], a[208:215], v[168:171]\n\t" \
  "ds_read_b64_tr_b8 v[124:125], v100 offset:16384\n\t" \
  "s_add_u32 m0, s24, 0x12400\n\t" \
  "s_add_u32 s53, s43, 0x420000\n\t" \
  "buffer_load_dwordx4 v2, s[8:11], s53 offen lds\n\t" \
  "v_mfma_f32_16x16x128_f8f6f4 v[172:175], v[104:111], a[216:223], v[172:175]\n\t" \
  "ds_read_b64_tr_b8 v[126:127], v100 offset:17408\n\t" \
  "s_waitcnt lgkmcnt(4)\n\t" \
  "v_mfma_f32_16x16x128_f8f6f4 v[188:191], v[112:119], a[216:223], v[188:191]\n\t" \
  "ds_read_b64_tr_b8 v[104:105], v85 offset:16384\n\t" \
  "s_add_u32 m0, s24, 0x14000\n\t" \
  "s_add_u32 s54, s44, 0x400000\n\t" \
  "buffer_load_dwordx4 v2, s[8:11], s54 offen lds\n\t" \
  "v_mfma_f32_16x16x128_f8f6f4 v[184:187], v[112:119], a[208:215], v[184:187]\n\t" \
  "ds_read_b64_tr_b8 v[106:107], v85 offset:17408\n\t" \
  "v_mfma_f32_16x16x128_f8f6f4 v[180:183], v[112:119], a[200:207], v[180:183]\n\t" \
  "ds_read_b64_tr_b8 v[108:109], v101 offset:16384\n\t" \
  "s_add_u32 m0, s24, 0x14400\n\t" \
  "s_add_u32 s55, s44, 0x420000\n\t" \
  "buffer_load_dwordx4 v2, s[8:11], s55 offen lds\n\t" \
  "v_mfma_f32_16x16x128_f8f6f4 v[176:179], v[112:119], a[192:199], v[176:179]\n\t" \
  "ds_read_b64_tr_b8 v[110:111], v101 offset:17408\n\t" \
  "s_waitcnt lgkmcnt(4)\n\t" \
  "v_mfma_f32_16x16x128_f8f6f4 v[192:195], v[120:127], a[192:199], v[192:195]\n\t" \
  "ds_read_b64_tr_b8 v[112:113], v86 offset:16384\n\t" \
  "s_add_u32 m0, s24, 0x16000\n\t" \
  "s_add_u32 s48, s45, 0x400000\n\t" \
  "buffer_load_dwordx4 v2, s[8:11], s48 offen lds\n\t" \
  "v_mfma_f32_16x16x128_f8f6f4 v[196:199], v[120:127], a[200:207], v[196:199]\n\t" \
  "ds_read_b64_tr_b8 v[114:115], v86 offset:17408\n\t" \
  "v_mfma_f32_16x16x128_f8f6f4 v[200:203], v[120:127], a[208:215], v[200:203]\n\t" \
  "ds_read_b64_tr_b8 v[116:117], v102 offset:16384\n\t" \
  "s_add_u32 m0, s24, 0x16400\n\t" \
  "s_add_u32 s49, s45, 0x420000\n\t" \
  "buffer_load_dwordx4 v2, s[8:11], s49 offen lds\n\t" \
  "v_mfma_f32_16x16x128_f8f6f4 v[204:207], v[120:127], a[216:223], v[204:207]\n\t" \
  "ds_read_b64_tr_b8 v[118:119], v102 offset:17408\n\t" \
  "s_waitcnt lgkmcnt(4)\n\t" \
  "v_mfma_f32_16x16x128_f8f6f4 v[220:223], v[104:111], a[216:223], v[220:223]\n\t" \
  "ds_read_b64_tr_b8 v[120:121], v87 offset:16384\n\t" \
  "v_mfma_f32_16x16x128_f8f6f4 v[216:219], v[104:111], a[208:215], v[216:219]\n\t" \
  "ds_read_b64_tr_b8 v[122:123], v87 offset:17408\n\t" \
  "v_mfma_f32_16x16x128_f8f6f4 v[212:215], v[104:111], a[200:207], v[212:215]\n\t" \
  "ds_read_b64_tr_b8 v[124:125], v103 offset:16384\n\t" \
  "v_mfma_f32_16x16x128_f8f6f4 v[208:211], v[104:111], a[192:199], v[208:211]\n\t" \
  "ds_read_b64_tr_b8 v[126:127], v103 offset:17408\n\t" \
  "s_waitcnt vmcnt(27) lgkmcnt(0)\n\t" \
  "s_barrier\n\t" \
  "v_mfma_f32_16x16x128_f8f6f4 v[224:227], v[112:119], a[192:199], v[224:227]\n\t" \
  "ds_read_b64_tr_b8 v[104:105], v80 offset:24576\n\t" \
  "v_mfma_f32_16x16x128_f8f6f4 v[228:231], v[112:119], a[200:207], v[228:231]\n\t" \
  "ds_read_b64_tr_b8 v[106:107], v80 offset:25600\n\t" \
  "v_mfma_f32_16x16x128_f8f6f4 v[232:235], v[112:119], a[208:215], v[232:235]\n\t" \
  "ds_read_b64_tr_b8 v[108:109], v96 offset:24576\n\t" \
  "v_mfma_f32_16x16x128_f8f6f4 v[236:239], v[112:119], a[216:223], v[236:239]\n\t" \
  "ds_read_b64_tr_b8 v[110:111], v96 offset:25600\n\t" \
  "v_mfma_f32_16x16x128_f8f6f4 v[252:255], v[120:127], a[216:223], v[252:255]\n\t" \
  "ds_read_b64_tr_b8 v[112:113], v81 offset:24576\n\t" \
  "v_mfma_f32_16x16x128_f8f6f4 v[248:251], v[120:127], a[208:215], v[248:251]\n\t" \
  "ds_read_b64_tr_b8 v[114:115], v81 offset:25600\n\t" \
  "v_mfma_f32_16x16x128_f8f6f4 v[244:247], v[120:127], a[200:207], v[244:247]\n\t" \
  "ds_read_b64_tr_b8 v[116:117], v97 offset:24576\n\t" \
  "v_mfma_f32_16x16x128_f8f6f4 v[240:243], v[120:127], a[192:199], v[240:243]\n\t" \
  "ds_read_b64_tr_b8 v[118:119], v97 offset:25600\n\t" \
  "s_waitcnt vmcnt(19) lgkmcnt(4)\n\t" \
  "v_mfma_f32_16x16x128_f8f6f4 v[128:131], v[104:111], a[224:231], v[128:131]\n\t" \
  "ds_read_b64_tr_b8 v[120:121], v82 offset:24576\n\t" \
  "v_mfma_f32_16x16x128_f8f6f4 v[132:135], v[104:111], a[232:239], v[132:135]\n\t" \
  "ds_read_b64_tr_b8 v[122:123], v82 offset:25600\n\t" \
  "v_mfma_f32_16x16x128_f8f6f4 v[136:139], v[104:111], a[240:247], v[136:139]\n\t" \
  "ds_read_b64_tr_b8 v[124:125], v98 offset:24576\n\t" \
  "s_add_u32 m0, s24, 0x1a000\n\t" \
  "s_add_u32 s50, s46, 0x200000\n\t" \
  "buffer_load_dwordx4 v3, s[8:11], s50 offen lds\n\t" \
  "v_mfma_f32_16x16x128_f8f6f4 v[140:143], v[104:111], a[248:255], v[140:143]\n\t" \
  "ds_read_b64_tr_b8 v[126:127], v98 offset:25600\n\t" \
  "s_waitcnt lgkmcnt(4)\n\t" \
  "v_mfma_f32_16x16x128_f8f6f4 v[156:159], v[112:119], a[248:255], v[156:159]\n\t" \
  "ds_read_b64_tr_b8 v[104:105], v83 offset:24576\n\t" \
  "s_add_u32 m0, s24, 0x1a400\n\t" \
  "s_add_u32 s51, s46, 0x220000\n\t" \
  "buffer_load_dwordx4 v3, s[8:11], s51 offen lds\n\t" \
  "v_mfma_f32_16x16x128_f8f6f4 v[152:155], v[112:119], a[240:247], v[152:155]\n\t" \
  "ds_read_b64_tr_b8 v[106:107], v83 offset:25600\n\t" \
  "v_mfma_f32_16x16x128_f8f6f4 v[148:151], v[112:119], a[232:239], v[148:151]\n\t" \
  "ds_read_b64_tr_b8 v[108:109], v99 offset:24576\n\t" \
  "s_add_u32 s52, s68, 0x110000\n\t" \
  "buffer_load_dwordx4 a[32:35], v4, s[12:15], s52 offen offset:0 sc0 sc1\n\t" \
  "v_mfma_f32_16x16x128_f8f6f4 v[144:147], v[112:119], a[224:231], v[144:147]\n\t" \
  "ds_read_b64_tr_b8 v[110:111], v99 offset:25600\n\t" \
  "s_waitcnt lgkmcnt(4)\n\t" \
  "v_mfma_f32_16x16x128_f8f6f4 v[160:163], v[120:127], a[224:231], v[160:163]\n\t" \
  "ds_read_b64_tr_b8 v[112:113], v84 offset:24576\n\t" \
  "s_add_u32 s53, s68, 0x110000\n\t" \
  "buffer_load_dwordx4 a[36:39], v4, s[12:15], s53 offen offset:1024 sc0 sc1\n\t" \
  "v_mfma_f32_16x16x128_f8f6f4 v[164:167], v[120:127], a[232:239], v[164:167]\n\t" \
  "ds_read_b64_tr_b8 v[114:115], v84 offset:25600\n\t" \
  "v_mfma_f32_16x16x128_f8f6f4 v[168:171], v[120:127], a[240:247], v[168:171]\n\t" \
  "ds_read_b64_tr_b8 v[116:117], v100 offset:24576\n\t" \
  "s_add_u32 s54, s68, 0x110000\n\t" \
  "buffer_load_dwordx4 a[40:43], v4, s[12:15], s54 offen offset:2048 sc0 sc1\n\t" \
  "v_mfma_f32_16x16x128_f8f6f4 v[172:175], v[120:127], a[248:255], v[172:175]\n\t" \
  "ds_read_b64_tr_b8 v[118:119], v100 offset:25600\n\t" \
  "s_waitcnt lgkmcnt(4)\n\t" \
  "v_mfma_f32_16x16x128_f8f6f4 v[188:191], v[104:111], a[248:255], v[188:191]\n\t" \
  "ds_read_b64_tr_b8 v[120:121], v85 offset:24576\n\t" \
  "s_add_u32 s55, s68, 0x110000\n\t" \
  "buffer_load_dwordx4 a[44:47], v4, s[12:15], s55 offen offset:3072 sc0 sc1\n\t" \
  "v_mfma_f32_16x16x128_f8f6f4 v[184:187], v[104:111], a[240:247], v[184:187]\n\t" \
  "ds_read_b64_tr_b8 v[122:123], v85 offset:25600\n\t" \
  "v_mfma_f32_16x16x128_f8f6f4 v[180:183], v[104:111], a[232:239], v[180:183]\n\t" \
  "ds_read_b64_tr_b8 v[124:125], v101 offset:24576\n\t" \
  "s_add_u32 s48, s68, 0x111000\n\t" \
  "buffer_load_dwordx4 a[48:51], v4, s[12:15], s48 offen offset:0 sc0 sc1\n\t" \
  "v_mfma_f32_16x16x128_f8f6f4 v[176:179], v[104:111], a[224:231], v[176:179]\n\t" \
  "ds_read_b64_tr_b8 v[126:127], v101 offset:25600\n\t" \
  "s_waitcnt lgkmcnt(4)\n\t" \
  "v_mfma_f32_16x16x128_f8f6f4 v[192:195], v[112:119], a[224:231], v[192:195]\n\t" \
  "ds_read_b64_tr_b8 v[104:105], v86 offset:24576\n\t" \
  "s_add_u32 s49, s68, 0x111000\n\t" \
  "buffer_load_dwordx4 a[52:55], v4, s[12:15], s49 offen offset:1024 sc0 sc1\n\t" \
  "v_mfma_f32_16x16x128_f8f6f4 v[196:199], v[112:119], a[232:239], v[196:199]\n\t" \
  "ds_read_b64_tr_b8 v[106:107], v86 offset:25600\n\t" \
  "v_mfma_f32_16x16x128_f8f6f4 v[200:203], v[112:119], a[240:247], v[200:203]\n\t" \
  "ds_read_b64_tr_b8 v[108:109], v102 offset:24576\n\t" \
  "s_add_u32 s50, s68, 0x111000\n\t" \
  "buffer_load_dwordx4 a[56:59], v4, s[12:15], s50 offen offset:2048 sc0 sc1\n\t" \
  "v_mfma_f32_16x16x128_f8f6f4 v[204:207], v[112:119], a[248:255], v[204:207]\n\t" \
  "ds_read_b64_tr_b8 v[110:111], v102 offset:25600\n\t" \
  "s_waitcnt lgkmcnt(4)\n\t" \
  "v_mfma_f32_16x16x128_f8f6f4 v[220:223], v[120:127], a[248:255], v[220:223]\n\t" \
  "ds_read_b64_tr_b8 v[112:113], v87 offset:24576\n\t" \
  "s_add_u32 s51, s68, 0x111000\n\t" \
  "buffer_load_dwordx4 a[60:63], v4, s[12:15], s51 offen offset:3072 sc0 sc1\n\t" \
  "v_mfma_f32_16x16x128_f8f6f4 v[216:219], v[120:127], a[240:247], v[216:219]\n\t" \
  "ds_read_b64_tr_b8 v[114:115], v87 offset:25600\n\t" \
  "v_mfma_f32_16x16x128_f8f6f4 v[212:215], v[120:127], a[232:239], v[212:215]\n\t" \
  "ds_read_b64_tr_b8 v[116:117], v103 offset:24576\n\t" \
  "s_add_u32 m0, s24, 0x1c000\n\t" \
  "s_add_u32 s52, s40, 0x400000\n\t" \
  "buffer_load_dwordx4 v3, s[8:11], s52 offen lds\n\t" \
  "v_mfma_f32_16x16x128_f8f6f4 v[208:211], v[120:127], a[224:231], v[208:211]\n\t" \
  "ds_read_b64_tr_b8 v[118:119], v103 offset:25600\n\t" \
  "s_waitcnt vmcnt(28) lgkmcnt(0)\n\t" \
  "s_barrier\n\t" \
  "v_mfma_f32_16x16x128_f8f6f4 v[224:227], v[104:111], a[224:231], v[224:227]\n\t" \
  "ds_read_b64_tr_b8 v[120:121], v80 offset:32768\n\t" \
  "s_add_u32 m0, s24, 0x1c400\n\t" \
  "s_add_u32 s53, s40, 0x420000\n\t" \
  "buffer_load_dwordx4 v3, s[8:11], s53 offen lds\n\t" \
  "v_mfma_f32_16x16x128_f8f6f4 v[228:231], v[104:111], a[232:239], v[228:231]\n\t" \
  "ds_read_b64_tr_b8 v[122:123], v80 offset:33792\n\t" \
  "v_mfma_f32_16x16x128_f8f6f4 v[232:235], v[104:111], a[240:247], v[232:235]\n\t" \
  "ds_read_b64_tr_b8 v[124:125], v88 offset:32768\n\t" \
  "s_waitcnt vmcnt(21)\n\t" \
  "v_min_u32 v68, v68, v69\n\t" \
  "v_min3_u32 v68, v68, v70, v71\n\t" \
  "v_cmp_le_u32 vcc, s57, v68\n\t" \
  "s_and_b32 s64, vcc_lo, 0x7f0000\n\t" \
  "s_or_b32 s56, s56, s64\n\t" \
  "s_and_b32 s64, s56, 0x7f0000\n\t" \
  "s_cmp_eq_u32 s64, 0x7f0000\n\t" \
  "s_cbranch_scc1 Lok8_1_%=\n\t" \
  "s_mov_b32 s62, 0x7f0000\n\t" \
  "s_mov_b32 s69, 0x7f0000\n\t" \
  "s_call_b64 s[60:61], Lpoll_1_%=\n\t" \
  "Lok8_1_%=:\n\t" \
  "s_add_u32 s54, s68, 0x120000\n\t" \
  "buffer_load_dwordx4 a[64:67], v4, s[12:15], s54 offen offset:0 sc0 sc1\n\t" \
  "v_mfma_f32_16x16x128_f8f6f4 v[236:239], v[104:111], a[248:255], v[236:239]\n\t" \
  "ds_read_b64_tr_b8 v[126:127], v88 offset:33792\n\t" \
  "v_mfma_f32_16x16x128_f8f6f4 v[252:255], v[112:119], a[248:255], v[252:255]\n\t" \
  "ds_read_b64_tr_b8 v[104:105], v81 offset:32768\n\t" \
  "s_add_u32 s55, s68, 0x120000\n\t" \
  "buffer_load_dwordx4 a[68:71], v4, s[12:15], s55 offen offset:1024 sc0 sc1\n\t" \
  "v_mfma_f32_16x16x128_f8f6f4 v[248:251], v[112:119], a[240:247], v[248:251]\n\t" \
  "ds_read_b64_tr_b8 v[106:107], v81 offset:33792\n\t" \
  "v_mfma_f32_16x16x128_f8f6f4 v[244:247], v[112:119], a[232:239], v[244:247]\n\t" \
  "ds_read_b64_tr_b8 v[108:109], v89 offset:32768\n\t" \
  "s_add_u32 s48, s68, 0x120000\n\t" \
  "buffer_load_dwordx4 a[72:75], v4, s[12:15], s48 offen offset:2048 sc0 sc1\n\t" \
  "v_mfma_f32_16x16x128_f8f6f4 v[240:243], v[112:119], a[224:231], v[240:243]\n\t" \
  "ds_read_b64_tr_b8 v[110:111], v89 offset:33792\n\t" \
  "s_waitcnt vmcnt(23) lgkmcnt(4)\n\t" \
  "v_mfma_f32_16x16x128_f8f6f4 v[128:131], v[120:127], a[0:7], v[128:131]\n\t" \
  "ds_read_b64_tr_b8 v[112:113], v82 offset:32768\n\t" \
  "s_add_u32 s49, s68, 0x120000\n\t" \
  "buffer_load_dwordx4 a[76:79], v4, s[12:15], s49 offen offset:3072 sc0 sc1\n\t" \
  "v_mfma_f32_16x16x128_f8f6f4 v[132:135], v[120:127], a[8:15], v[132:135]\n\t" \
  "ds_read_b64_tr_b8 v[114:115], v82 offset:33792\n\t" \
  "v_mfma_f32_16x16x128_f8f6f4 v[136:139], v[120:127], a[16:23], v[136:139]\n\t" \
  "ds_read_b64_tr_b8 v[116:117], v90 offset:32768\n\t" \
  "s_add_u32 s50, s68, 0x121000\n\t" \
  "buffer_load_dwordx4 a[80:83], v4, s[12:15], s50 offen offset:0 sc0 sc1\n\t" \
  "v_mfma_f32_16x16x128_f8f6f4 v[140:143], v[120:127], a[24:31], v[140:143]\n\t" \
  "ds_read_b64_tr_b8 v[118:119], v90 offset:33792\n\t" \
  "s_waitcnt lgkmcnt(4)\n\t" \
  "v_mfma_f32_16x16x128_f8f6f4 v[156:159], v[104:111], a[24:31], v[156:159]\n\t" \
  "ds_read_b64_tr_b8 v[120:121], v83 offset:32768\n\t" \
  "s_add_u32 s51, s68, 0x121000\n\t" \
  "buffer_load_dwordx4 a[84:87], v4, s[12:15], s51 offen offset:1024 sc0 sc1\n\t" \
  "v_mfma_f32_16x16x128_f8f6f4 v[152:155], v[104:111], a[16:23], v[152:155]\n\t" \
  "ds_read_b64_tr_b8 v[122:123], v83 offset:33792\n\t" \
  "v_mfma_f32_16x16x128_f8f6f4 v[148:151], v[104:111], a[8:15], v[148:151]\n\t" \
  "ds_read_b64_tr_b8 v[124:125], v91 offset:32768\n\t" \
  "s_add_u32 s52, s68, 0x121000\n\t" \
  "buffer_load_dwordx4 a[88:91], v4, s[12:15], s52 offen offset:2048 sc0 sc1\n\t" \
  "v_mfma_f32_16x16x128_f8f6f4 v[144:147], v[104:111], a[0:7], v[144:147]\n\t" \
  "ds_read_b64_tr_b8 v[126:127], v91 offset:33792\n\t" \
  "s_waitcnt lgkmcnt(4)\n\t" \
  "v_mfma_f32_16x16x128_f8f6f4 v[160:163], v[112:119], a[0:7], v[160:163]\n\t" \
  "ds_read_b64_tr_b8 v[104:105], v84 offset:32768\n\t" \
  "s_add_u32 s53, s68, 0x121000\n\t" \
  "buffer_load_dwordx4 a[92:95], v4, s[12:15], s53 offen offset:3072 sc0 sc1\n\t" \
  "v_mfma_f32_16x16x128_f8f6f4 v[164:167], v[112:119], a[8:15], v[164:167]\n\t" \
  "ds_read_b64_tr_b8 v[106:107], v84 offset:33792\n\t" \
  "v_mfma_f32_16x16x128_f8f6f4 v[168:171], v[112:119], a[16:23], v[168:171]\n\t" \
  "ds_read_b64_tr_b8 v[108:109], v92 offset:32768\n\t" \
  "s_add_u32 m0, s24, 0x1e000\n\t" \
  "s_add_u32 s54, s41, 0x400000\n\t" \
  "buffer_load_dwordx4 v3, s[8:11], s54 offen lds\n\t" \
  "v_mfma_f32_16x16x128_f8f6f4 v[172:175], v[112:119], a[24:31], v[172:175]\n\t" \
  "ds_read_b64_tr_b8 v[110:111], v92 offset:33792\n\t" \
  "s_waitcnt lgkmcnt(4)\n\t" \
  "v_mfma_f32_16x16x128_f8f6f4 v[188:191], v[120:127], a[24:31], v[188:191]\n\t" \
  "ds_read_b64_tr_b8 v[112:113], v85 offset:32768\n\t" \
  "s_add_u32 m0, s24, 0x1e400\n\t" \
  "s_add_u32 s55, s41, 0x420000\n\t" \
  "buffer_load_dwordx4 v3, s[8:11], s55 offen lds\n\t" \
  "v_mfma_f32_16x16x128_f8f6f4 v[184:187], v[120:127], a[16:23], v[184:187]\n\t" \
  "ds_read_b64_tr_b8 v[114:115], v85 offset:33792\n\t" \
  "v_mfma_f32_16x16x128_f8f6f4 v[180:183], v[120:127], a[8:15], v[180:183]\n\t" \
  "ds_read_b64_tr_b8 v[116:117], v93 offset:32768\n\t" \
  "s_add_u32 s48, s68, 0x130000\n\t" \
  "buffer_load_dwordx4 a[96:99], v4, s[12:15], s48 offen offset:0 sc0 sc1\n\t" \
  "v_mfma_f32_16x16x128_f8f6f4 v[176:179], v[120:127], a[0:7], v[176:179]\n\t" \
  "ds_read_b64_tr_b8 v[118:119], v93 offset:33792\n\t" \
  "s_waitcnt lgkmcnt(4)\n\t" \
  "v_mfma_f32_16x16x128_f8f6f4 v[192:195], v[104:111], a[0:7], v[192:195]\n\t" \
  "ds_read_b64_tr_b8 v[120:121], v86 offset:32768\n\t" \
  "s_add_u32 s49, s68, 0x130000\n\t" \
  "buffer_load_dwordx4 a[100:103], v4, s[12:15], s49 offen offset:1024 sc0 sc1\n\t" \
  "v_mfma_f32_16x16x128_f8f6f4 v[196:199], v[104:111], a[8:15], v[196:199]\n\t" \
  "ds_read_b64_tr_b8 v[122:123], v86 offset:33792\n\t" \
  "v_mfma_f32_16x16x128_f8f6f4 v[200:203], v[104:111], a[16:23], v[200:203]\n\t" \
  "ds_read_b64_tr_b8 v[124:125], v94 offset:32768\n\t" \
  "s_add_u32 s50, s68, 0x130000\n\t" \
  "buffer_load_dwordx4 a[104:107], v4, s[12:15], s50 offen offset:2048 sc0 sc1\n\t" \
  "v_mfma_f32_16x16x128_f8f6f4 v[204:207], v[104:111], a[24:31], v[204:207]\n\t" \
  "ds_read_b64_tr_b8 v[126:127], v94 offset:33792\n\t" \
  "s_waitcnt lgkmcnt(4)\n\t" \
  "v_mfma_f32_16x16x128_f8f6f4 v[220:223], v[112:119], a[24:31], v[220:223]\n\t" \
  "ds_read_b64_tr_b8 v[104:105], v87 offset:32768\n\t" \
  "s_add_u32 s51, s68, 0x130000\n\t" \
  "buffer_load_dwordx4 a[108:111], v4, s[12:15], s51 offen offset:3072 sc0 sc1\n\t" \
  "v_mfma_f32_16x16x128_f8f6f4 v[216:219], v[112:119], a[16:23], v[216:219]\n\t" \
  "ds_read_b64_tr_b8 v[106:107], v87 offset:33792\n\t" \
  "v_mfma_f32_16x16x128_f8f6f4 v[212:215], v[112:119], a[8:15], v[212:215]\n\t" \
  "ds_read_b64_tr_b8 v[108:109], v95 offset:32768\n\t" \
  "s_add_u32 s52, s68, 0x131000\n\t" \
  "buffer_load_dwordx4 a[112:115], v4, s[12:15], s52 offen offset:0 sc0 sc1\n\t" \
  "v_mfma_f32_16x16x128_f8f6f4 v[208:211], v[112:119], a[0:7], v[208:211]\n\t" \
  "ds_read_b64_tr_b8 v[110:111], v95 offset:33792\n\t" \
  "s_waitcnt vmcnt(25) lgkmcnt(0)\n\t" \
  "s_barrier\n\t" \
  "v_mfma_f32_16x16x128_f8f6f4 v[224:227], v[120:127], a[0:7], v[224:227]\n\t" \
  "ds_read_b64_tr_b8 v[112:113], v80 offset:40960\n\t" \
  "s_add_u32 s53, s68, 0x131000\n\t" \
  "buffer_load_dwordx4 a[116:119], v4, s[12:15], s53 offen offset:1024 sc0 sc1\n\t" \
  "v_mfma_f32_16x16x128_f8f6f4 v[228:231], v[120:127], a[8:15], v[228:231]\n\t" \
  "ds_read_b64_tr_b8 v[114:115], v80 offset:41984\n\t" \
  "v_mfma_f32_16x16x128_f8f6f4 v[232:235], v[120:127], a[16:23], v[232:235]\n\t" \
  "ds_read_b64_tr_b8 v[116:117], v88 offset:40960\n\t" \
  "s_add_u32 s54, s68, 0x131000\n\t" \
  "buffer_load_dwordx4 a[120:123], v4, s[12:15], s54 offen offset:2048 sc0 sc1\n\t" \
  "v_mfma_f32_16x16x128_f8f6f4 v[236:239], v[120:127], a[24:31], v[236:239]\n\t" \
  "ds_read_b64_tr_b8 v[118:119], v88 offset:41984\n\t" \
  "v_mfma_f32_16x16x128_f8f6f4 v[252:255], v[104:111], a[24:31], v[252:255]\n\t" \
  "ds_read_b64_tr_b8 v[120:121], v81 offset:40960\n\t" \
  "s_add_u32 s55, s68, 0x131000\n\t" \
  "buffer_load_dwordx4 a[124:127], v4, s[12:15], s55 offen offset:3072 sc0 sc1\n\t" \
  "v_mfma_f32_16x16x128_f8f6f4 v[248:251], v[104:111], a[16:23], v[248:251]\n\t" \
  "ds_read_b64_tr_b8 v[122:123], v81 offset:41984\n\t" \
  "v_mfma_f32_16x16x128_f8f6f4 v[244:247], v[104:111], a[8:15], v[244:247]\n\t" \
  "ds_read_b64_tr_b8 v[124:125], v89 offset:40960\n\t" \
  "s_add_u32 m0, s24, 0x0\n\t" \
  "s_add_u32 s48, s46, 0x400000\n\t" \
  "buffer_load_dwordx4 v2, s[8:11], s48 offen lds\n\t" \
  "v_mfma_f32_16x16x128_f8f6f4 v[240:243], v[104:111], a[0:7], v[240:243]\n\t" \
  "ds_read_b64_tr_b8 v[126:127], v89 offset:41984\n\t" \
  "s_waitcnt vmcnt(21) lgkmcnt(4)\n\t" \
  "v_mfma_f32_16x16x128_f8f6f4 v[128:131], v[112:119], a[32:39], v[128:131]\n\t" \
  "ds_read_b64_tr_b8 v[104:105], v82 offset:40960\n\t" \
  "s_add_u32 m0, s24, 0x400\n\t" \
  "s_add_u32 s49, s46, 0x420000\n\t" \
  "buffer_load_dwordx4 v2, s[8:11], s49 offen lds\n\t" \
  "v_mfma_f32_16x16x128_f8f6f4 v[132:135], v[112:119], a[40:47], v[132:135]\n\t" \
  "ds_read_b64_tr_b8 v[106:107], v82 offset:41984\n\t" \
  "v_mfma_f32_16x16x128_f8f6f4 v[136:139], v[112:119], a[48:55], v[136:139]\n\t" \
  "ds_read_b64_tr_b8 v[108:109], v90 offset:40960\n\t" \
  "s_add_u32 m0, s24, 0x2000\n\t" \
  "s_add_u32 s50, s40, 0x600000\n\t" \
  "buffer_load_dwordx4 v2, s[8:11], s50 offen lds\n\t" \
  "v_mfma_f32_16x16x128_f8f6f4 v[140:143], v[112:119], a[56:63], v[140:143]\n\t" \
  "ds_read_b64_tr_b8 v[110:111], v90 offset:41984\n\t" \
  "s_waitcnt lgkmcnt(4)\n\t" \
  "v_mfma_f32_16x16x128_f8f6f4 v[156:159], v[120:127], a[56:63], v[156:159]\n\t" \
  "ds_read_b64_tr_b8 v[112:113], v83 offset:40960\n\t" \
  "s_add_u32 m0, s24, 0x2400\n\t" \
  "s_add_u32 s51, s40, 0x620000\n\t" \
  "buffer_load_dwordx4 v2, s[8:11], s51 offen lds\n\t" \
  "v_mfma_f32_16x16x128_f8f6f4 v[152:155], v[120:127], a[48:55], v[152:155]\n\t" \
  "ds_read_b64_tr_b8 v[114:115], v83 offset:41984\n\t" \
  "v_mfma_f32_16x16x128_f8f6f4 v[148:151], v[120:127], a[40:47], v[148:151]\n\t" \
  "ds_read_b64_tr_b8 v[116:117], v91 offset:40960\n\t" \
  "s_add_u32 m0, s24, 0x4000\n\t" \
  "s_add_u32 s52, s41, 0x600000\n\t" \
  "buffer_load_dwordx4 v2, s[8:11], s52 offen lds\n\t" \
  "v_mfma_f32_16x16x128_f8f6f4 v[144:147], v[120:127], a[32:39], v[144:147]\n\t" \
  "ds_read_b64_tr_b8 v[118:119], v91 offset:41984\n\t" \
  "s_waitcnt lgkmcnt(4)\n\t" \
  "v_mfma_f32_16x16x128_f8f6f4 v[160:163], v[104:111], a[32:39], v[160:163]\n\t" \
  "ds_read_b64_tr_b8 v[120:121], v84 offset:40960\n\t" \
  "s_add_u32 m0, s24, 0x4400\n\t" \
  "s_add_u32 s53, s41, 0x620000\n\t" \
  "buffer_load_dwordx4 v2, s[8:11], s53 offen lds\n\t" \
  "v_mfma_f32_16x16x128_f8f6f4 v[164:167], v[104:111], a[40:47], v[164:167]\n\t" \
  "ds_read_b64_tr_b8 v[122:123], v84 offset:41984\n\t" \
  "v_mfma_f32_16x16x128_f8f6f4 v[168:171], v[104:111], a[48:55], v[168:171]\n\t" \
  "ds_read_b64_tr_b8 v[124:125], v92 offset:40960\n\t" \
  "s_add_u32 m0, s24, 0x6000\n\t" \
  "s_add_u32 s54, s42, 0x600000\n\t" \
  "buffer_load_dwordx4 v2, s[8:11], s54 offen lds\n\t" \
  "v_mfma_f32_16x16x128_f8f6f4 v[172:175], v[104:111], a[56:63], v[172:175]\n\t" \
  "ds_read_b64_tr_b8 v[126:127], v92 offset:41984\n\t" \
  "s_waitcnt lgkmcnt(4)\n\t" \
  "v_mfma_f32_16x16x128_f8f6f4 v[188:191], v[112:119], a[56:63], v[188:191]\n\t" \
  "ds_read_b64_tr_b8 v[104:105], v85 offset:40960\n\t" \
  "s_add_u32 m0, s24, 0x6400\n\t" \
  "s_add_u32 s55, s42, 0x620000\n\t" \
  "buffer_load_dwordx4 v2, s[8:11], s55 offen lds\n\t" \
  "v_mfma_f32_16x16x128_f8f6f4 v[184:187], v[112:119], a[48:55], v[184:187]\n\t" \
  "ds_read_b64_tr_b8 v[106:107], v85 offset:41984\n\t" \
  "v_mfma_f32_16x16x128_f8f6f4 v[180:183], v[112:119], a[40:47], v[180:183]\n\t" \
  "ds_read_b64_tr_b8 v[108:109], v93 offset:40960\n\t" \
  "v_mfma_f32_16x16x128_f8f6f4 v[176:179], v[112:119], a[32:39], v[176:179]\n\t" \
  "ds_read_b64_tr_b8 v[110:111], v93 offset:41984\n\t" \
  "s_waitcnt lgkmcnt(4)\n\t" \
  "v_mfma_f32_16x16x128_f8f6f4 v[192:195], v[120:127], a[32:39], v[192:195]\n\t" \
  "ds_read_b64_tr_b8 v[112:113], v86 offset:40960\n\t" \
  "v_mfma_f32_16x16x128_f8f6f4 v[196:199], v[120:127], a[40:47], v[196:199]\n\t" \
  "ds_read_b64_tr_b8 v[114:115], v86 offset:41984\n\t" \
  "v_mfma_f32_16x16x128_f8f6f4 v[200:203], v[120:127], a[48:55], v[200:203]\n\t" \
  "ds_read_b64_tr_b8 v[116:117], v94 offset:40960\n\t" \
  "v_mfma_f32_16x16x128_f8f6f4 v[204:207], v[120:127], a[56:63], v[204:207]\n\t" \
  "ds_read_b64_tr_b8 v[118:119], v94 offset:41984\n\t" \
  "s_waitcnt lgkmcnt(4)\n\t" \
  "v_mfma_f32_16x16x128_f8f6f4 v[220:223], v[104:111], a[56:63], v[220:223]\n\t" \
  "ds_read_b64_tr_b8 v[120:121], v87 offset:40960\n\t" \
  "v_mfma_f32_16x16x128_f8f6f4 v[216:219], v[104:111], a[48:55], v[216:219]\n\t" \
  "ds_read_b64_tr_b8 v[122:123], v87 offset:41984\n\t" \
  "v_mfma_f32_16x16x128_f8f6f4 v[212:215], v[104:111], a[40:47], v[212:215]\n\t" \
  "ds_read_b64_tr_b8 v[124:125], v95 offset:40960\n\t" \
  "v_mfma_f32_16x16x128_f8f6f4 v[208:211], v[104:111], a[32:39], v[208:211]\n\t" \
  "ds_read_b64_tr_b8 v[126:127], v95 offset:41984\n\t" \
  "s_waitcnt vmcnt(26) lgkmcnt(0)\n\t" \
  "s_barrier\n\t" \
  "v_mfma_f32_16x16x128_f8f6f4 v[224:227], v[112:119], a[32:39], v[224:227]\n\t" \
  "ds_read_b64_tr_b8 v[104:105], v80 offset:49152\n\t" \
  "v_mfma_f32_16x16x128_f8f6f4 v[228:231], v[112:119], a[40:47], v[228:231]\n\t" \
  "ds_read_b64_tr_b8 v[106:107], v80 offset:50176\n\t" \
  "v_mfma_f32_16x16x128_f8f6f4 v[232:235], v[112:119], a[48:55], v[232:235]\n\t" \
  "ds_read_b64_tr_b8 v[108:109], v88 offset:49152\n\t" \
  "v_mfma_f32_16x16x128_f8f6f4 v[236:239], v[112:119], a[56:63], v[236:239]\n\t" \
  "ds_read_b64_tr_b8 v[110:111], v88 offset:50176\n\t" \
  "v_mfma_f32_16x16x128_f8f6f4 v[252:255], v[120:127], a[56:63], v[252:255]\n\t" \
  "ds_read_b64_tr_b8 v[112:113], v81 offset:49152\n\t" \
  "s_add_u32 m0, s24, 0x20000\n\t" \
  "s_add_u32 s48, s42, 0x400000\n\t" \
  "buffer_load_dwordx4 v3, s[8:11], s48 offen lds\n\t" \
  "v_mfma_f32_16x16x128_f8f6f4 v[248:251], v[120:127], a[48:55], v[248:251]\n\t" \
  "ds_read_b64_tr_b8 v[114:115], v81 offset:50176\n\t" \
  "v_mfma_f32_16x16x128_f8f6f4 v[244:247], v[120:127], a[40:47], v[244:247]\n\t" \
  "ds_read_b64_tr_b8 v[116:117], v89 offset:49152\n\t" \
  "s_add_u32 m0, s24, 0x20400\n\t" \
  "s_add_u32 s49, s42, 0x420000\n\t" \
  "buffer_load_dwordx4 v3, s[8:11], s49 offen lds\n\t" \
  "v_mfma_f32_16x16x128_f8f6f4 v[240:243], v[120:127], a[32:39], v[240:243]\n\t" \
  "ds_read_b64_tr_b8 v[118:119], v89 offset:50176\n\t" \
  "s_waitcnt vmcnt(20) lgkmcnt(4)\n\t" \
  "v_mfma_f32_16x16x128_f8f6f4 v[128:131], v[104:111], a[64:71], v[128:131]\n\t" \
  "ds_read_b64_tr_b8 v[120:121], v82 offset:49152\n\t" \
  "s_add_u32 s50, s68, 0x140000\n\t" \
  "buffer_load_dwordx4 a[128:131], v4, s[12:15], s50 offen offset:0 sc0 sc1\n\t" \
  "v_mfma_f32_16x16x128_f8f6f4 v[132:135], v[104:111], a[72:79], v[132:135]\n\t" \
  "ds_read_b64_tr_b8 v[122:123], v82 offset:50176\n\t" \
  "v_mfma_f32_16x16x128_f8f6f4 v[136:139], v[104:111], a[80:87], v[136:139]\n\t" \
  "ds_read_b64_tr_b8 v[124:125], v90 offset:49152\n\t" \
  "s_add_u32 s51, s68, 0x140000\n\t" \
  "buffer_load_dwordx4 a[132:135], v4, s[12:15], s51 offen offset:1024 sc0 sc1\n\t" \
  "v_mfma_f32_16x16x128_f8f6f4 v[140:143], v[104:111], a[88:95], v[140:143]\n\t" \
  "ds_read_b64_tr_b8 v[126:127], v90 offset:50176\n\t" \
  "s_waitcnt lgkmcnt(4)\n\t" \
  "v_mfma_f32_16x16x128_f8f6f4 v[156:159], v[112:119], a[88:95], v[156:159]\n\t" \
  "ds_read_b64_tr_b8 v[104:105], v83 offset:49152\n\t" \
  "s_add_u32 s52, s68, 0x140000\n\t" \
  "buffer_load_dwordx4 a[136:139], v4, s[12:15], s52 offen offset:2048 sc0 sc1\n\t" \
  "v_mfma_f32_16x16x128_f8f6f4 v[152:155], v[112:119], a[80:87], v[152:155]\n\t" \
  "ds_read_b64_tr_b8 v[106:107], v83 offset:50176\n\t" \
  "v_mfma_f32_16x16x128_f8f6f4 v[148:151], v[112:119], a[72:79], v[148:151]\n\t" \
  "ds_read_b64_tr_b8 v[108:109], v91 offset:49152\n\t" \
  "s_add_u32 s53, s68, 0x140000\n\t" \
  "buffer_load_dwordx4 a[140:143], v4, s[12:15], s53 offen offset:3072 sc0 sc1\n\t" \
  "v_mfma_f32_16x16x128_f8f6f4 v[144:147], v[112:119], a[64:71], v[144:147]\n\t" \
  "ds_read_b64_tr_b8 v[110:111], v91 offset:50176\n\t" \
  "s_waitcnt lgkmcnt(4)\n\t" \
  "v_mfma_f32_16x16x128_f8f6f4 v[160:163], v[120:127], a[64:71], v[160:163]\n\t" \
  "ds_read_b64_tr_b8 v[112:113], v84 offset:49152\n\t" \
  "s_add_u32 s54, s68, 0x141000\n\t" \
  "buffer_load_dwordx4 a[144:147], v4, s[12:15], s54 offen offset:0 sc0 sc1\n\t" \
  "v_mfma_f32_16x16x128_f8f6f4 v[164:167], v[120:127], a[72:79], v[164:167]\n\t" \
  "ds_read_b64_tr_b8 v[114:115], v84 offset:50176\n\t" \
  "v_mfma_f32_16x16x128_f8f6f4 v[168:171], v[120:127], a[80:87], v[168:171]\n\t" \
  "ds_read_b64_tr_b8 v[116:117], v92 offset:49152\n\t" \
  "s_add_u32 s55, s68, 0x141000\n\t" \
  "buffer_load_dwordx4 a[148:151], v4, s[12:15], s55 offen offset:1024 sc0 sc1\n\t" \
  "v_mfma_f32_16x16x128_f8f6f4 v[172:175], v[120:127], a[88:95], v[172:175]\n\t" \
  "ds_read_b64_tr_b8 v[118:119], v92 offset:50176\n\t" \
  "s_waitcnt lgkmcnt(4)\n\t" \
  "v_mfma_f32_16x16x128_f8f6f4 v[188:191], v[104:111], a[88:95], v[188:191]\n\t" \
  "ds_read_b64_tr_b8 v[120:121], v85 offset:49152\n\t" \
  "s_add_u32 s48, s68, 0x141000\n\t" \
  "buffer_load_dwordx4 a[152:155], v4, s[12:15], s48 offen offset:2048 sc0 sc1\n\t" \
  "v_mfma_f32_16x16x128_f8f6f4 v[184:187], v[104:111], a[80:87], v[184:187]\n\t" \
  "ds_read_b64_tr_b8 v[122:123], v85 offset:50176\n\t" \
  "v_mfma_f32_16x16x128_f8f6f4 v[180:183], v[104:111], a[72:79], v[180:183]\n\t" \
  "ds_read_b64_tr_b8 v[124:125], v93 offset:49152\n\t" \
  "s_add_u32 s49, s68, 0x141000\n\t" \
  "buffer_load_dwordx4 a[156:159], v4, s[12:15], s49 offen offset:3072 sc0 sc1\n\t" \
  "v_mfma_f32_16x16x128_f8f6f4 v[176:179], v[104:111], a[64:71], v[176:179]\n\t" \
  "ds_read_b64_tr_b8 v[126:127], v93 offset:50176\n\t" \
  "s_waitcnt lgkmcnt(4)\n\t" \
  "v_mfma_f32_16x16x128_f8f6f4 v[192:195], v[112:119], a[64:71], v[192:195]\n\t" \
  "ds_read_b64_tr_b8 v[104:105], v86 offset:49152\n\t" \
  "s_add_u32 m0, s24, 0x22000\n\t" \
  "s_add_u32 s50, s43, 0x400000\n\t" \
  "buffer_load_dwordx4 v3, s[8:11], s50 offen lds\n\t" \
  "v_mfma_f32_16x16x128_f8f6f4 v[196:199], v[112:119], a[72:79], v[196:199]\n\t" \
  "ds_read_b64_tr_b8 v[106:107], v86 offset:50176\n\t" \
  "v_mfma_f32_16x16x128_f8f6f4 v[200:203], v[112:119], a[80:87], v[200:203]\n\t" \
  "ds_read_b64_tr_b8 v[108:109], v94 offset:49152\n\t" \
  "s_add_u32 m0, s24, 0x22400\n\t" \
  "s_add_u32 s51, s43, 0x420000\n\t" \
  "buffer_load_dwordx4 v3, s[8:11], s51 offen lds\n\t" \
  "v_mfma_f32_16x16x128_f8f6f4 v[204:207], v[112:119], a[88:95], v[204:207]\n\t" \
  "ds_read_b64_tr_b8 v[110:111], v94 offset:50176\n\t" \
  "s_waitcnt lgkmcnt(4)\n\t" \
  "v_mfma_f32_16x16x128_f8f6f4 v[220:223], v[120:127], a[88:95], v[220:223]\n\t" \
  "ds_read_b64_tr_b8 v[112:113], v87 offset:49152\n\t" \
  "s_add_u32 s52, s68, 0x150000\n\t" \
  "buffer_load_dwordx4 a[160:163], v4, s[12:15], s52 offen offset:0 sc0 sc1\n\t" \
  "v_mfma_f32_16x16x128_f8f6f4 v[216:219], v[120:127], a[80:87], v[216:219]\n\t" \
  "ds_read_b64_tr_b8 v[114:115], v87 offset:50176\n\t" \
  "v_mfma_f32_16x16x128_f8f6f4 v[212:215], v[120:127], a[72:79], v[212:215]\n\t" \
  "ds_read_b64_tr_b8 v[116:117], v95 offset:49152\n\t" \
  "s_add_u32 s53, s68, 0x150000\n\t" \
  "buffer_load_dwordx4 a[164:167], v4, s[12:15], s53 offen offset:1024 sc0 sc1\n\t" \
  "v_mfma_f32_16x16x128_f8f6f4 v[208:211], v[120:127], a[64:71], v[208:211]\n\t" \
  "ds_read_b64_tr_b8 v[118:119], v95 offset:50176\n\t" \
  "s_waitcnt vmcnt(30) lgkmcnt(0)\n\t" \
  "s_barrier\n\t" \
  "v_mfma_f32_16x16x128_f8f6f4 v[224:227], v[104:111], a[64:71], v[224:227]\n\t" \
  "ds_read_b64_tr_b8 v[120:121], v80 offset:57344\n\t" \
  "s_add_u32 s54, s68, 0x150000\n\t" \
  "buffer_load_dwordx4 a[168:171], v4, s[12:15], s54 offen offset:2048 sc0 sc1\n\t" \
  "v_mfma_f32_16x16x128_f8f6f4 v[228:231], v[104:111], a[72:79], v[228:231]\n\t" \
  "ds_read_b64_tr_b8 v[122:123], v80 offset:58368\n\t" \
  "v_mfma_f32_16x16x128_f8f6f4 v[232:235], v[104:111], a[80:87], v[232:235]\n\t" \
  "ds_read_b64_tr_b8 v[124:125], v88 offset:57344\n\t" \
  "s_add_u32 s55, s68, 0x150000\n\t" \
  "buffer_load_dwordx4 a[172:175], v4, s[12:15], s55 offen offset:3072 sc0 sc1\n\t" \
  "v_mfma_f32_16x16x128_f8f6f4 v[236:239], v[104:111], a[88:95], v[236:239]\n\t" \
  "ds_read_b64_tr_b8 v[126:127], v88 offset:58368\n\t" \
  "v_mfma_f32_16x16x128_f8f6f4 v[252:255], v[112:119], a[88:95], v[252:255]\n\t" \
  "ds_read_b64_tr_b8 v[104:105], v81 offset:57344\n\t" \
  "s_add_u32 s48, s68, 0x151000\n\t" \
  "buffer_load_dwordx4 a[176:179], v4, s[12:15], s48 offen offset:0 sc0 sc1\n\t" \
  "v_mfma_f32_16x16x128_f8f6f4 v[248:251], v[112:119], a[80:87], v[248:251]\n\t" \
  "ds_read_b64_tr_b8 v[106:107], v81 offset:58368\n\t" \
  "v_mfma_f32_16x16x128_f8f6f4 v[244:247], v[112:119], a[72:79], v[244:247]\n\t" \
  "ds_read_b64_tr_b8 v[108:109], v89 offset:57344\n\t" \
  "s_add_u32 s49, s68, 0x151000\n\t" \
  "buffer_load_dwordx4 a[180:183], v4, s[12:15], s49 offen offset:1024 sc0 sc1\n\t" \
  "v_mfma_f32_16x16x128_f8f6f4 v[240:243], v[112:119], a[64:71], v[240:243]\n\t" \
  "ds_read_b64_tr_b8 v[110:111], v89 offset:58368\n\t" \
  "s_waitcnt vmcnt(26) lgkmcnt(4)\n\t" \
  "v_mfma_f32_16x16x128_f8f6f4 v[128:131], v[120:127], a[96:103], v[128:131]\n\t" \
  "ds_read_b64_tr_b8 v[112:113], v82 offset:57344\n\t" \
  "s_add_u32 s50, s68, 0x151000\n\t" \
  "buffer_load_dwordx4 a[184:187], v4, s[12:15], s50 offen offset:2048 sc0 sc1\n\t" \
  "v_mfma_f32_16x16x128_f8f6f4 v[132:135], v[120:127], a[104:111], v[132:135]\n\t" \
  "ds_read_b64_tr_b8 v[114:115], v82 offset:58368\n\t" \
  "v_mfma_f32_16x16x128_f8f6f4 v[136:139], v[120:127], a[112:119], v[136:139]\n\t" \
  "ds_read_b64_tr_b8 v[116:117], v90 offset:57344\n\t" \
  "s_add_u32 s51, s68, 0x151000\n\t" \
  "buffer_load_dwordx4 a[188:191], v4, s[12:15], s51 offen offset:3072 sc0 sc1\n\t" \
  "v_mfma_f32_16x16x128_f8f6f4 v[140:143], v[120:127], a[120:127], v[140:143]\n\t" \
  "ds_read_b64_tr_b8 v[118:119], v90 offset:58368\n\t" \
  "s_waitcnt lgkmcnt(4)\n\t" \
  "v_mfma_f32_16x16x128_f8f6f4 v[156:159], v[104:111], a[120:127], v[156:159]\n\t" \
  "ds_read_b64_tr_b8 v[120:121], v83 offset:57344\n\t" \
  "s_add_u32 m0, s24, 0x24000\n\t" \
  "s_add_u32 s52, s44, 0x400000\n\t" \
  "buffer_load_dwordx4 v3, s[8:11], s52 offen lds\n\t" \
  "v_mfma_f32_16x16x128_f8f6f4 v[152:155], v[104:111], a[112:119], v[152:155]\n\t" \
  "ds_read_b64_tr_b8 v[122:123], v83 offset:58368\n\t" \
  "v_mfma_f32_16x16x128_f8f6f4 v[148:151], v[104:111], a[104:111], v[148:151]\n\t" \
  "ds_read_b64_tr_b8 v[124:125], v91 offset:57344\n\t" \
  "s_add_u32 m0, s24, 0x24400\n\t" \
  "s_add_u32 s53, s44, 0x420000\n\t" \
  "buffer_load_dwordx4 v3, s[8:11], s53 offen lds\n\t" \
  "v_mfma_f32_16x16x128_f8f6f4 v[144:147], v[104:111], a[96:103], v[144:147]\n\t" \
  "ds_read_b64_tr_b8 v[126:127], v91 offset:58368\n\t" \
  "s_waitcnt lgkmcnt(4)\n\t" \
  "v_mfma_f32_16x16x128_f8f6f4 v[160:163], v[112:119], a[96:103], v[160:163]\n\t" \
  "ds_read_b64_tr_b8 v[104:105], v84 offset:57344\n\t" \
  "s_add_u32 s54, s68, 0x160000\n\t" \
  "buffer_load_dwordx4 a[192:195], v4, s[12:15], s54 offen offset:0 sc0 sc1\n\t" \
  "v_mfma_f32_16x16x128_f8f6f4 v[164:167], v[112:119], a[104:111], v[164:167]\n\t" \
  "ds_read_b64_tr_b8 v[106:107], v84 offset:58368\n\t" \
  "v_mfma_f32_16x16x128_f8f6f4 v[168:171], v[112:119], a[112:119], v[168:171]\n\t" \
  "ds_read_b64_tr_b8 v[108:109], v92 offset:57344\n\t" \
  "s_add_u32 s55, s68, 0x160000\n\t" \
  "buffer_load_dwordx4 a[196:199], v4, s[12:15], s55 offen offset:1024 sc0 sc1\n\t" \
  "v_mfma_f32_16x16x128_f8f6f4 v[172:175], v[112:119], a[120:127], v[172:175]\n\t" \
  "ds_read_b64_tr_b8 v[110:111], v92 offset:58368\n\t" \
  "s_waitcnt lgkmcnt(4)\n\t" \
  "v_mfma_f32_16x16x128_f8f6f4 v[188:191], v[120:127], a[120:127], v[188:191]\n\t" \
  "ds_read_b64_tr_b8 v[112:113], v85 offset:57344\n\t" \
  "s_add_u32 s48, s68, 0x160000\n\t" \
  "buffer_load_dwordx4 a[200:203], v4, s[12:15], s48 offen offset:2048 sc0 sc1\n\t" \
  "v_mfma_f32_16x16x128_f8f6f4 v[184:187], v[120:127], a[112:119], v[184:187]\n\t" \
  "ds_read_b64_tr_b8 v[114:115], v85 offset:58368\n\t" \
  "v_mfma_f32_16x16x128_f8f6f4 v[180:183], v[120:127], a[104:111], v[180:183]\n\t" \
  "ds_read_b64_tr_b8 v[116:117], v93 offset:57344\n\t" \
  "s_add_u32 s49, s68, 0x160000\n\t" \
  "buffer_load_dwordx4 a[204:207], v4, s[12:15], s49 offen offset:3072 sc0 sc1\n\t" \
  "v_mfma_f32_16x16x128_f8f6f4 v[176:179], v[120:127], a[96:103], v[176:179]\n\t" \
  "ds_read_b64_tr_b8 v[118:119], v93 offset:58368\n\t" \
  "s_waitcnt lgkmcnt(4)\n\t" \
  "v_mfma_f32_16x16x128_f8f6f4 v[192:195], v[104:111], a[96:103], v[192:195]\n\t" \
  "ds_read_b64_tr_b8 v[120:121], v86 offset:57344\n\t" \
  "s_add_u32 s50, s68, 0x161000\n\t" \
  "buffer_load_dwordx4 a[208:211], v4, s[12:15], s50 offen offset:0 sc0 sc1\n\t" \
  "v_mfma_f32_16x16x128_f8f6f4 v[196:199], v[104:111], a[104:111], v[196:199]\n\t" \
  "ds_read_b64_tr_b8 v[122:123], v86 offset:58368\n\t" \
  "v_mfma_f32_16x16x128_f8f6f4 v[200:203], v[104:111], a[112:119], v[200:203]\n\t" \
  "ds_read_b64_tr_b8 v[124:125], v94 offset:57344\n\t" \
  "s_add_u32 s51, s68, 0x161000\n\t" \
  "buffer_load_dwordx4 a[212:215], v4, s[12:15], s51 offen offset:1024 sc0 sc1\n\t" \
  "v_mfma_f32_16x16x128_f8f6f4 v[204:207], v[104:111], a[120:127], v[204:207]\n\t" \
  "ds_read_b64_tr_b8 v[126:127], v94 offset:58368\n\t" \
  "s_waitcnt lgkmcnt(4)\n\t" \
  "v_mfma_f32_16x16x128_f8f6f4 v[220:223], v[112:119], a[120:127], v[220:223]\n\t" \
  "ds_read_b64_tr_b8 v[104:105], v87 offset:57344\n\t" \
  "s_add_u32 s52, s68, 0x161000\n\t" \
  "buffer_load_dwordx4 a[216:219], v4, s[12:15], s52 offen offset:2048 sc0 sc1\n\t" \
  "v_mfma_f32_16x16x128_f8f6f4 v[216:219], v[112:119], a[112:119], v[216:219]\n\t" \
  "ds_read_b64_tr_b8 v[106:107], v87 offset:58368\n\t" \
  "v_mfma_f32_16x16x128_f8f6f4 v[212:215], v[112:119], a[104:111], v[212:215]\n\t" \
  "ds_read_b64_tr_b8 v[108:109], v95 offset:57344\n\t" \
  "s_add_u32 s53, s68, 0x161000\n\t" \
  "buffer_load_dwordx4 a[220:223], v4, s[12:15], s53 offen offset:3072 sc0 sc1\n\t" \
  "v_mfma_f32_16x16x128_f8f6f4 v[208:211], v[112:119], a[96:103], v[208:211]\n\t" \
  "ds_read_b64_tr_b8 v[110:111], v95 offset:58368\n\t" \
  "s_waitcnt vmcnt(28) lgkmcnt(0)\n\t" \
  "s_barrier\n\t" \
  "v_mfma_f32_16x16x128_f8f6f4 v[224:227], v[120:127], a[96:103], v[224:227]\n\t" \
  "ds_read_b64_tr_b8 v[112:113], v88 offset:0\n\t" \
  "s_add_u32 m0, s24, 0x26000\n\t" \
  "s_add_u32 s54, s45, 0x400000\n\t" \
  "buffer_load_dwordx4 v3, s[8:11], s54 offen lds\n\t" \
  "v_mfma_f32_16x16x128_f8f6f4 v[228:231], v[120:127], a[104:111], v[228:231]\n\t" \
  "ds_read_b64_tr_b8 v[114:115], v88 offset:1024\n\t" \
  "v_mfma_f32_16x16x128_f8f6f4 v[232:235], v[120:127], a[112:119], v[232:235]\n\t" \
  "ds_read_b64_tr_b8 v[116:117], v96 offset:0\n\t" \
  "s_add_u32 m0, s24, 0x26400\n\t" \
  "s_add_u32 s55, s45, 0x420000\n\t" \
  "buffer_load_dwordx4 v3, s[8:11], s55 offen lds\n\t" \
  "v_mfma_f32_16x16x128_f8f6f4 v[236:239], v[120:127], a[120:127], v[236:239]\n\t" \
  "ds_read_b64_tr_b8 v[118:119], v96 offset:1024\n\t" \
  "v_mfma_f32_16x16x128_f8f6f4 v[252:255], v[104:111], a[120:127], v[252:255]\n\t" \
  "ds_read_b64_tr_b8 v[120:121], v89 offset:0\n\t" \
  "s_add_u32 s48, s68, 0x170000\n\t" \
  "buffer_load_dwordx4 a[224:227], v4, s[12:15], s48 offen offset:0 sc0 sc1\n\t" \
  "v_mfma_f32_16x16x128_f8f6f4 v[248:251], v[104:111], a[112:119], v[248:251]\n\t" \
  "ds_read_b64_tr_b8 v[122:123], v89 offset:1024\n\t" \
  "v_mfma_f32_16x16x128_f8f6f4 v[244:247], v[104:111], a[104:111], v[244:247]\n\t" \
  "ds_read_b64_tr_b8 v[124:125], v97 offset:0\n\t" \
  "s_add_u32 s49, s68, 0x170000\n\t" \
  "buffer_load_dwordx4 a[228:231], v4, s[12:15], s49 offen offset:1024 sc0 sc1\n\t" \
  "v_mfma_f32_16x16x128_f8f6f4 v[240:243], v[104:111], a[96:103], v[240:243]\n\t" \
  "ds_read_b64_tr_b8 v[126:127], v97 offset:1024\n\t" \
  "s_waitcnt vmcnt(24) lgkmcnt(4)\n\t" \
  "v_mfma_f32_16x16x128_f8f6f4 v[128:131], v[112:119], a[128:135], v[128:131]\n\t" \
  "ds_read_b64_tr_b8 v[104:105], v90 offset:0\n\t" \
  "s_add_u32 s50, s68, 0x170000\n\t" \
  "buffer_load_dwordx4 a[232:235], v4, s[12:15], s50 offen offset:2048 sc0 sc1\n\t" \
  "v_mfma_f32_16x16x128_f8f6f4 v[132:135], v[112:119], a[136:143], v[132:135]\n\t" \
  "ds_read_b64_tr_b8 v[106:107], v90 offset:1024\n\t" \
  "v_mfma_f32_16x16x128_f8f6f4 v[136:139], v[112:119], a[144:151], v[136:139]\n\t" \
  "ds_read_b64_tr_b8 v[108:109], v98 offset:0\n\t" \
  "s_add_u32 s51, s68, 0x170000\n\t" \
  "buffer_load_dwordx4 a[236:239], v4, s[12:15], s51 offen offset:3072 sc0 sc1\n\t" \
  "v_mfma_f32_16x16x128_f8f6f4 v[140:143], v[112:119], a[152:159], v[140:143]\n\t" \
  "ds_read_b64_tr_b8 v[110:111], v98 offset:1024\n\t" \
  "s_waitcnt lgkmcnt(4)\n\t" \
  "v_mfma_f32_16x16x128_f8f6f4 v[156:159], v[120:127], a[152:159], v[156:159]\n\t" \
  "ds_read_b64_tr_b8 v[112:113], v91 offset:0\n\t" \
  "s_add_u32 s52, s68, 0x171000\n\t" \
  "buffer_load_dwordx4 a[240:243], v4, s[12:15], s52 offen offset:0 sc0 sc1\n\t" \
  "v_mfma_f32_16x16x128_f8f6f4 v[152:155], v[120:127], a[144:151], v[152:155]\n\t" \
  "ds_read_b64_tr_b8 v[114:115], v91 offset:1024\n\t" \
  "v_mfma_f32_16x16x128_f8f6f4 v[148:151], v[120:127], a[136:143], v[148:151]\n\t" \
  "ds_read_b64_tr_b8 v[116:117], v99 offset:0\n\t" \
  "s_add_u32 s53, s68, 0x171000\n\t" \
  "buffer_load_dwordx4 a[244:247], v4, s[12:15], s53 offen offset:1024 sc0 sc1\n\t" \
  "v_mfma_f32_16x16x128_f8f6f4 v[144:147], v[120:127], a[128:135], v[144:147]\n\t" \
  "ds_read_b64_tr_b8 v[118:119], v99 offset:1024\n\t" \
  "s_waitcnt lgkmcnt(4)\n\t" \
  "v_mfma_f32_16x16x128_f8f6f4 v[160:163], v[104:111], a[128:135], v[160:163]\n\t" \
  "ds_read_b64_tr_b8 v[120:121], v92 offset:0\n\t" \
  "s_add_u32 s54, s68, 0x171000\n\t" \
  "buffer_load_dwordx4 a[248:251], v4, s[12:15], s54 offen offset:2048 sc0 sc1\n\t" \
  "v_mfma_f32_16x16x128_f8f6f4 v[164:167], v[104:111], a[136:143], v[164:167]\n\t" \
  "ds_read_b64_tr_b8 v[122:123], v92 offset:1024\n\t" \
  "v_mfma_f32_16x16x128_f8f6f4 v[168:171], v[104:111], a[144:151], v[168:171]\n\t" \
  "ds_read_b64_tr_b8 v[124:125], v100 offset:0\n\t" \
  "s_add_u32 s55, s68, 0x171000\n\t" \
  "buffer_load_dwordx4 a[252:255], v4, s[12:15], s55 offen offset:3072 sc0 sc1\n\t" \
  "v_mfma_f32_16x16x128_f8f6f4 v[172:175], v[104:111], a[152:159], v[172:175]\n\t" \
  "ds_read_b64_tr_b8 v[126:127], v100 offset:1024\n\t" \
  "s_waitcnt lgkmcnt(4)\n\t" \
  "v_mfma_f32_16x16x128_f8f6f4 v[188:191], v[112:119], a[152:159], v[188:191]\n\t" \
  "ds_read_b64_tr_b8 v[104:105], v93 offset:0\n\t" \
  "s_add_u32 m0, s24, 0x8000\n\t" \
  "s_add_u32 s48, s43, 0x600000\n\t" \
  "buffer_load_dwordx4 v2, s[8:11], s48 offen lds\n\t" \
  "v_mfma_f32_16x16x128_f8f6f4 v[184:187], v[112:119], a[144:151], v[184:187]\n\t" \
  "ds_read_b64_tr_b8 v[106:107], v93 offset:1024\n\t" \
  "v_mfma_f32_16x16x128_f8f6f4 v[180:183], v[112:119], a[136:143], v[180:183]\n\t" \
  "ds_read_b64_tr_b8 v[108:109], v101 offset:0\n\t" \
  "s_add_u32 m0, s24, 0x8400\n\t" \
  "s_add_u32 s49, s43, 0x620000\n\t" \
  "buffer_load_dwordx4 v2, s[8:11], s49 offen lds\n\t" \
  "v_mfma_f32_16x16x128_f8f6f4 v[176:179], v[112:119], a[128:135], v[176:179]\n\t" \
  "ds_read_b64_tr_b8 v[110:111], v101 offset:1024\n\t" \
  "s_waitcnt lgkmcnt(4)\n\t" \
  "v_mfma_f32_16x16x128_f8f6f4 v[192:195], v[120:127], a[128:135], v[192:195]\n\t" \
  "ds_read_b64_tr_b8 v[112:113], v94 offset:0\n\t" \
  "s_add_u32 m0, s24, 0xa000\n\t" \
  "s_add_u32 s50, s44, 0x600000\n\t" \
  "buffer_load_dwordx4 v2, s[8:11], s50 offen lds\n\t" \
  "v_mfma_f32_16x16x128_f8f6f4 v[196:199], v[120:127], a[136:143], v[196:199]\n\t" \
  "ds_read_b64_tr_b8 v[114:115], v94 offset:1024\n\t" \
  "v_mfma_f32_16x16x128_f8f6f4 v[200:203], v[120:127], a[144:151], v[200:203]\n\t" \
  "ds_read_b64_tr_b8 v[116:117], v102 offset:0\n\t" \
  "s_add_u32 m0, s24, 0xa400\n\t" \
  "s_add_u32 s51, s44, 0x620000\n\t" \
  "buffer_load_dwordx4 v2, s[8:11], s51 offen lds\n\t" \
  "v_mfma_f32_16x16x128_f8f6f4 v[204:207], v[120:127], a[152:159], v[204:207]\n\t" \
  "ds_read_b64_tr_b8 v[118:119], v102 offset:1024\n\t" \
  "s_waitcnt lgkmcnt(4)\n\t" \
  "v_mfma_f32_16x16x128_f8f6f4 v[220:223], v[104:111], a[152:159], v[220:223]\n\t" \
  "ds_read_b64_tr_b8 v[120:121], v95 offset:0\n\t" \
  "s_add_u32 m0, s24, 0xc000\n\t" \
  "s_add_u32 s52, s45, 0x600000\n\t" \
  "buffer_load_dwordx4 v2, s[8:11], s52 offen lds\n\t" \
  "v_mfma_f32_16x16x128_f8f6f4 v[216:219], v[104:111], a[144:151], v[216:219]\n\t" \
  "ds_read_b64_tr_b8 v[122:123], v95 offset:1024\n\t" \
  "v_mfma_f32_16x16x128_f8f6f4 v[212:215], v[104:111], a[136:143], v[212:215]\n\t" \
  "ds_read_b64_tr_b8 v[124:125], v103 offset:0\n\t" \
  "s_add_u32 m0, s24, 0xc400\n\t" \
  "s_add_u32 s53, s45, 0x620000\n\t" \
  "buffer_load_dwordx4 v2, s[8:11], s53 offen lds\n\t" \
  "v_mfma_f32_16x16x128_f8f6f4 v[208:211], v[104:111], a[128:135], v[208:211]\n\t" \
  "ds_read_b64_tr_b8 v[126:127], v103 offset:1024\n\t" \
  "s_waitcnt vmcnt(34) lgkmcnt(0)\n\t" \
  "s_barrier\n\t" \
  "v_mfma_f32_16x16x128_f8f6f4 v[224:227], v[112:119], a[128:135], v[224:227]\n\t" \
  "ds_read_b64_tr_b8 v[104:105], v88 offset:8192\n\t" \
  "s_add_u32 m0, s24, 0xe000\n\t" \
  "s_add_u32 s54, s46, 0x600000\n\t" \
  "buffer_load_dwordx4 v2, s[8:11], s54 offen lds\n\t" \
  "v_mfma_f32_16x16x128_f8f6f4 v[228:231], v[112:119], a[136:143], v[228:231]\n\t" \
  "ds_read_b64_tr_b8 v[106:107], v88 offset:9216\n\t" \
  "v_mfma_f32_16x16x128_f8f6f4 v[232:235], v[112:119], a[144:151], v[232:235]\n\t" \
  "ds_read_b64_tr_b8 v[108:109], v96 offset:8192\n\t" \
  "s_add_u32 m0, s24, 0xe400\n\t" \
  "s_add_u32 s55, s46, 0x620000\n\t" \
  "buffer_load_dwordx4 v2, s[8:11], s55 offen lds\n\t" \
  "v_mfma_f32_16x16x128_f8f6f4 v[236:239], v[112:119], a[152:159], v[236:239]\n\t" \
  "ds_read_b64_tr_b8 v[110:111], v96 offset:9216\n\t" \
  "v_mfma_f32_16x16x128_f8f6f4 v[252:255], v[120:127], a[152:159], v[252:255]\n\t" \
  "ds_read_b64_tr_b8 v[112:113], v89 offset:8192\n\t" \
  "v_mfma_f32_16x16x128_f8f6f4 v[248:251], v[120:127], a[144:151], v[248:251]\n\t" \
  "ds_read_b64_tr_b8 v[114:115], v89 offset:9216\n\t" \
  "v_mfma_f32_16x16x128_f8f6f4 v[244:247], v[120:127], a[136:143], v[244:247]\n\t" \
  "ds_read_b64_tr_b8 v[116:117], v97 offset:8192\n\t" \
  "v_mfma_f32_16x16x128_f8f6f4 v[240:243], v[120:127], a[128:135], v[240:243]\n\t" \
  "ds_read_b64_tr_b8 v[118:119], v97 offset:9216\n\t" \
  "s_waitcnt vmcnt(28) lgkmcnt(4)\n\t" \
  "v_mfma_f32_16x16x128_f8f6f4 v[128:131], v[104:111], a[160:167], v[128:131]\n\t" \
  "ds_read_b64_tr_b8 v[120:121], v90 offset:8192\n\t" \
  "s_mov_b32 exec_lo, 0x7f000000\n\t" \
  "s_mov_b32 exec_hi, 0\n\t" \
  "global_load_dwordx4 v[68:71], v10, s[6:7] sc0 sc1\n\t" \
  "s_mov_b64 exec, -1\n\t" \
  "v_mfma_f32_16x16x128_f8f6f4 v[132:135], v[104:111], a[168:175], v[132:135]\n\t" \
  "ds_read_b64_tr_b8 v[122:123], v90 offset:9216\n\t" \
  "v_mfma_f32_16x16x128_f8f6f4 v[136:139], v[104:111], a[176:183], v[136:139]\n\t" \
  "ds_read_b64_tr_b8 v[124:125], v98 offset:8192\n\t" \
  "v_mfma_f32_16x16x128_f8f6f4 v[140:143], v[104:111], a[184:191], v[140:143]\n\t" \
  "ds_read_b64_tr_b8 v[126:127], v98 offset:9216\n\t" \
  "s_waitcnt lgkmcnt(4)\n\t" \
  "v_mfma_f32_16x16x128_f8f6f4 v[156:159], v[112:119], a[184:191], v[156:159]\n\t" \
  "ds_read_b64_tr_b8 v[104:105], v91 offset:8192\n\t" \
  "v_mfma_f32_16x16x128_f8f6f4 v[152:155], v[112:119], a[176:183], v[152:155]\n\t" \
  "ds_read_b64_tr_b8 v[106:107], v91 offset:9216\n\t" \
  "v_mfma_f32_16x16x128_f8f6f4 v[148:151], v[112:119], a[168:175], v[148:151]\n\t" \
  "ds_read_b64_tr_b8 v[108:109], v99 offset:8192\n\t" \
  "v_mfma_f32_16x16x128_f8f6f4 v[144:147], v[112:119], a[160:167], v[144:147]\n\t" \
  "ds_read_b64_tr_b8 v[110:111], v99 offset:9216\n\t" \
  "s_waitcnt lgkmcnt(4)\n\t" \
  "v_mfma_f32_16x16x128_f8f6f4 v[160:163], v[120:127], a[160:167], v[160:163]\n\t" \
  "ds_read_b64_tr_b8 v[112:113], v92 offset:8192\n\t" \
  "v_mfma_f32_16x16x128_f8f6f4 v[164:167], v[120:127], a[168:175], v[164:167]\n\t" \
  "ds_read_b64_tr_b8 v[114:115], v92 offset:9216\n\t" \
  "v_mfma_f32_16x16x128_f8f6f4 v[168:171], v[120:127], a[176:183], v[168:171]\n\t" \
  "ds_read_b64_tr_b8 v[116:117], v100 offset:8192\n\t" \
  "s_add_u32 m0, s24, 0x18000\n\t" \
  "s_add_u32 s48, s46, 0x400000\n\t" \
  "buffer_load_dwordx4 v3, s[8:11], s48 offen lds\n\t" \
  "v_mfma_f32_16x16x128_f8f6f4 v[172:175], v[120:127], a[184:191], v[172:175]\n\t" \
  "ds_read_b64_tr_b8 v[118:119], v100 offset:9216\n\t" \
  "s_waitcnt lgkmcnt(4)\n\t" \
  "v_mfma_f32_16x16x128_f8f6f4 v[188:191], v[104:111], a[184:191], v[188:191]\n\t" \
  "ds_read_b64_tr_b8 v[120:121], v93 offset:8192\n\t" \
  "s_add_u32 m0, s24, 0x18400\n\t" \
  "s_add_u32 s49, s46, 0x420000\n\t" \
  "buffer_load_dwordx4 v3, s[8:11], s49 offen lds\n\t" \
  "v_mfma_f32_16x16x128_f8f6f4 v[184:187], v[104:111], a[176:183], v[184:187]\n\t" \
  "ds_read_b64_tr_b8 v[122:123], v93 offset:9216\n\t" \
  "v_mfma_f32_16x16x128_f8f6f4 v[180:183], v[104:111], a[168:175], v[180:183]\n\t" \
  "ds_read_b64_tr_b8 v[124:125], v101 offset:8192\n\t" \
  "s_add_u32 m0, s24, 0x1a000\n\t" \
  "s_add_u32 s50, s40, 0x600000\n\t" \
  "buffer_load_dwordx4 v3, s[8:11], s50 offen lds\n\t" \
  "v_mfma_f32_16x16x128_f8f6f4 v[176:179], v[104:111], a[160:167], v[176:179]\n\t" \
  "ds_read_b64_tr_b8 v[126:127], v101 offset:9216\n\t" \
  "s_waitcnt lgkmcnt(4)\n\t" \
  "v_mfma_f32_16x16x128_f8f6f4 v[192:195], v[112:119], a[160:167], v[192:195]\n\t" \
  "ds_read_b64_tr_b8 v[104:105], v94 offset:8192\n\t" \
  "s_add_u32 m0, s24, 0x1a400\n\t" \
  "s_add_u32 s51, s40, 0x620000\n\t" \
  "buffer_load_dwordx4 v3, s[8:11], s51 offen lds\n\t" \
  "v_mfma_f32_16x16x128_f8f6f4 v[196:199], v[112:119], a[168:175], v[196:199]\n\t" \
  "ds_read_b64_tr_b8 v[106:107], v94 offset:9216\n\t" \
  "v_mfma_f32_16x16x128_f8f6f4 v[200:203], v[112:119], a[176:183], v[200:203]\n\t" \
  "ds_read_b64_tr_b8 v[108:109], v102 offset:8192\n\t" \
  "s_add_u32 m0, s24, 0x1c000\n\t" \
  "s_add_u32 s52, s41, 0x600000\n\t" \
  "buffer_load_dwordx4 v3, s[8:11], s52 offen lds\n\t" \
  "v_mfma_f32_16x16x128_f8f6f4 v[204:207], v[112:119], a[184:191], v[204:207]\n\t" \
  "ds_read_b64_tr_b8 v[110:111], v102 offset:9216\n\t" \
  "s_waitcnt lgkmcnt(4)\n\t" \
  "v_mfma_f32_16x16x128_f8f6f4 v[220:223], v[120:127], a[184:191], v[220:223]\n\t" \
  "ds_read_b64_tr_b8 v[112:113], v95 offset:8192\n\t" \
  "s_add_u32 m0, s24, 0x1c400\n\t" \
  "s_add_u32 s53, s41, 0x620000\n\t" \
  "buffer_load_dwordx4 v3, s[8:11], s53 offen lds\n\t" \
  "v_mfma_f32_16x16x128_f8f6f4 v[216:219], v[120:127], a[176:183], v[216:219]\n\t" \
  "ds_read_b64_tr_b8 v[114:115], v95 offset:9216\n\t" \
  "v_mfma_f32_16x16x128_f8f6f4 v[212:215], v[120:127], a[168:175], v[212:215]\n\t" \
  "ds_read_b64_tr_b8 v[116:117], v103 offset:8192\n\t" \
  "s_add_u32 m0, s24, 0x1e000\n\t" \
  "s_add_u32 s54, s42, 0x600000\n\t" \
  "buffer_load_dwordx4 v3, s[8:11], s54 offen lds\n\t" \
  "v_mfma_f32_16x16x128_f8f6f4 v[208:211], v[120:127], a[160:167], v[208:211]\n\t" \
  "ds_read_b64_tr_b8 v[118:119], v103 offset:9216\n\t" \
  "s_waitcnt vmcnt(34) lgkmcnt(0)\n\t" \
  "s_barrier\n\t" \
  "v_mfma_f32_16x16x128_f8f6f4 v[224:227], v[104:111], a[160:167], v[224:227]\n\t" \
  "ds_read_b64_tr_b8 v[120:121], v88 offset:16384\n\t" \
  "s_add_u32 m0, s24, 0x1e400\n\t" \
  "s_add_u32 s55, s42, 0x620000\n\t" \
  "buffer_load_dwordx4 v3, s[8:11], s55 offen lds\n\t" \
  "v_mfma_f32_16x16x128_f8f6f4 v[228:231], v[104:111], a[168:175], v[228:231]\n\t" \
  "ds_read_b64_tr_b8 v[122:123], v88 offset:17408\n\t" \
  "v_mfma_f32_16x16x128_f8f6f4 v[232:235], v[104:111], a[176:183], v[232:235]\n\t" \
  "ds_read_b64_tr_b8 v[124:125], v96 offset:16384\n\t" \
  "s_add_u32 m0, s24, 0x20000\n\t" \
  "s_add_u32 s48, s43, 0x600000\n\t" \
  "buffer_load_dwordx4 v3, s[8:11], s48 offen lds\n\t" \
  "v_mfma_f32_16x16x128_f8f6f4 v[236:239], v[104:111], a[184:191], v[236:239]\n\t" \
  "ds_read_b64_tr_b8 v[126:127], v96 offset:17408\n\t" \
  "v_mfma_f32_16x16x128_f8f6f4 v[252:255], v[112:119], a[184:191], v[252:255]\n\t" \
  "ds_read_b64_tr_b8 v[104:105], v89 offset:16384\n\t" \
  "s_add_u32 m0, s24, 0x20400\n\t" \
  "s_add_u32 s49, s43, 0x620000\n\t" \
  "buffer_load_dwordx4 v3, s[8:11], s49 offen lds\n\t" \
  "v_mfma_f32_16x16x128_f8f6f4 v[248:251], v[112:119], a[176:183], v[248:251]\n\t" \
  "ds_read_b64_tr_b8 v[106:107], v89 offset:17408\n\t" \
  "v_mfma_f32_16x16x128_f8f6f4 v[244:247], v[112:119], a[168:175], v[244:247]\n\t" \
  "ds_read_b64_tr_b8 v[108:109], v97 offset:16384\n\t" \
  "s_add_u32 m0, s24, 0x22000\n\t" \
  "s_add_u32 s50, s44, 0x600000\n\t" \
  "buffer_load_dwordx4 v3, s[8:11], s50 offen lds\n\t" \
  "v_mfma_f32_16x16x128_f8f6f4 v[240:243], v[112:119], a[160:167], v[240:243]\n\t" \
  "ds_read_b64_tr_b8 v[110:111], v97 offset:17408\n\t" \
  "s_waitcnt vmcnt(30) lgkmcnt(4)\n\t" \
  "v_mfma_f32_16x16x128_f8f6f4 v[128:131], v[120:127], a[192:199], v[128:131]\n\t" \
  "ds_read_b64_tr_b8 v[112:113], v90 offset:16384\n\t" \
  "s_add_u32 m0, s24, 0x22400\n\t" \
  "s_add_u32 s51, s44, 0x620000\n\t" \
  "buffer_load_dwordx4 v3, s[8:11], s51 offen lds\n\t" \
  "v_mfma_f32_16x16x128_f8f6f4 v[132:135], v[120:127], a[200:207], v[132:135]\n\t" \
  "ds_read_b64_tr_b8 v[114:115], v90 offset:17408\n\t" \
  "v_mfma_f32_16x16x128_f8f6f4 v[136:139], v[120:127], a[208:215], v[136:139]\n\t" \
  "ds_read_b64_tr_b8 v[116:117], v98 offset:16384\n\t" \
  "s_add_u32 s52, s68, 0x180000\n\t" \
  "buffer_load_dwordx4 a[0:3], v4, s[12:15], s52 offen offset:0 sc0 sc1\n\t" \
  "v_mfma_f32_16x16x128_f8f6f4 v[140:143], v[120:127], a[216:223], v[140:143]\n\t" \
  "ds_read_b64_tr_b8 v[118:119], v98 offset:17408\n\t" \
  "s_waitcnt lgkmcnt(4)\n\t" \
  "v_mfma_f32_16x16x128_f8f6f4 v[156:159], v[104:111], a[216:223], v[156:159]\n\t" \
  "ds_read_b64_tr_b8 v[120:121], v91 offset:16384\n\t" \
  "s_add_u32 s53, s68, 0x180000\n\t" \
  "buffer_load_dwordx4 a[4:7], v4, s[12:15], s53 offen offset:1024 sc0 sc1\n\t" \
  "v_mfma_f32_16x16x128_f8f6f4 v[152:155], v[104:111], a[208:215], v[152:155]\n\t" \
  "ds_read_b64_tr_b8 v[122:123], v91 offset:17408\n\t" \
  "v_mfma_f32_16x16x128_f8f6f4 v[148:151], v[104:111], a[200:207], v[148:151]\n\t" \
  "ds_read_b64_tr_b8 v[124:125], v99 offset:16384\n\t" \
  "s_add_u32 s54, s68, 0x180000\n\t" \
  "buffer_load_dwordx4 a[8:11], v4, s[12:15], s54 offen offset:2048 sc0 sc1\n\t" \
  "v_mfma_f32_16x16x128_f8f6f4 v[144:147], v[104:111], a[192:199], v[144:147]\n\t" \
  "ds_read_b64_tr_b8 v[126:127], v99 offset:17408\n\t" \
  "s_waitcnt lgkmcnt(4)\n\t" \
  "v_mfma_f32_16x16x128_f8f6f4 v[160:163], v[112:119], a[192:199], v[160:163]\n\t" \
  "ds_read_b64_tr_b8 v[104:105], v92 offset:16384\n\t" \
  "s_add_u32 s55, s68, 0x180000\n\t" \
  "buffer_load_dwordx4 a[12:15], v4, s[12:15], s55 offen offset:3072 sc0 sc1\n\t" \
  "v_mfma_f32_16x16x128_f8f6f4 v[164:167], v[112:119], a[200:207], v[164:167]\n\t" \
  "ds_read_b64_tr_b8 v[106:107], v92 offset:17408\n\t" \
  "v_mfma_f32_16x16x128_f8f6f4 v[168:171], v[112:119], a[208:215], v[168:171]\n\t" \
  "ds_read_b64_tr_b8 v[108:109], v100 offset:16384\n\t" \
  "s_add_u32 s48, s68, 0x181000\n\t" \
  "buffer_load_dwordx4 a[16:19], v4, s[12:15], s48 offen offset:0 sc0 sc1\n\t" \
  "v_mfma_f32_16x16x128_f8f6f4 v[172:175], v[112:119], a[216:223], v[172:175]\n\t" \
  "ds_read_b64_tr_b8 v[110:111], v100 offset:17408\n\t" \
  "s_waitcnt lgkmcnt(4)\n\t" \
  "v_mfma_f32_16x16x128_f8f6f4 v[188:191], v[120:127], a[216:223], v[188:191]\n\t" \
  "ds_read_b64_tr_b8 v[112:113], v93 offset:16384\n\t" \
  "s_add_u32 s49, s68, 0x181000\n\t" \
  "buffer_load_dwordx4 a[20:23], v4, s[12:15], s49 offen offset:1024 sc0 sc1\n\t" \
  "v_mfma_f32_16x16x128_f8f6f4 v[184:187], v[120:127], a[208:215], v[184:187]\n\t" \
  "ds_read_b64_tr_b8 v[114:115], v93 offset:17408\n\t" \
  "v_mfma_f32_16x16x128_f8f6f4 v[180:183], v[120:127], a[200:207], v[180:183]\n\t" \
  "ds_read_b64_tr_b8 v[116:117], v101 offset:16384\n\t" \
  "s_add_u32 s50, s68, 0x181000\n\t" \
  "buffer_load_dwordx4 a[24:27], v4, s[12:15], s50 offen offset:2048 sc0 sc1\n\t" \
  "v_mfma_f32_16x16x128_f8f6f4 v[176:179], v[120:127], a[192:199], v[176:179]\n\t" \
  "ds_read_b64_tr_b8 v[118:119], v101 offset:17408\n\t" \
  "s_waitcnt lgkmcnt(4)\n\t" \
  "v_mfma_f32_16x16x128_f8f6f4 v[192:195], v[104:111], a[192:199], v[192:195]\n\t" \
  "ds_read_b64_tr_b8 v[120:121], v94 offset:16384\n\t" \
  "s_add_u32 s51, s68, 0x181000\n\t" \
  "buffer_load_dwordx4 a[28:31], v4, s[12:15], s51 offen offset:3072 sc0 sc1\n\t" \
  "v_mfma_f32_16x16x128_f8f6f4 v[196:199], v[104:111], a[200:207], v[196:199]\n\t" \
  "ds_read_b64_tr_b8 v[122:123], v94 offset:17408\n\t" \
  "v_mfma_f32_16x16x128_f8f6f4 v[200:203], v[104:111], a[208:215], v[200:203]\n\t" \
  "ds_read_b64_tr_b8 v[124:125], v102 offset:16384\n\t" \
  "s_waitcnt vmcnt(20)\n\t" \
  "v_min_u32 v68, v68, v69\n\t" \
  "v_min3_u32 v68, v68, v70, v71\n\t" \
  "v_cmp_le_u32 vcc, s57, v68\n\t" \
  "s_and_b32 s64, vcc_lo, 0x7f000000\n\t" \
  "s_or_b32 s56, s56, s64\n\t" \
  "s_and_b32 s64, s56, 0x7f000000\n\t" \
  "s_cmp_eq_u32 s64, 0x7f000000\n\t" \
  "s_cbranch_scc1 Lok9_1_%=\n\t" \
  "s_mov_b32 s62, 0x7f000000\n\t" \
  "s_mov_b32 s69, 0x7f000000\n\t" \
  "s_call_b64 s[60:61], Lpoll_1_%=\n\t" \
  "Lok9_1_%=:\n\t" \
  "s_add_u32 s52, s68, 0x190000\n\t" \
  "buffer_load_dwordx4 a[32:35], v4, s[12:15], s52 offen offset:0 sc0 sc1\n\t" \
  "v_mfma_f32_16x16x128_f8f6f4 v[204:207], v[104:111], a[216:223], v[204:207]\n\t" \
  "ds_read_b64_tr_b8 v[126:127], v102 offset:17408\n\t" \
  "s_waitcnt lgkmcnt(4)\n\t" \
  "v_mfma_f32_16x16x128_f8f6f4 v[220:223], v[112:119], a[216:223], v[220:223]\n\t" \
  "ds_read_b64_tr_b8 v[104:105], v95 offset:16384\n\t" \
  "s_add_u32 s53, s68, 0x190000\n\t" \
  "buffer_load_dwordx4 a[36:39], v4, s[12:15], s53 offen offset:1024 sc0 sc1\n\t" \
  "v_mfma_f32_16x16x128_f8f6f4 v[216:219], v[112:119], a[208:215], v[216:219]\n\t" \
  "ds_read_b64_tr_b8 v[106:107], v95 offset:17408\n\t" \
  "v_mfma_f32_16x16x128_f8f6f4 v[212:215], v[112:119], a[200:207], v[212:215]\n\t" \
  "ds_read_b64_tr_b8 v[108:109], v103 offset:16384\n\t" \
  "s_add_u32 s54, s68, 0x190000\n\t" \
  "buffer_load_dwordx4 a[40:43], v4, s[12:15], s54 offen offset:2048 sc0 sc1\n\t" \
  "v_mfma_f32_16x16x128_f8f6f4 v[208:211], v[112:119], a[192:199], v[208:211]\n\t" \
  "ds_read_b64_tr_b8 v[110:111], v103 offset:17408\n\t" \
  "s_waitcnt lgkmcnt(0)\n\t" \
  "s_barrier\n\t" \
  "v_mfma_f32_16x16x128_f8f6f4 v[224:227], v[120:127], a[192:199], v[224:227]\n\t" \
  "ds_read_b64_tr_b8 v[112:113], v88 offset:24576\n\t" \
  "s_add_u32 s55, s68, 0x190000\n\t" \
  "buffer_load_dwordx4 a[44:47], v4, s[12:15], s55 offen offset:3072 sc0 sc1\n\t" \
  "v_mfma_f32_16x16x128_f8f6f4 v[228:231], v[120:127], a[200:207], v[228:231]\n\t" \
  "ds_read_b64_tr_b8 v[114:115], v88 offset:25600\n\t" \
  "v_mfma_f32_16x16x128_f8f6f4 v[232:235], v[120:127], a[208:215], v[232:235]\n\t" \
  "ds_read_b64_tr_b8 v[116:117], v96 offset:24576\n\t" \
  "s_add_u32 s48, s68, 0x191000\n\t" \
  "buffer_load_dwordx4 a[48:51], v4, s[12:15], s48 offen offset:0 sc0 sc1\n\t" \
  "v_mfma_f32_16x16x128_f8f6f4 v[236:239], v[120:127], a[216:223], v[236:239]\n\t" \
  "ds_read_b64_tr_b8 v[118:119], v96 offset:25600\n\t" \
  "v_mfma_f32_16x16x128_f8f6f4 v[252:255], v[104:111], a[216:223], v[252:255]\n\t" \
  "ds_read_b64_tr_b8 v[120:121], v89 offset:24576\n\t" \
  "s_add_u32 s49, s68, 0x191000\n\t" \
  "buffer_load_dwordx4 a[52:55], v4, s[12:15], s49 offen offset:1024 sc0 sc1\n\t" \
  "v_mfma_f32_16x16x128_f8f6f4 v[248:251], v[104:111], a[208:215], v[248:251]\n\t" \
  "ds_read_b64_tr_b8 v[122:123], v89 offset:25600\n\t" \
  "v_mfma_f32_16x16x128_f8f6f4 v[244:247], v[104:111], a[200:207], v[244:247]\n\t" \
  "ds_read_b64_tr_b8 v[124:125], v97 offset:24576\n\t" \
  "s_add_u32 s50, s68, 0x191000\n\t" \
  "buffer_load_dwordx4 a[56:59], v4, s[12:15], s50 offen offset:2048 sc0 sc1\n\t" \
  "v_mfma_f32_16x16x128_f8f6f4 v[240:243], v[104:111], a[192:199], v[240:243]\n\t" \
  "ds_read_b64_tr_b8 v[126:127], v97 offset:25600\n\t" \
  "s_waitcnt lgkmcnt(4)\n\t" \
  "v_mfma_f32_16x16x128_f8f6f4 v[128:131], v[112:119], a[224:231], v[128:131]\n\t" \
  "ds_read_b64_tr_b8 v[104:105], v90 offset:24576\n\t" \
  "s_add_u32 s51, s68, 0x191000\n\t" \
  "buffer_load_dwordx4 a[60:63], v4, s[12:15], s51 offen offset:3072 sc0 sc1\n\t" \
  "v_mfma_f32_16x16x128_f8f6f4 v[132:135], v[112:119], a[232:239], v[132:135]\n\t" \
  "ds_read_b64_tr_b8 v[106:107], v90 offset:25600\n\t" \
  "v_mfma_f32_16x16x128_f8f6f4 v[136:139], v[112:119], a[240:247], v[136:139]\n\t" \
  "ds_read_b64_tr_b8 v[108:109], v98 offset:24576\n\t" \
  "s_add_u32 m0, s24, 0x24000\n\t" \
  "s_add_u32 s52, s45, 0x600000\n\t" \
  "buffer_load_dwordx4 v3, s[8:11], s52 offen lds\n\t" \
  "v_mfma_f32_16x16x128_f8f6f4 v[140:143], v[112:119], a[248:255], v[140:143]\n\t" \
  "ds_read_b64_tr_b8 v[110:111], v98 offset:25600\n\t" \
  "s_waitcnt lgkmcnt(4)\n\t" \
  "v_mfma_f32_16x16x128_f8f6f4 v[156:159], v[120:127], a[248:255], v[156:159]\n\t" \
  "ds_read_b64_tr_b8 v[112:113], v91 offset:24576\n\t" \
  "s_add_u32 m0, s24, 0x24400\n\t" \
  "s_add_u32 s53, s45, 0x620000\n\t" \
  "buffer_load_dwordx4 v3, s[8:11], s53 offen lds\n\t" \
  "v_mfma_f32_16x16x128_f8f6f4 v[152:155], v[120:127], a[240:247], v[152:155]\n\t" \
  "ds_read_b64_tr_b8 v[114:115], v91 offset:25600\n\t" \
  "v_mfma_f32_16x16x128_f8f6f4 v[148:151], v[120:127], a[232:239], v[148:151]\n\t" \
  "ds_read_b64_tr_b8 v[116:117], v99 offset:24576\n\t" \
  "s_add_u32 s54, s68, 0x1a0000\n\t" \
  "buffer_load_dwordx4 a[64:67], v4, s[12:15], s54 offen offset:0 sc0 sc1\n\t" \
  "v_mfma_f32_16x16x128_f8f6f4 v[144:147], v[120:127], a[224:231], v[144:147]\n\t" \
  "ds_read_b64_tr_b8 v[118:119], v99 offset:25600\n\t" \
  "s_waitcnt lgkmcnt(4)\n\t" \
  "v_mfma_f32_16x16x128_f8f6f4 v[160:163], v[104:111], a[224:231], v[160:163]\n\t" \
  "ds_read_b64_tr_b8 v[120:121], v92 offset:24576\n\t" \
  "s_add_u32 s55, s68, 0x1a0000\n\t" \
  "buffer_load_dwordx4 a[68:71], v4, s[12:15], s55 offen offset:1024 sc0 sc1\n\t" \
  "v_mfma_f32_16x16x128_f8f6f4 v[164:167], v[104:111], a[232:239], v[164:167]\n\t" \
  "ds_read_b64_tr_b8 v[122:123], v92 offset:25600\n\t" \
  "v_mfma_f32_16x16x128_f8f6f4 v[168:171], v[104:111], a[240:247], v[168:171]\n\t" \
  "ds_read_b64_tr_b8 v[124:125], v100 offset:24576\n\t" \
  "s_add_u32 s48, s68, 0x1a0000\n\t" \
  "buffer_load_dwordx4 a[72:75], v4, s[12:15], s48 offen offset:2048 sc0 sc1\n\t" \
  "v_mfma_f32_16x16x128_f8f6f4 v[172:175], v[104:111], a[248:255], v[172:175]\n\t" \
  "ds_read_b64_tr_b8 v[126:127], v100 offset:25600\n\t" \
  "s_waitcnt lgkmcnt(4)\n\t" \
  "v_mfma_f32_16x16x128_f8f6f4 v[188:191], v[112:119], a[248:255], v[188:191]\n\t" \
  "ds_read_b64_tr_b8 v[104:105], v93 offset:24576\n\t" \
  "s_add_u32 s49, s68, 0x1a0000\n\t" \
  "buffer_load_dwordx4 a[76:79], v4, s[12:15], s49 offen offset:3072 sc0 sc1\n\t" \
  "v_mfma_f32_16x16x128_f8f6f4 v[184:187], v[112:119], a[240:247], v[184:187]\n\t" \
  "ds_read_b64_tr_b8 v[106:107], v93 offset:25600\n\t" \
  "v_mfma_f32_16x16x128_f8f6f4 v[180:183], v[112:119], a[232:239], v[180:183]\n\t" \
  "ds_read_b64_tr_b8 v[108:109], v101 offset:24576\n\t" \
  "s_add_u32 s50, s68, 0x1a1000\n\t" \
  "buffer_load_dwordx4 a[80:83], v4, s[12:15], s50 offen offset:0 sc0 sc1\n\t" \
  "v_mfma_f32_16x16x128_f8f6f4 v[176:179], v[112:119], a[224:231], v[176:179]\n\t" \
  "ds_read_b64_tr_b8 v[110:111], v101 offset:25600\n\t" \
  "s_waitcnt lgkmcnt(4)\n\t" \
  "v_mfma_f32_16x16x128_f8f6f4 v[192:195], v[120:127], a[224:231], v[192:195]\n\t" \
  "ds_read_b64_tr_b8 v[112:113], v94 offset:24576\n\t" \
  "s_add_u32 s51, s68, 0x1a1000\n\t" \
  "buffer_load_dwordx4 a[84:87], v4, s[12:15], s51 offen offset:1024 sc0 sc1\n\t" \
  "v_mfma_f32_16x16x128_f8f6f4 v[196:199], v[120:127], a[232:239], v[196:199]\n\t" \
  "ds_read_b64_tr_b8 v[114:115], v94 offset:25600\n\t" \
  "v_mfma_f32_16x16x128_f8f6f4 v[200:203], v[120:127], a[240:247], v[200:203]\n\t" \
  "ds_read_b64_tr_b8 v[116:117], v102 offset:24576\n\t" \
  "s_add_u32 s52, s68, 0x1a1000\n\t" \
  "buffer_load_dwordx4 a[88:91], v4, s[12:15], s52 offen offset:2048 sc0 sc1\n\t" \
  "v_mfma_f32_16x16x128_f8f6f4 v[204:207], v[120:127], a[248:255], v[204:207]\n\t" \
  "ds_read_b64_tr_b8 v[118:119], v102 offset:25600\n\t" \
  "s_waitcnt lgkmcnt(4)\n\t" \
  "v_mfma_f32_16x16x128_f8f6f4 v[220:223], v[104:111], a[248:255], v[220:223]\n\t" \
  "ds_read_b64_tr_b8 v[120:121], v95 offset:24576\n\t" \
  "s_add_u32 s53, s68, 0x1a1000\n\t" \
  "buffer_load_dwordx4 a[92:95], v4, s[12:15], s53 offen offset:3072 sc0 sc1\n\t" \
  "v_mfma_f32_16x16x128_f8f6f4 v[216:219], v[104:111], a[240:247], v[216:219]\n\t" \
  "ds_read_b64_tr_b8 v[122:123], v95 offset:25600\n\t" \
  "v_mfma_f32_16x16x128_f8f6f4 v[212:215], v[104:111], a[232:239], v[212:215]\n\t" \
  "ds_read_b64_tr_b8 v[124:125], v103 offset:24576\n\t" \
  "s_add_u32 s54, s68, 0x1b0000\n\t" \
  "buffer_load_dwordx4 a[96:99], v4, s[12:15], s54 offen offset:0 sc0 sc1\n\t" \
  "v_mfma_f32_16x16x128_f8f6f4 v[208:211], v[104:111], a[224:231], v[208:211]\n\t" \
  "ds_read_b64_tr_b8 v[126:127], v103 offset:25600\n\t" \
  "s_waitcnt vmcnt(37) lgkmcnt(0)\n\t" \
  "s_barrier\n\t" \
  "v_mfma_f32_16x16x128_f8f6f4 v[224:227], v[112:119], a[224:231], v[224:227]\n\t" \
  "ds_read_b64_tr_b8 v[104:105], v80 offset:0\n\t" \
  "s_add_u32 s55, s68, 0x1b0000\n\t" \
  "buffer_load_dwordx4 a[100:103], v4, s[12:15], s55 offen offset:1024 sc0 sc1\n\t" \
  "v_mfma_f32_16x16x128_f8f6f4 v[228:231], v[112:119], a[232:239], v[228:231]\n\t" \
  "ds_read_b64_tr_b8 v[106:107], v80 offset:1024\n\t" \
  "v_mfma_f32_16x16x128_f8f6f4 v[232:235], v[112:119], a[240:247], v[232:235]\n\t" \
  "ds_read_b64_tr_b8 v[108:109], v88 offset:32768\n\t" \
  "s_add_u32 s48, s68, 0x1b0000\n\t" \
  "buffer_load_dwordx4 a[104:107], v4, s[12:15], s48 offen offset:2048 sc0 sc1\n\t" \
  "v_mfma_f32_16x16x128_f8f6f4 v[236:239], v[112:119], a[248:255], v[236:239]\n\t" \
  "ds_read_b64_tr_b8 v[110:111], v88 offset:33792\n\t" \
  "v_mfma_f32_16x16x128_f8f6f4 v[252:255], v[120:127], a[248:255], v[252:255]\n\t" \
  "ds_read_b64_tr_b8 v[112:113], v81 offset:0\n\t" \
  "s_add_u32 s49, s68, 0x1b0000\n\t" \
  "buffer_load_dwordx4 a[108:111], v4, s[12:15], s49 offen offset:3072 sc0 sc1\n\t" \
  "v_mfma_f32_16x16x128_f8f6f4 v[248:251], v[120:127], a[240:247], v[248:251]\n\t" \
  "ds_read_b64_tr_b8 v[114:115], v81 offset:1024\n\t" \
  "v_mfma_f32_16x16x128_f8f6f4 v[244:247], v[120:127], a[232:239], v[244:247]\n\t" \
  "ds_read_b64_tr_b8 v[116:117], v89 offset:32768\n\t" \
  "s_add_u32 s50, s68, 0x1b1000\n\t" \
  "buffer_load_dwordx4 a[112:115], v4, s[12:15], s50 offen offset:0 sc0 sc1\n\t" \
  "v_mfma_f32_16x16x128_f8f6f4 v[240:243], v[120:127], a[224:231], v[240:243]\n\t" \
  "ds_read_b64_tr_b8 v[118:119], v89 offset:33792\n\t" \
  "s_waitcnt vmcnt(23) lgkmcnt(4)\n\t" \
  "v_mfma_f32_16x16x128_f8f6f4 v[128:131], v[104:111], a[0:7], v[128:131]\n\t" \
  "ds_read_b64_tr_b8 v[120:121], v82 offset:0\n\t" \
  "s_add_u32 s51, s68, 0x1b1000\n\t" \
  "buffer_load_dwordx4 a[116:119], v4, s[12:15], s51 offen offset:1024 sc0 sc1\n\t" \
  "v_mfma_f32_16x16x128_f8f6f4 v[132:135], v[104:111], a[8:15], v[132:135]\n\t" \
  "ds_read_b64_tr_b8 v[122:123], v82 offset:1024\n\t" \
  "v_mfma_f32_16x16x128_f8f6f4 v[136:139], v[104:111], a[16:23], v[136:139]\n\t" \
  "ds_read_b64_tr_b8 v[124:125], v90 offset:32768\n\t" \
  "s_add_u32 s52, s68, 0x1b1000\n\t" \
  "buffer_load_dwordx4 a[120:123], v4, s[12:15], s52 offen offset:2048 sc0 sc1\n\t" \
  "v_mfma_f32_16x16x128_f8f6f4 v[140:143], v[104:111], a[24:31], v[140:143]\n\t" \
  "ds_read_b64_tr_b8 v[126:127], v90 offset:33792\n\t" \
  "s_waitcnt lgkmcnt(4)\n\t" \
  "v_mfma_f32_16x16x128_f8f6f4 v[156:159], v[112:119], a[24:31], v[156:159]\n\t" \
  "ds_read_b64_tr_b8 v[104:105], v83 offset:0\n\t" \
  "s_add_u32 s53, s68, 0x1b1000\n\t" \
  "buffer_load_dwordx4 a[124:127], v4, s[12:15], s53 offen offset:3072 sc0 sc1\n\t" \
  "v_mfma_f32_16x16x128_f8f6f4 v[152:155], v[112:119], a[16:23], v[152:155]\n\t" \
  "ds_read_b64_tr_b8 v[106:107], v83 offset:1024\n\t" \
  "v_mfma_f32_16x16x128_f8f6f4 v[148:151], v[112:119], a[8:15], v[148:151]\n\t" \
  "ds_read_b64_tr_b8 v[108:109], v91 offset:32768\n\t" \
  "s_add_u32 m0, s24, 0x26000\n\t" \
  "s_add_u32 s54, s46, 0x600000\n\t" \
  "buffer_load_dwordx4 v3, s[8:11], s54 offen lds\n\t" \
  "v_mfma_f32_16x16x128_f8f6f4 v[144:147], v[112:119], a[0:7], v[144:147]\n\t" \
  "ds_read_b64_tr_b8 v[110:111], v91 offset:33792\n\t" \
  "s_waitcnt lgkmcnt(4)\n\t" \
  "v_mfma_f32_16x16x128_f8f6f4 v[160:163], v[120:127], a[0:7], v[160:163]\n\t" \
  "ds_read_b64_tr_b8 v[112:113], v84 offset:0\n\t" \
  "s_add_u32 m0, s24, 0x26400\n\t" \
  "s_add_u32 s55, s46, 0x620000\n\t" \
  "buffer_load_dwordx4 v3, s[8:11], s55 offen lds\n\t" \
  "v_mfma_f32_16x16x128_f8f6f4 v[164:167], v[120:127], a[8:15], v[164:167]\n\t" \
  "ds_read_b64_tr_b8 v[114:115], v84 offset:1024\n\t" \
  "v_mfma_f32_16x16x128_f8f6f4 v[168:171], v[120:127], a[16:23], v[168:171]\n\t" \
  "ds_read_b64_tr_b8 v[116:117], v92 offset:32768\n\t" \
  "s_add_u32 s48, s68, 0x1c0000\n\t" \
  "buffer_load_dwordx4 a[128:131], v4, s[12:15], s48 offen offset:0 sc0 sc1\n\t" \
  "v_mfma_f32_16x16x128_f8f6f4 v[172:175], v[120:127], a[24:31], v[172:175]\n\t" \
  "ds_read_b64_tr_b8 v[118:119], v92 offset:33792\n\t" \
  "s_waitcnt lgkmcnt(4)\n\t" \
  "v_mfma_f32_16x16x128_f8f6f4 v[188:191], v[104:111], a[24:31], v[188:191]\n\t" \
  "ds_read_b64_tr_b8 v[120:121], v85 offset:0\n\t" \
  "s_add_u32 s49, s68, 0x1c0000\n\t" \
  "buffer_load_dwordx4 a[132:135], v4, s[12:15], s49 offen offset:1024 sc0 sc1\n\t" \
  "v_mfma_f32_16x16x128_f8f6f4 v[184:187], v[104:111], a[16:23], v[184:187]\n\t" \
  "ds_read_b64_tr_b8 v[122:123], v85 offset:1024\n\t" \
  "v_mfma_f32_16x16x128_f8f6f4 v[180:183], v[104:111], a[8:15], v[180:183]\n\t" \
  "ds_read_b64_tr_b8 v[124:125], v93 offset:32768\n\t" \
  "s_add_u32 s50, s68, 0x1c0000\n\t" \
  "buffer_load_dwordx4 a[136:139], v4, s[12:15], s50 offen offset:2048 sc0 sc1\n\t" \
  "v_mfma_f32_16x16x128_f8f6f4 v[176:179], v[104:111], a[0:7], v[176:179]\n\t" \
  "ds_read_b64_tr_b8 v[126:127], v93 offset:33792\n\t" \
  "s_waitcnt lgkmcnt(4)\n\t" \
  "v_mfma_f32_16x16x128_f8f6f4 v[192:195], v[112:119], a[0:7], v[192:195]\n\t" \
  "ds_read_b64_tr_b8 v[104:105], v86 offset:0\n\t" \
  "s_add_u32 s51, s68, 0x1c0000\n\t" \
  "buffer_load_dwordx4 a[140:143], v4, s[12:15], s51 offen offset:3072 sc0 sc1\n\t" \
  "v_mfma_f32_16x16x128_f8f6f4 v[196:199], v[112:119], a[8:15], v[196:199]\n\t" \
  "ds_read_b64_tr_b8 v[106:107], v86 offset:1024\n\t" \
  "v_mfma_f32_16x16x128_f8f6f4 v[200:203], v[112:119], a[16:23], v[200:203]\n\t" \
  "ds_read_b64_tr_b8 v[108:109], v94 offset:32768\n\t" \
  "s_add_u32 s52, s68, 0x1c1000\n\t" \
  "buffer_load_dwordx4 a[144:147], v4, s[12:15], s52 offen offset:0 sc0 sc1\n\t" \
  "v_mfma_f32_16x16x128_f8f6f4 v[204:207], v[112:119], a[24:31], v[204:207]\n\t" \
  "ds_read_b64_tr_b8 v[110:111], v94 offset:33792\n\t" \
  "s_waitcnt lgkmcnt(4)\n\t" \
  "v_mfma_f32_16x16x128_f8f6f4 v[220:223], v[120:127], a[24:31], v[220:223]\n\t" \
  "ds_read_b64_tr_b8 v[112:113], v87 offset:0\n\t" \
  "s_add_u32 s53, s68, 0x1c1000\n\t" \
  "buffer_load_dwordx4 a[148:151], v4, s[12:15], s53 offen offset:1024 sc0 sc1\n\t" \
  "v_mfma_f32_16x16x128_f8f6f4 v[216:219], v[120:127], a[16:23], v[216:219]\n\t" \
  "ds_read_b64_tr_b8 v[114:115], v87 offset:1024\n\t" \
  "v_mfma_f32_16x16x128_f8f6f4 v[212:215], v[120:127], a[8:15], v[212:215]\n\t" \
  "ds_read_b64_tr_b8 v[116:117], v95 offset:32768\n\t" \
  "s_add_u32 s54, s68, 0x1c1000\n\t" \
  "buffer_load_dwordx4 a[152:155], v4, s[12:15], s54 offen offset:2048 sc0 sc1\n\t" \
  "v_mfma_f32_16x16x128_f8f6f4 v[208:211], v[120:127], a[0:7], v[208:211]\n\t" \
  "ds_read_b64_tr_b8 v[118:119], v95 offset:33792\n\t" \
  "s_waitcnt lgkmcnt(0)\n\t" \
  "s_barrier\n\t" \
  "v_mfma_f32_16x16x128_f8f6f4 v[224:227], v[104:111], a[0:7], v[224:227]\n\t" \
  "ds_read_b64_tr_b8 v[120:121], v80 offset:8192\n\t" \
  "s_add_u32 s55, s68, 0x1c1000\n\t" \
  "buffer_load_dwordx4 a[156:159], v4, s[12:15], s55 offen offset:3072 sc0 sc1\n\t" \
  "v_mfma_f32_16x16x128_f8f6f4 v[228:231], v[104:111], a[8:15], v[228:231]\n\t" \
  "ds_read_b64_tr_b8 v[122:123], v80 offset:9216\n\t" \
  "v_mfma_f32_16x16x128_f8f6f4 v[232:235], v[104:111], a[16:23], v[232:235]\n\t" \
  "ds_read_b64_tr_b8 v[124:125], v88 offset:40960\n\t" \
  "s_add_u32 s48, s68, 0x1d0000\n\t" \
  "buffer_load_dwordx4 a[160:163], v4, s[12:15], s48 offen offset:0 sc0 sc1\n\t" \
  "v_mfma_f32_16x16x128_f8f6f4 v[236:239], v[104:111], a[24:31], v[236:239]\n\t" \
  "ds_read_b64_tr_b8 v[126:127], v88 offset:41984\n\t" \
  "v_mfma_f32_16x16x128_f8f6f4 v[252:255], v[112:119], a[24:31], v[252:255]\n\t" \
  "ds_read_b64_tr_b8 v[104:105], v81 offset:8192\n\t" \
  "s_add_u32 s49, s68, 0x1d0000\n\t" \
  "buffer_load_dwordx4 a[164:167], v4, s[12:15], s49 offen offset:1024 sc0 sc1\n\t" \
  "v_mfma_f32_16x16x128_f8f6f4 v[248:251], v[112:119], a[16:23], v[248:251]\n\t" \
  "ds_read_b64_tr_b8 v[106:107], v81 offset:9216\n\t" \
  "v_mfma_f32_16x16x128_f8f6f4 v[244:247], v[112:119], a[8:15], v[244:247]\n\t" \
  "ds_read_b64_tr_b8 v[108:109], v89 offset:40960\n\t" \
  "s_add_u32 s50, s68, 0x1d0000\n\t" \
  "buffer_load_dwordx4 a[168:171], v4, s[12:15], s50 offen offset:2048 sc0 sc1\n\t" \
  "v_mfma_f32_16x16x128_f8f6f4 v[240:243], v[112:119], a[0:7], v[240:243]\n\t" \
  "ds_read_b64_tr_b8 v[110:111], v89 offset:41984\n\t" \
  "s_waitcnt vmcnt(31) lgkmcnt(4)\n\t" \
  "v_mfma_f32_16x16x128_f8f6f4 v[128:131], v[120:127], a[32:39], v[128:131]\n\t" \
  "ds_read_b64_tr_b8 v[112:113], v82 offset:8192\n\t" \
  "s_add_u32 s51, s68, 0x1d0000\n\t" \
  "buffer_load_dwordx4 a[172:175], v4, s[12:15], s51 offen offset:3072 sc0 sc1\n\t" \
  "v_mfma_f32_16x16x128_f8f6f4 v[132:135], v[120:127], a[40:47], v[132:135]\n\t" \
  "ds_read_b64_tr_b8 v[114:115], v82 offset:9216\n\t" \
  "v_mfma_f32_16x16x128_f8f6f4 v[136:139], v[120:127], a[48:55], v[136:139]\n\t" \
  "ds_read_b64_tr_b8 v[116:117], v90 offset:40960\n\t" \
  "s_add_u32 s52, s68, 0x1d1000\n\t" \
  "buffer_load_dwordx4 a[176:179], v4, s[12:15], s52 offen offset:0 sc0 sc1\n\t" \
  "v_mfma_f32_16x16x128_f8f6f4 v[140:143], v[120:127], a[56:63], v[140:143]\n\t" \
  "ds_read_b64_tr_b8 v[118:119], v90 offset:41984\n\t" \
  "s_waitcnt lgkmcnt(4)\n\t" \
  "v_mfma_f32_16x16x128_f8f6f4 v[156:159], v[104:111], a[56:63], v[156:159]\n\t" \
  "ds_read_b64_tr_b8 v[120:121], v83 offset:8192\n\t" \
  "s_add_u32 s53, s68, 0x1d1000\n\t" \
  "buffer_load_dwordx4 a[180:183], v4, s[12:15], s53 offen offset:1024 sc0 sc1\n\t" \
  "v_mfma_f32_16x16x128_f8f6f4 v[152:155], v[104:111], a[48:55], v[152:155]\n\t" \
  "ds_read_b64_tr_b8 v[122:123], v83 offset:9216\n\t" \
  "v_mfma_f32_16x16x128_f8f6f4 v[148:151], v[104:111], a[40:47], v[148:151]\n\t" \
  "ds_read_b64_tr_b8 v[124:125], v91 offset:40960\n\t" \
  "s_add_u32 s54, s68, 0x1d1000\n\t" \
  "buffer_load_dwordx4 a[184:187], v4, s[12:15], s54 offen offset:2048 sc0 sc1\n\t" \
  "v_mfma_f32_16x16x128_f8f6f4 v[144:147], v[104:111], a[32:39], v[144:147]\n\t" \
  "ds_read_b64_tr_b8 v[126:127], v91 offset:41984\n\t" \
  "s_waitcnt lgkmcnt(4)\n\t" \
  "v_mfma_f32_16x16x128_f8f6f4 v[160:163], v[112:119], a[32:39], v[160:163]\n\t" \
  "ds_read_b64_tr_b8 v[104:105], v84 offset:8192\n\t" \
  "s_add_u32 s55, s68, 0x1d1000\n\t" \
  "buffer_load_dwordx4 a[188:191], v4, s[12:15], s55 offen offset:3072 sc0 sc1\n\t" \
  "v_mfma_f32_16x16x128_f8f6f4 v[164:167], v[112:119], a[40:47], v[164:167]\n\t" \
  "ds_read_b64_tr_b8 v[106:107], v84 offset:9216\n\t" \
  "v_mfma_f32_16x16x128_f8f6f4 v[168:171], v[112:119], a[48:55], v[168:171]\n\t" \
  "ds_read_b64_tr_b8 v[108:109], v92 offset:40960\n\t" \
  "s_add_u32 s48, s68, 0x1e0000\n\t" \
  "buffer_load_dwordx4 a[192:195], v4, s[12:15], s48 offen offset:0 sc0 sc1\n\t" \
  "v_mfma_f32_16x16x128_f8f6f4 v[172:175], v[112:119], a[56:63], v[172:175]\n\t" \
  "ds_read_b64_tr_b8 v[110:111], v92 offset:41984\n\t" \
  "s_waitcnt lgkmcnt(4)\n\t" \
  "v_mfma_f32_16x16x128_f8f6f4 v[188:191], v[120:127], a[56:63], v[188:191]\n\t" \
  "ds_read_b64_tr_b8 v[112:113], v85 offset:8192\n\t" \
  "s_add_u32 s49, s68, 0x1e0000\n\t" \
  "buffer_load_dwordx4 a[196:199], v4, s[12:15], s49 offen offset:1024 sc0 sc1\n\t" \
  "v_mfma_f32_16x16x128_f8f6f4 v[184:187], v[120:127], a[48:55], v[184:187]\n\t" \
  "ds_read_b64_tr_b8 v[114:115], v85 offset:9216\n\t" \
  "v_mfma_f32_16x16x128_f8f6f4 v[180:183], v[120:127], a[40:47], v[180:183]\n\t" \
  "ds_read_b64_tr_b8 v[116:117], v93 offset:40960\n\t" \
  "s_add_u32 s50, s68, 0x1e0000\n\t" \
  "buffer_load_dwordx4 a[200:203], v4, s[12:15], s50 offen offset:2048 sc0 sc1\n\t" \
  "v_mfma_f32_16x16x128_f8f6f4 v[176:179], v[120:127], a[32:39], v[176:179]\n\t" \
  "ds_read_b64_tr_b8 v[118:119], v93 offset:41984\n\t" \
  "s_waitcnt lgkmcnt(4)\n\t" \
  "v_mfma_f32_16x16x128_f8f6f4 v[192:195], v[104:111], a[32:39], v[192:195]\n\t" \
  "ds_read_b64_tr_b8 v[120:121], v86 offset:8192\n\t" \
  "s_add_u32 s51, s68, 0x1e0000\n\t" \
  "buffer_load_dwordx4 a[204:207], v4, s[12:15], s51 offen offset:3072 sc0 sc1\n\t" \
  "v_mfma_f32_16x16x128_f8f6f4 v[196:199], v[104:111], a[40:47], v[196:199]\n\t" \
  "ds_read_b64_tr_b8 v[122:123], v86 offset:9216\n\t" \
  "v_mfma_f32_16x16x128_f8f6f4 v[200:203], v[104:111], a[48:55], v[200:203]\n\t" \
  "ds_read_b64_tr_b8 v[124:125], v94 offset:40960\n\t" \
  "s_add_u32 s52, s68, 0x1e1000\n\t" \
  "buffer_load_dwordx4 a[208:211], v4, s[12:15], s52 offen offset:0 sc0 sc1\n\t" \
  "v_mfma_f32_16x16x128_f8f6f4 v[204:207], v[104:111], a[56:63], v[204:207]\n\t" \
  "ds_read_b64_tr_b8 v[126:127], v94 offset:41984\n\t" \
  "s_waitcnt lgkmcnt(4)\n\t" \
  "v_mfma_f32_16x16x128_f8f6f4 v[220:223], v[112:119], a[56:63], v[220:223]\n\t" \
  "ds_read_b64_tr_b8 v[104:105], v87 offset:8192\n\t" \
  "s_add_u32 s53, s68, 0x1e1000\n\t" \
  "buffer_load_dwordx4 a[212:215], v4, s[12:15], s53 offen offset:1024 sc0 sc1\n\t" \
  "v_mfma_f32_16x16x128_f8f6f4 v[216:219], v[112:119], a[48:55], v[216:219]\n\t" \
  "ds_read_b64_tr_b8 v[106:107], v87 offset:9216\n\t" \
  "v_mfma_f32_16x16x128_f8f6f4 v[212:215], v[112:119], a[40:47], v[212:215]\n\t" \
  "ds_read_b64_tr_b8 v[108:109], v95 offset:40960\n\t" \
  "s_add_u32 s54, s68, 0x1e1000\n\t" \
  "buffer_load_dwordx4 a[216:219], v4, s[12:15], s54 offen offset:2048 sc0 sc1\n\t" \
  "v_mfma_f32_16x16x128_f8f6f4 v[208:211], v[112:119], a[32:39], v[208:211]\n\t" \
  "ds_read_b64_tr_b8 v[110:111], v95 offset:41984\n\t" \
  "s_waitcnt lgkmcnt(0)\n\t" \
  "s_barrier\n\t" \
  "v_mfma_f32_16x16x128_f8f6f4 v[224:227], v[120:127], a[32:39], v[224:227]\n\t" \
  "ds_read_b64_tr_b8 v[112:113], v80 offset:16384\n\t" \
  "s_add_u32 s55, s68, 0x1e1000\n\t" \
  "buffer_load_dwordx4 a[220:223], v4, s[12:15], s55 offen offset:3072 sc0 sc1\n\t" \
  "v_mfma_f32_16x16x128_f8f6f4 v[228:231], v[120:127], a[40:47], v[228:231]\n\t" \
  "ds_read_b64_tr_b8 v[114:115], v80 offset:17408\n\t" \
  "v_mfma_f32_16x16x128_f8f6f4 v[232:235], v[120:127], a[48:55], v[232:235]\n\t" \
  "ds_read_b64_tr_b8 v[116:117], v88 offset:49152\n\t" \
  "s_add_u32 s48, s68, 0x1f0000\n\t" \
  "buffer_load_dwordx4 a[224:227], v4, s[12:15], s48 offen offset:0 sc0 sc1\n\t" \
  "v_mfma_f32_16x16x128_f8f6f4 v[236:239], v[120:127], a[56:63], v[236:239]\n\t" \
  "ds_read_b64_tr_b8 v[118:119], v88 offset:50176\n\t" \
  "v_mfma_f32_16x16x128_f8f6f4 v[252:255], v[104:111], a[56:63], v[252:255]\n\t" \
  "ds_read_b64_tr_b8 v[120:121], v81 offset:16384\n\t" \
  "s_add_u32 s49, s68, 0x1f0000\n\t" \
  "buffer_load_dwordx4 a[228:231], v4, s[12:15], s49 offen offset:1024 sc0 sc1\n\t" \
  "v_mfma_f32_16x16x128_f8f6f4 v[248:251], v[104:111], a[48:55], v[248:251]\n\t" \
  "ds_read_b64_tr_b8 v[122:123], v81 offset:17408\n\t" \
  "v_mfma_f32_16x16x128_f8f6f4 v[244:247], v[104:111], a[40:47], v[244:247]\n\t" \
  "ds_read_b64_tr_b8 v[124:125], v89 offset:49152\n\t" \
  "s_add_u32 s50, s68, 0x1f0000\n\t" \
  "buffer_load_dwordx4 a[232:235], v4, s[12:15], s50 offen offset:2048 sc0 sc1\n\t" \
  "v_mfma_f32_16x16x128_f8f6f4 v[240:243], v[104:111], a[32:39], v[240:243]\n\t" \
  "ds_read_b64_tr_b8 v[126:127], v89 offset:50176\n\t" \
  "s_waitcnt vmcnt(37) lgkmcnt(4)\n\t" \
  "v_mfma_f32_16x16x128_f8f6f4 v[128:131], v[112:119], a[64:71], v[128:131]\n\t" \
  "ds_read_b64_tr_b8 v[104:105], v82 offset:16384\n\t" \
  "s_add_u32 s51, s68, 0x1f0000\n\t" \
  "buffer_load_dwordx4 a[236:239], v4, s[12:15], s51 offen offset:3072 sc0 sc1\n\t" \
  "v_mfma_f32_16x16x128_f8f6f4 v[132:135], v[112:119], a[72:79], v[132:135]\n\t" \
  "ds_read_b64_tr_b8 v[106:107], v82 offset:17408\n\t" \
  "v_mfma_f32_16x16x128_f8f6f4 v[136:139], v[112:119], a[80:87], v[136:139]\n\t" \
  "ds_read_b64_tr_b8 v[108:109], v90 offset:49152\n\t" \
  "s_add_u32 s52, s68, 0x1f1000\n\t" \
  "buffer_load_dwordx4 a[240:243], v4, s[12:15], s52 offen offset:0 sc0 sc1\n\t" \
  "v_mfma_f32_16x16x128_f8f6f4 v[140:143], v[112:119], a[88:95], v[140:143]\n\t" \
  "ds_read_b64_tr_b8 v[110:111], v90 offset:50176\n\t" \
  "s_waitcnt lgkmcnt(4)\n\t" \
  "v_mfma_f32_16x16x128_f8f6f4 v[156:159], v[120:127], a[88:95], v[156:159]\n\t" \
  "ds_read_b64_tr_b8 v[112:113], v83 offset:16384\n\t" \
  "s_add_u32 s53, s68, 0x1f1000\n\t" \
  "buffer_load_dwordx4 a[244:247], v4, s[12:15], s53 offen offset:1024 sc0 sc1\n\t" \
  "v_mfma_f32_16x16x128_f8f6f4 v[152:155], v[120:127], a[80:87], v[152:155]\n\t" \
  "ds_read_b64_tr_b8 v[114:115], v83 offset:17408\n\t" \
  "v_mfma_f32_16x16x128_f8f6f4 v[148:151], v[120:127], a[72:79], v[148:151]\n\t" \
  "ds_read_b64_tr_b8 v[116:117], v91 offset:49152\n\t" \
  "s_add_u32 s54, s68, 0x1f1000\n\t" \
  "buffer_load_dwordx4 a[248:251], v4, s[12:15], s54 offen offset:2048 sc0 sc1\n\t" \
  "v_mfma_f32_16x16x128_f8f6f4 v[144:147], v[120:127], a[64:71], v[144:147]\n\t" \
  "ds_read_b64_tr_b8 v[118:119], v91 offset:50176\n\t" \
  "s_waitcnt lgkmcnt(4)\n\t" \
  "v_mfma_f32_16x16x128_f8f6f4 v[160:163], v[104:111], a[64:71], v[160:163]\n\t" \
  "ds_read_b64_tr_b8 v[120:121], v84 offset:16384\n\t" \
  "s_add_u32 s55, s68, 0x1f1000\n\t" \
  "buffer_load_dwordx4 a[252:255], v4, s[12:15], s55 offen offset:3072 sc0 sc1\n\t" \
  "v_mfma_f32_16x16x128_f8f6f4 v[164:167], v[104:111], a[72:79], v[164:167]\n\t" \
  "ds_read_b64_tr_b8 v[122:123], v84 offset:17408\n\t" \
  "v_mfma_f32_16x16x128_f8f6f4 v[168:171], v[104:111], a[80:87], v[168:171]\n\t" \
  "ds_read_b64_tr_b8 v[124:125], v92 offset:49152\n\t" \
  "v_mfma_f32_16x16x128_f8f6f4 v[172:175], v[104:111], a[88:95], v[172:175]\n\t" \
  "ds_read_b64_tr_b8 v[126:127], v92 offset:50176\n\t" \
  "s_waitcnt lgkmcnt(4)\n\t" \
  "v_mfma_f32_16x16x128_f8f6f4 v[188:191], v[112:119], a[88:95], v[188:191]\n\t" \
  "ds_read_b64_tr_b8 v[104:105], v85 offset:16384\n\t" \
  "v_mfma_f32_16x16x128_f8f6f4 v[184:187], v[112:119], a[80:87], v[184:187]\n\t" \
  "ds_read_b64_tr_b8 v[106:107], v85 offset:17408\n\t" \
  "v_mfma_f32_16x16x128_f8f6f4 v[180:183], v[112:119], a[72:79], v[180:183]\n\t" \
  "ds_read_b64_tr_b8 v[108:109], v93 offset:49152\n\t" \
  "v_mfma_f32_16x16x128_f8f6f4 v[176:179], v[112:119], a[64:71], v[176:179]\n\t" \
  "ds_read_b64_tr_b8 v[110:111], v93 offset:50176\n\t" \
  "s_waitcnt lgkmcnt(4)\n\t" \
  "v_mfma_f32_16x16x128_f8f6f4 v[192:195], v[120:127], a[64:71], v[192:195]\n\t" \
  "ds_read_b64_tr_b8 v[112:113], v86 offset:16384\n\t" \
  "v_mfma_f32_16x16x128_f8f6f4 v[196:199], v[120:127], a[72:79], v[196:199]\n\t" \
  "ds_read_b64_tr_b8 v[114:115], v86 offset:17408\n\t" \
  "v_mfma_f32_16x16x128_f8f6f4 v[200:203], v[120:127], a[80:87], v[200:203]\n\t" \
  "ds_read_b64_tr_b8 v[116:117], v94 offset:49152\n\t" \
  "v_mfma_f32_16x16x128_f8f6f4 v[204:207], v[120:127], a[88:95], v[204:207]\n\t" \
  "ds_read_b64_tr_b8 v[118:119], v94 offset:50176\n\t" \
  "s_waitcnt lgkmcnt(4)\n\t" \
  "v_mfma_f32_16x16x128_f8f6f4 v[220:223], v[104:111], a[88:95], v[220:223]\n\t" \
  "ds_read_b64_tr_b8 v[120:121], v87 offset:16384\n\t" \
  "v_mfma_f32_16x16x128_f8f6f4 v[216:219], v[104:111], a[80:87], v[216:219]\n\t" \
  "ds_read_b64_tr_b8 v[122:123], v87 offset:17408\n\t" \
  "v_mfma_f32_16x16x128_f8f6f4 v[212:215], v[104:111], a[72:79], v[212:215]\n\t" \
  "ds_read_b64_tr_b8 v[124:125], v95 offset:49152\n\t" \
  "v_mfma_f32_16x16x128_f8f6f4 v[208:211], v[104:111], a[64:71], v[208:211]\n\t" \
  "ds_read_b64_tr_b8 v[126:127], v95 offset:50176\n\t" \
  "s_waitcnt vmcnt(32) lgkmcnt(0)\n\t" \
  "s_barrier\n\t" \
  "v_mfma_f32_16x16x128_f8f6f4 v[224:227], v[112:119], a[64:71], v[224:227]\n\t" \
  "ds_read_b64_tr_b8 v[104:105], v80 offset:24576\n\t" \
  "v_mov_b32 v66, s66\n\t" \
  "v_mul_f32 v66, s67, v66\n\t" \
  "v_or_b32 v66, s63, v66\n\t" \
  "v_mov_b32 v67, v66\n\t" \
  "v_mfma_f32_16x16x128_f8f6f4 v[228:231], v[112:119], a[72:79], v[228:231]\n\t" \
  "ds_read_b64_tr_b8 v[106:107], v80 offset:25600\n\t" \
  "v_mfma_f32_16x16x128_f8f6f4 v[232:235], v[112:119], a[80:87], v[232:235]\n\t" \
  "ds_read_b64_tr_b8 v[108:109], v88 offset:57344\n\t" \
  "v_mfma_f32_16x16x128_f8f6f4 v[236:239], v[112:119], a[88:95], v[236:239]\n\t" \
  "ds_read_b64_tr_b8 v[110:111], v88 offset:58368\n\t" \
  "v_mfma_f32_16x16x128_f8f6f4 v[252:255], v[120:127], a[88:95], v[252:255]\n\t" \
  "ds_read_b64_tr_b8 v[112:113], v80 offset:32768\n\t" \
  "v_mfma_f32_16x16x128_f8f6f4 v[248:251], v[120:127], a[80:87], v[248:251]\n\t" \
  "ds_read_b64_tr_b8 v[114:115], v80 offset:33792\n\t" \
  "v_mfma_f32_16x16x128_f8f6f4 v[244:247], v[120:127], a[72:79], v[244:247]\n\t" \
  "ds_read_b64_tr_b8 v[116:117], v96 offset:0\n\t" \
  "v_mfma_f32_16x16x128_f8f6f4 v[240:243], v[120:127], a[64:71], v[240:243]\n\t" \
  "ds_read_b64_tr_b8 v[118:119], v96 offset:1024\n\t" \
  "s_waitcnt lgkmcnt(4)\n\t" \
  "v_mfma_f32_16x16x128_f8f6f4 v[128:131], v[104:111], a[96:103], v[128:131]\n\t" \
  "ds_read_b64_tr_b8 v[120:121], v80 offset:40960\n\t" \
  "v_mfma_f32_16x16x128_f8f6f4 v[132:135], v[104:111], a[104:111], v[132:135]\n\t" \
  "ds_read_b64_tr_b8 v[122:123], v80 offset:41984\n\t" \
  "v_mfma_f32_16x16x128_f8f6f4 v[136:139], v[104:111], a[112:119], v[136:139]\n\t" \
  "ds_read_b64_tr_b8 v[124:125], v96 offset:8192\n\t" \
  "v_mfma_f32_16x16x128_f8f6f4 v[140:143], v[104:111], a[120:127], v[140:143]\n\t" \
  "ds_read_b64_tr_b8 v[126:127], v96 offset:9216\n\t" \
  "s_waitcnt vmcnt(24) lgkmcnt(4)\n\t" \
  "v_mfma_f32_16x16x128_f8f6f4 v[140:143], v[112:119], a[152:159], v[140:143]\n\t" \
  "ds_read_b64_tr_b8 v[104:105], v80 offset:49152\n\t" \
  "v_mfma_f32_16x16x128_f8f6f4 v[136:139], v[112:119], a[144:151], v[136:139]\n\t" \
  "ds_read_b64_tr_b8 v[106:107], v80 offset:50176\n\t" \
  "v_mfma_f32_16x16x128_f8f6f4 v[132:135], v[112:119], a[136:143], v[132:135]\n\t" \
  "ds_read_b64_tr_b8 v[108:109], v96 offset:16384\n\t" \
  "v_mfma_f32_16x16x128_f8f6f4 v[128:131], v[112:119], a[128:135], v[128:131]\n\t" \
  "ds_read_b64_tr_b8 v[110:111], v96 offset:17408\n\t" \
  "s_waitcnt vmcnt(16) lgkmcnt(4)\n\t" \
  "v_mfma_f32_16x16x128_f8f6f4 v[128:131], v[120:127], a[160:167], v[128:131]\n\t" \
  "ds_read_b64_tr_b8 v[112:113], v80 offset:57344\n\t" \
  "v_mfma_f32_16x16x128_f8f6f4 v[132:135], v[120:127], a[168:175], v[132:135]\n\t" \
  "ds_read_b64_tr_b8 v[114:115], v80 offset:58368\n\t" \
  "v_mfma_f32_16x16x128_f8f6f4 v[136:139], v[120:127], a[176:183], v[136:139]\n\t" \
  "ds_read_b64_tr_b8 v[116:117], v96 offset:24576\n\t" \
  "v_mfma_f32_16x16x128_f8f6f4 v[140:143], v[120:127], a[184:191], v[140:143]\n\t" \
  "ds_read_b64_tr_b8 v[118:119], v96 offset:25600\n\t" \
  "s_waitcnt vmcnt(8) lgkmcnt(4)\n\t" \
  "v_mfma_f32_16x16x128_f8f6f4 v[140:143], v[104:111], a[216:223], v[140:143]\n\t" \
  "ds_read_b64_tr_b8 v[120:121], v81 offset:24576\n\t" \
  "v_mfma_f32_16x16x128_f8f6f4 v[136:139], v[104:111], a[208:215], v[136:139]\n\t" \
  "ds_read_b64_tr_b8 v[122:123], v81 offset:25600\n\t" \
  "v_mfma_f32_16x16x128_f8f6f4 v[132:135], v[104:111], a[200:207], v[132:135]\n\t" \
  "ds_read_b64_tr_b8 v[124:125], v89 offset:57344\n\t" \
  "v_mfma_f32_16x16x128_f8f6f4 v[128:131], v[104:111], a[192:199], v[128:131]\n\t" \
  "ds_read_b64_tr_b8 v[126:127], v89 offset:58368\n\t" \
  "s_waitcnt vmcnt(0) lgkmcnt(4)\n\t" \
  "v_mfma_f32_16x16x128_f8f6f4 v[128:131], v[112:119], a[224:231], v[128:131]\n\t" \
  "ds_read_b64_tr_b8 v[104:105], v81 offset:32768\n\t" \
  "v_mfma_f32_16x16x128_f8f6f4 v[132:135], v[112:119], a[232:239], v[132:135]\n\t" \
  "ds_read_b64_tr_b8 v[106:107], v81 offset:33792\n\t" \
  "v_mfma_f32_16x16x128_f8f6f4 v[136:139], v[112:119], a[240:247], v[136:139]\n\t" \
  "ds_read_b64_tr_b8 v[108:109], v97 offset:0\n\t" \
  "v_mfma_f32_16x16x128_f8f6f4 v[140:143], v[112:119], a[248:255], v[140:143]\n\t" \
  "ds_read_b64_tr_b8 v[110:111], v97 offset:1024\n\t" \
  "s_waitcnt lgkmcnt(4)\n\t" \
  "v_mfma_f32_16x16x128_f8f6f4 v[156:159], v[120:127], a[120:127], v[156:159]\n\t" \
  "ds_read_b64_tr_b8 v[112:113], v81 offset:40960\n\t" \
  "v_mfma_f32_16x16x128_f8f6f4 v[152:155], v[120:127], a[112:119], v[152:155]\n\t" \
  "ds_read_b64_tr_b8 v[114:115], v81 offset:41984\n\t" \
  "v_mfma_f32_16x16x128_f8f6f4 v[148:151], v[120:127], a[104:111], v[148:151]\n\t" \
  "ds_read_b64_tr_b8 v[116:117], v97 offset:8192\n\t" \
  "v_mfma_f32_16x16x128_f8f6f4 v[144:147], v[120:127], a[96:103], v[144:147]\n\t" \
  "ds_read_b64_tr_b8 v[118:119], v97 offset:9216\n\t" \
  "s_waitcnt lgkmcnt(4)\n\t" \
  "v_mfma_f32_16x16x128_f8f6f4 v[144:147], v[104:111], a[128:135], v[144:147]\n\t" \
  "ds_read_b64_tr_b8 v[120:121], v81 offset:49152\n\t" \
  "v_pk_mul_f32 v[128:129], v[128:129], v[66:67]\n\t" \
  "v_pk_mul_f32 v[130:131], v[130:131], v[66:67]\n\t" \
  "s_mov_b32 s48, 0x0\n\t" \
  "buffer_store_dwordx4 v[128:131], v13, s[16:19], s48 offen offset:0 sc1\n\t" \
  "v_mfma_f32_16x16x128_f8f6f4 v[148:151], v[104:111], a[136:143], v[148:151]\n\t" \
  "ds_read_b64_tr_b8 v[122:123], v81 offset:50176\n\t" \
  "v_mfma_f32_16x16x128_f8f6f4 v[152:155], v[104:111], a[144:151], v[152:155]\n\t" \
  "ds_read_b64_tr_b8 v[124:125], v97 offset:16384\n\t" \
  "v_mfma_f32_16x16x128_f8f6f4 v[156:159], v[104:111], a[152:159], v[156:159]\n\t" \
  "ds_read_b64_tr_b8 v[126:127], v97 offset:17408\n\t" \
  "s_waitcnt lgkmcnt(4)\n\t" \
  "v_mfma_f32_16x16x128_f8f6f4 v[156:159], v[112:119], a[184:191], v[156:159]\n\t" \
  "ds_read_b64_tr_b8 v[104:105], v81 offset:57344\n\t" \
  "v_pk_mul_f32 v[132:133], v[132:133], v[66:67]\n\t" \
  "v_pk_mul_f32 v[134:135], v[134:135], v[66:67]\n\t" \
  "s_mov_b32 s49, 0x100000\n\t" \
  "buffer_store_dwordx4 v[132:135], v13, s[16:19], s49 offen offset:0 sc1\n\t" \
  "v_mfma_f32_16x16x128_f8f6f4 v[152:155], v[112:119], a[176:183], v[152:155]\n\t" \
  "ds_read_b64_tr_b8 v[106:107], v81 offset:58368\n\t" \
  "v_mfma_f32_16x16x128_f8f6f4 v[148:151], v[112:119], a[168:175], v[148:151]\n\t" \
  "ds_read_b64_tr_b8 v[108:109], v97 offset:24576\n\t" \
  "v_mfma_f32_16x16x128_f8f6f4 v[144:147], v[112:119], a[160:167], v[144:147]\n\t" \
  "ds_read_b64_tr_b8 v[110:111], v97 offset:25600\n\t" \
  "s_waitcnt lgkmcnt(4)\n\t" \
  "v_mfma_f32_16x16x128_f8f6f4 v[144:147], v[120:127], a[192:199], v[144:147]\n\t" \
  "ds_read_b64_tr_b8 v[112:113], v82 offset:24576\n\t" \
  "v_pk_mul_f32 v[136:137], v[136:137], v[66:67]\n\t" \
  "v_pk_mul_f32 v[138:139], v[138:139], v[66:67]\n\t" \
  "s_mov_b32 s50, 0x200000\n\t" \
  "buffer_store_dwordx4 v[136:139], v13, s[16:19], s50 offen offset:0 sc1\n\t" \
  "v_mfma_f32_16x16x128_f8f6f4 v[148:151], v[120:127], a[200:207], v[148:151]\n\t" \
  "ds_read_b64_tr_b8 v[114:115], v82 offset:25600\n\t" \
  "v_mfma_f32_16x16x128_f8f6f4 v[152:155], v[120:127], a[208:215], v[152:155]\n\t" \
  "ds_read_b64_tr_b8 v[116:117], v90 offset:57344\n\t" \
  "v_mfma_f32_16x16x128_f8f6f4 v[156:159], v[120:127], a[216:223], v[156:159]\n\t" \
  "ds_read_b64_tr_b8 v[118:119], v90 offset:58368\n\t" \
  "s_waitcnt lgkmcnt(4)\n\t" \
  "v_mfma_f32_16x16x128_f8f6f4 v[156:159], v[104:111], a[248:255], v[156:159]\n\t" \
  "ds_read_b64_tr_b8 v[120:121], v82 offset:32768\n\t" \
  "v_pk_mul_f32 v[140:141], v[140:141], v[66:67]\n\t" \
  "v_pk_mul_f32 v[142:143], v[142:143], v[66:67]\n\t" \
  "s_mov_b32 s51, 0x300000\n\t" \
  "buffer_store_dwordx4 v[140:143], v13, s[16:19], s51 offen offset:0 sc1\n\t" \
  "v_mfma_f32_16x16x128_f8f6f4 v[152:155], v[104:111], a[240:247], v[152:155]\n\t" \
  "ds_read_b64_tr_b8 v[122:123], v82 offset:33792\n\t" \
  "v_mfma_f32_16x16x128_f8f6f4 v[148:151], v[104:111], a[232:239], v[148:151]\n\t" \
  "ds_read_b64_tr_b8 v[124:125], v98 offset:0\n\t" \
  "v_mfma_f32_16x16x128_f8f6f4 v[144:147], v[104:111], a[224:231], v[144:147]\n\t" \
  "ds_read_b64_tr_b8 v[126:127], v98 offset:1024\n\t" \
  "s_waitcnt lgkmcnt(4)\n\t" \
  "v_mfma_f32_16x16x128_f8f6f4 v[160:163], v[112:119], a[96:103], v[160:163]\n\t" \
  "ds_read_b64_tr_b8 v[104:105], v82 offset:40960\n\t" \
  "v_mfma_f32_16x16x128_f8f6f4 v[164:167], v[112:119], a[104:111], v[164:167]\n\t" \
  "ds_read_b64_tr_b8 v[106:107], v82 offset:41984\n\t" \
  "v_mfma_f32_16x16x128_f8f6f4 v[168:171], v[112:119], a[112:119], v[168:171]\n\t" \
  "ds_read_b64_tr_b8 v[108:109], v98 offset:8192\n\t" \
  "v_mfma_f32_16x16x128_f8f6f4 v[172:175], v[112:119], a[120:127], v[172:175]\n\t" \
  "ds_read_b64_tr_b8 v[110:111], v98 offset:9216\n\t" \
  "s_waitcnt lgkmcnt(4)\n\t" \
  "v_mfma_f32_16x16x128_f8f6f4 v[172:175], v[120:127], a[152:159], v[172:175]\n\t" \
  "ds_read_b64_tr_b8 v[112:113], v82 offset:49152\n\t" \
  "v_pk_mul_f32 v[144:145], v[144:145], v[66:67]\n\t" \
  "v_pk_mul_f32 v[146:147], v[146:147], v[66:67]\n\t" \
  "s_mov_b32 s52, 0x0\n\t" \
  "buffer_store_dwordx4 v[144:147], v13, s[16:19], s52 offen offset:64 sc1\n\t" \
  "v_mfma_f32_16x16x128_f8f6f4 v[168:171], v[120:127], a[144:151], v[168:171]\n\t" \
  "ds_read_b64_tr_b8 v[114:115], v82 offset:50176\n\t" \
  "v_mfma_f32_16x16x128_f8f6f4 v[164:167], v[120:127], a[136:143], v[164:167]\n\t" \
  "ds_read_b64_tr_b8 v[116:117], v98 offset:16384\n\t" \
  "v_mfma_f32_16x16x128_f8f6f4 v[160:163], v[120:127], a[128:135], v[160:163]\n\t" \
  "ds_read_b64_tr_b8 v[118:119], v98 offset:17408\n\t" \
  "s_waitcnt lgkmcnt(4)\n\t" \
  "v_mfma_f32_16x16x128_f8f6f4 v[160:163], v[104:111], a[160:167], v[160:163]\n\t" \
  "ds_read_b64_tr_b8 v[120:121], v82 offset:57344\n\t" \
  "v_pk_mul_f32 v[148:149], v[148:149], v[66:67]\n\t" \
  "v_pk_mul_f32 v[150:151], v[150:151], v[66:67]\n\t" \
  "s_mov_b32 s53, 0x100000\n\t" \
  "buffer_store_dwordx4 v[148:151], v13, s[16:19], s53 offen offset:64 sc1\n\t" \
  "v_mfma_f32_16x16x128_f8f6f4 v[164:167], v[104:111], a[168:175], v[164:167]\n\t" \
  "ds_read_b64_tr_b8 v[122:123], v82 offset:58368\n\t" \
  "v_mfma_f32_16x16x128_f8f6f4 v[168:171], v[104:111], a[176:183], v[168:171]\n\t" \
  "ds_read_b64_tr_b8 v[124:125], v98 offset:24576\n\t" \
  "v_mfma_f32_16x16x128_f8f6f4 v[172:175], v[104:111], a[184:191], v[172:175]\n\t" \
  "ds_read_b64_tr_b8 v[126:127], v98 offset:25600\n\t" \
  "s_waitcnt lgkmcnt(4)\n\t" \
  "v_mfma_f32_16x16x128_f8f6f4 v[172:175], v[112:119], a[216:223], v[172:175]\n\t" \
  "ds_read_b64_tr_b8 v[104:105], v83 offset:24576\n\t" \
  "v_pk_mul_f32 v[152:153], v[152:153], v[66:67]\n\t" \
  "v_pk_mul_f32 v[154:155], v[154:155], v[66:67]\n\t" \
  "s_mov_b32 s54, 0x200000\n\t" \
  "buffer_store_dwordx4 v[152:155], v13, s[16:19], s54 offen offset:64 sc1\n\t" \
  "v_mfma_f32_16x16x128_f8f6f4 v[168:171], v[112:119], a[208:215], v[168:171]\n\t" \
  "ds_read_b64_tr_b8 v[106:107], v83 offset:25600\n\t" \
  "v_mfma_f32_16x16x128_f8f6f4 v[164:167], v[112:119], a[200:207], v[164:167]\n\t" \
  "ds_read_b64_tr_b8 v[108:109], v91 offset:57344\n\t" \
  "v_mfma_f32_16x16x128_f8f6f4 v[160:163], v[112:119], a[192:199], v[160:163]\n\t" \
  "ds_read_b64_tr_b8 v[110:111], v91 offset:58368\n\t" \
  "s_waitcnt lgkmcnt(4)\n\t" \
  "v_mfma_f32_16x16x128_f8f6f4 v[160:163], v[120:127], a[224:231], v[160:163]\n\t" \
  "ds_read_b64_tr_b8 v[112:113], v83 offset:32768\n\t" \
  "v_pk_mul_f32 v[156:157], v[156:157], v[66:67]\n\t" \
  "v_pk_mul_f32 v[158:159], v[158:159], v[66:67]\n\t" \
  "s_mov_b32 s55, 0x300000\n\t" \
  "buffer_store_dwordx4 v[156:159], v13, s[16:19], s55 offen offset:64 sc1\n\t" \
  "v_mfma_f32_16x16x128_f8f6f4 v[164:167], v[120:127], a[232:239], v[164:167]\n\t" \
  "ds_read_b64_tr_b8 v[114:115], v83 offset:33792\n\t" \
  "v_mfma_f32_16x16x128_f8f6f4 v[168:171], v[120:127], a[240:247], v[168:171]\n\t" \
  "ds_read_b64_tr_b8 v[116:117], v99 offset:0\n\t" \
  "v_mfma_f32_16x16x128_f8f6f4 v[172:175], v[120:127], a[248:255], v[172:175]\n\t" \
  "ds_read_b64_tr_b8 v[118:119], v99 offset:1024\n\t" \
  "s_waitcnt lgkmcnt(4)\n\t" \
  "v_mfma_f32_16x16x128_f8f6f4 v[188:191], v[104:111], a[120:127], v[188:191]\n\t" \
  "ds_read_b64_tr_b8 v[120:121], v83 offset:40960\n\t" \
  "v_mfma_f32_16x16x128_f8f6f4 v[184:187], v[104:111], a[112:119], v[184:187]\n\t" \
  "ds_read_b64_tr_b8 v[122:123], v83 offset:41984\n\t" \
  "v_mfma_f32_16x16x128_f8f6f4 v[180:183], v[104:111], a[104:111], v[180:183]\n\t" \
  "ds_read_b64_tr_b8 v[124:125], v99 offset:8192\n\t" \
  "v_mfma_f32_16x16x128_f8f6f4 v[176:179], v[104:111], a[96:103], v[176:179]\n\t" \
  "ds_read_b64_tr_b8 v[126:127], v99 offset:9216\n\t" \
  "s_waitcnt lgkmcnt(4)\n\t" \
  "v_mfma_f32_16x16x128_f8f6f4 v[176:179], v[112:119], a[128:135], v[176:179]\n\t" \
  "ds_read_b64_tr_b8 v[104:105], v83 offset:49152\n\t" \
  "v_pk_mul_f32 v[160:161], v[160:161], v[66:67]\n\t" \
  "v_pk_mul_f32 v[162:163], v[162:163], v[66:67]\n\t" \
  "s_mov_b32 s48, 0x0\n\t" \
  "buffer_store_dwordx4 v[160:163], v13, s[16:19], s48 offen offset:128 sc1\n\t" \
  "v_mfma_f32_16x16x128_f8f6f4 v[180:183], v[112:119], a[136:143], v[180:183]\n\t" \
  "ds_read_b64_tr_b8 v[106:107], v83 offset:50176\n\t" \
  "v_mfma_f32_16x16x128_f8f6f4 v[184:187], v[112:119], a[144:151], v[184:187]\n\t" \
  "ds_read_b64_tr_b8 v[108:109], v99 offset:16384\n\t" \
  "v_mfma_f32_16x16x128_f8f6f4 v[188:191], v[112:119], a[152:159], v[188:191]\n\t" \
  "ds_read_b64_tr_b8 v[110:111], v99 offset:17408\n\t" \
  "s_waitcnt lgkmcnt(4)\n\t" \
  "v_mfma_f32_16x16x128_f8f6f4 v[188:191], v[120:127], a[184:191], v[188:191]\n\t" \
  "ds_read_b64_tr_b8 v[112:113], v83 offset:57344\n\t" \
  "v_pk_mul_f32 v[164:165], v[164:165], v[66:67]\n\t" \
  "v_pk_mul_f32 v[166:167], v[166:167], v[66:67]\n\t" \
  "s_mov_b32 s49, 0x100000\n\t" \
  "buffer_store_dwordx4 v[164:167], v13, s[16:19], s49 offen offset:128 sc1\n\t" \
  "v_mfma_f32_16x16x128_f8f6f4 v[184:187], v[120:127], a[176:183], v[184:187]\n\t" \
  "ds_read_b64_tr_b8 v[114:115], v83 offset:58368\n\t" \
  "v_mfma_f32_16x16x128_f8f6f4 v[180:183], v[120:127], a[168:175], v[180:183]\n\t" \
  "ds_read_b64_tr_b8 v[116:117], v99 offset:24576\n\t" \
  "v_mfma_f32_16x16x128_f8f6f4 v[176:179], v[120:127], a[160:167], v[176:179]\n\t" \
  "ds_read_b64_tr_b8 v[118:119], v99 offset:25600\n\t" \
  "s_waitcnt lgkmcnt(4)\n\t" \
  "v_mfma_f32_16x16x128_f8f6f4 v[176:179], v[104:111], a[192:199], v[176:179]\n\t" \
  "ds_read_b64_tr_b8 v[120:121], v84 offset:24576\n\t" \
  "v_pk_mul_f32 v[168:169], v[168:169], v[66:67]\n\t" \
  "v_pk_mul_f32 v[170:171], v[170:171], v[66:67]\n\t" \
  "s_mov_b32 s50, 0x200000\n\t" \
  "buffer_store_dwordx4 v[168:171], v13, s[16:19], s50 offen offset:128 sc1\n\t" \
  "v_mfma_f32_16x16x128_f8f6f4 v[180:183], v[104:111], a[200:207], v[180:183]\n\t" \
  "ds_read_b64_tr_b8 v[122:123], v84 offset:25600\n\t" \
  "v_mfma_f32_16x16x128_f8f6f4 v[184:187], v[104:111], a[208:215], v[184:187]\n\t" \
  "ds_read_b64_tr_b8 v[124:125], v92 offset:57344\n\t" \
  "v_mfma_f32_16x16x128_f8f6f4 v[188:191], v[104:111], a[216:223], v[188:191]\n\t" \
  "ds_read_b64_tr_b8 v[126:127], v92 offset:58368\n\t" \
  "s_waitcnt lgkmcnt(4)\n\t" \
  "v_mfma_f32_16x16x128_f8f6f4 v[188:191], v[112:119], a[248:255], v[188:191]\n\t" \
  "ds_read_b64_tr_b8 v[104:105], v84 offset:32768\n\t" \
  "v_pk_mul_f32 v[172:173], v[172:173], v[66:67]\n\t" \
  "v_pk_mul_f32 v[174:175], v[174:175], v[66:67]\n\t" \
  "s_mov_b32 s51, 0x300000\n\t" \
  "buffer_store_dwordx4 v[172:175], v13, s[16:19], s51 offen offset:128 sc1\n\t" \
  "v_mfma_f32_16x16x128_f8f6f4 v[184:187], v[112:119], a[240:247], v[184:187]\n\t" \
  "ds_read_b64_tr_b8 v[106:107], v84 offset:33792\n\t" \
  "v_mfma_f32_16x16x128_f8f6f4 v[180:183], v[112:119], a[232:239], v[180:183]\n\t" \
  "ds_read_b64_tr_b8 v[108:109], v100 offset:0\n\t" \
  "v_mfma_f32_16x16x128_f8f6f4 v[176:179], v[112:119], a[224:231], v[176:179]\n\t" \
  "ds_read_b64_tr_b8 v[110:111], v100 offset:1024\n\t" \
  "s_waitcnt lgkmcnt(4)\n\t" \
  "v_mfma_f32_16x16x128_f8f6f4 v[192:195], v[120:127], a[96:103], v[192:195]\n\t" \
  "ds_read_b64_tr_b8 v[112:113], v84 offset:40960\n\t" \
  "v_mfma_f32_16x16x128_f8f6f4 v[196:199], v[120:127], a[104:111], v[196:199]\n\t" \
  "ds_read_b64_tr_b8 v[114:115], v84 offset:41984\n\t" \
  "v_mfma_f32_16x16x128_f8f6f4 v[200:203], v[120:127], a[112:119], v[200:203]\n\t" \
  "ds_read_b64_tr_b8 v[116:117], v100 offset:8192\n\t" \
  "v_mfma_f32_16x16x128_f8f6f4 v[204:207], v[120:127], a[120:127], v[204:207]\n\t" \
  "ds_read_b64_tr_b8 v[118:119], v100 offset:9216\n\t" \
  "s_waitcnt lgkmcnt(4)\n\t" \
  "v_mfma_f32_16x16x128_f8f6f4 v[204:207], v[104:111], a[152:159], v[204:207]\n\t" \
  "ds_read_b64_tr_b8 v[120:121], v84 offset:49152\n\t" \
  "v_pk_mul_f32 v[176:177], v[176:177], v[66:67]\n\t" \
  "v_pk_mul_f32 v[178:179], v[178:179], v[66:67]\n\t" \
  "s_mov_b32 s52, 0x0\n\t" \
  "buffer_store_dwordx4 v[176:179], v13, s[16:19], s52 offen offset:192 sc1\n\t" \
  "v_mfma_f32_16x16x128_f8f6f4 v[200:203], v[104:111], a[144:151], v[200:203]\n\t" \
  "ds_read_b64_tr_b8 v[122:123], v84 offset:50176\n\t" \
  "v_mfma_f32_16x16x128_f8f6f4 v[196:199], v[104:111], a[136:143], v[196:199]\n\t" \
  "ds_read_b64_tr_b8 v[124:125], v100 offset:16384\n\t" \
  "v_mfma_f32_16x16x128_f8f6f4 v[192:195], v[104:111], a[128:135], v[192:195]\n\t" \
  "ds_read_b64_tr_b8 v[126:127], v100 offset:17408\n\t" \
  "s_waitcnt lgkmcnt(4)\n\t" \
  "v_mfma_f32_16x16x128_f8f6f4 v[192:195], v[112:119], a[160:167], v[192:195]\n\t" \
  "ds_read_b64_tr_b8 v[104:105], v84 offset:57344\n\t" \
  "v_pk_mul_f32 v[180:181], v[180:181], v[66:67]\n\t" \
  "v_pk_mul_f32 v[182:183], v[182:183], v[66:67]\n\t" \
  "s_mov_b32 s53, 0x100000\n\t" \
  "buffer_store_dwordx4 v[180:183], v13, s[16:19], s53 offen offset:192 sc1\n\t" \
  "v_mfma_f32_16x16x128_f8f6f4 v[196:199], v[112:119], a[168:175], v[196:199]\n\t" \
  "ds_read_b64_tr_b8 v[106:107], v84 offset:58368\n\t" \
  "v_mfma_f32_16x16x128_f8f6f4 v[200:203], v[112:119], a[176:183], v[200:203]\n\t" \
  "ds_read_b64_tr_b8 v[108:109], v100 offset:24576\n\t" \
  "v_mfma_f32_16x16x128_f8f6f4 v[204:207], v[112:119], a[184:191], v[204:207]\n\t" \
  "ds_read_b64_tr_b8 v[110:111], v100 offset:25600\n\t" \
  "s_waitcnt lgkmcnt(4)\n\t" \
  "v_mfma_f32_16x16x128_f8f6f4 v[204:207], v[120:127], a[216:223], v[204:207]\n\t" \
  "ds_read_b64_tr_b8 v[112:113], v85 offset:24576\n\t" \
  "v_pk_mul_f32 v[184:185], v[184:185], v[66:67]\n\t" \
  "v_pk_mul_f32 v[186:187], v[186:187], v[66:67]\n\t" \
  "s_mov_b32 s54, 0x200000\n\t" \
  "buffer_store_dwordx4 v[184:187], v13, s[16:19], s54 offen offset:192 sc1\n\t" \
  "v_mfma_f32_16x16x128_f8f6f4 v[200:203], v[120:127], a[208:215], v[200:203]\n\t" \
  "ds_read_b64_tr_b8 v[114:115], v85 offset:25600\n\t" \
  "v_mfma_f32_16x16x128_f8f6f4 v[196:199], v[120:127], a[200:207], v[196:199]\n\t" \
  "ds_read_b64_tr_b8 v[116:117], v93 offset:57344\n\t" \
  "v_mfma_f32_16x16x128_f8f6f4 v[192:195], v[120:127], a[192:199], v[192:195]\n\t" \
  "ds_read_b64_tr_b8 v[118:119], v93 offset:58368\n\t" \
  "s_waitcnt lgkmcnt(4)\n\t" \
  "v_mfma_f32_16x16x128_f8f6f4 v[192:195], v[104:111], a[224:231], v[192:195]\n\t" \
  "ds_read_b64_tr_b8 v[120:121], v85 offset:32768\n\t" \
  "v_pk_mul_f32 v[188:189], v[188:189], v[66:67]\n\t" \
  "v_pk_mul_f32 v[190:191], v[190:191], v[66:67]\n\t" \
  "s_mov_b32 s55, 0x300000\n\t" \
  "buffer_store_dwordx4 v[188:191], v13, s[16:19], s55 offen offset:192 sc1\n\t" \
  "v_mfma_f32_16x16x128_f8f6f4 v[196:199], v[104:111], a[232:239], v[196:199]\n\t" \
  "ds_read_b64_tr_b8 v[122:123], v85 offset:33792\n\t" \
  "v_mfma_f32_16x16x128_f8f6f4 v[200:203], v[104:111], a[240:247], v[200:203]\n\t" \
  "ds_read_b64_tr_b8 v[124:125], v101 offset:0\n\t" \
  "v_mfma_f32_16x16x128_f8f6f4 v[204:207], v[104:111], a[248:255], v[204:207]\n\t" \
  "ds_read_b64_tr_b8 v[126:127], v101 offset:1024\n\t" \
  "s_waitcnt lgkmcnt(4)\n\t" \
  "v_mfma_f32_16x16x128_f8f6f4 v[220:223], v[112:119], a[120:127], v[220:223]\n\t" \
  "ds_read_b64_tr_b8 v[104:105], v85 offset:40960\n\t" \
  "v_mfma_f32_16x16x128_f8f6f4 v[216:219], v[112:119], a[112:119], v[216:219]\n\t" \
  "ds_read_b64_tr_b8 v[106:107], v85 offset:41984\n\t" \
  "v_mfma_f32_16x16x128_f8f6f4 v[212:215], v[112:119], a[104:111], v[212:215]\n\t" \
  "ds_read_b64_tr_b8 v[108:109], v101 offset:8192\n\t" \
  "v_mfma_f32_16x16x128_f8f6f4 v[208:211], v[112:119], a[96:103], v[208:211]\n\t" \
  "ds_read_b64_tr_b8 v[110:111], v101 offset:9216\n\t" \
  "s_waitcnt lgkmcnt(4)\n\t" \
  "v_mfma_f32_16x16x128_f8f6f4 v[208:211], v[120:127], a[128:135], v[208:211]\n\t" \
  "ds_read_b64_tr_b8 v[112:113], v85 offset:49152\n\t" \
  "v_pk_mul_f32 v[192:193], v[192:193], v[66:67]\n\t" \
  "v_pk_mul_f32 v[194:195], v[194:195], v[66:67]\n\t" \
  "s_mov_b32 s48, 0x0\n\t" \
  "buffer_store_dwordx4 v[192:195], v13, s[16:19], s48 offen offset:256 sc1\n\t" \
  "v_mfma_f32_16x16x128_f8f6f4 v[212:215], v[120:127], a[136:143], v[212:215]\n\t" \
  "ds_read_b64_tr_b8 v[114:115], v85 offset:50176\n\t" \
  "v_mfma_f32_16x16x128_f8f6f4 v[216:219], v[120:127], a[144:151], v[216:219]\n\t" \
  "ds_read_b64_tr_b8 v[116:117], v101 offset:16384\n\t" \
  "v_mfma_f32_16x16x128_f8f6f4 v[220:223], v[120:127], a[152:159], v[220:223]\n\t" \
  "ds_read_b64_tr_b8 v[118:119], v101 offset:17408\n\t" \
  "s_waitcnt lgkmcnt(4)\n\t" \
  "v_mfma_f32_16x16x128_f8f6f4 v[220:223], v[104:111], a[184:191], v[220:223]\n\t" \
  "ds_read_b64_tr_b8 v[120:121], v85 offset:57344\n\t" \
  "v_pk_mul_f32 v[196:197], v[196:197], v[66:67]\n\t" \
  "v_pk_mul_f32 v[198:199], v[198:199], v[66:67]\n\t" \
  "s_mov_b32 s49, 0x100000\n\t" \
  "buffer_store_dwordx4 v[196:199], v13, s[16:19], s49 offen offset:256 sc1\n\t" \
  "v_mfma_f32_16x16x128_f8f6f4 v[216:219], v[104:111], a[176:183], v[216:219]\n\t" \
  "ds_read_b64_tr_b8 v[122:123], v85 offset:58368\n\t" \
  "v_mfma_f32_16x16x128_f8f6f4 v[212:215], v[104:111], a[168:175], v[212:215]\n\t" \
  "ds_read_b64_tr_b8 v[124:125], v101 offset:24576\n\t" \
  "v_mfma_f32_16x16x128_f8f6f4 v[208:211], v[104:111], a[160:167], v[208:211]\n\t" \
  "ds_read_b64_tr_b8 v[126:127], v101 offset:25600\n\t" \
  "s_waitcnt lgkmcnt(4)\n\t" \
  "v_mfma_f32_16x16x128_f8f6f4 v[208:211], v[112:119], a[192:199], v[208:211]\n\t" \
  "ds_read_b64_tr_b8 v[104:105], v86 offset:24576\n\t" \
  "v_pk_mul_f32 v[200:201], v[200:201], v[66:67]\n\t" \
  "v_pk_mul_f32 v[202:203], v[202:203], v[66:67]\n\t" \
  "s_mov_b32 s50, 0x200000\n\t" \
  "buffer_store_dwordx4 v[200:203], v13, s[16:19], s50 offen offset:256 sc1\n\t" \
  "v_mfma_f32_16x16x128_f8f6f4 v[212:215], v[112:119], a[200:207], v[212:215]\n\t" \
  "ds_read_b64_tr_b8 v[106:107], v86 offset:25600\n\t" \
  "v_mfma_f32_16x16x128_f8f6f4 v[216:219], v[112:119], a[208:215], v[216:219]\n\t" \
  "ds_read_b64_tr_b8 v[108:109], v94 offset:57344\n\t" \
  "v_mfma_f32_16x16x128_f8f6f4 v[220:223], v[112:119], a[216:223], v[220:223]\n\t" \
  "ds_read_b64_tr_b8 v[110:111], v94 offset:58368\n\t" \
  "s_waitcnt lgkmcnt(4)\n\t" \
  "v_mfma_f32_16x16x128_f8f6f4 v[220:223], v[120:127], a[248:255], v[220:223]\n\t" \
  "ds_read_b64_tr_b8 v[112:113], v86 offset:32768\n\t" \
  "v_pk_mul_f32 v[204:205], v[204:205], v[66:67]\n\t" \
  "v_pk_mul_f32 v[206:207], v[206:207], v[66:67]\n\t" \
  "s_mov_b32 s51, 0x300000\n\t" \
  "buffer_store_dwordx4 v[204:207], v13, s[16:19], s51 offen offset:256 sc1\n\t" \
  "v_mfma_f32_16x16x128_f8f6f4 v[216:219], v[120:127], a[240:247], v[216:219]\n\t" \
  "ds_read_b64_tr_b8 v[114:115], v86 offset:33792\n\t" \
  "v_mfma_f32_16x16x128_f8f6f4 v[212:215], v[120:127], a[232:239], v[212:215]\n\t" \
  "ds_read_b64_tr_b8 v[116:117], v102 offset:0\n\t" \
  "v_mfma_f32_16x16x128_f8f6f4 v[208:211], v[120:127], a[224:231], v[208:211]\n\t" \
  "ds_read_b64_tr_b8 v[118:119], v102 offset:1024\n\t" \
  "s_waitcnt lgkmcnt(4)\n\t" \
  "v_mfma_f32_16x16x128_f8f6f4 v[224:227], v[104:111], a[96:103], v[224:227]\n\t" \
  "ds_read_b64_tr_b8 v[120:121], v86 offset:40960\n\t" \
  "v_mfma_f32_16x16x128_f8f6f4 v[228:231], v[104:111], a[104:111], v[228:231]\n\t" \
  "ds_read_b64_tr_b8 v[122:123], v86 offset:41984\n\t" \
  "v_mfma_f32_16x16x128_f8f6f4 v[232:235], v[104:111], a[112:119], v[232:235]\n\t" \
  "ds_read_b64_tr_b8 v[124:125], v102 offset:8192\n\t" \
  "v_mfma_f32_16x16x128_f8f6f4 v[236:239], v[104:111], a[120:127], v[236:239]\n\t" \
  "ds_read_b64_tr_b8 v[126:127], v102 offset:9216\n\t" \
  "s_waitcnt lgkmcnt(4)\n\t" \
  "v_mfma_f32_16x16x128_f8f6f4 v[236:239], v[112:119], a[152:159], v[236:239]\n\t" \
  "ds_read_b64_tr_b8 v[104:105], v86 offset:49152\n\t" \
  "v_pk_mul_f32 v[208:209], v[208:209], v[66:67]\n\t" \
  "v_pk_mul_f32 v[210:211], v[210:211], v[66:67]\n\t" \
  "s_mov_b32 s52, 0x0\n\t" \
  "buffer_store_dwordx4 v[208:211], v13, s[16:19], s52 offen offset:320 sc1\n\t" \
  "v_mfma_f32_16x16x128_f8f6f4 v[232:235], v[112:119], a[144:151], v[232:235]\n\t" \
  "ds_read_b64_tr_b8 v[106:107], v86 offset:50176\n\t" \
  "v_mfma_f32_16x16x128_f8f6f4 v[228:231], v[112:119], a[136:143], v[228:231]\n\t" \
  "ds_read_b64_tr_b8 v[108:109], v102 offset:16384\n\t" \
  "v_mfma_f32_16x16x128_f8f6f4 v[224:227], v[112:119], a[128:135], v[224:227]\n\t" \
  "ds_read_b64_tr_b8 v[110:111], v102 offset:17408\n\t" \
  "s_waitcnt lgkmcnt(4)\n\t" \
  "v_mfma_f32_16x16x128_f8f6f4 v[224:227], v[120:127], a[160:167], v[224:227]\n\t" \
  "ds_read_b64_tr_b8 v[112:113], v86 offset:57344\n\t" \
  "v_pk_mul_f32 v[212:213], v[212:213], v[66:67]\n\t" \
  "v_pk_mul_f32 v[214:215], v[214:215], v[66:67]\n\t" \
  "s_mov_b32 s53, 0x100000\n\t" \
  "buffer_store_dwordx4 v[212:215], v13, s[16:19], s53 offen offset:320 sc1\n\t" \
  "v_mfma_f32_16x16x128_f8f6f4 v[228:231], v[120:127], a[168:175], v[228:231]\n\t" \
  "ds_read_b64_tr_b8 v[114:115], v86 offset:58368\n\t" \
  "v_mfma_f32_16x16x128_f8f6f4 v[232:235], v[120:127], a[176:183], v[232:235]\n\t" \
  "ds_read_b64_tr_b8 v[116:117], v102 offset:24576\n\t" \
  "v_mfma_f32_16x16x128_f8f6f4 v[236:239], v[120:127], a[184:191], v[236:239]\n\t" \
  "ds_read_b64_tr_b8 v[118:119], v102 offset:25600\n\t" \
  "s_waitcnt lgkmcnt(4)\n\t" \
  "v_mfma_f32_16x16x128_f8f6f4 v[236:239], v[104:111], a[216:223], v[236:239]\n\t" \
  "ds_read_b64_tr_b8 v[120:121], v87 offset:24576\n\t" \
  "v_pk_mul_f32 v[216:217], v[216:217], v[66:67]\n\t" \
  "v_pk_mul_f32 v[218:219], v[218:219], v[66:67]\n\t" \
  "s_mov_b32 s54, 0x200000\n\t" \
  "buffer_store_dwordx4 v[216:219], v13, s[16:19], s54 offen offset:320 sc1\n\t" \
  "v_mfma_f32_16x16x128_f8f6f4 v[232:235], v[104:111], a[208:215], v[232:235]\n\t" \
  "ds_read_b64_tr_b8 v[122:123], v87 offset:25600\n\t" \
  "v_mfma_f32_16x16x128_f8f6f4 v[228:231], v[104:111], a[200:207], v[228:231]\n\t" \
  "ds_read_b64_tr_b8 v[124:125], v95 offset:57344\n\t" \
  "v_mfma_f32_16x16x128_f8f6f4 v[224:227], v[104:111], a[192:199], v[224:227]\n\t" \
  "ds_read_b64_tr_b8 v[126:127], v95 offset:58368\n\t" \
  "s_waitcnt lgkmcnt(4)\n\t" \
  "v_mfma_f32_16x16x128_f8f6f4 v[224:227], v[112:119], a[224:231], v[224:227]\n\t" \
  "ds_read_b64_tr_b8 v[104:105], v87 offset:32768\n\t" \
  "v_pk_mul_f32 v[220:221], v[220:221], v[66:67]\n\t" \
  "v_pk_mul_f32 v[222:223], v[222:223], v[66:67]\n\t" \
  "s_mov_b32 s55, 0x300000\n\t" \
  "buffer_store_dwordx4 v[220:223], v13, s[16:19], s55 offen offset:320 sc1\n\t" \
  "v_mfma_f32_16x16x128_f8f6f4 v[228:231], v[112:119], a[232:239], v[228:231]\n\t" \
  "ds_read_b64_tr_b8 v[106:107], v87 offset:33792\n\t" \
  "v_mfma_f32_16x16x128_f8f6f4 v[232:235], v[112:119], a[240:247], v[232:235]\n\t" \
  "ds_read_b64_tr_b8 v[108:109], v103 offset:0\n\t" \
  "v_mfma_f32_16x16x128_f8f6f4 v[236:239], v[112:119], a[248:255], v[236:239]\n\t" \
  "ds_read_b64_tr_b8 v[110:111], v103 offset:1024\n\t" \
  "s_waitcnt lgkmcnt(4)\n\t" \
  "v_mfma_f32_16x16x128_f8f6f4 v[252:255], v[120:127], a[120:127], v[252:255]\n\t" \
  "ds_read_b64_tr_b8 v[112:113], v87 offset:40960\n\t" \
  "v_mfma_f32_16x16x128_f8f6f4 v[248:251], v[120:127], a[112:119], v[248:251]\n\t" \
  "ds_read_b64_tr_b8 v[114:115], v87 offset:41984\n\t" \
  "v_mfma_f32_16x16x128_f8f6f4 v[244:247], v[120:127], a[104:111], v[244:247]\n\t" \
  "ds_read_b64_tr_b8 v[116:117], v103 offset:8192\n\t" \
  "v_mfma_f32_16x16x128_f8f6f4 v[240:243], v[120:127], a[96:103], v[240:243]\n\t" \
  "ds_read_b64_tr_b8 v[118:119], v103 offset:9216\n\t" \
  "s_waitcnt lgkmcnt(4)\n\t" \
  "v_mfma_f32_16x16x128_f8f6f4 v[240:243], v[104:111], a[128:135], v[240:243]\n\t" \
  "ds_read_b64_tr_b8 v[120:121], v87 offset:49152\n\t" \
  "v_pk_mul_f32 v[224:225], v[224:225], v[66:67]\n\t" \
  "v_pk_mul_f32 v[226:227], v[226:227], v[66:67]\n\t" \
  "s_mov_b32 s48, 0x0\n\t" \
  "buffer_store_dwordx4 v[224:227], v13, s[16:19], s48 offen offset:384 sc1\n\t" \
  "v_mfma_f32_16x16x128_f8f6f4 v[244:247], v[104:111], a[136:143], v[244:247]\n\t" \
  "ds_read_b64_tr_b8 v[122:123], v87 offset:50176\n\t" \
  "v_mfma_f32_16x16x128_f8f6f4 v[248:251], v[104:111], a[144:151], v[248:251]\n\t" \
  "ds_read_b64_tr_b8 v[124:125], v103 offset:16384\n\t" \
  "v_mfma_f32_16x16x128_f8f6f4 v[252:255], v[104:111], a[152:159], v[252:255]\n\t" \
  "ds_read_b64_tr_b8 v[126:127], v103 offset:17408\n\t" \
  "s_waitcnt lgkmcnt(4)\n\t" \
  "v_mfma_f32_16x16x128_f8f6f4 v[252:255], v[112:119], a[184:191], v[252:255]\n\t" \
  "ds_read_b64_tr_b8 v[104:105], v87 offset:57344\n\t" \
  "v_pk_mul_f32 v[228:229], v[228:229], v[66:67]\n\t" \
  "v_pk_mul_f32 v[230:231], v[230:231], v[66:67]\n\t" \
  "s_mov_b32 s49, 0x100000\n\t" \
  "buffer_store_dwordx4 v[228:231], v13, s[16:19], s49 offen offset:384 sc1\n\t" \
  "v_mfma_f32_16x16x128_f8f6f4 v[248:251], v[112:119], a[176:183], v[248:251]\n\t" \
  "ds_read_b64_tr_b8 v[106:107], v87 offset:58368\n\t" \
  "v_mfma_f32_16x16x128_f8f6f4 v[244:247], v[112:119], a[168:175], v[244:247]\n\t" \
  "ds_read_b64_tr_b8 v[108:109], v103 offset:24576\n\t" \
  "v_mfma_f32_16x16x128_f8f6f4 v[240:243], v[112:119], a[160:167], v[240:243]\n\t" \
  "ds_read_b64_tr_b8 v[110:111], v103 offset:25600\n\t" \
  "s_waitcnt lgkmcnt(4)\n\t" \
  "v_mfma_f32_16x16x128_f8f6f4 v[240:243], v[120:127], a[192:199], v[240:243]\n\t" \
  "v_pk_mul_f32 v[232:233], v[232:233], v[66:67]\n\t" \
  "v_pk_mul_f32 v[234:235], v[234:235], v[66:67]\n\t" \
  "s_mov_b32 s50, 0x200000\n\t" \
  "buffer_store_dwordx4 v[232:235], v13, s[16:19], s50 offen offset:384 sc1\n\t" \
  "v_mfma_f32_16x16x128_f8f6f4 v[244:247], v[120:127], a[200:207], v[244:247]\n\t" \
  "v_mfma_f32_16x16x128_f8f6f4 v[248:251], v[120:127], a[208:215], v[248:251]\n\t" \
  "v_mfma_f32_16x16x128_f8f6f4 v[252:255], v[120:127], a[216:223], v[252:255]\n\t" \
  "s_waitcnt lgkmcnt(0)\n\t" \
  "v_mfma_f32_16x16x128_f8f6f4 v[252:255], v[104:111], a[248:255], v[252:255]\n\t" \
  "v_pk_mul_f32 v[236:237], v[236:237], v[66:67]\n\t" \
  "v_pk_mul_f32 v[238:239], v[238:239], v[66:67]\n\t" \
  "s_mov_b32 s51, 0x300000\n\t" \
  "buffer_store_dwordx4 v[236:239], v13, s[16:19], s51 offen offset:384 sc1\n\t" \
  "v_mfma_f32_16x16x128_f8f6f4 v[248:251], v[104:111], a[240:247], v[248:251]\n\t" \
  "v_mfma_f32_16x16x128_f8f6f4 v[244:247], v[104:111], a[232:239], v[244:247]\n\t" \
  "v_mfma_f32_16x16x128_f8f6f4 v[240:243], v[104:111], a[224:231], v[240:243]\n\t" \
  "s_nop 15\n\t" \
  "s_nop 7\n\t" \
  "v_pk_mul_f32 v[240:241], v[240:241], v[66:67]\n\t" \
  "v_pk_mul_f32 v[242:243], v[242:243], v[66:67]\n\t" \
  "s_mov_b32 s52, 0x0\n\t" \
  "buffer_store_dwordx4 v[240:243], v13, s[16:19], s52 offen offset:448 sc1\n\t" \
  "v_pk_mul_f32 v[244:245], v[244:245], v[66:67]\n\t" \
  "v_pk_mul_f32 v[246:247], v[246:247], v[66:67]\n\t" \
  "s_mov_b32 s53, 0x100000\n\t" \
  "buffer_store_dwordx4 v[244:247], v13, s[16:19], s53 offen offset:448 sc1\n\t" \
  "v_pk_mul_f32 v[248:249], v[248:249], v[66:67]\n\t" \
  "v_pk_mul_f32 v[250:251], v[250:251], v[66:67]\n\t" \
  "s_mov_b32 s54, 0x200000\n\t" \
  "buffer_store_dwordx4 v[248:251], v13, s[16:19], s54 offen offset:448 sc1\n\t" \
  "v_pk_mul_f32 v[252:253], v[252:253], v[66:67]\n\t" \
  "v_pk_mul_f32 v[254:255], v[254:255], v[66:67]\n\t" \
  "s_mov_b32 s55, 0x300000\n\t" \
  "buffer_store_dwordx4 v[252:255], v13, s[16:19], s55 offen offset:448 sc1\n\t" \
  "s_endpgm\n\t" \
  "Lpoll_1_%=:\n\t" \
  "s_mov_b32 exec_lo, s69\n\t" \
  "s_mov_b32 exec_hi, 1\n\t" \
  "global_load_dwordx4 v[68:71], v10, s[6:7] sc0 sc1\n\t" \
  "s_mov_b64 exec, -1\n\t" \
  "s_waitcnt vmcnt(0)\n\t" \
  "v_min_u32 v68, v68, v69\n\t" \
  "v_min3_u32 v68, v68, v70, v71\n\t" \
  "v_cmp_le_u32 vcc, s57, v68\n\t" \
  "s_and_b32 s64, vcc_lo, s69\n\t" \
  "s_or_b32 s56, s56, s64\n\t" \
  "s_and_b32 s64, s56, s62\n\t" \
  "s_cmp_eq_u32 s64, s62\n\t" \
  "s_cbranch_scc1 Lret_1_%=\n\t" \
  "s_bitcmp1_b32 vcc_hi, 0\n\t" \
  "s_cbranch_scc1 Lfail_1_%=\n\t" \
  "s_memrealtime s[64:65]\n\t" \
  "s_waitcnt lgkmcnt(0)\n\t" \
  "s_sub_u32 s64, s64, s58\n\t" \
  "s_subb_u32 s65, s65, s59\n\t" \
  "s_cmp_lg_u32 s65, 0\n\t" \
  "s_cbranch_scc1 Lfail_1_%=\n\t" \
  "s_cmp_gt_u32 s64, 50000000\n\t" \
  "s_cbranch_scc1 Lfail_1_%=\n\t" \
  "s_sleep 4\n\t" \
  "s_branch Lpoll_1_%=\n\t" \
  "Lfail_1_%=:\n\t" \
  "v_mov_b32 v68, s57\n\t" \
  "v_mov_b32 v69, s57\n\t" \
  "v_mov_b32 v70, s57\n\t" \
  "v_mov_b32 v71, s57\n\t" \
  "s_mov_b32 exec_lo, 0\n\t" \
  "s_mov_b32 exec_hi, 1\n\t" \
  "global_store_dwordx4 v10, v[68:71], s[6:7] sc0 sc1\n\t" \
  "s_mov_b64 exec, -1\n\t" \
  "s_waitcnt vmcnt(0)\n\t" \
  "s_mov_b32 s63, 0x7fc00000\n\t" \
  "s_mov_b32 s56, -1\n\t" \
  "Lret_1_%=:\n\t" \
  "s_setpc_b64 s[60:61]\n\t" \
  "Lend_1_%=:\n\t" \
  "Lvar_2_%=:\n\t" \
  "s_lshl_b32 s64, s4, 15\n\t" \
  "s_add_u32 s64, s64, s78\n\t" \
  "s_lshl_b32 s84, s4, 13\n\t" \
  "v_lshrrev_b32 v17, 3, v1\n\t" \
  "v_lshrrev_b32 v18, 4, v1\n\t" \
  "v_and_b32 v19, 7, v1\n\t" \
  "v_xor_b32 v18, v18, v19\n\t" \
  "v_lshlrev_b32 v18, 4, v18\n\t" \
  "v_lshl_or_b32 v5, v17, 9, v18\n\t" \
  "v_add_u32 v5, s64, v5\n\t" \
  "v_xor_b32 v6, 64, v5\n\t" \
  "v_lshlrev_b32 v4, 4, v1\n\t" \
  "s_lshl_b32 s64, s72, 15\n\t" \
  "s_lshl_b32 s65, s4, 13\n\t" \
  "s_add_u32 s64, s64, s65\n\t" \
  "s_add_u32 s12, s6, s64\n\t" \
  "s_addc_u32 s13, s7, 0\n\t" \
  "s_and_b32 s13, s13, 0xffff\n\t" \
  "s_sub_u32 s14, 0x400000, s64\n\t" \
  "s_mov_b32 s15, 0x20000\n\t" \
  "s_lshl_b32 s64, s72, 24\n\t" \
  "s_lshl_b32 s65, s4, 22\n\t" \
  "s_add_u32 s64, s64, s65\n\t" \
  "s_lshl_b32 s65, s74, 2\n\t" \
  "s_add_u32 s64, s64, s65\n\t" \
  "s_add_u32 s16, s16, s64\n\t" \
  "s_addc_u32 s17, s17, 0\n\t" \
  "s_and_b32 s17, s17, 0xffff\n\t" \
  "s_sub_u32 s18, 0x400000, s65\n\t" \
  "s_mov_b32 s19, 0x20000\n\t" \
  "s_bfe_u32 s26, s27, 0x4000a\n\t" \
  "s_sub_u32 s64, 6, s75\n\t" \
  "s_and_b32 s64, s64, 7\n\t" \
  "s_mul_i32 s65, s77, 7\n\t" \
  "s_add_u32 s65, s65, s64\n\t" \
  "s_add_u32 s65, s65, 4\n\t" \
  "s_cmp_eq_u32 s64, 7\n\t" \
  "s_cselect_b32 s65, s77, s65\n\t" \
  "s_lshl_b32 s65, s65, 16\n\t" \
  "s_lshl_b32 s82, s76, 11\n\t" \
  "s_add_u32 s65, s65, s82\n\t" \
  "v_add_u32 v9, s65, v4\n\t" \
  "s_lshr_b32 s82, s76, 2\n\t" \
  "s_lshl_b32 s82, s82, 9\n\t" \
  "s_lshl_b32 s83, s77, 7\n\t" \
  "s_add_u32 s82, s82, s83\n\t" \
  "s_lshl_b32 s83, s64, 4\n\t" \
  "s_add_u32 s82, s82, s83\n\t" \
  "s_and_b32 s83, s76, 3\n\t" \
  "s_lshl_b32 s83, s83, 2\n\t" \
  "s_add_u32 s82, s82, s83\n\t" \
  "s_add_u32 s82, s82, 0x400000\n\t" \
  "s_lshl_b32 s64, s72, 2\n\t" \
  "s_add_u32 s64, s64, s4\n\t" \
  "s_lshl_b32 s64, s64, 9\n\t" \
  "s_add_u32 s64, s64, 0x400000\n\t" \
  "v_lshl_add_u32 v10, v1, 4, s64\n\t" \
  "v_cmp_lt_u32 vcc, 31, v1\n\t" \
  "v_mov_b32 v18, 0x401000\n\t" \
  "s_nop 0\n\t" \
  "v_cndmask_b32 v10, v10, v18, vcc\n\t" \
  "v_and_b32 v17, 15, v1\n\t" \
  "v_lshrrev_b32 v18, 4, v1\n\t" \
  "v_lshlrev_b32 v19, 4, v18\n\t" \
  "v_lshl_or_b32 v13, v17, 16, v19\n\t" \
  "v_lshl_add_u32 v15, v17, 2, v18\n\t" \
  "v_lshlrev_b32 v15, 2, v15\n\t" \
  "v_lshrrev_b32 v19, 1, v17\n\t" \
  "v_lshl_add_u32 v19, v18, 4, v19\n\t" \
  "v_lshlrev_b32 v19, 7, v19\n\t" \
  "v_bfe_u32 v20, v1, 2, 3\n\t" \
  "v_lshl_or_b32 v19, v20, 4, v19\n\t" \
  "v_and_b32 v20, 1, v1\n\t" \
  "v_lshl_or_b32 v14, v20, 3, v19\n\t" \
  "v_lshrrev_b32 v17, 2, v1\n\t" \
  "v_min_u32 v17, 6, v17\n\t" \
  "v_lshlrev_b32 v17, 16, v17\n\t" \
  "v_and_b32 v18, 3, v1\n\t" \
  "v_lshl_or_b32 v17, v18, 7, v17\n\t" \
  "s_lshl_b32 s64, s73, 9\n\t" \
  "v_add_u32 v16, s64, v17\n\t" \
  "v_and_b32 v17, 15, v1\n\t" \
  "v_lshrrev_b32 v18, 4, v1\n\t" \
  "s_lshl_b32 s64, s72, 2\n\t" \
  "v_add_u32 v18, s64, v18\n\t" \
  "v_lshrrev_b32 v19, 1, v17\n\t" \
  "v_xor_b32 v18, v18, v19\n\t" \
  "v_lshlrev_b32 v18, 4, v18\n\t" \
  "v_lshl_or_b32 v17, v17, 7, v18\n\t" \
  "s_add_u32 s64, s84, 0x10000\n\t" \
  "v_add_u32 v20, s64, v17\n\t" \
  "v_xor_b32 v21, 64, v20\n\t" \
  "s_add_u32 s64, s84, 0x20000\n\t" \
  "v_add_u32 v22, s64, v17\n\t" \
  "v_xor_b32 v23, 64, v22\n\t" \
  "s_add_u32 s40, s25, 1\n\t" \
  "s_and_b32 s40, s40, 7\n\t" \
  "s_lshl_b32 s40, s40, 23\n\t" \
  "s_add_u32 s41, s25, 2\n\t" \
  "s_and_b32 s41, s41, 7\n\t" \
  "s_lshl_b32 s41, s41, 23\n\t" \
  "s_add_u32 s42, s25, 3\n\t" \
  "s_and_b32 s42, s42, 7\n\t" \
  "s_lshl_b32 s42, s42, 23\n\t" \
  "s_add_u32 s43, s25, 4\n\t" \
  "s_and_b32 s43, s43, 7\n\t" \
  "s_lshl_b32 s43, s43, 23\n\t" \
  "s_add_u32 s44, s25, 5\n\t" \
  "s_and_b32 s44, s44, 7\n\t" \
  "s_lshl_b32 s44, s44, 23\n\t" \
  "s_add_u32 s45, s25, 6\n\t" \
  "s_and_b32 s45, s45, 7\n\t" \
  "s_lshl_b32 s45, s45, 23\n\t" \
  "s_add_u32 s46, s25, 7\n\t" \
  "s_and_b32 s46, s46, 7\n\t" \
  "s_lshl_b32 s46, s46, 23\n\t" \
  "v_xor_b32 v80, 0x0, v14\n\t" \
  "v_xor_b32 v81, 0x10, v14\n\t" \
  "v_xor_b32 v82, 0x20, v14\n\t" \
  "v_xor_b32 v83, 0x30, v14\n\t" \
  "v_xor_b32 v84, 0x40, v14\n\t" \
  "v_xor_b32 v85, 0x50, v14\n\t" \
  "v_xor_b32 v86, 0x60, v14\n\t" \
  "v_xor_b32 v87, 0x70, v14\n\t" \
  "v_xor_b32 v88, 0x10000, v14\n\t" \
  "v_xor_b32 v89, 0x10010, v14\n\t" \
  "v_xor_b32 v90, 0x10020, v14\n\t" \
  "v_xor_b32 v91, 0x10030, v14\n\t" \
  "v_xor_b32 v92, 0x10040, v14\n\t" \
  "v_xor_b32 v93, 0x10050, v14\n\t" \
  "v_xor_b32 v94, 0x10060, v14\n\t" \
  "v_xor_b32 v95, 0x10070, v14\n\t" \
  "v_xor_b32 v96, 0x20000, v14\n\t" \
  "v_xor_b32 v97, 0x20010, v14\n\t" \
  "v_xor_b32 v98, 0x20020, v14\n\t" \
  "v_xor_b32 v99, 0x20030, v14\n\t" \
  "v_xor_b32 v100, 0x20040, v14\n\t" \
  "v_xor_b32 v101, 0x20050, v14\n\t" \
  "v_xor_b32 v102, 0x20060, v14\n\t" \
  "v_xor_b32 v103, 0x20070, v14\n\t" \
  "s_mov_b32 s56, 0\n\t" \
  "s_mov_b32 s63, 0\n\t" \
  "s_waitcnt lgkmcnt(0)\n\t" \
  "s_add_u32 s30, s28, s82\n\t" \
  "s_addc_u32 s31, s29, 0\n\t" \
  "s_add_u32 m0, s24, 0x18000\n\t" \
  "s_add_u32 s52, s47, 0x0\n\t" \
  "buffer_load_dwordx4 v3, s[8:11], s52 offen lds\n\t" \
  "s_add_u32 m0, s24, 0x18400\n\t" \
  "s_add_u32 s53, s47, 0x20000\n\t" \
  "buffer_load_dwordx4 v3, s[8:11], s53 offen lds\n\t" \
  "s_add_u32 m0, s84, 0x10000\n\t" \
  "s_mov_b32 s54, 0x0\n\t" \
  "buffer_load_dwordx4 v5, s[20:23], s54 offen lds\n\t" \
  "s_add_u32 m0, s84, 0x10400\n\t" \
  "s_mov_b32 s55, 0x1000\n\t" \
  "buffer_load_dwordx4 v6, s[20:23], s55 offen lds\n\t" \
  "s_add_u32 m0, s84, 0x10800\n\t" \
  "s_mov_b32 s48, 0x2000\n\t" \
  "buffer_load_dwordx4 v5, s[20:23], s48 offen lds\n\t" \
  "s_add_u32 m0, s84, 0x10c00\n\t" \
  "s_mov_b32 s49, 0x3000\n\t" \
  "buffer_load_dwordx4 v6, s[20:23], s49 offen lds\n\t" \
  "s_add_u32 m0, s84, 0x11000\n\t" \
  "s_mov_b32 s50, 0x4000\n\t" \
  "buffer_load_dwordx4 v5, s[20:23], s50 offen lds\n\t" \
  "s_add_u32 m0, s84, 0x11400\n\t" \
  "s_mov_b32 s51, 0x5000\n\t" \
  "buffer_load_dwordx4 v6, s[20:23], s51 offen lds\n\t" \
  "s_add_u32 m0, s84, 0x11800\n\t" \
  "s_mov_b32 s52, 0x6000\n\t" \
  "buffer_load_dwordx4 v5, s[20:23], s52 offen lds\n\t" \
  "s_add_u32 m0, s84, 0x11c00\n\t" \
  "s_mov_b32 s53, 0x7000\n\t" \
  "buffer_load_dwordx4 v6, s[20:23], s53 offen lds\n\t" \
  "s_waitcnt vmcnt(18)\n\t" \
  "v_add_u32 v64, 1, v64\n\t" \
  "s_nop 0\n\t" \
  "v_readfirstlane_b32 s57, v64\n\t" \
  "s_and_b32 s68, s57, 1\n\t" \
  "s_lshl_b32 s68, s68, 21\n\t" \
  "v_add_u32 v63, s68, v9\n\t" \
  "s_waitcnt lgkmcnt(0)\n\t" \
  "s_add_u32 m0, s24, 0x1a000\n\t" \
  "s_add_u32 s54, s47, 0x200000\n\t" \
  "buffer_load_dwordx4 v3, s[8:11], s54 offen lds\n\t" \
  "s_add_u32 m0, s24, 0x1a400\n\t" \
  "s_add_u32 s55, s47, 0x220000\n\t" \
  "buffer_load_dwordx4 v3, s[8:11], s55 offen lds\n\t" \
  "s_waitcnt vmcnt(2)\n\t" \
  "ds_read_b128 a[0:3], v20 offset:0\n\t" \
  "ds_read_b128 a[4:7], v21 offset:0\n\t" \
  "ds_read_b128 a[8:11], v20 offset:2048\n\t" \
  "ds_read_b128 a[12:15], v21 offset:2048\n\t" \
  "ds_read_b128 a[16:19], v20 offset:4096\n\t" \
  "ds_read_b128 a[20:23], v21 offset:4096\n\t" \
  "ds_read_b128 a[24:27], v20 offset:6144\n\t" \
  "ds_read_b128 a[28:31], v21 offset:6144\n\t" \
  "s_waitcnt lgkmcnt(0)\n\t" \
  "s_waitcnt lgkmcnt(0)\n\t" \
  "s_barrier\n\t" \
  "s_mov_b64 exec, 1\n\t" \
  "global_store_dword v65, v64, s[34:35] sc1\n\t" \
  "s_mov_b64 exec, -1\n\t" \
  "ds_read_b64_tr_b8 v[104:105], v80 offset:0\n\t" \
  "ds_read_b64_tr_b8 v[106:107], v80 offset:1024\n\t" \
  "ds_read_b64_tr_b8 v[108:109], v88 offset:32768\n\t" \
  "ds_read_b64_tr_b8 v[110:111], v88 offset:33792\n\t" \
  "ds_read_b64_tr_b8 v[112:113], v81 offset:0\n\t" \
  "ds_read_b64_tr_b8 v[114:115], v81 offset:1024\n\t" \
  "ds_read_b64_tr_b8 v[116:117], v89 offset:32768\n\t" \
  "ds_read_b64_tr_b8 v[118:119], v89 offset:33792\n\t" \
  "s_waitcnt lgkmcnt(4)\n\t" \
  "v_mfma_f32_16x16x128_f8f6f4 v[128:131], v[104:111], a[0:7], 0\n\t" \
  "ds_read_b64_tr_b8 v[120:121], v82 offset:0\n\t" \
  "s_add_u32 m0, s84, 0x20000\n\t" \
  "s_mov_b32 s48, 0x80\n\t" \
  "buffer_load_dwordx4 v5, s[20:23], s48 offen lds\n\t" \
  "v_mfma_f32_16x16x128_f8f6f4 v[132:135], v[104:111], a[8:15], 0\n\t" \
  "ds_read_b64_tr_b8 v[122:123], v82 offset:1024\n\t" \
  "s_add_u32 m0, s84, 0x20400\n\t" \
  "s_mov_b32 s49, 0x1080\n\t" \
  "buffer_load_dwordx4 v6, s[20:23], s49 offen lds\n\t" \
  "v_mfma_f32_16x16x128_f8f6f4 v[136:139], v[104:111], a[16:23], 0\n\t" \
  "ds_read_b64_tr_b8 v[124:125], v90 offset:32768\n\t" \
  "s_add_u32 m0, s84, 0x20800\n\t" \
  "s_mov_b32 s50, 0x2080\n\t" \
  "buffer_load_dwordx4 v5, s[20:23], s50 offen lds\n\t" \
  "v_mfma_f32_16x16x128_f8f6f4 v[140:143], v[104:111], a[24:31], 0\n\t" \
  "ds_read_b64_tr_b8 v[126:127], v90 offset:33792\n\t" \
  "s_add_u32 m0, s84, 0x20c00\n\t" \
  "s_mov_b32 s51, 0x3080\n\t" \
  "buffer_load_dwordx4 v6, s[20:23], s51 offen lds\n\t" \
  "s_waitcnt lgkmcnt(4)\n\t" \
  "v_mfma_f32_16x16x128_f8f6f4 v[156:159], v[112:119], a[24:31], 0\n\t" \
  "ds_read_b64_tr_b8 v[104:105], v83 offset:0\n\t" \
  "s_add_u32 m0, s84, 0x21000\n\t" \
  "s_mov_b32 s52, 0x4080\n\t" \
  "buffer_load_dwordx4 v5, s[20:23], s52 offen lds\n\t" \
  "v_mfma_f32_16x16x128_f8f6f4 v[152:155], v[112:119], a[16:23], 0\n\t" \
  "ds_read_b64_tr_b8 v[106:107], v83 offset:1024\n\t" \
  "s_add_u32 m0, s84, 0x21400\n\t" \
  "s_mov_b32 s53, 0x5080\n\t" \
  "buffer_load_dwordx4 v6, s[20:23], s53 offen lds\n\t" \
  "v_mfma_f32_16x16x128_f8f6f4 v[148:151], v[112:119], a[8:15], 0\n\t" \
  "ds_read_b64_tr_b8 v[108:109], v91 offset:32768\n\t" \
  "s_add_u32 m0, s84, 0x21800\n\t" \
  "s_mov_b32 s54, 0x6080\n\t" \
  "buffer_load_dwordx4 v5, s[20:23], s54 offen lds\n\t" \
  "v_mfma_f32_16x16x128_f8f6f4 v[144:147], v[112:119], a[0:7], 0\n\t" \
  "ds_read_b64_tr_b8 v[110:111], v91 offset:33792\n\t" \
  "s_add_u32 m0, s84, 0x21c00\n\t" \
  "s_mov_b32 s55, 0x7080\n\t" \
  "buffer_load_dwordx4 v6, s[20:23], s55 offen lds\n\t" \
  "s_waitcnt lgkmcnt(4)\n\t" \
  "v_mfma_f32_16x16x128_f8f6f4 v[160:163], v[120:127], a[0:7], 0\n\t" \
  "ds_read_b64_tr_b8 v[112:113], v84 offset:0\n\t" \
  "s_add_u32 m0, s24, 0x4000\n\t" \
  "s_add_u32 s48, s47, 0x400000\n\t" \
  "buffer_load_dwordx4 v2, s[8:11], s48 offen lds\n\t" \
  "v_mfma_f32_16x16x128_f8f6f4 v[164:167], v[120:127], a[8:15], 0\n\t" \
  "ds_read_b64_tr_b8 v[114:115], v84 offset:1024\n\t" \
  "s_add_u32 m0, s24, 0x4400\n\t" \
  "s_add_u32 s49, s47, 0x420000\n\t" \
  "buffer_load_dwordx4 v2, s[8:11], s49 offen lds\n\t" \
  "v_mfma_f32_16x16x128_f8f6f4 v[168:171], v[120:127], a[16:23], 0\n\t" \
  "ds_read_b64_tr_b8 v[116:117], v92 offset:32768\n\t" \
  "s_add_u32 m0, s24, 0x6000\n\t" \
  "s_add_u32 s50, s47, 0x600000\n\t" \
  "buffer_load_dwordx4 v2, s[8:11], s50 offen lds\n\t" \
  "v_mfma_f32_16x16x128_f8f6f4 v[172:175], v[120:127], a[24:31], 0\n\t" \
  "ds_read_b64_tr_b8 v[118:119], v92 offset:33792\n\t" \
  "s_add_u32 m0, s24, 0x6400\n\t" \
  "s_add_u32 s51, s47, 0x620000\n\t" \
  "buffer_load_dwordx4 v2, s[8:11], s51 offen lds\n\t" \
  "s_waitcnt lgkmcnt(4)\n\t" \
  "v_mfma_f32_16x16x128_f8f6f4 v[188:191], v[104:111], a[24:31], 0\n\t" \
  "ds_read_b64_tr_b8 v[120:121], v85 offset:0\n\t" \
  "s_add_u32 m0, s24, 0x8000\n\t" \
  "s_add_u32 s52, s40, 0x0\n\t" \
  "buffer_load_dwordx4 v2, s[8:11], s52 offen lds\n\t" \
  "v_mfma_f32_16x16x128_f8f6f4 v[184:187], v[104:111], a[16:23], 0\n\t" \
  "ds_read_b64_tr_b8 v[122:123], v85 offset:1024\n\t" \
  "s_add_u32 m0, s24, 0x8400\n\t" \
  "s_add_u32 s53, s40, 0x20000\n\t" \
  "buffer_load_dwordx4 v2, s[8:11], s53 offen lds\n\t" \
  "v_mfma_f32_16x16x128_f8f6f4 v[180:183], v[104:111], a[8:15], 0\n\t" \
  "ds_read_b64_tr_b8 v[124:125], v93 offset:32768\n\t" \
  "v_mfma_f32_16x16x128_f8f6f4 v[176:179], v[104:111], a[0:7], 0\n\t" \
  "ds_read_b64_tr_b8 v[126:127], v93 offset:33792\n\t" \
  "s_waitcnt lgkmcnt(4)\n\t" \
  "v_mfma_f32_16x16x128_f8f6f4 v[192:195], v[112:119], a[0:7], 0\n\t" \
  "ds_read_b64_tr_b8 v[104:105], v86 offset:0\n\t" \
  "v_mfma_f32_16x16x128_f8f6f4 v[196:199], v[112:119], a[8:15], 0\n\t" \
  "ds_read_b64_tr_b8 v[106:107], v86 offset:1024\n\t" \
  "v_mfma_f32_16x16x128_f8f6f4 v[200:203], v[112:119], a[16:23], 0\n\t" \
  "ds_read_b64_tr_b8 v[108:109], v94 offset:32768\n\t" \
  "v_mfma_f32_16x16x128_f8f6f4 v[204:207], v[112:119], a[24:31], 0\n\t" \
  "ds_read_b64_tr_b8 v[110:111], v94 offset:33792\n\t" \
  "s_waitcnt lgkmcnt(4)\n\t" \
  "v_mfma_f32_16x16x128_f8f6f4 v[220:223], v[120:127], a[24:31], 0\n\t" \
  "ds_read_b64_tr_b8 v[112:113], v87 offset:0\n\t" \
  "v_mfma_f32_16x16x128_f8f6f4 v[216:219], v[120:127], a[16:23], 0\n\t" \
  "ds_read_b64_tr_b8 v[114:115], v87 offset:1024\n\t" \
  "v_mfma_f32_16x16x128_f8f6f4 v[212:215], v[120:127], a[8:15], 0\n\t" \
  "ds_read_b64_tr_b8 v[116:117], v95 offset:32768\n\t" \
  "s_waitcnt vmcnt(6)\n\t" \
  "ds_read_b128 a[32:35], v22 offset:0\n\t" \
  "v_mfma_f32_16x16x128_f8f6f4 v[208:211], v[120:127], a[0:7], 0\n\t" \
  "ds_read_b64_tr_b8 v[118:119], v95 offset:33792\n\t" \
  "ds_read_b128 a[36:39], v23 offset:0\n\t" \
  "s_waitcnt lgkmcnt(0)\n\t" \
  "s_barrier\n\t" \
  "v_mfma_f32_16x16x128_f8f6f4 v[224:227], v[104:111], a[0:7], 0\n\t" \
  "ds_read_b64_tr_b8 v[120:121], v80 offset:8192\n\t" \
  "ds_read_b128 a[40:43], v22 offset:2048\n\t" \
  "v_mfma_f32_16x16x128_f8f6f4 v[228:231], v[104:111], a[8:15], 0\n\t" \
  "ds_read_b64_tr_b8 v[122:123], v80 offset:9216\n\t" \
  "ds_read_b128 a[44:47], v23 offset:2048\n\t" \
  "v_mfma_f32_16x16x128_f8f6f4 v[232:235], v[104:111], a[16:23], 0\n\t" \
  "ds_read_b64_tr_b8 v[124:125], v88 offset:40960\n\t" \
  "ds_read_b128 a[48:51], v22 offset:4096\n\t" \
  "v_mfma_f32_16x16x128_f8f6f4 v[236:239], v[104:111], a[24:31], 0\n\t" \
  "ds_read_b64_tr_b8 v[126:127], v88 offset:41984\n\t" \
  "ds_read_b128 a[52:55], v23 offset:4096\n\t" \
  "v_mfma_f32_16x16x128_f8f6f4 v[252:255], v[112:119], a[24:31], 0\n\t" \
  "ds_read_b64_tr_b8 v[104:105], v81 offset:8192\n\t" \
  "ds_read_b128 a[56:59], v22 offset:6144\n\t" \
  "v_mfma_f32_16x16x128_f8f6f4 v[248:251], v[112:119], a[16:23], 0\n\t" \
  "ds_read_b64_tr_b8 v[106:107], v81 offset:9216\n\t" \
  "ds_read_b128 a[60:63], v23 offset:6144\n\t" \
  "v_mfma_f32_16x16x128_f8f6f4 v[244:247], v[112:119], a[8:15], 0\n\t" \
  "ds_read_b64_tr_b8 v[108:109], v89 offset:40960\n\t" \
  "v_mfma_f32_16x16x128_f8f6f4 v[240:243], v[112:119], a[0:7], 0\n\t" \
  "ds_read_b64_tr_b8 v[110:111], v89 offset:41984\n\t" \
  "s_waitcnt lgkmcnt(2)\n\t" \
  "v_mfma_f32_16x16x128_f8f6f4 v[128:131], v[120:127], a[32:39], v[128:131]\n\t" \
  "ds_read_b64_tr_b8 v[112:113], v82 offset:8192\n\t" \
  "s_add_u32 m0, s24, 0x1c000\n\t" \
  "s_add_u32 s54, s47, 0x400000\n\t" \
  "buffer_load_dwordx4 v3, s[8:11], s54 offen lds\n\t" \
  "v_mfma_f32_16x16x128_f8f6f4 v[132:135], v[120:127], a[40:47], v[132:135]\n\t" \
  "ds_read_b64_tr_b8 v[114:115], v82 offset:9216\n\t" \
  "s_add_u32 m0, s24, 0x1c400\n\t" \
  "s_add_u32 s55, s47, 0x420000\n\t" \
  "buffer_load_dwordx4 v3, s[8:11], s55 offen lds\n\t" \
  "v_mfma_f32_16x16x128_f8f6f4 v[136:139], v[120:127], a[48:55], v[136:139]\n\t" \
  "ds_read_b64_tr_b8 v[116:117], v90 offset:40960\n\t" \
  "s_add_u32 m0, s84, 0x10000\n\t" \
  "s_mov_b32 s48, 0x100\n\t" \
  "buffer_load_dwordx4 v5, s[20:23], s48 offen lds\n\t" \
  "v_mfma_f32_16x16x128_f8f6f4 v[140:143], v[120:127], a[56:63], v[140:143]\n\t" \
  "ds_read_b64_tr_b8 v[118:119], v90 offset:41984\n\t" \
  "s_add_u32 m0, s84, 0x10400\n\t" \
  "s_mov_b32 s49, 0x1100\n\t" \
  "buffer_load_dwordx4 v6, s[20:23], s49 offen lds\n\t" \
  "s_waitcnt lgkmcnt(4)\n\t" \
  "v_mfma_f32_16x16x128_f8f6f4 v[156:159], v[104:111], a[56:63], v[156:159]\n\t" \
  "ds_read_b64_tr_b8 v[120:121], v83 offset:8192\n\t" \
  "s_add_u32 m0, s84, 0x10800\n\t" \
  "s_mov_b32 s50, 0x2100\n\t" \
  "buffer_load_dwordx4 v5, s[20:23], s50 offen lds\n\t" \
  "v_mfma_f32_16x16x128_f8f6f4 v[152:155], v[104:111], a[48:55], v[152:155]\n\t" \
  "ds_read_b64_tr_b8 v[122:123], v83 offset:9216\n\t" \
  "s_add_u32 m0, s84, 0x10c00\n\t" \
  "s_mov_b32 s51, 0x3100\n\t" \
  "buffer_load_dwordx4 v6, s[20:23], s51 offen lds\n\t" \
  "v_mfma_f32_16x16x128_f8f6f4 v[148:151], v[104:111], a[40:47], v[148:151]\n\t" \
  "ds_read_b64_tr_b8 v[124:125], v91 offset:40960\n\t" \
  "s_add_u32 m0, s84, 0x11000\n\t" \
  "s_mov_b32 s52, 0x4100\n\t" \
  "buffer_load_dwordx4 v5, s[20:23], s52 offen lds\n\t" \
  "v_mfma_f32_16x16x128_f8f6f4 v[144:147], v[104:111], a[32:39], v[144:147]\n\t" \
  "ds_read_b64_tr_b8 v[126:127], v91 offset:41984\n\t" \
  "s_add_u32 m0, s84, 0x11400\n\t" \
  "s_mov_b32 s53, 0x5100\n\t" \
  "buffer_load_dwordx4 v6, s[20:23], s53 offen lds\n\t" \
  "s_waitcnt lgkmcnt(4)\n\t" \
  "v_mfma_f32_16x16x128_f8f6f4 v[160:163], v[112:119], a[32:39], v[160:163]\n\t" \
  "ds_read_b64_tr_b8 v[104:105], v84 offset:8192\n\t" \
  "s_add_u32 m0, s84, 0x11800\n\t" \
  "s_mov_b32 s54, 0x6100\n\t" \
  "buffer_load_dwordx4 v5, s[20:23], s54 offen lds\n\t" \
  "v_mfma_f32_16x16x128_f8f6f4 v[164:167], v[112:119], a[40:47], v[164:167]\n\t" \
  "ds_read_b64_tr_b8 v[106:107], v84 offset:9216\n\t" \
  "s_add_u32 m0, s84, 0x11c00\n\t" \
  "s_mov_b32 s55, 0x7100\n\t" \
  "buffer_load_dwordx4 v6, s[20:23], s55 offen lds\n\t" \
  "v_mfma_f32_16x16x128_f8f6f4 v[168:171], v[112:119], a[48:55], v[168:171]\n\t" \
  "ds_read_b64_tr_b8 v[108:109], v92 offset:40960\n\t" \
  "s_add_u32 m0, s24, 0x1e000\n\t" \
  "s_add_u32 s48, s47, 0x600000\n\t" \
  "buffer_load_dwordx4 v3, s[8:11], s48 offen lds\n\t" \
  "v_mfma_f32_16x16x128_f8f6f4 v[172:175], v[112:119], a[56:63], v[172:175]\n\t" \
  "ds_read_b64_tr_b8 v[110:111], v92 offset:41984\n\t" \
  "s_add_u32 m0, s24, 0x1e400\n\t" \
  "s_add_u32 s49, s47, 0x620000\n\t" \
  "buffer_load_dwordx4 v3, s[8:11], s49 offen lds\n\t" \
  "s_waitcnt lgkmcnt(4)\n\t" \
  "v_mfma_f32_16x16x128_f8f6f4 v[188:191], v[120:127], a[56:63], v[188:191]\n\t" \
  "ds_read_b64_tr_b8 v[112:113], v85 offset:8192\n\t" \
  "s_add_u32 m0, s84, 0x20000\n\t" \
  "s_mov_b32 s50, 0x180\n\t" \
  "buffer_load_dwordx4 v5, s[20:23], s50 offen lds\n\t" \
  "v_mfma_f32_16x16x128_f8f6f4 v[184:187], v[120:127], a[48:55], v[184:187]\n\t" \
  "ds_read_b64_tr_b8 v[114:115], v85 offset:9216\n\t" \
  "s_add_u32 m0, s84, 0x20400\n\t" \
  "s_mov_b32 s51, 0x1180\n\t" \
  "buffer_load_dwordx4 v6, s[20:23], s51 offen lds\n\t" \
  "v_mfma_f32_16x16x128_f8f6f4 v[180:183], v[120:127], a[40:47], v[180:183]\n\t" \
  "ds_read_b64_tr_b8 v[116:117], v93 offset:40960\n\t" \
  "s_add_u32 m0, s84, 0x20800\n\t" \
  "s_mov_b32 s52, 0x2180\n\t" \
  "buffer_load_dwordx4 v5, s[20:23], s52 offen lds\n\t" \
  "v_mfma_f32_16x16x128_f8f6f4 v[176:179], v[120:127], a[32:39], v[176:179]\n\t" \
  "ds_read_b64_tr_b8 v[118:119], v93 offset:41984\n\t" \
  "s_add_u32 m0, s84, 0x20c00\n\t" \
  "s_mov_b32 s53, 0x3180\n\t" \
  "buffer_load_dwordx4 v6, s[20:23], s53 offen lds\n\t" \
  "s_waitcnt lgkmcnt(4)\n\t" \
  "v_mfma_f32_16x16x128_f8f6f4 v[192:195], v[104:111], a[32:39], v[192:195]\n\t" \
  "ds_read_b64_tr_b8 v[120:121], v86 offset:8192\n\t" \
  "s_add_u32 m0, s84, 0x21000\n\t" \
  "s_mov_b32 s54, 0x4180\n\t" \
  "buffer_load_dwordx4 v5, s[20:23], s54 offen lds\n\t" \
  "v_mfma_f32_16x16x128_f8f6f4 v[196:199], v[104:111], a[40:47], v[196:199]\n\t" \
  "ds_read_b64_tr_b8 v[122:123], v86 offset:9216\n\t" \
  "s_add_u32 m0, s84, 0x21400\n\t" \
  "s_mov_b32 s55, 0x5180\n\t" \
  "buffer_load_dwordx4 v6, s[20:23], s55 offen lds\n\t" \
  "v_mfma_f32_16x16x128_f8f6f4 v[200:203], v[104:111], a[48:55], v[200:203]\n\t" \
  "ds_read_b64_tr_b8 v[124:125], v94 offset:40960\n\t" \
  "s_add_u32 m0, s84, 0x21800\n\t" \
  "s_mov_b32 s48, 0x6180\n\t" \
  "buffer_load_dwordx4 v5, s[20:23], s48 offen lds\n\t" \
  "v_mfma_f32_16x16x128_f8f6f4 v[204:207], v[104:111], a[56:63], v[204:207]\n\t" \
  "ds_read_b64_tr_b8 v[126:127], v94 offset:41984\n\t" \
  "s_add_u32 m0, s84, 0x21c00\n\t" \
  "s_mov_b32 s49, 0x7180\n\t" \
  "buffer_load_dwordx4 v6, s[20:23], s49 offen lds\n\t" \
  "s_waitcnt lgkmcnt(4)\n\t" \
  "v_mfma_f32_16x16x128_f8f6f4 v[220:223], v[112:119], a[56:63], v[220:223]\n\t" \
  "ds_read_b64_tr_b8 v[104:105], v87 offset:8192\n\t" \
  "v_mfma_f32_16x16x128_f8f6f4 v[216:219], v[112:119], a[48:55], v[216:219]\n\t" \
  "ds_read_b64_tr_b8 v[106:107], v87 offset:9216\n\t" \
  "v_mfma_f32_16x16x128_f8f6f4 v[212:215], v[112:119], a[40:47], v[212:215]\n\t" \
  "ds_read_b64_tr_b8 v[108:109], v95 offset:40960\n\t" \
  "s_waitcnt vmcnt(10)\n\t" \
  "ds_read_b128 a[64:67], v20 offset:0\n\t" \
  "s_cmp_lg_u32 s26, 1\n\t" \
  "s_cbranch_scc1 Lns1_2_%=\n\t" \
  "global_store_dwordx4 v63, v[72:75], s[28:29] sc0 sc1\n\t" \
  "global_store_dwordx4 v63, v[76:79], s[28:29] offset:1024 sc0 sc1\n\t" \
  "s_branch Lms1_2_%=\n\t" \
  "Lns1_2_%=:\n\t" \
  "buffer_load_dword v62, v65, s[20:23], s22 offen\n\t" \
  "buffer_load_dword v62, v65, s[20:23], s22 offen\n\t" \
  "Lms1_2_%=:\n\t" \
  "s_add_u32 m0, s24, 0xa000\n\t" \
  "s_add_u32 s50, s41, 0x0\n\t" \
  "buffer_load_dwordx4 v2, s[8:11], s50 offen lds\n\t" \
  "v_mfma_f32_16x16x128_f8f6f4 v[208:211], v[112:119], a[32:39], v[208:211]\n\t" \
  "ds_read_b64_tr_b8 v[110:111], v95 offset:41984\n\t" \
  "ds_read_b128 a[68:71], v21 offset:0\n\t" \
  "s_waitcnt lgkmcnt(0)\n\t" \
  "s_barrier\n\t" \
  "v_mfma_f32_16x16x128_f8f6f4 v[224:227], v[120:127], a[32:39], v[224:227]\n\t" \
  "ds_read_b64_tr_b8 v[112:113], v80 offset:16384\n\t" \
  "ds_read_b128 a[72:75], v20 offset:2048\n\t" \
  "s_add_u32 m0, s24, 0xa400\n\t" \
  "s_add_u32 s51, s41, 0x20000\n\t" \
  "buffer_load_dwordx4 v2, s[8:11], s51 offen lds\n\t" \
  "v_mfma_f32_16x16x128_f8f6f4 v[228:231], v[120:127], a[40:47], v[228:231]\n\t" \
  "ds_read_b64_tr_b8 v[114:115], v80 offset:17408\n\t" \
  "ds_read_b128 a[76:79], v21 offset:2048\n\t" \
  "v_mfma_f32_16x16x128_f8f6f4 v[232:235], v[120:127], a[48:55], v[232:235]\n\t" \
  "ds_read_b64_tr_b8 v[116:117], v88 offset:49152\n\t" \
  "ds_read_b128 a[80:83], v20 offset:4096\n\t" \
  "s_add_u32 m0, s24, 0xc000\n\t" \
  "s_add_u32 s52, s42, 0x0\n\t" \
  "buffer_load_dwordx4 v2, s[8:11], s52 offen lds\n\t" \
  "v_mfma_f32_16x16x128_f8f6f4 v[236:239], v[120:127], a[56:63], v[236:239]\n\t" \
  "ds_read_b64_tr_b8 v[118:119], v88 offset:50176\n\t" \
  "ds_read_b128 a[84:87], v21 offset:4096\n\t" \
  "v_mfma_f32_16x16x128_f8f6f4 v[252:255], v[104:111], a[56:63], v[252:255]\n\t" \
  "ds_read_b64_tr_b8 v[120:121], v81 offset:16384\n\t" \
  "ds_read_b128 a[88:91], v20 offset:6144\n\t" \
  "s_add_u32 m0, s24, 0xc400\n\t" \
  "s_add_u32 s53, s42, 0x20000\n\t" \
  "buffer_load_dwordx4 v2, s[8:11], s53 offen lds\n\t" \
  "v_mfma_f32_16x16x128_f8f6f4 v[248:251], v[104:111], a[48:55], v[248:251]\n\t" \
  "ds_read_b64_tr_b8 v[122:123], v81 offset:17408\n\t" \
  "ds_read_b128 a[92:95], v21 offset:6144\n\t" \
  "v_mfma_f32_16x16x128_f8f6f4 v[244:247], v[104:111], a[40:47], v[244:247]\n\t" \
  "ds_read_b64_tr_b8 v[124:125], v89 offset:49152\n\t" \
  "s_add_u32 m0, s24, 0xe000\n\t" \
  "s_add_u32 s54, s43, 0x0\n\t" \
  "buffer_load_dwordx4 v2, s[8:11], s54 offen lds\n\t" \
  "v_mfma_f32_16x16x128_f8f6f4 v[240:243], v[104:111], a[32:39], v[240:243]\n\t" \
  "ds_read_b64_tr_b8 v[126:127], v89 offset:50176\n\t" \
  "s_waitcnt lgkmcnt(2)\n\t" \
  "v_mfma_f32_16x16x128_f8f6f4 v[128:131], v[112:119], a[64:71], v[128:131]\n\t" \
  "ds_read_b64_tr_b8 v[104:105], v82 offset:16384\n\t" \
  "s_waitcnt vmcnt(7)\n\t" \
  "ds_read_b128 a[96:99], v22 offset:0\n\t" \
  "s_add_u32 m0, s24, 0xe400\n\t" \
  "s_add_u32 s55, s43, 0x20000\n\t" \
  "buffer_load_dwordx4 v2, s[8:11], s55 offen lds\n\t" \
  "v_mfma_f32_16x16x128_f8f6f4 v[132:135], v[112:119], a[72:79], v[132:135]\n\t" \
  "ds_read_b64_tr_b8 v[106:107], v82 offset:17408\n\t" \
  "ds_read_b128 a[100:103], v23 offset:0\n\t" \
  "v_mfma_f32_16x16x128_f8f6f4 v[136:139], v[112:119], a[80:87], v[136:139]\n\t" \
  "ds_read_b64_tr_b8 v[108:109], v90 offset:49152\n\t" \
  "ds_read_b128 a[104:107], v22 offset:2048\n\t" \
  "v_mfma_f32_16x16x128_f8f6f4 v[140:143], v[112:119], a[88:95], v[140:143]\n\t" \
  "ds_read_b64_tr_b8 v[110:111], v90 offset:50176\n\t" \
  "ds_read_b128 a[108:111], v23 offset:2048\n\t" \
  "s_waitcnt lgkmcnt(8)\n\t" \
  "v_mfma_f32_16x16x128_f8f6f4 v[156:159], v[120:127], a[88:95], v[156:159]\n\t" \
  "ds_read_b64_tr_b8 v[112:113], v83 offset:16384\n\t" \
  "ds_read_b128 a[112:115], v22 offset:4096\n\t" \
  "v_mfma_f32_16x16x128_f8f6f4 v[152:155], v[120:127], a[80:87], v[152:155]\n\t" \
  "ds_read_b64_tr_b8 v[114:115], v83 offset:17408\n\t" \
  "ds_read_b128 a[116:119], v23 offset:4096\n\t" \
  "v_mfma_f32_16x16x128_f8f6f4 v[148:151], v[120:127], a[72:79], v[148:151]\n\t" \
  "ds_read_b64_tr_b8 v[116:117], v91 offset:49152\n\t" \
  "ds_read_b128 a[120:123], v22 offset:6144\n\t" \
  "v_mfma_f32_16x16x128_f8f6f4 v[144:147], v[120:127], a[64:71], v[144:147]\n\t" \
  "ds_read_b64_tr_b8 v[118:119], v91 offset:50176\n\t" \
  "s_waitcnt lgkmcnt(14)\n\t" \
  "ds_read_b128 a[124:127], v23 offset:6144\n\t" \
  "s_waitcnt lgkmcnt(9)\n\t" \
  "v_mfma_f32_16x16x128_f8f6f4 v[160:163], v[104:111], a[64:71], v[160:163]\n\t" \
  "ds_read_b64_tr_b8 v[120:121], v84 offset:16384\n\t" \
  "v_mfma_f32_16x16x128_f8f6f4 v[164:167], v[104:111], a[72:79], v[164:167]\n\t" \
  "ds_read_b64_tr_b8 v[122:123], v84 offset:17408\n\t" \
  "v_mfma_f32_16x16x128_f8f6f4 v[168:171], v[104:111], a[80:87], v[168:171]\n\t" \
  "ds_read_b64_tr_b8 v[124:125], v92 offset:49152\n\t" \
  "v_mfma_f32_16x16x128_f8f6f4 v[172:175], v[104:111], a[88:95], v[172:175]\n\t" \
  "ds_read_b64_tr_b8 v[126:127], v92 offset:50176\n\t" \
  "s_waitcnt lgkmcnt(5)\n\t" \
  "v_mfma_f32_16x16x128_f8f6f4 v[188:191], v[112:119], a[88:95], v[188:191]\n\t" \
  "ds_read_b64_tr_b8 v[104:105], v85 offset:16384\n\t" \
  "v_mfma_f32_16x16x128_f8f6f4 v[184:187], v[112:119], a[80:87], v[184:187]\n\t" \
  "ds_read_b64_tr_b8 v[106:107], v85 offset:17408\n\t" \
  "v_mfma_f32_16x16x128_f8f6f4 v[180:183], v[112:119], a[72:79], v[180:183]\n\t" \
  "ds_read_b64_tr_b8 v[108:109], v93 offset:49152\n\t" \
  "v_mfma_f32_16x16x128_f8f6f4 v[176:179], v[112:119], a[64:71], v[176:179]\n\t" \
  "ds_read_b64_tr_b8 v[110:111], v93 offset:50176\n\t" \
  "s_waitcnt lgkmcnt(4)\n\t" \
  "v_mfma_f32_16x16x128_f8f6f4 v[192:195], v[120:127], a[64:71], v[192:195]\n\t" \
  "ds_read_b64_tr_b8 v[112:113], v86 offset:16384\n\t" \
  "v_mfma_f32_16x16x128_f8f6f4 v[196:199], v[120:127], a[72:79], v[196:199]\n\t" \
  "ds_read_b64_tr_b8 v[114:115], v86 offset:17408\n\t" \
  "v_mfma_f32_16x16x128_f8f6f4 v[200:203], v[120:127], a[80:87], v[200:203]\n\t" \
  "ds_read_b64_tr_b8 v[116:117], v94 offset:49152\n\t" \
  "v_mfma_f32_16x16x128_f8f6f4 v[204:207], v[120:127], a[88:95], v[204:207]\n\t" \
  "ds_read_b64_tr_b8 v[118:119], v94 offset:50176\n\t" \
  "s_waitcnt lgkmcnt(4)\n\t" \
  "v_mfma_f32_16x16x128_f8f6f4 v[220:223], v[104:111], a[88:95], v[220:223]\n\t" \
  "ds_read_b64_tr_b8 v[120:121], v87 offset:16384\n\t" \
  "v_mfma_f32_16x16x128_f8f6f4 v[216:219], v[104:111], a[80:87], v[216:219]\n\t" \
  "ds_read_b64_tr_b8 v[122:123], v87 offset:17408\n\t" \
  "v_mfma_f32_16x16x128_f8f6f4 v[212:215], v[104:111], a[72:79], v[212:215]\n\t" \
  "ds_read_b64_tr_b8 v[124:125], v95 offset:49152\n\t" \
  "v_mfma_f32_16x16x128_f8f6f4 v[208:211], v[104:111], a[64:71], v[208:211]\n\t" \
  "ds_read_b64_tr_b8 v[126:127], v95 offset:50176\n\t" \
  "s_waitcnt lgkmcnt(0)\n\t" \
  "s_barrier\n\t" \
  "v_mfma_f32_16x16x128_f8f6f4 v[224:227], v[112:119], a[64:71], v[224:227]\n\t" \
  "ds_read_b64_tr_b8 v[104:105], v80 offset:24576\n\t" \
  "s_mov_b32 exec_lo, 0x7f\n\t" \
  "s_mov_b32 exec_hi, 0\n\t" \
  "global_load_dwordx4 v[68:71], v10, s[6:7] sc0 sc1\n\t" \
  "s_mov_b64 exec, -1\n\t" \
  "s_add_u32 m0, s24, 0x20000\n\t" \
  "s_add_u32 s48, s40, 0x0\n\t" \
  "buffer_load_dwordx4 v3, s[8:11], s48 offen lds\n\t" \
  "v_mfma_f32_16x16x128_f8f6f4 v[228:231], v[112:119], a[72:79], v[228:231]\n\t" \
  "ds_read_b64_tr_b8 v[106:107], v80 offset:25600\n\t" \
  "v_mfma_f32_16x16x128_f8f6f4 v[232:235], v[112:119], a[80:87], v[232:235]\n\t" \
  "ds_read_b64_tr_b8 v[108:109], v88 offset:57344\n\t" \
  "s_add_u32 m0, s24, 0x20400\n\t" \
  "s_add_u32 s49, s40, 0x20000\n\t" \
  "buffer_load_dwordx4 v3, s[8:11], s49 offen lds\n\t" \
  "v_mfma_f32_16x16x128_f8f6f4 v[236:239], v[112:119], a[88:95], v[236:239]\n\t" \
  "ds_read_b64_tr_b8 v[110:111], v88 offset:58368\n\t" \
  "v_mfma_f32_16x16x128_f8f6f4 v[252:255], v[120:127], a[88:95], v[252:255]\n\t" \
  "ds_read_b64_tr_b8 v[112:113], v81 offset:24576\n\t" \
  "s_add_u32 m0, s24, 0x22000\n\t" \
  "s_add_u32 s50, s41, 0x0\n\t" \
  "buffer_load_dwordx4 v3, s[8:11], s50 offen lds\n\t" \
  "v_mfma_f32_16x16x128_f8f6f4 v[248:251], v[120:127], a[80:87], v[248:251]\n\t" \
  "ds_read_b64_tr_b8 v[114:115], v81 offset:25600\n\t" \
  "v_mfma_f32_16x16x128_f8f6f4 v[244:247], v[120:127], a[72:79], v[244:247]\n\t" \
  "ds_read_b64_tr_b8 v[116:117], v89 offset:57344\n\t" \
  "s_add_u32 m0, s24, 0x22400\n\t" \
  "s_add_u32 s51, s41, 0x20000\n\t" \
  "buffer_load_dwordx4 v3, s[8:11], s51 offen lds\n\t" \
  "v_mfma_f32_16x16x128_f8f6f4 v[240:243], v[120:127], a[64:71], v[240:243]\n\t" \
  "ds_read_b64_tr_b8 v[118:119], v89 offset:58368\n\t" \
  "s_waitcnt lgkmcnt(4)\n\t" \
  "v_mfma_f32_16x16x128_f8f6f4 v[128:131], v[104:111], a[96:103], v[128:131]\n\t" \
  "ds_read_b64_tr_b8 v[120:121], v82 offset:24576\n\t" \
  "s_waitcnt vmcnt(4)\n\t" \
  "v_min_u32 v68, v68, v69\n\t" \
  "v_min3_u32 v68, v68, v70, v71\n\t" \
  "v_cmp_le_u32 vcc, s57, v68\n\t" \
  "s_and_b32 s64, vcc_lo, 0x7f\n\t" \
  "s_or_b32 s56, s56, s64\n\t" \
  "s_and_b32 s64, s56, 0x1\n\t" \
  "s_cmp_eq_u32 s64, 0x1\n\t" \
  "s_cbranch_scc1 Lok0_2_%=\n\t" \
  "s_mov_b32 s62, 0x1\n\t" \
  "s_mov_b32 s69, 0x7f\n\t" \
  "s_call_b64 s[60:61], Lpoll_2_%=\n\t" \
  "Lok0_2_%=:\n\t" \
  "s_add_u32 s52, s68, 0x40000\n\t" \
  "buffer_load_dwordx4 a[128:131], v4, s[12:15], s52 offen offset:0 sc0 sc1\n\t" \
  "v_mfma_f32_16x16x128_f8f6f4 v[132:135], v[104:111], a[104:111], v[132:135]\n\t" \
  "ds_read_b64_tr_b8 v[122:123], v82 offset:25600\n\t" \
  "v_mfma_f32_16x16x128_f8f6f4 v[136:139], v[104:111], a[112:119], v[136:139]\n\t" \
  "ds_read_b64_tr_b8 v[124:125], v90 offset:57344\n\t" \
  "s_add_u32 s53, s68, 0x40000\n\t" \
  "buffer_load_dwordx4 a[132:135], v4, s[12:15], s53 offen offset:1024 sc0 sc1\n\t" \
  "v_mfma_f32_16x16x128_f8f6f4 v[140:143], v[104:111], a[120:127], v[140:143]\n\t" \
  "ds_read_b64_tr_b8 v[126:127], v90 offset:58368\n\t" \
  "s_waitcnt lgkmcnt(4)\n\t" \
  "v_mfma_f32_16x16x128_f8f6f4 v[156:159], v[112:119], a[120:127], v[156:159]\n\t" \
  "ds_read_b64_tr_b8 v[104:105], v83 offset:24576\n\t" \
  "s_add_u32 s54, s68, 0x40000\n\t" \
  "buffer_load_dwordx4 a[136:139], v4, s[12:15], s54 offen offset:2048 sc0 sc1\n\t" \
  "v_mfma_f32_16x16x128_f8f6f4 v[152:155], v[112:119], a[112:119], v[152:155]\n\t" \
  "ds_read_b64_tr_b8 v[106:107], v83 offset:25600\n\t" \
  "v_mfma_f32_16x16x128_f8f6f4 v[148:151], v[112:119], a[104:111], v[148:151]\n\t" \
  "ds_read_b64_tr_b8 v[108:109], v91 offset:57344\n\t" \
  "s_add_u32 s55, s68, 0x40000\n\t" \
  "buffer_load_dwordx4 a[140:143], v4, s[12:15], s55 offen offset:3072 sc0 sc1\n\t" \
  "v_mfma_f32_16x16x128_f8f6f4 v[144:147], v[112:119], a[96:103], v[144:147]\n\t" \
  "ds_read_b64_tr_b8 v[110:111], v91 offset:58368\n\t" \
  "s_waitcnt lgkmcnt(4)\n\t" \
  "v_mfma_f32_16x16x128_f8f6f4 v[160:163], v[120:127], a[96:103], v[160:163]\n\t" \
  "ds_read_b64_tr_b8 v[112:113], v84 offset:24576\n\t" \
  "s_add_u32 s48, s68, 0x41000\n\t" \
  "buffer_load_dwordx4 a[144:147], v4, s[12:15], s48 offen offset:0 sc0 sc1\n\t" \
  "v_mfma_f32_16x16x128_f8f6f4 v[164:167], v[120:127], a[104:111], v[164:167]\n\t" \
  "ds_read_b64_tr_b8 v[114:115], v84 offset:25600\n\t" \
  "v_mfma_f32_16x16x128_f8f6f4 v[168:171], v[120:127], a[112:119], v[168:171]\n\t" \
  "ds_read_b64_tr_b8 v[116:117], v92 offset:57344\n\t" \
  "s_add_u32 s49, s68, 0x41000\n\t" \
  "buffer_load_dwordx4 a[148:151], v4, s[12:15], s49 offen offset:1024 sc0 sc1\n\t" \
  "v_mfma_f32_16x16x128_f8f6f4 v[172:175], v[120:127], a[120:127], v[172:175]\n\t" \
  "ds_read_b64_tr_b8 v[118:119], v92 offset:58368\n\t" \
  "s_waitcnt lgkmcnt(4)\n\t" \
  "v_mfma_f32_16x16x128_f8f6f4 v[188:191], v[104:111], a[120:127], v[188:191]\n\t" \
  "ds_read_b64_tr_b8 v[120:121], v85 offset:24576\n\t" \
  "s_add_u32 s50, s68, 0x41000\n\t" \
  "buffer_load_dwordx4 a[152:155], v4, s[12:15], s50 offen offset:2048 sc0 sc1\n\t" \
  "v_mfma_f32_16x16x128_f8f6f4 v[184:187], v[104:111], a[112:119], v[184:187]\n\t" \
  "ds_read_b64_tr_b8 v[122:123], v85 offset:25600\n\t" \
  "v_mfma_f32_16x16x128_f8f6f4 v[180:183], v[104:111], a[104:111], v[180:183]\n\t" \
  "ds_read_b64_tr_b8 v[124:125], v93 offset:57344\n\t" \
  "s_add_u32 s51, s68, 0x41000\n\t" \
  "buffer_load_dwordx4 a[156:159], v4, s[12:15], s51 offen offset:3072 sc0 sc1\n\t" \
  "v_mfma_f32_16x16x128_f8f6f4 v[176:179], v[104:111], a[96:103], v[176:179]\n\t" \
  "ds_read_b64_tr_b8 v[126:127], v93 offset:58368\n\t" \
  "s_waitcnt lgkmcnt(4)\n\t" \
  "v_mfma_f32_16x16x128_f8f6f4 v[192:195], v[112:119], a[96:103], v[192:195]\n\t" \
  "ds_read_b64_tr_b8 v[104:105], v86 offset:24576\n\t" \
  "s_and_b32 s64, s56, 0x2\n\t" \
  "s_cmp_eq_u32 s64, 0x2\n\t" \
  "s_cbranch_scc1 Lok1_2_%=\n\t" \
  "s_mov_b32 s62, 0x2\n\t" \
  "s_mov_b32 s69, 0x7f\n\t" \
  "s_call_b64 s[60:61], Lpoll_2_%=\n\t" \
  "Lok1_2_%=:\n\t" \
  "s_add_u32 s52, s68, 0x50000\n\t" \
  "buffer_load_dwordx4 a[160:163], v4, s[12:15], s52 offen offset:0 sc0 sc1\n\t" \
  "v_mfma_f32_16x16x128_f8f6f4 v[196:199], v[112:119], a[104:111], v[196:199]\n\t" \
  "ds_read_b64_tr_b8 v[106:107], v86 offset:25600\n\t" \
  "v_mfma_f32_16x16x128_f8f6f4 v[200:203], v[112:119], a[112:119], v[200:203]\n\t" \
  "ds_read_b64_tr_b8 v[108:109], v94 offset:57344\n\t" \
  "s_add_u32 s53, s68, 0x50000\n\t" \
  "buffer_load_dwordx4 a[164:167], v4, s[12:15], s53 offen offset:1024 sc0 sc1\n\t" \
  "v_mfma_f32_16x16x128_f8f6f4 v[204:207], v[112:119], a[120:127], v[204:207]\n\t" \
  "ds_read_b64_tr_b8 v[110:111], v94 offset:58368\n\t" \
  "s_waitcnt lgkmcnt(4)\n\t" \
  "v_mfma_f32_16x16x128_f8f6f4 v[220:223], v[120:127], a[120:127], v[220:223]\n\t" \
  "ds_read_b64_tr_b8 v[112:113], v87 offset:24576\n\t" \
  "s_add_u32 s54, s68, 0x50000\n\t" \
  "buffer_load_dwordx4 a[168:171], v4, s[12:15], s54 offen offset:2048 sc0 sc1\n\t" \
  "v_mfma_f32_16x16x128_f8f6f4 v[216:219], v[120:127], a[112:119], v[216:219]\n\t" \
  "ds_read_b64_tr_b8 v[114:115], v87 offset:25600\n\t" \
  "v_mfma_f32_16x16x128_f8f6f4 v[212:215], v[120:127], a[104:111], v[212:215]\n\t" \
  "ds_read_b64_tr_b8 v[116:117], v95 offset:57344\n\t" \
  "s_add_u32 s55, s68, 0x50000\n\t" \
  "buffer_load_dwordx4 a[172:175], v4, s[12:15], s55 offen offset:3072 sc0 sc1\n\t" \
  "v_mfma_f32_16x16x128_f8f6f4 v[208:211], v[120:127], a[96:103], v[208:211]\n\t" \
  "ds_read_b64_tr_b8 v[118:119], v95 offset:58368\n\t" \
  "s_waitcnt vmcnt(14) lgkmcnt(0)\n\t" \
  "s_barrier\n\t" \
  "v_mfma_f32_16x16x128_f8f6f4 v[224:227], v[104:111], a[96:103], v[224:227]\n\t" \
  "ds_read_b64_tr_b8 v[120:121], v80 offset:32768\n\t" \
  "s_add_u32 s48, s68, 0x51000\n\t" \
  "buffer_load_dwordx4 a[176:179], v4, s[12:15], s48 offen offset:0 sc0 sc1\n\t" \
  "v_mfma_f32_16x16x128_f8f6f4 v[228:231], v[104:111], a[104:111], v[228:231]\n\t" \
  "ds_read_b64_tr_b8 v[122:123], v80 offset:33792\n\t" \
  "v_mfma_f32_16x16x128_f8f6f4 v[232:235], v[104:111], a[112:119], v[232:235]\n\t" \
  "ds_read_b64_tr_b8 v[124:125], v96 offset:0\n\t" \
  "s_add_u32 s49, s68, 0x51000\n\t" \
  "buffer_load_dwordx4 a[180:183], v4, s[12:15], s49 offen offset:1024 sc0 sc1\n\t" \
  "v_mfma_f32_16x16x128_f8f6f4 v[236:239], v[104:111], a[120:127], v[236:239]\n\t" \
  "ds_read_b64_tr_b8 v[126:127], v96 offset:1024\n\t" \
  "v_mfma_f32_16x16x128_f8f6f4 v[252:255], v[112:119], a[120:127], v[252:255]\n\t" \
  "ds_read_b64_tr_b8 v[104:105], v81 offset:32768\n\t" \
  "s_add_u32 s50, s68, 0x51000\n\t" \
  "buffer_load_dwordx4 a[184:187], v4, s[12:15], s50 offen offset:2048 sc0 sc1\n\t" \
  "v_mfma_f32_16x16x128_f8f6f4 v[248:251], v[112:119], a[112:119], v[248:251]\n\t" \
  "ds_read_b64_tr_b8 v[106:107], v81 offset:33792\n\t" \
  "v_mfma_f32_16x16x128_f8f6f4 v[244:247], v[112:119], a[104:111], v[244:247]\n\t" \
  "ds_read_b64_tr_b8 v[108:109], v97 offset:0\n\t" \
  "s_add_u32 s51, s68, 0x51000\n\t" \
  "buffer_load_dwordx4 a[188:191], v4, s[12:15], s51 offen offset:3072 sc0 sc1\n\t" \
  "v_mfma_f32_16x16x128_f8f6f4 v[240:243], v[112:119], a[96:103], v[240:243]\n\t" \
  "ds_read_b64_tr_b8 v[110:111], v97 offset:1024\n\t" \
  "s_waitcnt vmcnt(8) lgkmcnt(4)\n\t" \
  "v_mfma_f32_16x16x128_f8f6f4 v[128:131], v[120:127], a[128:135], v[128:131]\n\t" \
  "ds_read_b64_tr_b8 v[112:113], v82 offset:32768\n\t" \
  "s_add_u32 m0, s24, 0x24000\n\t" \
  "s_add_u32 s52, s42, 0x0\n\t" \
  "buffer_load_dwordx4 v3, s[8:11], s52 offen lds\n\t" \
  "v_mfma_f32_16x16x128_f8f6f4 v[132:135], v[120:127], a[136:143], v[132:135]\n\t" \
  "ds_read_b64_tr_b8 v[114:115], v82 offset:33792\n\t" \
  "v_mfma_f32_16x16x128_f8f6f4 v[136:139], v[120:127], a[144:151], v[136:139]\n\t" \
  "ds_read_b64_tr_b8 v[116:117], v98 offset:0\n\t" \
  "s_add_u32 m0, s24, 0x24400\n\t" \
  "s_add_u32 s53, s42, 0x20000\n\t" \
  "buffer_load_dwordx4 v3, s[8:11], s53 offen lds\n\t" \
  "v_mfma_f32_16x16x128_f8f6f4 v[140:143], v[120:127], a[152:159], v[140:143]\n\t" \
  "ds_read_b64_tr_b8 v[118:119], v98 offset:1024\n\t" \
  "s_waitcnt lgkmcnt(4)\n\t" \
  "v_mfma_f32_16x16x128_f8f6f4 v[156:159], v[104:111], a[152:159], v[156:159]\n\t" \
  "ds_read_b64_tr_b8 v[120:121], v83 offset:32768\n\t" \
  "s_and_b32 s64, s56, 0x4\n\t" \
  "s_cmp_eq_u32 s64, 0x4\n\t" \
  "s_cbranch_scc1 Lok2_2_%=\n\t" \
  "s_mov_b32 s62, 0x4\n\t" \
  "s_mov_b32 s69, 0x7f\n\t" \
  "s_call_b64 s[60:61], Lpoll_2_%=\n\t" \
  "Lok2_2_%=:\n\t" \
  "s_add_u32 s54, s68, 0x60000\n\t" \
  "buffer_load_dwordx4 a[192:195], v4, s[12:15], s54 offen offset:0 sc0 sc1\n\t" \
  "v_mfma_f32_16x16x128_f8f6f4 v[152:155], v[104:111], a[144:151], v[152:155]\n\t" \
  "ds_read_b64_tr_b8 v[122:123], v83 offset:33792\n\t" \
  "v_mfma_f32_16x16x128_f8f6f4 v[148:151], v[104:111], a[136:143], v[148:151]\n\t" \
  "ds_read_b64_tr_b8 v[124:125], v99 offset:0\n\t" \
  "s_add_u32 s55, s68, 0x60000\n\t" \
  "buffer_load_dwordx4 a[196:199], v4, s[12:15], s55 offen offset:1024 sc0 sc1\n\t" \
  "v_mfma_f32_16x16x128_f8f6f4 v[144:147], v[104:111], a[128:135], v[144:147]\n\t" \
  "ds_read_b64_tr_b8 v[126:127], v99 offset:1024\n\t" \
  "s_waitcnt lgkmcnt(4)\n\t" \
  "v_mfma_f32_16x16x128_f8f6f4 v[160:163], v[112:119], a[128:135], v[160:163]\n\t" \
  "ds_read_b64_tr_b8 v[104:105], v84 offset:32768\n\t" \
  "s_add_u32 s48, s68, 0x60000\n\t" \
  "buffer_load_dwordx4 a[200:203], v4, s[12:15], s48 offen offset:2048 sc0 sc1\n\t" \
  "v_mfma_f32_16x16x128_f8f6f4 v[164:167], v[112:119], a[136:143], v[164:167]\n\t" \
  "ds_read_b64_tr_b8 v[106:107], v84 offset:33792\n\t" \
  "v_mfma_f32_16x16x128_f8f6f4 v[168:171], v[112:119], a[144:151], v[168:171]\n\t" \
  "ds_read_b64_tr_b8 v[108:109], v100 offset:0\n\t" \
  "s_add_u32 s49, s68, 0x60000\n\t" \
  "buffer_load_dwordx4 a[204:207], v4, s[12:15], s49 offen offset:3072 sc0 sc1\n\t" \
  "v_mfma_f32_16x16x128_f8f6f4 v[172:175], v[112:119], a[152:159], v[172:175]\n\t" \
  "ds_read_b64_tr_b8 v[110:111], v100 offset:1024\n\t" \
  "s_waitcnt lgkmcnt(4)\n\t" \
  "v_mfma_f32_16x16x128_f8f6f4 v[188:191], v[120:127], a[152:159], v[188:191]\n\t" \
  "ds_read_b64_tr_b8 v[112:113], v85 offset:32768\n\t" \
  "s_add_u32 s50, s68, 0x61000\n\t" \
  "buffer_load_dwordx4 a[208:211], v4, s[12:15], s50 offen offset:0 sc0 sc1\n\t" \
  "v_mfma_f32_16x16x128_f8f6f4 v[184:187], v[120:127], a[144:151], v[184:187]\n\t" \
  "ds_read_b64_tr_b8 v[114:115], v85 offset:33792\n\t" \
  "v_mfma_f32_16x16x128_f8f6f4 v[180:183], v[120:127], a[136:143], v[180:183]\n\t" \
  "ds_read_b64_tr_b8 v[116:117], v101 offset:0\n\t" \
  "s_add_u32 s51, s68, 0x61000\n\t" \
  "buffer_load_dwordx4 a[212:215], v4, s[12:15], s51 offen offset:1024 sc0 sc1\n\t" \
  "v_mfma_f32_16x16x128_f8f6f4 v[176:179], v[120:127], a[128:135], v[176:179]\n\t" \
  "ds_read_b64_tr_b8 v[118:119], v101 offset:1024\n\t" \
  "s_waitcnt lgkmcnt(4)\n\t" \
  "v_mfma_f32_16x16x128_f8f6f4 v[192:195], v[104:111], a[128:135], v[192:195]\n\t" \
  "ds_read_b64_tr_b8 v[120:121], v86 offset:32768\n\t" \
  "s_add_u32 s52, s68, 0x61000\n\t" \
  "buffer_load_dwordx4 a[216:219], v4, s[12:15], s52 offen offset:2048 sc0 sc1\n\t" \
  "v_mfma_f32_16x16x128_f8f6f4 v[196:199], v[104:111], a[136:143], v[196:199]\n\t" \
  "ds_read_b64_tr_b8 v[122:123], v86 offset:33792\n\t" \
  "v_mfma_f32_16x16x128_f8f6f4 v[200:203], v[104:111], a[144:151], v[200:203]\n\t" \
  "ds_read_b64_tr_b8 v[124:125], v102 offset:0\n\t" \
  "s_add_u32 s53, s68, 0x61000\n\t" \
  "buffer_load_dwordx4 a[220:223], v4, s[12:15], s53 offen offset:3072 sc0 sc1\n\t" \
  "v_mfma_f32_16x16x128_f8f6f4 v[204:207], v[104:111], a[152:159], v[204:207]\n\t" \
  "ds_read_b64_tr_b8 v[126:127], v102 offset:1024\n\t" \
  "s_waitcnt lgkmcnt(4)\n\t" \
  "v_mfma_f32_16x16x128_f8f6f4 v[220:223], v[112:119], a[152:159], v[220:223]\n\t" \
  "ds_read_b64_tr_b8 v[104:105], v87 offset:32768\n\t" \
  "s_add_u32 m0, s24, 0x26000\n\t" \
  "s_add_u32 s54, s43, 0x0\n\t" \
  "buffer_load_dwordx4 v3, s[8:11], s54 offen lds\n\t" \
  "v_mfma_f32_16x16x128_f8f6f4 v[216:219], v[112:119], a[144:151], v[216:219]\n\t" \
  "ds_read_b64_tr_b8 v[106:107], v87 offset:33792\n\t" \
  "v_mfma_f32_16x16x128_f8f6f4 v[212:215], v[112:119], a[136:143], v[212:215]\n\t" \
  "ds_read_b64_tr_b8 v[108:109], v103 offset:0\n\t" \
  "s_add_u32 m0, s24, 0x26400\n\t" \
  "s_add_u32 s55, s43, 0x20000\n\t" \
  "buffer_load_dwordx4 v3, s[8:11], s55 offen lds\n\t" \
  "v_mfma_f32_16x16x128_f8f6f4 v[208:211], v[112:119], a[128:135], v[208:211]\n\t" \
  "ds_read_b64_tr_b8 v[110:111], v103 offset:1024\n\t" \
  "s_waitcnt lgkmcnt(0)\n\t" \
  "s_barrier\n\t" \
  "v_mfma_f32_16x16x128_f8f6f4 v[224:227], v[120:127], a[128:135], v[224:227]\n\t" \
  "ds_read_b64_tr_b8 v[112:113], v80 offset:40960\n\t" \
  "s_and_b32 s64, s56, 0x8\n\t" \
  "s_cmp_eq_u32 s64, 0x8\n\t" \
  "s_cbranch_scc1 Lok3_2_%=\n\t" \
  "s_mov_b32 s62, 0x8\n\t" \
  "s_mov_b32 s69, 0x7f\n\t" \
  "s_call_b64 s[60:61], Lpoll_2_%=\n\t" \
  "Lok3_2_%=:\n\t" \
  "s_add_u32 s48, s68, 0x70000\n\t" \
  "buffer_load_dwordx4 a[224:227], v4, s[12:15], s48 offen offset:0 sc0 sc1\n\t" \
  "v_mfma_f32_16x16x128_f8f6f4 v[228:231], v[120:127], a[136:143], v[228:231]\n\t" \
  "ds_read_b64_tr_b8 v[114:115], v80 offset:41984\n\t" \
  "v_mfma_f32_16x16x128_f8f6f4 v[232:235], v[120:127], a[144:151], v[232:235]\n\t" \
  "ds_read_b64_tr_b8 v[116:117], v96 offset:8192\n\t" \
  "s_add_u32 s49, s68, 0x70000\n\t" \
  "buffer_load_dwordx4 a[228:231], v4, s[12:15], s49 offen offset:1024 sc0 sc1\n\t" \
  "v_mfma_f32_16x16x128_f8f6f4 v[236:239], v[120:127], a[152:159], v[236:239]\n\t" \
  "ds_read_b64_tr_b8 v[118:119], v96 offset:9216\n\t" \
  "v_mfma_f32_16x16x128_f8f6f4 v[252:255], v[104:111], a[152:159], v[252:255]\n\t" \
  "ds_read_b64_tr_b8 v[120:121], v81 offset:40960\n\t" \
  "s_add_u32 s50, s68, 0x70000\n\t" \
  "buffer_load_dwordx4 a[232:235], v4, s[12:15], s50 offen offset:2048 sc0 sc1\n\t" \
  "v_mfma_f32_16x16x128_f8f6f4 v[248:251], v[104:111], a[144:151], v[248:251]\n\t" \
  "ds_read_b64_tr_b8 v[122:123], v81 offset:41984\n\t" \
  "v_mfma_f32_16x16x128_f8f6f4 v[244:247], v[104:111], a[136:143], v[244:247]\n\t" \
  "ds_read_b64_tr_b8 v[124:125], v97 offset:8192\n\t" \
  "s_add_u32 s51, s68, 0x70000\n\t" \
  "buffer_load_dwordx4 a[236:239], v4, s[12:15], s51 offen offset:3072 sc0 sc1\n\t" \
  "v_mfma_f32_16x16x128_f8f6f4 v[240:243], v[104:111], a[128:135], v[240:243]\n\t" \
  "ds_read_b64_tr_b8 v[126:127], v97 offset:9216\n\t" \
  "s_waitcnt vmcnt(16) lgkmcnt(4)\n\t" \
  "v_mfma_f32_16x16x128_f8f6f4 v[128:131], v[112:119], a[160:167], v[128:131]\n\t" \
  "ds_read_b64_tr_b8 v[104:105], v82 offset:40960\n\t" \
  "s_add_u32 s52, s68, 0x71000\n\t" \
  "buffer_load_dwordx4 a[240:243], v4, s[12:15], s52 offen offset:0 sc0 sc1\n\t" \
  "v_mfma_f32_16x16x128_f8f6f4 v[132:135], v[112:119], a[168:175], v[132:135]\n\t" \
  "ds_read_b64_tr_b8 v[106:107], v82 offset:41984\n\t" \
  "v_mfma_f32_16x16x128_f8f6f4 v[136:139], v[112:119], a[176:183], v[136:139]\n\t" \
  "ds_read_b64_tr_b8 v[108:109], v98 offset:8192\n\t" \
  "s_add_u32 s53, s68, 0x71000\n\t" \
  "buffer_load_dwordx4 a[244:247], v4, s[12:15], s53 offen offset:1024 sc0 sc1\n\t" \
  "v_mfma_f32_16x16x128_f8f6f4 v[140:143], v[112:119], a[184:191], v[140:143]\n\t" \
  "ds_read_b64_tr_b8 v[110:111], v98 offset:9216\n\t" \
  "s_waitcnt lgkmcnt(4)\n\t" \
  "v_mfma_f32_16x16x128_f8f6f4 v[156:159], v[120:127], a[184:191], v[156:159]\n\t" \
  "ds_read_b64_tr_b8 v[112:113], v83 offset:40960\n\t" \
  "s_add_u32 s54, s68, 0x71000\n\t" \
  "buffer_load_dwordx4 a[248:251], v4, s[12:15], s54 offen offset:2048 sc0 sc1\n\t" \
  "v_mfma_f32_16x16x128_f8f6f4 v[152:155], v[120:127], a[176:183], v[152:155]\n\t" \
  "ds_read_b64_tr_b8 v[114:115], v83 offset:41984\n\t" \
  "v_mfma_f32_16x16x128_f8f6f4 v[148:151], v[120:127], a[168:175], v[148:151]\n\t" \
  "ds_read_b64_tr_b8 v[116:117], v99 offset:8192\n\t" \
  "s_add_u32 s55, s68, 0x71000\n\t" \
  "buffer_load_dwordx4 a[252:255], v4, s[12:15], s55 offen offset:3072 sc0 sc1\n\t" \
  "v_mfma_f32_16x16x128_f8f6f4 v[144:147], v[120:127], a[160:167], v[144:147]\n\t" \
  "ds_read_b64_tr_b8 v[118:119], v99 offset:9216\n\t" \
  "s_waitcnt lgkmcnt(4)\n\t" \
  "v_mfma_f32_16x16x128_f8f6f4 v[160:163], v[104:111], a[160:167], v[160:163]\n\t" \
  "ds_read_b64_tr_b8 v[120:121], v84 offset:40960\n\t" \
  "s_cmp_lg_u32 s26, 1\n\t" \
  "s_cbranch_scc1 Lnf1_2_%=\n\t" \
  "s_mov_b64 exec, 1\n\t" \
  "global_store_dword v65, v64, s[30:31] sc0 sc1\n\t" \
  "s_mov_b64 exec, -1\n\t" \
  "s_branch Lmf1_2_%=\n\t" \
  "Lnf1_2_%=:\n\t" \
  "buffer_load_dword v62, v65, s[20:23], s22 offen\n\t" \
  "Lmf1_2_%=:\n\t" \
  "s_cmp_lg_u32 s26, 2\n\t" \
  "s_cbranch_scc1 Lns2_2_%=\n\t" \
  "global_store_dwordx4 v63, v[72:75], s[28:29] sc0 sc1\n\t" \
  "global_store_dwordx4 v63, v[76:79], s[28:29] offset:1024 sc0 sc1\n\t" \
  "s_branch Lms2_2_%=\n\t" \
  "Lns2_2_%=:\n\t" \
  "buffer_load_dword v62, v65, s[20:23], s22 offen\n\t" \
  "buffer_load_dword v62, v65, s[20:23], s22 offen\n\t" \
  "Lms2_2_%=:\n\t" \
  "s_add_u32 m0, s24, 0x10000\n\t" \
  "s_add_u32 s48, s44, 0x0\n\t" \
  "buffer_load_dwordx4 v2, s[8:11], s48 offen lds\n\t" \
  "v_mfma_f32_16x16x128_f8f6f4 v[164:167], v[104:111], a[168:175], v[164:167]\n\t" \
  "ds_read_b64_tr_b8 v[122:123], v84 offset:41984\n\t" \
  "v_mfma_f32_16x16x128_f8f6f4 v[168:171], v[104:111], a[176:183], v[168:171]\n\t" \
  "ds_read_b64_tr_b8 v[124:125], v100 offset:8192\n\t" \
  "s_add_u32 m0, s24, 0x10400\n\t" \
  "s_add_u32 s49, s44, 0x20000\n\t" \
  "buffer_load_dwordx4 v2, s[8:11], s49 offen lds\n\t" \
  "v_mfma_f32_16x16x128_f8f6f4 v[172:175], v[104:111], a[184:191], v[172:175]\n\t" \
  "ds_read_b64_tr_b8 v[126:127], v100 offset:9216\n\t" \
  "s_waitcnt lgkmcnt(4)\n\t" \
  "v_mfma_f32_16x16x128_f8f6f4 v[188:191], v[112:119], a[184:191], v[188:191]\n\t" \
  "ds_read_b64_tr_b8 v[104:105], v85 offset:40960\n\t" \
  "s_add_u32 m0, s24, 0x12000\n\t" \
  "s_add_u32 s50, s45, 0x0\n\t" \
  "buffer_load_dwordx4 v2, s[8:11], s50 offen lds\n\t" \
  "v_mfma_f32_16x16x128_f8f6f4 v[184:187], v[112:119], a[176:183], v[184:187]\n\t" \
  "ds_read_b64_tr_b8 v[106:107], v85 offset:41984\n\t" \
  "v_mfma_f32_16x16x128_f8f6f4 v[180:183], v[112:119], a[168:175], v[180:183]\n\t" \
  "ds_read_b64_tr_b8 v[108:109], v101 offset:8192\n\t" \
  "s_add_u32 m0, s24, 0x12400\n\t" \
  "s_add_u32 s51, s45, 0x20000\n\t" \
  "buffer_load_dwordx4 v2, s[8:11], s51 offen lds\n\t" \
  "v_mfma_f32_16x16x128_f8f6f4 v[176:179], v[112:119], a[160:167], v[176:179]\n\t" \
  "ds_read_b64_tr_b8 v[110:111], v101 offset:9216\n\t" \
  "s_waitcnt lgkmcnt(4)\n\t" \
  "v_mfma_f32_16x16x128_f8f6f4 v[192:195], v[120:127], a[160:167], v[192:195]\n\t" \
  "ds_read_b64_tr_b8 v[112:113], v86 offset:40960\n\t" \
  "s_add_u32 m0, s24, 0x14000\n\t" \
  "s_add_u32 s52, s46, 0x0\n\t" \
  "buffer_load_dwordx4 v2, s[8:11], s52 offen lds\n\t" \
  "v_mfma_f32_16x16x128_f8f6f4 v[196:199], v[120:127], a[168:175], v[196:199]\n\t" \
  "ds_read_b64_tr_b8 v[114:115], v86 offset:41984\n\t" \
  "v_mfma_f32_16x16x128_f8f6f4 v[200:203], v[120:127], a[176:183], v[200:203]\n\t" \
  "ds_read_b64_tr_b8 v[116:117], v102 offset:8192\n\t" \
  "s_add_u32 m0, s24, 0x14400\n\t" \
  "s_add_u32 s53, s46, 0x20000\n\t" \
  "buffer_load_dwordx4 v2, s[8:11], s53 offen lds\n\t" \
  "v_mfma_f32_16x16x128_f8f6f4 v[204:207], v[120:127], a[184:191], v[204:207]\n\t" \
  "ds_read_b64_tr_b8 v[118:119], v102 offset:9216\n\t" \
  "s_waitcnt lgkmcnt(4)\n\t" \
  "v_mfma_f32_16x16x128_f8f6f4 v[220:223], v[104:111], a[184:191], v[220:223]\n\t" \
  "ds_read_b64_tr_b8 v[120:121], v87 offset:40960\n\t" \
  "s_add_u32 m0, s24, 0x16000\n\t" \
  "s_add_u32 s54, s40, 0x200000\n\t" \
  "buffer_load_dwordx4 v2, s[8:11], s54 offen lds\n\t" \
  "v_mfma_f32_16x16x128_f8f6f4 v[216:219], v[104:111], a[176:183], v[216:219]\n\t" \
  "ds_read_b64_tr_b8 v[122:123], v87 offset:41984\n\t" \
  "v_mfma_f32_16x16x128_f8f6f4 v[212:215], v[104:111], a[168:175], v[212:215]\n\t" \
  "ds_read_b64_tr_b8 v[124:125], v103 offset:8192\n\t" \
  "s_add_u32 m0, s24, 0x16400\n\t" \
  "s_add_u32 s55, s40, 0x220000\n\t" \
  "buffer_load_dwordx4 v2, s[8:11], s55 offen lds\n\t" \
  "v_mfma_f32_16x16x128_f8f6f4 v[208:211], v[104:111], a[160:167], v[208:211]\n\t" \
  "ds_read_b64_tr_b8 v[126:127], v103 offset:9216\n\t" \
  "s_waitcnt vmcnt(29) lgkmcnt(0)\n\t" \
  "s_barrier\n\t" \
  "v_mfma_f32_16x16x128_f8f6f4 v[224:227], v[112:119], a[160:167], v[224:227]\n\t" \
  "ds_read_b64_tr_b8 v[104:105], v80 offset:49152\n\t" \
  "v_mfma_f32_16x16x128_f8f6f4 v[228:231], v[112:119], a[168:175], v[228:231]\n\t" \
  "ds_read_b64_tr_b8 v[106:107], v80 offset:50176\n\t" \
  "v_mfma_f32_16x16x128_f8f6f4 v[232:235], v[112:119], a[176:183], v[232:235]\n\t" \
  "ds_read_b64_tr_b8 v[108:109], v96 offset:16384\n\t" \
  "v_mfma_f32_16x16x128_f8f6f4 v[236:239], v[112:119], a[184:191], v[236:239]\n\t" \
  "ds_read_b64_tr_b8 v[110:111], v96 offset:17408\n\t" \
  "v_mfma_f32_16x16x128_f8f6f4 v[252:255], v[120:127], a[184:191], v[252:255]\n\t" \
  "ds_read_b64_tr_b8 v[112:113], v81 offset:49152\n\t" \
  "v_mfma_f32_16x16x128_f8f6f4 v[248:251], v[120:127], a[176:183], v[248:251]\n\t" \
  "ds_read_b64_tr_b8 v[114:115], v81 offset:50176\n\t" \
  "v_mfma_f32_16x16x128_f8f6f4 v[244:247], v[120:127], a[168:175], v[244:247]\n\t" \
  "ds_read_b64_tr_b8 v[116:117], v97 offset:16384\n\t" \
  "v_mfma_f32_16x16x128_f8f6f4 v[240:243], v[120:127], a[160:167], v[240:243]\n\t" \
  "ds_read_b64_tr_b8 v[118:119], v97 offset:17408\n\t" \
  "s_waitcnt vmcnt(21) lgkmcnt(4)\n\t" \
  "v_mfma_f32_16x16x128_f8f6f4 v[128:131], v[104:111], a[192:199], v[128:131]\n\t" \
  "ds_read_b64_tr_b8 v[120:121], v82 offset:49152\n\t" \
  "v_mfma_f32_16x16x128_f8f6f4 v[132:135], v[104:111], a[200:207], v[132:135]\n\t" \
  "ds_read_b64_tr_b8 v[122:123], v82 offset:50176\n\t" \
  "v_mfma_f32_16x16x128_f8f6f4 v[136:139], v[104:111], a[208:215], v[136:139]\n\t" \
  "ds_read_b64_tr_b8 v[124:125], v98 offset:16384\n\t" \
  "v_mfma_f32_16x16x128_f8f6f4 v[140:143], v[104:111], a[216:223], v[140:143]\n\t" \
  "ds_read_b64_tr_b8 v[126:127], v98 offset:17408\n\t" \
  "s_waitcnt lgkmcnt(4)\n\t" \
  "v_mfma_f32_16x16x128_f8f6f4 v[156:159], v[112:119], a[216:223], v[156:159]\n\t" \
  "ds_read_b64_tr_b8 v[104:105], v83 offset:49152\n\t" \
  "v_mfma_f32_16x16x128_f8f6f4 v[152:155], v[112:119], a[208:215], v[152:155]\n\t" \
  "ds_read_b64_tr_b8 v[106:107], v83 offset:50176\n\t" \
  "v_mfma_f32_16x16x128_f8f6f4 v[148:151], v[112:119], a[200:207], v[148:151]\n\t" \
  "ds_read_b64_tr_b8 v[108:109], v99 offset:16384\n\t" \
  "v_mfma_f32_16x16x128_f8f6f4 v[144:147], v[112:119], a[192:199], v[144:147]\n\t" \
  "ds_read_b64_tr_b8 v[110:111], v99 offset:17408\n\t" \
  "s_waitcnt lgkmcnt(4)\n\t" \
  "v_mfma_f32_16x16x128_f8f6f4 v[160:163], v[120:127], a[192:199], v[160:163]\n\t" \
  "ds_read_b64_tr_b8 v[112:113], v84 offset:49152\n\t" \
  "v_mfma_f32_16x16x128_f8f6f4 v[164:167], v[120:127], a[200:207], v[164:167]\n\t" \
  "ds_read_b64_tr_b8 v[114:115], v84 offset:50176\n\t" \
  "v_mfma_f32_16x16x128_f8f6f4 v[168:171], v[120:127], a[208:215], v[168:171]\n\t" \
  "ds_read_b64_tr_b8 v[116:117], v100 offset:16384\n\t" \
  "v_mfma_f32_16x16x128_f8f6f4 v[172:175], v[120:127], a[216:223], v[172:175]\n\t" \
  "ds_read_b64_tr_b8 v[118:119], v100 offset:17408\n\t" \
  "s_waitcnt lgkmcnt(4)\n\t" \
  "v_mfma_f32_16x16x128_f8f6f4 v[188:191], v[104:111], a[216:223], v[188:191]\n\t" \
  "ds_read_b64_tr_b8 v[120:121], v85 offset:49152\n\t" \
  "s_add_u32 m0, s24, 0x18000\n\t" \
  "s_add_u32 s48, s44, 0x0\n\t" \
  "buffer_load_dwordx4 v3, s[8:11], s48 offen lds\n\t" \
  "v_mfma_f32_16x16x128_f8f6f4 v[184:187], v[104:111], a[208:215], v[184:187]\n\t" \
  "ds_read_b64_tr_b8 v[122:123], v85 offset:50176\n\t" \
  "v_mfma_f32_16x16x128_f8f6f4 v[180:183], v[104:111], a[200:207], v[180:183]\n\t" \
  "ds_read_b64_tr_b8 v[124:125], v101 offset:16384\n\t" \
  "s_add_u32 m0, s24, 0x18400\n\t" \
  "s_add_u32 s49, s44, 0x20000\n\t" \
  "buffer_load_dwordx4 v3, s[8:11], s49 offen lds\n\t" \
  "v_mfma_f32_16x16x128_f8f6f4 v[176:179], v[104:111], a[192:199], v[176:179]\n\t" \
  "ds_read_b64_tr_b8 v[126:127], v101 offset:17408\n\t" \
  "s_waitcnt lgkmcnt(4)\n\t" \
  "v_mfma_f32_16x16x128_f8f6f4 v[192:195], v[112:119], a[192:199], v[192:195]\n\t" \
  "ds_read_b64_tr_b8 v[104:105], v86 offset:49152\n\t" \
  "s_and_b32 s64, s56, 0x10\n\t" \
  "s_cmp_eq_u32 s64, 0x10\n\t" \
  "s_cbranch_scc1 Lok4_2_%=\n\t" \
  "s_mov_b32 s62, 0x10\n\t" \
  "s_mov_b32 s69, 0x7f\n\t" \
  "s_call_b64 s[60:61], Lpoll_2_%=\n\t" \
  "Lok4_2_%=:\n\t" \
  "s_add_u32 s50, s68, 0x80000\n\t" \
  "buffer_load_dwordx4 a[0:3], v4, s[12:15], s50 offen offset:0 sc0 sc1\n\t" \
  "v_mfma_f32_16x16x128_f8f6f4 v[196:199], v[112:119], a[200:207], v[196:199]\n\t" \
  "ds_read_b64_tr_b8 v[106:107], v86 offset:50176\n\t" \
  "v_mfma_f32_16x16x128_f8f6f4 v[200:203], v[112:119], a[208:215], v[200:203]\n\t" \
  "ds_read_b64_tr_b8 v[108:109], v102 offset:16384\n\t" \
  "s_add_u32 s51, s68, 0x80000\n\t" \
  "buffer_load_dwordx4 a[4:7], v4, s[12:15], s51 offen offset:1024 sc0 sc1\n\t" \
  "v_mfma_f32_16x16x128_f8f6f4 v[204:207], v[112:119], a[216:223], v[204:207]\n\t" \
  "ds_read_b64_tr_b8 v[110:111], v102 offset:17408\n\t" \
  "s_waitcnt lgkmcnt(4)\n\t" \
  "v_mfma_f32_16x16x128_f8f6f4 v[220:223], v[120:127], a[216:223], v[220:223]\n\t" \
  "ds_read_b64_tr_b8 v[112:113], v87 offset:49152\n\t" \
  "s_add_u32 s52, s68, 0x80000\n\t" \
  "buffer_load_dwordx4 a[8:11], v4, s[12:15], s52 offen offset:2048 sc0 sc1\n\t" \
  "v_mfma_f32_16x16x128_f8f6f4 v[216:219], v[120:127], a[208:215], v[216:219]\n\t" \
  "ds_read_b64_tr_b8 v[114:115], v87 offset:50176\n\t" \
  "v_mfma_f32_16x16x128_f8f6f4 v[212:215], v[120:127], a[200:207], v[212:215]\n\t" \
  "ds_read_b64_tr_b8 v[116:117], v103 offset:16384\n\t" \
  "s_add_u32 s53, s68, 0x80000\n\t" \
  "buffer_load_dwordx4 a[12:15], v4, s[12:15], s53 offen offset:3072 sc0 sc1\n\t" \
  "v_mfma_f32_16x16x128_f8f6f4 v[208:211], v[120:127], a[192:199], v[208:211]\n\t" \
  "ds_read_b64_tr_b8 v[118:119], v103 offset:17408\n\t" \
  "s_waitcnt vmcnt(25) lgkmcnt(0)\n\t" \
  "s_barrier\n\t" \
  "v_mfma_f32_16x16x128_f8f6f4 v[224:227], v[104:111], a[192:199], v[224:227]\n\t" \
  "ds_read_b64_tr_b8 v[120:121], v80 offset:57344\n\t" \
  "s_add_u32 s54, s68, 0x81000\n\t" \
  "buffer_load_dwordx4 a[16:19], v4, s[12:15], s54 offen offset:0 sc0 sc1\n\t" \
  "v_mfma_f32_16x16x128_f8f6f4 v[228:231], v[104:111], a[200:207], v[228:231]\n\t" \
  "ds_read_b64_tr_b8 v[122:123], v80 offset:58368\n\t" \
  "v_mfma_f32_16x16x128_f8f6f4 v[232:235], v[104:111], a[208:215], v[232:235]\n\t" \
  "ds_read_b64_tr_b8 v[124:125], v96 offset:24576\n\t" \
  "s_add_u32 s55, s68, 0x81000\n\t" \
  "buffer_load_dwordx4 a[20:23], v4, s[12:15], s55 offen offset:1024 sc0 sc1\n\t" \
  "v_mfma_f32_16x16x128_f8f6f4 v[236:239], v[104:111], a[216:223], v[236:239]\n\t" \
  "ds_read_b64_tr_b8 v[126:127], v96 offset:25600\n\t" \
  "v_mfma_f32_16x16x128_f8f6f4 v[252:255], v[112:119], a[216:223], v[252:255]\n\t" \
  "ds_read_b64_tr_b8 v[104:105], v81 offset:57344\n\t" \
  "s_add_u32 s48, s68, 0x81000\n\t" \
  "buffer_load_dwordx4 a[24:27], v4, s[12:15], s48 offen offset:2048 sc0 sc1\n\t" \
  "v_mfma_f32_16x16x128_f8f6f4 v[248:251], v[112:119], a[208:215], v[248:251]\n\t" \
  "ds_read_b64_tr_b8 v[106:107], v81 offset:58368\n\t" \
  "v_mfma_f32_16x16x128_f8f6f4 v[244:247], v[112:119], a[200:207], v[244:247]\n\t" \
  "ds_read_b64_tr_b8 v[108:109], v97 offset:24576\n\t" \
  "s_add_u32 s49, s68, 0x81000\n\t" \
  "buffer_load_dwordx4 a[28:31], v4, s[12:15], s49 offen offset:3072 sc0 sc1\n\t" \
  "v_mfma_f32_16x16x128_f8f6f4 v[240:243], v[112:119], a[192:199], v[240:243]\n\t" \
  "ds_read_b64_tr_b8 v[110:111], v97 offset:25600\n\t" \
  "s_waitcnt vmcnt(21) lgkmcnt(4)\n\t" \
  "v_mfma_f32_16x16x128_f8f6f4 v[128:131], v[120:127], a[224:231], v[128:131]\n\t" \
  "ds_read_b64_tr_b8 v[112:113], v82 offset:57344\n\t" \
  "s_add_u32 m0, s24, 0x1a000\n\t" \
  "s_add_u32 s50, s45, 0x0\n\t" \
  "buffer_load_dwordx4 v3, s[8:11], s50 offen lds\n\t" \
  "v_mfma_f32_16x16x128_f8f6f4 v[132:135], v[120:127], a[232:239], v[132:135]\n\t" \
  "ds_read_b64_tr_b8 v[114:115], v82 offset:58368\n\t" \
  "v_mfma_f32_16x16x128_f8f6f4 v[136:139], v[120:127], a[240:247], v[136:139]\n\t" \
  "ds_read_b64_tr_b8 v[116:117], v98 offset:24576\n\t" \
  "s_add_u32 m0, s24, 0x1a400\n\t" \
  "s_add_u32 s51, s45, 0x20000\n\t" \
  "buffer_load_dwordx4 v3, s[8:11], s51 offen lds\n\t" \
  "v_mfma_f32_16x16x128_f8f6f4 v[140:143], v[120:127], a[248:255], v[140:143]\n\t" \
  "ds_read_b64_tr_b8 v[118:119], v98 offset:25600\n\t" \
  "s_waitcnt lgkmcnt(4)\n\t" \
  "v_mfma_f32_16x16x128_f8f6f4 v[156:159], v[104:111], a[248:255], v[156:159]\n\t" \
  "ds_read_b64_tr_b8 v[120:121], v83 offset:57344\n\t" \
  "s_and_b32 s64, s56, 0x20\n\t" \
  "s_cmp_eq_u32 s64, 0x20\n\t" \
  "s_cbranch_scc1 Lok5_2_%=\n\t" \
  "s_mov_b32 s62, 0x20\n\t" \
  "s_mov_b32 s69, 0x7f\n\t" \
  "s_call_b64 s[60:61], Lpoll_2_%=\n\t" \
  "Lok5_2_%=:\n\t" \
  "s_add_u32 s52, s68, 0x90000\n\t" \
  "buffer_load_dwordx4 a[32:35], v4, s[12:15], s52 offen offset:0 sc0 sc1\n\t" \
  "v_mfma_f32_16x16x128_f8f6f4 v[152:155], v[104:111], a[240:247], v[152:155]\n\t" \
  "ds_read_b64_tr_b8 v[122:123], v83 offset:58368\n\t" \
  "v_mfma_f32_16x16x128_f8f6f4 v[148:151], v[104:111], a[232:239], v[148:151]\n\t" \
  "ds_read_b64_tr_b8 v[124:125], v99 offset:24576\n\t" \
  "s_add_u32 s53, s68, 0x90000\n\t" \
  "buffer_load_dwordx4 a[36:39], v4, s[12:15], s53 offen offset:1024 sc0 sc1\n\t" \
  "v_mfma_f32_16x16x128_f8f6f4 v[144:147], v[104:111], a[224:231], v[144:147]\n\t" \
  "ds_read_b64_tr_b8 v[126:127], v99 offset:25600\n\t" \
  "s_waitcnt lgkmcnt(4)\n\t" \
  "v_mfma_f32_16x16x128_f8f6f4 v[160:163], v[112:119], a[224:231], v[160:163]\n\t" \
  "ds_read_b64_tr_b8 v[104:105], v84 offset:57344\n\t" \
  "s_add_u32 s54, s68, 0x90000\n\t" \
  "buffer_load_dwordx4 a[40:43], v4, s[12:15], s54 offen offset:2048 sc0 sc1\n\t" \
  "v_mfma_f32_16x16x128_f8f6f4 v[164:167], v[112:119], a[232:239], v[164:167]\n\t" \
  "ds_read_b64_tr_b8 v[106:107], v84 offset:58368\n\t" \
  "v_mfma_f32_16x16x128_f8f6f4 v[168:171], v[112:119], a[240:247], v[168:171]\n\t" \
  "ds_read_b64_tr_b8 v[108:109], v100 offset:24576\n\t" \
  "s_add_u32 s55, s68, 0x90000\n\t" \
  "buffer_load_dwordx4 a[44:47], v4, s[12:15], s55 offen offset:3072 sc0 sc1\n\t" \
  "v_mfma_f32_16x16x128_f8f6f4 v[172:175], v[112:119], a[248:255], v[172:175]\n\t" \
  "ds_read_b64_tr_b8 v[110:111], v100 offset:25600\n\t" \
  "s_waitcnt lgkmcnt(4)\n\t" \
  "v_mfma_f32_16x16x128_f8f6f4 v[188:191], v[120:127], a[248:255], v[188:191]\n\t" \
  "ds_read_b64_tr_b8 v[112:113], v85 offset:57344\n\t" \
  "s_add_u32 s48, s68, 0x91000\n\t" \
  "buffer_load_dwordx4 a[48:51], v4, s[12:15], s48 offen offset:0 sc0 sc1\n\t" \
  "v_mfma_f32_16x16x128_f8f6f4 v[184:187], v[120:127], a[240:247], v[184:187]\n\t" \
  "ds_read_b64_tr_b8 v[114:115], v85 offset:58368\n\t" \
  "v_mfma_f32_16x16x128_f8f6f4 v[180:183], v[120:127], a[232:239], v[180:183]\n\t" \
  "ds_read_b64_tr_b8 v[116:117], v101 offset:24576\n\t" \
  "s_add_u32 s49, s68, 0x91000\n\t" \
  "buffer_load_dwordx4 a[52:55], v4, s[12:15], s49 offen offset:1024 sc0 sc1\n\t" \
  "v_mfma_f32_16x16x128_f8f6f4 v[176:179], v[120:127], a[224:231], v[176:179]\n\t" \
  "ds_read_b64_tr_b8 v[118:119], v101 offset:25600\n\t" \
  "s_waitcnt lgkmcnt(4)\n\t" \
  "v_mfma_f32_16x16x128_f8f6f4 v[192:195], v[104:111], a[224:231], v[192:195]\n\t" \
  "ds_read_b64_tr_b8 v[120:121], v86 offset:57344\n\t" \
  "s_add_u32 s50, s68, 0x91000\n\t" \
  "buffer_load_dwordx4 a[56:59], v4, s[12:15], s50 offen offset:2048 sc0 sc1\n\t" \
  "v_mfma_f32_16x16x128_f8f6f4 v[196:199], v[104:111], a[232:239], v[196:199]\n\t" \
  "ds_read_b64_tr_b8 v[122:123], v86 offset:58368\n\t" \
  "v_mfma_f32_16x16x128_f8f6f4 v[200:203], v[104:111], a[240:247], v[200:203]\n\t" \
  "ds_read_b64_tr_b8 v[124:125], v102 offset:24576\n\t" \
  "s_add_u32 s51, s68, 0x91000\n\t" \
  "buffer_load_dwordx4 a[60:63], v4, s[12:15], s51 offen offset:3072 sc0 sc1\n\t" \
  "v_mfma_f32_16x16x128_f8f6f4 v[204:207], v[104:111], a[248:255], v[204:207]\n\t" \
  "ds_read_b64_tr_b8 v[126:127], v102 offset:25600\n\t" \
  "s_waitcnt lgkmcnt(4)\n\t" \
  "v_mfma_f32_16x16x128_f8f6f4 v[220:223], v[112:119], a[248:255], v[220:223]\n\t" \
  "ds_read_b64_tr_b8 v[104:105], v87 offset:57344\n\t" \
  "s_add_u32 m0, s24, 0x1c000\n\t" \
  "s_add_u32 s52, s46, 0x0\n\t" \
  "buffer_load_dwordx4 v3, s[8:11], s52 offen lds\n\t" \
  "v_mfma_f32_16x16x128_f8f6f4 v[216:219], v[112:119], a[240:247], v[216:219]\n\t" \
  "ds_read_b64_tr_b8 v[106:107], v87 offset:58368\n\t" \
  "v_mfma_f32_16x16x128_f8f6f4 v[212:215], v[112:119], a[232:239], v[212:215]\n\t" \
  "ds_read_b64_tr_b8 v[108:109], v103 offset:24576\n\t" \
  "s_add_u32 m0, s24, 0x1c400\n\t" \
  "s_add_u32 s53, s46, 0x20000\n\t" \
  "buffer_load_dwordx4 v3, s[8:11], s53 offen lds\n\t" \
  "v_mfma_f32_16x16x128_f8f6f4 v[208:211], v[112:119], a[224:231], v[208:211]\n\t" \
  "ds_read_b64_tr_b8 v[110:111], v103 offset:25600\n\t" \
  "s_waitcnt vmcnt(20) lgkmcnt(0)\n\t" \
  "s_barrier\n\t" \
  "v_mfma_f32_16x16x128_f8f6f4 v[224:227], v[120:127], a[224:231], v[224:227]\n\t" \
  "ds_read_b64_tr_b8 v[112:113], v88 offset:0\n\t" \
  "s_and_b32 s64, s56, 0x40\n\t" \
  "s_cmp_eq_u32 s64, 0x40\n\t" \
  "s_cbranch_scc1 Lok6_2_%=\n\t" \
  "s_mov_b32 s62, 0x40\n\t" \
  "s_mov_b32 s69, 0x7f\n\t" \
  "s_call_b64 s[60:61], Lpoll_2_%=\n\t" \
  "Lok6_2_%=:\n\t" \
  "s_add_u32 s54, s68, 0xa0000\n\t" \
  "buffer_load_dwordx4 a[64:67], v4, s[12:15], s54 offen offset:0 sc0 sc1\n\t" \
  "v_mfma_f32_16x16x128_f8f6f4 v[228:231], v[120:127], a[232:239], v[228:231]\n\t" \
  "ds_read_b64_tr_b8 v[114:115], v88 offset:1024\n\t" \
  "v_mfma_f32_16x16x128_f8f6f4 v[232:235], v[120:127], a[240:247], v[232:235]\n\t" \
  "ds_read_b64_tr_b8 v[116:117], v88 offset:32768\n\t" \
  "s_add_u32 s55, s68, 0xa0000\n\t" \
  "buffer_load_dwordx4 a[68:71], v4, s[12:15], s55 offen offset:1024 sc0 sc1\n\t" \
  "v_mfma_f32_16x16x128_f8f6f4 v[236:239], v[120:127], a[248:255], v[236:239]\n\t" \
  "ds_read_b64_tr_b8 v[118:119], v88 offset:33792\n\t" \
  "v_mfma_f32_16x16x128_f8f6f4 v[252:255], v[104:111], a[248:255], v[252:255]\n\t" \
  "ds_read_b64_tr_b8 v[120:121], v89 offset:0\n\t" \
  "s_add_u32 s48, s68, 0xa0000\n\t" \
  "buffer_load_dwordx4 a[72:75], v4, s[12:15], s48 offen offset:2048 sc0 sc1\n\t" \
  "v_mfma_f32_16x16x128_f8f6f4 v[248:251], v[104:111], a[240:247], v[248:251]\n\t" \
  "ds_read_b64_tr_b8 v[122:123], v89 offset:1024\n\t" \
  "v_mfma_f32_16x16x128_f8f6f4 v[244:247], v[104:111], a[232:239], v[244:247]\n\t" \
  "ds_read_b64_tr_b8 v[124:125], v89 offset:32768\n\t" \
  "s_add_u32 s49, s68, 0xa0000\n\t" \
  "buffer_load_dwordx4 a[76:79], v4, s[12:15], s49 offen offset:3072 sc0 sc1\n\t" \
  "v_mfma_f32_16x16x128_f8f6f4 v[240:243], v[104:111], a[224:231], v[240:243]\n\t" \
  "ds_read_b64_tr_b8 v[126:127], v89 offset:33792\n\t" \
  "s_waitcnt vmcnt(16) lgkmcnt(4)\n\t" \
  "v_mfma_f32_16x16x128_f8f6f4 v[128:131], v[112:119], a[0:7], v[128:131]\n\t" \
  "ds_read_b64_tr_b8 v[104:105], v90 offset:0\n\t" \
  "s_add_u32 s50, s68, 0xa1000\n\t" \
  "buffer_load_dwordx4 a[80:83], v4, s[12:15], s50 offen offset:0 sc0 sc1\n\t" \
  "s_mov_b32 exec_lo, 0x7f00\n\t" \
  "s_mov_b32 exec_hi, 0\n\t" \
  "global_load_dwordx4 v[68:71], v10, s[6:7] sc0 sc1\n\t" \
  "s_mov_b64 exec, -1\n\t" \
  "v_mfma_f32_16x16x128_f8f6f4 v[132:135], v[112:119], a[8:15], v[132:135]\n\t" \
  "ds_read_b64_tr_b8 v[106:107], v90 offset:1024\n\t" \
  "v_mfma_f32_16x16x128_f8f6f4 v[136:139], v[112:119], a[16:23], v[136:139]\n\t" \
  "ds_read_b64_tr_b8 v[108:109], v90 offset:32768\n\t" \
  "s_add_u32 s51, s68, 0xa1000\n\t" \
  "buffer_load_dwordx4 a[84:87], v4, s[12:15], s51 offen offset:1024 sc0 sc1\n\t" \
  "v_mfma_f32_16x16x128_f8f6f4 v[140:143], v[112:119], a[24:31], v[140:143]\n\t" \
  "ds_read_b64_tr_b8 v[110:111], v90 offset:33792\n\t" \
  "s_waitcnt lgkmcnt(4)\n\t" \
  "v_mfma_f32_16x16x128_f8f6f4 v[156:159], v[120:127], a[24:31], v[156:159]\n\t" \
  "ds_read_b64_tr_b8 v[112:113], v91 offset:0\n\t" \
  "s_add_u32 s52, s68, 0xa1000\n\t" \
  "buffer_load_dwordx4 a[88:91], v4, s[12:15], s52 offen offset:2048 sc0 sc1\n\t" \
  "v_mfma_f32_16x16x128_f8f6f4 v[152:155], v[120:127], a[16:23], v[152:155]\n\t" \
  "ds_read_b64_tr_b8 v[114:115], v91 offset:1024\n\t" \
  "v_mfma_f32_16x16x128_f8f6f4 v[148:151], v[120:127], a[8:15], v[148:151]\n\t" \
  "ds_read_b64_tr_b8 v[116:117], v91 offset:32768\n\t" \
  "s_add_u32 s53, s68, 0xa1000\n\t" \
  "buffer_load_dwordx4 a[92:95], v4, s[12:15], s53 offen offset:3072 sc0 sc1\n\t" \
  "v_mfma_f32_16x16x128_f8f6f4 v[144:147], v[120:127], a[0:7], v[144:147]\n\t" \
  "ds_read_b64_tr_b8 v[118:119], v91 offset:33792\n\t" \
  "s_waitcnt lgkmcnt(4)\n\t" \
  "v_mfma_f32_16x16x128_f8f6f4 v[160:163], v[104:111], a[0:7], v[160:163]\n\t" \
  "ds_read_b64_tr_b8 v[120:121], v92 offset:0\n\t" \
  "s_cmp_lg_u32 s26, 2\n\t" \
  "s_cbranch_scc1 Lnf2_2_%=\n\t" \
  "s_mov_b64 exec, 1\n\t" \
  "global_store_dword v65, v64, s[30:31] sc0 sc1\n\t" \
  "s_mov_b64 exec, -1\n\t" \
  "s_branch Lmf2_2_%=\n\t" \
  "Lnf2_2_%=:\n\t" \
  "buffer_load_dword v62, v65, s[20:23], s22 offen\n\t" \
  "Lmf2_2_%=:\n\t" \
  "s_cmp_lg_u32 s26, 3\n\t" \
  "s_cbranch_scc1 Lns3_2_%=\n\t" \
  "global_store_dwordx4 v63, v[72:75], s[28:29] sc0 sc1\n\t" \
  "global_store_dwordx4 v63, v[76:79], s[28:29] offset:1024 sc0 sc1\n\t" \
  "s_branch Lms3_2_%=\n\t" \
  "Lns3_2_%=:\n\t" \
  "buffer_load_dword v62, v65, s[20:23], s22 offen\n\t" \
  "buffer_load_dword v62, v65, s[20:23], s22 offen\n\t" \
  "Lms3_2_%=:\n\t" \
  "s_add_u32 m0, s24, 0x0\n\t" \
  "s_add_u32 s54, s41, 0x200000\n\t" \
  "buffer_load_dwordx4 v2, s[8:11], s54 offen lds\n\t" \
  "v_mfma_f32_16x16x128_f8f6f4 v[164:167], v[104:111], a[8:15], v[164:167]\n\t" \
  "ds_read_b64_tr_b8 v[122:123], v92 offset:1024\n\t" \
  "v_mfma_f32_16x16x128_f8f6f4 v[168:171], v[104:111], a[16:23], v[168:171]\n\t" \
  "ds_read_b64_tr_b8 v[124:125], v92 offset:32768\n\t" \
  "s_add_u32 m0, s24, 0x400\n\t" \
  "s_add_u32 s55, s41, 0x220000\n\t" \
  "buffer_load_dwordx4 v2, s[8:11], s55 offen lds\n\t" \
  "v_mfma_f32_16x16x128_f8f6f4 v[172:175], v[104:111], a[24:31], v[172:175]\n\t" \
  "ds_read_b64_tr_b8 v[126:127], v92 offset:33792\n\t" \
  "s_waitcnt lgkmcnt(4)\n\t" \
  "v_mfma_f32_16x16x128_f8f6f4 v[188:191], v[112:119], a[24:31], v[188:191]\n\t" \
  "ds_read_b64_tr_b8 v[104:105], v93 offset:0\n\t" \
  "s_add_u32 m0, s24, 0x2000\n\t" \
  "s_add_u32 s48, s42, 0x200000\n\t" \
  "buffer_load_dwordx4 v2, s[8:11], s48 offen lds\n\t" \
  "v_mfma_f32_16x16x128_f8f6f4 v[184:187], v[112:119], a[16:23], v[184:187]\n\t" \
  "ds_read_b64_tr_b8 v[106:107], v93 offset:1024\n\t" \
  "v_mfma_f32_16x16x128_f8f6f4 v[180:183], v[112:119], a[8:15], v[180:183]\n\t" \
  "ds_read_b64_tr_b8 v[108:109], v93 offset:32768\n\t" \
  "s_add_u32 m0, s24, 0x2400\n\t" \
  "s_add_u32 s49, s42, 0x220000\n\t" \
  "buffer_load_dwordx4 v2, s[8:11], s49 offen lds\n\t" \
  "v_mfma_f32_16x16x128_f8f6f4 v[176:179], v[112:119], a[0:7], v[176:179]\n\t" \
  "ds_read_b64_tr_b8 v[110:111], v93 offset:33792\n\t" \
  "s_waitcnt lgkmcnt(4)\n\t" \
  "v_mfma_f32_16x16x128_f8f6f4 v[192:195], v[120:127], a[0:7], v[192:195]\n\t" \
  "ds_read_b64_tr_b8 v[112:113], v94 offset:0\n\t" \
  "s_add_u32 m0, s24, 0x4000\n\t" \
  "s_add_u32 s50, s43, 0x200000\n\t" \
  "buffer_load_dwordx4 v2, s[8:11], s50 offen lds\n\t" \
  "v_mfma_f32_16x16x128_f8f6f4 v[196:199], v[120:127], a[8:15], v[196:199]\n\t" \
  "ds_read_b64_tr_b8 v[114:115], v94 offset:1024\n\t" \
  "v_mfma_f32_16x16x128_f8f6f4 v[200:203], v[120:127], a[16:23], v[200:203]\n\t" \
  "ds_read_b64_tr_b8 v[116:117], v94 offset:32768\n\t" \
  "s_add_u32 m0, s24, 0x4400\n\t" \
  "s_add_u32 s51, s43, 0x220000\n\t" \
  "buffer_load_dwordx4 v2, s[8:11], s51 offen lds\n\t" \
  "v_mfma_f32_16x16x128_f8f6f4 v[204:207], v[120:127], a[24:31], v[204:207]\n\t" \
  "ds_read_b64_tr_b8 v[118:119], v94 offset:33792\n\t" \
  "s_waitcnt lgkmcnt(4)\n\t" \
  "v_mfma_f32_16x16x128_f8f6f4 v[220:223], v[104:111], a[24:31], v[220:223]\n\t" \
  "ds_read_b64_tr_b8 v[120:121], v95 offset:0\n\t" \
  "s_add_u32 m0, s24, 0x6000\n\t" \
  "s_add_u32 s52, s44, 0x200000\n\t" \
  "buffer_load_dwordx4 v2, s[8:11], s52 offen lds\n\t" \
  "v_mfma_f32_16x16x128_f8f6f4 v[216:219], v[104:111], a[16:23], v[216:219]\n\t" \
  "ds_read_b64_tr_b8 v[122:123], v95 offset:1024\n\t" \
  "v_mfma_f32_16x16x128_f8f6f4 v[212:215], v[104:111], a[8:15], v[212:215]\n\t" \
  "ds_read_b64_tr_b8 v[124:125], v95 offset:32768\n\t" \
  "s_add_u32 m0, s24, 0x6400\n\t" \
  "s_add_u32 s53, s44, 0x220000\n\t" \
  "buffer_load_dwordx4 v2, s[8:11], s53 offen lds\n\t" \
  "v_mfma_f32_16x16x128_f8f6f4 v[208:211], v[104:111], a[0:7], v[208:211]\n\t" \
  "ds_read_b64_tr_b8 v[126:127], v95 offset:33792\n\t" \
  "s_waitcnt vmcnt(30) lgkmcnt(0)\n\t" \
  "s_barrier\n\t" \
  "v_mfma_f32_16x16x128_f8f6f4 v[224:227], v[112:119], a[0:7], v[224:227]\n\t" \
  "ds_read_b64_tr_b8 v[104:105], v88 offset:8192\n\t" \
  "v_mfma_f32_16x16x128_f8f6f4 v[228:231], v[112:119], a[8:15], v[228:231]\n\t" \
  "ds_read_b64_tr_b8 v[106:107], v88 offset:9216\n\t" \
  "v_mfma_f32_16x16x128_f8f6f4 v[232:235], v[112:119], a[16:23], v[232:235]\n\t" \
  "ds_read_b64_tr_b8 v[108:109], v88 offset:40960\n\t" \
  "v_mfma_f32_16x16x128_f8f6f4 v[236:239], v[112:119], a[24:31], v[236:239]\n\t" \
  "ds_read_b64_tr_b8 v[110:111], v88 offset:41984\n\t" \
  "v_mfma_f32_16x16x128_f8f6f4 v[252:255], v[120:127], a[24:31], v[252:255]\n\t" \
  "ds_read_b64_tr_b8 v[112:113], v89 offset:8192\n\t" \
  "v_mfma_f32_16x16x128_f8f6f4 v[248:251], v[120:127], a[16:23], v[248:251]\n\t" \
  "ds_read_b64_tr_b8 v[114:115], v89 offset:9216\n\t" \
  "v_mfma_f32_16x16x128_f8f6f4 v[244:247], v[120:127], a[8:15], v[244:247]\n\t" \
  "ds_read_b64_tr_b8 v[116:117], v89 offset:40960\n\t" \
  "v_mfma_f32_16x16x128_f8f6f4 v[240:243], v[120:127], a[0:7], v[240:243]\n\t" \
  "ds_read_b64_tr_b8 v[118:119], v89 offset:41984\n\t" \
  "s_waitcnt vmcnt(22) lgkmcnt(4)\n\t" \
  "v_mfma_f32_16x16x128_f8f6f4 v[128:131], v[104:111], a[32:39], v[128:131]\n\t" \
  "ds_read_b64_tr_b8 v[120:121], v90 offset:8192\n\t" \
  "v_mfma_f32_16x16x128_f8f6f4 v[132:135], v[104:111], a[40:47], v[132:135]\n\t" \
  "ds_read_b64_tr_b8 v[122:123], v90 offset:9216\n\t" \
  "v_mfma_f32_16x16x128_f8f6f4 v[136:139], v[104:111], a[48:55], v[136:139]\n\t" \
  "ds_read_b64_tr_b8 v[124:125], v90 offset:40960\n\t" \
  "v_mfma_f32_16x16x128_f8f6f4 v[140:143], v[104:111], a[56:63], v[140:143]\n\t" \
  "ds_read_b64_tr_b8 v[126:127], v90 offset:41984\n\t" \
  "s_waitcnt lgkmcnt(4)\n\t" \
  "v_mfma_f32_16x16x128_f8f6f4 v[156:159], v[112:119], a[56:63], v[156:159]\n\t" \
  "ds_read_b64_tr_b8 v[104:105], v91 offset:8192\n\t" \
  "v_mfma_f32_16x16x128_f8f6f4 v[152:155], v[112:119], a[48:55], v[152:155]\n\t" \
  "ds_read_b64_tr_b8 v[106:107], v91 offset:9216\n\t" \
  "v_mfma_f32_16x16x128_f8f6f4 v[148:151], v[112:119], a[40:47], v[148:151]\n\t" \
  "ds_read_b64_tr_b8 v[108:109], v91 offset:40960\n\t" \
  "v_mfma_f32_16x16x128_f8f6f4 v[144:147], v[112:119], a[32:39], v[144:147]\n\t" \
  "ds_read_b64_tr_b8 v[110:111], v91 offset:41984\n\t" \
  "s_waitcnt lgkmcnt(4)\n\t" \
  "v_mfma_f32_16x16x128_f8f6f4 v[160:163], v[120:127], a[32:39], v[160:163]\n\t" \
  "ds_read_b64_tr_b8 v[112:113], v92 offset:8192\n\t" \
  "v_mfma_f32_16x16x128_f8f6f4 v[164:167], v[120:127], a[40:47], v[164:167]\n\t" \
  "ds_read_b64_tr_b8 v[114:115], v92 offset:9216\n\t" \
  "v_mfma_f32_16x16x128_f8f6f4 v[168:171], v[120:127], a[48:55], v[168:171]\n\t" \
  "ds_read_b64_tr_b8 v[116:117], v92 offset:40960\n\t" \
  "v_mfma_f32_16x16x128_f8f6f4 v[172:175], v[120:127], a[56:63], v[172:175]\n\t" \
  "ds_read_b64_tr_b8 v[118:119], v92 offset:41984\n\t" \
  "s_waitcnt lgkmcnt(4)\n\t" \
  "v_mfma_f32_16x16x128_f8f6f4 v[188:191], v[104:111], a[56:63], v[188:191]\n\t" \
  "ds_read_b64_tr_b8 v[120:121], v93 offset:8192\n\t" \
  "s_add_u32 m0, s24, 0x1e000\n\t" \
  "s_add_u32 s54, s40, 0x200000\n\t" \
  "buffer_load_dwordx4 v3, s[8:11], s54 offen lds\n\t" \
  "v_mfma_f32_16x16x128_f8f6f4 v[184:187], v[104:111], a[48:55], v[184:187]\n\t" \
  "ds_read_b64_tr_b8 v[122:123], v93 offset:9216\n\t" \
  "v_mfma_f32_16x16x128_f8f6f4 v[180:183], v[104:111], a[40:47], v[180:183]\n\t" \
  "ds_read_b64_tr_b8 v[124:125], v93 offset:40960\n\t" \
  "s_add_u32 m0, s24, 0x1e400\n\t" \
  "s_add_u32 s55, s40, 0x220000\n\t" \
  "buffer_load_dwordx4 v3, s[8:11], s55 offen lds\n\t" \
  "v_mfma_f32_16x16x128_f8f6f4 v[176:179], v[104:111], a[32:39], v[176:179]\n\t" \
  "ds_read_b64_tr_b8 v[126:127], v93 offset:41984\n\t" \
  "s_waitcnt lgkmcnt(4)\n\t" \
  "v_mfma_f32_16x16x128_f8f6f4 v[192:195], v[112:119], a[32:39], v[192:195]\n\t" \
  "ds_read_b64_tr_b8 v[104:105], v94 offset:8192\n\t" \
  "s_waitcnt vmcnt(16)\n\t" \
  "v_min_u32 v68, v68, v69\n\t" \
  "v_min3_u32 v68, v68, v70, v71\n\t" \
  "v_cmp_le_u32 vcc, s57, v68\n\t" \
  "s_and_b32 s64, vcc_lo, 0x7f00\n\t" \
  "s_or_b32 s56, s56, s64\n\t" \
  "s_and_b32 s64, s56, 0x7f00\n\t" \
  "s_cmp_eq_u32 s64, 0x7f00\n\t" \
  "s_cbranch_scc1 Lok7_2_%=\n\t" \
  "s_mov_b32 s62, 0x7f00\n\t" \
  "s_mov_b32 s69, 0x7f00\n\t" \
  "s_call_b64 s[60:61], Lpoll_2_%=\n\t" \
  "Lok7_2_%=:\n\t" \
  "s_add_u32 s48, s68, 0xb0000\n\t" \
  "buffer_load_dwordx4 a[96:99], v4, s[12:15], s48 offen offset:0 sc0 sc1\n\t" \
  "v_mfma_f32_16x16x128_f8f6f4 v[196:199], v[112:119], a[40:47], v[196:199]\n\t" \
  "ds_read_b64_tr_b8 v[106:107], v94 offset:9216\n\t" \
  "v_mfma_f32_16x16x128_f8f6f4 v[200:203], v[112:119], a[48:55], v[200:203]\n\t" \
  "ds_read_b64_tr_b8 v[108:109], v94 offset:40960\n\t" \
  "s_add_u32 s49, s68, 0xb0000\n\t" \
  "buffer_load_dwordx4 a[100:103], v4, s[12:15], s49 offen offset:1024 sc0 sc1\n\t" \
  "v_mfma_f32_16x16x128_f8f6f4 v[204:207], v[112:119], a[56:63], v[204:207]\n\t" \
  "ds_read_b64_tr_b8 v[110:111], v94 offset:41984\n\t" \
  "s_waitcnt lgkmcnt(4)\n\t" \
  "v_mfma_f32_16x16x128_f8f6f4 v[220:223], v[120:127], a[56:63], v[220:223]\n\t" \
  "ds_read_b64_tr_b8 v[112:113], v95 offset:8192\n\t" \
  "s_add_u32 s50, s68, 0xb0000\n\t" \
  "buffer_load_dwordx4 a[104:107], v4, s[12:15], s50 offen offset:2048 sc0 sc1\n\t" \
  "v_mfma_f32_16x16x128_f8f6f4 v[216:219], v[120:127], a[48:55], v[216:219]\n\t" \
  "ds_read_b64_tr_b8 v[114:115], v95 offset:9216\n\t" \
  "v_mfma_f32_16x16x128_f8f6f4 v[212:215], v[120:127], a[40:47], v[212:215]\n\t" \
  "ds_read_b64_tr_b8 v[116:117], v95 offset:40960\n\t" \
  "s_add_u32 s51, s68, 0xb0000\n\t" \
  "buffer_load_dwordx4 a[108:111], v4, s[12:15], s51 offen offset:3072 sc0 sc1\n\t" \
  "v_mfma_f32_16x16x128_f8f6f4 v[208:211], v[120:127], a[32:39], v[208:211]\n\t" \
  "ds_read_b64_tr_b8 v[118:119], v95 offset:41984\n\t" \
  "s_waitcnt lgkmcnt(0)\n\t" \
  "s_barrier\n\t" \
  "v_mfma_f32_16x16x128_f8f6f4 v[224:227], v[104:111], a[32:39], v[224:227]\n\t" \
  "ds_read_b64_tr_b8 v[120:121], v88 offset:16384\n\t" \
  "s_add_u32 s52, s68, 0xb1000\n\t" \
  "buffer_load_dwordx4 a[112:115], v4, s[12:15], s52 offen offset:0 sc0 sc1\n\t" \
  "v_mfma_f32_16x16x128_f8f6f4 v[228:231], v[104:111], a[40:47], v[228:231]\n\t" \
  "ds_read_b64_tr_b8 v[122:123], v88 offset:17408\n\t" \
  "v_mfma_f32_16x16x128_f8f6f4 v[232:235], v[104:111], a[48:55], v[232:235]\n\t" \
  "ds_read_b64_tr_b8 v[124:125], v88 offset:49152\n\t" \
  "s_add_u32 s53, s68, 0xb1000\n\t" \
  "buffer_load_dwordx4 a[116:119], v4, s[12:15], s53 offen offset:1024 sc0 sc1\n\t" \
  "v_mfma_f32_16x16x128_f8f6f4 v[236:239], v[104:111], a[56:63], v[236:239]\n\t" \
  "ds_read_b64_tr_b8 v[126:127], v88 offset:50176\n\t" \
  "v_mfma_f32_16x16x128_f8f6f4 v[252:255], v[112:119], a[56:63], v[252:255]\n\t" \
  "ds_read_b64_tr_b8 v[104:105], v89 offset:16384\n\t" \
  "s_add_u32 s54, s68, 0xb1000\n\t" \
  "buffer_load_dwordx4 a[120:123], v4, s[12:15], s54 offen offset:2048 sc0 sc1\n\t" \
  "v_mfma_f32_16x16x128_f8f6f4 v[248:251], v[112:119], a[48:55], v[248:251]\n\t" \
  "ds_read_b64_tr_b8 v[106:107], v89 offset:17408\n\t" \
  "v_mfma_f32_16x16x128_f8f6f4 v[244:247], v[112:119], a[40:47], v[244:247]\n\t" \
  "ds_read_b64_tr_b8 v[108:109], v89 offset:49152\n\t" \
  "s_add_u32 s55, s68, 0xb1000\n\t" \
  "buffer_load_dwordx4 a[124:127], v4, s[12:15], s55 offen offset:3072 sc0 sc1\n\t" \
  "v_mfma_f32_16x16x128_f8f6f4 v[240:243], v[112:119], a[32:39], v[240:243]\n\t" \
  "ds_read_b64_tr_b8 v[110:111], v89 offset:50176\n\t" \
  "s_waitcnt vmcnt(21) lgkmcnt(4)\n\t" \
  "v_mfma_f32_16x16x128_f8f6f4 v[128:131], v[120:127], a[64:71], v[128:131]\n\t" \
  "ds_read_b64_tr_b8 v[112:113], v90 offset:16384\n\t" \
  "s_add_u32 m0, s24, 0x20000\n\t" \
  "s_add_u32 s48, s41, 0x200000\n\t" \
  "buffer_load_dwordx4 v3, s[8:11], s48 offen lds\n\t" \
  "v_mfma_f32_16x16x128_f8f6f4 v[132:135], v[120:127], a[72:79], v[132:135]\n\t" \
  "ds_read_b64_tr_b8 v[114:115], v90 offset:17408\n\t" \
  "v_mfma_f32_16x16x128_f8f6f4 v[136:139], v[120:127], a[80:87], v[136:139]\n\t" \
  "ds_read_b64_tr_b8 v[116:117], v90 offset:49152\n\t" \
  "s_add_u32 m0, s24, 0x20400\n\t" \
  "s_add_u32 s49, s41, 0x220000\n\t" \
  "buffer_load_dwordx4 v3, s[8:11], s49 offen lds\n\t" \
  "v_mfma_f32_16x16x128_f8f6f4 v[140:143], v[120:127], a[88:95], v[140:143]\n\t" \
  "ds_read_b64_tr_b8 v[118:119], v90 offset:50176\n\t" \
  "s_waitcnt lgkmcnt(4)\n\t" \
  "v_mfma_f32_16x16x128_f8f6f4 v[156:159], v[104:111], a[88:95], v[156:159]\n\t" \
  "ds_read_b64_tr_b8 v[120:121], v91 offset:16384\n\t" \
  "s_add_u32 s50, s68, 0xc0000\n\t" \
  "buffer_load_dwordx4 a[128:131], v4, s[12:15], s50 offen offset:0 sc0 sc1\n\t" \
  "v_mfma_f32_16x16x128_f8f6f4 v[152:155], v[104:111], a[80:87], v[152:155]\n\t" \
  "ds_read_b64_tr_b8 v[122:123], v91 offset:17408\n\t" \
  "v_mfma_f32_16x16x128_f8f6f4 v[148:151], v[104:111], a[72:79], v[148:151]\n\t" \
  "ds_read_b64_tr_b8 v[124:125], v91 offset:49152\n\t" \
  "s_add_u32 s51, s68, 0xc0000\n\t" \
  "buffer_load_dwordx4 a[132:135], v4, s[12:15], s51 offen offset:1024 sc0 sc1\n\t" \
  "v_mfma_f32_16x16x128_f8f6f4 v[144:147], v[104:111], a[64:71], v[144:147]\n\t" \
  "ds_read_b64_tr_b8 v[126:127], v91 offset:50176\n\t" \
  "s_waitcnt lgkmcnt(4)\n\t" \
  "v_mfma_f32_16x16x128_f8f6f4 v[160:163], v[112:119], a[64:71], v[160:163]\n\t" \
  "ds_read_b64_tr_b8 v[104:105], v92 offset:16384\n\t" \
  "s_add_u32 s52, s68, 0xc0000\n\t" \
  "buffer_load_dwordx4 a[136:139], v4, s[12:15], s52 offen offset:2048 sc0 sc1\n\t" \
  "v_mfma_f32_16x16x128_f8f6f4 v[164:167], v[112:119], a[72:79], v[164:167]\n\t" \
  "ds_read_b64_tr_b8 v[106:107], v92 offset:17408\n\t" \
  "v_mfma_f32_16x16x128_f8f6f4 v[168:171], v[112:119], a[80:87], v[168:171]\n\t" \
  "ds_read_b64_tr_b8 v[108:109], v92 offset:49152\n\t" \
  "s_add_u32 s53, s68, 0xc0000\n\t" \
  "buffer_load_dwordx4 a[140:143], v4, s[12:15], s53 offen offset:3072 sc0 sc1\n\t" \
  "v_mfma_f32_16x16x128_f8f6f4 v[172:175], v[112:119], a[88:95], v[172:175]\n\t" \
  "ds_read_b64_tr_b8 v[110:111], v92 offset:50176\n\t" \
  "s_waitcnt lgkmcnt(4)\n\t" \
  "v_mfma_f32_16x16x128_f8f6f4 v[188:191], v[120:127], a[88:95], v[188:191]\n\t" \
  "ds_read_b64_tr_b8 v[112:113], v93 offset:16384\n\t" \
  "s_add_u32 s54, s68, 0xc1000\n\t" \
  "buffer_load_dwordx4 a[144:147], v4, s[12:15], s54 offen offset:0 sc0 sc1\n\t" \
  "v_mfma_f32_16x16x128_f8f6f4 v[184:187], v[120:127], a[80:87], v[184:187]\n\t" \
  "ds_read_b64_tr_b8 v[114:115], v93 offset:17408\n\t" \
  "v_mfma_f32_16x16x128_f8f6f4 v[180:183], v[120:127], a[72:79], v[180:183]\n\t" \
  "ds_read_b64_tr_b8 v[116:117], v93 offset:49152\n\t" \
  "s_add_u32 s55, s68, 0xc1000\n\t" \
  "buffer_load_dwordx4 a[148:151], v4, s[12:15], s55 offen offset:1024 sc0 sc1\n\t" \
  "v_mfma_f32_16x16x128_f8f6f4 v[176:179], v[120:127], a[64:71], v[176:179]\n\t" \
  "ds_read_b64_tr_b8 v[118:119], v93 offset:50176\n\t" \
  "s_waitcnt lgkmcnt(4)\n\t" \
  "v_mfma_f32_16x16x128_f8f6f4 v[192:195], v[104:111], a[64:71], v[192:195]\n\t" \
  "ds_read_b64_tr_b8 v[120:121], v94 offset:16384\n\t" \
  "s_add_u32 s48, s68, 0xc1000\n\t" \
  "buffer_load_dwordx4 a[152:155], v4, s[12:15], s48 offen offset:2048 sc0 sc1\n\t" \
  "v_mfma_f32_16x16x128_f8f6f4 v[196:199], v[104:111], a[72:79], v[196:199]\n\t" \
  "ds_read_b64_tr_b8 v[122:123], v94 offset:17408\n\t" \
  "v_mfma_f32_16x16x128_f8f6f4 v[200:203], v[104:111], a[80:87], v[200:203]\n\t" \
  "ds_read_b64_tr_b8 v[124:125], v94 offset:49152\n\t" \
  "s_add_u32 s49, s68, 0xc1000\n\t" \
  "buffer_load_dwordx4 a[156:159], v4, s[12:15], s49 offen offset:3072 sc0 sc1\n\t" \
  "v_mfma_f32_16x16x128_f8f6f4 v[204:207], v[104:111], a[88:95], v[204:207]\n\t" \
  "ds_read_b64_tr_b8 v[126:127], v94 offset:50176\n\t" \
  "s_waitcnt lgkmcnt(4)\n\t" \
  "v_mfma_f32_16x16x128_f8f6f4 v[220:223], v[112:119], a[88:95], v[220:223]\n\t" \
  "ds_read_b64_tr_b8 v[104:105], v95 offset:16384\n\t" \
  "s_add_u32 m0, s24, 0x22000\n\t" \
  "s_add_u32 s50, s42, 0x200000\n\t" \
  "buffer_load_dwordx4 v3, s[8:11], s50 offen lds\n\t" \
  "v_mfma_f32_16x16x128_f8f6f4 v[216:219], v[112:119], a[80:87], v[216:219]\n\t" \
  "ds_read_b64_tr_b8 v[106:107], v95 offset:17408\n\t" \
  "v_mfma_f32_16x16x128_f8f6f4 v[212:215], v[112:119], a[72:79], v[212:215]\n\t" \
  "ds_read_b64_tr_b8 v[108:109], v95 offset:49152\n\t" \
  "s_add_u32 m0, s24, 0x22400\n\t" \
  "s_add_u32 s51, s42, 0x220000\n\t" \
  "buffer_load_dwordx4 v3, s[8:11], s51 offen lds\n\t" \
  "v_mfma_f32_16x16x128_f8f6f4 v[208:211], v[112:119], a[64:71], v[208:211]\n\t" \
  "ds_read_b64_tr_b8 v[110:111], v95 offset:50176\n\t" \
  "s_waitcnt vmcnt(20) lgkmcnt(0)\n\t" \
  "s_barrier\n\t" \
  "v_mfma_f32_16x16x128_f8f6f4 v[224:227], v[120:127], a[64:71], v[224:227]\n\t" \
  "ds_read_b64_tr_b8 v[112:113], v88 offset:24576\n\t" \
  "s_add_u32 s52, s68, 0xd0000\n\t" \
  "buffer_load_dwordx4 a[160:163], v4, s[12:15], s52 offen offset:0 sc0 sc1\n\t" \
  "v_mfma_f32_16x16x128_f8f6f4 v[228:231], v[120:127], a[72:79], v[228:231]\n\t" \
  "ds_read_b64_tr_b8 v[114:115], v88 offset:25600\n\t" \
  "v_mfma_f32_16x16x128_f8f6f4 v[232:235], v[120:127], a[80:87], v[232:235]\n\t" \
  "ds_read_b64_tr_b8 v[116:117], v88 offset:57344\n\t" \
  "s_add_u32 s53, s68, 0xd0000\n\t" \
  "buffer_load_dwordx4 a[164:167], v4, s[12:15], s53 offen offset:1024 sc0 sc1\n\t" \
  "v_mfma_f32_16x16x128_f8f6f4 v[236:239], v[120:127], a[88:95], v[236:239]\n\t" \
  "ds_read_b64_tr_b8 v[118:119], v88 offset:58368\n\t" \
  "v_mfma_f32_16x16x128_f8f6f4 v[252:255], v[104:111], a[88:95], v[252:255]\n\t" \
  "ds_read_b64_tr_b8 v[120:121], v89 offset:24576\n\t" \
  "s_add_u32 s54, s68, 0xd0000\n\t" \
  "buffer_load_dwordx4 a[168:171], v4, s[12:15], s54 offen offset:2048 sc0 sc1\n\t" \
  "v_mfma_f32_16x16x128_f8f6f4 v[248:251], v[104:111], a[80:87], v[248:251]\n\t" \
  "ds_read_b64_tr_b8 v[122:123], v89 offset:25600\n\t" \
  "v_mfma_f32_16x16x128_f8f6f4 v[244:247], v[104:111], a[72:79], v[244:247]\n\t" \
  "ds_read_b64_tr_b8 v[124:125], v89 offset:57344\n\t" \
  "s_add_u32 s55, s68, 0xd0000\n\t" \
  "buffer_load_dwordx4 a[172:175], v4, s[12:15], s55 offen offset:3072 sc0 sc1\n\t" \
  "v_mfma_f32_16x16x128_f8f6f4 v[240:243], v[104:111], a[64:71], v[240:243]\n\t" \
  "ds_read_b64_tr_b8 v[126:127], v89 offset:58368\n\t" \
  "s_waitcnt vmcnt(16) lgkmcnt(4)\n\t" \
  "v_mfma_f32_16x16x128_f8f6f4 v[128:131], v[112:119], a[96:103], v[128:131]\n\t" \
  "ds_read_b64_tr_b8 v[104:105], v90 offset:24576\n\t" \
  "s_add_u32 s48, s68, 0xd1000\n\t" \
  "buffer_load_dwordx4 a[176:179], v4, s[12:15], s48 offen offset:0 sc0 sc1\n\t" \
  "v_mfma_f32_16x16x128_f8f6f4 v[132:135], v[112:119], a[104:111], v[132:135]\n\t" \
  "ds_read_b64_tr_b8 v[106:107], v90 offset:25600\n\t" \
  "v_mfma_f32_16x16x128_f8f6f4 v[136:139], v[112:119], a[112:119], v[136:139]\n\t" \
  "ds_read_b64_tr_b8 v[108:109], v90 offset:57344\n\t" \
  "s_add_u32 s49, s68, 0xd1000\n\t" \
  "buffer_load_dwordx4 a[180:183], v4, s[12:15], s49 offen offset:1024 sc0 sc1\n\t" \
  "v_mfma_f32_16x16x128_f8f6f4 v[140:143], v[112:119], a[120:127], v[140:143]\n\t" \
  "ds_read_b64_tr_b8 v[110:111], v90 offset:58368\n\t" \
  "s_waitcnt lgkmcnt(4)\n\t" \
  "v_mfma_f32_16x16x128_f8f6f4 v[156:159], v[120:127], a[120:127], v[156:159]\n\t" \
  "ds_read_b64_tr_b8 v[112:113], v91 offset:24576\n\t" \
  "s_add_u32 s50, s68, 0xd1000\n\t" \
  "buffer_load_dwordx4 a[184:187], v4, s[12:15], s50 offen offset:2048 sc0 sc1\n\t" \
  "v_mfma_f32_16x16x128_f8f6f4 v[152:155], v[120:127], a[112:119], v[152:155]\n\t" \
  "ds_read_b64_tr_b8 v[114:115], v91 offset:25600\n\t" \
  "v_mfma_f32_16x16x128_f8f6f4 v[148:151], v[120:127], a[104:111], v[148:151]\n\t" \
  "ds_read_b64_tr_b8 v[116:117], v91 offset:57344\n\t" \
  "s_add_u32 s51, s68, 0xd1000\n\t" \
  "buffer_load_dwordx4 a[188:191], v4, s[12:15], s51 offen offset:3072 sc0 sc1\n\t" \
  "v_mfma_f32_16x16x128_f8f6f4 v[144:147], v[120:127], a[96:103], v[144:147]\n\t" \
  "ds_read_b64_tr_b8 v[118:119], v91 offset:58368\n\t" \
  "s_waitcnt lgkmcnt(4)\n\t" \
  "v_mfma_f32_16x16x128_f8f6f4 v[160:163], v[104:111], a[96:103], v[160:163]\n\t" \
  "ds_read_b64_tr_b8 v[120:121], v92 offset:24576\n\t" \
  "s_cmp_lg_u32 s26, 3\n\t" \
  "s_cbranch_scc1 Lnf3_2_%=\n\t" \
  "s_mov_b64 exec, 1\n\t" \
  "global_store_dword v65, v64, s[30:31] sc0 sc1\n\t" \
  "s_mov_b64 exec, -1\n\t" \
  "s_branch Lmf3_2_%=\n\t" \
  "Lnf3_2_%=:\n\t" \
  "buffer_load_dword v62, v65, s[20:23], s22 offen\n\t" \
  "Lmf3_2_%=:\n\t" \
  "s_add_u32 m0, s24, 0x8000\n\t" \
  "s_add_u32 s52, s45, 0x200000\n\t" \
  "buffer_load_dwordx4 v2, s[8:11], s52 offen lds\n\t" \
  "v_mfma_f32_16x16x128_f8f6f4 v[164:167], v[104:111], a[104:111], v[164:167]\n\t" \
  "ds_read_b64_tr_b8 v[122:123], v92 offset:25600\n\t" \
  "v_mfma_f32_16x16x128_f8f6f4 v[168:171], v[104:111], a[112:119], v[168:171]\n\t" \
  "ds_read_b64_tr_b8 v[124:125], v92 offset:57344\n\t" \
  "s_add_u32 m0, s24, 0x8400\n\t" \
  "s_add_u32 s53, s45, 0x220000\n\t" \
  "buffer_load_dwordx4 v2, s[8:11], s53 offen lds\n\t" \
  "v_mfma_f32_16x16x128_f8f6f4 v[172:175], v[104:111], a[120:127], v[172:175]\n\t" \
  "ds_read_b64_tr_b8 v[126:127], v92 offset:58368\n\t" \
  "s_waitcnt lgkmcnt(4)\n\t" \
  "v_mfma_f32_16x16x128_f8f6f4 v[188:191], v[112:119], a[120:127], v[188:191]\n\t" \
  "ds_read_b64_tr_b8 v[104:105], v93 offset:24576\n\t" \
  "s_add_u32 m0, s24, 0xa000\n\t" \
  "s_add_u32 s54, s46, 0x200000\n\t" \
  "buffer_load_dwordx4 v2, s[8:11], s54 offen lds\n\t" \
  "v_mfma_f32_16x16x128_f8f6f4 v[184:187], v[112:119], a[112:119], v[184:187]\n\t" \
  "ds_read_b64_tr_b8 v[106:107], v93 offset:25600\n\t" \
  "v_mfma_f32_16x16x128_f8f6f4 v[180:183], v[112:119], a[104:111], v[180:183]\n\t" \
  "ds_read_b64_tr_b8 v[108:109], v93 offset:57344\n\t" \
  "s_add_u32 m0, s24, 0xa400\n\t" \
  "s_add_u32 s55, s46, 0x220000\n\t" \
  "buffer_load_dwordx4 v2, s[8:11], s55 offen lds\n\t" \
  "v_mfma_f32_16x16x128_f8f6f4 v[176:179], v[112:119], a[96:103], v[176:179]\n\t" \
  "ds_read_b64_tr_b8 v[110:111], v93 offset:58368\n\t" \
  "s_waitcnt lgkmcnt(4)\n\t" \
  "v_mfma_f32_16x16x128_f8f6f4 v[192:195], v[120:127], a[96:103], v[192:195]\n\t" \
  "ds_read_b64_tr_b8 v[112:113], v94 offset:24576\n\t" \
  "s_add_u32 m0, s24, 0xc000\n\t" \
  "s_add_u32 s48, s40, 0x400000\n\t" \
  "buffer_load_dwordx4 v2, s[8:11], s48 offen lds\n\t" \
  "v_mfma_f32_16x16x128_f8f6f4 v[196:199], v[120:127], a[104:111], v[196:199]\n\t" \
  "ds_read_b64_tr_b8 v[114:115], v94 offset:25600\n\t" \
  "v_mfma_f32_16x16x128_f8f6f4 v[200:203], v[120:127], a[112:119], v[200:203]\n\t" \
  "ds_read_b64_tr_b8 v[116:117], v94 offset:57344\n\t" \
  "s_add_u32 m0, s24, 0xc400\n\t" \
  "s_add_u32 s49, s40, 0x420000\n\t" \
  "buffer_load_dwordx4 v2, s[8:11], s49 offen lds\n\t" \
  "v_mfma_f32_16x16x128_f8f6f4 v[204:207], v[120:127], a[120:127], v[204:207]\n\t" \
  "ds_read_b64_tr_b8 v[118:119], v94 offset:58368\n\t" \
  "s_waitcnt lgkmcnt(4)\n\t" \
  "v_mfma_f32_16x16x128_f8f6f4 v[220:223], v[104:111], a[120:127], v[220:223]\n\t" \
  "ds_read_b64_tr_b8 v[120:121], v95 offset:24576\n\t" \
  "s_add_u32 m0, s24, 0xe000\n\t" \
  "s_add_u32 s50, s41, 0x400000\n\t" \
  "buffer_load_dwordx4 v2, s[8:11], s50 offen lds\n\t" \
  "v_mfma_f32_16x16x128_f8f6f4 v[216:219], v[104:111], a[112:119], v[216:219]\n\t" \
  "ds_read_b64_tr_b8 v[122:123], v95 offset:25600\n\t" \
  "v_mfma_f32_16x16x128_f8f6f4 v[212:215], v[104:111], a[104:111], v[212:215]\n\t" \
  "ds_read_b64_tr_b8 v[124:125], v95 offset:57344\n\t" \
  "s_add_u32 m0, s24, 0xe400\n\t" \
  "s_add_u32 s51, s41, 0x420000\n\t" \
  "buffer_load_dwordx4 v2, s[8:11], s51 offen lds\n\t" \
  "v_mfma_f32_16x16x128_f8f6f4 v[208:211], v[104:111], a[96:103], v[208:211]\n\t" \
  "ds_read_b64_tr_b8 v[126:127], v95 offset:58368\n\t" \
  "s_waitcnt vmcnt(27) lgkmcnt(0)\n\t" \
  "s_barrier\n\t" \
  "v_mfma_f32_16x16x128_f8f6f4 v[224:227], v[112:119], a[96:103], v[224:227]\n\t" \
  "ds_read_b64_tr_b8 v[104:105], v80 offset:0\n\t" \
  "v_mfma_f32_16x16x128_f8f6f4 v[228:231], v[112:119], a[104:111], v[228:231]\n\t" \
  "ds_read_b64_tr_b8 v[106:107], v80 offset:1024\n\t" \
  "v_mfma_f32_16x16x128_f8f6f4 v[232:235], v[112:119], a[112:119], v[232:235]\n\t" \
  "ds_read_b64_tr_b8 v[108:109], v96 offset:0\n\t" \
  "v_mfma_f32_16x16x128_f8f6f4 v[236:239], v[112:119], a[120:127], v[236:239]\n\t" \
  "ds_read_b64_tr_b8 v[110:111], v96 offset:1024\n\t" \
  "v_mfma_f32_16x16x128_f8f6f4 v[252:255], v[120:127], a[120:127], v[252:255]\n\t" \
  "ds_read_b64_tr_b8 v[112:113], v81 offset:0\n\t" \
  "v_mfma_f32_16x16x128_f8f6f4 v[248:251], v[120:127], a[112:119], v[248:251]\n\t" \
  "ds_read_b64_tr_b8 v[114:115], v81 offset:1024\n\t" \
  "v_mfma_f32_16x16x128_f8f6f4 v[244:247], v[120:127], a[104:111], v[244:247]\n\t" \
  "ds_read_b64_tr_b8 v[116:117], v97 offset:0\n\t" \
  "v_mfma_f32_16x16x128_f8f6f4 v[240:243], v[120:127], a[96:103], v[240:243]\n\t" \
  "ds_read_b64_tr_b8 v[118:119], v97 offset:1024\n\t" \
  "s_waitcnt vmcnt(19) lgkmcnt(4)\n\t" \
  "v_mfma_f32_16x16x128_f8f6f4 v[128:131], v[104:111], a[128:135], v[128:131]\n\t" \
  "ds_read_b64_tr_b8 v[120:121], v82 offset:0\n\t" \
  "v_mfma_f32_16x16x128_f8f6f4 v[132:135], v[104:111], a[136:143], v[132:135]\n\t" \
  "ds_read_b64_tr_b8 v[122:123], v82 offset:1024\n\t" \
  "v_mfma_f32_16x16x128_f8f6f4 v[136:139], v[104:111], a[144:151], v[136:139]\n\t" \
  "ds_read_b64_tr_b8 v[124:125], v98 offset:0\n\t" \
  "v_mfma_f32_16x16x128_f8f6f4 v[140:143], v[104:111], a[152:159], v[140:143]\n\t" \
  "ds_read_b64_tr_b8 v[126:127], v98 offset:1024\n\t" \
  "s_waitcnt lgkmcnt(4)\n\t" \
  "v_mfma_f32_16x16x128_f8f6f4 v[156:159], v[112:119], a[152:159], v[156:159]\n\t" \
  "ds_read_b64_tr_b8 v[104:105], v83 offset:0\n\t" \
  "v_mfma_f32_16x16x128_f8f6f4 v[152:155], v[112:119], a[144:151], v[152:155]\n\t" \
  "ds_read_b64_tr_b8 v[106:107], v83 offset:1024\n\t" \
  "v_mfma_f32_16x16x128_f8f6f4 v[148:151], v[112:119], a[136:143], v[148:151]\n\t" \
  "ds_read_b64_tr_b8 v[108:109], v99 offset:0\n\t" \
  "s_add_u32 m0, s24, 0x24000\n\t" \
  "s_add_u32 s52, s43, 0x200000\n\t" \
  "buffer_load_dwordx4 v3, s[8:11], s52 offen lds\n\t" \
  "v_mfma_f32_16x16x128_f8f6f4 v[144:147], v[112:119], a[128:135], v[144:147]\n\t" \
  "ds_read_b64_tr_b8 v[110:111], v99 offset:1024\n\t" \
  "s_waitcnt lgkmcnt(4)\n\t" \
  "v_mfma_f32_16x16x128_f8f6f4 v[160:163], v[120:127], a[128:135], v[160:163]\n\t" \
  "ds_read_b64_tr_b8 v[112:113], v84 offset:0\n\t" \
  "s_add_u32 m0, s24, 0x24400\n\t" \
  "s_add_u32 s53, s43, 0x220000\n\t" \
  "buffer_load_dwordx4 v3, s[8:11], s53 offen lds\n\t" \
  "v_mfma_f32_16x16x128_f8f6f4 v[164:167], v[120:127], a[136:143], v[164:167]\n\t" \
  "ds_read_b64_tr_b8 v[114:115], v84 offset:1024\n\t" \
  "v_mfma_f32_16x16x128_f8f6f4 v[168:171], v[120:127], a[144:151], v[168:171]\n\t" \
  "ds_read_b64_tr_b8 v[116:117], v100 offset:0\n\t" \
  "s_add_u32 s54, s68, 0xe0000\n\t" \
  "buffer_load_dwordx4 a[192:195], v4, s[12:15], s54 offen offset:0 sc0 sc1\n\t" \
  "v_mfma_f32_16x16x128_f8f6f4 v[172:175], v[120:127], a[152:159], v[172:175]\n\t" \
  "ds_read_b64_tr_b8 v[118:119], v100 offset:1024\n\t" \
  "s_waitcnt lgkmcnt(4)\n\t" \
  "v_mfma_f32_16x16x128_f8f6f4 v[188:191], v[104:111], a[152:159], v[188:191]\n\t" \
  "ds_read_b64_tr_b8 v[120:121], v85 offset:0\n\t" \
  "s_add_u32 s55, s68, 0xe0000\n\t" \
  "buffer_load_dwordx4 a[196:199], v4, s[12:15], s55 offen offset:1024 sc0 sc1\n\t" \
  "v_mfma_f32_16x16x128_f8f6f4 v[184:187], v[104:111], a[144:151], v[184:187]\n\t" \
  "ds_read_b64_tr_b8 v[122:123], v85 offset:1024\n\t" \
  "v_mfma_f32_16x16x128_f8f6f4 v[180:183], v[104:111], a[136:143], v[180:183]\n\t" \
  "ds_read_b64_tr_b8 v[124:125], v101 offset:0\n\t" \
  "s_add_u32 s48, s68, 0xe0000\n\t" \
  "buffer_load_dwordx4 a[200:203], v4, s[12:15], s48 offen offset:2048 sc0 sc1\n\t" \
  "v_mfma_f32_16x16x128_f8f6f4 v[176:179], v[104:111], a[128:135], v[176:179]\n\t" \
  "ds_read_b64_tr_b8 v[126:127], v101 offset:1024\n\t" \
  "s_waitcnt lgkmcnt(4)\n\t" \
  "v_mfma_f32_16x16x128_f8f6f4 v[192:195], v[112:119], a[128:135], v[192:195]\n\t" \
  "ds_read_b64_tr_b8 v[104:105], v86 offset:0\n\t" \
  "s_add_u32 s49, s68, 0xe0000\n\t" \
  "buffer_load_dwordx4 a[204:207], v4, s[12:15], s49 offen offset:3072 sc0 sc1\n\t" \
  "v_mfma_f32_16x16x128_f8f6f4 v[196:199], v[112:119], a[136:143], v[196:199]\n\t" \
  "ds_read_b64_tr_b8 v[106:107], v86 offset:1024\n\t" \
  "v_mfma_f32_16x16x128_f8f6f4 v[200:203], v[112:119], a[144:151], v[200:203]\n\t" \
  "ds_read_b64_tr_b8 v[108:109], v102 offset:0\n\t" \
  "s_add_u32 s50, s68, 0xe1000\n\t" \
  "buffer_load_dwordx4 a[208:211], v4, s[12:15], s50 offen offset:0 sc0 sc1\n\t" \
  "v_mfma_f32_16x16x128_f8f6f4 v[204:207], v[112:119], a[152:159], v[204:207]\n\t" \
  "ds_read_b64_tr_b8 v[110:111], v102 offset:1024\n\t" \
  "s_waitcnt lgkmcnt(4)\n\t" \
  "v_mfma_f32_16x16x128_f8f6f4 v[220:223], v[120:127], a[152:159], v[220:223]\n\t" \
  "ds_read_b64_tr_b8 v[112:113], v87 offset:0\n\t" \
  "s_add_u32 s51, s68, 0xe1000\n\t" \
  "buffer_load_dwordx4 a[212:215], v4, s[12:15], s51 offen offset:1024 sc0 sc1\n\t" \
  "v_mfma_f32_16x16x128_f8f6f4 v[216:219], v[120:127], a[144:151], v[216:219]\n\t" \
  "ds_read_b64_tr_b8 v[114:115], v87 offset:1024\n\t" \
  "v_mfma_f32_16x16x128_f8f6f4 v[212:215], v[120:127], a[136:143], v[212:215]\n\t" \
  "ds_read_b64_tr_b8 v[116:117], v103 offset:0\n\t" \
  "s_add_u32 s52, s68, 0xe1000\n\t" \
  "buffer_load_dwordx4 a[216:219], v4, s[12:15], s52 offen offset:2048 sc0 sc1\n\t" \
  "v_mfma_f32_16x16x128_f8f6f4 v[208:211], v[120:127], a[128:135], v[208:211]\n\t" \
  "ds_read_b64_tr_b8 v[118:119], v103 offset:1024\n\t" \
  "s_waitcnt vmcnt(26) lgkmcnt(0)\n\t" \
  "s_barrier\n\t" \
  "v_mfma_f32_16x16x128_f8f6f4 v[224:227], v[104:111], a[128:135], v[224:227]\n\t" \
  "ds_read_b64_tr_b8 v[120:121], v80 offset:8192\n\t" \
  "s_add_u32 s53, s68, 0xe1000\n\t" \
  "buffer_load_dwordx4 a[220:223], v4, s[12:15], s53 offen offset:3072 sc0 sc1\n\t" \
  "v_mfma_f32_16x16x128_f8f6f4 v[228:231], v[104:111], a[136:143], v[228:231]\n\t" \
  "ds_read_b64_tr_b8 v[122:123], v80 offset:9216\n\t" \
  "v_mfma_f32_16x16x128_f8f6f4 v[232:235], v[104:111], a[144:151], v[232:235]\n\t" \
  "ds_read_b64_tr_b8 v[124:125], v96 offset:8192\n\t" \
  "s_add_u32 m0, s24, 0x26000\n\t" \
  "s_add_u32 s54, s44, 0x200000\n\t" \
  "buffer_load_dwordx4 v3, s[8:11], s54 offen lds\n\t" \
  "v_mfma_f32_16x16x128_f8f6f4 v[236:239], v[104:111], a[152:159], v[236:239]\n\t" \
  "ds_read_b64_tr_b8 v[126:127], v96 offset:9216\n\t" \
  "v_mfma_f32_16x16x128_f8f6f4 v[252:255], v[112:119], a[152:159], v[252:255]\n\t" \
  "ds_read_b64_tr_b8 v[104:105], v81 offset:8192\n\t" \
  "s_add_u32 m0, s24, 0x26400\n\t" \
  "s_add_u32 s55, s44, 0x220000\n\t" \
  "buffer_load_dwordx4 v3, s[8:11], s55 offen lds\n\t" \
  "v_mfma_f32_16x16x128_f8f6f4 v[248:251], v[112:119], a[144:151], v[248:251]\n\t" \
  "ds_read_b64_tr_b8 v[106:107], v81 offset:9216\n\t" \
  "v_mfma_f32_16x16x128_f8f6f4 v[244:247], v[112:119], a[136:143], v[244:247]\n\t" \
  "ds_read_b64_tr_b8 v[108:109], v97 offset:8192\n\t" \
  "s_add_u32 s48, s68, 0xf0000\n\t" \
  "buffer_load_dwordx4 a[224:227], v4, s[12:15], s48 offen offset:0 sc0 sc1\n\t" \
  "v_mfma_f32_16x16x128_f8f6f4 v[240:243], v[112:119], a[128:135], v[240:243]\n\t" \
  "ds_read_b64_tr_b8 v[110:111], v97 offset:9216\n\t" \
  "s_waitcnt vmcnt(22) lgkmcnt(4)\n\t" \
  "v_mfma_f32_16x16x128_f8f6f4 v[128:131], v[120:127], a[160:167], v[128:131]\n\t" \
  "ds_read_b64_tr_b8 v[112:113], v82 offset:8192\n\t" \
  "s_add_u32 s49, s68, 0xf0000\n\t" \
  "buffer_load_dwordx4 a[228:231], v4, s[12:15], s49 offen offset:1024 sc0 sc1\n\t" \
  "v_mfma_f32_16x16x128_f8f6f4 v[132:135], v[120:127], a[168:175], v[132:135]\n\t" \
  "ds_read_b64_tr_b8 v[114:115], v82 offset:9216\n\t" \
  "v_mfma_f32_16x16x128_f8f6f4 v[136:139], v[120:127], a[176:183], v[136:139]\n\t" \
  "ds_read_b64_tr_b8 v[116:117], v98 offset:8192\n\t" \
  "s_add_u32 s50, s68, 0xf0000\n\t" \
  "buffer_load_dwordx4 a[232:235], v4, s[12:15], s50 offen offset:2048 sc0 sc1\n\t" \
  "v_mfma_f32_16x16x128_f8f6f4 v[140:143], v[120:127], a[184:191], v[140:143]\n\t" \
  "ds_read_b64_tr_b8 v[118:119], v98 offset:9216\n\t" \
  "s_waitcnt lgkmcnt(4)\n\t" \
  "v_mfma_f32_16x16x128_f8f6f4 v[156:159], v[104:111], a[184:191], v[156:159]\n\t" \
  "ds_read_b64_tr_b8 v[120:121], v83 offset:8192\n\t" \
  "s_add_u32 s51, s68, 0xf0000\n\t" \
  "buffer_load_dwordx4 a[236:239], v4, s[12:15], s51 offen offset:3072 sc0 sc1\n\t" \
  "v_mfma_f32_16x16x128_f8f6f4 v[152:155], v[104:111], a[176:183], v[152:155]\n\t" \
  "ds_read_b64_tr_b8 v[122:123], v83 offset:9216\n\t" \
  "v_mfma_f32_16x16x128_f8f6f4 v[148:151], v[104:111], a[168:175], v[148:151]\n\t" \
  "ds_read_b64_tr_b8 v[124:125], v99 offset:8192\n\t" \
  "s_add_u32 s52, s68, 0xf1000\n\t" \
  "buffer_load_dwordx4 a[240:243], v4, s[12:15], s52 offen offset:0 sc0 sc1\n\t" \
  "v_mfma_f32_16x16x128_f8f6f4 v[144:147], v[104:111], a[160:167], v[144:147]\n\t" \
  "ds_read_b64_tr_b8 v[126:127], v99 offset:9216\n\t" \
  "s_waitcnt lgkmcnt(4)\n\t" \
  "v_mfma_f32_16x16x128_f8f6f4 v[160:163], v[112:119], a[160:167], v[160:163]\n\t" \
  "ds_read_b64_tr_b8 v[104:105], v84 offset:8192\n\t" \
  "s_add_u32 s53, s68, 0xf1000\n\t" \
  "buffer_load_dwordx4 a[244:247], v4, s[12:15], s53 offen offset:1024 sc0 sc1\n\t" \
  "v_mfma_f32_16x16x128_f8f6f4 v[164:167], v[112:119], a[168:175], v[164:167]\n\t" \
  "ds_read_b64_tr_b8 v[106:107], v84 offset:9216\n\t" \
  "v_mfma_f32_16x16x128_f8f6f4 v[168:171], v[112:119], a[176:183], v[168:171]\n\t" \
  "ds_read_b64_tr_b8 v[108:109], v100 offset:8192\n\t" \
  "s_add_u32 s54, s68, 0xf1000\n\t" \
  "buffer_load_dwordx4 a[248:251], v4, s[12:15], s54 offen offset:2048 sc0 sc1\n\t" \
  "v_mfma_f32_16x16x128_f8f6f4 v[172:175], v[112:119], a[184:191], v[172:175]\n\t" \
  "ds_read_b64_tr_b8 v[110:111], v100 offset:9216\n\t" \
  "s_waitcnt lgkmcnt(4)\n\t" \
  "v_mfma_f32_16x16x128_f8f6f4 v[188:191], v[120:127], a[184:191], v[188:191]\n\t" \
  "ds_read_b64_tr_b8 v[112:113], v85 offset:8192\n\t" \
  "s_add_u32 s55, s68, 0xf1000\n\t" \
  "buffer_load_dwordx4 a[252:255], v4, s[12:15], s55 offen offset:3072 sc0 sc1\n\t" \
  "v_mfma_f32_16x16x128_f8f6f4 v[184:187], v[120:127], a[176:183], v[184:187]\n\t" \
  "ds_read_b64_tr_b8 v[114:115], v85 offset:9216\n\t" \
  "v_mfma_f32_16x16x128_f8f6f4 v[180:183], v[120:127], a[168:175], v[180:183]\n\t" \
  "ds_read_b64_tr_b8 v[116:117], v101 offset:8192\n\t" \
  "s_add_u32 m0, s24, 0x18000\n\t" \
  "s_add_u32 s48, s45, 0x200000\n\t" \
  "buffer_load_dwordx4 v3, s[8:11], s48 offen lds\n\t" \
  "v_mfma_f32_16x16x128_f8f6f4 v[176:179], v[120:127], a[160:167], v[176:179]\n\t" \
  "ds_read_b64_tr_b8 v[118:119], v101 offset:9216\n\t" \
  "s_waitcnt lgkmcnt(4)\n\t" \
  "v_mfma_f32_16x16x128_f8f6f4 v[192:195], v[104:111], a[160:167], v[192:195]\n\t" \
  "ds_read_b64_tr_b8 v[120:121], v86 offset:8192\n\t" \
  "s_add_u32 m0, s24, 0x18400\n\t" \
  "s_add_u32 s49, s45, 0x220000\n\t" \
  "buffer_load_dwordx4 v3, s[8:11], s49 offen lds\n\t" \
  "v_mfma_f32_16x16x128_f8f6f4 v[196:199], v[104:111], a[168:175], v[196:199]\n\t" \
  "ds_read_b64_tr_b8 v[122:123], v86 offset:9216\n\t" \
  "v_mfma_f32_16x16x128_f8f6f4 v[200:203], v[104:111], a[176:183], v[200:203]\n\t" \
  "ds_read_b64_tr_b8 v[124:125], v102 offset:8192\n\t" \
  "s_add_u32 s50, s68, 0x100000\n\t" \
  "buffer_load_dwordx4 a[0:3], v4, s[12:15], s50 offen offset:0 sc0 sc1\n\t" \
  "v_mfma_f32_16x16x128_f8f6f4 v[204:207], v[104:111], a[184:191], v[204:207]\n\t" \
  "ds_read_b64_tr_b8 v[126:127], v102 offset:9216\n\t" \
  "s_waitcnt lgkmcnt(4)\n\t" \
  "v_mfma_f32_16x16x128_f8f6f4 v[220:223], v[112:119], a[184:191], v[220:223]\n\t" \
  "ds_read_b64_tr_b8 v[104:105], v87 offset:8192\n\t" \
  "s_add_u32 s51, s68, 0x100000\n\t" \
  "buffer_load_dwordx4 a[4:7], v4, s[12:15], s51 offen offset:1024 sc0 sc1\n\t" \
  "v_mfma_f32_16x16x128_f8f6f4 v[216:219], v[112:119], a[176:183], v[216:219]\n\t" \
  "ds_read_b64_tr_b8 v[106:107], v87 offset:9216\n\t" \
  "v_mfma_f32_16x16x128_f8f6f4 v[212:215], v[112:119], a[168:175], v[212:215]\n\t" \
  "ds_read_b64_tr_b8 v[108:109], v103 offset:8192\n\t" \
  "s_add_u32 s52, s68, 0x100000\n\t" \
  "buffer_load_dwordx4 a[8:11], v4, s[12:15], s52 offen offset:2048 sc0 sc1\n\t" \
  "v_mfma_f32_16x16x128_f8f6f4 v[208:211], v[112:119], a[160:167], v[208:211]\n\t" \
  "ds_read_b64_tr_b8 v[110:111], v103 offset:9216\n\t" \
  "s_waitcnt vmcnt(23) lgkmcnt(0)\n\t" \
  "s_barrier\n\t" \
  "v_mfma_f32_16x16x128_f8f6f4 v[224:227], v[120:127], a[160:167], v[224:227]\n\t" \
  "ds_read_b64_tr_b8 v[112:113], v80 offset:16384\n\t" \
  "s_add_u32 s53, s68, 0x100000\n\t" \
  "buffer_load_dwordx4 a[12:15], v4, s[12:15], s53 offen offset:3072 sc0 sc1\n\t" \
  "v_mfma_f32_16x16x128_f8f6f4 v[228:231], v[120:127], a[168:175], v[228:231]\n\t" \
  "ds_read_b64_tr_b8 v[114:115], v80 offset:17408\n\t" \
  "v_mfma_f32_16x16x128_f8f6f4 v[232:235], v[120:127], a[176:183], v[232:235]\n\t" \
  "ds_read_b64_tr_b8 v[116:117], v96 offset:16384\n\t" \
  "s_add_u32 s54, s68, 0x101000\n\t" \
  "buffer_load_dwordx4 a[16:19], v4, s[12:15], s54 offen offset:0 sc0 sc1\n\t" \
  "v_mfma_f32_16x16x128_f8f6f4 v[236:239], v[120:127], a[184:191], v[236:239]\n\t" \
  "ds_read_b64_tr_b8 v[118:119], v96 offset:17408\n\t" \
  "v_mfma_f32_16x16x128_f8f6f4 v[252:255], v[104:111], a[184:191], v[252:255]\n\t" \
  "ds_read_b64_tr_b8 v[120:121], v81 offset:16384\n\t" \
  "s_add_u32 s55, s68, 0x101000\n\t" \
  "buffer_load_dwordx4 a[20:23], v4, s[12:15], s55 offen offset:1024 sc0 sc1\n\t" \
  "v_mfma_f32_16x16x128_f8f6f4 v[248:251], v[104:111], a[176:183], v[248:251]\n\t" \
  "ds_read_b64_tr_b8 v[122:123], v81 offset:17408\n\t" \
  "v_mfma_f32_16x16x128_f8f6f4 v[244:247], v[104:111], a[168:175], v[244:247]\n\t" \
  "ds_read_b64_tr_b8 v[124:125], v97 offset:16384\n\t" \
  "s_add_u32 s48, s68, 0x101000\n\t" \
  "buffer_load_dwordx4 a[24:27], v4, s[12:15], s48 offen offset:2048 sc0 sc1\n\t" \
  "v_mfma_f32_16x16x128_f8f6f4 v[240:243], v[104:111], a[160:167], v[240:243]\n\t" \
  "ds_read_b64_tr_b8 v[126:127], v97 offset:17408\n\t" \
  "s_waitcnt vmcnt(19) lgkmcnt(4)\n\t" \
  "v_mfma_f32_16x16x128_f8f6f4 v[128:131], v[112:119], a[192:199], v[128:131]\n\t" \
  "ds_read_b64_tr_b8 v[104:105], v82 offset:16384\n\t" \
  "s_add_u32 s49, s68, 0x101000\n\t" \
  "buffer_load_dwordx4 a[28:31], v4, s[12:15], s49 offen offset:3072 sc0 sc1\n\t" \
  "s_mov_b32 exec_lo, 0x7f0000\n\t" \
  "s_mov_b32 exec_hi, 0\n\t" \
  "global_load_dwordx4 v[68:71], v10, s[6:7] sc0 sc1\n\t" \
  "s_mov_b64 exec, -1\n\t" \
  "v_mfma_f32_16x16x128_f8f6f4 v[132:135], v[112:119], a[200:207], v[132:135]\n\t" \
  "ds_read_b64_tr_b8 v[106:107], v82 offset:17408\n\t" \
  "v_mfma_f32_16x16x128_f8f6f4 v[136:139], v[112:119], a[208:215], v[136:139]\n\t" \
  "ds_read_b64_tr_b8 v[108:109], v98 offset:16384\n\t" \
  "s_add_u32 m0, s24, 0x1a000\n\t" \
  "s_add_u32 s50, s46, 0x200000\n\t" \
  "buffer_load_dwordx4 v3, s[8:11], s50 offen lds\n\t" \
  "v_mfma_f32_16x16x128_f8f6f4 v[140:143], v[112:119], a[216:223], v[140:143]\n\t" \
  "ds_read_b64_tr_b8 v[110:111], v98 offset:17408\n\t" \
  "s_waitcnt lgkmcnt(4)\n\t" \
  "v_mfma_f32_16x16x128_f8f6f4 v[156:159], v[120:127], a[216:223], v[156:159]\n\t" \
  "ds_read_b64_tr_b8 v[112:113], v83 offset:16384\n\t" \
  "s_add_u32 m0, s24, 0x1a400\n\t" \
  "s_add_u32 s51, s46, 0x220000\n\t" \
  "buffer_load_dwordx4 v3, s[8:11], s51 offen lds\n\t" \
  "v_mfma_f32_16x16x128_f8f6f4 v[152:155], v[120:127], a[208:215], v[152:155]\n\t" \
  "ds_read_b64_tr_b8 v[114:115], v83 offset:17408\n\t" \
  "v_mfma_f32_16x16x128_f8f6f4 v[148:151], v[120:127], a[200:207], v[148:151]\n\t" \
  "ds_read_b64_tr_b8 v[116:117], v99 offset:16384\n\t" \
  "s_add_u32 s52, s68, 0x110000\n\t" \
  "buffer_load_dwordx4 a[32:35], v4, s[12:15], s52 offen offset:0 sc0 sc1\n\t" \
  "v_mfma_f32_16x16x128_f8f6f4 v[144:147], v[120:127], a[192:199], v[144:147]\n\t" \
  "ds_read_b64_tr_b8 v[118:119], v99 offset:17408\n\t" \
  "s_waitcnt lgkmcnt(4)\n\t" \
  "v_mfma_f32_16x16x128_f8f6f4 v[160:163], v[104:111], a[192:199], v[160:163]\n\t" \
  "ds_read_b64_tr_b8 v[120:121], v84 offset:16384\n\t" \
  "s_add_u32 s53, s68, 0x110000\n\t" \
  "buffer_load_dwordx4 a[36:39], v4, s[12:15], s53 offen offset:1024 sc0 sc1\n\t" \
  "v_mfma_f32_16x16x128_f8f6f4 v[164:167], v[104:111], a[200:207], v[164:167]\n\t" \
  "ds_read_b64_tr_b8 v[122:123], v84 offset:17408\n\t" \
  "v_mfma_f32_16x16x128_f8f6f4 v[168:171], v[104:111], a[208:215], v[168:171]\n\t" \
  "ds_read_b64_tr_b8 v[124:125], v100 offset:16384\n\t" \
  "s_add_u32 s54, s68, 0x110000\n\t" \
  "buffer_load_dwordx4 a[40:43], v4, s[12:15], s54 offen offset:2048 sc0 sc1\n\t" \
  "v_mfma_f32_16x16x128_f8f6f4 v[172:175], v[104:111], a[216:223], v[172:175]\n\t" \
  "ds_read_b64_tr_b8 v[126:127], v100 offset:17408\n\t" \
  "s_waitcnt lgkmcnt(4)\n\t" \
  "v_mfma_f32_16x16x128_f8f6f4 v[188:191], v[112:119], a[216:223], v[188:191]\n\t" \
  "ds_read_b64_tr_b8 v[104:105], v85 offset:16384\n\t" \
  "s_add_u32 s55, s68, 0x110000\n\t" \
  "buffer_load_dwordx4 a[44:47], v4, s[12:15], s55 offen offset:3072 sc0 sc1\n\t" \
  "v_mfma_f32_16x16x128_f8f6f4 v[184:187], v[112:119], a[208:215], v[184:187]\n\t" \
  "ds_read_b64_tr_b8 v[106:107], v85 offset:17408\n\t" \
  "v_mfma_f32_16x16x128_f8f6f4 v[180:183], v[112:119], a[200:207], v[180:183]\n\t" \
  "ds_read_b64_tr_b8 v[108:109], v101 offset:16384\n\t" \
  "s_add_u32 s48, s68, 0x111000\n\t" \
  "buffer_load_dwordx4 a[48:51], v4, s[12:15], s48 offen offset:0 sc0 sc1\n\t" \
  "v_mfma_f32_16x16x128_f8f6f4 v[176:179], v[112:119], a[192:199], v[176:179]\n\t" \
  "ds_read_b64_tr_b8 v[110:111], v101 offset:17408\n\t" \
  "s_waitcnt lgkmcnt(4)\n\t" \
  "v_mfma_f32_16x16x128_f8f6f4 v[192:195], v[120:127], a[192:199], v[192:195]\n\t" \
  "ds_read_b64_tr_b8 v[112:113], v86 offset:16384\n\t" \
  "s_add_u32 s49, s68, 0x111000\n\t" \
  "buffer_load_dwordx4 a[52:55], v4, s[12:15], s49 offen offset:1024 sc0 sc1\n\t" \
  "v_mfma_f32_16x16x128_f8f6f4 v[196:199], v[120:127], a[200:207], v[196:199]\n\t" \
  "ds_read_b64_tr_b8 v[114:115], v86 offset:17408\n\t" \
  "v_mfma_f32_16x16x128_f8f6f4 v[200:203], v[120:127], a[208:215], v[200:203]\n\t" \
  "ds_read_b64_tr_b8 v[116:117], v102 offset:16384\n\t" \
  "s_add_u32 s50, s68, 0x111000\n\t" \
  "buffer_load_dwordx4 a[56:59], v4, s[12:15], s50 offen offset:2048 sc0 sc1\n\t" \
  "v_mfma_f32_16x16x128_f8f6f4 v[204:207], v[120:127], a[216:223], v[204:207]\n\t" \
  "ds_read_b64_tr_b8 v[118:119], v102 offset:17408\n\t" \
  "s_waitcnt lgkmcnt(4)\n\t" \
  "v_mfma_f32_16x16x128_f8f6f4 v[220:223], v[104:111], a[216:223], v[220:223]\n\t" \
  "ds_read_b64_tr_b8 v[120:121], v87 offset:16384\n\t" \
  "s_add_u32 s51, s68, 0x111000\n\t" \
  "buffer_load_dwordx4 a[60:63], v4, s[12:15], s51 offen offset:3072 sc0 sc1\n\t" \
  "v_mfma_f32_16x16x128_f8f6f4 v[216:219], v[104:111], a[208:215], v[216:219]\n\t" \
  "ds_read_b64_tr_b8 v[122:123], v87 offset:17408\n\t" \
  "v_mfma_f32_16x16x128_f8f6f4 v[212:215], v[104:111], a[200:207], v[212:215]\n\t" \
  "ds_read_b64_tr_b8 v[124:125], v103 offset:16384\n\t" \
  "s_add_u32 m0, s24, 0x10000\n\t" \
  "s_add_u32 s52, s42, 0x400000\n\t" \
  "buffer_load_dwordx4 v2, s[8:11], s52 offen lds\n\t" \
  "v_mfma_f32_16x16x128_f8f6f4 v[208:211], v[104:111], a[192:199], v[208:211]\n\t" \
  "ds_read_b64_tr_b8 v[126:127], v103 offset:17408\n\t" \
  "s_waitcnt vmcnt(30) lgkmcnt(0)\n\t" \
  "s_barrier\n\t" \
  "v_mfma_f32_16x16x128_f8f6f4 v[224:227], v[112:119], a[192:199], v[224:227]\n\t" \
  "ds_read_b64_tr_b8 v[104:105], v80 offset:24576\n\t" \
  "s_add_u32 m0, s24, 0x10400\n\t" \
  "s_add_u32 s53, s42, 0x420000\n\t" \
  "buffer_load_dwordx4 v2, s[8:11], s53 offen lds\n\t" \
  "v_mfma_f32_16x16x128_f8f6f4 v[228:231], v[112:119], a[200:207], v[228:231]\n\t" \
  "ds_read_b64_tr_b8 v[106:107], v80 offset:25600\n\t" \
  "v_mfma_f32_16x16x128_f8f6f4 v[232:235], v[112:119], a[208:215], v[232:235]\n\t" \
  "ds_read_b64_tr_b8 v[108:109], v96 offset:24576\n\t" \
  "s_add_u32 m0, s24, 0x12000\n\t" \
  "s_add_u32 s54, s43, 0x400000\n\t" \
  "buffer_load_dwordx4 v2, s[8:11], s54 offen lds\n\t" \
  "v_mfma_f32_16x16x128_f8f6f4 v[236:239], v[112:119], a[216:223], v[236:239]\n\t" \
  "ds_read_b64_tr_b8 v[110:111], v96 offset:25600\n\t" \
  "v_mfma_f32_16x16x128_f8f6f4 v[252:255], v[120:127], a[216:223], v[252:255]\n\t" \
  "ds_read_b64_tr_b8 v[112:113], v81 offset:24576\n\t" \
  "s_add_u32 m0, s24, 0x12400\n\t" \
  "s_add_u32 s55, s43, 0x420000\n\t" \
  "buffer_load_dwordx4 v2, s[8:11], s55 offen lds\n\t" \
  "v_mfma_f32_16x16x128_f8f6f4 v[248:251], v[120:127], a[208:215], v[248:251]\n\t" \
  "ds_read_b64_tr_b8 v[114:115], v81 offset:25600\n\t" \
  "v_mfma_f32_16x16x128_f8f6f4 v[244:247], v[120:127], a[200:207], v[244:247]\n\t" \
  "ds_read_b64_tr_b8 v[116:117], v97 offset:24576\n\t" \
  "s_add_u32 m0, s24, 0x14000\n\t" \
  "s_add_u32 s48, s44, 0x400000\n\t" \
  "buffer_load_dwordx4 v2, s[8:11], s48 offen lds\n\t" \
  "v_mfma_f32_16x16x128_f8f6f4 v[240:243], v[120:127], a[192:199], v[240:243]\n\t" \
  "ds_read_b64_tr_b8 v[118:119], v97 offset:25600\n\t" \
  "s_waitcnt vmcnt(26) lgkmcnt(4)\n\t" \
  "v_mfma_f32_16x16x128_f8f6f4 v[128:131], v[104:111], a[224:231], v[128:131]\n\t" \
  "ds_read_b64_tr_b8 v[120:121], v82 offset:24576\n\t" \
  "s_add_u32 m0, s24, 0x14400\n\t" \
  "s_add_u32 s49, s44, 0x420000\n\t" \
  "buffer_load_dwordx4 v2, s[8:11], s49 offen lds\n\t" \
  "v_mfma_f32_16x16x128_f8f6f4 v[132:135], v[104:111], a[232:239], v[132:135]\n\t" \
  "ds_read_b64_tr_b8 v[122:123], v82 offset:25600\n\t" \
  "v_mfma_f32_16x16x128_f8f6f4 v[136:139], v[104:111], a[240:247], v[136:139]\n\t" \
  "ds_read_b64_tr_b8 v[124:125], v98 offset:24576\n\t" \
  "s_add_u32 m0, s24, 0x16000\n\t" \
  "s_add_u32 s50, s45, 0x400000\n\t" \
  "buffer_load_dwordx4 v2, s[8:11], s50 offen lds\n\t" \
  "v_mfma_f32_16x16x128_f8f6f4 v[140:143], v[104:111], a[248:255], v[140:143]\n\t" \
  "ds_read_b64_tr_b8 v[126:127], v98 offset:25600\n\t" \
  "s_waitcnt lgkmcnt(4)\n\t" \
  "v_mfma_f32_16x16x128_f8f6f4 v[156:159], v[112:119], a[248:255], v[156:159]\n\t" \
  "ds_read_b64_tr_b8 v[104:105], v83 offset:24576\n\t" \
  "s_add_u32 m0, s24, 0x16400\n\t" \
  "s_add_u32 s51, s45, 0x420000\n\t" \
  "buffer_load_dwordx4 v2, s[8:11], s51 offen lds\n\t" \
  "v_mfma_f32_16x16x128_f8f6f4 v[152:155], v[112:119], a[240:247], v[152:155]\n\t" \
  "ds_read_b64_tr_b8 v[106:107], v83 offset:25600\n\t" \
  "v_mfma_f32_16x16x128_f8f6f4 v[148:151], v[112:119], a[232:239], v[148:151]\n\t" \
  "ds_read_b64_tr_b8 v[108:109], v99 offset:24576\n\t" \
  "v_mfma_f32_16x16x128_f8f6f4 v[144:147], v[112:119], a[224:231], v[144:147]\n\t" \
  "ds_read_b64_tr_b8 v[110:111], v99 offset:25600\n\t" \
  "s_waitcnt lgkmcnt(4)\n\t" \
  "v_mfma_f32_16x16x128_f8f6f4 v[160:163], v[120:127], a[224:231], v[160:163]\n\t" \
  "ds_read_b64_tr_b8 v[112:113], v84 offset:24576\n\t" \
  "v_mfma_f32_16x16x128_f8f6f4 v[164:167], v[120:127], a[232:239], v[164:167]\n\t" \
  "ds_read_b64_tr_b8 v[114:115], v84 offset:25600\n\t" \
  "v_mfma_f32_16x16x128_f8f6f4 v[168:171], v[120:127], a[240:247], v[168:171]\n\t" \
  "ds_read_b64_tr_b8 v[116:117], v100 offset:24576\n\t" \
  "v_mfma_f32_16x16x128_f8f6f4 v[172:175], v[120:127], a[248:255], v[172:175]\n\t" \
  "ds_read_b64_tr_b8 v[118:119], v100 offset:25600\n\t" \
  "s_waitcnt lgkmcnt(4)\n\t" \
  "v_mfma_f32_16x16x128_f8f6f4 v[188:191], v[104:111], a[248:255], v[188:191]\n\t" \
  "ds_read_b64_tr_b8 v[120:121], v85 offset:24576\n\t" \
  "v_mfma_f32_16x16x128_f8f6f4 v[184:187], v[104:111], a[240:247], v[184:187]\n\t" \
  "ds_read_b64_tr_b8 v[122:123], v85 offset:25600\n\t" \
  "v_mfma_f32_16x16x128_f8f6f4 v[180:183], v[104:111], a[232:239], v[180:183]\n\t" \
  "ds_read_b64_tr_b8 v[124:125], v101 offset:24576\n\t" \
  "v_mfma_f32_16x16x128_f8f6f4 v[176:179], v[104:111], a[224:231], v[176:179]\n\t" \
  "ds_read_b64_tr_b8 v[126:127], v101 offset:25600\n\t" \
  "s_waitcnt lgkmcnt(4)\n\t" \
  "v_mfma_f32_16x16x128_f8f6f4 v[192:195], v[112:119], a[224:231], v[192:195]\n\t" \
  "ds_read_b64_tr_b8 v[104:105], v86 offset:24576\n\t" \
  "v_mfma_f32_16x16x128_f8f6f4 v[196:199], v[112:119], a[232:239], v[196:199]\n\t" \
  "ds_read_b64_tr_b8 v[106:107], v86 offset:25600\n\t" \
  "v_mfma_f32_16x16x128_f8f6f4 v[200:203], v[112:119], a[240:247], v[200:203]\n\t" \
  "ds_read_b64_tr_b8 v[108:109], v102 offset:24576\n\t" \
  "v_mfma_f32_16x16x128_f8f6f4 v[204:207], v[112:119], a[248:255], v[204:207]\n\t" \
  "ds_read_b64_tr_b8 v[110:111], v102 offset:25600\n\t" \
  "s_waitcnt lgkmcnt(4)\n\t" \
  "v_mfma_f32_16x16x128_f8f6f4 v[220:223], v[120:127], a[248:255], v[220:223]\n\t" \
  "ds_read_b64_tr_b8 v[112:113], v87 offset:24576\n\t" \
  "s_add_u32 m0, s24, 0x1c000\n\t" \
  "s_add_u32 s52, s40, 0x400000\n\t" \
  "buffer_load_dwordx4 v3, s[8:11], s52 offen lds\n\t" \
  "v_mfma_f32_16x16x128_f8f6f4 v[216:219], v[120:127], a[240:247], v[216:219]\n\t" \
  "ds_read_b64_tr_b8 v[114:115], v87 offset:25600\n\t" \
  "v_mfma_f32_16x16x128_f8f6f4 v[212:215], v[120:127], a[232:239], v[212:215]\n\t" \
  "ds_read_b64_tr_b8 v[116:117], v103 offset:24576\n\t" \
  "s_add_u32 m0, s24, 0x1c400\n\t" \
  "s_add_u32 s53, s40, 0x420000\n\t" \
  "buffer_load_dwordx4 v3, s[8:11], s53 offen lds\n\t" \
  "v_mfma_f32_16x16x128_f8f6f4 v[208:211], v[120:127], a[224:231], v[208:211]\n\t" \
  "ds_read_b64_tr_b8 v[118:119], v103 offset:25600\n\t" \
  "s_waitcnt vmcnt(29) lgkmcnt(0)\n\t" \
  "s_barrier\n\t" \
  "v_mfma_f32_16x16x128_f8f6f4 v[224:227], v[104:111], a[224:231], v[224:227]\n\t" \
  "ds_read_b64_tr_b8 v[120:121], v80 offset:32768\n\t" \
  "s_waitcnt vmcnt(20)\n\t" \
  "v_min_u32 v68, v68, v69\n\t" \
  "v_min3_u32 v68, v68, v70, v71\n\t" \
  "v_cmp_le_u32 vcc, s57, v68\n\t" \
  "s_and_b32 s64, vcc_lo, 0x7f0000\n\t" \
  "s_or_b32 s56, s56, s64\n\t" \
  "s_and_b32 s64, s56, 0x7f0000\n\t" \
  "s_cmp_eq_u32 s64, 0x7f0000\n\t" \
  "s_cbranch_scc1 Lok8_2_%=\n\t" \
  "s_mov_b32 s62, 0x7f0000\n\t" \
  "s_mov_b32 s69, 0x7f0000\n\t" \
  "s_call_b64 s[60:61], Lpoll_2_%=\n\t" \
  "Lok8_2_%=:\n\t" \
  "s_add_u32 s54, s68, 0x120000\n\t" \
  "buffer_load_dwordx4 a[64:67], v4, s[12:15], s54 offen offset:0 sc0 sc1\n\t" \
  "v_mfma_f32_16x16x128_f8f6f4 v[228:231], v[104:111], a[232:239], v[228:231]\n\t" \
  "ds_read_b64_tr_b8 v[122:123], v80 offset:33792\n\t" \
  "v_mfma_f32_16x16x128_f8f6f4 v[232:235], v[104:111], a[240:247], v[232:235]\n\t" \
  "ds_read_b64_tr_b8 v[124:125], v88 offset:32768\n\t" \
  "s_add_u32 s55, s68, 0x120000\n\t" \
  "buffer_load_dwordx4 a[68:71], v4, s[12:15], s55 offen offset:1024 sc0 sc1\n\t" \
  "v_mfma_f32_16x16x128_f8f6f4 v[236:239], v[104:111], a[248:255], v[236:239]\n\t" \
  "ds_read_b64_tr_b8 v[126:127], v88 offset:33792\n\t" \
  "v_mfma_f32_16x16x128_f8f6f4 v[252:255], v[112:119], a[248:255], v[252:255]\n\t" \
  "ds_read_b64_tr_b8 v[104:105], v81 offset:32768\n\t" \
  "s_add_u32 s48, s68, 0x120000\n\t" \
  "buffer_load_dwordx4 a[72:75], v4, s[12:15], s48 offen offset:2048 sc0 sc1\n\t" \
  "v_mfma_f32_16x16x128_f8f6f4 v[248:251], v[112:119], a[240:247], v[248:251]\n\t" \
  "ds_read_b64_tr_b8 v[106:107], v81 offset:33792\n\t" \
  "v_mfma_f32_16x16x128_f8f6f4 v[244:247], v[112:119], a[232:239], v[244:247]\n\t" \
  "ds_read_b64_tr_b8 v[108:109], v89 offset:32768\n\t" \
  "s_add_u32 s49, s68, 0x120000\n\t" \
  "buffer_load_dwordx4 a[76:79], v4, s[12:15], s49 offen offset:3072 sc0 sc1\n\t" \
  "v_mfma_f32_16x16x128_f8f6f4 v[240:243], v[112:119], a[224:231], v[240:243]\n\t" \
  "ds_read_b64_tr_b8 v[110:111], v89 offset:33792\n\t" \
  "s_waitcnt lgkmcnt(4)\n\t" \
  "v_mfma_f32_16x16x128_f8f6f4 v[128:131], v[120:127], a[0:7], v[128:131]\n\t" \
  "ds_read_b64_tr_b8 v[112:113], v82 offset:32768\n\t" \
  "s_add_u32 s50, s68, 0x121000\n\t" \
  "buffer_load_dwordx4 a[80:83], v4, s[12:15], s50 offen offset:0 sc0 sc1\n\t" \
  "v_mfma_f32_16x16x128_f8f6f4 v[132:135], v[120:127], a[8:15], v[132:135]\n\t" \
  "ds_read_b64_tr_b8 v[114:115], v82 offset:33792\n\t" \
  "v_mfma_f32_16x16x128_f8f6f4 v[136:139], v[120:127], a[16:23], v[136:139]\n\t" \
  "ds_read_b64_tr_b8 v[116:117], v90 offset:32768\n\t" \
  "s_add_u32 s51, s68, 0x121000\n\t" \
  "buffer_load_dwordx4 a[84:87], v4, s[12:15], s51 offen offset:1024 sc0 sc1\n\t" \
  "v_mfma_f32_16x16x128_f8f6f4 v[140:143], v[120:127], a[24:31], v[140:143]\n\t" \
  "ds_read_b64_tr_b8 v[118:119], v90 offset:33792\n\t" \
  "s_waitcnt lgkmcnt(4)\n\t" \
  "v_mfma_f32_16x16x128_f8f6f4 v[156:159], v[104:111], a[24:31], v[156:159]\n\t" \
  "ds_read_b64_tr_b8 v[120:121], v83 offset:32768\n\t" \
  "s_add_u32 s52, s68, 0x121000\n\t" \
  "buffer_load_dwordx4 a[88:91], v4, s[12:15], s52 offen offset:2048 sc0 sc1\n\t" \
  "v_mfma_f32_16x16x128_f8f6f4 v[152:155], v[104:111], a[16:23], v[152:155]\n\t" \
  "ds_read_b64_tr_b8 v[122:123], v83 offset:33792\n\t" \
  "v_mfma_f32_16x16x128_f8f6f4 v[148:151], v[104:111], a[8:15], v[148:151]\n\t" \
  "ds_read_b64_tr_b8 v[124:125], v91 offset:32768\n\t" \
  "s_add_u32 s53, s68, 0x121000\n\t" \
  "buffer_load_dwordx4 a[92:95], v4, s[12:15], s53 offen offset:3072 sc0 sc1\n\t" \
  "v_mfma_f32_16x16x128_f8f6f4 v[144:147], v[104:111], a[0:7], v[144:147]\n\t" \
  "ds_read_b64_tr_b8 v[126:127], v91 offset:33792\n\t" \
  "s_waitcnt lgkmcnt(4)\n\t" \
  "v_mfma_f32_16x16x128_f8f6f4 v[160:163], v[112:119], a[0:7], v[160:163]\n\t" \
  "ds_read_b64_tr_b8 v[104:105], v84 offset:32768\n\t" \
  "s_add_u32 m0, s24, 0x1e000\n\t" \
  "s_add_u32 s54, s41, 0x400000\n\t" \
  "buffer_load_dwordx4 v3, s[8:11], s54 offen lds\n\t" \
  "v_mfma_f32_16x16x128_f8f6f4 v[164:167], v[112:119], a[8:15], v[164:167]\n\t" \
  "ds_read_b64_tr_b8 v[106:107], v84 offset:33792\n\t" \
  "v_mfma_f32_16x16x128_f8f6f4 v[168:171], v[112:119], a[16:23], v[168:171]\n\t" \
  "ds_read_b64_tr_b8 v[108:109], v92 offset:32768\n\t" \
  "s_add_u32 m0, s24, 0x1e400\n\t" \
  "s_add_u32 s55, s41, 0x420000\n\t" \
  "buffer_load_dwordx4 v3, s[8:11], s55 offen lds\n\t" \
  "v_mfma_f32_16x16x128_f8f6f4 v[172:175], v[112:119], a[24:31], v[172:175]\n\t" \
  "ds_read_b64_tr_b8 v[110:111], v92 offset:33792\n\t" \
  "s_waitcnt lgkmcnt(4)\n\t" \
  "v_mfma_f32_16x16x128_f8f6f4 v[188:191], v[120:127], a[24:31], v[188:191]\n\t" \
  "ds_read_b64_tr_b8 v[112:113], v85 offset:32768\n\t" \
  "s_add_u32 s48, s68, 0x130000\n\t" \
  "buffer_load_dwordx4 a[96:99], v4, s[12:15], s48 offen offset:0 sc0 sc1\n\t" \
  "v_mfma_f32_16x16x128_f8f6f4 v[184:187], v[120:127], a[16:23], v[184:187]\n\t" \
  "ds_read_b64_tr_b8 v[114:115], v85 offset:33792\n\t" \
  "v_mfma_f32_16x16x128_f8f6f4 v[180:183], v[120:127], a[8:15], v[180:183]\n\t" \
  "ds_read_b64_tr_b8 v[116:117], v93 offset:32768\n\t" \
  "s_add_u32 s49, s68, 0x130000\n\t" \
  "buffer_load_dwordx4 a[100:103], v4, s[12:15], s49 offen offset:1024 sc0 sc1\n\t" \
  "v_mfma_f32_16x16x128_f8f6f4 v[176:179], v[120:127], a[0:7], v[176:179]\n\t" \
  "ds_read_b64_tr_b8 v[118:119], v93 offset:33792\n\t" \
  "s_waitcnt lgkmcnt(4)\n\t" \
  "v_mfma_f32_16x16x128_f8f6f4 v[192:195], v[104:111], a[0:7], v[192:195]\n\t" \
  "ds_read_b64_tr_b8 v[120:121], v86 offset:32768\n\t" \
  "s_add_u32 s50, s68, 0x130000\n\t" \
  "buffer_load_dwordx4 a[104:107], v4, s[12:15], s50 offen offset:2048 sc0 sc1\n\t" \
  "v_mfma_f32_16x16x128_f8f6f4 v[196:199], v[104:111], a[8:15], v[196:199]\n\t" \
  "ds_read_b64_tr_b8 v[122:123], v86 offset:33792\n\t" \
  "v_mfma_f32_16x16x128_f8f6f4 v[200:203], v[104:111], a[16:23], v[200:203]\n\t" \
  "ds_read_b64_tr_b8 v[124:125], v94 offset:32768\n\t" \
  "s_add_u32 s51, s68, 0x130000\n\t" \
  "buffer_load_dwordx4 a[108:111], v4, s[12:15], s51 offen offset:3072 sc0 sc1\n\t" \
  "v_mfma_f32_16x16x128_f8f6f4 v[204:207], v[104:111], a[24:31], v[204:207]\n\t" \
  "ds_read_b64_tr_b8 v[126:127], v94 offset:33792\n\t" \
  "s_waitcnt lgkmcnt(4)\n\t" \
  "v_mfma_f32_16x16x128_f8f6f4 v[220:223], v[112:119], a[24:31], v[220:223]\n\t" \
  "ds_read_b64_tr_b8 v[104:105], v87 offset:32768\n\t" \
  "s_add_u32 s52, s68, 0x131000\n\t" \
  "buffer_load_dwordx4 a[112:115], v4, s[12:15], s52 offen offset:0 sc0 sc1\n\t" \
  "v_mfma_f32_16x16x128_f8f6f4 v[216:219], v[112:119], a[16:23], v[216:219]\n\t" \
  "ds_read_b64_tr_b8 v[106:107], v87 offset:33792\n\t" \
  "v_mfma_f32_16x16x128_f8f6f4 v[212:215], v[112:119], a[8:15], v[212:215]\n\t" \
  "ds_read_b64_tr_b8 v[108:109], v95 offset:32768\n\t" \
  "s_add_u32 s53, s68, 0x131000\n\t" \
  "buffer_load_dwordx4 a[116:119], v4, s[12:15], s53 offen offset:1024 sc0 sc1\n\t" \
  "v_mfma_f32_16x16x128_f8f6f4 v[208:211], v[112:119], a[0:7], v[208:211]\n\t" \
  "ds_read_b64_tr_b8 v[110:111], v95 offset:33792\n\t" \
  "s_waitcnt vmcnt(34) lgkmcnt(0)\n\t" \
  "s_barrier\n\t" \
  "v_mfma_f32_16x16x128_f8f6f4 v[224:227], v[120:127], a[0:7], v[224:227]\n\t" \
  "ds_read_b64_tr_b8 v[112:113], v80 offset:40960\n\t" \
  "s_add_u32 s54, s68, 0x131000\n\t" \
  "buffer_load_dwordx4 a[120:123], v4, s[12:15], s54 offen offset:2048 sc0 sc1\n\t" \
  "v_mfma_f32_16x16x128_f8f6f4 v[228:231], v[120:127], a[8:15], v[228:231]\n\t" \
  "ds_read_b64_tr_b8 v[114:115], v80 offset:41984\n\t" \
  "v_mfma_f32_16x16x128_f8f6f4 v[232:235], v[120:127], a[16:23], v[232:235]\n\t" \
  "ds_read_b64_tr_b8 v[116:117], v88 offset:40960\n\t" \
  "s_add_u32 s55, s68, 0x131000\n\t" \
  "buffer_load_dwordx4 a[124:127], v4, s[12:15], s55 offen offset:3072 sc0 sc1\n\t" \
  "v_mfma_f32_16x16x128_f8f6f4 v[236:239], v[120:127], a[24:31], v[236:239]\n\t" \
  "ds_read_b64_tr_b8 v[118:119], v88 offset:41984\n\t" \
  "v_mfma_f32_16x16x128_f8f6f4 v[252:255], v[104:111], a[24:31], v[252:255]\n\t" \
  "ds_read_b64_tr_b8 v[120:121], v81 offset:40960\n\t" \
  "s_add_u32 m0, s24, 0x20000\n\t" \
  "s_add_u32 s48, s42, 0x400000\n\t" \
  "buffer_load_dwordx4 v3, s[8:11], s48 offen lds\n\t" \
  "v_mfma_f32_16x16x128_f8f6f4 v[248:251], v[104:111], a[16:23], v[248:251]\n\t" \
  "ds_read_b64_tr_b8 v[122:123], v81 offset:41984\n\t" \
  "v_mfma_f32_16x16x128_f8f6f4 v[244:247], v[104:111], a[8:15], v[244:247]\n\t" \
  "ds_read_b64_tr_b8 v[124:125], v89 offset:40960\n\t" \
  "s_add_u32 m0, s24, 0x20400\n\t" \
  "s_add_u32 s49, s42, 0x420000\n\t" \
  "buffer_load_dwordx4 v3, s[8:11], s49 offen lds\n\t" \
  "v_mfma_f32_16x16x128_f8f6f4 v[240:243], v[104:111], a[0:7], v[240:243]\n\t" \
  "ds_read_b64_tr_b8 v[126:127], v89 offset:41984\n\t" \
  "s_waitcnt vmcnt(30) lgkmcnt(4)\n\t" \
  "v_mfma_f32_16x16x128_f8f6f4 v[128:131], v[112:119], a[32:39], v[128:131]\n\t" \
  "ds_read_b64_tr_b8 v[104:105], v82 offset:40960\n\t" \
  "s_add_u32 s50, s68, 0x140000\n\t" \
  "buffer_load_dwordx4 a[128:131], v4, s[12:15], s50 offen offset:0 sc0 sc1\n\t" \
  "v_mfma_f32_16x16x128_f8f6f4 v[132:135], v[112:119], a[40:47], v[132:135]\n\t" \
  "ds_read_b64_tr_b8 v[106:107], v82 offset:41984\n\t" \
  "v_mfma_f32_16x16x128_f8f6f4 v[136:139], v[112:119], a[48:55], v[136:139]\n\t" \
  "ds_read_b64_tr_b8 v[108:109], v90 offset:40960\n\t" \
  "s_add_u32 s51, s68, 0x140000\n\t" \
  "buffer_load_dwordx4 a[132:135], v4, s[12:15], s51 offen offset:1024 sc0 sc1\n\t" \
  "v_mfma_f32_16x16x128_f8f6f4 v[140:143], v[112:119], a[56:63], v[140:143]\n\t" \
  "ds_read_b64_tr_b8 v[110:111], v90 offset:41984\n\t" \
  "s_waitcnt lgkmcnt(4)\n\t" \
  "v_mfma_f32_16x16x128_f8f6f4 v[156:159], v[120:127], a[56:63], v[156:159]\n\t" \
  "ds_read_b64_tr_b8 v[112:113], v83 offset:40960\n\t" \
  "s_add_u32 s52, s68, 0x140000\n\t" \
  "buffer_load_dwordx4 a[136:139], v4, s[12:15], s52 offen offset:2048 sc0 sc1\n\t" \
  "v_mfma_f32_16x16x128_f8f6f4 v[152:155], v[120:127], a[48:55], v[152:155]\n\t" \
  "ds_read_b64_tr_b8 v[114:115], v83 offset:41984\n\t" \
  "v_mfma_f32_16x16x128_f8f6f4 v[148:151], v[120:127], a[40:47], v[148:151]\n\t" \
  "ds_read_b64_tr_b8 v[116:117], v91 offset:40960\n\t" \
  "s_add_u32 s53, s68, 0x140000\n\t" \
  "buffer_load_dwordx4 a[140:143], v4, s[12:15], s53 offen offset:3072 sc0 sc1\n\t" \
  "v_mfma_f32_16x16x128_f8f6f4 v[144:147], v[120:127], a[32:39], v[144:147]\n\t" \
  "ds_read_b64_tr_b8 v[118:119], v91 offset:41984\n\t" \
  "s_waitcnt lgkmcnt(4)\n\t" \
  "v_mfma_f32_16x16x128_f8f6f4 v[160:163], v[104:111], a[32:39], v[160:163]\n\t" \
  "ds_read_b64_tr_b8 v[120:121], v84 offset:40960\n\t" \
  "s_add_u32 s54, s68, 0x141000\n\t" \
  "buffer_load_dwordx4 a[144:147], v4, s[12:15], s54 offen offset:0 sc0 sc1\n\t" \
  "v_mfma_f32_16x16x128_f8f6f4 v[164:167], v[104:111], a[40:47], v[164:167]\n\t" \
  "ds_read_b64_tr_b8 v[122:123], v84 offset:41984\n\t" \
  "v_mfma_f32_16x16x128_f8f6f4 v[168:171], v[104:111], a[48:55], v[168:171]\n\t" \
  "ds_read_b64_tr_b8 v[124:125], v92 offset:40960\n\t" \
  "s_add_u32 s55, s68, 0x141000\n\t" \
  "buffer_load_dwordx4 a[148:151], v4, s[12:15], s55 offen offset:1024 sc0 sc1\n\t" \
  "v_mfma_f32_16x16x128_f8f6f4 v[172:175], v[104:111], a[56:63], v[172:175]\n\t" \
  "ds_read_b64_tr_b8 v[126:127], v92 offset:41984\n\t" \
  "s_waitcnt lgkmcnt(4)\n\t" \
  "v_mfma_f32_16x16x128_f8f6f4 v[188:191], v[112:119], a[56:63], v[188:191]\n\t" \
  "ds_read_b64_tr_b8 v[104:105], v85 offset:40960\n\t" \
  "s_add_u32 s48, s68, 0x141000\n\t" \
  "buffer_load_dwordx4 a[152:155], v4, s[12:15], s48 offen offset:2048 sc0 sc1\n\t" \
  "v_mfma_f32_16x16x128_f8f6f4 v[184:187], v[112:119], a[48:55], v[184:187]\n\t" \
  "ds_read_b64_tr_b8 v[106:107], v85 offset:41984\n\t" \
  "v_mfma_f32_16x16x128_f8f6f4 v[180:183], v[112:119], a[40:47], v[180:183]\n\t" \
  "ds_read_b64_tr_b8 v[108:109], v93 offset:40960\n\t" \
  "s_add_u32 s49, s68, 0x141000\n\t" \
  "buffer_load_dwordx4 a[156:159], v4, s[12:15], s49 offen offset:3072 sc0 sc1\n\t" \
  "v_mfma_f32_16x16x128_f8f6f4 v[176:179], v[112:119], a[32:39], v[176:179]\n\t" \
  "ds_read_b64_tr_b8 v[110:111], v93 offset:41984\n\t" \
  "s_waitcnt lgkmcnt(4)\n\t" \
  "v_mfma_f32_16x16x128_f8f6f4 v[192:195], v[120:127], a[32:39], v[192:195]\n\t" \
  "ds_read_b64_tr_b8 v[112:113], v86 offset:40960\n\t" \
  "s_add_u32 m0, s24, 0x22000\n\t" \
  "s_add_u32 s50, s43, 0x400000\n\t" \
  "buffer_load_dwordx4 v3, s[8:11], s50 offen lds\n\t" \
  "v_mfma_f32_16x16x128_f8f6f4 v[196:199], v[120:127], a[40:47], v[196:199]\n\t" \
  "ds_read_b64_tr_b8 v[114:115], v86 offset:41984\n\t" \
  "v_mfma_f32_16x16x128_f8f6f4 v[200:203], v[120:127], a[48:55], v[200:203]\n\t" \
  "ds_read_b64_tr_b8 v[116:117], v94 offset:40960\n\t" \
  "s_add_u32 m0, s24, 0x22400\n\t" \
  "s_add_u32 s51, s43, 0x420000\n\t" \
  "buffer_load_dwordx4 v3, s[8:11], s51 offen lds\n\t" \
  "v_mfma_f32_16x16x128_f8f6f4 v[204:207], v[120:127], a[56:63], v[204:207]\n\t" \
  "ds_read_b64_tr_b8 v[118:119], v94 offset:41984\n\t" \
  "s_waitcnt lgkmcnt(4)\n\t" \
  "v_mfma_f32_16x16x128_f8f6f4 v[220:223], v[104:111], a[56:63], v[220:223]\n\t" \
  "ds_read_b64_tr_b8 v[120:121], v87 offset:40960\n\t" \
  "s_add_u32 s52, s68, 0x150000\n\t" \
  "buffer_load_dwordx4 a[160:163], v4, s[12:15], s52 offen offset:0 sc0 sc1\n\t" \
  "v_mfma_f32_16x16x128_f8f6f4 v[216:219], v[104:111], a[48:55], v[216:219]\n\t" \
  "ds_read_b64_tr_b8 v[122:123], v87 offset:41984\n\t" \
  "v_mfma_f32_16x16x128_f8f6f4 v[212:215], v[104:111], a[40:47], v[212:215]\n\t" \
  "ds_read_b64_tr_b8 v[124:125], v95 offset:40960\n\t" \
  "s_add_u32 s53, s68, 0x150000\n\t" \
  "buffer_load_dwordx4 a[164:167], v4, s[12:15], s53 offen offset:1024 sc0 sc1\n\t" \
  "v_mfma_f32_16x16x128_f8f6f4 v[208:211], v[104:111], a[32:39], v[208:211]\n\t" \
  "ds_read_b64_tr_b8 v[126:127], v95 offset:41984\n\t" \
  "s_waitcnt vmcnt(32) lgkmcnt(0)\n\t" \
  "s_barrier\n\t" \
  "v_mfma_f32_16x16x128_f8f6f4 v[224:227], v[112:119], a[32:39], v[224:227]\n\t" \
  "ds_read_b64_tr_b8 v[104:105], v80 offset:49152\n\t" \
  "s_add_u32 s54, s68, 0x150000\n\t" \
  "buffer_load_dwordx4 a[168:171], v4, s[12:15], s54 offen offset:2048 sc0 sc1\n\t" \
  "v_mfma_f32_16x16x128_f8f6f4 v[228:231], v[112:119], a[40:47], v[228:231]\n\t" \
  "ds_read_b64_tr_b8 v[106:107], v80 offset:50176\n\t" \
  "v_mfma_f32_16x16x128_f8f6f4 v[232:235], v[112:119], a[48:55], v[232:235]\n\t" \
  "ds_read_b64_tr_b8 v[108:109], v88 offset:49152\n\t" \
  "s_add_u32 s55, s68, 0x150000\n\t" \
  "buffer_load_dwordx4 a[172:175], v4, s[12:15], s55 offen offset:3072 sc0 sc1\n\t" \
  "v_mfma_f32_16x16x128_f8f6f4 v[236:239], v[112:119], a[56:63], v[236:239]\n\t" \
  "ds_read_b64_tr_b8 v[110:111], v88 offset:50176\n\t" \
  "v_mfma_f32_16x16x128_f8f6f4 v[252:255], v[120:127], a[56:63], v[252:255]\n\t" \
  "ds_read_b64_tr_b8 v[112:113], v81 offset:49152\n\t" \
  "s_add_u32 s48, s68, 0x151000\n\t" \
  "buffer_load_dwordx4 a[176:179], v4, s[12:15], s48 offen offset:0 sc0 sc1\n\t" \
  "v_mfma_f32_16x16x128_f8f6f4 v[248:251], v[120:127], a[48:55], v[248:251]\n\t" \
  "ds_read_b64_tr_b8 v[114:115], v81 offset:50176\n\t" \
  "v_mfma_f32_16x16x128_f8f6f4 v[244:247], v[120:127], a[40:47], v[244:247]\n\t" \
  "ds_read_b64_tr_b8 v[116:117], v89 offset:49152\n\t" \
  "s_add_u32 s49, s68, 0x151000\n\t" \
  "buffer_load_dwordx4 a[180:183], v4, s[12:15], s49 offen offset:1024 sc0 sc1\n\t" \
  "v_mfma_f32_16x16x128_f8f6f4 v[240:243], v[120:127], a[32:39], v[240:243]\n\t" \
  "ds_read_b64_tr_b8 v[118:119], v89 offset:50176\n\t" \
  "s_waitcnt vmcnt(28) lgkmcnt(4)\n\t" \
  "v_mfma_f32_16x16x128_f8f6f4 v[128:131], v[104:111], a[64:71], v[128:131]\n\t" \
  "ds_read_b64_tr_b8 v[120:121], v82 offset:49152\n\t" \
  "s_add_u32 s50, s68, 0x151000\n\t" \
  "buffer_load_dwordx4 a[184:187], v4, s[12:15], s50 offen offset:2048 sc0 sc1\n\t" \
  "v_mfma_f32_16x16x128_f8f6f4 v[132:135], v[104:111], a[72:79], v[132:135]\n\t" \
  "ds_read_b64_tr_b8 v[122:123], v82 offset:50176\n\t" \
  "v_mfma_f32_16x16x128_f8f6f4 v[136:139], v[104:111], a[80:87], v[136:139]\n\t" \
  "ds_read_b64_tr_b8 v[124:125], v90 offset:49152\n\t" \
  "s_add_u32 s51, s68, 0x151000\n\t" \
  "buffer_load_dwordx4 a[188:191], v4, s[12:15], s51 offen offset:3072 sc0 sc1\n\t" \
  "v_mfma_f32_16x16x128_f8f6f4 v[140:143], v[104:111], a[88:95], v[140:143]\n\t" \
  "ds_read_b64_tr_b8 v[126:127], v90 offset:50176\n\t" \
  "s_waitcnt lgkmcnt(4)\n\t" \
  "v_mfma_f32_16x16x128_f8f6f4 v[156:159], v[112:119], a[88:95], v[156:159]\n\t" \
  "ds_read_b64_tr_b8 v[104:105], v83 offset:49152\n\t" \
  "s_add_u32 m0, s24, 0x0\n\t" \
  "s_add_u32 s52, s46, 0x400000\n\t" \
  "buffer_load_dwordx4 v2, s[8:11], s52 offen lds\n\t" \
  "v_mfma_f32_16x16x128_f8f6f4 v[152:155], v[112:119], a[80:87], v[152:155]\n\t" \
  "ds_read_b64_tr_b8 v[106:107], v83 offset:50176\n\t" \
  "v_mfma_f32_16x16x128_f8f6f4 v[148:151], v[112:119], a[72:79], v[148:151]\n\t" \
  "ds_read_b64_tr_b8 v[108:109], v91 offset:49152\n\t" \
  "s_add_u32 m0, s24, 0x400\n\t" \
  "s_add_u32 s53, s46, 0x420000\n\t" \
  "buffer_load_dwordx4 v2, s[8:11], s53 offen lds\n\t" \
  "v_mfma_f32_16x16x128_f8f6f4 v[144:147], v[112:119], a[64:71], v[144:147]\n\t" \
  "ds_read_b64_tr_b8 v[110:111], v91 offset:50176\n\t" \
  "s_waitcnt lgkmcnt(4)\n\t" \
  "v_mfma_f32_16x16x128_f8f6f4 v[160:163], v[120:127], a[64:71], v[160:163]\n\t" \
  "ds_read_b64_tr_b8 v[112:113], v84 offset:49152\n\t" \
  "s_add_u32 m0, s24, 0x2000\n\t" \
  "s_add_u32 s54, s40, 0x600000\n\t" \
  "buffer_load_dwordx4 v2, s[8:11], s54 offen lds\n\t" \
  "v_mfma_f32_16x16x128_f8f6f4 v[164:167], v[120:127], a[72:79], v[164:167]\n\t" \
  "ds_read_b64_tr_b8 v[114:115], v84 offset:50176\n\t" \
  "v_mfma_f32_16x16x128_f8f6f4 v[168:171], v[120:127], a[80:87], v[168:171]\n\t" \
  "ds_read_b64_tr_b8 v[116:117], v92 offset:49152\n\t" \
  "s_add_u32 m0, s24, 0x2400\n\t" \
  "s_add_u32 s55, s40, 0x620000\n\t" \
  "buffer_load_dwordx4 v2, s[8:11], s55 offen lds\n\t" \
  "v_mfma_f32_16x16x128_f8f6f4 v[172:175], v[120:127], a[88:95], v[172:175]\n\t" \
  "ds_read_b64_tr_b8 v[118:119], v92 offset:50176\n\t" \
  "s_waitcnt lgkmcnt(4)\n\t" \
  "v_mfma_f32_16x16x128_f8f6f4 v[188:191], v[104:111], a[88:95], v[188:191]\n\t" \
  "ds_read_b64_tr_b8 v[120:121], v85 offset:49152\n\t" \
  "s_add_u32 m0, s24, 0x4000\n\t" \
  "s_add_u32 s48, s41, 0x600000\n\t" \
  "buffer_load_dwordx4 v2, s[8:11], s48 offen lds\n\t" \
  "v_mfma_f32_16x16x128_f8f6f4 v[184:187], v[104:111], a[80:87], v[184:187]\n\t" \
  "ds_read_b64_tr_b8 v[122:123], v85 offset:50176\n\t" \
  "v_mfma_f32_16x16x128_f8f6f4 v[180:183], v[104:111], a[72:79], v[180:183]\n\t" \
  "ds_read_b64_tr_b8 v[124:125], v93 offset:49152\n\t" \
  "s_add_u32 m0, s24, 0x4400\n\t" \
  "s_add_u32 s49, s41, 0x620000\n\t" \
  "buffer_load_dwordx4 v2, s[8:11], s49 offen lds\n\t" \
  "v_mfma_f32_16x16x128_f8f6f4 v[176:179], v[104:111], a[64:71], v[176:179]\n\t" \
  "ds_read_b64_tr_b8 v[126:127], v93 offset:50176\n\t" \
  "s_waitcnt lgkmcnt(4)\n\t" \
  "v_mfma_f32_16x16x128_f8f6f4 v[192:195], v[112:119], a[64:71], v[192:195]\n\t" \
  "ds_read_b64_tr_b8 v[104:105], v86 offset:49152\n\t" \
  "s_add_u32 m0, s24, 0x6000\n\t" \
  "s_add_u32 s50, s42, 0x600000\n\t" \
  "buffer_load_dwordx4 v2, s[8:11], s50 offen lds\n\t" \
  "v_mfma_f32_16x16x128_f8f6f4 v[196:199], v[112:119], a[72:79], v[196:199]\n\t" \
  "ds_read_b64_tr_b8 v[106:107], v86 offset:50176\n\t" \
  "v_mfma_f32_16x16x128_f8f6f4 v[200:203], v[112:119], a[80:87], v[200:203]\n\t" \
  "ds_read_b64_tr_b8 v[108:109], v94 offset:49152\n\t" \
  "s_add_u32 m0, s24, 0x6400\n\t" \
  "s_add_u32 s51, s42, 0x620000\n\t" \
  "buffer_load_dwordx4 v2, s[8:11], s51 offen lds\n\t" \
  "v_mfma_f32_16x16x128_f8f6f4 v[204:207], v[112:119], a[88:95], v[204:207]\n\t" \
  "ds_read_b64_tr_b8 v[110:111], v94 offset:50176\n\t" \
  "s_waitcnt lgkmcnt(4)\n\t" \
  "v_mfma_f32_16x16x128_f8f6f4 v[220:223], v[120:127], a[88:95], v[220:223]\n\t" \
  "ds_read_b64_tr_b8 v[112:113], v87 offset:49152\n\t" \
  "v_mfma_f32_16x16x128_f8f6f4 v[216:219], v[120:127], a[80:87], v[216:219]\n\t" \
  "ds_read_b64_tr_b8 v[114:115], v87 offset:50176\n\t" \
  "v_mfma_f32_16x16x128_f8f6f4 v[212:215], v[120:127], a[72:79], v[212:215]\n\t" \
  "ds_read_b64_tr_b8 v[116:117], v95 offset:49152\n\t" \
  "v_mfma_f32_16x16x128_f8f6f4 v[208:211], v[120:127], a[64:71], v[208:211]\n\t" \
  "ds_read_b64_tr_b8 v[118:119], v95 offset:50176\n\t" \
  "s_waitcnt vmcnt(36) lgkmcnt(0)\n\t" \
  "s_barrier\n\t" \
  "v_mfma_f32_16x16x128_f8f6f4 v[224:227], v[104:111], a[64:71], v[224:227]\n\t" \
  "ds_read_b64_tr_b8 v[120:121], v80 offset:57344\n\t" \
  "v_mfma_f32_16x16x128_f8f6f4 v[228:231], v[104:111], a[72:79], v[228:231]\n\t" \
  "ds_read_b64_tr_b8 v[122:123], v80 offset:58368\n\t" \
  "v_mfma_f32_16x16x128_f8f6f4 v[232:235], v[104:111], a[80:87], v[232:235]\n\t" \
  "ds_read_b64_tr_b8 v[124:125], v88 offset:57344\n\t" \
  "v_mfma_f32_16x16x128_f8f6f4 v[236:239], v[104:111], a[88:95], v[236:239]\n\t" \
  "ds_read_b64_tr_b8 v[126:127], v88 offset:58368\n\t" \
  "v_mfma_f32_16x16x128_f8f6f4 v[252:255], v[112:119], a[88:95], v[252:255]\n\t" \
  "ds_read_b64_tr_b8 v[104:105], v81 offset:57344\n\t" \
  "v_mfma_f32_16x16x128_f8f6f4 v[248:251], v[112:119], a[80:87], v[248:251]\n\t" \
  "ds_read_b64_tr_b8 v[106:107], v81 offset:58368\n\t" \
  "v_mfma_f32_16x16x128_f8f6f4 v[244:247], v[112:119], a[72:79], v[244:247]\n\t" \
  "ds_read_b64_tr_b8 v[108:109], v89 offset:57344\n\t" \
  "v_mfma_f32_16x16x128_f8f6f4 v[240:243], v[112:119], a[64:71], v[240:243]\n\t" \
  "ds_read_b64_tr_b8 v[110:111], v89 offset:58368\n\t" \
  "s_waitcnt vmcnt(28) lgkmcnt(4)\n\t" \
  "v_mfma_f32_16x16x128_f8f6f4 v[128:131], v[120:127], a[96:103], v[128:131]\n\t" \
  "ds_read_b64_tr_b8 v[112:113], v82 offset:57344\n\t" \
  "v_mfma_f32_16x16x128_f8f6f4 v[132:135], v[120:127], a[104:111], v[132:135]\n\t" \
  "ds_read_b64_tr_b8 v[114:115], v82 offset:58368\n\t" \
  "v_mfma_f32_16x16x128_f8f6f4 v[136:139], v[120:127], a[112:119], v[136:139]\n\t" \
  "ds_read_b64_tr_b8 v[116:117], v90 offset:57344\n\t" \
  "s_add_u32 m0, s24, 0x24000\n\t" \
  "s_add_u32 s52, s44, 0x400000\n\t" \
  "buffer_load_dwordx4 v3, s[8:11], s52 offen lds\n\t" \
  "v_mfma_f32_16x16x128_f8f6f4 v[140:143], v[120:127], a[120:127], v[140:143]\n\t" \
  "ds_read_b64_tr_b8 v[118:119], v90 offset:58368\n\t" \
  "s_waitcnt lgkmcnt(4)\n\t" \
  "v_mfma_f32_16x16x128_f8f6f4 v[156:159], v[104:111], a[120:127], v[156:159]\n\t" \
  "ds_read_b64_tr_b8 v[120:121], v83 offset:57344\n\t" \
  "s_add_u32 m0, s24, 0x24400\n\t" \
  "s_add_u32 s53, s44, 0x420000\n\t" \
  "buffer_load_dwordx4 v3, s[8:11], s53 offen lds\n\t" \
  "v_mfma_f32_16x16x128_f8f6f4 v[152:155], v[104:111], a[112:119], v[152:155]\n\t" \
  "ds_read_b64_tr_b8 v[122:123], v83 offset:58368\n\t" \
  "v_mfma_f32_16x16x128_f8f6f4 v[148:151], v[104:111], a[104:111], v[148:151]\n\t" \
  "ds_read_b64_tr_b8 v[124:125], v91 offset:57344\n\t" \
  "s_add_u32 s54, s68, 0x160000\n\t" \
  "buffer_load_dwordx4 a[192:195], v4, s[12:15], s54 offen offset:0 sc0 sc1\n\t" \
  "v_mfma_f32_16x16x128_f8f6f4 v[144:147], v[104:111], a[96:103], v[144:147]\n\t" \
  "ds_read_b64_tr_b8 v[126:127], v91 offset:58368\n\t" \
  "s_waitcnt lgkmcnt(4)\n\t" \
  "v_mfma_f32_16x16x128_f8f6f4 v[160:163], v[112:119], a[96:103], v[160:163]\n\t" \
  "ds_read_b64_tr_b8 v[104:105], v84 offset:57344\n\t" \
  "s_add_u32 s55, s68, 0x160000\n\t" \
  "buffer_load_dwordx4 a[196:199], v4, s[12:15], s55 offen offset:1024 sc0 sc1\n\t" \
  "v_mfma_f32_16x16x128_f8f6f4 v[164:167], v[112:119], a[104:111], v[164:167]\n\t" \
  "ds_read_b64_tr_b8 v[106:107], v84 offset:58368\n\t" \
  "v_mfma_f32_16x16x128_f8f6f4 v[168:171], v[112:119], a[112:119], v[168:171]\n\t" \
  "ds_read_b64_tr_b8 v[108:109], v92 offset:57344\n\t" \
  "s_add_u32 s48, s68, 0x160000\n\t" \
  "buffer_load_dwordx4 a[200:203], v4, s[12:15], s48 offen offset:2048 sc0 sc1\n\t" \
  "v_mfma_f32_16x16x128_f8f6f4 v[172:175], v[112:119], a[120:127], v[172:175]\n\t" \
  "ds_read_b64_tr_b8 v[110:111], v92 offset:58368\n\t" \
  "s_waitcnt lgkmcnt(4)\n\t" \
  "v_mfma_f32_16x16x128_f8f6f4 v[188:191], v[120:127], a[120:127], v[188:191]\n\t" \
  "ds_read_b64_tr_b8 v[112:113], v85 offset:57344\n\t" \
  "s_add_u32 s49, s68, 0x160000\n\t" \
  "buffer_load_dwordx4 a[204:207], v4, s[12:15], s49 offen offset:3072 sc0 sc1\n\t" \
  "v_mfma_f32_16x16x128_f8f6f4 v[184:187], v[120:127], a[112:119], v[184:187]\n\t" \
  "ds_read_b64_tr_b8 v[114:115], v85 offset:58368\n\t" \
  "v_mfma_f32_16x16x128_f8f6f4 v[180:183], v[120:127], a[104:111], v[180:183]\n\t" \
  "ds_read_b64_tr_b8 v[116:117], v93 offset:57344\n\t" \
  "s_add_u32 s50, s68, 0x161000\n\t" \
  "buffer_load_dwordx4 a[208:211], v4, s[12:15], s50 offen offset:0 sc0 sc1\n\t" \
  "v_mfma_f32_16x16x128_f8f6f4 v[176:179], v[120:127], a[96:103], v[176:179]\n\t" \
  "ds_read_b64_tr_b8 v[118:119], v93 offset:58368\n\t" \
  "s_waitcnt lgkmcnt(4)\n\t" \
  "v_mfma_f32_16x16x128_f8f6f4 v[192:195], v[104:111], a[96:103], v[192:195]\n\t" \
  "ds_read_b64_tr_b8 v[120:121], v86 offset:57344\n\t" \
  "s_add_u32 s51, s68, 0x161000\n\t" \
  "buffer_load_dwordx4 a[212:215], v4, s[12:15], s51 offen offset:1024 sc0 sc1\n\t" \
  "v_mfma_f32_16x16x128_f8f6f4 v[196:199], v[104:111], a[104:111], v[196:199]\n\t" \
  "ds_read_b64_tr_b8 v[122:123], v86 offset:58368\n\t" \
  "v_mfma_f32_16x16x128_f8f6f4 v[200:203], v[104:111], a[112:119], v[200:203]\n\t" \
  "ds_read_b64_tr_b8 v[124:125], v94 offset:57344\n\t" \
  "s_add_u32 s52, s68, 0x161000\n\t" \
  "buffer_load_dwordx4 a[216:219], v4, s[12:15], s52 offen offset:2048 sc0 sc1\n\t" \
  "v_mfma_f32_16x16x128_f8f6f4 v[204:207], v[104:111], a[120:127], v[204:207]\n\t" \
  "ds_read_b64_tr_b8 v[126:127], v94 offset:58368\n\t" \
  "s_waitcnt lgkmcnt(4)\n\t" \
  "v_mfma_f32_16x16x128_f8f6f4 v[220:223], v[112:119], a[120:127], v[220:223]\n\t" \
  "ds_read_b64_tr_b8 v[104:105], v87 offset:57344\n\t" \
  "s_add_u32 s53, s68, 0x161000\n\t" \
  "buffer_load_dwordx4 a[220:223], v4, s[12:15], s53 offen offset:3072 sc0 sc1\n\t" \
  "v_mfma_f32_16x16x128_f8f6f4 v[216:219], v[112:119], a[112:119], v[216:219]\n\t" \
  "ds_read_b64_tr_b8 v[106:107], v87 offset:58368\n\t" \
  "v_mfma_f32_16x16x128_f8f6f4 v[212:215], v[112:119], a[104:111], v[212:215]\n\t" \
  "ds_read_b64_tr_b8 v[108:109], v95 offset:57344\n\t" \
  "s_add_u32 m0, s24, 0x26000\n\t" \
  "s_add_u32 s54, s45, 0x400000\n\t" \
  "buffer_load_dwordx4 v3, s[8:11], s54 offen lds\n\t" \
  "v_mfma_f32_16x16x128_f8f6f4 v[208:211], v[112:119], a[96:103], v[208:211]\n\t" \
  "ds_read_b64_tr_b8 v[110:111], v95 offset:58368\n\t" \
  "s_waitcnt vmcnt(37) lgkmcnt(0)\n\t" \
  "s_barrier\n\t" \
  "v_mfma_f32_16x16x128_f8f6f4 v[224:227], v[120:127], a[96:103], v[224:227]\n\t" \
  "ds_read_b64_tr_b8 v[112:113], v88 offset:0\n\t" \
  "s_add_u32 m0, s24, 0x26400\n\t" \
  "s_add_u32 s55, s45, 0x420000\n\t" \
  "buffer_load_dwordx4 v3, s[8:11], s55 offen lds\n\t" \
  "v_mfma_f32_16x16x128_f8f6f4 v[228:231], v[120:127], a[104:111], v[228:231]\n\t" \
  "ds_read_b64_tr_b8 v[114:115], v88 offset:1024\n\t" \
  "v_mfma_f32_16x16x128_f8f6f4 v[232:235], v[120:127], a[112:119], v[232:235]\n\t" \
  "ds_read_b64_tr_b8 v[116:117], v96 offset:0\n\t" \
  "s_add_u32 s48, s68, 0x170000\n\t" \
  "buffer_load_dwordx4 a[224:227], v4, s[12:15], s48 offen offset:0 sc0 sc1\n\t" \
  "v_mfma_f32_16x16x128_f8f6f4 v[236:239], v[120:127], a[120:127], v[236:239]\n\t" \
  "ds_read_b64_tr_b8 v[118:119], v96 offset:1024\n\t" \
  "v_mfma_f32_16x16x128_f8f6f4 v[252:255], v[104:111], a[120:127], v[252:255]\n\t" \
  "ds_read_b64_tr_b8 v[120:121], v89 offset:0\n\t" \
  "s_add_u32 s49, s68, 0x170000\n\t" \
  "buffer_load_dwordx4 a[228:231], v4, s[12:15], s49 offen offset:1024 sc0 sc1\n\t" \
  "v_mfma_f32_16x16x128_f8f6f4 v[248:251], v[104:111], a[112:119], v[248:251]\n\t" \
  "ds_read_b64_tr_b8 v[122:123], v89 offset:1024\n\t" \
  "v_mfma_f32_16x16x128_f8f6f4 v[244:247], v[104:111], a[104:111], v[244:247]\n\t" \
  "ds_read_b64_tr_b8 v[124:125], v97 offset:0\n\t" \
  "s_add_u32 s50, s68, 0x170000\n\t" \
  "buffer_load_dwordx4 a[232:235], v4, s[12:15], s50 offen offset:2048 sc0 sc1\n\t" \
  "v_mfma_f32_16x16x128_f8f6f4 v[240:243], v[104:111], a[96:103], v[240:243]\n\t" \
  "ds_read_b64_tr_b8 v[126:127], v97 offset:1024\n\t" \
  "s_waitcnt vmcnt(33) lgkmcnt(4)\n\t" \
  "v_mfma_f32_16x16x128_f8f6f4 v[128:131], v[112:119], a[128:135], v[128:131]\n\t" \
  "ds_read_b64_tr_b8 v[104:105], v90 offset:0\n\t" \
  "s_add_u32 s51, s68, 0x170000\n\t" \
  "buffer_load_dwordx4 a[236:239], v4, s[12:15], s51 offen offset:3072 sc0 sc1\n\t" \
  "v_mfma_f32_16x16x128_f8f6f4 v[132:135], v[112:119], a[136:143], v[132:135]\n\t" \
  "ds_read_b64_tr_b8 v[106:107], v90 offset:1024\n\t" \
  "v_mfma_f32_16x16x128_f8f6f4 v[136:139], v[112:119], a[144:151], v[136:139]\n\t" \
  "ds_read_b64_tr_b8 v[108:109], v98 offset:0\n\t" \
  "s_add_u32 s52, s68, 0x171000\n\t" \
  "buffer_load_dwordx4 a[240:243], v4, s[12:15], s52 offen offset:0 sc0 sc1\n\t" \
  "v_mfma_f32_16x16x128_f8f6f4 v[140:143], v[112:119], a[152:159], v[140:143]\n\t" \
  "ds_read_b64_tr_b8 v[110:111], v98 offset:1024\n\t" \
  "s_waitcnt lgkmcnt(4)\n\t" \
  "v_mfma_f32_16x16x128_f8f6f4 v[156:159], v[120:127], a[152:159], v[156:159]\n\t" \
  "ds_read_b64_tr_b8 v[112:113], v91 offset:0\n\t" \
  "s_add_u32 s53, s68, 0x171000\n\t" \
  "buffer_load_dwordx4 a[244:247], v4, s[12:15], s53 offen offset:1024 sc0 sc1\n\t" \
  "v_mfma_f32_16x16x128_f8f6f4 v[152:155], v[120:127], a[144:151], v[152:155]\n\t" \
  "ds_read_b64_tr_b8 v[114:115], v91 offset:1024\n\t" \
  "v_mfma_f32_16x16x128_f8f6f4 v[148:151], v[120:127], a[136:143], v[148:151]\n\t" \
  "ds_read_b64_tr_b8 v[116:117], v99 offset:0\n\t" \
  "s_add_u32 s54, s68, 0x171000\n\t" \
  "buffer_load_dwordx4 a[248:251], v4, s[12:15], s54 offen offset:2048 sc0 sc1\n\t" \
  "v_mfma_f32_16x16x128_f8f6f4 v[144:147], v[120:127], a[128:135], v[144:147]\n\t" \
  "ds_read_b64_tr_b8 v[118:119], v99 offset:1024\n\t" \
  "s_waitcnt lgkmcnt(4)\n\t" \
  "v_mfma_f32_16x16x128_f8f6f4 v[160:163], v[104:111], a[128:135], v[160:163]\n\t" \
  "ds_read_b64_tr_b8 v[120:121], v92 offset:0\n\t" \
  "s_add_u32 s55, s68, 0x171000\n\t" \
  "buffer_load_dwordx4 a[252:255], v4, s[12:15], s55 offen offset:3072 sc0 sc1\n\t" \
  "v_mfma_f32_16x16x128_f8f6f4 v[164:167], v[104:111], a[136:143], v[164:167]\n\t" \
  "ds_read_b64_tr_b8 v[122:123], v92 offset:1024\n\t" \
  "v_mfma_f32_16x16x128_f8f6f4 v[168:171], v[104:111], a[144:151], v[168:171]\n\t" \
  "ds_read_b64_tr_b8 v[124:125], v100 offset:0\n\t" \
  "s_add_u32 m0, s24, 0x18000\n\t" \
  "s_add_u32 s48, s46, 0x400000\n\t" \
  "buffer_load_dwordx4 v3, s[8:11], s48 offen lds\n\t" \
  "v_mfma_f32_16x16x128_f8f6f4 v[172:175], v[104:111], a[152:159], v[172:175]\n\t" \
  "ds_read_b64_tr_b8 v[126:127], v100 offset:1024\n\t" \
  "s_waitcnt lgkmcnt(4)\n\t" \
  "v_mfma_f32_16x16x128_f8f6f4 v[188:191], v[112:119], a[152:159], v[188:191]\n\t" \
  "ds_read_b64_tr_b8 v[104:105], v93 offset:0\n\t" \
  "s_add_u32 m0, s24, 0x18400\n\t" \
  "s_add_u32 s49, s46, 0x420000\n\t" \
  "buffer_load_dwordx4 v3, s[8:11], s49 offen lds\n\t" \
  "v_mfma_f32_16x16x128_f8f6f4 v[184:187], v[112:119], a[144:151], v[184:187]\n\t" \
  "ds_read_b64_tr_b8 v[106:107], v93 offset:1024\n\t" \
  "v_mfma_f32_16x16x128_f8f6f4 v[180:183], v[112:119], a[136:143], v[180:183]\n\t" \
  "ds_read_b64_tr_b8 v[108:109], v101 offset:0\n\t" \
  "s_add_u32 s50, s68, 0x180000\n\t" \
  "buffer_load_dwordx4 a[0:3], v4, s[12:15], s50 offen offset:0 sc0 sc1\n\t" \
  "v_mfma_f32_16x16x128_f8f6f4 v[176:179], v[112:119], a[128:135], v[176:179]\n\t" \
  "ds_read_b64_tr_b8 v[110:111], v101 offset:1024\n\t" \
  "s_waitcnt lgkmcnt(4)\n\t" \
  "v_mfma_f32_16x16x128_f8f6f4 v[192:195], v[120:127], a[128:135], v[192:195]\n\t" \
  "ds_read_b64_tr_b8 v[112:113], v94 offset:0\n\t" \
  "s_add_u32 s51, s68, 0x180000\n\t" \
  "buffer_load_dwordx4 a[4:7], v4, s[12:15], s51 offen offset:1024 sc0 sc1\n\t" \
  "v_mfma_f32_16x16x128_f8f6f4 v[196:199], v[120:127], a[136:143], v[196:199]\n\t" \
  "ds_read_b64_tr_b8 v[114:115], v94 offset:1024\n\t" \
  "v_mfma_f32_16x16x128_f8f6f4 v[200:203], v[120:127], a[144:151], v[200:203]\n\t" \
  "ds_read_b64_tr_b8 v[116:117], v102 offset:0\n\t" \
  "s_add_u32 s52, s68, 0x180000\n\t" \
  "buffer_load_dwordx4 a[8:11], v4, s[12:15], s52 offen offset:2048 sc0 sc1\n\t" \
  "v_mfma_f32_16x16x128_f8f6f4 v[204:207], v[120:127], a[152:159], v[204:207]\n\t" \
  "ds_read_b64_tr_b8 v[118:119], v102 offset:1024\n\t" \
  "s_waitcnt lgkmcnt(4)\n\t" \
  "v_mfma_f32_16x16x128_f8f6f4 v[220:223], v[104:111], a[152:159], v[220:223]\n\t" \
  "ds_read_b64_tr_b8 v[120:121], v95 offset:0\n\t" \
  "s_add_u32 s53, s68, 0x180000\n\t" \
  "buffer_load_dwordx4 a[12:15], v4, s[12:15], s53 offen offset:3072 sc0 sc1\n\t" \
  "v_mfma_f32_16x16x128_f8f6f4 v[216:219], v[104:111], a[144:151], v[216:219]\n\t" \
  "ds_read_b64_tr_b8 v[122:123], v95 offset:1024\n\t" \
  "v_mfma_f32_16x16x128_f8f6f4 v[212:215], v[104:111], a[136:143], v[212:215]\n\t" \
  "ds_read_b64_tr_b8 v[124:125], v103 offset:0\n\t" \
  "s_add_u32 s54, s68, 0x181000\n\t" \
  "buffer_load_dwordx4 a[16:19], v4, s[12:15], s54 offen offset:0 sc0 sc1\n\t" \
  "v_mfma_f32_16x16x128_f8f6f4 v[208:211], v[104:111], a[128:135], v[208:211]\n\t" \
  "ds_read_b64_tr_b8 v[126:127], v103 offset:1024\n\t" \
  "s_waitcnt vmcnt(43) lgkmcnt(0)\n\t" \
  "s_barrier\n\t" \
  "v_mfma_f32_16x16x128_f8f6f4 v[224:227], v[112:119], a[128:135], v[224:227]\n\t" \
  "ds_read_b64_tr_b8 v[104:105], v88 offset:8192\n\t" \
  "s_add_u32 s55, s68, 0x181000\n\t" \
  "buffer_load_dwordx4 a[20:23], v4, s[12:15], s55 offen offset:1024 sc0 sc1\n\t" \
  "v_mfma_f32_16x16x128_f8f6f4 v[228:231], v[112:119], a[136:143], v[228:231]\n\t" \
  "ds_read_b64_tr_b8 v[106:107], v88 offset:9216\n\t" \
  "v_mfma_f32_16x16x128_f8f6f4 v[232:235], v[112:119], a[144:151], v[232:235]\n\t" \
  "ds_read_b64_tr_b8 v[108:109], v96 offset:8192\n\t" \
  "s_add_u32 s48, s68, 0x181000\n\t" \
  "buffer_load_dwordx4 a[24:27], v4, s[12:15], s48 offen offset:2048 sc0 sc1\n\t" \
  "v_mfma_f32_16x16x128_f8f6f4 v[236:239], v[112:119], a[152:159], v[236:239]\n\t" \
  "ds_read_b64_tr_b8 v[110:111], v96 offset:9216\n\t" \
  "v_mfma_f32_16x16x128_f8f6f4 v[252:255], v[120:127], a[152:159], v[252:255]\n\t" \
  "ds_read_b64_tr_b8 v[112:113], v89 offset:8192\n\t" \
  "s_add_u32 s49, s68, 0x181000\n\t" \
  "buffer_load_dwordx4 a[28:31], v4, s[12:15], s49 offen offset:3072 sc0 sc1\n\t" \
  "v_mfma_f32_16x16x128_f8f6f4 v[248:251], v[120:127], a[144:151], v[248:251]\n\t" \
  "ds_read_b64_tr_b8 v[114:115], v89 offset:9216\n\t" \
  "v_mfma_f32_16x16x128_f8f6f4 v[244:247], v[120:127], a[136:143], v[244:247]\n\t" \
  "ds_read_b64_tr_b8 v[116:117], v97 offset:8192\n\t" \
  "s_add_u32 m0, s24, 0x8000\n\t" \
  "s_add_u32 s50, s43, 0x600000\n\t" \
  "buffer_load_dwordx4 v2, s[8:11], s50 offen lds\n\t" \
  "v_mfma_f32_16x16x128_f8f6f4 v[240:243], v[120:127], a[128:135], v[240:243]\n\t" \
  "ds_read_b64_tr_b8 v[118:119], v97 offset:9216\n\t" \
  "s_waitcnt vmcnt(39) lgkmcnt(4)\n\t" \
  "v_mfma_f32_16x16x128_f8f6f4 v[128:131], v[104:111], a[160:167], v[128:131]\n\t" \
  "ds_read_b64_tr_b8 v[120:121], v90 offset:8192\n\t" \
  "s_add_u32 m0, s24, 0x8400\n\t" \
  "s_add_u32 s51, s43, 0x620000\n\t" \
  "buffer_load_dwordx4 v2, s[8:11], s51 offen lds\n\t" \
  "s_mov_b32 exec_lo, 0x7f000000\n\t" \
  "s_mov_b32 exec_hi, 0\n\t" \
  "global_load_dwordx4 v[68:71], v10, s[6:7] sc0 sc1\n\t" \
  "s_mov_b64 exec, -1\n\t" \
  "v_mfma_f32_16x16x128_f8f6f4 v[132:135], v[104:111], a[168:175], v[132:135]\n\t" \
  "ds_read_b64_tr_b8 v[122:123], v90 offset:9216\n\t" \
  "v_mfma_f32_16x16x128_f8f6f4 v[136:139], v[104:111], a[176:183], v[136:139]\n\t" \
  "ds_read_b64_tr_b8 v[124:125], v98 offset:8192\n\t" \
  "s_add_u32 m0, s24, 0xa000\n\t" \
  "s_add_u32 s52, s44, 0x600000\n\t" \
  "buffer_load_dwordx4 v2, s[8:11], s52 offen lds\n\t" \
  "v_mfma_f32_16x16x128_f8f6f4 v[140:143], v[104:111], a[184:191], v[140:143]\n\t" \
  "ds_read_b64_tr_b8 v[126:127], v98 offset:9216\n\t" \
  "s_waitcnt lgkmcnt(4)\n\t" \
  "v_mfma_f32_16x16x128_f8f6f4 v[156:159], v[112:119], a[184:191], v[156:159]\n\t" \
  "ds_read_b64_tr_b8 v[104:105], v91 offset:8192\n\t" \
  "s_add_u32 m0, s24, 0xa400\n\t" \
  "s_add_u32 s53, s44, 0x620000\n\t" \
  "buffer_load_dwordx4 v2, s[8:11], s53 offen lds\n\t" \
  "v_mfma_f32_16x16x128_f8f6f4 v[152:155], v[112:119], a[176:183], v[152:155]\n\t" \
  "ds_read_b64_tr_b8 v[106:107], v91 offset:9216\n\t" \
  "v_mfma_f32_16x16x128_f8f6f4 v[148:151], v[112:119], a[168:175], v[148:151]\n\t" \
  "ds_read_b64_tr_b8 v[108:109], v99 offset:8192\n\t" \
  "s_add_u32 m0, s24, 0xc000\n\t" \
  "s_add_u32 s54, s45, 0x600000\n\t" \
  "buffer_load_dwordx4 v2, s[8:11], s54 offen lds\n\t" \
  "v_mfma_f32_16x16x128_f8f6f4 v[144:147], v[112:119], a[160:167], v[144:147]\n\t" \
  "ds_read_b64_tr_b8 v[110:111], v99 offset:9216\n\t" \
  "s_waitcnt lgkmcnt(4)\n\t" \
  "v_mfma_f32_16x16x128_f8f6f4 v[160:163], v[120:127], a[160:167], v[160:163]\n\t" \
  "ds_read_b64_tr_b8 v[112:113], v92 offset:8192\n\t" \
  "s_add_u32 m0, s24, 0xc400\n\t" \
  "s_add_u32 s55, s45, 0x620000\n\t" \
  "buffer_load_dwordx4 v2, s[8:11], s55 offen lds\n\t" \
  "v_mfma_f32_16x16x128_f8f6f4 v[164:167], v[120:127], a[168:175], v[164:167]\n\t" \
  "ds_read_b64_tr_b8 v[114:115], v92 offset:9216\n\t" \
  "v_mfma_f32_16x16x128_f8f6f4 v[168:171], v[120:127], a[176:183], v[168:171]\n\t" \
  "ds_read_b64_tr_b8 v[116:117], v100 offset:8192\n\t" \
  "s_add_u32 m0, s24, 0xe000\n\t" \
  "s_add_u32 s48, s46, 0x600000\n\t" \
  "buffer_load_dwordx4 v2, s[8:11], s48 offen lds\n\t" \
  "v_mfma_f32_16x16x128_f8f6f4 v[172:175], v[120:127], a[184:191], v[172:175]\n\t" \
  "ds_read_b64_tr_b8 v[118:119], v100 offset:9216\n\t" \
  "s_waitcnt lgkmcnt(4)\n\t" \
  "v_mfma_f32_16x16x128_f8f6f4 v[188:191], v[104:111], a[184:191], v[188:191]\n\t" \
  "ds_read_b64_tr_b8 v[120:121], v93 offset:8192\n\t" \
  "s_add_u32 m0, s24, 0xe400\n\t" \
  "s_add_u32 s49, s46, 0x620000\n\t" \
  "buffer_load_dwordx4 v2, s[8:11], s49 offen lds\n\t" \
  "v_mfma_f32_16x16x128_f8f6f4 v[184:187], v[104:111], a[176:183], v[184:187]\n\t" \
  "ds_read_b64_tr_b8 v[122:123], v93 offset:9216\n\t" \
  "v_mfma_f32_16x16x128_f8f6f4 v[180:183], v[104:111], a[168:175], v[180:183]\n\t" \
  "ds_read_b64_tr_b8 v[124:125], v101 offset:8192\n\t" \
  "v_mfma_f32_16x16x128_f8f6f4 v[176:179], v[104:111], a[160:167], v[176:179]\n\t" \
  "ds_read_b64_tr_b8 v[126:127], v101 offset:9216\n\t" \
  "s_waitcnt lgkmcnt(4)\n\t" \
  "v_mfma_f32_16x16x128_f8f6f4 v[192:195], v[112:119], a[160:167], v[192:195]\n\t" \
  "ds_read_b64_tr_b8 v[104:105], v94 offset:8192\n\t" \
  "v_mfma_f32_16x16x128_f8f6f4 v[196:199], v[112:119], a[168:175], v[196:199]\n\t" \
  "ds_read_b64_tr_b8 v[106:107], v94 offset:9216\n\t" \
  "v_mfma_f32_16x16x128_f8f6f4 v[200:203], v[112:119], a[176:183], v[200:203]\n\t" \
  "ds_read_b64_tr_b8 v[108:109], v102 offset:8192\n\t" \
  "v_mfma_f32_16x16x128_f8f6f4 v[204:207], v[112:119], a[184:191], v[204:207]\n\t" \
  "ds_read_b64_tr_b8 v[110:111], v102 offset:9216\n\t" \
  "s_waitcnt lgkmcnt(4)\n\t" \
  "v_mfma_f32_16x16x128_f8f6f4 v[220:223], v[120:127], a[184:191], v[220:223]\n\t" \
  "ds_read_b64_tr_b8 v[112:113], v95 offset:8192\n\t" \
  "v_mfma_f32_16x16x128_f8f6f4 v[216:219], v[120:127], a[176:183], v[216:219]\n\t" \
  "ds_read_b64_tr_b8 v[114:115], v95 offset:9216\n\t" \
  "v_mfma_f32_16x16x128_f8f6f4 v[212:215], v[120:127], a[168:175], v[212:215]\n\t" \
  "ds_read_b64_tr_b8 v[116:117], v103 offset:8192\n\t" \
  "v_mfma_f32_16x16x128_f8f6f4 v[208:211], v[120:127], a[160:167], v[208:211]\n\t" \
  "ds_read_b64_tr_b8 v[118:119], v103 offset:9216\n\t" \
  "s_waitcnt vmcnt(37) lgkmcnt(0)\n\t" \
  "s_barrier\n\t" \
  "v_mfma_f32_16x16x128_f8f6f4 v[224:227], v[104:111], a[160:167], v[224:227]\n\t" \
  "ds_read_b64_tr_b8 v[120:121], v88 offset:16384\n\t" \
  "v_mfma_f32_16x16x128_f8f6f4 v[228:231], v[104:111], a[168:175], v[228:231]\n\t" \
  "ds_read_b64_tr_b8 v[122:123], v88 offset:17408\n\t" \
  "v_mfma_f32_16x16x128_f8f6f4 v[232:235], v[104:111], a[176:183], v[232:235]\n\t" \
  "ds_read_b64_tr_b8 v[124:125], v96 offset:16384\n\t" \
  "v_mfma_f32_16x16x128_f8f6f4 v[236:239], v[104:111], a[184:191], v[236:239]\n\t" \
  "ds_read_b64_tr_b8 v[126:127], v96 offset:17408\n\t" \
  "v_mfma_f32_16x16x128_f8f6f4 v[252:255], v[112:119], a[184:191], v[252:255]\n\t" \
  "ds_read_b64_tr_b8 v[104:105], v89 offset:16384\n\t" \
  "s_add_u32 m0, s24, 0x1a000\n\t" \
  "s_add_u32 s50, s40, 0x600000\n\t" \
  "buffer_load_dwordx4 v3, s[8:11], s50 offen lds\n\t" \
  "v_mfma_f32_16x16x128_f8f6f4 v[248:251], v[112:119], a[176:183], v[248:251]\n\t" \
  "ds_read_b64_tr_b8 v[106:107], v89 offset:17408\n\t" \
  "v_mfma_f32_16x16x128_f8f6f4 v[244:247], v[112:119], a[168:175], v[244:247]\n\t" \
  "ds_read_b64_tr_b8 v[108:109], v97 offset:16384\n\t" \
  "s_add_u32 m0, s24, 0x1a400\n\t" \
  "s_add_u32 s51, s40, 0x620000\n\t" \
  "buffer_load_dwordx4 v3, s[8:11], s51 offen lds\n\t" \
  "v_mfma_f32_16x16x128_f8f6f4 v[240:243], v[112:119], a[160:167], v[240:243]\n\t" \
  "ds_read_b64_tr_b8 v[110:111], v97 offset:17408\n\t" \
  "s_waitcnt vmcnt(31) lgkmcnt(4)\n\t" \
  "v_mfma_f32_16x16x128_f8f6f4 v[128:131], v[120:127], a[192:199], v[128:131]\n\t" \
  "ds_read_b64_tr_b8 v[112:113], v90 offset:16384\n\t" \
  "s_add_u32 m0, s24, 0x1c000\n\t" \
  "s_add_u32 s52, s41, 0x600000\n\t" \
  "buffer_load_dwordx4 v3, s[8:11], s52 offen lds\n\t" \
  "v_mfma_f32_16x16x128_f8f6f4 v[132:135], v[120:127], a[200:207], v[132:135]\n\t" \
  "ds_read_b64_tr_b8 v[114:115], v90 offset:17408\n\t" \
  "v_mfma_f32_16x16x128_f8f6f4 v[136:139], v[120:127], a[208:215], v[136:139]\n\t" \
  "ds_read_b64_tr_b8 v[116:117], v98 offset:16384\n\t" \
  "s_add_u32 m0, s24, 0x1c400\n\t" \
  "s_add_u32 s53, s41, 0x620000\n\t" \
  "buffer_load_dwordx4 v3, s[8:11], s53 offen lds\n\t" \
  "v_mfma_f32_16x16x128_f8f6f4 v[140:143], v[120:127], a[216:223], v[140:143]\n\t" \
  "ds_read_b64_tr_b8 v[118:119], v98 offset:17408\n\t" \
  "s_waitcnt lgkmcnt(4)\n\t" \
  "v_mfma_f32_16x16x128_f8f6f4 v[156:159], v[104:111], a[216:223], v[156:159]\n\t" \
  "ds_read_b64_tr_b8 v[120:121], v91 offset:16384\n\t" \
  "s_add_u32 m0, s24, 0x1e000\n\t" \
  "s_add_u32 s54, s42, 0x600000\n\t" \
  "buffer_load_dwordx4 v3, s[8:11], s54 offen lds\n\t" \
  "v_mfma_f32_16x16x128_f8f6f4 v[152:155], v[104:111], a[208:215], v[152:155]\n\t" \
  "ds_read_b64_tr_b8 v[122:123], v91 offset:17408\n\t" \
  "v_mfma_f32_16x16x128_f8f6f4 v[148:151], v[104:111], a[200:207], v[148:151]\n\t" \
  "ds_read_b64_tr_b8 v[124:125], v99 offset:16384\n\t" \
  "s_add_u32 m0, s24, 0x1e400\n\t" \
  "s_add_u32 s55, s42, 0x620000\n\t" \
  "buffer_load_dwordx4 v3, s[8:11], s55 offen lds\n\t" \
  "v_mfma_f32_16x16x128_f8f6f4 v[144:147], v[104:111], a[192:199], v[144:147]\n\t" \
  "ds_read_b64_tr_b8 v[126:127], v99 offset:17408\n\t" \
  "s_waitcnt lgkmcnt(4)\n\t" \
  "v_mfma_f32_16x16x128_f8f6f4 v[160:163], v[112:119], a[192:199], v[160:163]\n\t" \
  "ds_read_b64_tr_b8 v[104:105], v92 offset:16384\n\t" \
  "s_add_u32 m0, s24, 0x20000\n\t" \
  "s_add_u32 s48, s43, 0x600000\n\t" \
  "buffer_load_dwordx4 v3, s[8:11], s48 offen lds\n\t" \
  "v_mfma_f32_16x16x128_f8f6f4 v[164:167], v[112:119], a[200:207], v[164:167]\n\t" \
  "ds_read_b64_tr_b8 v[106:107], v92 offset:17408\n\t" \
  "v_mfma_f32_16x16x128_f8f6f4 v[168:171], v[112:119], a[208:215], v[168:171]\n\t" \
  "ds_read_b64_tr_b8 v[108:109], v100 offset:16384\n\t" \
  "s_add_u32 m0, s24, 0x20400\n\t" \
  "s_add_u32 s49, s43, 0x620000\n\t" \
  "buffer_load_dwordx4 v3, s[8:11], s49 offen lds\n\t" \
  "v_mfma_f32_16x16x128_f8f6f4 v[172:175], v[112:119], a[216:223], v[172:175]\n\t" \
  "ds_read_b64_tr_b8 v[110:111], v100 offset:17408\n\t" \
  "s_waitcnt lgkmcnt(4)\n\t" \
  "v_mfma_f32_16x16x128_f8f6f4 v[188:191], v[120:127], a[216:223], v[188:191]\n\t" \
  "ds_read_b64_tr_b8 v[112:113], v93 offset:16384\n\t" \
  "s_add_u32 m0, s24, 0x22000\n\t" \
  "s_add_u32 s50, s44, 0x600000\n\t" \
  "buffer_load_dwordx4 v3, s[8:11], s50 offen lds\n\t" \
  "v_mfma_f32_16x16x128_f8f6f4 v[184:187], v[120:127], a[208:215], v[184:187]\n\t" \
  "ds_read_b64_tr_b8 v[114:115], v93 offset:17408\n\t" \
  "v_mfma_f32_16x16x128_f8f6f4 v[180:183], v[120:127], a[200:207], v[180:183]\n\t" \
  "ds_read_b64_tr_b8 v[116:117], v101 offset:16384\n\t" \
  "s_add_u32 m0, s24, 0x22400\n\t" \
  "s_add_u32 s51, s44, 0x620000\n\t" \
  "buffer_load_dwordx4 v3, s[8:11], s51 offen lds\n\t" \
  "v_mfma_f32_16x16x128_f8f6f4 v[176:179], v[120:127], a[192:199], v[176:179]\n\t" \
  "ds_read_b64_tr_b8 v[118:119], v101 offset:17408\n\t" \
  "s_waitcnt lgkmcnt(4)\n\t" \
  "v_mfma_f32_16x16x128_f8f6f4 v[192:195], v[104:111], a[192:199], v[192:195]\n\t" \
  "ds_read_b64_tr_b8 v[120:121], v94 offset:16384\n\t" \
  "s_waitcnt vmcnt(16)\n\t" \
  "v_min_u32 v68, v68, v69\n\t" \
  "v_min3_u32 v68, v68, v70, v71\n\t" \
  "v_cmp_le_u32 vcc, s57, v68\n\t" \
  "s_and_b32 s64, vcc_lo, 0x7f000000\n\t" \
  "s_or_b32 s56, s56, s64\n\t" \
  "s_and_b32 s64, s56, 0x7f000000\n\t" \
  "s_cmp_eq_u32 s64, 0x7f000000\n\t" \
  "s_cbranch_scc1 Lok9_2_%=\n\t" \
  "s_mov_b32 s62, 0x7f000000\n\t" \
  "s_mov_b32 s69, 0x7f000000\n\t" \
  "s_call_b64 s[60:61], Lpoll_2_%=\n\t" \
  "Lok9_2_%=:\n\t" \
  "s_add_u32 s52, s68, 0x190000\n\t" \
  "buffer_load_dwordx4 a[32:35], v4, s[12:15], s52 offen offset:0 sc0 sc1\n\t" \
  "v_mfma_f32_16x16x128_f8f6f4 v[196:199], v[104:111], a[200:207], v[196:199]\n\t" \
  "ds_read_b64_tr_b8 v[122:123], v94 offset:17408\n\t" \
  "v_mfma_f32_16x16x128_f8f6f4 v[200:203], v[104:111], a[208:215], v[200:203]\n\t" \
  "ds_read_b64_tr_b8 v[124:125], v102 offset:16384\n\t" \
  "s_add_u32 s53, s68, 0x190000\n\t" \
  "buffer_load_dwordx4 a[36:39], v4, s[12:15], s53 offen offset:1024 sc0 sc1\n\t" \
  "v_mfma_f32_16x16x128_f8f6f4 v[204:207], v[104:111], a[216:223], v[204:207]\n\t" \
  "ds_read_b64_tr_b8 v[126:127], v102 offset:17408\n\t" \
  "s_waitcnt lgkmcnt(4)\n\t" \
  "v_mfma_f32_16x16x128_f8f6f4 v[220:223], v[112:119], a[216:223], v[220:223]\n\t" \
  "ds_read_b64_tr_b8 v[104:105], v95 offset:16384\n\t" \
  "s_add_u32 s54, s68, 0x190000\n\t" \
  "buffer_load_dwordx4 a[40:43], v4, s[12:15], s54 offen offset:2048 sc0 sc1\n\t" \
  "v_mfma_f32_16x16x128_f8f6f4 v[216:219], v[112:119], a[208:215], v[216:219]\n\t" \
  "ds_read_b64_tr_b8 v[106:107], v95 offset:17408\n\t" \
  "v_mfma_f32_16x16x128_f8f6f4 v[212:215], v[112:119], a[200:207], v[212:215]\n\t" \
  "ds_read_b64_tr_b8 v[108:109], v103 offset:16384\n\t" \
  "s_add_u32 s55, s68, 0x190000\n\t" \
  "buffer_load_dwordx4 a[44:47], v4, s[12:15], s55 offen offset:3072 sc0 sc1\n\t" \
  "v_mfma_f32_16x16x128_f8f6f4 v[208:211], v[112:119], a[192:199], v[208:211]\n\t" \
  "ds_read_b64_tr_b8 v[110:111], v103 offset:17408\n\t" \
  "s_waitcnt lgkmcnt(0)\n\t" \
  "s_barrier\n\t" \
  "v_mfma_f32_16x16x128_f8f6f4 v[224:227], v[120:127], a[192:199], v[224:227]\n\t" \
  "ds_read_b64_tr_b8 v[112:113], v88 offset:24576\n\t" \
  "s_add_u32 s48, s68, 0x191000\n\t" \
  "buffer_load_dwordx4 a[48:51], v4, s[12:15], s48 offen offset:0 sc0 sc1\n\t" \
  "v_mfma_f32_16x16x128_f8f6f4 v[228:231], v[120:127], a[200:207], v[228:231]\n\t" \
  "ds_read_b64_tr_b8 v[114:115], v88 offset:25600\n\t" \
  "v_mfma_f32_16x16x128_f8f6f4 v[232:235], v[120:127], a[208:215], v[232:235]\n\t" \
  "ds_read_b64_tr_b8 v[116:117], v96 offset:24576\n\t" \
  "s_add_u32 s49, s68, 0x191000\n\t" \
  "buffer_load_dwordx4 a[52:55], v4, s[12:15], s49 offen offset:1024 sc0 sc1\n\t" \
  "v_mfma_f32_16x16x128_f8f6f4 v[236:239], v[120:127], a[216:223], v[236:239]\n\t" \
  "ds_read_b64_tr_b8 v[118:119], v96 offset:25600\n\t" \
  "v_mfma_f32_16x16x128_f8f6f4 v[252:255], v[104:111], a[216:223], v[252:255]\n\t" \
  "ds_read_b64_tr_b8 v[120:121], v89 offset:24576\n\t" \
  "s_add_u32 s50, s68, 0x191000\n\t" \
  "buffer_load_dwordx4 a[56:59], v4, s[12:15], s50 offen offset:2048 sc0 sc1\n\t" \
  "v_mfma_f32_16x16x128_f8f6f4 v[248:251], v[104:111], a[208:215], v[248:251]\n\t" \
  "ds_read_b64_tr_b8 v[122:123], v89 offset:25600\n\t" \
  "v_mfma_f32_16x16x128_f8f6f4 v[244:247], v[104:111], a[200:207], v[244:247]\n\t" \
  "ds_read_b64_tr_b8 v[124:125], v97 offset:24576\n\t" \
  "s_add_u32 s51, s68, 0x191000\n\t" \
  "buffer_load_dwordx4 a[60:63], v4, s[12:15], s51 offen offset:3072 sc0 sc1\n\t" \
  "v_mfma_f32_16x16x128_f8f6f4 v[240:243], v[104:111], a[192:199], v[240:243]\n\t" \
  "ds_read_b64_tr_b8 v[126:127], v97 offset:25600\n\t" \
  "s_waitcnt lgkmcnt(4)\n\t" \
  "v_mfma_f32_16x16x128_f8f6f4 v[128:131], v[112:119], a[224:231], v[128:131]\n\t" \
  "ds_read_b64_tr_b8 v[104:105], v90 offset:24576\n\t" \
  "s_add_u32 m0, s24, 0x24000\n\t" \
  "s_add_u32 s52, s45, 0x600000\n\t" \
  "buffer_load_dwordx4 v3, s[8:11], s52 offen lds\n\t" \
  "v_mfma_f32_16x16x128_f8f6f4 v[132:135], v[112:119], a[232:239], v[132:135]\n\t" \
  "ds_read_b64_tr_b8 v[106:107], v90 offset:25600\n\t" \
  "v_mfma_f32_16x16x128_f8f6f4 v[136:139], v[112:119], a[240:247], v[136:139]\n\t" \
  "ds_read_b64_tr_b8 v[108:109], v98 offset:24576\n\t" \
  "s_add_u32 m0, s24, 0x24400\n\t" \
  "s_add_u32 s53, s45, 0x620000\n\t" \
  "buffer_load_dwordx4 v3, s[8:11], s53 offen lds\n\t" \
  "v_mfma_f32_16x16x128_f8f6f4 v[140:143], v[112:119], a[248:255], v[140:143]\n\t" \
  "ds_read_b64_tr_b8 v[110:111], v98 offset:25600\n\t" \
  "s_waitcnt lgkmcnt(4)\n\t" \
  "v_mfma_f32_16x16x128_f8f6f4 v[156:159], v[120:127], a[248:255], v[156:159]\n\t" \
  "ds_read_b64_tr_b8 v[112:113], v91 offset:24576\n\t" \
  "s_add_u32 s54, s68, 0x1a0000\n\t" \
  "buffer_load_dwordx4 a[64:67], v4, s[12:15], s54 offen offset:0 sc0 sc1\n\t" \
  "v_mfma_f32_16x16x128_f8f6f4 v[152:155], v[120:127], a[240:247], v[152:155]\n\t" \
  "ds_read_b64_tr_b8 v[114:115], v91 offset:25600\n\t" \
  "v_mfma_f32_16x16x128_f8f6f4 v[148:151], v[120:127], a[232:239], v[148:151]\n\t" \
  "ds_read_b64_tr_b8 v[116:117], v99 offset:24576\n\t" \
  "s_add_u32 s55, s68, 0x1a0000\n\t" \
  "buffer_load_dwordx4 a[68:71], v4, s[12:15], s55 offen offset:1024 sc0 sc1\n\t" \
  "v_mfma_f32_16x16x128_f8f6f4 v[144:147], v[120:127], a[224:231], v[144:147]\n\t" \
  "ds_read_b64_tr_b8 v[118:119], v99 offset:25600\n\t" \
  "s_waitcnt lgkmcnt(4)\n\t" \
  "v_mfma_f32_16x16x128_f8f6f4 v[160:163], v[104:111], a[224:231], v[160:163]\n\t" \
  "ds_read_b64_tr_b8 v[120:121], v92 offset:24576\n\t" \
  "s_add_u32 s48, s68, 0x1a0000\n\t" \
  "buffer_load_dwordx4 a[72:75], v4, s[12:15], s48 offen offset:2048 sc0 sc1\n\t" \
  "v_mfma_f32_16x16x128_f8f6f4 v[164:167], v[104:111], a[232:239], v[164:167]\n\t" \
  "ds_read_b64_tr_b8 v[122:123], v92 offset:25600\n\t" \
  "v_mfma_f32_16x16x128_f8f6f4 v[168:171], v[104:111], a[240:247], v[168:171]\n\t" \
  "ds_read_b64_tr_b8 v[124:125], v100 offset:24576\n\t" \
  "s_add_u32 s49, s68, 0x1a0000\n\t" \
  "buffer_load_dwordx4 a[76:79], v4, s[12:15], s49 offen offset:3072 sc0 sc1\n\t" \
  "v_mfma_f32_16x16x128_f8f6f4 v[172:175], v[104:111], a[248:255], v[172:175]\n\t" \
  "ds_read_b64_tr_b8 v[126:127], v100 offset:25600\n\t" \
  "s_waitcnt lgkmcnt(4)\n\t" \
  "v_mfma_f32_16x16x128_f8f6f4 v[188:191], v[112:119], a[248:255], v[188:191]\n\t" \
  "ds_read_b64_tr_b8 v[104:105], v93 offset:24576\n\t" \
  "s_add_u32 s50, s68, 0x1a1000\n\t" \
  "buffer_load_dwordx4 a[80:83], v4, s[12:15], s50 offen offset:0 sc0 sc1\n\t" \
  "v_mfma_f32_16x16x128_f8f6f4 v[184:187], v[112:119], a[240:247], v[184:187]\n\t" \
  "ds_read_b64_tr_b8 v[106:107], v93 offset:25600\n\t" \
  "v_mfma_f32_16x16x128_f8f6f4 v[180:183], v[112:119], a[232:239], v[180:183]\n\t" \
  "ds_read_b64_tr_b8 v[108:109], v101 offset:24576\n\t" \
  "s_add_u32 s51, s68, 0x1a1000\n\t" \
  "buffer_load_dwordx4 a[84:87], v4, s[12:15], s51 offen offset:1024 sc0 sc1\n\t" \
  "v_mfma_f32_16x16x128_f8f6f4 v[176:179], v[112:119], a[224:231], v[176:179]\n\t" \
  "ds_read_b64_tr_b8 v[110:111], v101 offset:25600\n\t" \
  "s_waitcnt lgkmcnt(4)\n\t" \
  "v_mfma_f32_16x16x128_f8f6f4 v[192:195], v[120:127], a[224:231], v[192:195]\n\t" \
  "ds_read_b64_tr_b8 v[112:113], v94 offset:24576\n\t" \
  "s_add_u32 s52, s68, 0x1a1000\n\t" \
  "buffer_load_dwordx4 a[88:91], v4, s[12:15], s52 offen offset:2048 sc0 sc1\n\t" \
  "v_mfma_f32_16x16x128_f8f6f4 v[196:199], v[120:127], a[232:239], v[196:199]\n\t" \
  "ds_read_b64_tr_b8 v[114:115], v94 offset:25600\n\t" \
  "v_mfma_f32_16x16x128_f8f6f4 v[200:203], v[120:127], a[240:247], v[200:203]\n\t" \
  "ds_read_b64_tr_b8 v[116:117], v102 offset:24576\n\t" \
  "s_add_u32 s53, s68, 0x1a1000\n\t" \
  "buffer_load_dwordx4 a[92:95], v4, s[12:15], s53 offen offset:3072 sc0 sc1\n\t" \
  "v_mfma_f32_16x16x128_f8f6f4 v[204:207], v[120:127], a[248:255], v[204:207]\n\t" \
  "ds_read_b64_tr_b8 v[118:119], v102 offset:25600\n\t" \
  "s_waitcnt lgkmcnt(4)\n\t" \
  "v_mfma_f32_16x16x128_f8f6f4 v[220:223], v[104:111], a[248:255], v[220:223]\n\t" \
  "ds_read_b64_tr_b8 v[120:121], v95 offset:24576\n\t" \
  "s_add_u32 s54, s68, 0x1b0000\n\t" \
  "buffer_load_dwordx4 a[96:99], v4, s[12:15], s54 offen offset:0 sc0 sc1\n\t" \
  "v_mfma_f32_16x16x128_f8f6f4 v[216:219], v[104:111], a[240:247], v[216:219]\n\t" \
  "ds_read_b64_tr_b8 v[122:123], v95 offset:25600\n\t" \
  "v_mfma_f32_16x16x128_f8f6f4 v[212:215], v[104:111], a[232:239], v[212:215]\n\t" \
  "ds_read_b64_tr_b8 v[124:125], v103 offset:24576\n\t" \
  "s_add_u32 s55, s68, 0x1b0000\n\t" \
  "buffer_load_dwordx4 a[100:103], v4, s[12:15], s55 offen offset:1024 sc0 sc1\n\t" \
  "v_mfma_f32_16x16x128_f8f6f4 v[208:211], v[104:111], a[224:231], v[208:211]\n\t" \
  "ds_read_b64_tr_b8 v[126:127], v103 offset:25600\n\t" \
  "s_waitcnt lgkmcnt(0)\n\t" \
  "s_barrier\n\t" \
  "v_mfma_f32_16x16x128_f8f6f4 v[224:227], v[112:119], a[224:231], v[224:227]\n\t" \
  "ds_read_b64_tr_b8 v[104:105], v80 offset:0\n\t" \
  "s_add_u32 s48, s68, 0x1b0000\n\t" \
  "buffer_load_dwordx4 a[104:107], v4, s[12:15], s48 offen offset:2048 sc0 sc1\n\t" \
  "v_mfma_f32_16x16x128_f8f6f4 v[228:231], v[112:119], a[232:239], v[228:231]\n\t" \
  "ds_read_b64_tr_b8 v[106:107], v80 offset:1024\n\t" \
  "v_mfma_f32_16x16x128_f8f6f4 v[232:235], v[112:119], a[240:247], v[232:235]\n\t" \
  "ds_read_b64_tr_b8 v[108:109], v88 offset:32768\n\t" \
  "s_add_u32 s49, s68, 0x1b0000\n\t" \
  "buffer_load_dwordx4 a[108:111], v4, s[12:15], s49 offen offset:3072 sc0 sc1\n\t" \
  "v_mfma_f32_16x16x128_f8f6f4 v[236:239], v[112:119], a[248:255], v[236:239]\n\t" \
  "ds_read_b64_tr_b8 v[110:111], v88 offset:33792\n\t" \
  "v_mfma_f32_16x16x128_f8f6f4 v[252:255], v[120:127], a[248:255], v[252:255]\n\t" \
  "ds_read_b64_tr_b8 v[112:113], v81 offset:0\n\t" \
  "s_add_u32 s50, s68, 0x1b1000\n\t" \
  "buffer_load_dwordx4 a[112:115], v4, s[12:15], s50 offen offset:0 sc0 sc1\n\t" \
  "v_mfma_f32_16x16x128_f8f6f4 v[248:251], v[120:127], a[240:247], v[248:251]\n\t" \
  "ds_read_b64_tr_b8 v[114:115], v81 offset:1024\n\t" \
  "v_mfma_f32_16x16x128_f8f6f4 v[244:247], v[120:127], a[232:239], v[244:247]\n\t" \
  "ds_read_b64_tr_b8 v[116:117], v89 offset:32768\n\t" \
  "s_add_u32 s51, s68, 0x1b1000\n\t" \
  "buffer_load_dwordx4 a[116:119], v4, s[12:15], s51 offen offset:1024 sc0 sc1\n\t" \
  "v_mfma_f32_16x16x128_f8f6f4 v[240:243], v[120:127], a[224:231], v[240:243]\n\t" \
  "ds_read_b64_tr_b8 v[118:119], v89 offset:33792\n\t" \
  "s_waitcnt lgkmcnt(4)\n\t" \
  "v_mfma_f32_16x16x128_f8f6f4 v[128:131], v[104:111], a[0:7], v[128:131]\n\t" \
  "ds_read_b64_tr_b8 v[120:121], v82 offset:0\n\t" \
  "s_add_u32 s52, s68, 0x1b1000\n\t" \
  "buffer_load_dwordx4 a[120:123], v4, s[12:15], s52 offen offset:2048 sc0 sc1\n\t" \
  "v_mfma_f32_16x16x128_f8f6f4 v[132:135], v[104:111], a[8:15], v[132:135]\n\t" \
  "ds_read_b64_tr_b8 v[122:123], v82 offset:1024\n\t" \
  "v_mfma_f32_16x16x128_f8f6f4 v[136:139], v[104:111], a[16:23], v[136:139]\n\t" \
  "ds_read_b64_tr_b8 v[124:125], v90 offset:32768\n\t" \
  "s_add_u32 s53, s68, 0x1b1000\n\t" \
  "buffer_load_dwordx4 a[124:127], v4, s[12:15], s53 offen offset:3072 sc0 sc1\n\t" \
  "v_mfma_f32_16x16x128_f8f6f4 v[140:143], v[104:111], a[24:31], v[140:143]\n\t" \
  "ds_read_b64_tr_b8 v[126:127], v90 offset:33792\n\t" \
  "s_waitcnt lgkmcnt(4)\n\t" \
  "v_mfma_f32_16x16x128_f8f6f4 v[156:159], v[112:119], a[24:31], v[156:159]\n\t" \
  "ds_read_b64_tr_b8 v[104:105], v83 offset:0\n\t" \
  "s_add_u32 m0, s24, 0x26000\n\t" \
  "s_add_u32 s54, s46, 0x600000\n\t" \
  "buffer_load_dwordx4 v3, s[8:11], s54 offen lds\n\t" \
  "v_mfma_f32_16x16x128_f8f6f4 v[152:155], v[112:119], a[16:23], v[152:155]\n\t" \
  "ds_read_b64_tr_b8 v[106:107], v83 offset:1024\n\t" \
  "v_mfma_f32_16x16x128_f8f6f4 v[148:151], v[112:119], a[8:15], v[148:151]\n\t" \
  "ds_read_b64_tr_b8 v[108:109], v91 offset:32768\n\t" \
  "s_add_u32 m0, s24, 0x26400\n\t" \
  "s_add_u32 s55, s46, 0x620000\n\t" \
  "buffer_load_dwordx4 v3, s[8:11], s55 offen lds\n\t" \
  "v_mfma_f32_16x16x128_f8f6f4 v[144:147], v[112:119], a[0:7], v[144:147]\n\t" \
  "ds_read_b64_tr_b8 v[110:111], v91 offset:33792\n\t" \
  "s_waitcnt lgkmcnt(4)\n\t" \
  "v_mfma_f32_16x16x128_f8f6f4 v[160:163], v[120:127], a[0:7], v[160:163]\n\t" \
  "ds_read_b64_tr_b8 v[112:113], v84 offset:0\n\t" \
  "s_add_u32 s48, s68, 0x1c0000\n\t" \
  "buffer_load_dwordx4 a[128:131], v4, s[12:15], s48 offen offset:0 sc0 sc1\n\t" \
  "v_mfma_f32_16x16x128_f8f6f4 v[164:167], v[120:127], a[8:15], v[164:167]\n\t" \
  "ds_read_b64_tr_b8 v[114:115], v84 offset:1024\n\t" \
  "v_mfma_f32_16x16x128_f8f6f4 v[168:171], v[120:127], a[16:23], v[168:171]\n\t" \
  "ds_read_b64_tr_b8 v[116:117], v92 offset:32768\n\t" \
  "s_add_u32 s49, s68, 0x1c0000\n\t" \
  "buffer_load_dwordx4 a[132:135], v4, s[12:15], s49 offen offset:1024 sc0 sc1\n\t" \
  "v_mfma_f32_16x16x128_f8f6f4 v[172:175], v[120:127], a[24:31], v[172:175]\n\t" \
  "ds_read_b64_tr_b8 v[118:119], v92 offset:33792\n\t" \
  "s_waitcnt lgkmcnt(4)\n\t" \
  "v_mfma_f32_16x16x128_f8f6f4 v[188:191], v[104:111], a[24:31], v[188:191]\n\t" \
  "ds_read_b64_tr_b8 v[120:121], v85 offset:0\n\t" \
  "s_add_u32 s50, s68, 0x1c0000\n\t" \
  "buffer_load_dwordx4 a[136:139], v4, s[12:15], s50 offen offset:2048 sc0 sc1\n\t" \
  "v_mfma_f32_16x16x128_f8f6f4 v[184:187], v[104:111], a[16:23], v[184:187]\n\t" \
  "ds_read_b64_tr_b8 v[122:123], v85 offset:1024\n\t" \
  "v_mfma_f32_16x16x128_f8f6f4 v[180:183], v[104:111], a[8:15], v[180:183]\n\t" \
  "ds_read_b64_tr_b8 v[124:125], v93 offset:32768\n\t" \
  "s_add_u32 s51, s68, 0x1c0000\n\t" \
  "buffer_load_dwordx4 a[140:143], v4, s[12:15], s51 offen offset:3072 sc0 sc1\n\t" \
  "v_mfma_f32_16x16x128_f8f6f4 v[176:179], v[104:111], a[0:7], v[176:179]\n\t" \
  "ds_read_b64_tr_b8 v[126:127], v93 offset:33792\n\t" \
  "s_waitcnt lgkmcnt(4)\n\t" \
  "v_mfma_f32_16x16x128_f8f6f4 v[192:195], v[112:119], a[0:7], v[192:195]\n\t" \
  "ds_read_b64_tr_b8 v[104:105], v86 offset:0\n\t" \
  "s_add_u32 s52, s68, 0x1c1000\n\t" \
  "buffer_load_dwordx4 a[144:147], v4, s[12:15], s52 offen offset:0 sc0 sc1\n\t" \
  "v_mfma_f32_16x16x128_f8f6f4 v[196:199], v[112:119], a[8:15], v[196:199]\n\t" \
  "ds_read_b64_tr_b8 v[106:107], v86 offset:1024\n\t" \
  "v_mfma_f32_16x16x128_f8f6f4 v[200:203], v[112:119], a[16:23], v[200:203]\n\t" \
  "ds_read_b64_tr_b8 v[108:109], v94 offset:32768\n\t" \
  "s_add_u32 s53, s68, 0x1c1000\n\t" \
  "buffer_load_dwordx4 a[148:151], v4, s[12:15], s53 offen offset:1024 sc0 sc1\n\t" \
  "v_mfma_f32_16x16x128_f8f6f4 v[204:207], v[112:119], a[24:31], v[204:207]\n\t" \
  "ds_read_b64_tr_b8 v[110:111], v94 offset:33792\n\t" \
  "s_waitcnt lgkmcnt(4)\n\t" \
  "v_mfma_f32_16x16x128_f8f6f4 v[220:223], v[120:127], a[24:31], v[220:223]\n\t" \
  "ds_read_b64_tr_b8 v[112:113], v87 offset:0\n\t" \
  "s_add_u32 s54, s68, 0x1c1000\n\t" \
  "buffer_load_dwordx4 a[152:155], v4, s[12:15], s54 offen offset:2048 sc0 sc1\n\t" \
  "v_mfma_f32_16x16x128_f8f6f4 v[216:219], v[120:127], a[16:23], v[216:219]\n\t" \
  "ds_read_b64_tr_b8 v[114:115], v87 offset:1024\n\t" \
  "v_mfma_f32_16x16x128_f8f6f4 v[212:215], v[120:127], a[8:15], v[212:215]\n\t" \
  "ds_read_b64_tr_b8 v[116:117], v95 offset:32768\n\t" \
  "s_add_u32 s55, s68, 0x1c1000\n\t" \
  "buffer_load_dwordx4 a[156:159], v4, s[12:15], s55 offen offset:3072 sc0 sc1\n\t" \
  "v_mfma_f32_16x16x128_f8f6f4 v[208:211], v[120:127], a[0:7], v[208:211]\n\t" \
  "ds_read_b64_tr_b8 v[118:119], v95 offset:33792\n\t" \
  "s_waitcnt vmcnt(44) lgkmcnt(0)\n\t" \
  "s_barrier\n\t" \
  "v_mfma_f32_16x16x128_f8f6f4 v[224:227], v[104:111], a[0:7], v[224:227]\n\t" \
  "ds_read_b64_tr_b8 v[120:121], v80 offset:8192\n\t" \
  "s_add_u32 s48, s68, 0x1d0000\n\t" \
  "buffer_load_dwordx4 a[160:163], v4, s[12:15], s48 offen offset:0 sc0 sc1\n\t" \
  "v_mfma_f32_16x16x128_f8f6f4 v[228:231], v[104:111], a[8:15], v[228:231]\n\t" \
  "ds_read_b64_tr_b8 v[122:123], v80 offset:9216\n\t" \
  "v_mfma_f32_16x16x128_f8f6f4 v[232:235], v[104:111], a[16:23], v[232:235]\n\t" \
  "ds_read_b64_tr_b8 v[124:125], v88 offset:40960\n\t" \
  "s_add_u32 s49, s68, 0x1d0000\n\t" \
  "buffer_load_dwordx4 a[164:167], v4, s[12:15], s49 offen offset:1024 sc0 sc1\n\t" \
  "v_mfma_f32_16x16x128_f8f6f4 v[236:239], v[104:111], a[24:31], v[236:239]\n\t" \
  "ds_read_b64_tr_b8 v[126:127], v88 offset:41984\n\t" \
  "v_mfma_f32_16x16x128_f8f6f4 v[252:255], v[112:119], a[24:31], v[252:255]\n\t" \
  "ds_read_b64_tr_b8 v[104:105], v81 offset:8192\n\t" \
  "s_add_u32 s50, s68, 0x1d0000\n\t" \
  "buffer_load_dwordx4 a[168:171], v4, s[12:15], s50 offen offset:2048 sc0 sc1\n\t" \
  "v_mfma_f32_16x16x128_f8f6f4 v[248:251], v[112:119], a[16:23], v[248:251]\n\t" \
  "ds_read_b64_tr_b8 v[106:107], v81 offset:9216\n\t" \
  "v_mfma_f32_16x16x128_f8f6f4 v[244:247], v[112:119], a[8:15], v[244:247]\n\t" \
  "ds_read_b64_tr_b8 v[108:109], v89 offset:40960\n\t" \
  "s_add_u32 s51, s68, 0x1d0000\n\t" \
  "buffer_load_dwordx4 a[172:175], v4, s[12:15], s51 offen offset:3072 sc0 sc1\n\t" \
  "v_mfma_f32_16x16x128_f8f6f4 v[240:243], v[112:119], a[0:7], v[240:243]\n\t" \
  "ds_read_b64_tr_b8 v[110:111], v89 offset:41984\n\t" \
  "s_waitcnt vmcnt(32) lgkmcnt(4)\n\t" \
  "v_mfma_f32_16x16x128_f8f6f4 v[128:131], v[120:127], a[32:39], v[128:131]\n\t" \
  "ds_read_b64_tr_b8 v[112:113], v82 offset:8192\n\t" \
  "s_add_u32 s52, s68, 0x1d1000\n\t" \
  "buffer_load_dwordx4 a[176:179], v4, s[12:15], s52 offen offset:0 sc0 sc1\n\t" \
  "v_mfma_f32_16x16x128_f8f6f4 v[132:135], v[120:127], a[40:47], v[132:135]\n\t" \
  "ds_read_b64_tr_b8 v[114:115], v82 offset:9216\n\t" \
  "v_mfma_f32_16x16x128_f8f6f4 v[136:139], v[120:127], a[48:55], v[136:139]\n\t" \
  "ds_read_b64_tr_b8 v[116:117], v90 offset:40960\n\t" \
  "s_add_u32 s53, s68, 0x1d1000\n\t" \
  "buffer_load_dwordx4 a[180:183], v4, s[12:15], s53 offen offset:1024 sc0 sc1\n\t" \
  "v_mfma_f32_16x16x128_f8f6f4 v[140:143], v[120:127], a[56:63], v[140:143]\n\t" \
  "ds_read_b64_tr_b8 v[118:119], v90 offset:41984\n\t" \
  "s_waitcnt lgkmcnt(4)\n\t" \
  "v_mfma_f32_16x16x128_f8f6f4 v[156:159], v[104:111], a[56:63], v[156:159]\n\t" \
  "ds_read_b64_tr_b8 v[120:121], v83 offset:8192\n\t" \
  "s_add_u32 s54, s68, 0x1d1000\n\t" \
  "buffer_load_dwordx4 a[184:187], v4, s[12:15], s54 offen offset:2048 sc0 sc1\n\t" \
  "v_mfma_f32_16x16x128_f8f6f4 v[152:155], v[104:111], a[48:55], v[152:155]\n\t" \
  "ds_read_b64_tr_b8 v[122:123], v83 offset:9216\n\t" \
  "v_mfma_f32_16x16x128_f8f6f4 v[148:151], v[104:111], a[40:47], v[148:151]\n\t" \
  "ds_read_b64_tr_b8 v[124:125], v91 offset:40960\n\t" \
  "s_add_u32 s55, s68, 0x1d1000\n\t" \
  "buffer_load_dwordx4 a[188:191], v4, s[12:15], s55 offen offset:3072 sc0 sc1\n\t" \
  "v_mfma_f32_16x16x128_f8f6f4 v[144:147], v[104:111], a[32:39], v[144:147]\n\t" \
  "ds_read_b64_tr_b8 v[126:127], v91 offset:41984\n\t" \
  "s_waitcnt lgkmcnt(4)\n\t" \
  "v_mfma_f32_16x16x128_f8f6f4 v[160:163], v[112:119], a[32:39], v[160:163]\n\t" \
  "ds_read_b64_tr_b8 v[104:105], v84 offset:8192\n\t" \
  "s_add_u32 s48, s68, 0x1e0000\n\t" \
  "buffer_load_dwordx4 a[192:195], v4, s[12:15], s48 offen offset:0 sc0 sc1\n\t" \
  "v_mfma_f32_16x16x128_f8f6f4 v[164:167], v[112:119], a[40:47], v[164:167]\n\t" \
  "ds_read_b64_tr_b8 v[106:107], v84 offset:9216\n\t" \
  "v_mfma_f32_16x16x128_f8f6f4 v[168:171], v[112:119], a[48:55], v[168:171]\n\t" \
  "ds_read_b64_tr_b8 v[108:109], v92 offset:40960\n\t" \
  "s_add_u32 s49, s68, 0x1e0000\n\t" \
  "buffer_load_dwordx4 a[196:199], v4, s[12:15], s49 offen offset:1024 sc0 sc1\n\t" \
  "v_mfma_f32_16x16x128_f8f6f4 v[172:175], v[112:119], a[56:63], v[172:175]\n\t" \
  "ds_read_b64_tr_b8 v[110:111], v92 offset:41984\n\t" \
  "s_waitcnt lgkmcnt(4)\n\t" \
  "v_mfma_f32_16x16x128_f8f6f4 v[188:191], v[120:127], a[56:63], v[188:191]\n\t" \
  "ds_read_b64_tr_b8 v[112:113], v85 offset:8192\n\t" \
  "s_add_u32 s50, s68, 0x1e0000\n\t" \
  "buffer_load_dwordx4 a[200:203], v4, s[12:15], s50 offen offset:2048 sc0 sc1\n\t" \
  "v_mfma_f32_16x16x128_f8f6f4 v[184:187], v[120:127], a[48:55], v[184:187]\n\t" \
  "ds_read_b64_tr_b8 v[114:115], v85 offset:9216\n\t" \
  "v_mfma_f32_16x16x128_f8f6f4 v[180:183], v[120:127], a[40:47], v[180:183]\n\t" \
  "ds_read_b64_tr_b8 v[116:117], v93 offset:40960\n\t" \
  "s_add_u32 s51, s68, 0x1e0000\n\t" \
  "buffer_load_dwordx4 a[204:207], v4, s[12:15], s51 offen offset:3072 sc0 sc1\n\t" \
  "v_mfma_f32_16x16x128_f8f6f4 v[176:179], v[120:127], a[32:39], v[176:179]\n\t" \
  "ds_read_b64_tr_b8 v[118:119], v93 offset:41984\n\t" \
  "s_waitcnt lgkmcnt(4)\n\t" \
  "v_mfma_f32_16x16x128_f8f6f4 v[192:195], v[104:111], a[32:39], v[192:195]\n\t" \
  "ds_read_b64_tr_b8 v[120:121], v86 offset:8192\n\t" \
  "s_add_u32 s52, s68, 0x1e1000\n\t" \
  "buffer_load_dwordx4 a[208:211], v4, s[12:15], s52 offen offset:0 sc0 sc1\n\t" \
  "v_mfma_f32_16x16x128_f8f6f4 v[196:199], v[104:111], a[40:47], v[196:199]\n\t" \
  "ds_read_b64_tr_b8 v[122:123], v86 offset:9216\n\t" \
  "v_mfma_f32_16x16x128_f8f6f4 v[200:203], v[104:111], a[48:55], v[200:203]\n\t" \
  "ds_read_b64_tr_b8 v[124:125], v94 offset:40960\n\t" \
  "s_add_u32 s53, s68, 0x1e1000\n\t" \
  "buffer_load_dwordx4 a[212:215], v4, s[12:15], s53 offen offset:1024 sc0 sc1\n\t" \
  "v_mfma_f32_16x16x128_f8f6f4 v[204:207], v[104:111], a[56:63], v[204:207]\n\t" \
  "ds_read_b64_tr_b8 v[126:127], v94 offset:41984\n\t" \
  "s_waitcnt lgkmcnt(4)\n\t" \
  "v_mfma_f32_16x16x128_f8f6f4 v[220:223], v[112:119], a[56:63], v[220:223]\n\t" \
  "ds_read_b64_tr_b8 v[104:105], v87 offset:8192\n\t" \
  "s_add_u32 s54, s68, 0x1e1000\n\t" \
  "buffer_load_dwordx4 a[216:219], v4, s[12:15], s54 offen offset:2048 sc0 sc1\n\t" \
  "v_mfma_f32_16x16x128_f8f6f4 v[216:219], v[112:119], a[48:55], v[216:219]\n\t" \
  "ds_read_b64_tr_b8 v[106:107], v87 offset:9216\n\t" \
  "v_mfma_f32_16x16x128_f8f6f4 v[212:215], v[112:119], a[40:47], v[212:215]\n\t" \
  "ds_read_b64_tr_b8 v[108:109], v95 offset:40960\n\t" \
  "s_add_u32 s55, s68, 0x1e1000\n\t" \
  "buffer_load_dwordx4 a[220:223], v4, s[12:15], s55 offen offset:3072 sc0 sc1\n\t" \
  "v_mfma_f32_16x16x128_f8f6f4 v[208:211], v[112:119], a[32:39], v[208:211]\n\t" \
  "ds_read_b64_tr_b8 v[110:111], v95 offset:41984\n\t" \
  "s_waitcnt lgkmcnt(0)\n\t" \
  "s_barrier\n\t" \
  "v_mfma_f32_16x16x128_f8f6f4 v[224:227], v[120:127], a[32:39], v[224:227]\n\t" \
  "ds_read_b64_tr_b8 v[112:113], v80 offset:16384\n\t" \
  "s_add_u32 s48, s68, 0x1f0000\n\t" \
  "buffer_load_dwordx4 a[224:227], v4, s[12:15], s48 offen offset:0 sc0 sc1\n\t" \
  "v_mfma_f32_16x16x128_f8f6f4 v[228:231], v[120:127], a[40:47], v[228:231]\n\t" \
  "ds_read_b64_tr_b8 v[114:115], v80 offset:17408\n\t" \
  "v_mfma_f32_16x16x128_f8f6f4 v[232:235], v[120:127], a[48:55], v[232:235]\n\t" \
  "ds_read_b64_tr_b8 v[116:117], v88 offset:49152\n\t" \
  "s_add_u32 s49, s68, 0x1f0000\n\t" \
  "buffer_load_dwordx4 a[228:231], v4, s[12:15], s49 offen offset:1024 sc0 sc1\n\t" \
  "v_mfma_f32_16x16x128_f8f6f4 v[236:239], v[120:127], a[56:63], v[236:239]\n\t" \
  "ds_read_b64_tr_b8 v[118:119], v88 offset:50176\n\t" \
  "v_mfma_f32_16x16x128_f8f6f4 v[252:255], v[104:111], a[56:63], v[252:255]\n\t" \
  "ds_read_b64_tr_b8 v[120:121], v81 offset:16384\n\t" \
  "s_add_u32 s50, s68, 0x1f0000\n\t" \
  "buffer_load_dwordx4 a[232:235], v4, s[12:15], s50 offen offset:2048 sc0 sc1\n\t" \
  "v_mfma_f32_16x16x128_f8f6f4 v[248:251], v[104:111], a[48:55], v[248:251]\n\t" \
  "ds_read_b64_tr_b8 v[122:123], v81 offset:17408\n\t" \
  "v_mfma_f32_16x16x128_f8f6f4 v[244:247], v[104:111], a[40:47], v[244:247]\n\t" \
  "ds_read_b64_tr_b8 v[124:125], v89 offset:49152\n\t" \
  "s_add_u32 s51, s68, 0x1f0000\n\t" \
  "buffer_load_dwordx4 a[236:239], v4, s[12:15], s51 offen offset:3072 sc0 sc1\n\t" \
  "v_mfma_f32_16x16x128_f8f6f4 v[240:243], v[104:111], a[32:39], v[240:243]\n\t" \
  "ds_read_b64_tr_b8 v[126:127], v89 offset:50176\n\t" \
  "s_waitcnt vmcnt(38) lgkmcnt(4)\n\t" \
  "v_mfma_f32_16x16x128_f8f6f4 v[128:131], v[112:119], a[64:71], v[128:131]\n\t" \
  "ds_read_b64_tr_b8 v[104:105], v82 offset:16384\n\t" \
  "s_add_u32 s52, s68, 0x1f1000\n\t" \
  "buffer_load_dwordx4 a[240:243], v4, s[12:15], s52 offen offset:0 sc0 sc1\n\t" \
  "v_mfma_f32_16x16x128_f8f6f4 v[132:135], v[112:119], a[72:79], v[132:135]\n\t" \
  "ds_read_b64_tr_b8 v[106:107], v82 offset:17408\n\t" \
  "v_mfma_f32_16x16x128_f8f6f4 v[136:139], v[112:119], a[80:87], v[136:139]\n\t" \
  "ds_read_b64_tr_b8 v[108:109], v90 offset:49152\n\t" \
  "s_add_u32 s53, s68, 0x1f1000\n\t" \
  "buffer_load_dwordx4 a[244:247], v4, s[12:15], s53 offen offset:1024 sc0 sc1\n\t" \
  "v_mfma_f32_16x16x128_f8f6f4 v[140:143], v[112:119], a[88:95], v[140:143]\n\t" \
  "ds_read_b64_tr_b8 v[110:111], v90 offset:50176\n\t" \
  "s_waitcnt lgkmcnt(4)\n\t" \
  "v_mfma_f32_16x16x128_f8f6f4 v[156:159], v[120:127], a[88:95], v[156:159]\n\t" \
  "ds_read_b64_tr_b8 v[112:113], v83 offset:16384\n\t" \
  "s_add_u32 s54, s68, 0x1f1000\n\t" \
  "buffer_load_dwordx4 a[248:251], v4, s[12:15], s54 offen offset:2048 sc0 sc1\n\t" \
  "v_mfma_f32_16x16x128_f8f6f4 v[152:155], v[120:127], a[80:87], v[152:155]\n\t" \
  "ds_read_b64_tr_b8 v[114:115], v83 offset:17408\n\t" \
  "v_mfma_f32_16x16x128_f8f6f4 v[148:151], v[120:127], a[72:79], v[148:151]\n\t" \
  "ds_read_b64_tr_b8 v[116:117], v91 offset:49152\n\t" \
  "s_add_u32 s55, s68, 0x1f1000\n\t" \
  "buffer_load_dwordx4 a[252:255], v4, s[12:15], s55 offen offset:3072 sc0 sc1\n\t" \
  "v_mfma_f32_16x16x128_f8f6f4 v[144:147], v[120:127], a[64:71], v[144:147]\n\t" \
  "ds_read_b64_tr_b8 v[118:119], v91 offset:50176\n\t" \
  "s_waitcnt lgkmcnt(4)\n\t" \
  "v_mfma_f32_16x16x128_f8f6f4 v[160:163], v[104:111], a[64:71], v[160:163]\n\t" \
  "ds_read_b64_tr_b8 v[120:121], v84 offset:16384\n\t" \
  "v_mfma_f32_16x16x128_f8f6f4 v[164:167], v[104:111], a[72:79], v[164:167]\n\t" \
  "ds_read_b64_tr_b8 v[122:123], v84 offset:17408\n\t" \
  "v_mfma_f32_16x16x128_f8f6f4 v[168:171], v[104:111], a[80:87], v[168:171]\n\t" \
  "ds_read_b64_tr_b8 v[124:125], v92 offset:49152\n\t" \
  "v_mfma_f32_16x16x128_f8f6f4 v[172:175], v[104:111], a[88:95], v[172:175]\n\t" \
  "ds_read_b64_tr_b8 v[126:127], v92 offset:50176\n\t" \
  "s_waitcnt lgkmcnt(4)\n\t" \
  "v_mfma_f32_16x16x128_f8f6f4 v[188:191], v[112:119], a[88:95], v[188:191]\n\t" \
  "ds_read_b64_tr_b8 v[104:105], v85 offset:16384\n\t" \
  "v_mfma_f32_16x16x128_f8f6f4 v[184:187], v[112:119], a[80:87], v[184:187]\n\t" \
  "ds_read_b64_tr_b8 v[106:107], v85 offset:17408\n\t" \
  "v_mfma_f32_16x16x128_f8f6f4 v[180:183], v[112:119], a[72:79], v[180:183]\n\t" \
  "ds_read_b64_tr_b8 v[108:109], v93 offset:49152\n\t" \
  "v_mfma_f32_16x16x128_f8f6f4 v[176:179], v[112:119], a[64:71], v[176:179]\n\t" \
  "ds_read_b64_tr_b8 v[110:111], v93 offset:50176\n\t" \
  "s_waitcnt lgkmcnt(4)\n\t" \
  "v_mfma_f32_16x16x128_f8f6f4 v[192:195], v[120:127], a[64:71], v[192:195]\n\t" \
  "ds_read_b64_tr_b8 v[112:113], v86 offset:16384\n\t" \
  "v_mfma_f32_16x16x128_f8f6f4 v[196:199], v[120:127], a[72:79], v[196:199]\n\t" \
  "ds_read_b64_tr_b8 v[114:115], v86 offset:17408\n\t" \
  "v_mfma_f32_16x16x128_f8f6f4 v[200:203], v[120:127], a[80:87], v[200:203]\n\t" \
  "ds_read_b64_tr_b8 v[116:117], v94 offset:49152\n\t" \
  "v_mfma_f32_16x16x128_f8f6f4 v[204:207], v[120:127], a[88:95], v[204:207]\n\t" \
  "ds_read_b64_tr_b8 v[118:119], v94 offset:50176\n\t" \
  "s_waitcnt lgkmcnt(4)\n\t" \
  "v_mfma_f32_16x16x128_f8f6f4 v[220:223], v[104:111], a[88:95], v[220:223]\n\t" \
  "ds_read_b64_tr_b8 v[120:121], v87 offset:16384\n\t" \
  "v_mfma_f32_16x16x128_f8f6f4 v[216:219], v[104:111], a[80:87], v[216:219]\n\t" \
  "ds_read_b64_tr_b8 v[122:123], v87 offset:17408\n\t" \
  "v_mfma_f32_16x16x128_f8f6f4 v[212:215], v[104:111], a[72:79], v[212:215]\n\t" \
  "ds_read_b64_tr_b8 v[124:125], v95 offset:49152\n\t" \
  "v_mfma_f32_16x16x128_f8f6f4 v[208:211], v[104:111], a[64:71], v[208:211]\n\t" \
  "ds_read_b64_tr_b8 v[126:127], v95 offset:50176\n\t" \
  "s_waitcnt vmcnt(32) lgkmcnt(0)\n\t" \
  "s_barrier\n\t" \
  "v_mfma_f32_16x16x128_f8f6f4 v[224:227], v[112:119], a[64:71], v[224:227]\n\t" \
  "ds_read_b64_tr_b8 v[104:105], v80 offset:24576\n\t" \
  "v_mov_b32 v66, s66\n\t" \
  "v_mul_f32 v66, s67, v66\n\t" \
  "v_or_b32 v66, s63, v66\n\t" \
  "v_mov_b32 v67, v66\n\t" \
  "v_mfma_f32_16x16x128_f8f6f4 v[228:231], v[112:119], a[72:79], v[228:231]\n\t" \
  "ds_read_b64_tr_b8 v[106:107], v80 offset:25600\n\t" \
  "v_mfma_f32_16x16x128_f8f6f4 v[232:235], v[112:119], a[80:87], v[232:235]\n\t" \
  "ds_read_b64_tr_b8 v[108:109], v88 offset:57344\n\t" \
  "v_mfma_f32_16x16x128_f8f6f4 v[236:239], v[112:119], a[88:95], v[236:239]\n\t" \
  "ds_read_b64_tr_b8 v[110:111], v88 offset:58368\n\t" \
  "v_mfma_f32_16x16x128_f8f6f4 v[252:255], v[120:127], a[88:95], v[252:255]\n\t" \
  "ds_read_b64_tr_b8 v[112:113], v80 offset:32768\n\t" \
  "v_mfma_f32_16x16x128_f8f6f4 v[248:251], v[120:127], a[80:87], v[248:251]\n\t" \
  "ds_read_b64_tr_b8 v[114:115], v80 offset:33792\n\t" \
  "v_mfma_f32_16x16x128_f8f6f4 v[244:247], v[120:127], a[72:79], v[244:247]\n\t" \
  "ds_read_b64_tr_b8 v[116:117], v96 offset:0\n\t" \
  "v_mfma_f32_16x16x128_f8f6f4 v[240:243], v[120:127], a[64:71], v[240:243]\n\t" \
  "ds_read_b64_tr_b8 v[118:119], v96 offset:1024\n\t" \
  "s_waitcnt lgkmcnt(4)\n\t" \
  "v_mfma_f32_16x16x128_f8f6f4 v[128:131], v[104:111], a[96:103], v[128:131]\n\t" \
  "ds_read_b64_tr_b8 v[120:121], v80 offset:40960\n\t" \
  "v_mfma_f32_16x16x128_f8f6f4 v[132:135], v[104:111], a[104:111], v[132:135]\n\t" \
  "ds_read_b64_tr_b8 v[122:123], v80 offset:41984\n\t" \
  "v_mfma_f32_16x16x128_f8f6f4 v[136:139], v[104:111], a[112:119], v[136:139]\n\t" \
  "ds_read_b64_tr_b8 v[124:125], v96 offset:8192\n\t" \
  "v_mfma_f32_16x16x128_f8f6f4 v[140:143], v[104:111], a[120:127], v[140:143]\n\t" \
  "ds_read_b64_tr_b8 v[126:127], v96 offset:9216\n\t" \
  "s_waitcnt vmcnt(24) lgkmcnt(4)\n\t" \
  "v_mfma_f32_16x16x128_f8f6f4 v[140:143], v[112:119], a[152:159], v[140:143]\n\t" \
  "ds_read_b64_tr_b8 v[104:105], v80 offset:49152\n\t" \
  "v_mfma_f32_16x16x128_f8f6f4 v[136:139], v[112:119], a[144:151], v[136:139]\n\t" \
  "ds_read_b64_tr_b8 v[106:107], v80 offset:50176\n\t" \
  "v_mfma_f32_16x16x128_f8f6f4 v[132:135], v[112:119], a[136:143], v[132:135]\n\t" \
  "ds_read_b64_tr_b8 v[108:109], v96 offset:16384\n\t" \
  "v_mfma_f32_16x16x128_f8f6f4 v[128:131], v[112:119], a[128:135], v[128:131]\n\t" \
  "ds_read_b64_tr_b8 v[110:111], v96 offset:17408\n\t" \
  "s_waitcnt vmcnt(16) lgkmcnt(4)\n\t" \
  "v_mfma_f32_16x16x128_f8f6f4 v[128:131], v[120:127], a[160:167], v[128:131]\n\t" \
  "ds_read_b64_tr_b8 v[112:113], v80 offset:57344\n\t" \
  "v_mfma_f32_16x16x128_f8f6f4 v[132:135], v[120:127], a[168:175], v[132:135]\n\t" \
  "ds_read_b64_tr_b8 v[114:115], v80 offset:58368\n\t" \
  "v_mfma_f32_16x16x128_f8f6f4 v[136:139], v[120:127], a[176:183], v[136:139]\n\t" \
  "ds_read_b64_tr_b8 v[116:117], v96 offset:24576\n\t" \
  "v_mfma_f32_16x16x128_f8f6f4 v[140:143], v[120:127], a[184:191], v[140:143]\n\t" \
  "ds_read_b64_tr_b8 v[118:119], v96 offset:25600\n\t" \
  "s_waitcnt vmcnt(8) lgkmcnt(4)\n\t" \
  "v_mfma_f32_16x16x128_f8f6f4 v[140:143], v[104:111], a[216:223], v[140:143]\n\t" \
  "ds_read_b64_tr_b8 v[120:121], v81 offset:24576\n\t" \
  "v_mfma_f32_16x16x128_f8f6f4 v[136:139], v[104:111], a[208:215], v[136:139]\n\t" \
  "ds_read_b64_tr_b8 v[122:123], v81 offset:25600\n\t" \
  "v_mfma_f32_16x16x128_f8f6f4 v[132:135], v[104:111], a[200:207], v[132:135]\n\t" \
  "ds_read_b64_tr_b8 v[124:125], v89 offset:57344\n\t" \
  "v_mfma_f32_16x16x128_f8f6f4 v[128:131], v[104:111], a[192:199], v[128:131]\n\t" \
  "ds_read_b64_tr_b8 v[126:127], v89 offset:58368\n\t" \
  "s_waitcnt vmcnt(0) lgkmcnt(4)\n\t" \
  "v_mfma_f32_16x16x128_f8f6f4 v[128:131], v[112:119], a[224:231], v[128:131]\n\t" \
  "ds_read_b64_tr_b8 v[104:105], v81 offset:32768\n\t" \
  "v_mfma_f32_16x16x128_f8f6f4 v[132:135], v[112:119], a[232:239], v[132:135]\n\t" \
  "ds_read_b64_tr_b8 v[106:107], v81 offset:33792\n\t" \
  "v_mfma_f32_16x16x128_f8f6f4 v[136:139], v[112:119], a[240:247], v[136:139]\n\t" \
  "ds_read_b64_tr_b8 v[108:109], v97 offset:0\n\t" \
  "v_mfma_f32_16x16x128_f8f6f4 v[140:143], v[112:119], a[248:255], v[140:143]\n\t" \
  "ds_read_b64_tr_b8 v[110:111], v97 offset:1024\n\t" \
  "s_waitcnt lgkmcnt(4)\n\t" \
  "v_mfma_f32_16x16x128_f8f6f4 v[156:159], v[120:127], a[120:127], v[156:159]\n\t" \
  "ds_read_b64_tr_b8 v[112:113], v81 offset:40960\n\t" \
  "v_mfma_f32_16x16x128_f8f6f4 v[152:155], v[120:127], a[112:119], v[152:155]\n\t" \
  "ds_read_b64_tr_b8 v[114:115], v81 offset:41984\n\t" \
  "v_mfma_f32_16x16x128_f8f6f4 v[148:151], v[120:127], a[104:111], v[148:151]\n\t" \
  "ds_read_b64_tr_b8 v[116:117], v97 offset:8192\n\t" \
  "v_mfma_f32_16x16x128_f8f6f4 v[144:147], v[120:127], a[96:103], v[144:147]\n\t" \
  "ds_read_b64_tr_b8 v[118:119], v97 offset:9216\n\t" \
  "s_waitcnt lgkmcnt(4)\n\t" \
  "v_mfma_f32_16x16x128_f8f6f4 v[144:147], v[104:111], a[128:135], v[144:147]\n\t" \
  "ds_read_b64_tr_b8 v[120:121], v81 offset:49152\n\t" \
  "v_pk_mul_f32 v[128:129], v[128:129], v[66:67]\n\t" \
  "v_pk_mul_f32 v[130:131], v[130:131], v[66:67]\n\t" \
  "s_mov_b32 s48, 0x0\n\t" \
  "buffer_store_dwordx4 v[128:131], v13, s[16:19], s48 offen offset:0 sc1\n\t" \
  "v_mfma_f32_16x16x128_f8f6f4 v[148:151], v[104:111], a[136:143], v[148:151]\n\t" \
  "ds_read_b64_tr_b8 v[122:123], v81 offset:50176\n\t" \
  "v_mfma_f32_16x16x128_f8f6f4 v[152:155], v[104:111], a[144:151], v[152:155]\n\t" \
  "ds_read_b64_tr_b8 v[124:125], v97 offset:16384\n\t" \
  "v_mfma_f32_16x16x128_f8f6f4 v[156:159], v[104:111], a[152:159], v[156:159]\n\t" \
  "ds_read_b64_tr_b8 v[126:127], v97 offset:17408\n\t" \
  "s_waitcnt lgkmcnt(4)\n\t" \
  "v_mfma_f32_16x16x128_f8f6f4 v[156:159], v[112:119], a[184:191], v[156:159]\n\t" \
  "ds_read_b64_tr_b8 v[104:105], v81 offset:57344\n\t" \
  "v_pk_mul_f32 v[132:133], v[132:133], v[66:67]\n\t" \
  "v_pk_mul_f32 v[134:135], v[134:135], v[66:67]\n\t" \
  "s_mov_b32 s49, 0x100000\n\t" \
  "buffer_store_dwordx4 v[132:135], v13, s[16:19], s49 offen offset:0 sc1\n\t" \
  "v_mfma_f32_16x16x128_f8f6f4 v[152:155], v[112:119], a[176:183], v[152:155]\n\t" \
  "ds_read_b64_tr_b8 v[106:107], v81 offset:58368\n\t" \
  "v_mfma_f32_16x16x128_f8f6f4 v[148:151], v[112:119], a[168:175], v[148:151]\n\t" \
  "ds_read_b64_tr_b8 v[108:109], v97 offset:24576\n\t" \
  "v_mfma_f32_16x16x128_f8f6f4 v[144:147], v[112:119], a[160:167], v[144:147]\n\t" \
  "ds_read_b64_tr_b8 v[110:111], v97 offset:25600\n\t" \
  "s_waitcnt lgkmcnt(4)\n\t" \
  "v_mfma_f32_16x16x128_f8f6f4 v[144:147], v[120:127], a[192:199], v[144:147]\n\t" \
  "ds_read_b64_tr_b8 v[112:113], v82 offset:24576\n\t" \
  "v_pk_mul_f32 v[136:137], v[136:137], v[66:67]\n\t" \
  "v_pk_mul_f32 v[138:139], v[138:139], v[66:67]\n\t" \
  "s_mov_b32 s50, 0x200000\n\t" \
  "buffer_store_dwordx4 v[136:139], v13, s[16:19], s50 offen offset:0 sc1\n\t" \
  "v_mfma_f32_16x16x128_f8f6f4 v[148:151], v[120:127], a[200:207], v[148:151]\n\t" \
  "ds_read_b64_tr_b8 v[114:115], v82 offset:25600\n\t" \
  "v_mfma_f32_16x16x128_f8f6f4 v[152:155], v[120:127], a[208:215], v[152:155]\n\t" \
  "ds_read_b64_tr_b8 v[116:117], v90 offset:57344\n\t" \
  "v_mfma_f32_16x16x128_f8f6f4 v[156:159], v[120:127], a[216:223], v[156:159]\n\t" \
  "ds_read_b64_tr_b8 v[118:119], v90 offset:58368\n\t" \
  "s_waitcnt lgkmcnt(4)\n\t" \
  "v_mfma_f32_16x16x128_f8f6f4 v[156:159], v[104:111], a[248:255], v[156:159]\n\t" \
  "ds_read_b64_tr_b8 v[120:121], v82 offset:32768\n\t" \
  "v_pk_mul_f32 v[140:141], v[140:141], v[66:67]\n\t" \
  "v_pk_mul_f32 v[142:143], v[142:143], v[66:67]\n\t" \
  "s_mov_b32 s51, 0x300000\n\t" \
  "buffer_store_dwordx4 v[140:143], v13, s[16:19], s51 offen offset:0 sc1\n\t" \
  "v_mfma_f32_16x16x128_f8f6f4 v[152:155], v[104:111], a[240:247], v[152:155]\n\t" \
  "ds_read_b64_tr_b8 v[122:123], v82 offset:33792\n\t" \
  "v_mfma_f32_16x16x128_f8f6f4 v[148:151], v[104:111], a[232:239], v[148:151]\n\t" \
  "ds_read_b64_tr_b8 v[124:125], v98 offset:0\n\t" \
  "v_mfma_f32_16x16x128_f8f6f4 v[144:147], v[104:111], a[224:231], v[144:147]\n\t" \
  "ds_read_b64_tr_b8 v[126:127], v98 offset:1024\n\t" \
  "s_waitcnt lgkmcnt(4)\n\t" \
  "v_mfma_f32_16x16x128_f8f6f4 v[160:163], v[112:119], a[96:103], v[160:163]\n\t" \
  "ds_read_b64_tr_b8 v[104:105], v82 offset:40960\n\t" \
  "v_mfma_f32_16x16x128_f8f6f4 v[164:167], v[112:119], a[104:111], v[164:167]\n\t" \
  "ds_read_b64_tr_b8 v[106:107], v82 offset:41984\n\t" \
  "v_mfma_f32_16x16x128_f8f6f4 v[168:171], v[112:119], a[112:119], v[168:171]\n\t" \
  "ds_read_b64_tr_b8 v[108:109], v98 offset:8192\n\t" \
  "v_mfma_f32_16x16x128_f8f6f4 v[172:175], v[112:119], a[120:127], v[172:175]\n\t" \
  "ds_read_b64_tr_b8 v[110:111], v98 offset:9216\n\t" \
  "s_waitcnt lgkmcnt(4)\n\t" \
  "v_mfma_f32_16x16x128_f8f6f4 v[172:175], v[120:127], a[152:159], v[172:175]\n\t" \
  "ds_read_b64_tr_b8 v[112:113], v82 offset:49152\n\t" \
  "v_pk_mul_f32 v[144:145], v[144:145], v[66:67]\n\t" \
  "v_pk_mul_f32 v[146:147], v[146:147], v[66:67]\n\t" \
  "s_mov_b32 s52, 0x0\n\t" \
  "buffer_store_dwordx4 v[144:147], v13, s[16:19], s52 offen offset:64 sc1\n\t" \
  "v_mfma_f32_16x16x128_f8f6f4 v[168:171], v[120:127], a[144:151], v[168:171]\n\t" \
  "ds_read_b64_tr_b8 v[114:115], v82 offset:50176\n\t" \
  "v_mfma_f32_16x16x128_f8f6f4 v[164:167], v[120:127], a[136:143], v[164:167]\n\t" \
  "ds_read_b64_tr_b8 v[116:117], v98 offset:16384\n\t" \
  "v_mfma_f32_16x16x128_f8f6f4 v[160:163], v[120:127], a[128:135], v[160:163]\n\t" \
  "ds_read_b64_tr_b8 v[118:119], v98 offset:17408\n\t" \
  "s_waitcnt lgkmcnt(4)\n\t" \
  "v_mfma_f32_16x16x128_f8f6f4 v[160:163], v[104:111], a[160:167], v[160:163]\n\t" \
  "ds_read_b64_tr_b8 v[120:121], v82 offset:57344\n\t" \
  "v_pk_mul_f32 v[148:149], v[148:149], v[66:67]\n\t" \
  "v_pk_mul_f32 v[150:151], v[150:151], v[66:67]\n\t" \
  "s_mov_b32 s53, 0x100000\n\t" \
  "buffer_store_dwordx4 v[148:151], v13, s[16:19], s53 offen offset:64 sc1\n\t" \
  "v_mfma_f32_16x16x128_f8f6f4 v[164:167], v[104:111], a[168:175], v[164:167]\n\t" \
  "ds_read_b64_tr_b8 v[122:123], v82 offset:58368\n\t" \
  "v_mfma_f32_16x16x128_f8f6f4 v[168:171], v[104:111], a[176:183], v[168:171]\n\t" \
  "ds_read_b64_tr_b8 v[124:125], v98 offset:24576\n\t" \
  "v_mfma_f32_16x16x128_f8f6f4 v[172:175], v[104:111], a[184:191], v[172:175]\n\t" \
  "ds_read_b64_tr_b8 v[126:127], v98 offset:25600\n\t" \
  "s_waitcnt lgkmcnt(4)\n\t" \
  "v_mfma_f32_16x16x128_f8f6f4 v[172:175], v[112:119], a[216:223], v[172:175]\n\t" \
  "ds_read_b64_tr_b8 v[104:105], v83 offset:24576\n\t" \
  "v_pk_mul_f32 v[152:153], v[152:153], v[66:67]\n\t" \
  "v_pk_mul_f32 v[154:155], v[154:155], v[66:67]\n\t" \
  "s_mov_b32 s54, 0x200000\n\t" \
  "buffer_store_dwordx4 v[152:155], v13, s[16:19], s54 offen offset:64 sc1\n\t" \
  "v_mfma_f32_16x16x128_f8f6f4 v[168:171], v[112:119], a[208:215], v[168:171]\n\t" \
  "ds_read_b64_tr_b8 v[106:107], v83 offset:25600\n\t" \
  "v_mfma_f32_16x16x128_f8f6f4 v[164:167], v[112:119], a[200:207], v[164:167]\n\t" \
  "ds_read_b64_tr_b8 v[108:109], v91 offset:57344\n\t" \
  "v_mfma_f32_16x16x128_f8f6f4 v[160:163], v[112:119], a[192:199], v[160:163]\n\t" \
  "ds_read_b64_tr_b8 v[110:111], v91 offset:58368\n\t" \
  "s_waitcnt lgkmcnt(4)\n\t" \
  "v_mfma_f32_16x16x128_f8f6f4 v[160:163], v[120:127], a[224:231], v[160:163]\n\t" \
  "ds_read_b64_tr_b8 v[112:113], v83 offset:32768\n\t" \
  "v_pk_mul_f32 v[156:157], v[156:157], v[66:67]\n\t" \
  "v_pk_mul_f32 v[158:159], v[158:159], v[66:67]\n\t" \
  "s_mov_b32 s55, 0x300000\n\t" \
  "buffer_store_dwordx4 v[156:159], v13, s[16:19], s55 offen offset:64 sc1\n\t" \
  "v_mfma_f32_16x16x128_f8f6f4 v[164:167], v[120:127], a[232:239], v[164:167]\n\t" \
  "ds_read_b64_tr_b8 v[114:115], v83 offset:33792\n\t" \
  "v_mfma_f32_16x16x128_f8f6f4 v[168:171], v[120:127], a[240:247], v[168:171]\n\t" \
  "ds_read_b64_tr_b8 v[116:117], v99 offset:0\n\t" \
  "v_mfma_f32_16x16x128_f8f6f4 v[172:175], v[120:127], a[248:255], v[172:175]\n\t" \
  "ds_read_b64_tr_b8 v[118:119], v99 offset:1024\n\t" \
  "s_waitcnt lgkmcnt(4)\n\t" \
  "v_mfma_f32_16x16x128_f8f6f4 v[188:191], v[104:111], a[120:127], v[188:191]\n\t" \
  "ds_read_b64_tr_b8 v[120:121], v83 offset:40960\n\t" \
  "v_mfma_f32_16x16x128_f8f6f4 v[184:187], v[104:111], a[112:119], v[184:187]\n\t" \
  "ds_read_b64_tr_b8 v[122:123], v83 offset:41984\n\t" \
  "v_mfma_f32_16x16x128_f8f6f4 v[180:183], v[104:111], a[104:111], v[180:183]\n\t" \
  "ds_read_b64_tr_b8 v[124:125], v99 offset:8192\n\t" \
  "v_mfma_f32_16x16x128_f8f6f4 v[176:179], v[104:111], a[96:103], v[176:179]\n\t" \
  "ds_read_b64_tr_b8 v[126:127], v99 offset:9216\n\t" \
  "s_waitcnt lgkmcnt(4)\n\t" \
  "v_mfma_f32_16x16x128_f8f6f4 v[176:179], v[112:119], a[128:135], v[176:179]\n\t" \
  "ds_read_b64_tr_b8 v[104:105], v83 offset:49152\n\t" \
  "v_pk_mul_f32 v[160:161], v[160:161], v[66:67]\n\t" \
  "v_pk_mul_f32 v[162:163], v[162:163], v[66:67]\n\t" \
  "s_mov_b32 s48, 0x0\n\t" \
  "buffer_store_dwordx4 v[160:163], v13, s[16:19], s48 offen offset:128 sc1\n\t" \
  "v_mfma_f32_16x16x128_f8f6f4 v[180:183], v[112:119], a[136:143], v[180:183]\n\t" \
  "ds_read_b64_tr_b8 v[106:107], v83 offset:50176\n\t" \
  "v_mfma_f32_16x16x128_f8f6f4 v[184:187], v[112:119], a[144:151], v[184:187]\n\t" \
  "ds_read_b64_tr_b8 v[108:109], v99 offset:16384\n\t" \
  "v_mfma_f32_16x16x128_f8f6f4 v[188:191], v[112:119], a[152:159], v[188:191]\n\t" \
  "ds_read_b64_tr_b8 v[110:111], v99 offset:17408\n\t" \
  "s_waitcnt lgkmcnt(4)\n\t" \
  "v_mfma_f32_16x16x128_f8f6f4 v[188:191], v[120:127], a[184:191], v[188:191]\n\t" \
  "ds_read_b64_tr_b8 v[112:113], v83 offset:57344\n\t" \
  "v_pk_mul_f32 v[164:165], v[164:165], v[66:67]\n\t" \
  "v_pk_mul_f32 v[166:167], v[166:167], v[66:67]\n\t" \
  "s_mov_b32 s49, 0x100000\n\t" \
  "buffer_store_dwordx4 v[164:167], v13, s[16:19], s49 offen offset:128 sc1\n\t" \
  "v_mfma_f32_16x16x128_f8f6f4 v[184:187], v[120:127], a[176:183], v[184:187]\n\t" \
  "ds_read_b64_tr_b8 v[114:115], v83 offset:58368\n\t" \
  "v_mfma_f32_16x16x128_f8f6f4 v[180:183], v[120:127], a[168:175], v[180:183]\n\t" \
  "ds_read_b64_tr_b8 v[116:117], v99 offset:24576\n\t" \
  "v_mfma_f32_16x16x128_f8f6f4 v[176:179], v[120:127], a[160:167], v[176:179]\n\t" \
  "ds_read_b64_tr_b8 v[118:119], v99 offset:25600\n\t" \
  "s_waitcnt lgkmcnt(4)\n\t" \
  "v_mfma_f32_16x16x128_f8f6f4 v[176:179], v[104:111], a[192:199], v[176:179]\n\t" \
  "ds_read_b64_tr_b8 v[120:121], v84 offset:24576\n\t" \
  "v_pk_mul_f32 v[168:169], v[168:169], v[66:67]\n\t" \
  "v_pk_mul_f32 v[170:171], v[170:171], v[66:67]\n\t" \
  "s_mov_b32 s50, 0x200000\n\t" \
  "buffer_store_dwordx4 v[168:171], v13, s[16:19], s50 offen offset:128 sc1\n\t" \
  "v_mfma_f32_16x16x128_f8f6f4 v[180:183], v[104:111], a[200:207], v[180:183]\n\t" \
  "ds_read_b64_tr_b8 v[122:123], v84 offset:25600\n\t" \
  "v_mfma_f32_16x16x128_f8f6f4 v[184:187], v[104:111], a[208:215], v[184:187]\n\t" \
  "ds_read_b64_tr_b8 v[124:125], v92 offset:57344\n\t" \
  "v_mfma_f32_16x16x128_f8f6f4 v[188:191], v[104:111], a[216:223], v[188:191]\n\t" \
  "ds_read_b64_tr_b8 v[126:127], v92 offset:58368\n\t" \
  "s_waitcnt lgkmcnt(4)\n\t" \
  "v_mfma_f32_16x16x128_f8f6f4 v[188:191], v[112:119], a[248:255], v[188:191]\n\t" \
  "ds_read_b64_tr_b8 v[104:105], v84 offset:32768\n\t" \
  "v_pk_mul_f32 v[172:173], v[172:173], v[66:67]\n\t" \
  "v_pk_mul_f32 v[174:175], v[174:175], v[66:67]\n\t" \
  "s_mov_b32 s51, 0x300000\n\t" \
  "buffer_store_dwordx4 v[172:175], v13, s[16:19], s51 offen offset:128 sc1\n\t" \
  "v_mfma_f32_16x16x128_f8f6f4 v[184:187], v[112:119], a[240:247], v[184:187]\n\t" \
  "ds_read_b64_tr_b8 v[106:107], v84 offset:33792\n\t" \
  "v_mfma_f32_16x16x128_f8f6f4 v[180:183], v[112:119], a[232:239], v[180:183]\n\t" \
  "ds_read_b64_tr_b8 v[108:109], v100 offset:0\n\t" \
  "v_mfma_f32_16x16x128_f8f6f4 v[176:179], v[112:119], a[224:231], v[176:179]\n\t" \
  "ds_read_b64_tr_b8 v[110:111], v100 offset:1024\n\t" \
  "s_waitcnt lgkmcnt(4)\n\t" \
  "v_mfma_f32_16x16x128_f8f6f4 v[192:195], v[120:127], a[96:103], v[192:195]\n\t" \
  "ds_read_b64_tr_b8 v[112:113], v84 offset:40960\n\t" \
  "v_mfma_f32_16x16x128_f8f6f4 v[196:199], v[120:127], a[104:111], v[196:199]\n\t" \
  "ds_read_b64_tr_b8 v[114:115], v84 offset:41984\n\t" \
  "v_mfma_f32_16x16x128_f8f6f4 v[200:203], v[120:127], a[112:119], v[200:203]\n\t" \
  "ds_read_b64_tr_b8 v[116:117], v100 offset:8192\n\t" \
  "v_mfma_f32_16x16x128_f8f6f4 v[204:207], v[120:127], a[120:127], v[204:207]\n\t" \
  "ds_read_b64_tr_b8 v[118:119], v100 offset:9216\n\t" \
  "s_waitcnt lgkmcnt(4)\n\t" \
  "v_mfma_f32_16x16x128_f8f6f4 v[204:207], v[104:111], a[152:159], v[204:207]\n\t" \
  "ds_read_b64_tr_b8 v[120:121], v84 offset:49152\n\t" \
  "v_pk_mul_f32 v[176:177], v[176:177], v[66:67]\n\t" \
  "v_pk_mul_f32 v[178:179], v[178:179], v[66:67]\n\t" \
  "s_mov_b32 s52, 0x0\n\t" \
  "buffer_store_dwordx4 v[176:179], v13, s[16:19], s52 offen offset:192 sc1\n\t" \
  "v_mfma_f32_16x16x128_f8f6f4 v[200:203], v[104:111], a[144:151], v[200:203]\n\t" \
  "ds_read_b64_tr_b8 v[122:123], v84 offset:50176\n\t" \
  "v_mfma_f32_16x16x128_f8f6f4 v[196:199], v[104:111], a[136:143], v[196:199]\n\t" \
  "ds_read_b64_tr_b8 v[124:125], v100 offset:16384\n\t" \
  "v_mfma_f32_16x16x128_f8f6f4 v[192:195], v[104:111], a[128:135], v[192:195]\n\t" \
  "ds_read_b64_tr_b8 v[126:127], v100 offset:17408\n\t" \
  "s_waitcnt lgkmcnt(4)\n\t" \
  "v_mfma_f32_16x16x128_f8f6f4 v[192:195], v[112:119], a[160:167], v[192:195]\n\t" \
  "ds_read_b64_tr_b8 v[104:105], v84 offset:57344\n\t" \
  "v_pk_mul_f32 v[180:181], v[180:181], v[66:67]\n\t" \
  "v_pk_mul_f32 v[182:183], v[182:183], v[66:67]\n\t" \
  "s_mov_b32 s53, 0x100000\n\t" \
  "buffer_store_dwordx4 v[180:183], v13, s[16:19], s53 offen offset:192 sc1\n\t" \
  "v_mfma_f32_16x16x128_f8f6f4 v[196:199], v[112:119], a[168:175], v[196:199]\n\t" \
  "ds_read_b64_tr_b8 v[106:107], v84 offset:58368\n\t" \
  "v_mfma_f32_16x16x128_f8f6f4 v[200:203], v[112:119], a[176:183], v[200:203]\n\t" \
  "ds_read_b64_tr_b8 v[108:109], v100 offset:24576\n\t" \
  "v_mfma_f32_16x16x128_f8f6f4 v[204:207], v[112:119], a[184:191], v[204:207]\n\t" \
  "ds_read_b64_tr_b8 v[110:111], v100 offset:25600\n\t" \
  "s_waitcnt lgkmcnt(4)\n\t" \
  "v_mfma_f32_16x16x128_f8f6f4 v[204:207], v[120:127], a[216:223], v[204:207]\n\t" \
  "ds_read_b64_tr_b8 v[112:113], v85 offset:24576\n\t" \
  "v_pk_mul_f32 v[184:185], v[184:185], v[66:67]\n\t" \
  "v_pk_mul_f32 v[186:187], v[186:187], v[66:67]\n\t" \
  "s_mov_b32 s54, 0x200000\n\t" \
  "buffer_store_dwordx4 v[184:187], v13, s[16:19], s54 offen offset:192 sc1\n\t" \
  "v_mfma_f32_16x16x128_f8f6f4 v[200:203], v[120:127], a[208:215], v[200:203]\n\t" \
  "ds_read_b64_tr_b8 v[114:115], v85 offset:25600\n\t" \
  "v_mfma_f32_16x16x128_f8f6f4 v[196:199], v[120:127], a[200:207], v[196:199]\n\t" \
  "ds_read_b64_tr_b8 v[116:117], v93 offset:57344\n\t" \
  "v_mfma_f32_16x16x128_f8f6f4 v[192:195], v[120:127], a[192:199], v[192:195]\n\t" \
  "ds_read_b64_tr_b8 v[118:119], v93 offset:58368\n\t" \
  "s_waitcnt lgkmcnt(4)\n\t" \
  "v_mfma_f32_16x16x128_f8f6f4 v[192:195], v[104:111], a[224:231], v[192:195]\n\t" \
  "ds_read_b64_tr_b8 v[120:121], v85 offset:32768\n\t" \
  "v_pk_mul_f32 v[188:189], v[188:189], v[66:67]\n\t" \
  "v_pk_mul_f32 v[190:191], v[190:191], v[66:67]\n\t" \
  "s_mov_b32 s55, 0x300000\n\t" \
  "buffer_store_dwordx4 v[188:191], v13, s[16:19], s55 offen offset:192 sc1\n\t" \
  "v_mfma_f32_16x16x128_f8f6f4 v[196:199], v[104:111], a[232:239], v[196:199]\n\t" \
  "ds_read_b64_tr_b8 v[122:123], v85 offset:33792\n\t" \
  "v_mfma_f32_16x16x128_f8f6f4 v[200:203], v[104:111], a[240:247], v[200:203]\n\t" \
  "ds_read_b64_tr_b8 v[124:125], v101 offset:0\n\t" \
  "v_mfma_f32_16x16x128_f8f6f4 v[204:207], v[104:111], a[248:255], v[204:207]\n\t" \
  "ds_read_b64_tr_b8 v[126:127], v101 offset:1024\n\t" \
  "s_waitcnt lgkmcnt(4)\n\t" \
  "v_mfma_f32_16x16x128_f8f6f4 v[220:223], v[112:119], a[120:127], v[220:223]\n\t" \
  "ds_read_b64_tr_b8 v[104:105], v85 offset:40960\n\t" \
  "v_mfma_f32_16x16x128_f8f6f4 v[216:219], v[112:119], a[112:119], v[216:219]\n\t" \
  "ds_read_b64_tr_b8 v[106:107], v85 offset:41984\n\t" \
  "v_mfma_f32_16x16x128_f8f6f4 v[212:215], v[112:119], a[104:111], v[212:215]\n\t" \
  "ds_read_b64_tr_b8 v[108:109], v101 offset:8192\n\t" \
  "v_mfma_f32_16x16x128_f8f6f4 v[208:211], v[112:119], a[96:103], v[208:211]\n\t" \
  "ds_read_b64_tr_b8 v[110:111], v101 offset:9216\n\t" \
  "s_waitcnt lgkmcnt(4)\n\t" \
  "v_mfma_f32_16x16x128_f8f6f4 v[208:211], v[120:127], a[128:135], v[208:211]\n\t" \
  "ds_read_b64_tr_b8 v[112:113], v85 offset:49152\n\t" \
  "v_pk_mul_f32 v[192:193], v[192:193], v[66:67]\n\t" \
  "v_pk_mul_f32 v[194:195], v[194:195], v[66:67]\n\t" \
  "s_mov_b32 s48, 0x0\n\t" \
  "buffer_store_dwordx4 v[192:195], v13, s[16:19], s48 offen offset:256 sc1\n\t" \
  "v_mfma_f32_16x16x128_f8f6f4 v[212:215], v[120:127], a[136:143], v[212:215]\n\t" \
  "ds_read_b64_tr_b8 v[114:115], v85 offset:50176\n\t" \
  "v_mfma_f32_16x16x128_f8f6f4 v[216:219], v[120:127], a[144:151], v[216:219]\n\t" \
  "ds_read_b64_tr_b8 v[116:117], v101 offset:16384\n\t" \
  "v_mfma_f32_16x16x128_f8f6f4 v[220:223], v[120:127], a[152:159], v[220:223]\n\t" \
  "ds_read_b64_tr_b8 v[118:119], v101 offset:17408\n\t" \
  "s_waitcnt lgkmcnt(4)\n\t" \
  "v_mfma_f32_16x16x128_f8f6f4 v[220:223], v[104:111], a[184:191], v[220:223]\n\t" \
  "ds_read_b64_tr_b8 v[120:121], v85 offset:57344\n\t" \
  "v_pk_mul_f32 v[196:197], v[196:197], v[66:67]\n\t" \
  "v_pk_mul_f32 v[198:199], v[198:199], v[66:67]\n\t" \
  "s_mov_b32 s49, 0x100000\n\t" \
  "buffer_store_dwordx4 v[196:199], v13, s[16:19], s49 offen offset:256 sc1\n\t" \
  "v_mfma_f32_16x16x128_f8f6f4 v[216:219], v[104:111], a[176:183], v[216:219]\n\t" \
  "ds_read_b64_tr_b8 v[122:123], v85 offset:58368\n\t" \
  "v_mfma_f32_16x16x128_f8f6f4 v[212:215], v[104:111], a[168:175], v[212:215]\n\t" \
  "ds_read_b64_tr_b8 v[124:125], v101 offset:24576\n\t" \
  "v_mfma_f32_16x16x128_f8f6f4 v[208:211], v[104:111], a[160:167], v[208:211]\n\t" \
  "ds_read_b64_tr_b8 v[126:127], v101 offset:25600\n\t" \
  "s_waitcnt lgkmcnt(4)\n\t" \
  "v_mfma_f32_16x16x128_f8f6f4 v[208:211], v[112:119], a[192:199], v[208:211]\n\t" \
  "ds_read_b64_tr_b8 v[104:105], v86 offset:24576\n\t" \
  "v_pk_mul_f32 v[200:201], v[200:201], v[66:67]\n\t" \
  "v_pk_mul_f32 v[202:203], v[202:203], v[66:67]\n\t" \
  "s_mov_b32 s50, 0x200000\n\t" \
  "buffer_store_dwordx4 v[200:203], v13, s[16:19], s50 offen offset:256 sc1\n\t" \
  "v_mfma_f32_16x16x128_f8f6f4 v[212:215], v[112:119], a[200:207], v[212:215]\n\t" \
  "ds_read_b64_tr_b8 v[106:107], v86 offset:25600\n\t" \
  "v_mfma_f32_16x16x128_f8f6f4 v[216:219], v[112:119], a[208:215], v[216:219]\n\t" \
  "ds_read_b64_tr_b8 v[108:109], v94 offset:57344\n\t" \
  "v_mfma_f32_16x16x128_f8f6f4 v[220:223], v[112:119], a[216:223], v[220:223]\n\t" \
  "ds_read_b64_tr_b8 v[110:111], v94 offset:58368\n\t" \
  "s_waitcnt lgkmcnt(4)\n\t" \
  "v_mfma_f32_16x16x128_f8f6f4 v[220:223], v[120:127], a[248:255], v[220:223]\n\t" \
  "ds_read_b64_tr_b8 v[112:113], v86 offset:32768\n\t" \
  "v_pk_mul_f32 v[204:205], v[204:205], v[66:67]\n\t" \
  "v_pk_mul_f32 v[206:207], v[206:207], v[66:67]\n\t" \
  "s_mov_b32 s51, 0x300000\n\t" \
  "buffer_store_dwordx4 v[204:207], v13, s[16:19], s51 offen offset:256 sc1\n\t" \
  "v_mfma_f32_16x16x128_f8f6f4 v[216:219], v[120:127], a[240:247], v[216:219]\n\t" \
  "ds_read_b64_tr_b8 v[114:115], v86 offset:33792\n\t" \
  "v_mfma_f32_16x16x128_f8f6f4 v[212:215], v[120:127], a[232:239], v[212:215]\n\t" \
  "ds_read_b64_tr_b8 v[116:117], v102 offset:0\n\t" \
  "v_mfma_f32_16x16x128_f8f6f4 v[208:211], v[120:127], a[224:231], v[208:211]\n\t" \
  "ds_read_b64_tr_b8 v[118:119], v102 offset:1024\n\t" \
  "s_waitcnt lgkmcnt(4)\n\t" \
  "v_mfma_f32_16x16x128_f8f6f4 v[224:227], v[104:111], a[96:103], v[224:227]\n\t" \
  "ds_read_b64_tr_b8 v[120:121], v86 offset:40960\n\t" \
  "v_mfma_f32_16x16x128_f8f6f4 v[228:231], v[104:111], a[104:111], v[228:231]\n\t" \
  "ds_read_b64_tr_b8 v[122:123], v86 offset:41984\n\t" \
  "v_mfma_f32_16x16x128_f8f6f4 v[232:235], v[104:111], a[112:119], v[232:235]\n\t" \
  "ds_read_b64_tr_b8 v[124:125], v102 offset:8192\n\t" \
  "v_mfma_f32_16x16x128_f8f6f4 v[236:239], v[104:111], a[120:127], v[236:239]\n\t" \
  "ds_read_b64_tr_b8 v[126:127], v102 offset:9216\n\t" \
  "s_waitcnt lgkmcnt(4)\n\t" \
  "v_mfma_f32_16x16x128_f8f6f4 v[236:239], v[112:119], a[152:159], v[236:239]\n\t" \
  "ds_read_b64_tr_b8 v[104:105], v86 offset:49152\n\t" \
  "v_pk_mul_f32 v[208:209], v[208:209], v[66:67]\n\t" \
  "v_pk_mul_f32 v[210:211], v[210:211], v[66:67]\n\t" \
  "s_mov_b32 s52, 0x0\n\t" \
  "buffer_store_dwordx4 v[208:211], v13, s[16:19], s52 offen offset:320 sc1\n\t" \
  "v_mfma_f32_16x16x128_f8f6f4 v[232:235], v[112:119], a[144:151], v[232:235]\n\t" \
  "ds_read_b64_tr_b8 v[106:107], v86 offset:50176\n\t" \
  "v_mfma_f32_16x16x128_f8f6f4 v[228:231], v[112:119], a[136:143], v[228:231]\n\t" \
  "ds_read_b64_tr_b8 v[108:109], v102 offset:16384\n\t" \
  "v_mfma_f32_16x16x128_f8f6f4 v[224:227], v[112:119], a[128:135], v[224:227]\n\t" \
  "ds_read_b64_tr_b8 v[110:111], v102 offset:17408\n\t" \
  "s_waitcnt lgkmcnt(4)\n\t" \
  "v_mfma_f32_16x16x128_f8f6f4 v[224:227], v[120:127], a[160:167], v[224:227]\n\t" \
  "ds_read_b64_tr_b8 v[112:113], v86 offset:57344\n\t" \
  "v_pk_mul_f32 v[212:213], v[212:213], v[66:67]\n\t" \
  "v_pk_mul_f32 v[214:215], v[214:215], v[66:67]\n\t" \
  "s_mov_b32 s53, 0x100000\n\t" \
  "buffer_store_dwordx4 v[212:215], v13, s[16:19], s53 offen offset:320 sc1\n\t" \
  "v_mfma_f32_16x16x128_f8f6f4 v[228:231], v[120:127], a[168:175], v[228:231]\n\t" \
  "ds_read_b64_tr_b8 v[114:115], v86 offset:58368\n\t" \
  "v_mfma_f32_16x16x128_f8f6f4 v[232:235], v[120:127], a[176:183], v[232:235]\n\t" \
  "ds_read_b64_tr_b8 v[116:117], v102 offset:24576\n\t" \
  "v_mfma_f32_16x16x128_f8f6f4 v[236:239], v[120:127], a[184:191], v[236:239]\n\t" \
  "ds_read_b64_tr_b8 v[118:119], v102 offset:25600\n\t" \
  "s_waitcnt lgkmcnt(4)\n\t" \
  "v_mfma_f32_16x16x128_f8f6f4 v[236:239], v[104:111], a[216:223], v[236:239]\n\t" \
  "ds_read_b64_tr_b8 v[120:121], v87 offset:24576\n\t" \
  "v_pk_mul_f32 v[216:217], v[216:217], v[66:67]\n\t" \
  "v_pk_mul_f32 v[218:219], v[218:219], v[66:67]\n\t" \
  "s_mov_b32 s54, 0x200000\n\t" \
  "buffer_store_dwordx4 v[216:219], v13, s[16:19], s54 offen offset:320 sc1\n\t" \
  "v_mfma_f32_16x16x128_f8f6f4 v[232:235], v[104:111], a[208:215], v[232:235]\n\t" \
  "ds_read_b64_tr_b8 v[122:123], v87 offset:25600\n\t" \
  "v_mfma_f32_16x16x128_f8f6f4 v[228:231], v[104:111], a[200:207], v[228:231]\n\t" \
  "ds_read_b64_tr_b8 v[124:125], v95 offset:57344\n\t" \
  "v_mfma_f32_16x16x128_f8f6f4 v[224:227], v[104:111], a[192:199], v[224:227]\n\t" \
  "ds_read_b64_tr_b8 v[126:127], v95 offset:58368\n\t" \
  "s_waitcnt lgkmcnt(4)\n\t" \
  "v_mfma_f32_16x16x128_f8f6f4 v[224:227], v[112:119], a[224:231], v[224:227]\n\t" \
  "ds_read_b64_tr_b8 v[104:105], v87 offset:32768\n\t" \
  "v_pk_mul_f32 v[220:221], v[220:221], v[66:67]\n\t" \
  "v_pk_mul_f32 v[222:223], v[222:223], v[66:67]\n\t" \
  "s_mov_b32 s55, 0x300000\n\t" \
  "buffer_store_dwordx4 v[220:223], v13, s[16:19], s55 offen offset:320 sc1\n\t" \
  "v_mfma_f32_16x16x128_f8f6f4 v[228:231], v[112:119], a[232:239], v[228:231]\n\t" \
  "ds_read_b64_tr_b8 v[106:107], v87 offset:33792\n\t" \
  "v_mfma_f32_16x16x128_f8f6f4 v[232:235], v[112:119], a[240:247], v[232:235]\n\t" \
  "ds_read_b64_tr_b8 v[108:109], v103 offset:0\n\t" \
  "v_mfma_f32_16x16x128_f8f6f4 v[236:239], v[112:119], a[248:255], v[236:239]\n\t" \
  "ds_read_b64_tr_b8 v[110:111], v103 offset:1024\n\t" \
  "s_waitcnt lgkmcnt(4)\n\t" \
  "v_mfma_f32_16x16x128_f8f6f4 v[252:255], v[120:127], a[120:127], v[252:255]\n\t" \
  "ds_read_b64_tr_b8 v[112:113], v87 offset:40960\n\t" \
  "v_mfma_f32_16x16x128_f8f6f4 v[248:251], v[120:127], a[112:119], v[248:251]\n\t" \
  "ds_read_b64_tr_b8 v[114:115], v87 offset:41984\n\t" \
  "v_mfma_f32_16x16x128_f8f6f4 v[244:247], v[120:127], a[104:111], v[244:247]\n\t" \
  "ds_read_b64_tr_b8 v[116:117], v103 offset:8192\n\t" \
  "v_mfma_f32_16x16x128_f8f6f4 v[240:243], v[120:127], a[96:103], v[240:243]\n\t" \
  "ds_read_b64_tr_b8 v[118:119], v103 offset:9216\n\t" \
  "s_waitcnt lgkmcnt(4)\n\t" \
  "v_mfma_f32_16x16x128_f8f6f4 v[240:243], v[104:111], a[128:135], v[240:243]\n\t" \
  "ds_read_b64_tr_b8 v[120:121], v87 offset:49152\n\t" \
  "v_pk_mul_f32 v[224:225], v[224:225], v[66:67]\n\t" \
  "v_pk_mul_f32 v[226:227], v[226:227], v[66:67]\n\t" \
  "s_mov_b32 s48, 0x0\n\t" \
  "buffer_store_dwordx4 v[224:227], v13, s[16:19], s48 offen offset:384 sc1\n\t" \
  "v_mfma_f32_16x16x128_f8f6f4 v[244:247], v[104:111], a[136:143], v[244:247]\n\t" \
  "ds_read_b64_tr_b8 v[122:123], v87 offset:50176\n\t" \
  "v_mfma_f32_16x16x128_f8f6f4 v[248:251], v[104:111], a[144:151], v[248:251]\n\t" \
  "ds_read_b64_tr_b8 v[124:125], v103 offset:16384\n\t" \
  "v_mfma_f32_16x16x128_f8f6f4 v[252:255], v[104:111], a[152:159], v[252:255]\n\t" \
  "ds_read_b64_tr_b8 v[126:127], v103 offset:17408\n\t" \
  "s_waitcnt lgkmcnt(4)\n\t" \
  "v_mfma_f32_16x16x128_f8f6f4 v[252:255], v[112:119], a[184:191], v[252:255]\n\t" \
  "ds_read_b64_tr_b8 v[104:105], v87 offset:57344\n\t" \
  "v_pk_mul_f32 v[228:229], v[228:229], v[66:67]\n\t" \
  "v_pk_mul_f32 v[230:231], v[230:231], v[66:67]\n\t" \
  "s_mov_b32 s49, 0x100000\n\t" \
  "buffer_store_dwordx4 v[228:231], v13, s[16:19], s49 offen offset:384 sc1\n\t" \
  "v_mfma_f32_16x16x128_f8f6f4 v[248:251], v[112:119], a[176:183], v[248:251]\n\t" \
  "ds_read_b64_tr_b8 v[106:107], v87 offset:58368\n\t" \
  "v_mfma_f32_16x16x128_f8f6f4 v[244:247], v[112:119], a[168:175], v[244:247]\n\t" \
  "ds_read_b64_tr_b8 v[108:109], v103 offset:24576\n\t" \
  "v_mfma_f32_16x16x128_f8f6f4 v[240:243], v[112:119], a[160:167], v[240:243]\n\t" \
  "ds_read_b64_tr_b8 v[110:111], v103 offset:25600\n\t" \
  "s_waitcnt lgkmcnt(4)\n\t" \
  "v_mfma_f32_16x16x128_f8f6f4 v[240:243], v[120:127], a[192:199], v[240:243]\n\t" \
  "v_pk_mul_f32 v[232:233], v[232:233], v[66:67]\n\t" \
  "v_pk_mul_f32 v[234:235], v[234:235], v[66:67]\n\t" \
  "s_mov_b32 s50, 0x200000\n\t" \
  "buffer_store_dwordx4 v[232:235], v13, s[16:19], s50 offen offset:384 sc1\n\t" \
  "v_mfma_f32_16x16x128_f8f6f4 v[244:247], v[120:127], a[200:207], v[244:247]\n\t" \
  "v_mfma_f32_16x16x128_f8f6f4 v[248:251], v[120:127], a[208:215], v[248:251]\n\t" \
  "v_mfma_f32_16x16x128_f8f6f4 v[252:255], v[120:127], a[216:223], v[252:255]\n\t" \
  "s_waitcnt lgkmcnt(0)\n\t" \
  "v_mfma_f32_16x16x128_f8f6f4 v[252:255], v[104:111], a[248:255], v[252:255]\n\t" \
  "v_pk_mul_f32 v[236:237], v[236:237], v[66:67]\n\t" \
  "v_pk_mul_f32 v[238:239], v[238:239], v[66:67]\n\t" \
  "s_mov_b32 s51, 0x300000\n\t" \
  "buffer_store_dwordx4 v[236:239], v13, s[16:19], s51 offen offset:384 sc1\n\t" \
  "v_mfma_f32_16x16x128_f8f6f4 v[248:251], v[104:111], a[240:247], v[248:251]\n\t" \
  "v_mfma_f32_16x16x128_f8f6f4 v[244:247], v[104:111], a[232:239], v[244:247]\n\t" \
  "v_mfma_f32_16x16x128_f8f6f4 v[240:243], v[104:111], a[224:231], v[240:243]\n\t" \
  "s_nop 15\n\t" \
  "s_nop 7\n\t" \
  "v_pk_mul_f32 v[240:241], v[240:241], v[66:67]\n\t" \
  "v_pk_mul_f32 v[242:243], v[242:243], v[66:67]\n\t" \
  "s_mov_b32 s52, 0x0\n\t" \
  "buffer_store_dwordx4 v[240:243], v13, s[16:19], s52 offen offset:448 sc1\n\t" \
  "v_pk_mul_f32 v[244:245], v[244:245], v[66:67]\n\t" \
  "v_pk_mul_f32 v[246:247], v[246:247], v[66:67]\n\t" \
  "s_mov_b32 s53, 0x100000\n\t" \
  "buffer_store_dwordx4 v[244:247], v13, s[16:19], s53 offen offset:448 sc1\n\t" \
  "v_pk_mul_f32 v[248:249], v[248:249], v[66:67]\n\t" \
  "v_pk_mul_f32 v[250:251], v[250:251], v[66:67]\n\t" \
  "s_mov_b32 s54, 0x200000\n\t" \
  "buffer_store_dwordx4 v[248:251], v13, s[16:19], s54 offen offset:448 sc1\n\t" \
  "v_pk_mul_f32 v[252:253], v[252:253], v[66:67]\n\t" \
  "v_pk_mul_f32 v[254:255], v[254:255], v[66:67]\n\t" \
  "s_mov_b32 s55, 0x300000\n\t" \
  "buffer_store_dwordx4 v[252:255], v13, s[16:19], s55 offen offset:448 sc1\n\t" \
  "s_endpgm\n\t" \
  "Lpoll_2_%=:\n\t" \
  "s_mov_b32 exec_lo, s69\n\t" \
  "s_mov_b32 exec_hi, 1\n\t" \
  "global_load_dwordx4 v[68:71], v10, s[6:7] sc0 sc1\n\t" \
  "s_mov_b64 exec, -1\n\t" \
  "s_waitcnt vmcnt(0)\n\t" \
  "v_min_u32 v68, v68, v69\n\t" \
  "v_min3_u32 v68, v68, v70, v71\n\t" \
  "v_cmp_le_u32 vcc, s57, v68\n\t" \
  "s_and_b32 s64, vcc_lo, s69\n\t" \
  "s_or_b32 s56, s56, s64\n\t" \
  "s_and_b32 s64, s56, s62\n\t" \
  "s_cmp_eq_u32 s64, s62\n\t" \
  "s_cbranch_scc1 Lret_2_%=\n\t" \
  "s_bitcmp1_b32 vcc_hi, 0\n\t" \
  "s_cbranch_scc1 Lfail_2_%=\n\t" \
  "s_memrealtime s[64:65]\n\t" \
  "s_waitcnt lgkmcnt(0)\n\t" \
  "s_sub_u32 s64, s64, s58\n\t" \
  "s_subb_u32 s65, s65, s59\n\t" \
  "s_cmp_lg_u32 s65, 0\n\t" \
  "s_cbranch_scc1 Lfail_2_%=\n\t" \
  "s_cmp_gt_u32 s64, 50000000\n\t" \
  "s_cbranch_scc1 Lfail_2_%=\n\t" \
  "s_sleep 4\n\t" \
  "s_branch Lpoll_2_%=\n\t" \
  "Lfail_2_%=:\n\t" \
  "v_mov_b32 v68, s57\n\t" \
  "v_mov_b32 v69, s57\n\t" \
  "v_mov_b32 v70, s57\n\t" \
  "v_mov_b32 v71, s57\n\t" \
  "s_mov_b32 exec_lo, 0\n\t" \
  "s_mov_b32 exec_hi, 1\n\t" \
  "global_store_dwordx4 v10, v[68:71], s[6:7] sc0 sc1\n\t" \
  "s_mov_b64 exec, -1\n\t" \
  "s_waitcnt vmcnt(0)\n\t" \
  "s_mov_b32 s63, 0x7fc00000\n\t" \
  "s_mov_b32 s56, -1\n\t" \
  "Lret_2_%=:\n\t" \
  "s_setpc_b64 s[60:61]\n\t" \
  "Lend_2_%=:\n\t" \
  "Lvar_3_%=:\n\t" \
  "s_lshl_b32 s64, s4, 15\n\t" \
  "s_add_u32 s64, s64, s78\n\t" \
  "s_lshl_b32 s84, s4, 13\n\t" \
  "v_lshrrev_b32 v17, 3, v1\n\t" \
  "v_lshrrev_b32 v18, 4, v1\n\t" \
  "v_and_b32 v19, 7, v1\n\t" \
  "v_xor_b32 v18, v18, v19\n\t" \
  "v_lshlrev_b32 v18, 4, v18\n\t" \
  "v_lshl_or_b32 v5, v17, 9, v18\n\t" \
  "v_add_u32 v5, s64, v5\n\t" \
  "v_xor_b32 v6, 64, v5\n\t" \
  "v_lshlrev_b32 v4, 4, v1\n\t" \
  "s_lshl_b32 s64, s72, 15\n\t" \
  "s_lshl_b32 s65, s4, 13\n\t" \
  "s_add_u32 s64, s64, s65\n\t" \
  "s_add_u32 s12, s6, s64\n\t" \
  "s_addc_u32 s13, s7, 0\n\t" \
  "s_and_b32 s13, s13, 0xffff\n\t" \
  "s_sub_u32 s14, 0x400000, s64\n\t" \
  "s_mov_b32 s15, 0x20000\n\t" \
  "s_lshl_b32 s64, s72, 24\n\t" \
  "s_lshl_b32 s65, s4, 22\n\t" \
  "s_add_u32 s64, s64, s65\n\t" \
  "s_lshl_b32 s65, s74, 2\n\t" \
  "s_add_u32 s64, s64, s65\n\t" \
  "s_add_u32 s16, s16, s64\n\t" \
  "s_addc_u32 s17, s17, 0\n\t" \
  "s_and_b32 s17, s17, 0xffff\n\t" \
  "s_sub_u32 s18, 0x400000, s65\n\t" \
  "s_mov_b32 s19, 0x20000\n\t" \
  "s_bfe_u32 s26, s27, 0x4000a\n\t" \
  "s_sub_u32 s64, 6, s75\n\t" \
  "s_and_b32 s64, s64, 7\n\t" \
  "s_mul_i32 s65, s77, 7\n\t" \
  "s_add_u32 s65, s65, s64\n\t" \
  "s_add_u32 s65, s65, 4\n\t" \
  "s_cmp_eq_u32 s64, 7\n\t" \
  "s_cselect_b32 s65, s77, s65\n\t" \
  "s_lshl_b32 s65, s65, 16\n\t" \
  "s_lshl_b32 s82, s76, 11\n\t" \
  "s_add_u32 s65, s65, s82\n\t" \
  "v_add_u32 v9, s65, v4\n\t" \
  "s_lshr_b32 s82, s76, 2\n\t" \
  "s_lshl_b32 s82, s82, 9\n\t" \
  "s_lshl_b32 s83, s77, 7\n\t" \
  "s_add_u32 s82, s82, s83\n\t" \
  "s_lshl_b32 s83, s64, 4\n\t" \
  "s_add_u32 s82, s82, s83\n\t" \
  "s_and_b32 s83, s76, 3\n\t" \
  "s_lshl_b32 s83, s83, 2\n\t" \
  "s_add_u32 s82, s82, s83\n\t" \
  "s_add_u32 s82, s82, 0x400000\n\t" \
  "s_lshl_b32 s64, s72, 2\n\t" \
  "s_add_u32 s64, s64, s4\n\t" \
  "s_lshl_b32 s64, s64, 9\n\t" \
  "s_add_u32 s64, s64, 0x400000\n\t" \
  "v_lshl_add_u32 v10, v1, 4, s64\n\t" \
  "v_cmp_lt_u32 vcc, 31, v1\n\t" \
  "v_mov_b32 v18, 0x401000\n\t" \
  "s_nop 0\n\t" \
  "v_cndmask_b32 v10, v10, v18, vcc\n\t" \
  "v_and_b32 v17, 15, v1\n\t" \
  "v_lshrrev_b32 v18, 4, v1\n\t" \
  "v_lshlrev_b32 v19, 4, v18\n\t" \
  "v_lshl_or_b32 v13, v17, 16, v19\n\t" \
  "v_lshl_add_u32 v15, v17, 2, v18\n\t" \
  "v_lshlrev_b32 v15, 2, v15\n\t" \
  "v_lshrrev_b32 v19, 1, v17\n\t" \
  "v_lshl_add_u32 v19, v18, 4, v19\n\t" \
  "v_lshlrev_b32 v19, 7, v19\n\t" \
  "v_bfe_u32 v20, v1, 2, 3\n\t" \
  "v_lshl_or_b32 v19, v20, 4, v19\n\t" \
  "v_and_b32 v20, 1, v1\n\t" \
  "v_lshl_or_b32 v14, v20, 3, v19\n\t" \
  "v_lshrrev_b32 v17, 2, v1\n\t" \
  "v_min_u32 v17, 6, v17\n\t" \
  "v_lshlrev_b32 v17, 16, v17\n\t" \
  "v_and_b32 v18, 3, v1\n\t" \
  "v_lshl_or_b32 v17, v18, 7, v17\n\t" \
  "s_lshl_b32 s64, s73, 9\n\t" \
  "v_add_u32 v16, s64, v17\n\t" \
  "v_and_b32 v17, 15, v1\n\t" \
  "v_lshrrev_b32 v18, 4, v1\n\t" \
  "s_lshl_b32 s64, s72, 2\n\t" \
  "v_add_u32 v18, s64, v18\n\t" \
  "v_lshrrev_b32 v19, 1, v17\n\t" \
  "v_xor_b32 v18, v18, v19\n\t" \
  "v_lshlrev_b32 v18, 4, v18\n\t" \
  "v_lshl_or_b32 v17, v17, 7, v18\n\t" \
  "s_add_u32 s64, s84, 0x10000\n\t" \
  "v_add_u32 v20, s64, v17\n\t" \
  "v_xor_b32 v21, 64, v20\n\t" \
  "s_add_u32 s64, s84, 0x20000\n\t" \
  "v_add_u32 v22, s64, v17\n\t" \
  "v_xor_b32 v23, 64, v22\n\t" \
  "s_add_u32 s40, s25, 1\n\t" \
  "s_and_b32 s40, s40, 7\n\t" \
  "s_lshl_b32 s40, s40, 23\n\t" \
  "s_add_u32 s41, s25, 2\n\t" \
  "s_and_b32 s41, s41, 7\n\t" \
  "s_lshl_b32 s41, s41, 23\n\t" \
  "s_add_u32 s42, s25, 3\n\t" \
  "s_and_b32 s42, s42, 7\n\t" \
  "s_lshl_b32 s42, s42, 23\n\t" \
  "s_add_u32 s43, s25, 4\n\t" \
  "s_and_b32 s43, s43, 7\n\t" \
  "s_lshl_b32 s43, s43, 23\n\t" \
  "s_add_u32 s44, s25, 5\n\t" \
  "s_and_b32 s44, s44, 7\n\t" \
  "s_lshl_b32 s44, s44, 23\n\t" \
  "s_add_u32 s45, s25, 6\n\t" \
  "s_and_b32 s45, s45, 7\n\t" \
  "s_lshl_b32 s45, s45, 23\n\t" \
  "s_add_u32 s46, s25, 7\n\t" \
  "s_and_b32 s46, s46, 7\n\t" \
  "s_lshl_b32 s46, s46, 23\n\t" \
  "v_xor_b32 v80, 0x0, v14\n\t" \
  "v_xor_b32 v81, 0x10, v14\n\t" \
  "v_xor_b32 v82, 0x20, v14\n\t" \
  "v_xor_b32 v83, 0x30, v14\n\t" \
  "v_xor_b32 v84, 0x40, v14\n\t" \
  "v_xor_b32 v85, 0x50, v14\n\t" \
  "v_xor_b32 v86, 0x60, v14\n\t" \
  "v_xor_b32 v87, 0x70, v14\n\t" \
  "v_xor_b32 v88, 0x10000, v14\n\t" \
  "v_xor_b32 v89, 0x10010, v14\n\t" \
  "v_xor_b32 v90, 0x10020, v14\n\t" \
  "v_xor_b32 v91, 0x10030, v14\n\t" \
  "v_xor_b32 v92, 0x10040, v14\n\t" \
  "v_xor_b32 v93, 0x10050, v14\n\t" \
  "v_xor_b32 v94, 0x10060, v14\n\t" \
  "v_xor_b32 v95, 0x10070, v14\n\t" \
  "v_xor_b32 v96, 0x20000, v14\n\t" \
  "v_xor_b32 v97, 0x20010, v14\n\t" \
  "v_xor_b32 v98, 0x20020, v14\n\t" \
  "v_xor_b32 v99, 0x20030, v14\n\t" \
  "v_xor_b32 v100, 0x20040, v14\n\t" \
  "v_xor_b32 v101, 0x20050, v14\n\t" \
  "v_xor_b32 v102, 0x20060, v14\n\t" \
  "v_xor_b32 v103, 0x20070, v14\n\t" \
  "s_mov_b32 s56, 0\n\t" \
  "s_mov_b32 s63, 0\n\t" \
  "s_waitcnt lgkmcnt(0)\n\t" \
  "s_add_u32 s30, s28, s82\n\t" \
  "s_addc_u32 s31, s29, 0\n\t" \
  "s_add_u32 m0, s24, 0x18000\n\t" \
  "s_add_u32 s52, s47, 0x0\n\t" \
  "buffer_load_dwordx4 v3, s[8:11], s52 offen lds\n\t" \
  "s_add_u32 m0, s24, 0x18400\n\t" \
  "s_add_u32 s53, s47, 0x20000\n\t" \
  "buffer_load_dwordx4 v3, s[8:11], s53 offen lds\n\t" \
  "s_add_u32 m0, s84, 0x10000\n\t" \
  "s_mov_b32 s54, 0x0\n\t" \
  "buffer_load_dwordx4 v5, s[20:23], s54 offen lds\n\t" \
  "s_add_u32 m0, s84, 0x10400\n\t" \
  "s_mov_b32 s55, 0x1000\n\t" \
  "buffer_load_dwordx4 v6, s[20:23], s55 offen lds\n\t" \
  "s_add_u32 m0, s84, 0x10800\n\t" \
  "s_mov_b32 s48, 0x2000\n\t" \
  "buffer_load_dwordx4 v5, s[20:23], s48 offen lds\n\t" \
  "s_add_u32 m0, s84, 0x10c00\n\t" \
  "s_mov_b32 s49, 0x3000\n\t" \
  "buffer_load_dwordx4 v6, s[20:23], s49 offen lds\n\t" \
  "s_add_u32 m0, s84, 0x11000\n\t" \
  "s_mov_b32 s50, 0x4000\n\t" \
  "buffer_load_dwordx4 v5, s[20:23], s50 offen lds\n\t" \
  "s_add_u32 m0, s84, 0x11400\n\t" \
  "s_mov_b32 s51, 0x5000\n\t" \
  "buffer_load_dwordx4 v6, s[20:23], s51 offen lds\n\t" \
  "s_add_u32 m0, s84, 0x11800\n\t" \
  "s_mov_b32 s52, 0x6000\n\t" \
  "buffer_load_dwordx4 v5, s[20:23], s52 offen lds\n\t" \
  "s_add_u32 m0, s84, 0x11c00\n\t" \
  "s_mov_b32 s53, 0x7000\n\t" \
  "buffer_load_dwordx4 v6, s[20:23], s53 offen lds\n\t" \
  "s_waitcnt vmcnt(18)\n\t" \
  "v_add_u32 v64, 1, v64\n\t" \
  "s_nop 0\n\t" \
  "v_readfirstlane_b32 s57, v64\n\t" \
  "s_and_b32 s68, s57, 1\n\t" \
  "s_lshl_b32 s68, s68, 21\n\t" \
  "v_add_u32 v63, s68, v9\n\t" \
  "s_waitcnt lgkmcnt(0)\n\t" \
  "s_add_u32 m0, s24, 0x1a000\n\t" \
  "s_add_u32 s54, s47, 0x200000\n\t" \
  "buffer_load_dwordx4 v3, s[8:11], s54 offen lds\n\t" \
  "s_add_u32 m0, s24, 0x1a400\n\t" \
  "s_add_u32 s55, s47, 0x220000\n\t" \
  "buffer_load_dwordx4 v3, s[8:11], s55 offen lds\n\t" \
  "s_waitcnt vmcnt(2)\n\t" \
  "ds_read_b128 a[0:3], v20 offset:0\n\t" \
  "ds_read_b128 a[4:7], v21 offset:0\n\t" \
  "ds_read_b128 a[8:11], v20 offset:2048\n\t" \
  "ds_read_b128 a[12:15], v21 offset:2048\n\t" \
  "ds_read_b128 a[16:19], v20 offset:4096\n\t" \
  "ds_read_b128 a[20:23], v21 offset:4096\n\t" \
  "ds_read_b128 a[24:27], v20 offset:6144\n\t" \
  "ds_read_b128 a[28:31], v21 offset:6144\n\t" \
  "s_waitcnt lgkmcnt(0)\n\t" \
  "s_waitcnt lgkmcnt(0)\n\t" \
  "s_barrier\n\t" \
  "s_mov_b64 exec, 1\n\t" \
  "global_store_dword v65, v64, s[34:35] sc1\n\t" \
  "s_mov_b64 exec, -1\n\t" \
  "ds_read_b64_tr_b8 v[104:105], v80 offset:0\n\t" \
  "ds_read_b64_tr_b8 v[106:107], v80 offset:1024\n\t" \
  "ds_read_b64_tr_b8 v[108:109], v88 offset:32768\n\t" \
  "ds_read_b64_tr_b8 v[110:111], v88 offset:33792\n\t" \
  "ds_read_b64_tr_b8 v[112:113], v81 offset:0\n\t" \
  "ds_read_b64_tr_b8 v[114:115], v81 offset:1024\n\t" \
  "ds_read_b64_tr_b8 v[116:117], v89 offset:32768\n\t" \
  "ds_read_b64_tr_b8 v[118:119], v89 offset:33792\n\t" \
  "s_waitcnt lgkmcnt(4)\n\t" \
  "v_mfma_f32_16x16x128_f8f6f4 v[128:131], v[104:111], a[0:7], 0\n\t" \
  "ds_read_b64_tr_b8 v[120:121], v82 offset:0\n\t" \
  "s_add_u32 m0, s84, 0x20000\n\t" \
  "s_mov_b32 s48, 0x80\n\t" \
  "buffer_load_dwordx4 v5, s[20:23], s48 offen lds\n\t" \
  "v_mfma_f32_16x16x128_f8f6f4 v[132:135], v[104:111], a[8:15], 0\n\t" \
  "ds_read_b64_tr_b8 v[122:123], v82 offset:1024\n\t" \
  "s_add_u32 m0, s84, 0x20400\n\t" \
  "s_mov_b32 s49, 0x1080\n\t" \
  "buffer_load_dwordx4 v6, s[20:23], s49 offen lds\n\t" \
  "v_mfma_f32_16x16x128_f8f6f4 v[136:139], v[104:111], a[16:23], 0\n\t" \
  "ds_read_b64_tr_b8 v[124:125], v90 offset:32768\n\t" \
  "s_add_u32 m0, s84, 0x20800\n\t" \
  "s_mov_b32 s50, 0x2080\n\t" \
  "buffer_load_dwordx4 v5, s[20:23], s50 offen lds\n\t" \
  "v_mfma_f32_16x16x128_f8f6f4 v[140:143], v[104:111], a[24:31], 0\n\t" \
  "ds_read_b64_tr_b8 v[126:127], v90 offset:33792\n\t" \
  "s_add_u32 m0, s84, 0x20c00\n\t" \
  "s_mov_b32 s51, 0x3080\n\t" \
  "buffer_load_dwordx4 v6, s[20:23], s51 offen lds\n\t" \
  "s_waitcnt lgkmcnt(4)\n\t" \
  "v_mfma_f32_16x16x128_f8f6f4 v[156:159], v[112:119], a[24:31], 0\n\t" \
  "ds_read_b64_tr_b8 v[104:105], v83 offset:0\n\t" \
  "s_add_u32 m0, s84, 0x21000\n\t" \
  "s_mov_b32 s52, 0x4080\n\t" \
  "buffer_load_dwordx4 v5, s[20:23], s52 offen lds\n\t" \
  "v_mfma_f32_16x16x128_f8f6f4 v[152:155], v[112:119], a[16:23], 0\n\t" \
  "ds_read_b64_tr_b8 v[106:107], v83 offset:1024\n\t" \
  "s_add_u32 m0, s84, 0x21400\n\t" \
  "s_mov_b32 s53, 0x5080\n\t" \
  "buffer_load_dwordx4 v6, s[20:23], s53 offen lds\n\t" \
  "v_mfma_f32_16x16x128_f8f6f4 v[148:151], v[112:119], a[8:15], 0\n\t" \
  "ds_read_b64_tr_b8 v[108:109], v91 offset:32768\n\t" \
  "s_add_u32 m0, s84, 0x21800\n\t" \
  "s_mov_b32 s54, 0x6080\n\t" \
  "buffer_load_dwordx4 v5, s[20:23], s54 offen lds\n\t" \
  "v_mfma_f32_16x16x128_f8f6f4 v[144:147], v[112:119], a[0:7], 0\n\t" \
  "ds_read_b64_tr_b8 v[110:111], v91 offset:33792\n\t" \
  "s_add_u32 m0, s84, 0x21c00\n\t" \
  "s_mov_b32 s55, 0x7080\n\t" \
  "buffer_load_dwordx4 v6, s[20:23], s55 offen lds\n\t" \
  "s_waitcnt lgkmcnt(4)\n\t" \
  "v_mfma_f32_16x16x128_f8f6f4 v[160:163], v[120:127], a[0:7], 0\n\t" \
  "ds_read_b64_tr_b8 v[112:113], v84 offset:0\n\t" \
  "s_add_u32 m0, s24, 0x4000\n\t" \
  "s_add_u32 s48, s47, 0x400000\n\t" \
  "buffer_load_dwordx4 v2, s[8:11], s48 offen lds\n\t" \
  "v_mfma_f32_16x16x128_f8f6f4 v[164:167], v[120:127], a[8:15], 0\n\t" \
  "ds_read_b64_tr_b8 v[114:115], v84 offset:1024\n\t" \
  "s_add_u32 m0, s24, 0x4400\n\t" \
  "s_add_u32 s49, s47, 0x420000\n\t" \
  "buffer_load_dwordx4 v2, s[8:11], s49 offen lds\n\t" \
  "v_mfma_f32_16x16x128_f8f6f4 v[168:171], v[120:127], a[16:23], 0\n\t" \
  "ds_read_b64_tr_b8 v[116:117], v92 offset:32768\n\t" \
  "s_add_u32 m0, s24, 0x6000\n\t" \
  "s_add_u32 s50, s47, 0x600000\n\t" \
  "buffer_load_dwordx4 v2, s[8:11], s50 offen lds\n\t" \
  "v_mfma_f32_16x16x128_f8f6f4 v[172:175], v[120:127], a[24:31], 0\n\t" \
  "ds_read_b64_tr_b8 v[118:119], v92 offset:33792\n\t" \
  "s_add_u32 m0, s24, 0x6400\n\t" \
  "s_add_u32 s51, s47, 0x620000\n\t" \
  "buffer_load_dwordx4 v2, s[8:11], s51 offen lds\n\t" \
  "s_waitcnt lgkmcnt(4)\n\t" \
  "v_mfma_f32_16x16x128_f8f6f4 v[188:191], v[104:111], a[24:31], 0\n\t" \
  "ds_read_b64_tr_b8 v[120:121], v85 offset:0\n\t" \
  "s_add_u32 m0, s24, 0x8000\n\t" \
  "s_add_u32 s52, s40, 0x0\n\t" \
  "buffer_load_dwordx4 v2, s[8:11], s52 offen lds\n\t" \
  "v_mfma_f32_16x16x128_f8f6f4 v[184:187], v[104:111], a[16:23], 0\n\t" \
  "ds_read_b64_tr_b8 v[122:123], v85 offset:1024\n\t" \
  "s_add_u32 m0, s24, 0x8400\n\t" \
  "s_add_u32 s53, s40, 0x20000\n\t" \
  "buffer_load_dwordx4 v2, s[8:11], s53 offen lds\n\t" \
  "v_mfma_f32_16x16x128_f8f6f4 v[180:183], v[104:111], a[8:15], 0\n\t" \
  "ds_read_b64_tr_b8 v[124:125], v93 offset:32768\n\t" \
  "s_add_u32 m0, s24, 0xa000\n\t" \
  "s_add_u32 s54, s41, 0x0\n\t" \
  "buffer_load_dwordx4 v2, s[8:11], s54 offen lds\n\t" \
  "v_mfma_f32_16x16x128_f8f6f4 v[176:179], v[104:111], a[0:7], 0\n\t" \
  "ds_read_b64_tr_b8 v[126:127], v93 offset:33792\n\t" \
  "s_add_u32 m0, s24, 0xa400\n\t" \
  "s_add_u32 s55, s41, 0x20000\n\t" \
  "buffer_load_dwordx4 v2, s[8:11], s55 offen lds\n\t" \
  "s_waitcnt lgkmcnt(4)\n\t" \
  "v_mfma_f32_16x16x128_f8f6f4 v[192:195], v[112:119], a[0:7], 0\n\t" \
  "ds_read_b64_tr_b8 v[104:105], v86 offset:0\n\t" \
  "v_mfma_f32_16x16x128_f8f6f4 v[196:199], v[112:119], a[8:15], 0\n\t" \
  "ds_read_b64_tr_b8 v[106:107], v86 offset:1024\n\t" \
  "v_mfma_f32_16x16x128_f8f6f4 v[200:203], v[112:119], a[16:23], 0\n\t" \
  "ds_read_b64_tr_b8 v[108:109], v94 offset:32768\n\t" \
  "v_mfma_f32_16x16x128_f8f6f4 v[204:207], v[112:119], a[24:31], 0\n\t" \
  "ds_read_b64_tr_b8 v[110:111], v94 offset:33792\n\t" \
  "s_waitcnt lgkmcnt(4)\n\t" \
  "v_mfma_f32_16x16x128_f8f6f4 v[220:223], v[120:127], a[24:31], 0\n\t" \
  "ds_read_b64_tr_b8 v[112:113], v87 offset:0\n\t" \
  "v_mfma_f32_16x16x128_f8f6f4 v[216:219], v[120:127], a[16:23], 0\n\t" \
  "ds_read_b64_tr_b8 v[114:115], v87 offset:1024\n\t" \
  "v_mfma_f32_16x16x128_f8f6f4 v[212:215], v[120:127], a[8:15], 0\n\t" \
  "ds_read_b64_tr_b8 v[116:117], v95 offset:32768\n\t" \
  "s_waitcnt vmcnt(8)\n\t" \
  "ds_read_b128 a[32:35], v22 offset:0\n\t" \
  "v_mfma_f32_16x16x128_f8f6f4 v[208:211], v[120:127], a[0:7], 0\n\t" \
  "ds_read_b64_tr_b8 v[118:119], v95 offset:33792\n\t" \
  "ds_read_b128 a[36:39], v23 offset:0\n\t" \
  "s_waitcnt lgkmcnt(0)\n\t" \
  "s_barrier\n\t" \
  "v_mfma_f32_16x16x128_f8f6f4 v[224:227], v[104:111], a[0:7], 0\n\t" \
  "ds_read_b64_tr_b8 v[120:121], v80 offset:8192\n\t" \
  "ds_read_b128 a[40:43], v22 offset:2048\n\t" \
  "v_mfma_f32_16x16x128_f8f6f4 v[228:231], v[104:111], a[8:15], 0\n\t" \
  "ds_read_b64_tr_b8 v[122:123], v80 offset:9216\n\t" \
  "ds_read_b128 a[44:47], v23 offset:2048\n\t" \
  "v_mfma_f32_16x16x128_f8f6f4 v[232:235], v[104:111], a[16:23], 0\n\t" \
  "ds_read_b64_tr_b8 v[124:125], v88 offset:40960\n\t" \
  "ds_read_b128 a[48:51], v22 offset:4096\n\t" \
  "v_mfma_f32_16x16x128_f8f6f4 v[236:239], v[104:111], a[24:31], 0\n\t" \
  "ds_read_b64_tr_b8 v[126:127], v88 offset:41984\n\t" \
  "ds_read_b128 a[52:55], v23 offset:4096\n\t" \
  "v_mfma_f32_16x16x128_f8f6f4 v[252:255], v[112:119], a[24:31], 0\n\t" \
  "ds_read_b64_tr_b8 v[104:105], v81 offset:8192\n\t" \
  "ds_read_b128 a[56:59], v22 offset:6144\n\t" \
  "v_mfma_f32_16x16x128_f8f6f4 v[248:251], v[112:119], a[16:23], 0\n\t" \
  "ds_read_b64_tr_b8 v[106:107], v81 offset:9216\n\t" \
  "ds_read_b128 a[60:63], v23 offset:6144\n\t" \
  "v_mfma_f32_16x16x128_f8f6f4 v[244:247], v[112:119], a[8:15], 0\n\t" \
  "ds_read_b64_tr_b8 v[108:109], v89 offset:40960\n\t" \
  "v_mfma_f32_16x16x128_f8f6f4 v[240:243], v[112:119], a[0:7], 0\n\t" \
  "ds_read_b64_tr_b8 v[110:111], v89 offset:41984\n\t" \
  "s_waitcnt lgkmcnt(2)\n\t" \
  "v_mfma_f32_16x16x128_f8f6f4 v[128:131], v[120:127], a[32:39], v[128:131]\n\t" \
  "ds_read_b64_tr_b8 v[112:113], v82 offset:8192\n\t" \
  "s_add_u32 m0, s24, 0x1c000\n\t" \
  "s_add_u32 s48, s47, 0x400000\n\t" \
  "buffer_load_dwordx4 v3, s[8:11], s48 offen lds\n\t" \
  "v_mfma_f32_16x16x128_f8f6f4 v[132:135], v[120:127], a[40:47], v[132:135]\n\t" \
  "ds_read_b64_tr_b8 v[114:115], v82 offset:9216\n\t" \
  "s_add_u32 m0, s24, 0x1c400\n\t" \
  "s_add_u32 s49, s47, 0x420000\n\t" \
  "buffer_load_dwordx4 v3, s[8:11], s49 offen lds\n\t" \
  "v_mfma_f32_16x16x128_f8f6f4 v[136:139], v[120:127], a[48:55], v[136:139]\n\t" \
  "ds_read_b64_tr_b8 v[116:117], v90 offset:40960\n\t" \
  "s_add_u32 m0, s84, 0x10000\n\t" \
  "s_mov_b32 s50, 0x100\n\t" \
  "buffer_load_dwordx4 v5, s[20:23], s50 offen lds\n\t" \
  "v_mfma_f32_16x16x128_f8f6f4 v[140:143], v[120:127], a[56:63], v[140:143]\n\t" \
  "ds_read_b64_tr_b8 v[118:119], v90 offset:41984\n\t" \
  "s_add_u32 m0, s84, 0x10400\n\t" \
  "s_mov_b32 s51, 0x1100\n\t" \
  "buffer_load_dwordx4 v6, s[20:23], s51 offen lds\n\t" \
  "s_waitcnt lgkmcnt(4)\n\t" \
  "v_mfma_f32_16x16x128_f8f6f4 v[156:159], v[104:111], a[56:63], v[156:159]\n\t" \
  "ds_read_b64_tr_b8 v[120:121], v83 offset:8192\n\t" \
  "s_add_u32 m0, s84, 0x10800\n\t" \
  "s_mov_b32 s52, 0x2100\n\t" \
  "buffer_load_dwordx4 v5, s[20:23], s52 offen lds\n\t" \
  "v_mfma_f32_16x16x128_f8f6f4 v[152:155], v[104:111], a[48:55], v[152:155]\n\t" \
  "ds_read_b64_tr_b8 v[122:123], v83 offset:9216\n\t" \
  "s_add_u32 m0, s84, 0x10c00\n\t" \
  "s_mov_b32 s53, 0x3100\n\t" \
  "buffer_load_dwordx4 v6, s[20:23], s53 offen lds\n\t" \
  "v_mfma_f32_16x16x128_f8f6f4 v[148:151], v[104:111], a[40:47], v[148:151]\n\t" \
  "ds_read_b64_tr_b8 v[124:125], v91 offset:40960\n\t" \
  "s_add_u32 m0, s84, 0x11000\n\t" \
  "s_mov_b32 s54, 0x4100\n\t" \
  "buffer_load_dwordx4 v5, s[20:23], s54 offen lds\n\t" \
  "v_mfma_f32_16x16x128_f8f6f4 v[144:147], v[104:111], a[32:39], v[144:147]\n\t" \
  "ds_read_b64_tr_b8 v[126:127], v91 offset:41984\n\t" \
  "s_add_u32 m0, s84, 0x11400\n\t" \
  "s_mov_b32 s55, 0x5100\n\t" \
  "buffer_load_dwordx4 v6, s[20:23], s55 offen lds\n\t" \
  "s_waitcnt lgkmcnt(4)\n\t" \
  "v_mfma_f32_16x16x128_f8f6f4 v[160:163], v[112:119], a[32:39], v[160:163]\n\t" \
  "ds_read_b64_tr_b8 v[104:105], v84 offset:8192\n\t" \
  "s_add_u32 m0, s84, 0x11800\n\t" \
  "s_mov_b32 s48, 0x6100\n\t" \
  "buffer_load_dwordx4 v5, s[20:23], s48 offen lds\n\t" \
  "v_mfma_f32_16x16x128_f8f6f4 v[164:167], v[112:119], a[40:47], v[164:167]\n\t" \
  "ds_read_b64_tr_b8 v[106:107], v84 offset:9216\n\t" \
  "s_add_u32 m0, s84, 0x11c00\n\t" \
  "s_mov_b32 s49, 0x7100\n\t" \
  "buffer_load_dwordx4 v6, s[20:23], s49 offen lds\n\t" \
  "v_mfma_f32_16x16x128_f8f6f4 v[168:171], v[112:119], a[48:55], v[168:171]\n\t" \
  "ds_read_b64_tr_b8 v[108:109], v92 offset:40960\n\t" \
  "s_add_u32 m0, s24, 0x1e000\n\t" \
  "s_add_u32 s50, s47, 0x600000\n\t" \
  "buffer_load_dwordx4 v3, s[8:11], s50 offen lds\n\t" \
  "v_mfma_f32_16x16x128_f8f6f4 v[172:175], v[112:119], a[56:63], v[172:175]\n\t" \
  "ds_read_b64_tr_b8 v[110:111], v92 offset:41984\n\t" \
  "s_add_u32 m0, s24, 0x1e400\n\t" \
  "s_add_u32 s51, s47, 0x620000\n\t" \
  "buffer_load_dwordx4 v3, s[8:11], s51 offen lds\n\t" \
  "s_waitcnt lgkmcnt(4)\n\t" \
  "v_mfma_f32_16x16x128_f8f6f4 v[188:191], v[120:127], a[56:63], v[188:191]\n\t" \
  "ds_read_b64_tr_b8 v[112:113], v85 offset:8192\n\t" \
  "s_add_u32 m0, s84, 0x20000\n\t" \
  "s_mov_b32 s52, 0x180\n\t" \
  "buffer_load_dwordx4 v5, s[20:23], s52 offen lds\n\t" \
  "v_mfma_f32_16x16x128_f8f6f4 v[184:187], v[120:127], a[48:55], v[184:187]\n\t" \
  "ds_read_b64_tr_b8 v[114:115], v85 offset:9216\n\t" \
  "s_add_u32 m0, s84, 0x20400\n\t" \
  "s_mov_b32 s53, 0x1180\n\t" \
  "buffer_load_dwordx4 v6, s[20:23], s53 offen lds\n\t" \
  "v_mfma_f32_16x16x128_f8f6f4 v[180:183], v[120:127], a[40:47], v[180:183]\n\t" \
  "ds_read_b64_tr_b8 v[116:117], v93 offset:40960\n\t" \
  "s_add_u32 m0, s84, 0x20800\n\t" \
  "s_mov_b32 s54, 0x2180\n\t" \
  "buffer_load_dwordx4 v5, s[20:23], s54 offen lds\n\t" \
  "v_mfma_f32_16x16x128_f8f6f4 v[176:179], v[120:127], a[32:39], v[176:179]\n\t" \
  "ds_read_b64_tr_b8 v[118:119], v93 offset:41984\n\t" \
  "s_add_u32 m0, s84, 0x20c00\n\t" \
  "s_mov_b32 s55, 0x3180\n\t" \
  "buffer_load_dwordx4 v6, s[20:23], s55 offen lds\n\t" \
  "s_waitcnt lgkmcnt(4)\n\t" \
  "v_mfma_f32_16x16x128_f8f6f4 v[192:195], v[104:111], a[32:39], v[192:195]\n\t" \
  "ds_read_b64_tr_b8 v[120:121], v86 offset:8192\n\t" \
  "s_add_u32 m0, s84, 0x21000\n\t" \
  "s_mov_b32 s48, 0x4180\n\t" \
  "buffer_load_dwordx4 v5, s[20:23], s48 offen lds\n\t" \
  "v_mfma_f32_16x16x128_f8f6f4 v[196:199], v[104:111], a[40:47], v[196:199]\n\t" \
  "ds_read_b64_tr_b8 v[122:123], v86 offset:9216\n\t" \
  "s_add_u32 m0, s84, 0x21400\n\t" \
  "s_mov_b32 s49, 0x5180\n\t" \
  "buffer_load_dwordx4 v6, s[20:23], s49 offen lds\n\t" \
  "v_mfma_f32_16x16x128_f8f6f4 v[200:203], v[104:111], a[48:55], v[200:203]\n\t" \
  "ds_read_b64_tr_b8 v[124:125], v94 offset:40960\n\t" \
  "s_add_u32 m0, s84, 0x21800\n\t" \
  "s_mov_b32 s50, 0x6180\n\t" \
  "buffer_load_dwordx4 v5, s[20:23], s50 offen lds\n\t" \
  "v_mfma_f32_16x16x128_f8f6f4 v[204:207], v[104:111], a[56:63], v[204:207]\n\t" \
  "ds_read_b64_tr_b8 v[126:127], v94 offset:41984\n\t" \
  "s_add_u32 m0, s84, 0x21c00\n\t" \
  "s_mov_b32 s51, 0x7180\n\t" \
  "buffer_load_dwordx4 v6, s[20:23], s51 offen lds\n\t" \
  "s_waitcnt lgkmcnt(4)\n\t" \
  "v_mfma_f32_16x16x128_f8f6f4 v[220:223], v[112:119], a[56:63], v[220:223]\n\t" \
  "ds_read_b64_tr_b8 v[104:105], v87 offset:8192\n\t" \
  "v_mfma_f32_16x16x128_f8f6f4 v[216:219], v[112:119], a[48:55], v[216:219]\n\t" \
  "ds_read_b64_tr_b8 v[106:107], v87 offset:9216\n\t" \
  "v_mfma_f32_16x16x128_f8f6f4 v[212:215], v[112:119], a[40:47], v[212:215]\n\t" \
  "ds_read_b64_tr_b8 v[108:109], v95 offset:40960\n\t" \
  "s_waitcnt vmcnt(10)\n\t" \
  "ds_read_b128 a[64:67], v20 offset:0\n\t" \
  "s_cmp_lg_u32 s26, 1\n\t" \
  "s_cbranch_scc1 Lns1_3_%=\n\t" \
  "global_store_dwordx4 v63, v[72:75], s[28:29] sc0 sc1\n\t" \
  "global_store_dwordx4 v63, v[76:79], s[28:29] offset:1024 sc0 sc1\n\t" \
  "s_branch Lms1_3_%=\n\t" \
  "Lns1_3_%=:\n\t" \
  "buffer_load_dword v62, v65, s[20:23], s22 offen\n\t" \
  "buffer_load_dword v62, v65, s[20:23], s22 offen\n\t" \
  "Lms1_3_%=:\n\t" \
  "s_add_u32 m0, s24, 0xc000\n\t" \
  "s_add_u32 s52, s42, 0x0\n\t" \
  "buffer_load_dwordx4 v2, s[8:11], s52 offen lds\n\t" \
  "v_mfma_f32_16x16x128_f8f6f4 v[208:211], v[112:119], a[32:39], v[208:211]\n\t" \
  "ds_read_b64_tr_b8 v[110:111], v95 offset:41984\n\t" \
  "ds_read_b128 a[68:71], v21 offset:0\n\t" \
  "s_waitcnt lgkmcnt(0)\n\t" \
  "s_barrier\n\t" \
  "v_mfma_f32_16x16x128_f8f6f4 v[224:227], v[120:127], a[32:39], v[224:227]\n\t" \
  "ds_read_b64_tr_b8 v[112:113], v80 offset:16384\n\t" \
  "ds_read_b128 a[72:75], v20 offset:2048\n\t" \
  "s_add_u32 m0, s24, 0xc400\n\t" \
  "s_add_u32 s53, s42, 0x20000\n\t" \
  "buffer_load_dwordx4 v2, s[8:11], s53 offen lds\n\t" \
  "v_mfma_f32_16x16x128_f8f6f4 v[228:231], v[120:127], a[40:47], v[228:231]\n\t" \
  "ds_read_b64_tr_b8 v[114:115], v80 offset:17408\n\t" \
  "ds_read_b128 a[76:79], v21 offset:2048\n\t" \
  "v_mfma_f32_16x16x128_f8f6f4 v[232:235], v[120:127], a[48:55], v[232:235]\n\t" \
  "ds_read_b64_tr_b8 v[116:117], v88 offset:49152\n\t" \
  "ds_read_b128 a[80:83], v20 offset:4096\n\t" \
  "s_add_u32 m0, s24, 0xe000\n\t" \
  "s_add_u32 s54, s43, 0x0\n\t" \
  "buffer_load_dwordx4 v2, s[8:11], s54 offen lds\n\t" \
  "v_mfma_f32_16x16x128_f8f6f4 v[236:239], v[120:127], a[56:63], v[236:239]\n\t" \
  "ds_read_b64_tr_b8 v[118:119], v88 offset:50176\n\t" \
  "ds_read_b128 a[84:87], v21 offset:4096\n\t" \
  "v_mfma_f32_16x16x128_f8f6f4 v[252:255], v[104:111], a[56:63], v[252:255]\n\t" \
  "ds_read_b64_tr_b8 v[120:121], v81 offset:16384\n\t" \
  "ds_read_b128 a[88:91], v20 offset:6144\n\t" \
  "s_add_u32 m0, s24, 0xe400\n\t" \
  "s_add_u32 s55, s43, 0x20000\n\t" \
  "buffer_load_dwordx4 v2, s[8:11], s55 offen lds\n\t" \
  "v_mfma_f32_16x16x128_f8f6f4 v[248:251], v[104:111], a[48:55], v[248:251]\n\t" \
  "ds_read_b64_tr_b8 v[122:123], v81 offset:17408\n\t" \
  "ds_read_b128 a[92:95], v21 offset:6144\n\t" \
  "v_mfma_f32_16x16x128_f8f6f4 v[244:247], v[104:111], a[40:47], v[244:247]\n\t" \
  "ds_read_b64_tr_b8 v[124:125], v89 offset:49152\n\t" \
  "v_mfma_f32_16x16x128_f8f6f4 v[240:243], v[104:111], a[32:39], v[240:243]\n\t" \
  "ds_read_b64_tr_b8 v[126:127], v89 offset:50176\n\t" \
  "s_waitcnt lgkmcnt(2)\n\t" \
  "v_mfma_f32_16x16x128_f8f6f4 v[128:131], v[112:119], a[64:71], v[128:131]\n\t" \
  "ds_read_b64_tr_b8 v[104:105], v82 offset:16384\n\t" \
  "s_waitcnt vmcnt(6)\n\t" \
  "ds_read_b128 a[96:99], v22 offset:0\n\t" \
  "v_mfma_f32_16x16x128_f8f6f4 v[132:135], v[112:119], a[72:79], v[132:135]\n\t" \
  "ds_read_b64_tr_b8 v[106:107], v82 offset:17408\n\t" \
  "ds_read_b128 a[100:103], v23 offset:0\n\t" \
  "v_mfma_f32_16x16x128_f8f6f4 v[136:139], v[112:119], a[80:87], v[136:139]\n\t" \
  "ds_read_b64_tr_b8 v[108:109], v90 offset:49152\n\t" \
  "ds_read_b128 a[104:107], v22 offset:2048\n\t" \
  "v_mfma_f32_16x16x128_f8f6f4 v[140:143], v[112:119], a[88:95], v[140:143]\n\t" \
  "ds_read_b64_tr_b8 v[110:111], v90 offset:50176\n\t" \
  "ds_read_b128 a[108:111], v23 offset:2048\n\t" \
  "s_waitcnt lgkmcnt(8)\n\t" \
  "v_mfma_f32_16x16x128_f8f6f4 v[156:159], v[120:127], a[88:95], v[156:159]\n\t" \
  "ds_read_b64_tr_b8 v[112:113], v83 offset:16384\n\t" \
  "ds_read_b128 a[112:115], v22 offset:4096\n\t" \
  "v_mfma_f32_16x16x128_f8f6f4 v[152:155], v[120:127], a[80:87], v[152:155]\n\t" \
  "ds_read_b64_tr_b8 v[114:115], v83 offset:17408\n\t" \
  "ds_read_b128 a[116:119], v23 offset:4096\n\t" \
  "v_mfma_f32_16x16x128_f8f6f4 v[148:151], v[120:127], a[72:79], v[148:151]\n\t" \
  "ds_read_b64_tr_b8 v[116:117], v91 offset:49152\n\t" \
  "ds_read_b128 a[120:123], v22 offset:6144\n\t" \
  "v_mfma_f32_16x16x128_f8f6f4 v[144:147], v[120:127], a[64:71], v[144:147]\n\t" \
  "ds_read_b64_tr_b8 v[118:119], v91 offset:50176\n\t" \
  "s_waitcnt lgkmcnt(14)\n\t" \
  "ds_read_b128 a[124:127], v23 offset:6144\n\t" \
  "s_waitcnt lgkmcnt(9)\n\t" \
  "v_mfma_f32_16x16x128_f8f6f4 v[160:163], v[104:111], a[64:71], v[160:163]\n\t" \
  "ds_read_b64_tr_b8 v[120:121], v84 offset:16384\n\t" \
  "v_mfma_f32_16x16x128_f8f6f4 v[164:167], v[104:111], a[72:79], v[164:167]\n\t" \
  "ds_read_b64_tr_b8 v[122:123], v84 offset:17408\n\t" \
  "v_mfma_f32_16x16x128_f8f6f4 v[168:171], v[104:111], a[80:87], v[168:171]\n\t" \
  "ds_read_b64_tr_b8 v[124:125], v92 offset:49152\n\t" \
  "v_mfma_f32_16x16x128_f8f6f4 v[172:175], v[104:111], a[88:95], v[172:175]\n\t" \
  "ds_read_b64_tr_b8 v[126:127], v92 offset:50176\n\t" \
  "s_waitcnt lgkmcnt(5)\n\t" \
  "v_mfma_f32_16x16x128_f8f6f4 v[188:191], v[112:119], a[88:95], v[188:191]\n\t" \
  "ds_read_b64_tr_b8 v[104:105], v85 offset:16384\n\t" \
  "v_mfma_f32_16x16x128_f8f6f4 v[184:187], v[112:119], a[80:87], v[184:187]\n\t" \
  "ds_read_b64_tr_b8 v[106:107], v85 offset:17408\n\t" \
  "v_mfma_f32_16x16x128_f8f6f4 v[180:183], v[112:119], a[72:79], v[180:183]\n\t" \
  "ds_read_b64_tr_b8 v[108:109], v93 offset:49152\n\t" \
  "v_mfma_f32_16x16x128_f8f6f4 v[176:179], v[112:119], a[64:71], v[176:179]\n\t" \
  "ds_read_b64_tr_b8 v[110:111], v93 offset:50176\n\t" \
  "s_waitcnt lgkmcnt(4)\n\t" \
  "v_mfma_f32_16x16x128_f8f6f4 v[192:195], v[120:127], a[64:71], v[192:195]\n\t" \
  "ds_read_b64_tr_b8 v[112:113], v86 offset:16384\n\t" \
  "v_mfma_f32_16x16x128_f8f6f4 v[196:199], v[120:127], a[72:79], v[196:199]\n\t" \
  "ds_read_b64_tr_b8 v[114:115], v86 offset:17408\n\t" \
  "v_mfma_f32_16x16x128_f8f6f4 v[200:203], v[120:127], a[80:87], v[200:203]\n\t" \
  "ds_read_b64_tr_b8 v[116:117], v94 offset:49152\n\t" \
  "v_mfma_f32_16x16x128_f8f6f4 v[204:207], v[120:127], a[88:95], v[204:207]\n\t" \
  "ds_read_b64_tr_b8 v[118:119], v94 offset:50176\n\t" \
  "s_waitcnt lgkmcnt(4)\n\t" \
  "v_mfma_f32_16x16x128_f8f6f4 v[220:223], v[104:111], a[88:95], v[220:223]\n\t" \
  "ds_read_b64_tr_b8 v[120:121], v87 offset:16384\n\t" \
  "v_mfma_f32_16x16x128_f8f6f4 v[216:219], v[104:111], a[80:87], v[216:219]\n\t" \
  "ds_read_b64_tr_b8 v[122:123], v87 offset:17408\n\t" \
  "v_mfma_f32_16x16x128_f8f6f4 v[212:215], v[104:111], a[72:79], v[212:215]\n\t" \
  "ds_read_b64_tr_b8 v[124:125], v95 offset:49152\n\t" \
  "v_mfma_f32_16x16x128_f8f6f4 v[208:211], v[104:111], a[64:71], v[208:211]\n\t" \
  "ds_read_b64_tr_b8 v[126:127], v95 offset:50176\n\t" \
  "s_waitcnt lgkmcnt(0)\n\t" \
  "s_barrier\n\t" \
  "v_mfma_f32_16x16x128_f8f6f4 v[224:227], v[112:119], a[64:71], v[224:227]\n\t" \
  "ds_read_b64_tr_b8 v[104:105], v80 offset:24576\n\t" \
  "s_mov_b32 exec_lo, 0x7f\n\t" \
  "s_mov_b32 exec_hi, 0\n\t" \
  "global_load_dwordx4 v[68:71], v10, s[6:7] sc0 sc1\n\t" \
  "s_mov_b64 exec, -1\n\t" \
  "s_add_u32 m0, s24, 0x20000\n\t" \
  "s_add_u32 s48, s40, 0x0\n\t" \
  "buffer_load_dwordx4 v3, s[8:11], s48 offen lds\n\t" \
  "v_mfma_f32_16x16x128_f8f6f4 v[228:231], v[112:119], a[72:79], v[228:231]\n\t" \
  "ds_read_b64_tr_b8 v[106:107], v80 offset:25600\n\t" \
  "v_mfma_f32_16x16x128_f8f6f4 v[232:235], v[112:119], a[80:87], v[232:235]\n\t" \
  "ds_read_b64_tr_b8 v[108:109], v88 offset:57344\n\t" \
  "s_add_u32 m0, s24, 0x20400\n\t" \
  "s_add_u32 s49, s40, 0x20000\n\t" \
  "buffer_load_dwordx4 v3, s[8:11], s49 offen lds\n\t" \
  "v_mfma_f32_16x16x128_f8f6f4 v[236:239], v[112:119], a[88:95], v[236:239]\n\t" \
  "ds_read_b64_tr_b8 v[110:111], v88 offset:58368\n\t" \
  "v_mfma_f32_16x16x128_f8f6f4 v[252:255], v[120:127], a[88:95], v[252:255]\n\t" \
  "ds_read_b64_tr_b8 v[112:113], v81 offset:24576\n\t" \
  "s_add_u32 m0, s24, 0x22000\n\t" \
  "s_add_u32 s50, s41, 0x0\n\t" \
  "buffer_load_dwordx4 v3, s[8:11], s50 offen lds\n\t" \
  "v_mfma_f32_16x16x128_f8f6f4 v[248:251], v[120:127], a[80:87], v[248:251]\n\t" \
  "ds_read_b64_tr_b8 v[114:115], v81 offset:25600\n\t" \
  "v_mfma_f32_16x16x128_f8f6f4 v[244:247], v[120:127], a[72:79], v[244:247]\n\t" \
  "ds_read_b64_tr_b8 v[116:117], v89 offset:57344\n\t" \
  "s_add_u32 m0, s24, 0x22400\n\t" \
  "s_add_u32 s51, s41, 0x20000\n\t" \
  "buffer_load_dwordx4 v3, s[8:11], s51 offen lds\n\t" \
  "v_mfma_f32_16x16x128_f8f6f4 v[240:243], v[120:127], a[64:71], v[240:243]\n\t" \
  "ds_read_b64_tr_b8 v[118:119], v89 offset:58368\n\t" \
  "s_waitcnt lgkmcnt(4)\n\t" \
  "v_mfma_f32_16x16x128_f8f6f4 v[128:131], v[104:111], a[96:103], v[128:131]\n\t" \
  "ds_read_b64_tr_b8 v[120:121], v82 offset:24576\n\t" \
  "s_waitcnt vmcnt(4)\n\t" \
  "v_min_u32 v68, v68, v69\n\t" \
  "v_min3_u32 v68, v68, v70, v71\n\t" \
  "v_cmp_le_u32 vcc, s57, v68\n\t" \
  "s_and_b32 s64, vcc_lo, 0x7f\n\t" \
  "s_or_b32 s56, s56, s64\n\t" \
  "s_and_b32 s64, s56, 0x1\n\t" \
  "s_cmp_eq_u32 s64, 0x1\n\t" \
  "s_cbranch_scc1 Lok0_3_%=\n\t" \
  "s_mov_b32 s62, 0x1\n\t" \
  "s_mov_b32 s69, 0x7f\n\t" \
  "s_call_b64 s[60:61], Lpoll_3_%=\n\t" \
  "Lok0_3_%=:\n\t" \
  "s_add_u32 s52, s68, 0x40000\n\t" \
  "buffer_load_dwordx4 a[128:131], v4, s[12:15], s52 offen offset:0 sc0 sc1\n\t" \
  "v_mfma_f32_16x16x128_f8f6f4 v[132:135], v[104:111], a[104:111], v[132:135]\n\t" \
  "ds_read_b64_tr_b8 v[122:123], v82 offset:25600\n\t" \
  "v_mfma_f32_16x16x128_f8f6f4 v[136:139], v[104:111], a[112:119], v[136:139]\n\t" \
  "ds_read_b64_tr_b8 v[124:125], v90 offset:57344\n\t" \
  "s_add_u32 s53, s68, 0x40000\n\t" \
  "buffer_load_dwordx4 a[132:135], v4, s[12:15], s53 offen offset:1024 sc0 sc1\n\t" \
  "v_mfma_f32_16x16x128_f8f6f4 v[140:143], v[104:111], a[120:127], v[140:143]\n\t" \
  "ds_read_b64_tr_b8 v[126:127], v90 offset:58368\n\t" \
  "s_waitcnt lgkmcnt(4)\n\t" \
  "v_mfma_f32_16x16x128_f8f6f4 v[156:159], v[112:119], a[120:127], v[156:159]\n\t" \
  "ds_read_b64_tr_b8 v[104:105], v83 offset:24576\n\t" \
  "s_add_u32 s54, s68, 0x40000\n\t" \
  "buffer_load_dwordx4 a[136:139], v4, s[12:15], s54 offen offset:2048 sc0 sc1\n\t" \
  "v_mfma_f32_16x16x128_f8f6f4 v[152:155], v[112:119], a[112:119], v[152:155]\n\t" \
  "ds_read_b64_tr_b8 v[106:107], v83 offset:25600\n\t" \
  "v_mfma_f32_16x16x128_f8f6f4 v[148:151], v[112:119], a[104:111], v[148:151]\n\t" \
  "ds_read_b64_tr_b8 v[108:109], v91 offset:57344\n\t" \
  "s_add_u32 s55, s68, 0x40000\n\t" \
  "buffer_load_dwordx4 a[140:143], v4, s[12:15], s55 offen offset:3072 sc0 sc1\n\t" \
  "v_mfma_f32_16x16x128_f8f6f4 v[144:147], v[112:119], a[96:103], v[144:147]\n\t" \
  "ds_read_b64_tr_b8 v[110:111], v91 offset:58368\n\t" \
  "s_waitcnt lgkmcnt(4)\n\t" \
  "v_mfma_f32_16x16x128_f8f6f4 v[160:163], v[120:127], a[96:103], v[160:163]\n\t" \
  "ds_read_b64_tr_b8 v[112:113], v84 offset:24576\n\t" \
  "s_add_u32 s48, s68, 0x41000\n\t" \
  "buffer_load_dwordx4 a[144:147], v4, s[12:15], s48 offen offset:0 sc0 sc1\n\t" \
  "v_mfma_f32_16x16x128_f8f6f4 v[164:167], v[120:127], a[104:111], v[164:167]\n\t" \
  "ds_read_b64_tr_b8 v[114:115], v84 offset:25600\n\t" \
  "v_mfma_f32_16x16x128_f8f6f4 v[168:171], v[120:127], a[112:119], v[168:171]\n\t" \
  "ds_read_b64_tr_b8 v[116:117], v92 offset:57344\n\t" \
  "s_add_u32 s49, s68, 0x41000\n\t" \
  "buffer_load_dwordx4 a[148:151], v4, s[12:15], s49 offen offset:1024 sc0 sc1\n\t" \
  "v_mfma_f32_16x16x128_f8f6f4 v[172:175], v[120:127], a[120:127], v[172:175]\n\t" \
  "ds_read_b64_tr_b8 v[118:119], v92 offset:58368\n\t" \
  "s_waitcnt lgkmcnt(4)\n\t" \
  "v_mfma_f32_16x16x128_f8f6f4 v[188:191], v[104:111], a[120:127], v[188:191]\n\t" \
  "ds_read_b64_tr_b8 v[120:121], v85 offset:24576\n\t" \
  "s_add_u32 s50, s68, 0x41000\n\t" \
  "buffer_load_dwordx4 a[152:155], v4, s[12:15], s50 offen offset:2048 sc0 sc1\n\t" \
  "v_mfma_f32_16x16x128_f8f6f4 v[184:187], v[104:111], a[112:119], v[184:187]\n\t" \
  "ds_read_b64_tr_b8 v[122:123], v85 offset:25600\n\t" \
  "v_mfma_f32_16x16x128_f8f6f4 v[180:183], v[104:111], a[104:111], v[180:183]\n\t" \
  "ds_read_b64_tr_b8 v[124:125], v93 offset:57344\n\t" \
  "s_add_u32 s51, s68, 0x41000\n\t" \
  "buffer_load_dwordx4 a[156:159], v4, s[12:15], s51 offen offset:3072 sc0 sc1\n\t" \
  "v_mfma_f32_16x16x128_f8f6f4 v[176:179], v[104:111], a[96:103], v[176:179]\n\t" \
  "ds_read_b64_tr_b8 v[126:127], v93 offset:58368\n\t" \
  "s_waitcnt lgkmcnt(4)\n\t" \
  "v_mfma_f32_16x16x128_f8f6f4 v[192:195], v[112:119], a[96:103], v[192:195]\n\t" \
  "ds_read_b64_tr_b8 v[104:105], v86 offset:24576\n\t" \
  "s_and_b32 s64, s56, 0x2\n\t" \
  "s_cmp_eq_u32 s64, 0x2\n\t" \
  "s_cbranch_scc1 Lok1_3_%=\n\t" \
  "s_mov_b32 s62, 0x2\n\t" \
  "s_mov_b32 s69, 0x7f\n\t" \
  "s_call_b64 s[60:61], Lpoll_3_%=\n\t" \
  "Lok1_3_%=:\n\t" \
  "s_add_u32 s52, s68, 0x50000\n\t" \
  "buffer_load_dwordx4 a[160:163], v4, s[12:15], s52 offen offset:0 sc0 sc1\n\t" \
  "v_mfma_f32_16x16x128_f8f6f4 v[196:199], v[112:119], a[104:111], v[196:199]\n\t" \
  "ds_read_b64_tr_b8 v[106:107], v86 offset:25600\n\t" \
  "v_mfma_f32_16x16x128_f8f6f4 v[200:203], v[112:119], a[112:119], v[200:203]\n\t" \
  "ds_read_b64_tr_b8 v[108:109], v94 offset:57344\n\t" \
  "s_add_u32 s53, s68, 0x50000\n\t" \
  "buffer_load_dwordx4 a[164:167], v4, s[12:15], s53 offen offset:1024 sc0 sc1\n\t" \
  "v_mfma_f32_16x16x128_f8f6f4 v[204:207], v[112:119], a[120:127], v[204:207]\n\t" \
  "ds_read_b64_tr_b8 v[110:111], v94 offset:58368\n\t" \
  "s_waitcnt lgkmcnt(4)\n\t" \
  "v_mfma_f32_16x16x128_f8f6f4 v[220:223], v[120:127], a[120:127], v[220:223]\n\t" \
  "ds_read_b64_tr_b8 v[112:113], v87 offset:24576\n\t" \
  "s_add_u32 s54, s68, 0x50000\n\t" \
  "buffer_load_dwordx4 a[168:171], v4, s[12:15], s54 offen offset:2048 sc0 sc1\n\t" \
  "v_mfma_f32_16x16x128_f8f6f4 v[216:219], v[120:127], a[112:119], v[216:219]\n\t" \
  "ds_read_b64_tr_b8 v[114:115], v87 offset:25600\n\t" \
  "v_mfma_f32_16x16x128_f8f6f4 v[212:215], v[120:127], a[104:111], v[212:215]\n\t" \
  "ds_read_b64_tr_b8 v[116:117], v95 offset:57344\n\t" \
  "s_add_u32 s55, s68, 0x50000\n\t" \
  "buffer_load_dwordx4 a[172:175], v4, s[12:15], s55 offen offset:3072 sc0 sc1\n\t" \
  "v_mfma_f32_16x16x128_f8f6f4 v[208:211], v[120:127], a[96:103], v[208:211]\n\t" \
  "ds_read_b64_tr_b8 v[118:119], v95 offset:58368\n\t" \
  "s_waitcnt vmcnt(14) lgkmcnt(0)\n\t" \
  "s_barrier\n\t" \
  "v_mfma_f32_16x16x128_f8f6f4 v[224:227], v[104:111], a[96:103], v[224:227]\n\t" \
  "ds_read_b64_tr_b8 v[120:121], v80 offset:32768\n\t" \
  "s_add_u32 s48, s68, 0x51000\n\t" \
  "buffer_load_dwordx4 a[176:179], v4, s[12:15], s48 offen offset:0 sc0 sc1\n\t" \
  "v_mfma_f32_16x16x128_f8f6f4 v[228:231], v[104:111], a[104:111], v[228:231]\n\t" \
  "ds_read_b64_tr_b8 v[122:123], v80 offset:33792\n\t" \
  "v_mfma_f32_16x16x128_f8f6f4 v[232:235], v[104:111], a[112:119], v[232:235]\n\t" \
  "ds_read_b64_tr_b8 v[124:125], v96 offset:0\n\t" \
  "s_add_u32 s49, s68, 0x51000\n\t" \
  "buffer_load_dwordx4 a[180:183], v4, s[12:15], s49 offen offset:1024 sc0 sc1\n\t" \
  "v_mfma_f32_16x16x128_f8f6f4 v[236:239], v[104:111], a[120:127], v[236:239]\n\t" \
  "ds_read_b64_tr_b8 v[126:127], v96 offset:1024\n\t" \
  "v_mfma_f32_16x16x128_f8f6f4 v[252:255], v[112:119], a[120:127], v[252:255]\n\t" \
  "ds_read_b64_tr_b8 v[104:105], v81 offset:32768\n\t" \
  "s_add_u32 s50, s68, 0x51000\n\t" \
  "buffer_load_dwordx4 a[184:187], v4, s[12:15], s50 offen offset:2048 sc0 sc1\n\t" \
  "v_mfma_f32_16x16x128_f8f6f4 v[248:251], v[112:119], a[112:119], v[248:251]\n\t" \
  "ds_read_b64_tr_b8 v[106:107], v81 offset:33792\n\t" \
  "v_mfma_f32_16x16x128_f8f6f4 v[244:247], v[112:119], a[104:111], v[244:247]\n\t" \
  "ds_read_b64_tr_b8 v[108:109], v97 offset:0\n\t" \
  "s_add_u32 s51, s68, 0x51000\n\t" \
  "buffer_load_dwordx4 a[188:191], v4, s[12:15], s51 offen offset:3072 sc0 sc1\n\t" \
  "v_mfma_f32_16x16x128_f8f6f4 v[240:243], v[112:119], a[96:103], v[240:243]\n\t" \
  "ds_read_b64_tr_b8 v[110:111], v97 offset:1024\n\t" \
  "s_waitcnt vmcnt(8) lgkmcnt(4)\n\t" \
  "v_mfma_f32_16x16x128_f8f6f4 v[128:131], v[120:127], a[128:135], v[128:131]\n\t" \
  "ds_read_b64_tr_b8 v[112:113], v82 offset:32768\n\t" \
  "s_add_u32 m0, s24, 0x24000\n\t" \
  "s_add_u32 s52, s42, 0x0\n\t" \
  "buffer_load_dwordx4 v3, s[8:11], s52 offen lds\n\t" \
  "v_mfma_f32_16x16x128_f8f6f4 v[132:135], v[120:127], a[136:143], v[132:135]\n\t" \
  "ds_read_b64_tr_b8 v[114:115], v82 offset:33792\n\t" \
  "v_mfma_f32_16x16x128_f8f6f4 v[136:139], v[120:127], a[144:151], v[136:139]\n\t" \
  "ds_read_b64_tr_b8 v[116:117], v98 offset:0\n\t" \
  "s_add_u32 m0, s24, 0x24400\n\t" \
  "s_add_u32 s53, s42, 0x20000\n\t" \
  "buffer_load_dwordx4 v3, s[8:11], s53 offen lds\n\t" \
  "v_mfma_f32_16x16x128_f8f6f4 v[140:143], v[120:127], a[152:159], v[140:143]\n\t" \
  "ds_read_b64_tr_b8 v[118:119], v98 offset:1024\n\t" \
  "s_waitcnt lgkmcnt(4)\n\t" \
  "v_mfma_f32_16x16x128_f8f6f4 v[156:159], v[104:111], a[152:159], v[156:159]\n\t" \
  "ds_read_b64_tr_b8 v[120:121], v83 offset:32768\n\t" \
  "s_and_b32 s64, s56, 0x4\n\t" \
  "s_cmp_eq_u32 s64, 0x4\n\t" \
  "s_cbranch_scc1 Lok2_3_%=\n\t" \
  "s_mov_b32 s62, 0x4\n\t" \
  "s_mov_b32 s69, 0x7f\n\t" \
  "s_call_b64 s[60:61], Lpoll_3_%=\n\t" \
  "Lok2_3_%=:\n\t" \
  "s_add_u32 s54, s68, 0x60000\n\t" \
  "buffer_load_dwordx4 a[192:195], v4, s[12:15], s54 offen offset:0 sc0 sc1\n\t" \
  "v_mfma_f32_16x16x128_f8f6f4 v[152:155], v[104:111], a[144:151], v[152:155]\n\t" \
  "ds_read_b64_tr_b8 v[122:123], v83 offset:33792\n\t" \
  "v_mfma_f32_16x16x128_f8f6f4 v[148:151], v[104:111], a[136:143], v[148:151]\n\t" \
  "ds_read_b64_tr_b8 v[124:125], v99 offset:0\n\t" \
  "s_add_u32 s55, s68, 0x60000\n\t" \
  "buffer_load_dwordx4 a[196:199], v4, s[12:15], s55 offen offset:1024 sc0 sc1\n\t" \
  "v_mfma_f32_16x16x128_f8f6f4 v[144:147], v[104:111], a[128:135], v[144:147]\n\t" \
  "ds_read_b64_tr_b8 v[126:127], v99 offset:1024\n\t" \
  "s_waitcnt lgkmcnt(4)\n\t" \
  "v_mfma_f32_16x16x128_f8f6f4 v[160:163], v[112:119], a[128:135], v[160:163]\n\t" \
  "ds_read_b64_tr_b8 v[104:105], v84 offset:32768\n\t" \
  "s_add_u32 s48, s68, 0x60000\n\t" \
  "buffer_load_dwordx4 a[200:203], v4, s[12:15], s48 offen offset:2048 sc0 sc1\n\t" \
  "v_mfma_f32_16x16x128_f8f6f4 v[164:167], v[112:119], a[136:143], v[164:167]\n\t" \
  "ds_read_b64_tr_b8 v[106:107], v84 offset:33792\n\t" \
  "v_mfma_f32_16x16x128_f8f6f4 v[168:171], v[112:119], a[144:151], v[168:171]\n\t" \
  "ds_read_b64_tr_b8 v[108:109], v100 offset:0\n\t" \
  "s_add_u32 s49, s68, 0x60000\n\t" \
  "buffer_load_dwordx4 a[204:207], v4, s[12:15], s49 offen offset:3072 sc0 sc1\n\t" \
  "v_mfma_f32_16x16x128_f8f6f4 v[172:175], v[112:119], a[152:159], v[172:175]\n\t" \
  "ds_read_b64_tr_b8 v[110:111], v100 offset:1024\n\t" \
  "s_waitcnt lgkmcnt(4)\n\t" \
  "v_mfma_f32_16x16x128_f8f6f4 v[188:191], v[120:127], a[152:159], v[188:191]\n\t" \
  "ds_read_b64_tr_b8 v[112:113], v85 offset:32768\n\t" \
  "s_add_u32 s50, s68, 0x61000\n\t" \
  "buffer_load_dwordx4 a[208:211], v4, s[12:15], s50 offen offset:0 sc0 sc1\n\t" \
  "v_mfma_f32_16x16x128_f8f6f4 v[184:187], v[120:127], a[144:151], v[184:187]\n\t" \
  "ds_read_b64_tr_b8 v[114:115], v85 offset:33792\n\t" \
  "v_mfma_f32_16x16x128_f8f6f4 v[180:183], v[120:127], a[136:143], v[180:183]\n\t" \
  "ds_read_b64_tr_b8 v[116:117], v101 offset:0\n\t" \
  "s_add_u32 s51, s68, 0x61000\n\t" \
  "buffer_load_dwordx4 a[212:215], v4, s[12:15], s51 offen offset:1024 sc0 sc1\n\t" \
  "v_mfma_f32_16x16x128_f8f6f4 v[176:179], v[120:127], a[128:135], v[176:179]\n\t" \
  "ds_read_b64_tr_b8 v[118:119], v101 offset:1024\n\t" \
  "s_waitcnt lgkmcnt(4)\n\t" \
  "v_mfma_f32_16x16x128_f8f6f4 v[192:195], v[104:111], a[128:135], v[192:195]\n\t" \
  "ds_read_b64_tr_b8 v[120:121], v86 offset:32768\n\t" \
  "s_add_u32 s52, s68, 0x61000\n\t" \
  "buffer_load_dwordx4 a[216:219], v4, s[12:15], s52 offen offset:2048 sc0 sc1\n\t" \
  "v_mfma_f32_16x16x128_f8f6f4 v[196:199], v[104:111], a[136:143], v[196:199]\n\t" \
  "ds_read_b64_tr_b8 v[122:123], v86 offset:33792\n\t" \
  "v_mfma_f32_16x16x128_f8f6f4 v[200:203], v[104:111], a[144:151], v[200:203]\n\t" \
  "ds_read_b64_tr_b8 v[124:125], v102 offset:0\n\t" \
  "s_add_u32 s53, s68, 0x61000\n\t" \
  "buffer_load_dwordx4 a[220:223], v4, s[12:15], s53 offen offset:3072 sc0 sc1\n\t" \
  "v_mfma_f32_16x16x128_f8f6f4 v[204:207], v[104:111], a[152:159], v[204:207]\n\t" \
  "ds_read_b64_tr_b8 v[126:127], v102 offset:1024\n\t" \
  "s_waitcnt lgkmcnt(4)\n\t" \
  "v_mfma_f32_16x16x128_f8f6f4 v[220:223], v[112:119], a[152:159], v[220:223]\n\t" \
  "ds_read_b64_tr_b8 v[104:105], v87 offset:32768\n\t" \
  "s_add_u32 m0, s24, 0x26000\n\t" \
  "s_add_u32 s54, s43, 0x0\n\t" \
  "buffer_load_dwordx4 v3, s[8:11], s54 offen lds\n\t" \
  "v_mfma_f32_16x16x128_f8f6f4 v[216:219], v[112:119], a[144:151], v[216:219]\n\t" \
  "ds_read_b64_tr_b8 v[106:107], v87 offset:33792\n\t" \
  "v_mfma_f32_16x16x128_f8f6f4 v[212:215], v[112:119], a[136:143], v[212:215]\n\t" \
  "ds_read_b64_tr_b8 v[108:109], v103 offset:0\n\t" \
  "s_add_u32 m0, s24, 0x26400\n\t" \
  "s_add_u32 s55, s43, 0x20000\n\t" \
  "buffer_load_dwordx4 v3, s[8:11], s55 offen lds\n\t" \
  "v_mfma_f32_16x16x128_f8f6f4 v[208:211], v[112:119], a[128:135], v[208:211]\n\t" \
  "ds_read_b64_tr_b8 v[110:111], v103 offset:1024\n\t" \
  "s_waitcnt lgkmcnt(0)\n\t" \
  "s_barrier\n\t" \
  "v_mfma_f32_16x16x128_f8f6f4 v[224:227], v[120:127], a[128:135], v[224:227]\n\t" \
  "ds_read_b64_tr_b8 v[112:113], v80 offset:40960\n\t" \
  "s_and_b32 s64, s56, 0x8\n\t" \
  "s_cmp_eq_u32 s64, 0x8\n\t" \
  "s_cbranch_scc1 Lok3_3_%=\n\t" \
  "s_mov_b32 s62, 0x8\n\t" \
  "s_mov_b32 s69, 0x7f\n\t" \
  "s_call_b64 s[60:61], Lpoll_3_%=\n\t" \
  "Lok3_3_%=:\n\t" \
  "s_add_u32 s48, s68, 0x70000\n\t" \
  "buffer_load_dwordx4 a[224:227], v4, s[12:15], s48 offen offset:0 sc0 sc1\n\t" \
  "v_mfma_f32_16x16x128_f8f6f4 v[228:231], v[120:127], a[136:143], v[228:231]\n\t" \
  "ds_read_b64_tr_b8 v[114:115], v80 offset:41984\n\t" \
  "v_mfma_f32_16x16x128_f8f6f4 v[232:235], v[120:127], a[144:151], v[232:235]\n\t" \
  "ds_read_b64_tr_b8 v[116:117], v96 offset:8192\n\t" \
  "s_add_u32 s49, s68, 0x70000\n\t" \
  "buffer_load_dwordx4 a[228:231], v4, s[12:15], s49 offen offset:1024 sc0 sc1\n\t" \
  "v_mfma_f32_16x16x128_f8f6f4 v[236:239], v[120:127], a[152:159], v[236:239]\n\t" \
  "ds_read_b64_tr_b8 v[118:119], v96 offset:9216\n\t" \
  "v_mfma_f32_16x16x128_f8f6f4 v[252:255], v[104:111], a[152:159], v[252:255]\n\t" \
  "ds_read_b64_tr_b8 v[120:121], v81 offset:40960\n\t" \
  "s_add_u32 s50, s68, 0x70000\n\t" \
  "buffer_load_dwordx4 a[232:235], v4, s[12:15], s50 offen offset:2048 sc0 sc1\n\t" \
  "v_mfma_f32_16x16x128_f8f6f4 v[248:251], v[104:111], a[144:151], v[248:251]\n\t" \
  "ds_read_b64_tr_b8 v[122:123], v81 offset:41984\n\t" \
  "v_mfma_f32_16x16x128_f8f6f4 v[244:247], v[104:111], a[136:143], v[244:247]\n\t" \
  "ds_read_b64_tr_b8 v[124:125], v97 offset:8192\n\t" \
  "s_add_u32 s51, s68, 0x70000\n\t" \
  "buffer_load_dwordx4 a[236:239], v4, s[12:15], s51 offen offset:3072 sc0 sc1\n\t" \
  "v_mfma_f32_16x16x128_f8f6f4 v[240:243], v[104:111], a[128:135], v[240:243]\n\t" \
  "ds_read_b64_tr_b8 v[126:127], v97 offset:9216\n\t" \
  "s_waitcnt vmcnt(16) lgkmcnt(4)\n\t" \
  "v_mfma_f32_16x16x128_f8f6f4 v[128:131], v[112:119], a[160:167], v[128:131]\n\t" \
  "ds_read_b64_tr_b8 v[104:105], v82 offset:40960\n\t" \
  "s_add_u32 s52, s68, 0x71000\n\t" \
  "buffer_load_dwordx4 a[240:243], v4, s[12:15], s52 offen offset:0 sc0 sc1\n\t" \
  "v_mfma_f32_16x16x128_f8f6f4 v[132:135], v[112:119], a[168:175], v[132:135]\n\t" \
  "ds_read_b64_tr_b8 v[106:107], v82 offset:41984\n\t" \
  "v_mfma_f32_16x16x128_f8f6f4 v[136:139], v[112:119], a[176:183], v[136:139]\n\t" \
  "ds_read_b64_tr_b8 v[108:109], v98 offset:8192\n\t" \
  "s_add_u32 s53, s68, 0x71000\n\t" \
  "buffer_load_dwordx4 a[244:247], v4, s[12:15], s53 offen offset:1024 sc0 sc1\n\t" \
  "v_mfma_f32_16x16x128_f8f6f4 v[140:143], v[112:119], a[184:191], v[140:143]\n\t" \
  "ds_read_b64_tr_b8 v[110:111], v98 offset:9216\n\t" \
  "s_waitcnt lgkmcnt(4)\n\t" \
  "v_mfma_f32_16x16x128_f8f6f4 v[156:159], v[120:127], a[184:191], v[156:159]\n\t" \
  "ds_read_b64_tr_b8 v[112:113], v83 offset:40960\n\t" \
  "s_add_u32 s54, s68, 0x71000\n\t" \
  "buffer_load_dwordx4 a[248:251], v4, s[12:15], s54 offen offset:2048 sc0 sc1\n\t" \
  "v_mfma_f32_16x16x128_f8f6f4 v[152:155], v[120:127], a[176:183], v[152:155]\n\t" \
  "ds_read_b64_tr_b8 v[114:115], v83 offset:41984\n\t" \
  "v_mfma_f32_16x16x128_f8f6f4 v[148:151], v[120:127], a[168:175], v[148:151]\n\t" \
  "ds_read_b64_tr_b8 v[116:117], v99 offset:8192\n\t" \
  "s_add_u32 s55, s68, 0x71000\n\t" \
  "buffer_load_dwordx4 a[252:255], v4, s[12:15], s55 offen offset:3072 sc0 sc1\n\t" \
  "v_mfma_f32_16x16x128_f8f6f4 v[144:147], v[120:127], a[160:167], v[144:147]\n\t" \
  "ds_read_b64_tr_b8 v[118:119], v99 offset:9216\n\t" \
  "s_waitcnt lgkmcnt(4)\n\t" \
  "v_mfma_f32_16x16x128_f8f6f4 v[160:163], v[104:111], a[160:167], v[160:163]\n\t" \
  "ds_read_b64_tr_b8 v[120:121], v84 offset:40960\n\t" \
  "s_cmp_lg_u32 s26, 1\n\t" \
  "s_cbranch_scc1 Lnf1_3_%=\n\t" \
  "s_mov_b64 exec, 1\n\t" \
  "global_store_dword v65, v64, s[30:31] sc0 sc1\n\t" \
  "s_mov_b64 exec, -1\n\t" \
  "s_branch Lmf1_3_%=\n\t" \
  "Lnf1_3_%=:\n\t" \
  "buffer_load_dword v62, v65, s[20:23], s22 offen\n\t" \
  "Lmf1_3_%=:\n\t" \
  "s_cmp_lg_u32 s26, 2\n\t" \
  "s_cbranch_scc1 Lns2_3_%=\n\t" \
  "global_store_dwordx4 v63, v[72:75], s[28:29] sc0 sc1\n\t" \
  "global_store_dwordx4 v63, v[76:79], s[28:29] offset:1024 sc0 sc1\n\t" \
  "s_branch Lms2_3_%=\n\t" \
  "Lns2_3_%=:\n\t" \
  "buffer_load_dword v62, v65, s[20:23], s22 offen\n\t" \
  "buffer_load_dword v62, v65, s[20:23], s22 offen\n\t" \
  "Lms2_3_%=:\n\t" \
  "s_add_u32 m0, s24, 0x10000\n\t" \
  "s_add_u32 s48, s44, 0x0\n\t" \
  "buffer_load_dwordx4 v2, s[8:11], s48 offen lds\n\t" \
  "v_mfma_f32_16x16x128_f8f6f4 v[164:167], v[104:111], a[168:175], v[164:167]\n\t" \
  "ds_read_b64_tr_b8 v[122:123], v84 offset:41984\n\t" \
  "v_mfma_f32_16x16x128_f8f6f4 v[168:171], v[104:111], a[176:183], v[168:171]\n\t" \
  "ds_read_b64_tr_b8 v[124:125], v100 offset:8192\n\t" \
  "s_add_u32 m0, s24, 0x10400\n\t" \
  "s_add_u32 s49, s44, 0x20000\n\t" \
  "buffer_load_dwordx4 v2, s[8:11], s49 offen lds\n\t" \
  "v_mfma_f32_16x16x128_f8f6f4 v[172:175], v[104:111], a[184:191], v[172:175]\n\t" \
  "ds_read_b64_tr_b8 v[126:127], v100 offset:9216\n\t" \
  "s_waitcnt lgkmcnt(4)\n\t" \
  "v_mfma_f32_16x16x128_f8f6f4 v[188:191], v[112:119], a[184:191], v[188:191]\n\t" \
  "ds_read_b64_tr_b8 v[104:105], v85 offset:40960\n\t" \
  "s_add_u32 m0, s24, 0x12000\n\t" \
  "s_add_u32 s50, s45, 0x0\n\t" \
  "buffer_load_dwordx4 v2, s[8:11], s50 offen lds\n\t" \
  "v_mfma_f32_16x16x128_f8f6f4 v[184:187], v[112:119], a[176:183], v[184:187]\n\t" \
  "ds_read_b64_tr_b8 v[106:107], v85 offset:41984\n\t" \
  "v_mfma_f32_16x16x128_f8f6f4 v[180:183], v[112:119], a[168:175], v[180:183]\n\t" \
  "ds_read_b64_tr_b8 v[108:109], v101 offset:8192\n\t" \
  "s_add_u32 m0, s24, 0x12400\n\t" \
  "s_add_u32 s51, s45, 0x20000\n\t" \
  "buffer_load_dwordx4 v2, s[8:11], s51 offen lds\n\t" \
  "v_mfma_f32_16x16x128_f8f6f4 v[176:179], v[112:119], a[160:167], v[176:179]\n\t" \
  "ds_read_b64_tr_b8 v[110:111], v101 offset:9216\n\t" \
  "s_waitcnt lgkmcnt(4)\n\t" \
  "v_mfma_f32_16x16x128_f8f6f4 v[192:195], v[120:127], a[160:167], v[192:195]\n\t" \
  "ds_read_b64_tr_b8 v[112:113], v86 offset:40960\n\t" \
  "s_add_u32 m0, s24, 0x14000\n\t" \
  "s_add_u32 s52, s46, 0x0\n\t" \
  "buffer_load_dwordx4 v2, s[8:11], s52 offen lds\n\t" \
  "v_mfma_f32_16x16x128_f8f6f4 v[196:199], v[120:127], a[168:175], v[196:199]\n\t" \
  "ds_read_b64_tr_b8 v[114:115], v86 offset:41984\n\t" \
  "v_mfma_f32_16x16x128_f8f6f4 v[200:203], v[120:127], a[176:183], v[200:203]\n\t" \
  "ds_read_b64_tr_b8 v[116:117], v102 offset:8192\n\t" \
  "s_add_u32 m0, s24, 0x14400\n\t" \
  "s_add_u32 s53, s46, 0x20000\n\t" \
  "buffer_load_dwordx4 v2, s[8:11], s53 offen lds\n\t" \
  "v_mfma_f32_16x16x128_f8f6f4 v[204:207], v[120:127], a[184:191], v[204:207]\n\t" \
  "ds_read_b64_tr_b8 v[118:119], v102 offset:9216\n\t" \
  "s_waitcnt lgkmcnt(4)\n\t" \
  "v_mfma_f32_16x16x128_f8f6f4 v[220:223], v[104:111], a[184:191], v[220:223]\n\t" \
  "ds_read_b64_tr_b8 v[120:121], v87 offset:40960\n\t" \
  "s_add_u32 m0, s24, 0x16000\n\t" \
  "s_add_u32 s54, s40, 0x200000\n\t" \
  "buffer_load_dwordx4 v2, s[8:11], s54 offen lds\n\t" \
  "v_mfma_f32_16x16x128_f8f6f4 v[216:219], v[104:111], a[176:183], v[216:219]\n\t" \
  "ds_read_b64_tr_b8 v[122:123], v87 offset:41984\n\t" \
  "v_mfma_f32_16x16x128_f8f6f4 v[212:215], v[104:111], a[168:175], v[212:215]\n\t" \
  "ds_read_b64_tr_b8 v[124:125], v103 offset:8192\n\t" \
  "s_add_u32 m0, s24, 0x16400\n\t" \
  "s_add_u32 s55, s40, 0x220000\n\t" \
  "buffer_load_dwordx4 v2, s[8:11], s55 offen lds\n\t" \
  "v_mfma_f32_16x16x128_f8f6f4 v[208:211], v[104:111], a[160:167], v[208:211]\n\t" \
  "ds_read_b64_tr_b8 v[126:127], v103 offset:9216\n\t" \
  "s_waitcnt vmcnt(29) lgkmcnt(0)\n\t" \
  "s_barrier\n\t" \
  "v_mfma_f32_16x16x128_f8f6f4 v[224:227], v[112:119], a[160:167], v[224:227]\n\t" \
  "ds_read_b64_tr_b8 v[104:105], v80 offset:49152\n\t" \
  "v_mfma_f32_16x16x128_f8f6f4 v[228:231], v[112:119], a[168:175], v[228:231]\n\t" \
  "ds_read_b64_tr_b8 v[106:107], v80 offset:50176\n\t" \
  "v_mfma_f32_16x16x128_f8f6f4 v[232:235], v[112:119], a[176:183], v[232:235]\n\t" \
  "ds_read_b64_tr_b8 v[108:109], v96 offset:16384\n\t" \
  "v_mfma_f32_16x16x128_f8f6f4 v[236:239], v[112:119], a[184:191], v[236:239]\n\t" \
  "ds_read_b64_tr_b8 v[110:111], v96 offset:17408\n\t" \
  "v_mfma_f32_16x16x128_f8f6f4 v[252:255], v[120:127], a[184:191], v[252:255]\n\t" \
  "ds_read_b64_tr_b8 v[112:113], v81 offset:49152\n\t" \
  "v_mfma_f32_16x16x128_f8f6f4 v[248:251], v[120:127], a[176:183], v[248:251]\n\t" \
  "ds_read_b64_tr_b8 v[114:115], v81 offset:50176\n\t" \
  "v_mfma_f32_16x16x128_f8f6f4 v[244:247], v[120:127], a[168:175], v[244:247]\n\t" \
  "ds_read_b64_tr_b8 v[116:117], v97 offset:16384\n\t" \
  "v_mfma_f32_16x16x128_f8f6f4 v[240:243], v[120:127], a[160:167], v[240:243]\n\t" \
  "ds_read_b64_tr_b8 v[118:119], v97 offset:17408\n\t" \
  "s_waitcnt vmcnt(21) lgkmcnt(4)\n\t" \
  "v_mfma_f32_16x16x128_f8f6f4 v[128:131], v[104:111], a[192:199], v[128:131]\n\t" \
  "ds_read_b64_tr_b8 v[120:121], v82 offset:49152\n\t" \
  "v_mfma_f32_16x16x128_f8f6f4 v[132:135], v[104:111], a[200:207], v[132:135]\n\t" \
  "ds_read_b64_tr_b8 v[122:123], v82 offset:50176\n\t" \
  "v_mfma_f32_16x16x128_f8f6f4 v[136:139], v[104:111], a[208:215], v[136:139]\n\t" \
  "ds_read_b64_tr_b8 v[124:125], v98 offset:16384\n\t" \
  "v_mfma_f32_16x16x128_f8f6f4 v[140:143], v[104:111], a[216:223], v[140:143]\n\t" \
  "ds_read_b64_tr_b8 v[126:127], v98 offset:17408\n\t" \
  "s_waitcnt lgkmcnt(4)\n\t" \
  "v_mfma_f32_16x16x128_f8f6f4 v[156:159], v[112:119], a[216:223], v[156:159]\n\t" \
  "ds_read_b64_tr_b8 v[104:105], v83 offset:49152\n\t" \
  "v_mfma_f32_16x16x128_f8f6f4 v[152:155], v[112:119], a[208:215], v[152:155]\n\t" \
  "ds_read_b64_tr_b8 v[106:107], v83 offset:50176\n\t" \
  "v_mfma_f32_16x16x128_f8f6f4 v[148:151], v[112:119], a[200:207], v[148:151]\n\t" \
  "ds_read_b64_tr_b8 v[108:109], v99 offset:16384\n\t" \
  "v_mfma_f32_16x16x128_f8f6f4 v[144:147], v[112:119], a[192:199], v[144:147]\n\t" \
  "ds_read_b64_tr_b8 v[110:111], v99 offset:17408\n\t" \
  "s_waitcnt lgkmcnt(4)\n\t" \
  "v_mfma_f32_16x16x128_f8f6f4 v[160:163], v[120:127], a[192:199], v[160:163]\n\t" \
  "ds_read_b64_tr_b8 v[112:113], v84 offset:49152\n\t" \
  "v_mfma_f32_16x16x128_f8f6f4 v[164:167], v[120:127], a[200:207], v[164:167]\n\t" \
  "ds_read_b64_tr_b8 v[114:115], v84 offset:50176\n\t" \
  "v_mfma_f32_16x16x128_f8f6f4 v[168:171], v[120:127], a[208:215], v[168:171]\n\t" \
  "ds_read_b64_tr_b8 v[116:117], v100 offset:16384\n\t" \
  "v_mfma_f32_16x16x128_f8f6f4 v[172:175], v[120:127], a[216:223], v[172:175]\n\t" \
  "ds_read_b64_tr_b8 v[118:119], v100 offset:17408\n\t" \
  "s_waitcnt lgkmcnt(4)\n\t" \
  "v_mfma_f32_16x16x128_f8f6f4 v[188:191], v[104:111], a[216:223], v[188:191]\n\t" \
  "ds_read_b64_tr_b8 v[120:121], v85 offset:49152\n\t" \
  "s_add_u32 m0, s24, 0x18000\n\t" \
  "s_add_u32 s48, s44, 0x0\n\t" \
  "buffer_load_dwordx4 v3, s[8:11], s48 offen lds\n\t" \
  "v_mfma_f32_16x16x128_f8f6f4 v[184:187], v[104:111], a[208:215], v[184:187]\n\t" \
  "ds_read_b64_tr_b8 v[122:123], v85 offset:50176\n\t" \
  "v_mfma_f32_16x16x128_f8f6f4 v[180:183], v[104:111], a[200:207], v[180:183]\n\t" \
  "ds_read_b64_tr_b8 v[124:125], v101 offset:16384\n\t" \
  "s_add_u32 m0, s24, 0x18400\n\t" \
  "s_add_u32 s49, s44, 0x20000\n\t" \
  "buffer_load_dwordx4 v3, s[8:11], s49 offen lds\n\t" \
  "v_mfma_f32_16x16x128_f8f6f4 v[176:179], v[104:111], a[192:199], v[176:179]\n\t" \
  "ds_read_b64_tr_b8 v[126:127], v101 offset:17408\n\t" \
  "s_waitcnt lgkmcnt(4)\n\t" \
  "v_mfma_f32_16x16x128_f8f6f4 v[192:195], v[112:119], a[192:199], v[192:195]\n\t" \
  "ds_read_b64_tr_b8 v[104:105], v86 offset:49152\n\t" \
  "s_and_b32 s64, s56, 0x10\n\t" \
  "s_cmp_eq_u32 s64, 0x10\n\t" \
  "s_cbranch_scc1 Lok4_3_%=\n\t" \
  "s_mov_b32 s62, 0x10\n\t" \
  "s_mov_b32 s69, 0x7f\n\t" \
  "s_call_b64 s[60:61], Lpoll_3_%=\n\t" \
  "Lok4_3_%=:\n\t" \
  "s_add_u32 s50, s68, 0x80000\n\t" \
  "buffer_load_dwordx4 a[0:3], v4, s[12:15], s50 offen offset:0 sc0 sc1\n\t" \
  "v_mfma_f32_16x16x128_f8f6f4 v[196:199], v[112:119], a[200:207], v[196:199]\n\t" \
  "ds_read_b64_tr_b8 v[106:107], v86 offset:50176\n\t" \
  "v_mfma_f32_16x16x128_f8f6f4 v[200:203], v[112:119], a[208:215], v[200:203]\n\t" \
  "ds_read_b64_tr_b8 v[108:109], v102 offset:16384\n\t" \
  "s_add_u32 s51, s68, 0x80000\n\t" \
  "buffer_load_dwordx4 a[4:7], v4, s[12:15], s51 offen offset:1024 sc0 sc1\n\t" \
  "v_mfma_f32_16x16x128_f8f6f4 v[204:207], v[112:119], a[216:223], v[204:207]\n\t" \
  "ds_read_b64_tr_b8 v[110:111], v102 offset:17408\n\t" \
  "s_waitcnt lgkmcnt(4)\n\t" \
  "v_mfma_f32_16x16x128_f8f6f4 v[220:223], v[120:127], a[216:223], v[220:223]\n\t" \
  "ds_read_b64_tr_b8 v[112:113], v87 offset:49152\n\t" \
  "s_add_u32 s52, s68, 0x80000\n\t" \
  "buffer_load_dwordx4 a[8:11], v4, s[12:15], s52 offen offset:2048 sc0 sc1\n\t" \
  "v_mfma_f32_16x16x128_f8f6f4 v[216:219], v[120:127], a[208:215], v[216:219]\n\t" \
  "ds_read_b64_tr_b8 v[114:115], v87 offset:50176\n\t" \
  "v_mfma_f32_16x16x128_f8f6f4 v[212:215], v[120:127], a[200:207], v[212:215]\n\t" \
  "ds_read_b64_tr_b8 v[116:117], v103 offset:16384\n\t" \
  "s_add_u32 s53, s68, 0x80000\n\t" \
  "buffer_load_dwordx4 a[12:15], v4, s[12:15], s53 offen offset:3072 sc0 sc1\n\t" \
  "v_mfma_f32_16x16x128_f8f6f4 v[208:211], v[120:127], a[192:199], v[208:211]\n\t" \
  "ds_read_b64_tr_b8 v[118:119], v103 offset:17408\n\t" \
  "s_waitcnt vmcnt(25) lgkmcnt(0)\n\t" \
  "s_barrier\n\t" \
  "v_mfma_f32_16x16x128_f8f6f4 v[224:227], v[104:111], a[192:199], v[224:227]\n\t" \
  "ds_read_b64_tr_b8 v[120:121], v80 offset:57344\n\t" \
  "s_add_u32 s54, s68, 0x81000\n\t" \
  "buffer_load_dwordx4 a[16:19], v4, s[12:15], s54 offen offset:0 sc0 sc1\n\t" \
  "v_mfma_f32_16x16x128_f8f6f4 v[228:231], v[104:111], a[200:207], v[228:231]\n\t" \
  "ds_read_b64_tr_b8 v[122:123], v80 offset:58368\n\t" \
  "v_mfma_f32_16x16x128_f8f6f4 v[232:235], v[104:111], a[208:215], v[232:235]\n\t" \
  "ds_read_b64_tr_b8 v[124:125], v96 offset:24576\n\t" \
  "s_add_u32 s55, s68, 0x81000\n\t" \
  "buffer_load_dwordx4 a[20:23], v4, s[12:15], s55 offen offset:1024 sc0 sc1\n\t" \
  "v_mfma_f32_16x16x128_f8f6f4 v[236:239], v[104:111], a[216:223], v[236:239]\n\t" \
  "ds_read_b64_tr_b8 v[126:127], v96 offset:25600\n\t" \
  "v_mfma_f32_16x16x128_f8f6f4 v[252:255], v[112:119], a[216:223], v[252:255]\n\t" \
  "ds_read_b64_tr_b8 v[104:105], v81 offset:57344\n\t" \
  "s_add_u32 s48, s68, 0x81000\n\t" \
  "buffer_load_dwordx4 a[24:27], v4, s[12:15], s48 offen offset:2048 sc0 sc1\n\t" \
  "v_mfma_f32_16x16x128_f8f6f4 v[248:251], v[112:119], a[208:215], v[248:251]\n\t" \
  "ds_read_b64_tr_b8 v[106:107], v81 offset:58368\n\t" \
  "v_mfma_f32_16x16x128_f8f6f4 v[244:247], v[112:119], a[200:207], v[244:247]\n\t" \
  "ds_read_b64_tr_b8 v[108:109], v97 offset:24576\n\t" \
  "s_add_u32 s49, s68, 0x81000\n\t" \
  "buffer_load_dwordx4 a[28:31], v4, s[12:15], s49 offen offset:3072 sc0 sc1\n\t" \
  "v_mfma_f32_16x16x128_f8f6f4 v[240:243], v[112:119], a[192:199], v[240:243]\n\t" \
  "ds_read_b64_tr_b8 v[110:111], v97 offset:25600\n\t" \
  "s_waitcnt vmcnt(21) lgkmcnt(4)\n\t" \
  "v_mfma_f32_16x16x128_f8f6f4 v[128:131], v[120:127], a[224:231], v[128:131]\n\t" \
  "ds_read_b64_tr_b8 v[112:113], v82 offset:57344\n\t" \
  "s_add_u32 m0, s24, 0x1a000\n\t" \
  "s_add_u32 s50, s45, 0x0\n\t" \
  "buffer_load_dwordx4 v3, s[8:11], s50 offen lds\n\t" \
  "s_mov_b32 exec_lo, 0x7f00\n\t" \
  "s_mov_b32 exec_hi, 0\n\t" \
  "global_load_dwordx4 v[68:71], v10, s[6:7] sc0 sc1\n\t" \
  "s_mov_b64 exec, -1\n\t" \
  "v_mfma_f32_16x16x128_f8f6f4 v[132:135], v[120:127], a[232:239], v[132:135]\n\t" \
  "ds_read_b64_tr_b8 v[114:115], v82 offset:58368\n\t" \
  "v_mfma_f32_16x16x128_f8f6f4 v[136:139], v[120:127], a[240:247], v[136:139]\n\t" \
  "ds_read_b64_tr_b8 v[116:117], v98 offset:24576\n\t" \
  "s_add_u32 m0, s24, 0x1a400\n\t" \
  "s_add_u32 s51, s45, 0x20000\n\t" \
  "buffer_load_dwordx4 v3, s[8:11], s51 offen lds\n\t" \
  "v_mfma_f32_16x16x128_f8f6f4 v[140:143], v[120:127], a[248:255], v[140:143]\n\t" \
  "ds_read_b64_tr_b8 v[118:119], v98 offset:25600\n\t" \
  "s_waitcnt lgkmcnt(4)\n\t" \
  "v_mfma_f32_16x16x128_f8f6f4 v[156:159], v[104:111], a[248:255], v[156:159]\n\t" \
  "ds_read_b64_tr_b8 v[120:121], v83 offset:57344\n\t" \
  "s_and_b32 s64, s56, 0x20\n\t" \
  "s_cmp_eq_u32 s64, 0x20\n\t" \
  "s_cbranch_scc1 Lok5_3_%=\n\t" \
  "s_mov_b32 s62, 0x20\n\t" \
  "s_mov_b32 s69, 0x7f\n\t" \
  "s_call_b64 s[60:61], Lpoll_3_%=\n\t" \
  "Lok5_3_%=:\n\t" \
  "s_add_u32 s52, s68, 0x90000\n\t" \
  "buffer_load_dwordx4 a[32:35], v4, s[12:15], s52 offen offset:0 sc0 sc1\n\t" \
  "v_mfma_f32_16x16x128_f8f6f4 v[152:155], v[104:111], a[240:247], v[152:155]\n\t" \
  "ds_read_b64_tr_b8 v[122:123], v83 offset:58368\n\t" \
  "v_mfma_f32_16x16x128_f8f6f4 v[148:151], v[104:111], a[232:239], v[148:151]\n\t" \
  "ds_read_b64_tr_b8 v[124:125], v99 offset:24576\n\t" \
  "s_add_u32 s53, s68, 0x90000\n\t" \
  "buffer_load_dwordx4 a[36:39], v4, s[12:15], s53 offen offset:1024 sc0 sc1\n\t" \
  "v_mfma_f32_16x16x128_f8f6f4 v[144:147], v[104:111], a[224:231], v[144:147]\n\t" \
  "ds_read_b64_tr_b8 v[126:127], v99 offset:25600\n\t" \
  "s_waitcnt lgkmcnt(4)\n\t" \
  "v_mfma_f32_16x16x128_f8f6f4 v[160:163], v[112:119], a[224:231], v[160:163]\n\t" \
  "ds_read_b64_tr_b8 v[104:105], v84 offset:57344\n\t" \
  "s_add_u32 s54, s68, 0x90000\n\t" \
  "buffer_load_dwordx4 a[40:43], v4, s[12:15], s54 offen offset:2048 sc0 sc1\n\t" \
  "v_mfma_f32_16x16x128_f8f6f4 v[164:167], v[112:119], a[232:239], v[164:167]\n\t" \
  "ds_read_b64_tr_b8 v[106:107], v84 offset:58368\n\t" \
  "v_mfma_f32_16x16x128_f8f6f4 v[168:171], v[112:119], a[240:247], v[168:171]\n\t" \
  "ds_read_b64_tr_b8 v[108:109], v100 offset:24576\n\t" \
  "s_add_u32 s55, s68, 0x90000\n\t" \
  "buffer_load_dwordx4 a[44:47], v4, s[12:15], s55 offen offset:3072 sc0 sc1\n\t" \
  "v_mfma_f32_16x16x128_f8f6f4 v[172:175], v[112:119], a[248:255], v[172:175]\n\t" \
  "ds_read_b64_tr_b8 v[110:111], v100 offset:25600\n\t" \
  "s_waitcnt lgkmcnt(4)\n\t" \
  "v_mfma_f32_16x16x128_f8f6f4 v[188:191], v[120:127], a[248:255], v[188:191]\n\t" \
  "ds_read_b64_tr_b8 v[112:113], v85 offset:57344\n\t" \
  "s_add_u32 s48, s68, 0x91000\n\t" \
  "buffer_load_dwordx4 a[48:51], v4, s[12:15], s48 offen offset:0 sc0 sc1\n\t" \
  "v_mfma_f32_16x16x128_f8f6f4 v[184:187], v[120:127], a[240:247], v[184:187]\n\t" \
  "ds_read_b64_tr_b8 v[114:115], v85 offset:58368\n\t" \
  "v_mfma_f32_16x16x128_f8f6f4 v[180:183], v[120:127], a[232:239], v[180:183]\n\t" \
  "ds_read_b64_tr_b8 v[116:117], v101 offset:24576\n\t" \
  "s_add_u32 s49, s68, 0x91000\n\t" \
  "buffer_load_dwordx4 a[52:55], v4, s[12:15], s49 offen offset:1024 sc0 sc1\n\t" \
  "v_mfma_f32_16x16x128_f8f6f4 v[176:179], v[120:127], a[224:231], v[176:179]\n\t" \
  "ds_read_b64_tr_b8 v[118:119], v101 offset:25600\n\t" \
  "s_waitcnt lgkmcnt(4)\n\t" \
  "v_mfma_f32_16x16x128_f8f6f4 v[192:195], v[104:111], a[224:231], v[192:195]\n\t" \
  "ds_read_b64_tr_b8 v[120:121], v86 offset:57344\n\t" \
  "s_add_u32 s50, s68, 0x91000\n\t" \
  "buffer_load_dwordx4 a[56:59], v4, s[12:15], s50 offen offset:2048 sc0 sc1\n\t" \
  "v_mfma_f32_16x16x128_f8f6f4 v[196:199], v[104:111], a[232:239], v[196:199]\n\t" \
  "ds_read_b64_tr_b8 v[122:123], v86 offset:58368\n\t" \
  "v_mfma_f32_16x16x128_f8f6f4 v[200:203], v[104:111], a[240:247], v[200:203]\n\t" \
  "ds_read_b64_tr_b8 v[124:125], v102 offset:24576\n\t" \
  "s_add_u32 s51, s68, 0x91000\n\t" \
  "buffer_load_dwordx4 a[60:63], v4, s[12:15], s51 offen offset:3072 sc0 sc1\n\t" \
  "v_mfma_f32_16x16x128_f8f6f4 v[204:207], v[104:111], a[248:255], v[204:207]\n\t" \
  "ds_read_b64_tr_b8 v[126:127], v102 offset:25600\n\t" \
  "s_waitcnt lgkmcnt(4)\n\t" \
  "v_mfma_f32_16x16x128_f8f6f4 v[220:223], v[112:119], a[248:255], v[220:223]\n\t" \
  "ds_read_b64_tr_b8 v[104:105], v87 offset:57344\n\t" \
  "s_add_u32 m0, s24, 0x1c000\n\t" \
  "s_add_u32 s52, s46, 0x0\n\t" \
  "buffer_load_dwordx4 v3, s[8:11], s52 offen lds\n\t" \
  "v_mfma_f32_16x16x128_f8f6f4 v[216:219], v[112:119], a[240:247], v[216:219]\n\t" \
  "ds_read_b64_tr_b8 v[106:107], v87 offset:58368\n\t" \
  "v_mfma_f32_16x16x128_f8f6f4 v[212:215], v[112:119], a[232:239], v[212:215]\n\t" \
  "ds_read_b64_tr_b8 v[108:109], v103 offset:24576\n\t" \
  "s_add_u32 m0, s24, 0x1c400\n\t" \
  "s_add_u32 s53, s46, 0x20000\n\t" \
  "buffer_load_dwordx4 v3, s[8:11], s53 offen lds\n\t" \
  "v_mfma_f32_16x16x128_f8f6f4 v[208:211], v[112:119], a[224:231], v[208:211]\n\t" \
  "ds_read_b64_tr_b8 v[110:111], v103 offset:25600\n\t" \
  "s_waitcnt vmcnt(21) lgkmcnt(0)\n\t" \
  "s_barrier\n\t" \
  "v_mfma_f32_16x16x128_f8f6f4 v[224:227], v[120:127], a[224:231], v[224:227]\n\t" \
  "ds_read_b64_tr_b8 v[112:113], v88 offset:0\n\t" \
  "s_and_b32 s64, s56, 0x40\n\t" \
  "s_cmp_eq_u32 s64, 0x40\n\t" \
  "s_cbranch_scc1 Lok6_3_%=\n\t" \
  "s_mov_b32 s62, 0x40\n\t" \
  "s_mov_b32 s69, 0x7f\n\t" \
  "s_call_b64 s[60:61], Lpoll_3_%=\n\t" \
  "Lok6_3_%=:\n\t" \
  "s_add_u32 s54, s68, 0xa0000\n\t" \
  "buffer_load_dwordx4 a[64:67], v4, s[12:15], s54 offen offset:0 sc0 sc1\n\t" \
  "v_mfma_f32_16x16x128_f8f6f4 v[228:231], v[120:127], a[232:239], v[228:231]\n\t" \
  "ds_read_b64_tr_b8 v[114:115], v88 offset:1024\n\t" \
  "v_mfma_f32_16x16x128_f8f6f4 v[232:235], v[120:127], a[240:247], v[232:235]\n\t" \
  "ds_read_b64_tr_b8 v[116:117], v88 offset:32768\n\t" \
  "s_add_u32 s55, s68, 0xa0000\n\t" \
  "buffer_load_dwordx4 a[68:71], v4, s[12:15], s55 offen offset:1024 sc0 sc1\n\t" \
  "v_mfma_f32_16x16x128_f8f6f4 v[236:239], v[120:127], a[248:255], v[236:239]\n\t" \
  "ds_read_b64_tr_b8 v[118:119], v88 offset:33792\n\t" \
  "v_mfma_f32_16x16x128_f8f6f4 v[252:255], v[104:111], a[248:255], v[252:255]\n\t" \
  "ds_read_b64_tr_b8 v[120:121], v89 offset:0\n\t" \
  "s_add_u32 s48, s68, 0xa0000\n\t" \
  "buffer_load_dwordx4 a[72:75], v4, s[12:15], s48 offen offset:2048 sc0 sc1\n\t" \
  "v_mfma_f32_16x16x128_f8f6f4 v[248:251], v[104:111], a[240:247], v[248:251]\n\t" \
  "ds_read_b64_tr_b8 v[122:123], v89 offset:1024\n\t" \
  "v_mfma_f32_16x16x128_f8f6f4 v[244:247], v[104:111], a[232:239], v[244:247]\n\t" \
  "ds_read_b64_tr_b8 v[124:125], v89 offset:32768\n\t" \
  "s_add_u32 s49, s68, 0xa0000\n\t" \
  "buffer_load_dwordx4 a[76:79], v4, s[12:15], s49 offen offset:3072 sc0 sc1\n\t" \
  "v_mfma_f32_16x16x128_f8f6f4 v[240:243], v[104:111], a[224:231], v[240:243]\n\t" \
  "ds_read_b64_tr_b8 v[126:127], v89 offset:33792\n\t" \
  "s_waitcnt vmcnt(17) lgkmcnt(4)\n\t" \
  "v_mfma_f32_16x16x128_f8f6f4 v[128:131], v[112:119], a[0:7], v[128:131]\n\t" \
  "ds_read_b64_tr_b8 v[104:105], v90 offset:0\n\t" \
  "s_add_u32 s50, s68, 0xa1000\n\t" \
  "buffer_load_dwordx4 a[80:83], v4, s[12:15], s50 offen offset:0 sc0 sc1\n\t" \
  "v_mfma_f32_16x16x128_f8f6f4 v[132:135], v[112:119], a[8:15], v[132:135]\n\t" \
  "ds_read_b64_tr_b8 v[106:107], v90 offset:1024\n\t" \
  "v_mfma_f32_16x16x128_f8f6f4 v[136:139], v[112:119], a[16:23], v[136:139]\n\t" \
  "ds_read_b64_tr_b8 v[108:109], v90 offset:32768\n\t" \
  "s_add_u32 s51, s68, 0xa1000\n\t" \
  "buffer_load_dwordx4 a[84:87], v4, s[12:15], s51 offen offset:1024 sc0 sc1\n\t" \
  "v_mfma_f32_16x16x128_f8f6f4 v[140:143], v[112:119], a[24:31], v[140:143]\n\t" \
  "ds_read_b64_tr_b8 v[110:111], v90 offset:33792\n\t" \
  "s_waitcnt lgkmcnt(4)\n\t" \
  "v_mfma_f32_16x16x128_f8f6f4 v[156:159], v[120:127], a[24:31], v[156:159]\n\t" \
  "ds_read_b64_tr_b8 v[112:113], v91 offset:0\n\t" \
  "s_add_u32 s52, s68, 0xa1000\n\t" \
  "buffer_load_dwordx4 a[88:91], v4, s[12:15], s52 offen offset:2048 sc0 sc1\n\t" \
  "v_mfma_f32_16x16x128_f8f6f4 v[152:155], v[120:127], a[16:23], v[152:155]\n\t" \
  "ds_read_b64_tr_b8 v[114:115], v91 offset:1024\n\t" \
  "v_mfma_f32_16x16x128_f8f6f4 v[148:151], v[120:127], a[8:15], v[148:151]\n\t" \
  "ds_read_b64_tr_b8 v[116:117], v91 offset:32768\n\t" \
  "s_add_u32 s53, s68, 0xa1000\n\t" \
  "buffer_load_dwordx4 a[92:95], v4, s[12:15], s53 offen offset:3072 sc0 sc1\n\t" \
  "v_mfma_f32_16x16x128_f8f6f4 v[144:147], v[120:127], a[0:7], v[144:147]\n\t" \
  "ds_read_b64_tr_b8 v[118:119], v91 offset:33792\n\t" \
  "s_waitcnt lgkmcnt(4)\n\t" \
  "v_mfma_f32_16x16x128_f8f6f4 v[160:163], v[104:111], a[0:7], v[160:163]\n\t" \
  "ds_read_b64_tr_b8 v[120:121], v92 offset:0\n\t" \
  "s_add_u32 m0, s24, 0x1e000\n\t" \
  "s_add_u32 s54, s40, 0x200000\n\t" \
  "buffer_load_dwordx4 v3, s[8:11], s54 offen lds\n\t" \
  "v_mfma_f32_16x16x128_f8f6f4 v[164:167], v[104:111], a[8:15], v[164:167]\n\t" \
  "ds_read_b64_tr_b8 v[122:123], v92 offset:1024\n\t" \
  "v_mfma_f32_16x16x128_f8f6f4 v[168:171], v[104:111], a[16:23], v[168:171]\n\t" \
  "ds_read_b64_tr_b8 v[124:125], v92 offset:32768\n\t" \
  "s_add_u32 m0, s24, 0x1e400\n\t" \
  "s_add_u32 s55, s40, 0x220000\n\t" \
  "buffer_load_dwordx4 v3, s[8:11], s55 offen lds\n\t" \
  "v_mfma_f32_16x16x128_f8f6f4 v[172:175], v[104:111], a[24:31], v[172:175]\n\t" \
  "ds_read_b64_tr_b8 v[126:127], v92 offset:33792\n\t" \
  "s_waitcnt lgkmcnt(4)\n\t" \
  "v_mfma_f32_16x16x128_f8f6f4 v[188:191], v[112:119], a[24:31], v[188:191]\n\t" \
  "ds_read_b64_tr_b8 v[104:105], v93 offset:0\n\t" \
  "s_waitcnt vmcnt(21)\n\t" \
  "v_min_u32 v68, v68, v69\n\t" \
  "v_min3_u32 v68, v68, v70, v71\n\t" \
  "v_cmp_le_u32 vcc, s57, v68\n\t" \
  "s_and_b32 s64, vcc_lo, 0x7f00\n\t" \
  "s_or_b32 s56, s56, s64\n\t" \
  "s_and_b32 s64, s56, 0x7f00\n\t" \
  "s_cmp_eq_u32 s64, 0x7f00\n\t" \
  "s_cbranch_scc1 Lok7_3_%=\n\t" \
  "s_mov_b32 s62, 0x7f00\n\t" \
  "s_mov_b32 s69, 0x7f00\n\t" \
  "s_call_b64 s[60:61], Lpoll_3_%=\n\t" \
  "Lok7_3_%=:\n\t" \
  "s_add_u32 s48, s68, 0xb0000\n\t" \
  "buffer_load_dwordx4 a[96:99], v4, s[12:15], s48 offen offset:0 sc0 sc1\n\t" \
  "v_mfma_f32_16x16x128_f8f6f4 v[184:187], v[112:119], a[16:23], v[184:187]\n\t" \
  "ds_read_b64_tr_b8 v[106:107], v93 offset:1024\n\t" \
  "v_mfma_f32_16x16x128_f8f6f4 v[180:183], v[112:119], a[8:15], v[180:183]\n\t" \
  "ds_read_b64_tr_b8 v[108:109], v93 offset:32768\n\t" \
  "s_add_u32 s49, s68, 0xb0000\n\t" \
  "buffer_load_dwordx4 a[100:103], v4, s[12:15], s49 offen offset:1024 sc0 sc1\n\t" \
  "v_mfma_f32_16x16x128_f8f6f4 v[176:179], v[112:119], a[0:7], v[176:179]\n\t" \
  "ds_read_b64_tr_b8 v[110:111], v93 offset:33792\n\t" \
  "s_waitcnt lgkmcnt(4)\n\t" \
  "v_mfma_f32_16x16x128_f8f6f4 v[192:195], v[120:127], a[0:7], v[192:195]\n\t" \
  "ds_read_b64_tr_b8 v[112:113], v94 offset:0\n\t" \
  "s_add_u32 s50, s68, 0xb0000\n\t" \
  "buffer_load_dwordx4 a[104:107], v4, s[12:15], s50 offen offset:2048 sc0 sc1\n\t" \
  "v_mfma_f32_16x16x128_f8f6f4 v[196:199], v[120:127], a[8:15], v[196:199]\n\t" \
  "ds_read_b64_tr_b8 v[114:115], v94 offset:1024\n\t" \
  "v_mfma_f32_16x16x128_f8f6f4 v[200:203], v[120:127], a[16:23], v[200:203]\n\t" \
  "ds_read_b64_tr_b8 v[116:117], v94 offset:32768\n\t" \
  "s_add_u32 s51, s68, 0xb0000\n\t" \
  "buffer_load_dwordx4 a[108:111], v4, s[12:15], s51 offen offset:3072 sc0 sc1\n\t" \
  "v_mfma_f32_16x16x128_f8f6f4 v[204:207], v[120:127], a[24:31], v[204:207]\n\t" \
  "ds_read_b64_tr_b8 v[118:119], v94 offset:33792\n\t" \
  "s_waitcnt lgkmcnt(4)\n\t" \
  "v_mfma_f32_16x16x128_f8f6f4 v[220:223], v[104:111], a[24:31], v[220:223]\n\t" \
  "ds_read_b64_tr_b8 v[120:121], v95 offset:0\n\t" \
  "s_add_u32 s52, s68, 0xb1000\n\t" \
  "buffer_load_dwordx4 a[112:115], v4, s[12:15], s52 offen offset:0 sc0 sc1\n\t" \
  "v_mfma_f32_16x16x128_f8f6f4 v[216:219], v[104:111], a[16:23], v[216:219]\n\t" \
  "ds_read_b64_tr_b8 v[122:123], v95 offset:1024\n\t" \
  "v_mfma_f32_16x16x128_f8f6f4 v[212:215], v[104:111], a[8:15], v[212:215]\n\t" \
  "ds_read_b64_tr_b8 v[124:125], v95 offset:32768\n\t" \
  "s_add_u32 s53, s68, 0xb1000\n\t" \
  "buffer_load_dwordx4 a[116:119], v4, s[12:15], s53 offen offset:1024 sc0 sc1\n\t" \
  "v_mfma_f32_16x16x128_f8f6f4 v[208:211], v[104:111], a[0:7], v[208:211]\n\t" \
  "ds_read_b64_tr_b8 v[126:127], v95 offset:33792\n\t" \
  "s_waitcnt vmcnt(26) lgkmcnt(0)\n\t" \
  "s_barrier\n\t" \
  "v_mfma_f32_16x16x128_f8f6f4 v[224:227], v[112:119], a[0:7], v[224:227]\n\t" \
  "ds_read_b64_tr_b8 v[104:105], v88 offset:8192\n\t" \
  "s_add_u32 s54, s68, 0xb1000\n\t" \
  "buffer_load_dwordx4 a[120:123], v4, s[12:15], s54 offen offset:2048 sc0 sc1\n\t" \
  "v_mfma_f32_16x16x128_f8f6f4 v[228:231], v[112:119], a[8:15], v[228:231]\n\t" \
  "ds_read_b64_tr_b8 v[106:107], v88 offset:9216\n\t" \
  "v_mfma_f32_16x16x128_f8f6f4 v[232:235], v[112:119], a[16:23], v[232:235]\n\t" \
  "ds_read_b64_tr_b8 v[108:109], v88 offset:40960\n\t" \
  "s_add_u32 s55, s68, 0xb1000\n\t" \
  "buffer_load_dwordx4 a[124:127], v4, s[12:15], s55 offen offset:3072 sc0 sc1\n\t" \
  "v_mfma_f32_16x16x128_f8f6f4 v[236:239], v[112:119], a[24:31], v[236:239]\n\t" \
  "ds_read_b64_tr_b8 v[110:111], v88 offset:41984\n\t" \
  "v_mfma_f32_16x16x128_f8f6f4 v[252:255], v[120:127], a[24:31], v[252:255]\n\t" \
  "ds_read_b64_tr_b8 v[112:113], v89 offset:8192\n\t" \
  "s_cmp_lg_u32 s26, 2\n\t" \
  "s_cbranch_scc1 Lnf2_3_%=\n\t" \
  "s_mov_b64 exec, 1\n\t" \
  "global_store_dword v65, v64, s[30:31] sc0 sc1\n\t" \
  "s_mov_b64 exec, -1\n\t" \
  "s_branch Lmf2_3_%=\n\t" \
  "Lnf2_3_%=:\n\t" \
  "buffer_load_dword v62, v65, s[20:23], s22 offen\n\t" \
  "Lmf2_3_%=:\n\t" \
  "s_cmp_lg_u32 s26, 3\n\t" \
  "s_cbranch_scc1 Lns3_3_%=\n\t" \
  "global_store_dwordx4 v63, v[72:75], s[28:29] sc0 sc1\n\t" \
  "global_store_dwordx4 v63, v[76:79], s[28:29] offset:1024 sc0 sc1\n\t" \
  "s_branch Lms3_3_%=\n\t" \
  "Lns3_3_%=:\n\t" \
  "buffer_load_dword v62, v65, s[20:23], s22 offen\n\t" \
  "buffer_load_dword v62, v65, s[20:23], s22 offen\n\t" \
  "Lms3_3_%=:\n\t" \
  "s_add_u32 m0, s24, 0x0\n\t" \
  "s_add_u32 s48, s41, 0x200000\n\t" \
  "buffer_load_dwordx4 v2, s[8:11], s48 offen lds\n\t" \
  "v_mfma_f32_16x16x128_f8f6f4 v[248:251], v[120:127], a[16:23], v[248:251]\n\t" \
  "ds_read_b64_tr_b8 v[114:115], v89 offset:9216\n\t" \
  "v_mfma_f32_16x16x128_f8f6f4 v[244:247], v[120:127], a[8:15], v[244:247]\n\t" \
  "ds_read_b64_tr_b8 v[116:117], v89 offset:40960\n\t" \
  "s_add_u32 m0, s24, 0x400\n\t" \
  "s_add_u32 s49, s41, 0x220000\n\t" \
  "buffer_load_dwordx4 v2, s[8:11], s49 offen lds\n\t" \
  "v_mfma_f32_16x16x128_f8f6f4 v[240:243], v[120:127], a[0:7], v[240:243]\n\t" \
  "ds_read_b64_tr_b8 v[118:119], v89 offset:41984\n\t" \
  "s_waitcnt vmcnt(25) lgkmcnt(4)\n\t" \
  "v_mfma_f32_16x16x128_f8f6f4 v[128:131], v[104:111], a[32:39], v[128:131]\n\t" \
  "ds_read_b64_tr_b8 v[120:121], v90 offset:8192\n\t" \
  "s_add_u32 m0, s24, 0x2000\n\t" \
  "s_add_u32 s50, s42, 0x200000\n\t" \
  "buffer_load_dwordx4 v2, s[8:11], s50 offen lds\n\t" \
  "v_mfma_f32_16x16x128_f8f6f4 v[132:135], v[104:111], a[40:47], v[132:135]\n\t" \
  "ds_read_b64_tr_b8 v[122:123], v90 offset:9216\n\t" \
  "v_mfma_f32_16x16x128_f8f6f4 v[136:139], v[104:111], a[48:55], v[136:139]\n\t" \
  "ds_read_b64_tr_b8 v[124:125], v90 offset:40960\n\t" \
  "s_add_u32 m0, s24, 0x2400\n\t" \
  "s_add_u32 s51, s42, 0x220000\n\t" \
  "buffer_load_dwordx4 v2, s[8:11], s51 offen lds\n\t" \
  "v_mfma_f32_16x16x128_f8f6f4 v[140:143], v[104:111], a[56:63], v[140:143]\n\t" \
  "ds_read_b64_tr_b8 v[126:127], v90 offset:41984\n\t" \
  "s_waitcnt lgkmcnt(4)\n\t" \
  "v_mfma_f32_16x16x128_f8f6f4 v[156:159], v[112:119], a[56:63], v[156:159]\n\t" \
  "ds_read_b64_tr_b8 v[104:105], v91 offset:8192\n\t" \
  "s_add_u32 m0, s24, 0x4000\n\t" \
  "s_add_u32 s52, s43, 0x200000\n\t" \
  "buffer_load_dwordx4 v2, s[8:11], s52 offen lds\n\t" \
  "v_mfma_f32_16x16x128_f8f6f4 v[152:155], v[112:119], a[48:55], v[152:155]\n\t" \
  "ds_read_b64_tr_b8 v[106:107], v91 offset:9216\n\t" \
  "v_mfma_f32_16x16x128_f8f6f4 v[148:151], v[112:119], a[40:47], v[148:151]\n\t" \
  "ds_read_b64_tr_b8 v[108:109], v91 offset:40960\n\t" \
  "s_add_u32 m0, s24, 0x4400\n\t" \
  "s_add_u32 s53, s43, 0x220000\n\t" \
  "buffer_load_dwordx4 v2, s[8:11], s53 offen lds\n\t" \
  "v_mfma_f32_16x16x128_f8f6f4 v[144:147], v[112:119], a[32:39], v[144:147]\n\t" \
  "ds_read_b64_tr_b8 v[110:111], v91 offset:41984\n\t" \
  "s_waitcnt lgkmcnt(4)\n\t" \
  "v_mfma_f32_16x16x128_f8f6f4 v[160:163], v[120:127], a[32:39], v[160:163]\n\t" \
  "ds_read_b64_tr_b8 v[112:113], v92 offset:8192\n\t" \
  "s_add_u32 m0, s24, 0x6000\n\t" \
  "s_add_u32 s54, s44, 0x200000\n\t" \
  "buffer_load_dwordx4 v2, s[8:11], s54 offen lds\n\t" \
  "v_mfma_f32_16x16x128_f8f6f4 v[164:167], v[120:127], a[40:47], v[164:167]\n\t" \
  "ds_read_b64_tr_b8 v[114:115], v92 offset:9216\n\t" \
  "v_mfma_f32_16x16x128_f8f6f4 v[168:171], v[120:127], a[48:55], v[168:171]\n\t" \
  "ds_read_b64_tr_b8 v[116:117], v92 offset:40960\n\t" \
  "s_add_u32 m0, s24, 0x6400\n\t" \
  "s_add_u32 s55, s44, 0x220000\n\t" \
  "buffer_load_dwordx4 v2, s[8:11], s55 offen lds\n\t" \
  "v_mfma_f32_16x16x128_f8f6f4 v[172:175], v[120:127], a[56:63], v[172:175]\n\t" \
  "ds_read_b64_tr_b8 v[118:119], v92 offset:41984\n\t" \
  "s_waitcnt lgkmcnt(4)\n\t" \
  "v_mfma_f32_16x16x128_f8f6f4 v[188:191], v[104:111], a[56:63], v[188:191]\n\t" \
  "ds_read_b64_tr_b8 v[120:121], v93 offset:8192\n\t" \
  "v_mfma_f32_16x16x128_f8f6f4 v[184:187], v[104:111], a[48:55], v[184:187]\n\t" \
  "ds_read_b64_tr_b8 v[122:123], v93 offset:9216\n\t" \
  "v_mfma_f32_16x16x128_f8f6f4 v[180:183], v[104:111], a[40:47], v[180:183]\n\t" \
  "ds_read_b64_tr_b8 v[124:125], v93 offset:40960\n\t" \
  "v_mfma_f32_16x16x128_f8f6f4 v[176:179], v[104:111], a[32:39], v[176:179]\n\t" \
  "ds_read_b64_tr_b8 v[126:127], v93 offset:41984\n\t" \
  "s_waitcnt lgkmcnt(4)\n\t" \
  "v_mfma_f32_16x16x128_f8f6f4 v[192:195], v[112:119], a[32:39], v[192:195]\n\t" \
  "ds_read_b64_tr_b8 v[104:105], v94 offset:8192\n\t" \
  "v_mfma_f32_16x16x128_f8f6f4 v[196:199], v[112:119], a[40:47], v[196:199]\n\t" \
  "ds_read_b64_tr_b8 v[106:107], v94 offset:9216\n\t" \
  "v_mfma_f32_16x16x128_f8f6f4 v[200:203], v[112:119], a[48:55], v[200:203]\n\t" \
  "ds_read_b64_tr_b8 v[108:109], v94 offset:40960\n\t" \
  "v_mfma_f32_16x16x128_f8f6f4 v[204:207], v[112:119], a[56:63], v[204:207]\n\t" \
  "ds_read_b64_tr_b8 v[110:111], v94 offset:41984\n\t" \
  "s_waitcnt lgkmcnt(4)\n\t" \
  "v_mfma_f32_16x16x128_f8f6f4 v[220:223], v[120:127], a[56:63], v[220:223]\n\t" \
  "ds_read_b64_tr_b8 v[112:113], v95 offset:8192\n\t" \
  "v_mfma_f32_16x16x128_f8f6f4 v[216:219], v[120:127], a[48:55], v[216:219]\n\t" \
  "ds_read_b64_tr_b8 v[114:115], v95 offset:9216\n\t" \
  "v_mfma_f32_16x16x128_f8f6f4 v[212:215], v[120:127], a[40:47], v[212:215]\n\t" \
  "ds_read_b64_tr_b8 v[116:117], v95 offset:40960\n\t" \
  "v_mfma_f32_16x16x128_f8f6f4 v[208:211], v[120:127], a[32:39], v[208:211]\n\t" \
  "ds_read_b64_tr_b8 v[118:119], v95 offset:41984\n\t" \
  "s_waitcnt vmcnt(29) lgkmcnt(0)\n\t" \
  "s_barrier\n\t" \
  "v_mfma_f32_16x16x128_f8f6f4 v[224:227], v[104:111], a[32:39], v[224:227]\n\t" \
  "ds_read_b64_tr_b8 v[120:121], v88 offset:16384\n\t" \
  "v_mfma_f32_16x16x128_f8f6f4 v[228:231], v[104:111], a[40:47], v[228:231]\n\t" \
  "ds_read_b64_tr_b8 v[122:123], v88 offset:17408\n\t" \
  "v_mfma_f32_16x16x128_f8f6f4 v[232:235], v[104:111], a[48:55], v[232:235]\n\t" \
  "ds_read_b64_tr_b8 v[124:125], v88 offset:49152\n\t" \
  "v_mfma_f32_16x16x128_f8f6f4 v[236:239], v[104:111], a[56:63], v[236:239]\n\t" \
  "ds_read_b64_tr_b8 v[126:127], v88 offset:50176\n\t" \
  "v_mfma_f32_16x16x128_f8f6f4 v[252:255], v[112:119], a[56:63], v[252:255]\n\t" \
  "ds_read_b64_tr_b8 v[104:105], v89 offset:16384\n\t" \
  "v_mfma_f32_16x16x128_f8f6f4 v[248:251], v[112:119], a[48:55], v[248:251]\n\t" \
  "ds_read_b64_tr_b8 v[106:107], v89 offset:17408\n\t" \
  "v_mfma_f32_16x16x128_f8f6f4 v[244:247], v[112:119], a[40:47], v[244:247]\n\t" \
  "ds_read_b64_tr_b8 v[108:109], v89 offset:49152\n\t" \
  "v_mfma_f32_16x16x128_f8f6f4 v[240:243], v[112:119], a[32:39], v[240:243]\n\t" \
  "ds_read_b64_tr_b8 v[110:111], v89 offset:50176\n\t" \
  "s_waitcnt vmcnt(21) lgkmcnt(4)\n\t" \
  "v_mfma_f32_16x16x128_f8f6f4 v[128:131], v[120:127], a[64:71], v[128:131]\n\t" \
  "ds_read_b64_tr_b8 v[112:113], v90 offset:16384\n\t" \
  "s_add_u32 m0, s24, 0x20000\n\t" \
  "s_add_u32 s48, s41, 0x200000\n\t" \
  "buffer_load_dwordx4 v3, s[8:11], s48 offen lds\n\t" \
  "v_mfma_f32_16x16x128_f8f6f4 v[132:135], v[120:127], a[72:79], v[132:135]\n\t" \
  "ds_read_b64_tr_b8 v[114:115], v90 offset:17408\n\t" \
  "v_mfma_f32_16x16x128_f8f6f4 v[136:139], v[120:127], a[80:87], v[136:139]\n\t" \
  "ds_read_b64_tr_b8 v[116:117], v90 offset:49152\n\t" \
  "s_add_u32 m0, s24, 0x20400\n\t" \
  "s_add_u32 s49, s41, 0x220000\n\t" \
  "buffer_load_dwordx4 v3, s[8:11], s49 offen lds\n\t" \
  "v_mfma_f32_16x16x128_f8f6f4 v[140:143], v[120:127], a[88:95], v[140:143]\n\t" \
  "ds_read_b64_tr_b8 v[118:119], v90 offset:50176\n\t" \
  "s_waitcnt lgkmcnt(4)\n\t" \
  "v_mfma_f32_16x16x128_f8f6f4 v[156:159], v[104:111], a[88:95], v[156:159]\n\t" \
  "ds_read_b64_tr_b8 v[120:121], v91 offset:16384\n\t" \
  "s_add_u32 s50, s68, 0xc0000\n\t" \
  "buffer_load_dwordx4 a[128:131], v4, s[12:15], s50 offen offset:0 sc0 sc1\n\t" \
  "v_mfma_f32_16x16x128_f8f6f4 v[152:155], v[104:111], a[80:87], v[152:155]\n\t" \
  "ds_read_b64_tr_b8 v[122:123], v91 offset:17408\n\t" \
  "v_mfma_f32_16x16x128_f8f6f4 v[148:151], v[104:111], a[72:79], v[148:151]\n\t" \
  "ds_read_b64_tr_b8 v[124:125], v91 offset:49152\n\t" \
  "s_add_u32 s51, s68, 0xc0000\n\t" \
  "buffer_load_dwordx4 a[132:135], v4, s[12:15], s51 offen offset:1024 sc0 sc1\n\t" \
  "v_mfma_f32_16x16x128_f8f6f4 v[144:147], v[104:111], a[64:71], v[144:147]\n\t" \
  "ds_read_b64_tr_b8 v[126:127], v91 offset:50176\n\t" \
  "s_waitcnt lgkmcnt(4)\n\t" \
  "v_mfma_f32_16x16x128_f8f6f4 v[160:163], v[112:119], a[64:71], v[160:163]\n\t" \
  "ds_read_b64_tr_b8 v[104:105], v92 offset:16384\n\t" \
  "s_add_u32 s52, s68, 0xc0000\n\t" \
  "buffer_load_dwordx4 a[136:139], v4, s[12:15], s52 offen offset:2048 sc0 sc1\n\t" \
  "v_mfma_f32_16x16x128_f8f6f4 v[164:167], v[112:119], a[72:79], v[164:167]\n\t" \
  "ds_read_b64_tr_b8 v[106:107], v92 offset:17408\n\t" \
  "v_mfma_f32_16x16x128_f8f6f4 v[168:171], v[112:119], a[80:87], v[168:171]\n\t" \
  "ds_read_b64_tr_b8 v[108:109], v92 offset:49152\n\t" \
  "s_add_u32 s53, s68, 0xc0000\n\t" \
  "buffer_load_dwordx4 a[140:143], v4, s[12:15], s53 offen offset:3072 sc0 sc1\n\t" \
  "v_mfma_f32_16x16x128_f8f6f4 v[172:175], v[112:119], a[88:95], v[172:175]\n\t" \
  "ds_read_b64_tr_b8 v[110:111], v92 offset:50176\n\t" \
  "s_waitcnt lgkmcnt(4)\n\t" \
  "v_mfma_f32_16x16x128_f8f6f4 v[188:191], v[120:127], a[88:95], v[188:191]\n\t" \
  "ds_read_b64_tr_b8 v[112:113], v93 offset:16384\n\t" \
  "s_add_u32 s54, s68, 0xc1000\n\t" \
  "buffer_load_dwordx4 a[144:147], v4, s[12:15], s54 offen offset:0 sc0 sc1\n\t" \
  "v_mfma_f32_16x16x128_f8f6f4 v[184:187], v[120:127], a[80:87], v[184:187]\n\t" \
  "ds_read_b64_tr_b8 v[114:115], v93 offset:17408\n\t" \
  "v_mfma_f32_16x16x128_f8f6f4 v[180:183], v[120:127], a[72:79], v[180:183]\n\t" \
  "ds_read_b64_tr_b8 v[116:117], v93 offset:49152\n\t" \
  "s_add_u32 s55, s68, 0xc1000\n\t" \
  "buffer_load_dwordx4 a[148:151], v4, s[12:15], s55 offen offset:1024 sc0 sc1\n\t" \
  "v_mfma_f32_16x16x128_f8f6f4 v[176:179], v[120:127], a[64:71], v[176:179]\n\t" \
  "ds_read_b64_tr_b8 v[118:119], v93 offset:50176\n\t" \
  "s_waitcnt lgkmcnt(4)\n\t" \
  "v_mfma_f32_16x16x128_f8f6f4 v[192:195], v[104:111], a[64:71], v[192:195]\n\t" \
  "ds_read_b64_tr_b8 v[120:121], v94 offset:16384\n\t" \
  "s_add_u32 s48, s68, 0xc1000\n\t" \
  "buffer_load_dwordx4 a[152:155], v4, s[12:15], s48 offen offset:2048 sc0 sc1\n\t" \
  "v_mfma_f32_16x16x128_f8f6f4 v[196:199], v[104:111], a[72:79], v[196:199]\n\t" \
  "ds_read_b64_tr_b8 v[122:123], v94 offset:17408\n\t" \
  "v_mfma_f32_16x16x128_f8f6f4 v[200:203], v[104:111], a[80:87], v[200:203]\n\t" \
  "ds_read_b64_tr_b8 v[124:125], v94 offset:49152\n\t" \
  "s_add_u32 s49, s68, 0xc1000\n\t" \
  "buffer_load_dwordx4 a[156:159], v4, s[12:15], s49 offen offset:3072 sc0 sc1\n\t" \
  "v_mfma_f32_16x16x128_f8f6f4 v[204:207], v[104:111], a[88:95], v[204:207]\n\t" \
  "ds_read_b64_tr_b8 v[126:127], v94 offset:50176\n\t" \
  "s_waitcnt lgkmcnt(4)\n\t" \
  "v_mfma_f32_16x16x128_f8f6f4 v[220:223], v[112:119], a[88:95], v[220:223]\n\t" \
  "ds_read_b64_tr_b8 v[104:105], v95 offset:16384\n\t" \
  "s_add_u32 m0, s24, 0x22000\n\t" \
  "s_add_u32 s50, s42, 0x200000\n\t" \
  "buffer_load_dwordx4 v3, s[8:11], s50 offen lds\n\t" \
  "v_mfma_f32_16x16x128_f8f6f4 v[216:219], v[112:119], a[80:87], v[216:219]\n\t" \
  "ds_read_b64_tr_b8 v[106:107], v95 offset:17408\n\t" \
  "v_mfma_f32_16x16x128_f8f6f4 v[212:215], v[112:119], a[72:79], v[212:215]\n\t" \
  "ds_read_b64_tr_b8 v[108:109], v95 offset:49152\n\t" \
  "s_add_u32 m0, s24, 0x22400\n\t" \
  "s_add_u32 s51, s42, 0x220000\n\t" \
  "buffer_load_dwordx4 v3, s[8:11], s51 offen lds\n\t" \
  "v_mfma_f32_16x16x128_f8f6f4 v[208:211], v[112:119], a[64:71], v[208:211]\n\t" \
  "ds_read_b64_tr_b8 v[110:111], v95 offset:50176\n\t" \
  "s_waitcnt vmcnt(31) lgkmcnt(0)\n\t" \
  "s_barrier\n\t" \
  "v_mfma_f32_16x16x128_f8f6f4 v[224:227], v[120:127], a[64:71], v[224:227]\n\t" \
  "ds_read_b64_tr_b8 v[112:113], v88 offset:24576\n\t" \
  "s_add_u32 s52, s68, 0xd0000\n\t" \
  "buffer_load_dwordx4 a[160:163], v4, s[12:15], s52 offen offset:0 sc0 sc1\n\t" \
  "v_mfma_f32_16x16x128_f8f6f4 v[228:231], v[120:127], a[72:79], v[228:231]\n\t" \
  "ds_read_b64_tr_b8 v[114:115], v88 offset:25600\n\t" \
  "v_mfma_f32_16x16x128_f8f6f4 v[232:235], v[120:127], a[80:87], v[232:235]\n\t" \
  "ds_read_b64_tr_b8 v[116:117], v88 offset:57344\n\t" \
  "s_add_u32 s53, s68, 0xd0000\n\t" \
  "buffer_load_dwordx4 a[164:167], v4, s[12:15], s53 offen offset:1024 sc0 sc1\n\t" \
  "v_mfma_f32_16x16x128_f8f6f4 v[236:239], v[120:127], a[88:95], v[236:239]\n\t" \
  "ds_read_b64_tr_b8 v[118:119], v88 offset:58368\n\t" \
  "v_mfma_f32_16x16x128_f8f6f4 v[252:255], v[104:111], a[88:95], v[252:255]\n\t" \
  "ds_read_b64_tr_b8 v[120:121], v89 offset:24576\n\t" \
  "s_add_u32 s54, s68, 0xd0000\n\t" \
  "buffer_load_dwordx4 a[168:171], v4, s[12:15], s54 offen offset:2048 sc0 sc1\n\t" \
  "v_mfma_f32_16x16x128_f8f6f4 v[248:251], v[104:111], a[80:87], v[248:251]\n\t" \
  "ds_read_b64_tr_b8 v[122:123], v89 offset:25600\n\t" \
  "v_mfma_f32_16x16x128_f8f6f4 v[244:247], v[104:111], a[72:79], v[244:247]\n\t" \
  "ds_read_b64_tr_b8 v[124:125], v89 offset:57344\n\t" \
  "s_add_u32 s55, s68, 0xd0000\n\t" \
  "buffer_load_dwordx4 a[172:175], v4, s[12:15], s55 offen offset:3072 sc0 sc1\n\t" \
  "v_mfma_f32_16x16x128_f8f6f4 v[240:243], v[104:111], a[64:71], v[240:243]\n\t" \
  "ds_read_b64_tr_b8 v[126:127], v89 offset:58368\n\t" \
  "s_waitcnt vmcnt(27) lgkmcnt(4)\n\t" \
  "v_mfma_f32_16x16x128_f8f6f4 v[128:131], v[112:119], a[96:103], v[128:131]\n\t" \
  "ds_read_b64_tr_b8 v[104:105], v90 offset:24576\n\t" \
  "s_add_u32 s48, s68, 0xd1000\n\t" \
  "buffer_load_dwordx4 a[176:179], v4, s[12:15], s48 offen offset:0 sc0 sc1\n\t" \
  "v_mfma_f32_16x16x128_f8f6f4 v[132:135], v[112:119], a[104:111], v[132:135]\n\t" \
  "ds_read_b64_tr_b8 v[106:107], v90 offset:25600\n\t" \
  "v_mfma_f32_16x16x128_f8f6f4 v[136:139], v[112:119], a[112:119], v[136:139]\n\t" \
  "ds_read_b64_tr_b8 v[108:109], v90 offset:57344\n\t" \
  "s_add_u32 s49, s68, 0xd1000\n\t" \
  "buffer_load_dwordx4 a[180:183], v4, s[12:15], s49 offen offset:1024 sc0 sc1\n\t" \
  "v_mfma_f32_16x16x128_f8f6f4 v[140:143], v[112:119], a[120:127], v[140:143]\n\t" \
  "ds_read_b64_tr_b8 v[110:111], v90 offset:58368\n\t" \
  "s_waitcnt lgkmcnt(4)\n\t" \
  "v_mfma_f32_16x16x128_f8f6f4 v[156:159], v[120:127], a[120:127], v[156:159]\n\t" \
  "ds_read_b64_tr_b8 v[112:113], v91 offset:24576\n\t" \
  "s_add_u32 s50, s68, 0xd1000\n\t" \
  "buffer_load_dwordx4 a[184:187], v4, s[12:15], s50 offen offset:2048 sc0 sc1\n\t" \
  "v_mfma_f32_16x16x128_f8f6f4 v[152:155], v[120:127], a[112:119], v[152:155]\n\t" \
  "ds_read_b64_tr_b8 v[114:115], v91 offset:25600\n\t" \
  "v_mfma_f32_16x16x128_f8f6f4 v[148:151], v[120:127], a[104:111], v[148:151]\n\t" \
  "ds_read_b64_tr_b8 v[116:117], v91 offset:57344\n\t" \
  "s_add_u32 s51, s68, 0xd1000\n\t" \
  "buffer_load_dwordx4 a[188:191], v4, s[12:15], s51 offen offset:3072 sc0 sc1\n\t" \
  "v_mfma_f32_16x16x128_f8f6f4 v[144:147], v[120:127], a[96:103], v[144:147]\n\t" \
  "ds_read_b64_tr_b8 v[118:119], v91 offset:58368\n\t" \
  "s_waitcnt lgkmcnt(4)\n\t" \
  "v_mfma_f32_16x16x128_f8f6f4 v[160:163], v[104:111], a[96:103], v[160:163]\n\t" \
  "ds_read_b64_tr_b8 v[120:121], v92 offset:24576\n\t" \
  "s_add_u32 m0, s24, 0x24000\n\t" \
  "s_add_u32 s52, s43, 0x200000\n\t" \
  "buffer_load_dwordx4 v3, s[8:11], s52 offen lds\n\t" \
  "v_mfma_f32_16x16x128_f8f6f4 v[164:167], v[104:111], a[104:111], v[164:167]\n\t" \
  "ds_read_b64_tr_b8 v[122:123], v92 offset:25600\n\t" \
  "v_mfma_f32_16x16x128_f8f6f4 v[168:171], v[104:111], a[112:119], v[168:171]\n\t" \
  "ds_read_b64_tr_b8 v[124:125], v92 offset:57344\n\t" \
  "s_add_u32 m0, s24, 0x24400\n\t" \
  "s_add_u32 s53, s43, 0x220000\n\t" \
  "buffer_load_dwordx4 v3, s[8:11], s53 offen lds\n\t" \
  "v_mfma_f32_16x16x128_f8f6f4 v[172:175], v[104:111], a[120:127], v[172:175]\n\t" \
  "ds_read_b64_tr_b8 v[126:127], v92 offset:58368\n\t" \
  "s_waitcnt lgkmcnt(4)\n\t" \
  "v_mfma_f32_16x16x128_f8f6f4 v[188:191], v[112:119], a[120:127], v[188:191]\n\t" \
  "ds_read_b64_tr_b8 v[104:105], v93 offset:24576\n\t" \
  "s_add_u32 s54, s68, 0xe0000\n\t" \
  "buffer_load_dwordx4 a[192:195], v4, s[12:15], s54 offen offset:0 sc0 sc1\n\t" \
  "v_mfma_f32_16x16x128_f8f6f4 v[184:187], v[112:119], a[112:119], v[184:187]\n\t" \
  "ds_read_b64_tr_b8 v[106:107], v93 offset:25600\n\t" \
  "v_mfma_f32_16x16x128_f8f6f4 v[180:183], v[112:119], a[104:111], v[180:183]\n\t" \
  "ds_read_b64_tr_b8 v[108:109], v93 offset:57344\n\t" \
  "s_add_u32 s55, s68, 0xe0000\n\t" \
  "buffer_load_dwordx4 a[196:199], v4, s[12:15], s55 offen offset:1024 sc0 sc1\n\t" \
  "v_mfma_f32_16x16x128_f8f6f4 v[176:179], v[112:119], a[96:103], v[176:179]\n\t" \
  "ds_read_b64_tr_b8 v[110:111], v93 offset:58368\n\t" \
  "s_waitcnt lgkmcnt(4)\n\t" \
  "v_mfma_f32_16x16x128_f8f6f4 v[192:195], v[120:127], a[96:103], v[192:195]\n\t" \
  "ds_read_b64_tr_b8 v[112:113], v94 offset:24576\n\t" \
  "s_add_u32 s48, s68, 0xe0000\n\t" \
  "buffer_load_dwordx4 a[200:203], v4, s[12:15], s48 offen offset:2048 sc0 sc1\n\t" \
  "v_mfma_f32_16x16x128_f8f6f4 v[196:199], v[120:127], a[104:111], v[196:199]\n\t" \
  "ds_read_b64_tr_b8 v[114:115], v94 offset:25600\n\t" \
  "v_mfma_f32_16x16x128_f8f6f4 v[200:203], v[120:127], a[112:119], v[200:203]\n\t" \
  "ds_read_b64_tr_b8 v[116:117], v94 offset:57344\n\t" \
  "s_add_u32 s49, s68, 0xe0000\n\t" \
  "buffer_load_dwordx4 a[204:207], v4, s[12:15], s49 offen offset:3072 sc0 sc1\n\t" \
  "v_mfma_f32_16x16x128_f8f6f4 v[204:207], v[120:127], a[120:127], v[204:207]\n\t" \
  "ds_read_b64_tr_b8 v[118:119], v94 offset:58368\n\t" \
  "s_waitcnt lgkmcnt(4)\n\t" \
  "v_mfma_f32_16x16x128_f8f6f4 v[220:223], v[104:111], a[120:127], v[220:223]\n\t" \
  "ds_read_b64_tr_b8 v[120:121], v95 offset:24576\n\t" \
  "s_add_u32 s50, s68, 0xe1000\n\t" \
  "buffer_load_dwordx4 a[208:211], v4, s[12:15], s50 offen offset:0 sc0 sc1\n\t" \
  "v_mfma_f32_16x16x128_f8f6f4 v[216:219], v[104:111], a[112:119], v[216:219]\n\t" \
  "ds_read_b64_tr_b8 v[122:123], v95 offset:25600\n\t" \
  "v_mfma_f32_16x16x128_f8f6f4 v[212:215], v[104:111], a[104:111], v[212:215]\n\t" \
  "ds_read_b64_tr_b8 v[124:125], v95 offset:57344\n\t" \
  "s_add_u32 s51, s68, 0xe1000\n\t" \
  "buffer_load_dwordx4 a[212:215], v4, s[12:15], s51 offen offset:1024 sc0 sc1\n\t" \
  "v_mfma_f32_16x16x128_f8f6f4 v[208:211], v[104:111], a[96:103], v[208:211]\n\t" \
  "ds_read_b64_tr_b8 v[126:127], v95 offset:58368\n\t" \
  "s_waitcnt vmcnt(26) lgkmcnt(0)\n\t" \
  "s_barrier\n\t" \
  "v_mfma_f32_16x16x128_f8f6f4 v[224:227], v[112:119], a[96:103], v[224:227]\n\t" \
  "ds_read_b64_tr_b8 v[104:105], v80 offset:0\n\t" \
  "s_add_u32 s52, s68, 0xe1000\n\t" \
  "buffer_load_dwordx4 a[216:219], v4, s[12:15], s52 offen offset:2048 sc0 sc1\n\t" \
  "v_mfma_f32_16x16x128_f8f6f4 v[228:231], v[112:119], a[104:111], v[228:231]\n\t" \
  "ds_read_b64_tr_b8 v[106:107], v80 offset:1024\n\t" \
  "v_mfma_f32_16x16x128_f8f6f4 v[232:235], v[112:119], a[112:119], v[232:235]\n\t" \
  "ds_read_b64_tr_b8 v[108:109], v96 offset:0\n\t" \
  "s_add_u32 s53, s68, 0xe1000\n\t" \
  "buffer_load_dwordx4 a[220:223], v4, s[12:15], s53 offen offset:3072 sc0 sc1\n\t" \
  "v_mfma_f32_16x16x128_f8f6f4 v[236:239], v[112:119], a[120:127], v[236:239]\n\t" \
  "ds_read_b64_tr_b8 v[110:111], v96 offset:1024\n\t" \
  "v_mfma_f32_16x16x128_f8f6f4 v[252:255], v[120:127], a[120:127], v[252:255]\n\t" \
  "ds_read_b64_tr_b8 v[112:113], v81 offset:0\n\t" \
  "s_add_u32 m0, s24, 0x26000\n\t" \
  "s_add_u32 s54, s44, 0x200000\n\t" \
  "buffer_load_dwordx4 v3, s[8:11], s54 offen lds\n\t" \
  "v_mfma_f32_16x16x128_f8f6f4 v[248:251], v[120:127], a[112:119], v[248:251]\n\t" \
  "ds_read_b64_tr_b8 v[114:115], v81 offset:1024\n\t" \
  "v_mfma_f32_16x16x128_f8f6f4 v[244:247], v[120:127], a[104:111], v[244:247]\n\t" \
  "ds_read_b64_tr_b8 v[116:117], v97 offset:0\n\t" \
  "s_add_u32 m0, s24, 0x26400\n\t" \
  "s_add_u32 s55, s44, 0x220000\n\t" \
  "buffer_load_dwordx4 v3, s[8:11], s55 offen lds\n\t" \
  "v_mfma_f32_16x16x128_f8f6f4 v[240:243], v[120:127], a[96:103], v[240:243]\n\t" \
  "ds_read_b64_tr_b8 v[118:119], v97 offset:1024\n\t" \
  "s_waitcnt vmcnt(22) lgkmcnt(4)\n\t" \
  "v_mfma_f32_16x16x128_f8f6f4 v[128:131], v[104:111], a[128:135], v[128:131]\n\t" \
  "ds_read_b64_tr_b8 v[120:121], v82 offset:0\n\t" \
  "s_add_u32 s48, s68, 0xf0000\n\t" \
  "buffer_load_dwordx4 a[224:227], v4, s[12:15], s48 offen offset:0 sc0 sc1\n\t" \
  "v_mfma_f32_16x16x128_f8f6f4 v[132:135], v[104:111], a[136:143], v[132:135]\n\t" \
  "ds_read_b64_tr_b8 v[122:123], v82 offset:1024\n\t" \
  "v_mfma_f32_16x16x128_f8f6f4 v[136:139], v[104:111], a[144:151], v[136:139]\n\t" \
  "ds_read_b64_tr_b8 v[124:125], v98 offset:0\n\t" \
  "s_add_u32 s49, s68, 0xf0000\n\t" \
  "buffer_load_dwordx4 a[228:231], v4, s[12:15], s49 offen offset:1024 sc0 sc1\n\t" \
  "v_mfma_f32_16x16x128_f8f6f4 v[140:143], v[104:111], a[152:159], v[140:143]\n\t" \
  "ds_read_b64_tr_b8 v[126:127], v98 offset:1024\n\t" \
  "s_waitcnt lgkmcnt(4)\n\t" \
  "v_mfma_f32_16x16x128_f8f6f4 v[156:159], v[112:119], a[152:159], v[156:159]\n\t" \
  "ds_read_b64_tr_b8 v[104:105], v83 offset:0\n\t" \
  "s_add_u32 s50, s68, 0xf0000\n\t" \
  "buffer_load_dwordx4 a[232:235], v4, s[12:15], s50 offen offset:2048 sc0 sc1\n\t" \
  "v_mfma_f32_16x16x128_f8f6f4 v[152:155], v[112:119], a[144:151], v[152:155]\n\t" \
  "ds_read_b64_tr_b8 v[106:107], v83 offset:1024\n\t" \
  "v_mfma_f32_16x16x128_f8f6f4 v[148:151], v[112:119], a[136:143], v[148:151]\n\t" \
  "ds_read_b64_tr_b8 v[108:109], v99 offset:0\n\t" \
  "s_add_u32 s51, s68, 0xf0000\n\t" \
  "buffer_load_dwordx4 a[236:239], v4, s[12:15], s51 offen offset:3072 sc0 sc1\n\t" \
  "v_mfma_f32_16x16x128_f8f6f4 v[144:147], v[112:119], a[128:135], v[144:147]\n\t" \
  "ds_read_b64_tr_b8 v[110:111], v99 offset:1024\n\t" \
  "s_waitcnt lgkmcnt(4)\n\t" \
  "v_mfma_f32_16x16x128_f8f6f4 v[160:163], v[120:127], a[128:135], v[160:163]\n\t" \
  "ds_read_b64_tr_b8 v[112:113], v84 offset:0\n\t" \
  "s_add_u32 s52, s68, 0xf1000\n\t" \
  "buffer_load_dwordx4 a[240:243], v4, s[12:15], s52 offen offset:0 sc0 sc1\n\t" \
  "v_mfma_f32_16x16x128_f8f6f4 v[164:167], v[120:127], a[136:143], v[164:167]\n\t" \
  "ds_read_b64_tr_b8 v[114:115], v84 offset:1024\n\t" \
  "v_mfma_f32_16x16x128_f8f6f4 v[168:171], v[120:127], a[144:151], v[168:171]\n\t" \
  "ds_read_b64_tr_b8 v[116:117], v100 offset:0\n\t" \
  "s_add_u32 s53, s68, 0xf1000\n\t" \
  "buffer_load_dwordx4 a[244:247], v4, s[12:15], s53 offen offset:1024 sc0 sc1\n\t" \
  "v_mfma_f32_16x16x128_f8f6f4 v[172:175], v[120:127], a[152:159], v[172:175]\n\t" \
  "ds_read_b64_tr_b8 v[118:119], v100 offset:1024\n\t" \
  "s_waitcnt lgkmcnt(4)\n\t" \
  "v_mfma_f32_16x16x128_f8f6f4 v[188:191], v[104:111], a[152:159], v[188:191]\n\t" \
  "ds_read_b64_tr_b8 v[120:121], v85 offset:0\n\t" \
  "s_add_u32 s54, s68, 0xf1000\n\t" \
  "buffer_load_dwordx4 a[248:251], v4, s[12:15], s54 offen offset:2048 sc0 sc1\n\t" \
  "v_mfma_f32_16x16x128_f8f6f4 v[184:187], v[104:111], a[144:151], v[184:187]\n\t" \
  "ds_read_b64_tr_b8 v[122:123], v85 offset:1024\n\t" \
  "v_mfma_f32_16x16x128_f8f6f4 v[180:183], v[104:111], a[136:143], v[180:183]\n\t" \
  "ds_read_b64_tr_b8 v[124:125], v101 offset:0\n\t" \
  "s_add_u32 s55, s68, 0xf1000\n\t" \
  "buffer_load_dwordx4 a[252:255], v4, s[12:15], s55 offen offset:3072 sc0 sc1\n\t" \
  "v_mfma_f32_16x16x128_f8f6f4 v[176:179], v[104:111], a[128:135], v[176:179]\n\t" \
  "ds_read_b64_tr_b8 v[126:127], v101 offset:1024\n\t" \
  "s_waitcnt lgkmcnt(4)\n\t" \
  "v_mfma_f32_16x16x128_f8f6f4 v[192:195], v[112:119], a[128:135], v[192:195]\n\t" \
  "ds_read_b64_tr_b8 v[104:105], v86 offset:0\n\t" \
  "s_cmp_lg_u32 s26, 3\n\t" \
  "s_cbranch_scc1 Lnf3_3_%=\n\t" \
  "s_mov_b64 exec, 1\n\t" \
  "global_store_dword v65, v64, s[30:31] sc0 sc1\n\t" \
  "s_mov_b64 exec, -1\n\t" \
  "s_branch Lmf3_3_%=\n\t" \
  "Lnf3_3_%=:\n\t" \
  "buffer_load_dword v62, v65, s[20:23], s22 offen\n\t" \
  "Lmf3_3_%=:\n\t" \
  "s_add_u32 m0, s24, 0x8000\n\t" \
  "s_add_u32 s48, s45, 0x200000\n\t" \
  "buffer_load_dwordx4 v2, s[8:11], s48 offen lds\n\t" \
  "v_mfma_f32_16x16x128_f8f6f4 v[196:199], v[112:119], a[136:143], v[196:199]\n\t" \
  "ds_read_b64_tr_b8 v[106:107], v86 offset:1024\n\t" \
  "v_mfma_f32_16x16x128_f8f6f4 v[200:203], v[112:119], a[144:151], v[200:203]\n\t" \
  "ds_read_b64_tr_b8 v[108:109], v102 offset:0\n\t" \
  "s_add_u32 m0, s24, 0x8400\n\t" \
  "s_add_u32 s49, s45, 0x220000\n\t" \
  "buffer_load_dwordx4 v2, s[8:11], s49 offen lds\n\t" \
  "v_mfma_f32_16x16x128_f8f6f4 v[204:207], v[112:119], a[152:159], v[204:207]\n\t" \
  "ds_read_b64_tr_b8 v[110:111], v102 offset:1024\n\t" \
  "s_waitcnt lgkmcnt(4)\n\t" \
  "v_mfma_f32_16x16x128_f8f6f4 v[220:223], v[120:127], a[152:159], v[220:223]\n\t" \
  "ds_read_b64_tr_b8 v[112:113], v87 offset:0\n\t" \
  "s_add_u32 m0, s24, 0xa000\n\t" \
  "s_add_u32 s50, s46, 0x200000\n\t" \
  "buffer_load_dwordx4 v2, s[8:11], s50 offen lds\n\t" \
  "v_mfma_f32_16x16x128_f8f6f4 v[216:219], v[120:127], a[144:151], v[216:219]\n\t" \
  "ds_read_b64_tr_b8 v[114:115], v87 offset:1024\n\t" \
  "v_mfma_f32_16x16x128_f8f6f4 v[212:215], v[120:127], a[136:143], v[212:215]\n\t" \
  "ds_read_b64_tr_b8 v[116:117], v103 offset:0\n\t" \
  "s_add_u32 m0, s24, 0xa400\n\t" \
  "s_add_u32 s51, s46, 0x220000\n\t" \
  "buffer_load_dwordx4 v2, s[8:11], s51 offen lds\n\t" \
  "v_mfma_f32_16x16x128_f8f6f4 v[208:211], v[120:127], a[128:135], v[208:211]\n\t" \
  "ds_read_b64_tr_b8 v[118:119], v103 offset:1024\n\t" \
  "s_waitcnt vmcnt(33) lgkmcnt(0)\n\t" \
  "s_barrier\n\t" \
  "v_mfma_f32_16x16x128_f8f6f4 v[224:227], v[104:111], a[128:135], v[224:227]\n\t" \
  "ds_read_b64_tr_b8 v[120:121], v80 offset:8192\n\t" \
  "s_add_u32 m0, s24, 0xc000\n\t" \
  "s_add_u32 s52, s40, 0x400000\n\t" \
  "buffer_load_dwordx4 v2, s[8:11], s52 offen lds\n\t" \
  "v_mfma_f32_16x16x128_f8f6f4 v[228:231], v[104:111], a[136:143], v[228:231]\n\t" \
  "ds_read_b64_tr_b8 v[122:123], v80 offset:9216\n\t" \
  "v_mfma_f32_16x16x128_f8f6f4 v[232:235], v[104:111], a[144:151], v[232:235]\n\t" \
  "ds_read_b64_tr_b8 v[124:125], v96 offset:8192\n\t" \
  "s_add_u32 m0, s24, 0xc400\n\t" \
  "s_add_u32 s53, s40, 0x420000\n\t" \
  "buffer_load_dwordx4 v2, s[8:11], s53 offen lds\n\t" \
  "v_mfma_f32_16x16x128_f8f6f4 v[236:239], v[104:111], a[152:159], v[236:239]\n\t" \
  "ds_read_b64_tr_b8 v[126:127], v96 offset:9216\n\t" \
  "v_mfma_f32_16x16x128_f8f6f4 v[252:255], v[112:119], a[152:159], v[252:255]\n\t" \
  "ds_read_b64_tr_b8 v[104:105], v81 offset:8192\n\t" \
  "s_add_u32 m0, s24, 0xe000\n\t" \
  "s_add_u32 s54, s41, 0x400000\n\t" \
  "buffer_load_dwordx4 v2, s[8:11], s54 offen lds\n\t" \
  "v_mfma_f32_16x16x128_f8f6f4 v[248:251], v[112:119], a[144:151], v[248:251]\n\t" \
  "ds_read_b64_tr_b8 v[106:107], v81 offset:9216\n\t" \
  "v_mfma_f32_16x16x128_f8f6f4 v[244:247], v[112:119], a[136:143], v[244:247]\n\t" \
  "ds_read_b64_tr_b8 v[108:109], v97 offset:8192\n\t" \
  "s_add_u32 m0, s24, 0xe400\n\t" \
  "s_add_u32 s55, s41, 0x420000\n\t" \
  "buffer_load_dwordx4 v2, s[8:11], s55 offen lds\n\t" \
  "v_mfma_f32_16x16x128_f8f6f4 v[240:243], v[112:119], a[128:135], v[240:243]\n\t" \
  "ds_read_b64_tr_b8 v[110:111], v97 offset:9216\n\t" \
  "s_waitcnt vmcnt(29) lgkmcnt(4)\n\t" \
  "v_mfma_f32_16x16x128_f8f6f4 v[128:131], v[120:127], a[160:167], v[128:131]\n\t" \
  "ds_read_b64_tr_b8 v[112:113], v82 offset:8192\n\t" \
  "s_mov_b32 exec_lo, 0x7f0000\n\t" \
  "s_mov_b32 exec_hi, 0\n\t" \
  "global_load_dwordx4 v[68:71], v10, s[6:7] sc0 sc1\n\t" \
  "s_mov_b64 exec, -1\n\t" \
  "v_mfma_f32_16x16x128_f8f6f4 v[132:135], v[120:127], a[168:175], v[132:135]\n\t" \
  "ds_read_b64_tr_b8 v[114:115], v82 offset:9216\n\t" \
  "v_mfma_f32_16x16x128_f8f6f4 v[136:139], v[120:127], a[176:183], v[136:139]\n\t" \
  "ds_read_b64_tr_b8 v[116:117], v98 offset:8192\n\t" \
  "v_mfma_f32_16x16x128_f8f6f4 v[140:143], v[120:127], a[184:191], v[140:143]\n\t" \
  "ds_read_b64_tr_b8 v[118:119], v98 offset:9216\n\t" \
  "s_waitcnt lgkmcnt(4)\n\t" \
  "v_mfma_f32_16x16x128_f8f6f4 v[156:159], v[104:111], a[184:191], v[156:159]\n\t" \
  "ds_read_b64_tr_b8 v[120:121], v83 offset:8192\n\t" \
  "v_mfma_f32_16x16x128_f8f6f4 v[152:155], v[104:111], a[176:183], v[152:155]\n\t" \
  "ds_read_b64_tr_b8 v[122:123], v83 offset:9216\n\t" \
  "v_mfma_f32_16x16x128_f8f6f4 v[148:151], v[104:111], a[168:175], v[148:151]\n\t" \
  "ds_read_b64_tr_b8 v[124:125], v99 offset:8192\n\t" \
  "v_mfma_f32_16x16x128_f8f6f4 v[144:147], v[104:111], a[160:167], v[144:147]\n\t" \
  "ds_read_b64_tr_b8 v[126:127], v99 offset:9216\n\t" \
  "s_waitcnt lgkmcnt(4)\n\t" \
  "v_mfma_f32_16x16x128_f8f6f4 v[160:163], v[112:119], a[160:167], v[160:163]\n\t" \
  "ds_read_b64_tr_b8 v[104:105], v84 offset:8192\n\t" \
  "v_mfma_f32_16x16x128_f8f6f4 v[164:167], v[112:119], a[168:175], v[164:167]\n\t" \
  "ds_read_b64_tr_b8 v[106:107], v84 offset:9216\n\t" \
  "v_mfma_f32_16x16x128_f8f6f4 v[168:171], v[112:119], a[176:183], v[168:171]\n\t" \
  "ds_read_b64_tr_b8 v[108:109], v100 offset:8192\n\t" \
  "v_mfma_f32_16x16x128_f8f6f4 v[172:175], v[112:119], a[184:191], v[172:175]\n\t" \
  "ds_read_b64_tr_b8 v[110:111], v100 offset:9216\n\t" \
  "s_waitcnt lgkmcnt(4)\n\t" \
  "v_mfma_f32_16x16x128_f8f6f4 v[188:191], v[120:127], a[184:191], v[188:191]\n\t" \
  "ds_read_b64_tr_b8 v[112:113], v85 offset:8192\n\t" \
  "v_mfma_f32_16x16x128_f8f6f4 v[184:187], v[120:127], a[176:183], v[184:187]\n\t" \
  "ds_read_b64_tr_b8 v[114:115], v85 offset:9216\n\t" \
  "v_mfma_f32_16x16x128_f8f6f4 v[180:183], v[120:127], a[168:175], v[180:183]\n\t" \
  "ds_read_b64_tr_b8 v[116:117], v101 offset:8192\n\t" \
  "s_add_u32 m0, s24, 0x18000\n\t" \
  "s_add_u32 s48, s45, 0x200000\n\t" \
  "buffer_load_dwordx4 v3, s[8:11], s48 offen lds\n\t" \
  "v_mfma_f32_16x16x128_f8f6f4 v[176:179], v[120:127], a[160:167], v[176:179]\n\t" \
  "ds_read_b64_tr_b8 v[118:119], v101 offset:9216\n\t" \
  "s_waitcnt lgkmcnt(4)\n\t" \
  "v_mfma_f32_16x16x128_f8f6f4 v[192:195], v[104:111], a[160:167], v[192:195]\n\t" \
  "ds_read_b64_tr_b8 v[120:121], v86 offset:8192\n\t" \
  "s_add_u32 m0, s24, 0x18400\n\t" \
  "s_add_u32 s49, s45, 0x220000\n\t" \
  "buffer_load_dwordx4 v3, s[8:11], s49 offen lds\n\t" \
  "v_mfma_f32_16x16x128_f8f6f4 v[196:199], v[104:111], a[168:175], v[196:199]\n\t" \
  "ds_read_b64_tr_b8 v[122:123], v86 offset:9216\n\t" \
  "v_mfma_f32_16x16x128_f8f6f4 v[200:203], v[104:111], a[176:183], v[200:203]\n\t" \
  "ds_read_b64_tr_b8 v[124:125], v102 offset:8192\n\t" \
  "s_add_u32 s50, s68, 0x100000\n\t" \
  "buffer_load_dwordx4 a[0:3], v4, s[12:15], s50 offen offset:0 sc0 sc1\n\t" \
  "v_mfma_f32_16x16x128_f8f6f4 v[204:207], v[104:111], a[184:191], v[204:207]\n\t" \
  "ds_read_b64_tr_b8 v[126:127], v102 offset:9216\n\t" \
  "s_waitcnt lgkmcnt(4)\n\t" \
  "v_mfma_f32_16x16x128_f8f6f4 v[220:223], v[112:119], a[184:191], v[220:223]\n\t" \
  "ds_read_b64_tr_b8 v[104:105], v87 offset:8192\n\t" \
  "s_add_u32 s51, s68, 0x100000\n\t" \
  "buffer_load_dwordx4 a[4:7], v4, s[12:15], s51 offen offset:1024 sc0 sc1\n\t" \
  "v_mfma_f32_16x16x128_f8f6f4 v[216:219], v[112:119], a[176:183], v[216:219]\n\t" \
  "ds_read_b64_tr_b8 v[106:107], v87 offset:9216\n\t" \
  "v_mfma_f32_16x16x128_f8f6f4 v[212:215], v[112:119], a[168:175], v[212:215]\n\t" \
  "ds_read_b64_tr_b8 v[108:109], v103 offset:8192\n\t" \
  "s_add_u32 s52, s68, 0x100000\n\t" \
  "buffer_load_dwordx4 a[8:11], v4, s[12:15], s52 offen offset:2048 sc0 sc1\n\t" \
  "v_mfma_f32_16x16x128_f8f6f4 v[208:211], v[112:119], a[160:167], v[208:211]\n\t" \
  "ds_read_b64_tr_b8 v[110:111], v103 offset:9216\n\t" \
  "s_waitcnt vmcnt(33) lgkmcnt(0)\n\t" \
  "s_barrier\n\t" \
  "v_mfma_f32_16x16x128_f8f6f4 v[224:227], v[120:127], a[160:167], v[224:227]\n\t" \
  "ds_read_b64_tr_b8 v[112:113], v80 offset:16384\n\t" \
  "s_add_u32 s53, s68, 0x100000\n\t" \
  "buffer_load_dwordx4 a[12:15], v4, s[12:15], s53 offen offset:3072 sc0 sc1\n\t" \
  "v_mfma_f32_16x16x128_f8f6f4 v[228:231], v[120:127], a[168:175], v[228:231]\n\t" \
  "ds_read_b64_tr_b8 v[114:115], v80 offset:17408\n\t" \
  "v_mfma_f32_16x16x128_f8f6f4 v[232:235], v[120:127], a[176:183], v[232:235]\n\t" \
  "ds_read_b64_tr_b8 v[116:117], v96 offset:16384\n\t" \
  "s_add_u32 s54, s68, 0x101000\n\t" \
  "buffer_load_dwordx4 a[16:19], v4, s[12:15], s54 offen offset:0 sc0 sc1\n\t" \
  "v_mfma_f32_16x16x128_f8f6f4 v[236:239], v[120:127], a[184:191], v[236:239]\n\t" \
  "ds_read_b64_tr_b8 v[118:119], v96 offset:17408\n\t" \
  "v_mfma_f32_16x16x128_f8f6f4 v[252:255], v[104:111], a[184:191], v[252:255]\n\t" \
  "ds_read_b64_tr_b8 v[120:121], v81 offset:16384\n\t" \
  "s_add_u32 s55, s68, 0x101000\n\t" \
  "buffer_load_dwordx4 a[20:23], v4, s[12:15], s55 offen offset:1024 sc0 sc1\n\t" \
  "v_mfma_f32_16x16x128_f8f6f4 v[248:251], v[104:111], a[176:183], v[248:251]\n\t" \
  "ds_read_b64_tr_b8 v[122:123], v81 offset:17408\n\t" \
  "v_mfma_f32_16x16x128_f8f6f4 v[244:247], v[104:111], a[168:175], v[244:247]\n\t" \
  "ds_read_b64_tr_b8 v[124:125], v97 offset:16384\n\t" \
  "s_add_u32 s48, s68, 0x101000\n\t" \
  "buffer_load_dwordx4 a[24:27], v4, s[12:15], s48 offen offset:2048 sc0 sc1\n\t" \
  "v_mfma_f32_16x16x128_f8f6f4 v[240:243], v[104:111], a[160:167], v[240:243]\n\t" \
  "ds_read_b64_tr_b8 v[126:127], v97 offset:17408\n\t" \
  "s_waitcnt vmcnt(29) lgkmcnt(4)\n\t" \
  "v_mfma_f32_16x16x128_f8f6f4 v[128:131], v[112:119], a[192:199], v[128:131]\n\t" \
  "ds_read_b64_tr_b8 v[104:105], v82 offset:16384\n\t" \
  "s_add_u32 s49, s68, 0x101000\n\t" \
  "buffer_load_dwordx4 a[28:31], v4, s[12:15], s49 offen offset:3072 sc0 sc1\n\t" \
  "v_mfma_f32_16x16x128_f8f6f4 v[132:135], v[112:119], a[200:207], v[132:135]\n\t" \
  "ds_read_b64_tr_b8 v[106:107], v82 offset:17408\n\t" \
  "v_mfma_f32_16x16x128_f8f6f4 v[136:139], v[112:119], a[208:215], v[136:139]\n\t" \
  "ds_read_b64_tr_b8 v[108:109], v98 offset:16384\n\t" \
  "s_add_u32 m0, s24, 0x1a000\n\t" \
  "s_add_u32 s50, s46, 0x200000\n\t" \
  "buffer_load_dwordx4 v3, s[8:11], s50 offen lds\n\t" \
  "v_mfma_f32_16x16x128_f8f6f4 v[140:143], v[112:119], a[216:223], v[140:143]\n\t" \
  "ds_read_b64_tr_b8 v[110:111], v98 offset:17408\n\t" \
  "s_waitcnt lgkmcnt(4)\n\t" \
  "v_mfma_f32_16x16x128_f8f6f4 v[156:159], v[120:127], a[216:223], v[156:159]\n\t" \
  "ds_read_b64_tr_b8 v[112:113], v83 offset:16384\n\t" \
  "s_add_u32 m0, s24, 0x1a400\n\t" \
  "s_add_u32 s51, s46, 0x220000\n\t" \
  "buffer_load_dwordx4 v3, s[8:11], s51 offen lds\n\t" \
  "v_mfma_f32_16x16x128_f8f6f4 v[152:155], v[120:127], a[208:215], v[152:155]\n\t" \
  "ds_read_b64_tr_b8 v[114:115], v83 offset:17408\n\t" \
  "v_mfma_f32_16x16x128_f8f6f4 v[148:151], v[120:127], a[200:207], v[148:151]\n\t" \
  "ds_read_b64_tr_b8 v[116:117], v99 offset:16384\n\t" \
  "s_add_u32 s52, s68, 0x110000\n\t" \
  "buffer_load_dwordx4 a[32:35], v4, s[12:15], s52 offen offset:0 sc0 sc1\n\t" \
  "v_mfma_f32_16x16x128_f8f6f4 v[144:147], v[120:127], a[192:199], v[144:147]\n\t" \
  "ds_read_b64_tr_b8 v[118:119], v99 offset:17408\n\t" \
  "s_waitcnt lgkmcnt(4)\n\t" \
  "v_mfma_f32_16x16x128_f8f6f4 v[160:163], v[104:111], a[192:199], v[160:163]\n\t" \
  "ds_read_b64_tr_b8 v[120:121], v84 offset:16384\n\t" \
  "s_add_u32 s53, s68, 0x110000\n\t" \
  "buffer_load_dwordx4 a[36:39], v4, s[12:15], s53 offen offset:1024 sc0 sc1\n\t" \
  "v_mfma_f32_16x16x128_f8f6f4 v[164:167], v[104:111], a[200:207], v[164:167]\n\t" \
  "ds_read_b64_tr_b8 v[122:123], v84 offset:17408\n\t" \
  "v_mfma_f32_16x16x128_f8f6f4 v[168:171], v[104:111], a[208:215], v[168:171]\n\t" \
  "ds_read_b64_tr_b8 v[124:125], v100 offset:16384\n\t" \
  "s_add_u32 s54, s68, 0x110000\n\t" \
  "buffer_load_dwordx4 a[40:43], v4, s[12:15], s54 offen offset:2048 sc0 sc1\n\t" \
  "v_mfma_f32_16x16x128_f8f6f4 v[172:175], v[104:111], a[216:223], v[172:175]\n\t" \
  "ds_read_b64_tr_b8 v[126:127], v100 offset:17408\n\t" \
  "s_waitcnt lgkmcnt(4)\n\t" \
  "v_mfma_f32_16x16x128_f8f6f4 v[188:191], v[112:119], a[216:223], v[188:191]\n\t" \
  "ds_read_b64_tr_b8 v[104:105], v85 offset:16384\n\t" \
  "s_add_u32 s55, s68, 0x110000\n\t" \
  "buffer_load_dwordx4 a[44:47], v4, s[12:15], s55 offen offset:3072 sc0 sc1\n\t" \
  "v_mfma_f32_16x16x128_f8f6f4 v[184:187], v[112:119], a[208:215], v[184:187]\n\t" \
  "ds_read_b64_tr_b8 v[106:107], v85 offset:17408\n\t" \
  "v_mfma_f32_16x16x128_f8f6f4 v[180:183], v[112:119], a[200:207], v[180:183]\n\t" \
  "ds_read_b64_tr_b8 v[108:109], v101 offset:16384\n\t" \
  "s_add_u32 s48, s68, 0x111000\n\t" \
  "buffer_load_dwordx4 a[48:51], v4, s[12:15], s48 offen offset:0 sc0 sc1\n\t" \
  "v_mfma_f32_16x16x128_f8f6f4 v[176:179], v[112:119], a[192:199], v[176:179]\n\t" \
  "ds_read_b64_tr_b8 v[110:111], v101 offset:17408\n\t" \
  "s_waitcnt lgkmcnt(4)\n\t" \
  "v_mfma_f32_16x16x128_f8f6f4 v[192:195], v[120:127], a[192:199], v[192:195]\n\t" \
  "ds_read_b64_tr_b8 v[112:113], v86 offset:16384\n\t" \
  "s_add_u32 s49, s68, 0x111000\n\t" \
  "buffer_load_dwordx4 a[52:55], v4, s[12:15], s49 offen offset:1024 sc0 sc1\n\t" \
  "v_mfma_f32_16x16x128_f8f6f4 v[196:199], v[120:127], a[200:207], v[196:199]\n\t" \
  "ds_read_b64_tr_b8 v[114:115], v86 offset:17408\n\t" \
  "v_mfma_f32_16x16x128_f8f6f4 v[200:203], v[120:127], a[208:215], v[200:203]\n\t" \
  "ds_read_b64_tr_b8 v[116:117], v102 offset:16384\n\t" \
  "s_add_u32 s50, s68, 0x111000\n\t" \
  "buffer_load_dwordx4 a[56:59], v4, s[12:15], s50 offen offset:2048 sc0 sc1\n\t" \
  "v_mfma_f32_16x16x128_f8f6f4 v[204:207], v[120:127], a[216:223], v[204:207]\n\t" \
  "ds_read_b64_tr_b8 v[118:119], v102 offset:17408\n\t" \
  "s_waitcnt lgkmcnt(4)\n\t" \
  "v_mfma_f32_16x16x128_f8f6f4 v[220:223], v[104:111], a[216:223], v[220:223]\n\t" \
  "ds_read_b64_tr_b8 v[120:121], v87 offset:16384\n\t" \
  "s_add_u32 s51, s68, 0x111000\n\t" \
  "buffer_load_dwordx4 a[60:63], v4, s[12:15], s51 offen offset:3072 sc0 sc1\n\t" \
  "v_mfma_f32_16x16x128_f8f6f4 v[216:219], v[104:111], a[208:215], v[216:219]\n\t" \
  "ds_read_b64_tr_b8 v[122:123], v87 offset:17408\n\t" \
  "v_mfma_f32_16x16x128_f8f6f4 v[212:215], v[104:111], a[200:207], v[212:215]\n\t" \
  "ds_read_b64_tr_b8 v[124:125], v103 offset:16384\n\t" \
  "s_add_u32 m0, s24, 0x1c000\n\t" \
  "s_add_u32 s52, s40, 0x400000\n\t" \
  "buffer_load_dwordx4 v3, s[8:11], s52 offen lds\n\t" \
  "v_mfma_f32_16x16x128_f8f6f4 v[208:211], v[104:111], a[192:199], v[208:211]\n\t" \
  "ds_read_b64_tr_b8 v[126:127], v103 offset:17408\n\t" \
  "s_waitcnt vmcnt(39) lgkmcnt(0)\n\t" \
  "s_barrier\n\t" \
  "v_mfma_f32_16x16x128_f8f6f4 v[224:227], v[112:119], a[192:199], v[224:227]\n\t" \
  "ds_read_b64_tr_b8 v[104:105], v80 offset:24576\n\t" \
  "s_add_u32 m0, s24, 0x1c400\n\t" \
  "s_add_u32 s53, s40, 0x420000\n\t" \
  "buffer_load_dwordx4 v3, s[8:11], s53 offen lds\n\t" \
  "v_mfma_f32_16x16x128_f8f6f4 v[228:231], v[112:119], a[200:207], v[228:231]\n\t" \
  "ds_read_b64_tr_b8 v[106:107], v80 offset:25600\n\t" \
  "v_mfma_f32_16x16x128_f8f6f4 v[232:235], v[112:119], a[208:215], v[232:235]\n\t" \
  "ds_read_b64_tr_b8 v[108:109], v96 offset:24576\n\t" \
  "s_waitcnt vmcnt(22)\n\t" \
  "v_min_u32 v68, v68, v69\n\t" \
  "v_min3_u32 v68, v68, v70, v71\n\t" \
  "v_cmp_le_u32 vcc, s57, v68\n\t" \
  "s_and_b32 s64, vcc_lo, 0x7f0000\n\t" \
  "s_or_b32 s56, s56, s64\n\t" \
  "s_and_b32 s64, s56, 0x7f0000\n\t" \
  "s_cmp_eq_u32 s64, 0x7f0000\n\t" \
  "s_cbranch_scc1 Lok8_3_%=\n\t" \
  "s_mov_b32 s62, 0x7f0000\n\t" \
  "s_mov_b32 s69, 0x7f0000\n\t" \
  "s_call_b64 s[60:61], Lpoll_3_%=\n\t" \
  "Lok8_3_%=:\n\t" \
  "s_add_u32 s54, s68, 0x120000\n\t" \
  "buffer_load_dwordx4 a[64:67], v4, s[12:15], s54 offen offset:0 sc0 sc1\n\t" \
  "v_mfma_f32_16x16x128_f8f6f4 v[236:239], v[112:119], a[216:223], v[236:239]\n\t" \
  "ds_read_b64_tr_b8 v[110:111], v96 offset:25600\n\t" \
  "v_mfma_f32_16x16x128_f8f6f4 v[252:255], v[120:127], a[216:223], v[252:255]\n\t" \
  "ds_read_b64_tr_b8 v[112:113], v81 offset:24576\n\t" \
  "s_add_u32 s55, s68, 0x120000\n\t" \
  "buffer_load_dwordx4 a[68:71], v4, s[12:15], s55 offen offset:1024 sc0 sc1\n\t" \
  "v_mfma_f32_16x16x128_f8f6f4 v[248:251], v[120:127], a[208:215], v[248:251]\n\t" \
  "ds_read_b64_tr_b8 v[114:115], v81 offset:25600\n\t" \
  "v_mfma_f32_16x16x128_f8f6f4 v[244:247], v[120:127], a[200:207], v[244:247]\n\t" \
  "ds_read_b64_tr_b8 v[116:117], v97 offset:24576\n\t" \
  "s_add_u32 s48, s68, 0x120000\n\t" \
  "buffer_load_dwordx4 a[72:75], v4, s[12:15], s48 offen offset:2048 sc0 sc1\n\t" \
  "v_mfma_f32_16x16x128_f8f6f4 v[240:243], v[120:127], a[192:199], v[240:243]\n\t" \
  "ds_read_b64_tr_b8 v[118:119], v97 offset:25600\n\t" \
  "s_waitcnt lgkmcnt(4)\n\t" \
  "v_mfma_f32_16x16x128_f8f6f4 v[128:131], v[104:111], a[224:231], v[128:131]\n\t" \
  "ds_read_b64_tr_b8 v[120:121], v82 offset:24576\n\t" \
  "s_add_u32 s49, s68, 0x120000\n\t" \
  "buffer_load_dwordx4 a[76:79], v4, s[12:15], s49 offen offset:3072 sc0 sc1\n\t" \
  "v_mfma_f32_16x16x128_f8f6f4 v[132:135], v[104:111], a[232:239], v[132:135]\n\t" \
  "ds_read_b64_tr_b8 v[122:123], v82 offset:25600\n\t" \
  "v_mfma_f32_16x16x128_f8f6f4 v[136:139], v[104:111], a[240:247], v[136:139]\n\t" \
  "ds_read_b64_tr_b8 v[124:125], v98 offset:24576\n\t" \
  "s_add_u32 s50, s68, 0x121000\n\t" \
  "buffer_load_dwordx4 a[80:83], v4, s[12:15], s50 offen offset:0 sc0 sc1\n\t" \
  "v_mfma_f32_16x16x128_f8f6f4 v[140:143], v[104:111], a[248:255], v[140:143]\n\t" \
  "ds_read_b64_tr_b8 v[126:127], v98 offset:25600\n\t" \
  "s_waitcnt lgkmcnt(4)\n\t" \
  "v_mfma_f32_16x16x128_f8f6f4 v[156:159], v[112:119], a[248:255], v[156:159]\n\t" \
  "ds_read_b64_tr_b8 v[104:105], v83 offset:24576\n\t" \
  "s_add_u32 s51, s68, 0x121000\n\t" \
  "buffer_load_dwordx4 a[84:87], v4, s[12:15], s51 offen offset:1024 sc0 sc1\n\t" \
  "v_mfma_f32_16x16x128_f8f6f4 v[152:155], v[112:119], a[240:247], v[152:155]\n\t" \
  "ds_read_b64_tr_b8 v[106:107], v83 offset:25600\n\t" \
  "v_mfma_f32_16x16x128_f8f6f4 v[148:151], v[112:119], a[232:239], v[148:151]\n\t" \
  "ds_read_b64_tr_b8 v[108:109], v99 offset:24576\n\t" \
  "s_add_u32 s52, s68, 0x121000\n\t" \
  "buffer_load_dwordx4 a[88:91], v4, s[12:15], s52 offen offset:2048 sc0 sc1\n\t" \
  "v_mfma_f32_16x16x128_f8f6f4 v[144:147], v[112:119], a[224:231], v[144:147]\n\t" \
  "ds_read_b64_tr_b8 v[110:111], v99 offset:25600\n\t" \
  "s_waitcnt lgkmcnt(4)\n\t" \
  "v_mfma_f32_16x16x128_f8f6f4 v[160:163], v[120:127], a[224:231], v[160:163]\n\t" \
  "ds_read_b64_tr_b8 v[112:113], v84 offset:24576\n\t" \
  "s_add_u32 s53, s68, 0x121000\n\t" \
  "buffer_load_dwordx4 a[92:95], v4, s[12:15], s53 offen offset:3072 sc0 sc1\n\t" \
  "v_mfma_f32_16x16x128_f8f6f4 v[164:167], v[120:127], a[232:239], v[164:167]\n\t" \
  "ds_read_b64_tr_b8 v[114:115], v84 offset:25600\n\t" \
  "v_mfma_f32_16x16x128_f8f6f4 v[168:171], v[120:127], a[240:247], v[168:171]\n\t" \
  "ds_read_b64_tr_b8 v[116:117], v100 offset:24576\n\t" \
  "s_add_u32 m0, s24, 0x10000\n\t" \
  "s_add_u32 s54, s42, 0x400000\n\t" \
  "buffer_load_dwordx4 v2, s[8:11], s54 offen lds\n\t" \
  "v_mfma_f32_16x16x128_f8f6f4 v[172:175], v[120:127], a[248:255], v[172:175]\n\t" \
  "ds_read_b64_tr_b8 v[118:119], v100 offset:25600\n\t" \
  "s_waitcnt lgkmcnt(4)\n\t" \
  "v_mfma_f32_16x16x128_f8f6f4 v[188:191], v[104:111], a[248:255], v[188:191]\n\t" \
  "ds_read_b64_tr_b8 v[120:121], v85 offset:24576\n\t" \
  "s_add_u32 m0, s24, 0x10400\n\t" \
  "s_add_u32 s55, s42, 0x420000\n\t" \
  "buffer_load_dwordx4 v2, s[8:11], s55 offen lds\n\t" \
  "v_mfma_f32_16x16x128_f8f6f4 v[184:187], v[104:111], a[240:247], v[184:187]\n\t" \
  "ds_read_b64_tr_b8 v[122:123], v85 offset:25600\n\t" \
  "v_mfma_f32_16x16x128_f8f6f4 v[180:183], v[104:111], a[232:239], v[180:183]\n\t" \
  "ds_read_b64_tr_b8 v[124:125], v101 offset:24576\n\t" \
  "s_add_u32 m0, s24, 0x12000\n\t" \
  "s_add_u32 s48, s43, 0x400000\n\t" \
  "buffer_load_dwordx4 v2, s[8:11], s48 offen lds\n\t" \
  "v_mfma_f32_16x16x128_f8f6f4 v[176:179], v[104:111], a[224:231], v[176:179]\n\t" \
  "ds_read_b64_tr_b8 v[126:127], v101 offset:25600\n\t" \
  "s_waitcnt lgkmcnt(4)\n\t" \
  "v_mfma_f32_16x16x128_f8f6f4 v[192:195], v[112:119], a[224:231], v[192:195]\n\t" \
  "ds_read_b64_tr_b8 v[104:105], v86 offset:24576\n\t" \
  "s_add_u32 m0, s24, 0x12400\n\t" \
  "s_add_u32 s49, s43, 0x420000\n\t" \
  "buffer_load_dwordx4 v2, s[8:11], s49 offen lds\n\t" \
  "v_mfma_f32_16x16x128_f8f6f4 v[196:199], v[112:119], a[232:239], v[196:199]\n\t" \
  "ds_read_b64_tr_b8 v[106:107], v86 offset:25600\n\t" \
  "v_mfma_f32_16x16x128_f8f6f4 v[200:203], v[112:119], a[240:247], v[200:203]\n\t" \
  "ds_read_b64_tr_b8 v[108:109], v102 offset:24576\n\t" \
  "s_add_u32 m0, s24, 0x14000\n\t" \
  "s_add_u32 s50, s44, 0x400000\n\t" \
  "buffer_load_dwordx4 v2, s[8:11], s50 offen lds\n\t" \
  "v_mfma_f32_16x16x128_f8f6f4 v[204:207], v[112:119], a[248:255], v[204:207]\n\t" \
  "ds_read_b64_tr_b8 v[110:111], v102 offset:25600\n\t" \
  "s_waitcnt lgkmcnt(4)\n\t" \
  "v_mfma_f32_16x16x128_f8f6f4 v[220:223], v[120:127], a[248:255], v[220:223]\n\t" \
  "ds_read_b64_tr_b8 v[112:113], v87 offset:24576\n\t" \
  "s_add_u32 m0, s24, 0x14400\n\t" \
  "s_add_u32 s51, s44, 0x420000\n\t" \
  "buffer_load_dwordx4 v2, s[8:11], s51 offen lds\n\t" \
  "v_mfma_f32_16x16x128_f8f6f4 v[216:219], v[120:127], a[240:247], v[216:219]\n\t" \
  "ds_read_b64_tr_b8 v[114:115], v87 offset:25600\n\t" \
  "v_mfma_f32_16x16x128_f8f6f4 v[212:215], v[120:127], a[232:239], v[212:215]\n\t" \
  "ds_read_b64_tr_b8 v[116:117], v103 offset:24576\n\t" \
  "s_add_u32 m0, s24, 0x16000\n\t" \
  "s_add_u32 s52, s45, 0x400000\n\t" \
  "buffer_load_dwordx4 v2, s[8:11], s52 offen lds\n\t" \
  "v_mfma_f32_16x16x128_f8f6f4 v[208:211], v[120:127], a[224:231], v[208:211]\n\t" \
  "ds_read_b64_tr_b8 v[118:119], v103 offset:25600\n\t" \
  "s_waitcnt vmcnt(35) lgkmcnt(0)\n\t" \
  "s_barrier\n\t" \
  "v_mfma_f32_16x16x128_f8f6f4 v[224:227], v[104:111], a[224:231], v[224:227]\n\t" \
  "ds_read_b64_tr_b8 v[120:121], v80 offset:32768\n\t" \
  "s_add_u32 m0, s24, 0x16400\n\t" \
  "s_add_u32 s53, s45, 0x420000\n\t" \
  "buffer_load_dwordx4 v2, s[8:11], s53 offen lds\n\t" \
  "v_mfma_f32_16x16x128_f8f6f4 v[228:231], v[104:111], a[232:239], v[228:231]\n\t" \
  "ds_read_b64_tr_b8 v[122:123], v80 offset:33792\n\t" \
  "v_mfma_f32_16x16x128_f8f6f4 v[232:235], v[104:111], a[240:247], v[232:235]\n\t" \
  "ds_read_b64_tr_b8 v[124:125], v88 offset:32768\n\t" \
  "v_mfma_f32_16x16x128_f8f6f4 v[236:239], v[104:111], a[248:255], v[236:239]\n\t" \
  "ds_read_b64_tr_b8 v[126:127], v88 offset:33792\n\t" \
  "v_mfma_f32_16x16x128_f8f6f4 v[252:255], v[112:119], a[248:255], v[252:255]\n\t" \
  "ds_read_b64_tr_b8 v[104:105], v81 offset:32768\n\t" \
  "v_mfma_f32_16x16x128_f8f6f4 v[248:251], v[112:119], a[240:247], v[248:251]\n\t" \
  "ds_read_b64_tr_b8 v[106:107], v81 offset:33792\n\t" \
  "v_mfma_f32_16x16x128_f8f6f4 v[244:247], v[112:119], a[232:239], v[244:247]\n\t" \
  "ds_read_b64_tr_b8 v[108:109], v89 offset:32768\n\t" \
  "v_mfma_f32_16x16x128_f8f6f4 v[240:243], v[112:119], a[224:231], v[240:243]\n\t" \
  "ds_read_b64_tr_b8 v[110:111], v89 offset:33792\n\t" \
  "s_waitcnt vmcnt(28) lgkmcnt(4)\n\t" \
  "v_mfma_f32_16x16x128_f8f6f4 v[128:131], v[120:127], a[0:7], v[128:131]\n\t" \
  "ds_read_b64_tr_b8 v[112:113], v82 offset:32768\n\t" \
  "v_mfma_f32_16x16x128_f8f6f4 v[132:135], v[120:127], a[8:15], v[132:135]\n\t" \
  "ds_read_b64_tr_b8 v[114:115], v82 offset:33792\n\t" \
  "v_mfma_f32_16x16x128_f8f6f4 v[136:139], v[120:127], a[16:23], v[136:139]\n\t" \
  "ds_read_b64_tr_b8 v[116:117], v90 offset:32768\n\t" \
  "v_mfma_f32_16x16x128_f8f6f4 v[140:143], v[120:127], a[24:31], v[140:143]\n\t" \
  "ds_read_b64_tr_b8 v[118:119], v90 offset:33792\n\t" \
  "s_waitcnt lgkmcnt(4)\n\t" \
  "v_mfma_f32_16x16x128_f8f6f4 v[156:159], v[104:111], a[24:31], v[156:159]\n\t" \
  "ds_read_b64_tr_b8 v[120:121], v83 offset:32768\n\t" \
  "v_mfma_f32_16x16x128_f8f6f4 v[152:155], v[104:111], a[16:23], v[152:155]\n\t" \
  "ds_read_b64_tr_b8 v[122:123], v83 offset:33792\n\t" \
  "v_mfma_f32_16x16x128_f8f6f4 v[148:151], v[104:111], a[8:15], v[148:151]\n\t" \
  "ds_read_b64_tr_b8 v[124:125], v91 offset:32768\n\t" \
  "v_mfma_f32_16x16x128_f8f6f4 v[144:147], v[104:111], a[0:7], v[144:147]\n\t" \
  "ds_read_b64_tr_b8 v[126:127], v91 offset:33792\n\t" \
  "s_waitcnt lgkmcnt(4)\n\t" \
  "v_mfma_f32_16x16x128_f8f6f4 v[160:163], v[112:119], a[0:7], v[160:163]\n\t" \
  "ds_read_b64_tr_b8 v[104:105], v84 offset:32768\n\t" \
  "s_add_u32 m0, s24, 0x1e000\n\t" \
  "s_add_u32 s54, s41, 0x400000\n\t" \
  "buffer_load_dwordx4 v3, s[8:11], s54 offen lds\n\t" \
  "v_mfma_f32_16x16x128_f8f6f4 v[164:167], v[112:119], a[8:15], v[164:167]\n\t" \
  "ds_read_b64_tr_b8 v[106:107], v84 offset:33792\n\t" \
  "v_mfma_f32_16x16x128_f8f6f4 v[168:171], v[112:119], a[16:23], v[168:171]\n\t" \
  "ds_read_b64_tr_b8 v[108:109], v92 offset:32768\n\t" \
  "s_add_u32 m0, s24, 0x1e400\n\t" \
  "s_add_u32 s55, s41, 0x420000\n\t" \
  "buffer_load_dwordx4 v3, s[8:11], s55 offen lds\n\t" \
  "v_mfma_f32_16x16x128_f8f6f4 v[172:175], v[112:119], a[24:31], v[172:175]\n\t" \
  "ds_read_b64_tr_b8 v[110:111], v92 offset:33792\n\t" \
  "s_waitcnt lgkmcnt(4)\n\t" \
  "v_mfma_f32_16x16x128_f8f6f4 v[188:191], v[120:127], a[24:31], v[188:191]\n\t" \
  "ds_read_b64_tr_b8 v[112:113], v85 offset:32768\n\t" \
  "s_add_u32 s48, s68, 0x130000\n\t" \
  "buffer_load_dwordx4 a[96:99], v4, s[12:15], s48 offen offset:0 sc0 sc1\n\t" \
  "v_mfma_f32_16x16x128_f8f6f4 v[184:187], v[120:127], a[16:23], v[184:187]\n\t" \
  "ds_read_b64_tr_b8 v[114:115], v85 offset:33792\n\t" \
  "v_mfma_f32_16x16x128_f8f6f4 v[180:183], v[120:127], a[8:15], v[180:183]\n\t" \
  "ds_read_b64_tr_b8 v[116:117], v93 offset:32768\n\t" \
  "s_add_u32 s49, s68, 0x130000\n\t" \
  "buffer_load_dwordx4 a[100:103], v4, s[12:15], s49 offen offset:1024 sc0 sc1\n\t" \
  "v_mfma_f32_16x16x128_f8f6f4 v[176:179], v[120:127], a[0:7], v[176:179]\n\t" \
  "ds_read_b64_tr_b8 v[118:119], v93 offset:33792\n\t" \
  "s_waitcnt lgkmcnt(4)\n\t" \
  "v_mfma_f32_16x16x128_f8f6f4 v[192:195], v[104:111], a[0:7], v[192:195]\n\t" \
  "ds_read_b64_tr_b8 v[120:121], v86 offset:32768\n\t" \
  "s_add_u32 s50, s68, 0x130000\n\t" \
  "buffer_load_dwordx4 a[104:107], v4, s[12:15], s50 offen offset:2048 sc0 sc1\n\t" \
  "v_mfma_f32_16x16x128_f8f6f4 v[196:199], v[104:111], a[8:15], v[196:199]\n\t" \
  "ds_read_b64_tr_b8 v[122:123], v86 offset:33792\n\t" \
  "v_mfma_f32_16x16x128_f8f6f4 v[200:203], v[104:111], a[16:23], v[200:203]\n\t" \
  "ds_read_b64_tr_b8 v[124:125], v94 offset:32768\n\t" \
  "s_add_u32 s51, s68, 0x130000\n\t" \
  "buffer_load_dwordx4 a[108:111], v4, s[12:15], s51 offen offset:3072 sc0 sc1\n\t" \
  "v_mfma_f32_16x16x128_f8f6f4 v[204:207], v[104:111], a[24:31], v[204:207]\n\t" \
  "ds_read_b64_tr_b8 v[126:127], v94 offset:33792\n\t" \
  "s_waitcnt lgkmcnt(4)\n\t" \
  "v_mfma_f32_16x16x128_f8f6f4 v[220:223], v[112:119], a[24:31], v[220:223]\n\t" \
  "ds_read_b64_tr_b8 v[104:105], v87 offset:32768\n\t" \
  "s_add_u32 s52, s68, 0x131000\n\t" \
  "buffer_load_dwordx4 a[112:115], v4, s[12:15], s52 offen offset:0 sc0 sc1\n\t" \
  "v_mfma_f32_16x16x128_f8f6f4 v[216:219], v[112:119], a[16:23], v[216:219]\n\t" \
  "ds_read_b64_tr_b8 v[106:107], v87 offset:33792\n\t" \
  "v_mfma_f32_16x16x128_f8f6f4 v[212:215], v[112:119], a[8:15], v[212:215]\n\t" \
  "ds_read_b64_tr_b8 v[108:109], v95 offset:32768\n\t" \
  "s_add_u32 s53, s68, 0x131000\n\t" \
  "buffer_load_dwordx4 a[116:119], v4, s[12:15], s53 offen offset:1024 sc0 sc1\n\t" \
  "v_mfma_f32_16x16x128_f8f6f4 v[208:211], v[112:119], a[0:7], v[208:211]\n\t" \
  "ds_read_b64_tr_b8 v[110:111], v95 offset:33792\n\t" \
  "s_waitcnt vmcnt(34) lgkmcnt(0)\n\t" \
  "s_barrier\n\t" \
  "v_mfma_f32_16x16x128_f8f6f4 v[224:227], v[120:127], a[0:7], v[224:227]\n\t" \
  "ds_read_b64_tr_b8 v[112:113], v80 offset:40960\n\t" \
  "s_add_u32 s54, s68, 0x131000\n\t" \
  "buffer_load_dwordx4 a[120:123], v4, s[12:15], s54 offen offset:2048 sc0 sc1\n\t" \
  "v_mfma_f32_16x16x128_f8f6f4 v[228:231], v[120:127], a[8:15], v[228:231]\n\t" \
  "ds_read_b64_tr_b8 v[114:115], v80 offset:41984\n\t" \
  "v_mfma_f32_16x16x128_f8f6f4 v[232:235], v[120:127], a[16:23], v[232:235]\n\t" \
  "ds_read_b64_tr_b8 v[116:117], v88 offset:40960\n\t" \
  "s_add_u32 s55, s68, 0x131000\n\t" \
  "buffer_load_dwordx4 a[124:127], v4, s[12:15], s55 offen offset:3072 sc0 sc1\n\t" \
  "v_mfma_f32_16x16x128_f8f6f4 v[236:239], v[120:127], a[24:31], v[236:239]\n\t" \
  "ds_read_b64_tr_b8 v[118:119], v88 offset:41984\n\t" \
  "v_mfma_f32_16x16x128_f8f6f4 v[252:255], v[104:111], a[24:31], v[252:255]\n\t" \
  "ds_read_b64_tr_b8 v[120:121], v81 offset:40960\n\t" \
  "s_add_u32 m0, s24, 0x20000\n\t" \
  "s_add_u32 s48, s42, 0x400000\n\t" \
  "buffer_load_dwordx4 v3, s[8:11], s48 offen lds\n\t" \
  "v_mfma_f32_16x16x128_f8f6f4 v[248:251], v[104:111], a[16:23], v[248:251]\n\t" \
  "ds_read_b64_tr_b8 v[122:123], v81 offset:41984\n\t" \
  "v_mfma_f32_16x16x128_f8f6f4 v[244:247], v[104:111], a[8:15], v[244:247]\n\t" \
  "ds_read_b64_tr_b8 v[124:125], v89 offset:40960\n\t" \
  "s_add_u32 m0, s24, 0x20400\n\t" \
  "s_add_u32 s49, s42, 0x420000\n\t" \
  "buffer_load_dwordx4 v3, s[8:11], s49 offen lds\n\t" \
  "v_mfma_f32_16x16x128_f8f6f4 v[240:243], v[104:111], a[0:7], v[240:243]\n\t" \
  "ds_read_b64_tr_b8 v[126:127], v89 offset:41984\n\t" \
  "s_waitcnt vmcnt(30) lgkmcnt(4)\n\t" \
  "v_mfma_f32_16x16x128_f8f6f4 v[128:131], v[112:119], a[32:39], v[128:131]\n\t" \
  "ds_read_b64_tr_b8 v[104:105], v82 offset:40960\n\t" \
  "s_add_u32 s50, s68, 0x140000\n\t" \
  "buffer_load_dwordx4 a[128:131], v4, s[12:15], s50 offen offset:0 sc0 sc1\n\t" \
  "v_mfma_f32_16x16x128_f8f6f4 v[132:135], v[112:119], a[40:47], v[132:135]\n\t" \
  "ds_read_b64_tr_b8 v[106:107], v82 offset:41984\n\t" \
  "v_mfma_f32_16x16x128_f8f6f4 v[136:139], v[112:119], a[48:55], v[136:139]\n\t" \
  "ds_read_b64_tr_b8 v[108:109], v90 offset:40960\n\t" \
  "s_add_u32 s51, s68, 0x140000\n\t" \
  "buffer_load_dwordx4 a[132:135], v4, s[12:15], s51 offen offset:1024 sc0 sc1\n\t" \
  "v_mfma_f32_16x16x128_f8f6f4 v[140:143], v[112:119], a[56:63], v[140:143]\n\t" \
  "ds_read_b64_tr_b8 v[110:111], v90 offset:41984\n\t" \
  "s_waitcnt lgkmcnt(4)\n\t" \
  "v_mfma_f32_16x16x128_f8f6f4 v[156:159], v[120:127], a[56:63], v[156:159]\n\t" \
  "ds_read_b64_tr_b8 v[112:113], v83 offset:40960\n\t" \
  "s_add_u32 s52, s68, 0x140000\n\t" \
  "buffer_load_dwordx4 a[136:139], v4, s[12:15], s52 offen offset:2048 sc0 sc1\n\t" \
  "v_mfma_f32_16x16x128_f8f6f4 v[152:155], v[120:127], a[48:55], v[152:155]\n\t" \
  "ds_read_b64_tr_b8 v[114:115], v83 offset:41984\n\t" \
  "v_mfma_f32_16x16x128_f8f6f4 v[148:151], v[120:127], a[40:47], v[148:151]\n\t" \
  "ds_read_b64_tr_b8 v[116:117], v91 offset:40960\n\t" \
  "s_add_u32 s53, s68, 0x140000\n\t" \
  "buffer_load_dwordx4 a[140:143], v4, s[12:15], s53 offen offset:3072 sc0 sc1\n\t" \
  "v_mfma_f32_16x16x128_f8f6f4 v[144:147], v[120:127], a[32:39], v[144:147]\n\t" \
  "ds_read_b64_tr_b8 v[118:119], v91 offset:41984\n\t" \
  "s_waitcnt lgkmcnt(4)\n\t" \
  "v_mfma_f32_16x16x128_f8f6f4 v[160:163], v[104:111], a[32:39], v[160:163]\n\t" \
  "ds_read_b64_tr_b8 v[120:121], v84 offset:40960\n\t" \
  "s_add_u32 s54, s68, 0x141000\n\t" \
  "buffer_load_dwordx4 a[144:147], v4, s[12:15], s54 offen offset:0 sc0 sc1\n\t" \
  "v_mfma_f32_16x16x128_f8f6f4 v[164:167], v[104:111], a[40:47], v[164:167]\n\t" \
  "ds_read_b64_tr_b8 v[122:123], v84 offset:41984\n\t" \
  "v_mfma_f32_16x16x128_f8f6f4 v[168:171], v[104:111], a[48:55], v[168:171]\n\t" \
  "ds_read_b64_tr_b8 v[124:125], v92 offset:40960\n\t" \
  "s_add_u32 s55, s68, 0x141000\n\t" \
  "buffer_load_dwordx4 a[148:151], v4, s[12:15], s55 offen offset:1024 sc0 sc1\n\t" \
  "v_mfma_f32_16x16x128_f8f6f4 v[172:175], v[104:111], a[56:63], v[172:175]\n\t" \
  "ds_read_b64_tr_b8 v[126:127], v92 offset:41984\n\t" \
  "s_waitcnt lgkmcnt(4)\n\t" \
  "v_mfma_f32_16x16x128_f8f6f4 v[188:191], v[112:119], a[56:63], v[188:191]\n\t" \
  "ds_read_b64_tr_b8 v[104:105], v85 offset:40960\n\t" \
  "s_add_u32 s48, s68, 0x141000\n\t" \
  "buffer_load_dwordx4 a[152:155], v4, s[12:15], s48 offen offset:2048 sc0 sc1\n\t" \
  "v_mfma_f32_16x16x128_f8f6f4 v[184:187], v[112:119], a[48:55], v[184:187]\n\t" \
  "ds_read_b64_tr_b8 v[106:107], v85 offset:41984\n\t" \
  "v_mfma_f32_16x16x128_f8f6f4 v[180:183], v[112:119], a[40:47], v[180:183]\n\t" \
  "ds_read_b64_tr_b8 v[108:109], v93 offset:40960\n\t" \
  "s_add_u32 s49, s68, 0x141000\n\t" \
  "buffer_load_dwordx4 a[156:159], v4, s[12:15], s49 offen offset:3072 sc0 sc1\n\t" \
  "v_mfma_f32_16x16x128_f8f6f4 v[176:179], v[112:119], a[32:39], v[176:179]\n\t" \
  "ds_read_b64_tr_b8 v[110:111], v93 offset:41984\n\t" \
  "s_waitcnt lgkmcnt(4)\n\t" \
  "v_mfma_f32_16x16x128_f8f6f4 v[192:195], v[120:127], a[32:39], v[192:195]\n\t" \
  "ds_read_b64_tr_b8 v[112:113], v86 offset:40960\n\t" \
  "s_add_u32 m0, s24, 0x22000\n\t" \
  "s_add_u32 s50, s43, 0x400000\n\t" \
  "buffer_load_dwordx4 v3, s[8:11], s50 offen lds\n\t" \
  "v_mfma_f32_16x16x128_f8f6f4 v[196:199], v[120:127], a[40:47], v[196:199]\n\t" \
  "ds_read_b64_tr_b8 v[114:115], v86 offset:41984\n\t" \
  "v_mfma_f32_16x16x128_f8f6f4 v[200:203], v[120:127], a[48:55], v[200:203]\n\t" \
  "ds_read_b64_tr_b8 v[116:117], v94 offset:40960\n\t" \
  "s_add_u32 m0, s24, 0x22400\n\t" \
  "s_add_u32 s51, s43, 0x420000\n\t" \
  "buffer_load_dwordx4 v3, s[8:11], s51 offen lds\n\t" \
  "v_mfma_f32_16x16x128_f8f6f4 v[204:207], v[120:127], a[56:63], v[204:207]\n\t" \
  "ds_read_b64_tr_b8 v[118:119], v94 offset:41984\n\t" \
  "s_waitcnt lgkmcnt(4)\n\t" \
  "v_mfma_f32_16x16x128_f8f6f4 v[220:223], v[104:111], a[56:63], v[220:223]\n\t" \
  "ds_read_b64_tr_b8 v[120:121], v87 offset:40960\n\t" \
  "s_add_u32 s52, s68, 0x150000\n\t" \
  "buffer_load_dwordx4 a[160:163], v4, s[12:15], s52 offen offset:0 sc0 sc1\n\t" \
  "v_mfma_f32_16x16x128_f8f6f4 v[216:219], v[104:111], a[48:55], v[216:219]\n\t" \
  "ds_read_b64_tr_b8 v[122:123], v87 offset:41984\n\t" \
  "v_mfma_f32_16x16x128_f8f6f4 v[212:215], v[104:111], a[40:47], v[212:215]\n\t" \
  "ds_read_b64_tr_b8 v[124:125], v95 offset:40960\n\t" \
  "s_add_u32 s53, s68, 0x150000\n\t" \
  "buffer_load_dwordx4 a[164:167], v4, s[12:15], s53 offen offset:1024 sc0 sc1\n\t" \
  "v_mfma_f32_16x16x128_f8f6f4 v[208:211], v[104:111], a[32:39], v[208:211]\n\t" \
  "ds_read_b64_tr_b8 v[126:127], v95 offset:41984\n\t" \
  "s_waitcnt vmcnt(40) lgkmcnt(0)\n\t" \
  "s_barrier\n\t" \
  "v_mfma_f32_16x16x128_f8f6f4 v[224:227], v[112:119], a[32:39], v[224:227]\n\t" \
  "ds_read_b64_tr_b8 v[104:105], v80 offset:49152\n\t" \
  "s_add_u32 s54, s68, 0x150000\n\t" \
  "buffer_load_dwordx4 a[168:171], v4, s[12:15], s54 offen offset:2048 sc0 sc1\n\t" \
  "v_mfma_f32_16x16x128_f8f6f4 v[228:231], v[112:119], a[40:47], v[228:231]\n\t" \
  "ds_read_b64_tr_b8 v[106:107], v80 offset:50176\n\t" \
  "v_mfma_f32_16x16x128_f8f6f4 v[232:235], v[112:119], a[48:55], v[232:235]\n\t" \
  "ds_read_b64_tr_b8 v[108:109], v88 offset:49152\n\t" \
  "s_add_u32 s55, s68, 0x150000\n\t" \
  "buffer_load_dwordx4 a[172:175], v4, s[12:15], s55 offen offset:3072 sc0 sc1\n\t" \
  "v_mfma_f32_16x16x128_f8f6f4 v[236:239], v[112:119], a[56:63], v[236:239]\n\t" \
  "ds_read_b64_tr_b8 v[110:111], v88 offset:50176\n\t" \
  "v_mfma_f32_16x16x128_f8f6f4 v[252:255], v[120:127], a[56:63], v[252:255]\n\t" \
  "ds_read_b64_tr_b8 v[112:113], v81 offset:49152\n\t" \
  "s_add_u32 s48, s68, 0x151000\n\t" \
  "buffer_load_dwordx4 a[176:179], v4, s[12:15], s48 offen offset:0 sc0 sc1\n\t" \
  "v_mfma_f32_16x16x128_f8f6f4 v[248:251], v[120:127], a[48:55], v[248:251]\n\t" \
  "ds_read_b64_tr_b8 v[114:115], v81 offset:50176\n\t" \
  "v_mfma_f32_16x16x128_f8f6f4 v[244:247], v[120:127], a[40:47], v[244:247]\n\t" \
  "ds_read_b64_tr_b8 v[116:117], v89 offset:49152\n\t" \
  "s_add_u32 s49, s68, 0x151000\n\t" \
  "buffer_load_dwordx4 a[180:183], v4, s[12:15], s49 offen offset:1024 sc0 sc1\n\t" \
  "v_mfma_f32_16x16x128_f8f6f4 v[240:243], v[120:127], a[32:39], v[240:243]\n\t" \
  "ds_read_b64_tr_b8 v[118:119], v89 offset:50176\n\t" \
  "s_waitcnt vmcnt(36) lgkmcnt(4)\n\t" \
  "v_mfma_f32_16x16x128_f8f6f4 v[128:131], v[104:111], a[64:71], v[128:131]\n\t" \
  "ds_read_b64_tr_b8 v[120:121], v82 offset:49152\n\t" \
  "s_add_u32 s50, s68, 0x151000\n\t" \
  "buffer_load_dwordx4 a[184:187], v4, s[12:15], s50 offen offset:2048 sc0 sc1\n\t" \
  "v_mfma_f32_16x16x128_f8f6f4 v[132:135], v[104:111], a[72:79], v[132:135]\n\t" \
  "ds_read_b64_tr_b8 v[122:123], v82 offset:50176\n\t" \
  "v_mfma_f32_16x16x128_f8f6f4 v[136:139], v[104:111], a[80:87], v[136:139]\n\t" \
  "ds_read_b64_tr_b8 v[124:125], v90 offset:49152\n\t" \
  "s_add_u32 s51, s68, 0x151000\n\t" \
  "buffer_load_dwordx4 a[188:191], v4, s[12:15], s51 offen offset:3072 sc0 sc1\n\t" \
  "v_mfma_f32_16x16x128_f8f6f4 v[140:143], v[104:111], a[88:95], v[140:143]\n\t" \
  "ds_read_b64_tr_b8 v[126:127], v90 offset:50176\n\t" \
  "s_waitcnt lgkmcnt(4)\n\t" \
  "v_mfma_f32_16x16x128_f8f6f4 v[156:159], v[112:119], a[88:95], v[156:159]\n\t" \
  "ds_read_b64_tr_b8 v[104:105], v83 offset:49152\n\t" \
  "s_add_u32 m0, s24, 0x24000\n\t" \
  "s_add_u32 s52, s44, 0x400000\n\t" \
  "buffer_load_dwordx4 v3, s[8:11], s52 offen lds\n\t" \
  "v_mfma_f32_16x16x128_f8f6f4 v[152:155], v[112:119], a[80:87], v[152:155]\n\t" \
  "ds_read_b64_tr_b8 v[106:107], v83 offset:50176\n\t" \
  "v_mfma_f32_16x16x128_f8f6f4 v[148:151], v[112:119], a[72:79], v[148:151]\n\t" \
  "ds_read_b64_tr_b8 v[108:109], v91 offset:49152\n\t" \
  "s_add_u32 m0, s24, 0x24400\n\t" \
  "s_add_u32 s53, s44, 0x420000\n\t" \
  "buffer_load_dwordx4 v3, s[8:11], s53 offen lds\n\t" \
  "v_mfma_f32_16x16x128_f8f6f4 v[144:147], v[112:119], a[64:71], v[144:147]\n\t" \
  "ds_read_b64_tr_b8 v[110:111], v91 offset:50176\n\t" \
  "s_waitcnt lgkmcnt(4)\n\t" \
  "v_mfma_f32_16x16x128_f8f6f4 v[160:163], v[120:127], a[64:71], v[160:163]\n\t" \
  "ds_read_b64_tr_b8 v[112:113], v84 offset:49152\n\t" \
  "s_add_u32 s54, s68, 0x160000\n\t" \
  "buffer_load_dwordx4 a[192:195], v4, s[12:15], s54 offen offset:0 sc0 sc1\n\t" \
  "v_mfma_f32_16x16x128_f8f6f4 v[164:167], v[120:127], a[72:79], v[164:167]\n\t" \
  "ds_read_b64_tr_b8 v[114:115], v84 offset:50176\n\t" \
  "v_mfma_f32_16x16x128_f8f6f4 v[168:171], v[120:127], a[80:87], v[168:171]\n\t" \
  "ds_read_b64_tr_b8 v[116:117], v92 offset:49152\n\t" \
  "s_add_u32 s55, s68, 0x160000\n\t" \
  "buffer_load_dwordx4 a[196:199], v4, s[12:15], s55 offen offset:1024 sc0 sc1\n\t" \
  "v_mfma_f32_16x16x128_f8f6f4 v[172:175], v[120:127], a[88:95], v[172:175]\n\t" \
  "ds_read_b64_tr_b8 v[118:119], v92 offset:50176\n\t" \
  "s_waitcnt lgkmcnt(4)\n\t" \
  "v_mfma_f32_16x16x128_f8f6f4 v[188:191], v[104:111], a[88:95], v[188:191]\n\t" \
  "ds_read_b64_tr_b8 v[120:121], v85 offset:49152\n\t" \
  "s_add_u32 s48, s68, 0x160000\n\t" \
  "buffer_load_dwordx4 a[200:203], v4, s[12:15], s48 offen offset:2048 sc0 sc1\n\t" \
  "v_mfma_f32_16x16x128_f8f6f4 v[184:187], v[104:111], a[80:87], v[184:187]\n\t" \
  "ds_read_b64_tr_b8 v[122:123], v85 offset:50176\n\t" \
  "v_mfma_f32_16x16x128_f8f6f4 v[180:183], v[104:111], a[72:79], v[180:183]\n\t" \
  "ds_read_b64_tr_b8 v[124:125], v93 offset:49152\n\t" \
  "s_add_u32 s49, s68, 0x160000\n\t" \
  "buffer_load_dwordx4 a[204:207], v4, s[12:15], s49 offen offset:3072 sc0 sc1\n\t" \
  "v_mfma_f32_16x16x128_f8f6f4 v[176:179], v[104:111], a[64:71], v[176:179]\n\t" \
  "ds_read_b64_tr_b8 v[126:127], v93 offset:50176\n\t" \
  "s_waitcnt lgkmcnt(4)\n\t" \
  "v_mfma_f32_16x16x128_f8f6f4 v[192:195], v[112:119], a[64:71], v[192:195]\n\t" \
  "ds_read_b64_tr_b8 v[104:105], v86 offset:49152\n\t" \
  "s_add_u32 s50, s68, 0x161000\n\t" \
  "buffer_load_dwordx4 a[208:211], v4, s[12:15], s50 offen offset:0 sc0 sc1\n\t" \
  "v_mfma_f32_16x16x128_f8f6f4 v[196:199], v[112:119], a[72:79], v[196:199]\n\t" \
  "ds_read_b64_tr_b8 v[106:107], v86 offset:50176\n\t" \
  "v_mfma_f32_16x16x128_f8f6f4 v[200:203], v[112:119], a[80:87], v[200:203]\n\t" \
  "ds_read_b64_tr_b8 v[108:109], v94 offset:49152\n\t" \
  "s_add_u32 s51, s68, 0x161000\n\t" \
  "buffer_load_dwordx4 a[212:215], v4, s[12:15], s51 offen offset:1024 sc0 sc1\n\t" \
  "v_mfma_f32_16x16x128_f8f6f4 v[204:207], v[112:119], a[88:95], v[204:207]\n\t" \
  "ds_read_b64_tr_b8 v[110:111], v94 offset:50176\n\t" \
  "s_waitcnt lgkmcnt(4)\n\t" \
  "v_mfma_f32_16x16x128_f8f6f4 v[220:223], v[120:127], a[88:95], v[220:223]\n\t" \
  "ds_read_b64_tr_b8 v[112:113], v87 offset:49152\n\t" \
  "s_add_u32 s52, s68, 0x161000\n\t" \
  "buffer_load_dwordx4 a[216:219], v4, s[12:15], s52 offen offset:2048 sc0 sc1\n\t" \
  "v_mfma_f32_16x16x128_f8f6f4 v[216:219], v[120:127], a[80:87], v[216:219]\n\t" \
  "ds_read_b64_tr_b8 v[114:115], v87 offset:50176\n\t" \
  "v_mfma_f32_16x16x128_f8f6f4 v[212:215], v[120:127], a[72:79], v[212:215]\n\t" \
  "ds_read_b64_tr_b8 v[116:117], v95 offset:49152\n\t" \
  "s_add_u32 s53, s68, 0x161000\n\t" \
  "buffer_load_dwordx4 a[220:223], v4, s[12:15], s53 offen offset:3072 sc0 sc1\n\t" \
  "v_mfma_f32_16x16x128_f8f6f4 v[208:211], v[120:127], a[64:71], v[208:211]\n\t" \
  "ds_read_b64_tr_b8 v[118:119], v95 offset:50176\n\t" \
  "s_waitcnt vmcnt(38) lgkmcnt(0)\n\t" \
  "s_barrier\n\t" \
  "v_mfma_f32_16x16x128_f8f6f4 v[224:227], v[104:111], a[64:71], v[224:227]\n\t" \
  "ds_read_b64_tr_b8 v[120:121], v80 offset:57344\n\t" \
  "s_add_u32 m0, s24, 0x0\n\t" \
  "s_add_u32 s54, s46, 0x400000\n\t" \
  "buffer_load_dwordx4 v2, s[8:11], s54 offen lds\n\t" \
  "v_mfma_f32_16x16x128_f8f6f4 v[228:231], v[104:111], a[72:79], v[228:231]\n\t" \
  "ds_read_b64_tr_b8 v[122:123], v80 offset:58368\n\t" \
  "v_mfma_f32_16x16x128_f8f6f4 v[232:235], v[104:111], a[80:87], v[232:235]\n\t" \
  "ds_read_b64_tr_b8 v[124:125], v88 offset:57344\n\t" \
  "s_add_u32 m0, s24, 0x400\n\t" \
  "s_add_u32 s55, s46, 0x420000\n\t" \
  "buffer_load_dwordx4 v2, s[8:11], s55 offen lds\n\t" \
  "v_mfma_f32_16x16x128_f8f6f4 v[236:239], v[104:111], a[88:95], v[236:239]\n\t" \
  "ds_read_b64_tr_b8 v[126:127], v88 offset:58368\n\t" \
  "v_mfma_f32_16x16x128_f8f6f4 v[252:255], v[112:119], a[88:95], v[252:255]\n\t" \
  "ds_read_b64_tr_b8 v[104:105], v81 offset:57344\n\t" \
  "s_add_u32 m0, s24, 0x2000\n\t" \
  "s_add_u32 s48, s40, 0x600000\n\t" \
  "buffer_load_dwordx4 v2, s[8:11], s48 offen lds\n\t" \
  "v_mfma_f32_16x16x128_f8f6f4 v[248:251], v[112:119], a[80:87], v[248:251]\n\t" \
  "ds_read_b64_tr_b8 v[106:107], v81 offset:58368\n\t" \
  "v_mfma_f32_16x16x128_f8f6f4 v[244:247], v[112:119], a[72:79], v[244:247]\n\t" \
  "ds_read_b64_tr_b8 v[108:109], v89 offset:57344\n\t" \
  "s_add_u32 m0, s24, 0x2400\n\t" \
  "s_add_u32 s49, s40, 0x620000\n\t" \
  "buffer_load_dwordx4 v2, s[8:11], s49 offen lds\n\t" \
  "v_mfma_f32_16x16x128_f8f6f4 v[240:243], v[112:119], a[64:71], v[240:243]\n\t" \
  "ds_read_b64_tr_b8 v[110:111], v89 offset:58368\n\t" \
  "s_waitcnt vmcnt(34) lgkmcnt(4)\n\t" \
  "v_mfma_f32_16x16x128_f8f6f4 v[128:131], v[120:127], a[96:103], v[128:131]\n\t" \
  "ds_read_b64_tr_b8 v[112:113], v82 offset:57344\n\t" \
  "s_add_u32 m0, s24, 0x4000\n\t" \
  "s_add_u32 s50, s41, 0x600000\n\t" \
  "buffer_load_dwordx4 v2, s[8:11], s50 offen lds\n\t" \
  "v_mfma_f32_16x16x128_f8f6f4 v[132:135], v[120:127], a[104:111], v[132:135]\n\t" \
  "ds_read_b64_tr_b8 v[114:115], v82 offset:58368\n\t" \
  "v_mfma_f32_16x16x128_f8f6f4 v[136:139], v[120:127], a[112:119], v[136:139]\n\t" \
  "ds_read_b64_tr_b8 v[116:117], v90 offset:57344\n\t" \
  "s_add_u32 m0, s24, 0x4400\n\t" \
  "s_add_u32 s51, s41, 0x620000\n\t" \
  "buffer_load_dwordx4 v2, s[8:11], s51 offen lds\n\t" \
  "v_mfma_f32_16x16x128_f8f6f4 v[140:143], v[120:127], a[120:127], v[140:143]\n\t" \
  "ds_read_b64_tr_b8 v[118:119], v90 offset:58368\n\t" \
  "s_waitcnt lgkmcnt(4)\n\t" \
  "v_mfma_f32_16x16x128_f8f6f4 v[156:159], v[104:111], a[120:127], v[156:159]\n\t" \
  "ds_read_b64_tr_b8 v[120:121], v83 offset:57344\n\t" \
  "s_add_u32 m0, s24, 0x6000\n\t" \
  "s_add_u32 s52, s42, 0x600000\n\t" \
  "buffer_load_dwordx4 v2, s[8:11], s52 offen lds\n\t" \
  "v_mfma_f32_16x16x128_f8f6f4 v[152:155], v[104:111], a[112:119], v[152:155]\n\t" \
  "ds_read_b64_tr_b8 v[122:123], v83 offset:58368\n\t" \
  "v_mfma_f32_16x16x128_f8f6f4 v[148:151], v[104:111], a[104:111], v[148:151]\n\t" \
  "ds_read_b64_tr_b8 v[124:125], v91 offset:57344\n\t" \
  "s_add_u32 m0, s24, 0x6400\n\t" \
  "s_add_u32 s53, s42, 0x620000\n\t" \
  "buffer_load_dwordx4 v2, s[8:11], s53 offen lds\n\t" \
  "v_mfma_f32_16x16x128_f8f6f4 v[144:147], v[104:111], a[96:103], v[144:147]\n\t" \
  "ds_read_b64_tr_b8 v[126:127], v91 offset:58368\n\t" \
  "s_waitcnt lgkmcnt(4)\n\t" \
  "v_mfma_f32_16x16x128_f8f6f4 v[160:163], v[112:119], a[96:103], v[160:163]\n\t" \
  "ds_read_b64_tr_b8 v[104:105], v84 offset:57344\n\t" \
  "v_mfma_f32_16x16x128_f8f6f4 v[164:167], v[112:119], a[104:111], v[164:167]\n\t" \
  "ds_read_b64_tr_b8 v[106:107], v84 offset:58368\n\t" \
  "v_mfma_f32_16x16x128_f8f6f4 v[168:171], v[112:119], a[112:119], v[168:171]\n\t" \
  "ds_read_b64_tr_b8 v[108:109], v92 offset:57344\n\t" \
  "v_mfma_f32_16x16x128_f8f6f4 v[172:175], v[112:119], a[120:127], v[172:175]\n\t" \
  "ds_read_b64_tr_b8 v[110:111], v92 offset:58368\n\t" \
  "s_waitcnt lgkmcnt(4)\n\t" \
  "v_mfma_f32_16x16x128_f8f6f4 v[188:191], v[120:127], a[120:127], v[188:191]\n\t" \
  "ds_read_b64_tr_b8 v[112:113], v85 offset:57344\n\t" \
  "v_mfma_f32_16x16x128_f8f6f4 v[184:187], v[120:127], a[112:119], v[184:187]\n\t" \
  "ds_read_b64_tr_b8 v[114:115], v85 offset:58368\n\t" \
  "v_mfma_f32_16x16x128_f8f6f4 v[180:183], v[120:127], a[104:111], v[180:183]\n\t" \
  "ds_read_b64_tr_b8 v[116:117], v93 offset:57344\n\t" \
  "v_mfma_f32_16x16x128_f8f6f4 v[176:179], v[120:127], a[96:103], v[176:179]\n\t" \
  "ds_read_b64_tr_b8 v[118:119], v93 offset:58368\n\t" \
  "s_waitcnt lgkmcnt(4)\n\t" \
  "v_mfma_f32_16x16x128_f8f6f4 v[192:195], v[104:111], a[96:103], v[192:195]\n\t" \
  "ds_read_b64_tr_b8 v[120:121], v86 offset:57344\n\t" \
  "v_mfma_f32_16x16x128_f8f6f4 v[196:199], v[104:111], a[104:111], v[196:199]\n\t" \
  "ds_read_b64_tr_b8 v[122:123], v86 offset:58368\n\t" \
  "v_mfma_f32_16x16x128_f8f6f4 v[200:203], v[104:111], a[112:119], v[200:203]\n\t" \
  "ds_read_b64_tr_b8 v[124:125], v94 offset:57344\n\t" \
  "v_mfma_f32_16x16x128_f8f6f4 v[204:207], v[104:111], a[120:127], v[204:207]\n\t" \
  "ds_read_b64_tr_b8 v[126:127], v94 offset:58368\n\t" \
  "s_waitcnt lgkmcnt(4)\n\t" \
  "v_mfma_f32_16x16x128_f8f6f4 v[220:223], v[112:119], a[120:127], v[220:223]\n\t" \
  "ds_read_b64_tr_b8 v[104:105], v87 offset:57344\n\t" \
  "v_mfma_f32_16x16x128_f8f6f4 v[216:219], v[112:119], a[112:119], v[216:219]\n\t" \
  "ds_read_b64_tr_b8 v[106:107], v87 offset:58368\n\t" \
  "v_mfma_f32_16x16x128_f8f6f4 v[212:215], v[112:119], a[104:111], v[212:215]\n\t" \
  "ds_read_b64_tr_b8 v[108:109], v95 offset:57344\n\t" \
  "s_add_u32 m0, s24, 0x26000\n\t" \
  "s_add_u32 s54, s45, 0x400000\n\t" \
  "buffer_load_dwordx4 v3, s[8:11], s54 offen lds\n\t" \
  "v_mfma_f32_16x16x128_f8f6f4 v[208:211], v[112:119], a[96:103], v[208:211]\n\t" \
  "ds_read_b64_tr_b8 v[110:111], v95 offset:58368\n\t" \
  "s_waitcnt vmcnt(37) lgkmcnt(0)\n\t" \
  "s_barrier\n\t" \
  "v_mfma_f32_16x16x128_f8f6f4 v[224:227], v[120:127], a[96:103], v[224:227]\n\t" \
  "ds_read_b64_tr_b8 v[112:113], v88 offset:0\n\t" \
  "s_add_u32 m0, s24, 0x26400\n\t" \
  "s_add_u32 s55, s45, 0x420000\n\t" \
  "buffer_load_dwordx4 v3, s[8:11], s55 offen lds\n\t" \
  "v_mfma_f32_16x16x128_f8f6f4 v[228:231], v[120:127], a[104:111], v[228:231]\n\t" \
  "ds_read_b64_tr_b8 v[114:115], v88 offset:1024\n\t" \
  "v_mfma_f32_16x16x128_f8f6f4 v[232:235], v[120:127], a[112:119], v[232:235]\n\t" \
  "ds_read_b64_tr_b8 v[116:117], v96 offset:0\n\t" \
  "s_add_u32 s48, s68, 0x170000\n\t" \
  "buffer_load_dwordx4 a[224:227], v4, s[12:15], s48 offen offset:0 sc0 sc1\n\t" \
  "v_mfma_f32_16x16x128_f8f6f4 v[236:239], v[120:127], a[120:127], v[236:239]\n\t" \
  "ds_read_b64_tr_b8 v[118:119], v96 offset:1024\n\t" \
  "v_mfma_f32_16x16x128_f8f6f4 v[252:255], v[104:111], a[120:127], v[252:255]\n\t" \
  "ds_read_b64_tr_b8 v[120:121], v89 offset:0\n\t" \
  "s_add_u32 s49, s68, 0x170000\n\t" \
  "buffer_load_dwordx4 a[228:231], v4, s[12:15], s49 offen offset:1024 sc0 sc1\n\t" \
  "v_mfma_f32_16x16x128_f8f6f4 v[248:251], v[104:111], a[112:119], v[248:251]\n\t" \
  "ds_read_b64_tr_b8 v[122:123], v89 offset:1024\n\t" \
  "v_mfma_f32_16x16x128_f8f6f4 v[244:247], v[104:111], a[104:111], v[244:247]\n\t" \
  "ds_read_b64_tr_b8 v[124:125], v97 offset:0\n\t" \
  "s_add_u32 s50, s68, 0x170000\n\t" \
  "buffer_load_dwordx4 a[232:235], v4, s[12:15], s50 offen offset:2048 sc0 sc1\n\t" \
  "v_mfma_f32_16x16x128_f8f6f4 v[240:243], v[104:111], a[96:103], v[240:243]\n\t" \
  "ds_read_b64_tr_b8 v[126:127], v97 offset:1024\n\t" \
  "s_waitcnt vmcnt(33) lgkmcnt(4)\n\t" \
  "v_mfma_f32_16x16x128_f8f6f4 v[128:131], v[112:119], a[128:135], v[128:131]\n\t" \
  "ds_read_b64_tr_b8 v[104:105], v90 offset:0\n\t" \
  "s_add_u32 s51, s68, 0x170000\n\t" \
  "buffer_load_dwordx4 a[236:239], v4, s[12:15], s51 offen offset:3072 sc0 sc1\n\t" \
  "s_mov_b32 exec_lo, 0x7f000000\n\t" \
  "s_mov_b32 exec_hi, 0\n\t" \
  "global_load_dwordx4 v[68:71], v10, s[6:7] sc0 sc1\n\t" \
  "s_mov_b64 exec, -1\n\t" \
  "v_mfma_f32_16x16x128_f8f6f4 v[132:135], v[112:119], a[136:143], v[132:135]\n\t" \
  "ds_read_b64_tr_b8 v[106:107], v90 offset:1024\n\t" \
  "v_mfma_f32_16x16x128_f8f6f4 v[136:139], v[112:119], a[144:151], v[136:139]\n\t" \
  "ds_read_b64_tr_b8 v[108:109], v98 offset:0\n\t" \
  "s_add_u32 s52, s68, 0x171000\n\t" \
  "buffer_load_dwordx4 a[240:243], v4, s[12:15], s52 offen offset:0 sc0 sc1\n\t" \
  "v_mfma_f32_16x16x128_f8f6f4 v[140:143], v[112:119], a[152:159], v[140:143]\n\t" \
  "ds_read_b64_tr_b8 v[110:111], v98 offset:1024\n\t" \
  "s_waitcnt lgkmcnt(4)\n\t" \
  "v_mfma_f32_16x16x128_f8f6f4 v[156:159], v[120:127], a[152:159], v[156:159]\n\t" \
  "ds_read_b64_tr_b8 v[112:113], v91 offset:0\n\t" \
  "s_add_u32 s53, s68, 0x171000\n\t" \
  "buffer_load_dwordx4 a[244:247], v4, s[12:15], s53 offen offset:1024 sc0 sc1\n\t" \
  "v_mfma_f32_16x16x128_f8f6f4 v[152:155], v[120:127], a[144:151], v[152:155]\n\t" \
  "ds_read_b64_tr_b8 v[114:115], v91 offset:1024\n\t" \
  "v_mfma_f32_16x16x128_f8f6f4 v[148:151], v[120:127], a[136:143], v[148:151]\n\t" \
  "ds_read_b64_tr_b8 v[116:117], v99 offset:0\n\t" \
  "s_add_u32 s54, s68, 0x171000\n\t" \
  "buffer_load_dwordx4 a[248:251], v4, s[12:15], s54 offen offset:2048 sc0 sc1\n\t" \
  "v_mfma_f32_16x16x128_f8f6f4 v[144:147], v[120:127], a[128:135], v[144:147]\n\t" \
  "ds_read_b64_tr_b8 v[118:119], v99 offset:1024\n\t" \
  "s_waitcnt lgkmcnt(4)\n\t" \
  "v_mfma_f32_16x16x128_f8f6f4 v[160:163], v[104:111], a[128:135], v[160:163]\n\t" \
  "ds_read_b64_tr_b8 v[120:121], v92 offset:0\n\t" \
  "s_add_u32 s55, s68, 0x171000\n\t" \
  "buffer_load_dwordx4 a[252:255], v4, s[12:15], s55 offen offset:3072 sc0 sc1\n\t" \
  "v_mfma_f32_16x16x128_f8f6f4 v[164:167], v[104:111], a[136:143], v[164:167]\n\t" \
  "ds_read_b64_tr_b8 v[122:123], v92 offset:1024\n\t" \
  "v_mfma_f32_16x16x128_f8f6f4 v[168:171], v[104:111], a[144:151], v[168:171]\n\t" \
  "ds_read_b64_tr_b8 v[124:125], v100 offset:0\n\t" \
  "s_add_u32 m0, s24, 0x18000\n\t" \
  "s_add_u32 s48, s46, 0x400000\n\t" \
  "buffer_load_dwordx4 v3, s[8:11], s48 offen lds\n\t" \
  "v_mfma_f32_16x16x128_f8f6f4 v[172:175], v[104:111], a[152:159], v[172:175]\n\t" \
  "ds_read_b64_tr_b8 v[126:127], v100 offset:1024\n\t" \
  "s_waitcnt lgkmcnt(4)\n\t" \
  "v_mfma_f32_16x16x128_f8f6f4 v[188:191], v[112:119], a[152:159], v[188:191]\n\t" \
  "ds_read_b64_tr_b8 v[104:105], v93 offset:0\n\t" \
  "s_add_u32 m0, s24, 0x18400\n\t" \
  "s_add_u32 s49, s46, 0x420000\n\t" \
  "buffer_load_dwordx4 v3, s[8:11], s49 offen lds\n\t" \
  "v_mfma_f32_16x16x128_f8f6f4 v[184:187], v[112:119], a[144:151], v[184:187]\n\t" \
  "ds_read_b64_tr_b8 v[106:107], v93 offset:1024\n\t" \
  "v_mfma_f32_16x16x128_f8f6f4 v[180:183], v[112:119], a[136:143], v[180:183]\n\t" \
  "ds_read_b64_tr_b8 v[108:109], v101 offset:0\n\t" \
  "s_add_u32 s50, s68, 0x180000\n\t" \
  "buffer_load_dwordx4 a[0:3], v4, s[12:15], s50 offen offset:0 sc0 sc1\n\t" \
  "v_mfma_f32_16x16x128_f8f6f4 v[176:179], v[112:119], a[128:135], v[176:179]\n\t" \
  "ds_read_b64_tr_b8 v[110:111], v101 offset:1024\n\t" \
  "s_waitcnt lgkmcnt(4)\n\t" \
  "v_mfma_f32_16x16x128_f8f6f4 v[192:195], v[120:127], a[128:135], v[192:195]\n\t" \
  "ds_read_b64_tr_b8 v[112:113], v94 offset:0\n\t" \
  "s_add_u32 s51, s68, 0x180000\n\t" \
  "buffer_load_dwordx4 a[4:7], v4, s[12:15], s51 offen offset:1024 sc0 sc1\n\t" \
  "v_mfma_f32_16x16x128_f8f6f4 v[196:199], v[120:127], a[136:143], v[196:199]\n\t" \
  "ds_read_b64_tr_b8 v[114:115], v94 offset:1024\n\t" \
  "v_mfma_f32_16x16x128_f8f6f4 v[200:203], v[120:127], a[144:151], v[200:203]\n\t" \
  "ds_read_b64_tr_b8 v[116:117], v102 offset:0\n\t" \
  "s_add_u32 s52, s68, 0x180000\n\t" \
  "buffer_load_dwordx4 a[8:11], v4, s[12:15], s52 offen offset:2048 sc0 sc1\n\t" \
  "v_mfma_f32_16x16x128_f8f6f4 v[204:207], v[120:127], a[152:159], v[204:207]\n\t" \
  "ds_read_b64_tr_b8 v[118:119], v102 offset:1024\n\t" \
  "s_waitcnt lgkmcnt(4)\n\t" \
  "v_mfma_f32_16x16x128_f8f6f4 v[220:223], v[104:111], a[152:159], v[220:223]\n\t" \
  "ds_read_b64_tr_b8 v[120:121], v95 offset:0\n\t" \
  "s_add_u32 s53, s68, 0x180000\n\t" \
  "buffer_load_dwordx4 a[12:15], v4, s[12:15], s53 offen offset:3072 sc0 sc1\n\t" \
  "v_mfma_f32_16x16x128_f8f6f4 v[216:219], v[104:111], a[144:151], v[216:219]\n\t" \
  "ds_read_b64_tr_b8 v[122:123], v95 offset:1024\n\t" \
  "v_mfma_f32_16x16x128_f8f6f4 v[212:215], v[104:111], a[136:143], v[212:215]\n\t" \
  "ds_read_b64_tr_b8 v[124:125], v103 offset:0\n\t" \
  "s_add_u32 s54, s68, 0x181000\n\t" \
  "buffer_load_dwordx4 a[16:19], v4, s[12:15], s54 offen offset:0 sc0 sc1\n\t" \
  "v_mfma_f32_16x16x128_f8f6f4 v[208:211], v[104:111], a[128:135], v[208:211]\n\t" \
  "ds_read_b64_tr_b8 v[126:127], v103 offset:1024\n\t" \
  "s_waitcnt vmcnt(44) lgkmcnt(0)\n\t" \
  "s_barrier\n\t" \
  "v_mfma_f32_16x16x128_f8f6f4 v[224:227], v[112:119], a[128:135], v[224:227]\n\t" \
  "ds_read_b64_tr_b8 v[104:105], v88 offset:8192\n\t" \
  "s_add_u32 s55, s68, 0x181000\n\t" \
  "buffer_load_dwordx4 a[20:23], v4, s[12:15], s55 offen offset:1024 sc0 sc1\n\t" \
  "v_mfma_f32_16x16x128_f8f6f4 v[228:231], v[112:119], a[136:143], v[228:231]\n\t" \
  "ds_read_b64_tr_b8 v[106:107], v88 offset:9216\n\t" \
  "v_mfma_f32_16x16x128_f8f6f4 v[232:235], v[112:119], a[144:151], v[232:235]\n\t" \
  "ds_read_b64_tr_b8 v[108:109], v96 offset:8192\n\t" \
  "s_add_u32 s48, s68, 0x181000\n\t" \
  "buffer_load_dwordx4 a[24:27], v4, s[12:15], s48 offen offset:2048 sc0 sc1\n\t" \
  "v_mfma_f32_16x16x128_f8f6f4 v[236:239], v[112:119], a[152:159], v[236:239]\n\t" \
  "ds_read_b64_tr_b8 v[110:111], v96 offset:9216\n\t" \
  "v_mfma_f32_16x16x128_f8f6f4 v[252:255], v[120:127], a[152:159], v[252:255]\n\t" \
  "ds_read_b64_tr_b8 v[112:113], v89 offset:8192\n\t" \
  "s_add_u32 s49, s68, 0x181000\n\t" \
  "buffer_load_dwordx4 a[28:31], v4, s[12:15], s49 offen offset:3072 sc0 sc1\n\t" \
  "v_mfma_f32_16x16x128_f8f6f4 v[248:251], v[120:127], a[144:151], v[248:251]\n\t" \
  "ds_read_b64_tr_b8 v[114:115], v89 offset:9216\n\t" \
  "v_mfma_f32_16x16x128_f8f6f4 v[244:247], v[120:127], a[136:143], v[244:247]\n\t" \
  "ds_read_b64_tr_b8 v[116:117], v97 offset:8192\n\t" \
  "s_add_u32 m0, s24, 0x1a000\n\t" \
  "s_add_u32 s50, s40, 0x600000\n\t" \
  "buffer_load_dwordx4 v3, s[8:11], s50 offen lds\n\t" \
  "v_mfma_f32_16x16x128_f8f6f4 v[240:243], v[120:127], a[128:135], v[240:243]\n\t" \
  "ds_read_b64_tr_b8 v[118:119], v97 offset:9216\n\t" \
  "s_waitcnt vmcnt(40) lgkmcnt(4)\n\t" \
  "v_mfma_f32_16x16x128_f8f6f4 v[128:131], v[104:111], a[160:167], v[128:131]\n\t" \
  "ds_read_b64_tr_b8 v[120:121], v90 offset:8192\n\t" \
  "s_add_u32 m0, s24, 0x1a400\n\t" \
  "s_add_u32 s51, s40, 0x620000\n\t" \
  "buffer_load_dwordx4 v3, s[8:11], s51 offen lds\n\t" \
  "v_mfma_f32_16x16x128_f8f6f4 v[132:135], v[104:111], a[168:175], v[132:135]\n\t" \
  "ds_read_b64_tr_b8 v[122:123], v90 offset:9216\n\t" \
  "v_mfma_f32_16x16x128_f8f6f4 v[136:139], v[104:111], a[176:183], v[136:139]\n\t" \
  "ds_read_b64_tr_b8 v[124:125], v98 offset:8192\n\t" \
  "s_waitcnt vmcnt(16)\n\t" \
  "v_min_u32 v68, v68, v69\n\t" \
  "v_min3_u32 v68, v68, v70, v71\n\t" \
  "v_cmp_le_u32 vcc, s57, v68\n\t" \
  "s_and_b32 s64, vcc_lo, 0x7f000000\n\t" \
  "s_or_b32 s56, s56, s64\n\t" \
  "s_and_b32 s64, s56, 0x7f000000\n\t" \
  "s_cmp_eq_u32 s64, 0x7f000000\n\t" \
  "s_cbranch_scc1 Lok9_3_%=\n\t" \
  "s_mov_b32 s62, 0x7f000000\n\t" \
  "s_mov_b32 s69, 0x7f000000\n\t" \
  "s_call_b64 s[60:61], Lpoll_3_%=\n\t" \
  "Lok9_3_%=:\n\t" \
  "s_add_u32 s52, s68, 0x190000\n\t" \
  "buffer_load_dwordx4 a[32:35], v4, s[12:15], s52 offen offset:0 sc0 sc1\n\t" \
  "v_mfma_f32_16x16x128_f8f6f4 v[140:143], v[104:111], a[184:191], v[140:143]\n\t" \
  "ds_read_b64_tr_b8 v[126:127], v98 offset:9216\n\t" \
  "s_waitcnt lgkmcnt(4)\n\t" \
  "v_mfma_f32_16x16x128_f8f6f4 v[156:159], v[112:119], a[184:191], v[156:159]\n\t" \
  "ds_read_b64_tr_b8 v[104:105], v91 offset:8192\n\t" \
  "s_add_u32 s53, s68, 0x190000\n\t" \
  "buffer_load_dwordx4 a[36:39], v4, s[12:15], s53 offen offset:1024 sc0 sc1\n\t" \
  "v_mfma_f32_16x16x128_f8f6f4 v[152:155], v[112:119], a[176:183], v[152:155]\n\t" \
  "ds_read_b64_tr_b8 v[106:107], v91 offset:9216\n\t" \
  "v_mfma_f32_16x16x128_f8f6f4 v[148:151], v[112:119], a[168:175], v[148:151]\n\t" \
  "ds_read_b64_tr_b8 v[108:109], v99 offset:8192\n\t" \
  "s_add_u32 s54, s68, 0x190000\n\t" \
  "buffer_load_dwordx4 a[40:43], v4, s[12:15], s54 offen offset:2048 sc0 sc1\n\t" \
  "v_mfma_f32_16x16x128_f8f6f4 v[144:147], v[112:119], a[160:167], v[144:147]\n\t" \
  "ds_read_b64_tr_b8 v[110:111], v99 offset:9216\n\t" \
  "s_waitcnt lgkmcnt(4)\n\t" \
  "v_mfma_f32_16x16x128_f8f6f4 v[160:163], v[120:127], a[160:167], v[160:163]\n\t" \
  "ds_read_b64_tr_b8 v[112:113], v92 offset:8192\n\t" \
  "s_add_u32 s55, s68, 0x190000\n\t" \
  "buffer_load_dwordx4 a[44:47], v4, s[12:15], s55 offen offset:3072 sc0 sc1\n\t" \
  "v_mfma_f32_16x16x128_f8f6f4 v[164:167], v[120:127], a[168:175], v[164:167]\n\t" \
  "ds_read_b64_tr_b8 v[114:115], v92 offset:9216\n\t" \
  "v_mfma_f32_16x16x128_f8f6f4 v[168:171], v[120:127], a[176:183], v[168:171]\n\t" \
  "ds_read_b64_tr_b8 v[116:117], v100 offset:8192\n\t" \
  "s_add_u32 s48, s68, 0x191000\n\t" \
  "buffer_load_dwordx4 a[48:51], v4, s[12:15], s48 offen offset:0 sc0 sc1\n\t" \
  "v_mfma_f32_16x16x128_f8f6f4 v[172:175], v[120:127], a[184:191], v[172:175]\n\t" \
  "ds_read_b64_tr_b8 v[118:119], v100 offset:9216\n\t" \
  "s_waitcnt lgkmcnt(4)\n\t" \
  "v_mfma_f32_16x16x128_f8f6f4 v[188:191], v[104:111], a[184:191], v[188:191]\n\t" \
  "ds_read_b64_tr_b8 v[120:121], v93 offset:8192\n\t" \
  "s_add_u32 s49, s68, 0x191000\n\t" \
  "buffer_load_dwordx4 a[52:55], v4, s[12:15], s49 offen offset:1024 sc0 sc1\n\t" \
  "v_mfma_f32_16x16x128_f8f6f4 v[184:187], v[104:111], a[176:183], v[184:187]\n\t" \
  "ds_read_b64_tr_b8 v[122:123], v93 offset:9216\n\t" \
  "v_mfma_f32_16x16x128_f8f6f4 v[180:183], v[104:111], a[168:175], v[180:183]\n\t" \
  "ds_read_b64_tr_b8 v[124:125], v101 offset:8192\n\t" \
  "s_add_u32 s50, s68, 0x191000\n\t" \
  "buffer_load_dwordx4 a[56:59], v4, s[12:15], s50 offen offset:2048 sc0 sc1\n\t" \
  "v_mfma_f32_16x16x128_f8f6f4 v[176:179], v[104:111], a[160:167], v[176:179]\n\t" \
  "ds_read_b64_tr_b8 v[126:127], v101 offset:9216\n\t" \
  "s_waitcnt lgkmcnt(4)\n\t" \
  "v_mfma_f32_16x16x128_f8f6f4 v[192:195], v[112:119], a[160:167], v[192:195]\n\t" \
  "ds_read_b64_tr_b8 v[104:105], v94 offset:8192\n\t" \
  "s_add_u32 s51, s68, 0x191000\n\t" \
  "buffer_load_dwordx4 a[60:63], v4, s[12:15], s51 offen offset:3072 sc0 sc1\n\t" \
  "v_mfma_f32_16x16x128_f8f6f4 v[196:199], v[112:119], a[168:175], v[196:199]\n\t" \
  "ds_read_b64_tr_b8 v[106:107], v94 offset:9216\n\t" \
  "v_mfma_f32_16x16x128_f8f6f4 v[200:203], v[112:119], a[176:183], v[200:203]\n\t" \
  "ds_read_b64_tr_b8 v[108:109], v102 offset:8192\n\t" \
  "s_add_u32 m0, s24, 0x1c000\n\t" \
  "s_add_u32 s52, s41, 0x600000\n\t" \
  "buffer_load_dwordx4 v3, s[8:11], s52 offen lds\n\t" \
  "v_mfma_f32_16x16x128_f8f6f4 v[204:207], v[112:119], a[184:191], v[204:207]\n\t" \
  "ds_read_b64_tr_b8 v[110:111], v102 offset:9216\n\t" \
  "s_waitcnt lgkmcnt(4)\n\t" \
  "v_mfma_f32_16x16x128_f8f6f4 v[220:223], v[120:127], a[184:191], v[220:223]\n\t" \
  "ds_read_b64_tr_b8 v[112:113], v95 offset:8192\n\t" \
  "s_add_u32 m0, s24, 0x1c400\n\t" \
  "s_add_u32 s53, s41, 0x620000\n\t" \
  "buffer_load_dwordx4 v3, s[8:11], s53 offen lds\n\t" \
  "v_mfma_f32_16x16x128_f8f6f4 v[216:219], v[120:127], a[176:183], v[216:219]\n\t" \
  "ds_read_b64_tr_b8 v[114:115], v95 offset:9216\n\t" \
  "v_mfma_f32_16x16x128_f8f6f4 v[212:215], v[120:127], a[168:175], v[212:215]\n\t" \
  "ds_read_b64_tr_b8 v[116:117], v103 offset:8192\n\t" \
  "s_add_u32 m0, s24, 0x8000\n\t" \
  "s_add_u32 s54, s43, 0x600000\n\t" \
  "buffer_load_dwordx4 v2, s[8:11], s54 offen lds\n\t" \
  "v_mfma_f32_16x16x128_f8f6f4 v[208:211], v[120:127], a[160:167], v[208:211]\n\t" \
  "ds_read_b64_tr_b8 v[118:119], v103 offset:9216\n\t" \
  "s_waitcnt lgkmcnt(0)\n\t" \
  "s_barrier\n\t" \
  "v_mfma_f32_16x16x128_f8f6f4 v[224:227], v[104:111], a[160:167], v[224:227]\n\t" \
  "ds_read_b64_tr_b8 v[120:121], v88 offset:16384\n\t" \
  "s_add_u32 m0, s24, 0x8400\n\t" \
  "s_add_u32 s55, s43, 0x620000\n\t" \
  "buffer_load_dwordx4 v2, s[8:11], s55 offen lds\n\t" \
  "v_mfma_f32_16x16x128_f8f6f4 v[228:231], v[104:111], a[168:175], v[228:231]\n\t" \
  "ds_read_b64_tr_b8 v[122:123], v88 offset:17408\n\t" \
  "v_mfma_f32_16x16x128_f8f6f4 v[232:235], v[104:111], a[176:183], v[232:235]\n\t" \
  "ds_read_b64_tr_b8 v[124:125], v96 offset:16384\n\t" \
  "s_add_u32 m0, s24, 0xa000\n\t" \
  "s_add_u32 s48, s44, 0x600000\n\t" \
  "buffer_load_dwordx4 v2, s[8:11], s48 offen lds\n\t" \
  "v_mfma_f32_16x16x128_f8f6f4 v[236:239], v[104:111], a[184:191], v[236:239]\n\t" \
  "ds_read_b64_tr_b8 v[126:127], v96 offset:17408\n\t" \
  "v_mfma_f32_16x16x128_f8f6f4 v[252:255], v[112:119], a[184:191], v[252:255]\n\t" \
  "ds_read_b64_tr_b8 v[104:105], v89 offset:16384\n\t" \
  "s_add_u32 m0, s24, 0xa400\n\t" \
  "s_add_u32 s49, s44, 0x620000\n\t" \
  "buffer_load_dwordx4 v2, s[8:11], s49 offen lds\n\t" \
  "v_mfma_f32_16x16x128_f8f6f4 v[248:251], v[112:119], a[176:183], v[248:251]\n\t" \
  "ds_read_b64_tr_b8 v[106:107], v89 offset:17408\n\t" \
  "v_mfma_f32_16x16x128_f8f6f4 v[244:247], v[112:119], a[168:175], v[244:247]\n\t" \
  "ds_read_b64_tr_b8 v[108:109], v97 offset:16384\n\t" \
  "s_add_u32 m0, s24, 0xc000\n\t" \
  "s_add_u32 s50, s45, 0x600000\n\t" \
  "buffer_load_dwordx4 v2, s[8:11], s50 offen lds\n\t" \
  "v_mfma_f32_16x16x128_f8f6f4 v[240:243], v[112:119], a[160:167], v[240:243]\n\t" \
  "ds_read_b64_tr_b8 v[110:111], v97 offset:17408\n\t" \
  "s_waitcnt lgkmcnt(4)\n\t" \
  "v_mfma_f32_16x16x128_f8f6f4 v[128:131], v[120:127], a[192:199], v[128:131]\n\t" \
  "ds_read_b64_tr_b8 v[112:113], v90 offset:16384\n\t" \
  "s_add_u32 m0, s24, 0xc400\n\t" \
  "s_add_u32 s51, s45, 0x620000\n\t" \
  "buffer_load_dwordx4 v2, s[8:11], s51 offen lds\n\t" \
  "v_mfma_f32_16x16x128_f8f6f4 v[132:135], v[120:127], a[200:207], v[132:135]\n\t" \
  "ds_read_b64_tr_b8 v[114:115], v90 offset:17408\n\t" \
  "v_mfma_f32_16x16x128_f8f6f4 v[136:139], v[120:127], a[208:215], v[136:139]\n\t" \
  "ds_read_b64_tr_b8 v[116:117], v98 offset:16384\n\t" \
  "s_add_u32 m0, s24, 0xe000\n\t" \
  "s_add_u32 s52, s46, 0x600000\n\t" \
  "buffer_load_dwordx4 v2, s[8:11], s52 offen lds\n\t" \
  "v_mfma_f32_16x16x128_f8f6f4 v[140:143], v[120:127], a[216:223], v[140:143]\n\t" \
  "ds_read_b64_tr_b8 v[118:119], v98 offset:17408\n\t" \
  "s_waitcnt lgkmcnt(4)\n\t" \
  "v_mfma_f32_16x16x128_f8f6f4 v[156:159], v[104:111], a[216:223], v[156:159]\n\t" \
  "ds_read_b64_tr_b8 v[120:121], v91 offset:16384\n\t" \
  "s_add_u32 m0, s24, 0xe400\n\t" \
  "s_add_u32 s53, s46, 0x620000\n\t" \
  "buffer_load_dwordx4 v2, s[8:11], s53 offen lds\n\t" \
  "v_mfma_f32_16x16x128_f8f6f4 v[152:155], v[104:111], a[208:215], v[152:155]\n\t" \
  "ds_read_b64_tr_b8 v[122:123], v91 offset:17408\n\t" \
  "v_mfma_f32_16x16x128_f8f6f4 v[148:151], v[104:111], a[200:207], v[148:151]\n\t" \
  "ds_read_b64_tr_b8 v[124:125], v99 offset:16384\n\t" \
  "v_mfma_f32_16x16x128_f8f6f4 v[144:147], v[104:111], a[192:199], v[144:147]\n\t" \
  "ds_read_b64_tr_b8 v[126:127], v99 offset:17408\n\t" \
  "s_waitcnt lgkmcnt(4)\n\t" \
  "v_mfma_f32_16x16x128_f8f6f4 v[160:163], v[112:119], a[192:199], v[160:163]\n\t" \
  "ds_read_b64_tr_b8 v[104:105], v92 offset:16384\n\t" \
  "v_mfma_f32_16x16x128_f8f6f4 v[164:167], v[112:119], a[200:207], v[164:167]\n\t" \
  "ds_read_b64_tr_b8 v[106:107], v92 offset:17408\n\t" \
  "v_mfma_f32_16x16x128_f8f6f4 v[168:171], v[112:119], a[208:215], v[168:171]\n\t" \
  "ds_read_b64_tr_b8 v[108:109], v100 offset:16384\n\t" \
  "v_mfma_f32_16x16x128_f8f6f4 v[172:175], v[112:119], a[216:223], v[172:175]\n\t" \
  "ds_read_b64_tr_b8 v[110:111], v100 offset:17408\n\t" \
  "s_waitcnt lgkmcnt(4)\n\t" \
  "v_mfma_f32_16x16x128_f8f6f4 v[188:191], v[120:127], a[216:223], v[188:191]\n\t" \
  "ds_read_b64_tr_b8 v[112:113], v93 offset:16384\n\t" \
  "v_mfma_f32_16x16x128_f8f6f4 v[184:187], v[120:127], a[208:215], v[184:187]\n\t" \
  "ds_read_b64_tr_b8 v[114:115], v93 offset:17408\n\t" \
  "v_mfma_f32_16x16x128_f8f6f4 v[180:183], v[120:127], a[200:207], v[180:183]\n\t" \
  "ds_read_b64_tr_b8 v[116:117], v101 offset:16384\n\t" \
  "v_mfma_f32_16x16x128_f8f6f4 v[176:179], v[120:127], a[192:199], v[176:179]\n\t" \
  "ds_read_b64_tr_b8 v[118:119], v101 offset:17408\n\t" \
  "s_waitcnt lgkmcnt(4)\n\t" \
  "v_mfma_f32_16x16x128_f8f6f4 v[192:195], v[104:111], a[192:199], v[192:195]\n\t" \
  "ds_read_b64_tr_b8 v[120:121], v94 offset:16384\n\t" \
  "v_mfma_f32_16x16x128_f8f6f4 v[196:199], v[104:111], a[200:207], v[196:199]\n\t" \
  "ds_read_b64_tr_b8 v[122:123], v94 offset:17408\n\t" \
  "v_mfma_f32_16x16x128_f8f6f4 v[200:203], v[104:111], a[208:215], v[200:203]\n\t" \
  "ds_read_b64_tr_b8 v[124:125], v102 offset:16384\n\t" \
  "v_mfma_f32_16x16x128_f8f6f4 v[204:207], v[104:111], a[216:223], v[204:207]\n\t" \
  "ds_read_b64_tr_b8 v[126:127], v102 offset:17408\n\t" \
  "s_waitcnt lgkmcnt(4)\n\t" \
  "v_mfma_f32_16x16x128_f8f6f4 v[220:223], v[112:119], a[216:223], v[220:223]\n\t" \
  "ds_read_b64_tr_b8 v[104:105], v95 offset:16384\n\t" \
  "s_add_u32 m0, s24, 0x1e000\n\t" \
  "s_add_u32 s54, s42, 0x600000\n\t" \
  "buffer_load_dwordx4 v3, s[8:11], s54 offen lds\n\t" \
  "v_mfma_f32_16x16x128_f8f6f4 v[216:219], v[112:119], a[208:215], v[216:219]\n\t" \
  "ds_read_b64_tr_b8 v[106:107], v95 offset:17408\n\t" \
  "v_mfma_f32_16x16x128_f8f6f4 v[212:215], v[112:119], a[200:207], v[212:215]\n\t" \
  "ds_read_b64_tr_b8 v[108:109], v103 offset:16384\n\t" \
  "s_add_u32 m0, s24, 0x1e400\n\t" \
  "s_add_u32 s55, s42, 0x620000\n\t" \
  "buffer_load_dwordx4 v3, s[8:11], s55 offen lds\n\t" \
  "v_mfma_f32_16x16x128_f8f6f4 v[208:211], v[112:119], a[192:199], v[208:211]\n\t" \
  "ds_read_b64_tr_b8 v[110:111], v103 offset:17408\n\t" \
  "s_waitcnt lgkmcnt(0)\n\t" \
  "s_barrier\n\t" \
  "v_mfma_f32_16x16x128_f8f6f4 v[224:227], v[120:127], a[192:199], v[224:227]\n\t" \
  "ds_read_b64_tr_b8 v[112:113], v88 offset:24576\n\t" \
  "s_add_u32 m0, s24, 0x20000\n\t" \
  "s_add_u32 s48, s43, 0x600000\n\t" \
  "buffer_load_dwordx4 v3, s[8:11], s48 offen lds\n\t" \
  "v_mfma_f32_16x16x128_f8f6f4 v[228:231], v[120:127], a[200:207], v[228:231]\n\t" \
  "ds_read_b64_tr_b8 v[114:115], v88 offset:25600\n\t" \
  "v_mfma_f32_16x16x128_f8f6f4 v[232:235], v[120:127], a[208:215], v[232:235]\n\t" \
  "ds_read_b64_tr_b8 v[116:117], v96 offset:24576\n\t" \
  "s_add_u32 m0, s24, 0x20400\n\t" \
  "s_add_u32 s49, s43, 0x620000\n\t" \
  "buffer_load_dwordx4 v3, s[8:11], s49 offen lds\n\t" \
  "v_mfma_f32_16x16x128_f8f6f4 v[236:239], v[120:127], a[216:223], v[236:239]\n\t" \
  "ds_read_b64_tr_b8 v[118:119], v96 offset:25600\n\t" \
  "v_mfma_f32_16x16x128_f8f6f4 v[252:255], v[104:111], a[216:223], v[252:255]\n\t" \
  "ds_read_b64_tr_b8 v[120:121], v89 offset:24576\n\t" \
  "s_add_u32 m0, s24, 0x22000\n\t" \
  "s_add_u32 s50, s44, 0x600000\n\t" \
  "buffer_load_dwordx4 v3, s[8:11], s50 offen lds\n\t" \
  "v_mfma_f32_16x16x128_f8f6f4 v[248:251], v[104:111], a[208:215], v[248:251]\n\t" \
  "ds_read_b64_tr_b8 v[122:123], v89 offset:25600\n\t" \
  "v_mfma_f32_16x16x128_f8f6f4 v[244:247], v[104:111], a[200:207], v[244:247]\n\t" \
  "ds_read_b64_tr_b8 v[124:125], v97 offset:24576\n\t" \
  "s_add_u32 m0, s24, 0x22400\n\t" \
  "s_add_u32 s51, s44, 0x620000\n\t" \
  "buffer_load_dwordx4 v3, s[8:11], s51 offen lds\n\t" \
  "v_mfma_f32_16x16x128_f8f6f4 v[240:243], v[104:111], a[192:199], v[240:243]\n\t" \
  "ds_read_b64_tr_b8 v[126:127], v97 offset:25600\n\t" \
  "s_waitcnt vmcnt(36) lgkmcnt(4)\n\t" \
  "v_mfma_f32_16x16x128_f8f6f4 v[128:131], v[112:119], a[224:231], v[128:131]\n\t" \
  "ds_read_b64_tr_b8 v[104:105], v90 offset:24576\n\t" \
  "s_add_u32 m0, s24, 0x24000\n\t" \
  "s_add_u32 s52, s45, 0x600000\n\t" \
  "buffer_load_dwordx4 v3, s[8:11], s52 offen lds\n\t" \
  "v_mfma_f32_16x16x128_f8f6f4 v[132:135], v[112:119], a[232:239], v[132:135]\n\t" \
  "ds_read_b64_tr_b8 v[106:107], v90 offset:25600\n\t" \
  "v_mfma_f32_16x16x128_f8f6f4 v[136:139], v[112:119], a[240:247], v[136:139]\n\t" \
  "ds_read_b64_tr_b8 v[108:109], v98 offset:24576\n\t" \
  "s_add_u32 m0, s24, 0x24400\n\t" \
  "s_add_u32 s53, s45, 0x620000\n\t" \
  "buffer_load_dwordx4 v3, s[8:11], s53 offen lds\n\t" \
  "v_mfma_f32_16x16x128_f8f6f4 v[140:143], v[112:119], a[248:255], v[140:143]\n\t" \
  "ds_read_b64_tr_b8 v[110:111], v98 offset:25600\n\t" \
  "s_waitcnt lgkmcnt(4)\n\t" \
  "v_mfma_f32_16x16x128_f8f6f4 v[156:159], v[120:127], a[248:255], v[156:159]\n\t" \
  "ds_read_b64_tr_b8 v[112:113], v91 offset:24576\n\t" \
  "s_add_u32 s54, s68, 0x1a0000\n\t" \
  "buffer_load_dwordx4 a[64:67], v4, s[12:15], s54 offen offset:0 sc0 sc1\n\t" \
  "v_mfma_f32_16x16x128_f8f6f4 v[152:155], v[120:127], a[240:247], v[152:155]\n\t" \
  "ds_read_b64_tr_b8 v[114:115], v91 offset:25600\n\t" \
  "v_mfma_f32_16x16x128_f8f6f4 v[148:151], v[120:127], a[232:239], v[148:151]\n\t" \
  "ds_read_b64_tr_b8 v[116:117], v99 offset:24576\n\t" \
  "s_add_u32 s55, s68, 0x1a0000\n\t" \
  "buffer_load_dwordx4 a[68:71], v4, s[12:15], s55 offen offset:1024 sc0 sc1\n\t" \
  "v_mfma_f32_16x16x128_f8f6f4 v[144:147], v[120:127], a[224:231], v[144:147]\n\t" \
  "ds_read_b64_tr_b8 v[118:119], v99 offset:25600\n\t" \
  "s_waitcnt lgkmcnt(4)\n\t" \
  "v_mfma_f32_16x16x128_f8f6f4 v[160:163], v[104:111], a[224:231], v[160:163]\n\t" \
  "ds_read_b64_tr_b8 v[120:121], v92 offset:24576\n\t" \
  "s_add_u32 s48, s68, 0x1a0000\n\t" \
  "buffer_load_dwordx4 a[72:75], v4, s[12:15], s48 offen offset:2048 sc0 sc1\n\t" \
  "v_mfma_f32_16x16x128_f8f6f4 v[164:167], v[104:111], a[232:239], v[164:167]\n\t" \
  "ds_read_b64_tr_b8 v[122:123], v92 offset:25600\n\t" \
  "v_mfma_f32_16x16x128_f8f6f4 v[168:171], v[104:111], a[240:247], v[168:171]\n\t" \
  "ds_read_b64_tr_b8 v[124:125], v100 offset:24576\n\t" \
  "s_add_u32 s49, s68, 0x1a0000\n\t" \
  "buffer_load_dwordx4 a[76:79], v4, s[12:15], s49 offen offset:3072 sc0 sc1\n\t" \
  "v_mfma_f32_16x16x128_f8f6f4 v[172:175], v[104:111], a[248:255], v[172:175]\n\t" \
  "ds_read_b64_tr_b8 v[126:127], v100 offset:25600\n\t" \
  "s_waitcnt lgkmcnt(4)\n\t" \
  "v_mfma_f32_16x16x128_f8f6f4 v[188:191], v[112:119], a[248:255], v[188:191]\n\t" \
  "ds_read_b64_tr_b8 v[104:105], v93 offset:24576\n\t" \
  "s_add_u32 s50, s68, 0x1a1000\n\t" \
  "buffer_load_dwordx4 a[80:83], v4, s[12:15], s50 offen offset:0 sc0 sc1\n\t" \
  "v_mfma_f32_16x16x128_f8f6f4 v[184:187], v[112:119], a[240:247], v[184:187]\n\t" \
  "ds_read_b64_tr_b8 v[106:107], v93 offset:25600\n\t" \
  "v_mfma_f32_16x16x128_f8f6f4 v[180:183], v[112:119], a[232:239], v[180:183]\n\t" \
  "ds_read_b64_tr_b8 v[108:109], v101 offset:24576\n\t" \
  "s_add_u32 s51, s68, 0x1a1000\n\t" \
  "buffer_load_dwordx4 a[84:87], v4, s[12:15], s51 offen offset:1024 sc0 sc1\n\t" \
  "v_mfma_f32_16x16x128_f8f6f4 v[176:179], v[112:119], a[224:231], v[176:179]\n\t" \
  "ds_read_b64_tr_b8 v[110:111], v101 offset:25600\n\t" \
  "s_waitcnt lgkmcnt(4)\n\t" \
  "v_mfma_f32_16x16x128_f8f6f4 v[192:195], v[120:127], a[224:231], v[192:195]\n\t" \
  "ds_read_b64_tr_b8 v[112:113], v94 offset:24576\n\t" \
  "s_add_u32 s52, s68, 0x1a1000\n\t" \
  "buffer_load_dwordx4 a[88:91], v4, s[12:15], s52 offen offset:2048 sc0 sc1\n\t" \
  "v_mfma_f32_16x16x128_f8f6f4 v[196:199], v[120:127], a[232:239], v[196:199]\n\t" \
  "ds_read_b64_tr_b8 v[114:115], v94 offset:25600\n\t" \
  "v_mfma_f32_16x16x128_f8f6f4 v[200:203], v[120:127], a[240:247], v[200:203]\n\t" \
  "ds_read_b64_tr_b8 v[116:117], v102 offset:24576\n\t" \
  "s_add_u32 s53, s68, 0x1a1000\n\t" \
  "buffer_load_dwordx4 a[92:95], v4, s[12:15], s53 offen offset:3072 sc0 sc1\n\t" \
  "v_mfma_f32_16x16x128_f8f6f4 v[204:207], v[120:127], a[248:255], v[204:207]\n\t" \
  "ds_read_b64_tr_b8 v[118:119], v102 offset:25600\n\t" \
  "s_waitcnt lgkmcnt(4)\n\t" \
  "v_mfma_f32_16x16x128_f8f6f4 v[220:223], v[104:111], a[248:255], v[220:223]\n\t" \
  "ds_read_b64_tr_b8 v[120:121], v95 offset:24576\n\t" \
  "s_add_u32 s54, s68, 0x1b0000\n\t" \
  "buffer_load_dwordx4 a[96:99], v4, s[12:15], s54 offen offset:0 sc0 sc1\n\t" \
  "v_mfma_f32_16x16x128_f8f6f4 v[216:219], v[104:111], a[240:247], v[216:219]\n\t" \
  "ds_read_b64_tr_b8 v[122:123], v95 offset:25600\n\t" \
  "v_mfma_f32_16x16x128_f8f6f4 v[212:215], v[104:111], a[232:239], v[212:215]\n\t" \
  "ds_read_b64_tr_b8 v[124:125], v103 offset:24576\n\t" \
  "s_add_u32 s55, s68, 0x1b0000\n\t" \
  "buffer_load_dwordx4 a[100:103], v4, s[12:15], s55 offen offset:1024 sc0 sc1\n\t" \
  "v_mfma_f32_16x16x128_f8f6f4 v[208:211], v[104:111], a[224:231], v[208:211]\n\t" \
  "ds_read_b64_tr_b8 v[126:127], v103 offset:25600\n\t" \
  "s_waitcnt vmcnt(46) lgkmcnt(0)\n\t" \
  "s_barrier\n\t" \
  "v_mfma_f32_16x16x128_f8f6f4 v[224:227], v[112:119], a[224:231], v[224:227]\n\t" \
  "ds_read_b64_tr_b8 v[104:105], v80 offset:0\n\t" \
  "s_add_u32 s48, s68, 0x1b0000\n\t" \
  "buffer_load_dwordx4 a[104:107], v4, s[12:15], s48 offen offset:2048 sc0 sc1\n\t" \
  "v_mfma_f32_16x16x128_f8f6f4 v[228:231], v[112:119], a[232:239], v[228:231]\n\t" \
  "ds_read_b64_tr_b8 v[106:107], v80 offset:1024\n\t" \
  "v_mfma_f32_16x16x128_f8f6f4 v[232:235], v[112:119], a[240:247], v[232:235]\n\t" \
  "ds_read_b64_tr_b8 v[108:109], v88 offset:32768\n\t" \
  "s_add_u32 s49, s68, 0x1b0000\n\t" \
  "buffer_load_dwordx4 a[108:111], v4, s[12:15], s49 offen offset:3072 sc0 sc1\n\t" \
  "v_mfma_f32_16x16x128_f8f6f4 v[236:239], v[112:119], a[248:255], v[236:239]\n\t" \
  "ds_read_b64_tr_b8 v[110:111], v88 offset:33792\n\t" \
  "v_mfma_f32_16x16x128_f8f6f4 v[252:255], v[120:127], a[248:255], v[252:255]\n\t" \
  "ds_read_b64_tr_b8 v[112:113], v81 offset:0\n\t" \
  "s_add_u32 s50, s68, 0x1b1000\n\t" \
  "buffer_load_dwordx4 a[112:115], v4, s[12:15], s50 offen offset:0 sc0 sc1\n\t" \
  "v_mfma_f32_16x16x128_f8f6f4 v[248:251], v[120:127], a[240:247], v[248:251]\n\t" \
  "ds_read_b64_tr_b8 v[114:115], v81 offset:1024\n\t" \
  "v_mfma_f32_16x16x128_f8f6f4 v[244:247], v[120:127], a[232:239], v[244:247]\n\t" \
  "ds_read_b64_tr_b8 v[116:117], v89 offset:32768\n\t" \
  "s_add_u32 s51, s68, 0x1b1000\n\t" \
  "buffer_load_dwordx4 a[116:119], v4, s[12:15], s51 offen offset:1024 sc0 sc1\n\t" \
  "v_mfma_f32_16x16x128_f8f6f4 v[240:243], v[120:127], a[224:231], v[240:243]\n\t" \
  "ds_read_b64_tr_b8 v[118:119], v89 offset:33792\n\t" \
  "s_waitcnt vmcnt(42) lgkmcnt(4)\n\t" \
  "v_mfma_f32_16x16x128_f8f6f4 v[128:131], v[104:111], a[0:7], v[128:131]\n\t" \
  "ds_read_b64_tr_b8 v[120:121], v82 offset:0\n\t" \
  "s_add_u32 s52, s68, 0x1b1000\n\t" \
  "buffer_load_dwordx4 a[120:123], v4, s[12:15], s52 offen offset:2048 sc0 sc1\n\t" \
  "v_mfma_f32_16x16x128_f8f6f4 v[132:135], v[104:111], a[8:15], v[132:135]\n\t" \
  "ds_read_b64_tr_b8 v[122:123], v82 offset:1024\n\t" \
  "v_mfma_f32_16x16x128_f8f6f4 v[136:139], v[104:111], a[16:23], v[136:139]\n\t" \
  "ds_read_b64_tr_b8 v[124:125], v90 offset:32768\n\t" \
  "s_add_u32 s53, s68, 0x1b1000\n\t" \
  "buffer_load_dwordx4 a[124:127], v4, s[12:15], s53 offen offset:3072 sc0 sc1\n\t" \
  "v_mfma_f32_16x16x128_f8f6f4 v[140:143], v[104:111], a[24:31], v[140:143]\n\t" \
  "ds_read_b64_tr_b8 v[126:127], v90 offset:33792\n\t" \
  "s_waitcnt lgkmcnt(4)\n\t" \
  "v_mfma_f32_16x16x128_f8f6f4 v[156:159], v[112:119], a[24:31], v[156:159]\n\t" \
  "ds_read_b64_tr_b8 v[104:105], v83 offset:0\n\t" \
  "s_add_u32 m0, s24, 0x26000\n\t" \
  "s_add_u32 s54, s46, 0x600000\n\t" \
  "buffer_load_dwordx4 v3, s[8:11], s54 offen lds\n\t" \
  "v_mfma_f32_16x16x128_f8f6f4 v[152:155], v[112:119], a[16:23], v[152:155]\n\t" \
  "ds_read_b64_tr_b8 v[106:107], v83 offset:1024\n\t" \
  "v_mfma_f32_16x16x128_f8f6f4 v[148:151], v[112:119], a[8:15], v[148:151]\n\t" \
  "ds_read_b64_tr_b8 v[108:109], v91 offset:32768\n\t" \
  "s_add_u32 m0, s24, 0x26400\n\t" \
  "s_add_u32 s55, s46, 0x620000\n\t" \
  "buffer_load_dwordx4 v3, s[8:11], s55 offen lds\n\t" \
  "v_mfma_f32_16x16x128_f8f6f4 v[144:147], v[112:119], a[0:7], v[144:147]\n\t" \
  "ds_read_b64_tr_b8 v[110:111], v91 offset:33792\n\t" \
  "s_waitcnt lgkmcnt(4)\n\t" \
  "v_mfma_f32_16x16x128_f8f6f4 v[160:163], v[120:127], a[0:7], v[160:163]\n\t" \
  "ds_read_b64_tr_b8 v[112:113], v84 offset:0\n\t" \
  "s_add_u32 s48, s68, 0x1c0000\n\t" \
  "buffer_load_dwordx4 a[128:131], v4, s[12:15], s48 offen offset:0 sc0 sc1\n\t" \
  "v_mfma_f32_16x16x128_f8f6f4 v[164:167], v[120:127], a[8:15], v[164:167]\n\t" \
  "ds_read_b64_tr_b8 v[114:115], v84 offset:1024\n\t" \
  "v_mfma_f32_16x16x128_f8f6f4 v[168:171], v[120:127], a[16:23], v[168:171]\n\t" \
  "ds_read_b64_tr_b8 v[116:117], v92 offset:32768\n\t" \
  "s_add_u32 s49, s68, 0x1c0000\n\t" \
  "buffer_load_dwordx4 a[132:135], v4, s[12:15], s49 offen offset:1024 sc0 sc1\n\t" \
  "v_mfma_f32_16x16x128_f8f6f4 v[172:175], v[120:127], a[24:31], v[172:175]\n\t" \
  "ds_read_b64_tr_b8 v[118:119], v92 offset:33792\n\t" \
  "s_waitcnt lgkmcnt(4)\n\t" \
  "v_mfma_f32_16x16x128_f8f6f4 v[188:191], v[104:111], a[24:31], v[188:191]\n\t" \
  "ds_read_b64_tr_b8 v[120:121], v85 offset:0\n\t" \
  "s_add_u32 s50, s68, 0x1c0000\n\t" \
  "buffer_load_dwordx4 a[136:139], v4, s[12:15], s50 offen offset:2048 sc0 sc1\n\t" \
  "v_mfma_f32_16x16x128_f8f6f4 v[184:187], v[104:111], a[16:23], v[184:187]\n\t" \
  "ds_read_b64_tr_b8 v[122:123], v85 offset:1024\n\t" \
  "v_mfma_f32_16x16x128_f8f6f4 v[180:183], v[104:111], a[8:15], v[180:183]\n\t" \
  "ds_read_b64_tr_b8 v[124:125], v93 offset:32768\n\t" \
  "s_add_u32 s51, s68, 0x1c0000\n\t" \
  "buffer_load_dwordx4 a[140:143], v4, s[12:15], s51 offen offset:3072 sc0 sc1\n\t" \
  "v_mfma_f32_16x16x128_f8f6f4 v[176:179], v[104:111], a[0:7], v[176:179]\n\t" \
  "ds_read_b64_tr_b8 v[126:127], v93 offset:33792\n\t" \
  "s_waitcnt lgkmcnt(4)\n\t" \
  "v_mfma_f32_16x16x128_f8f6f4 v[192:195], v[112:119], a[0:7], v[192:195]\n\t" \
  "ds_read_b64_tr_b8 v[104:105], v86 offset:0\n\t" \
  "s_add_u32 s52, s68, 0x1c1000\n\t" \
  "buffer_load_dwordx4 a[144:147], v4, s[12:15], s52 offen offset:0 sc0 sc1\n\t" \
  "v_mfma_f32_16x16x128_f8f6f4 v[196:199], v[112:119], a[8:15], v[196:199]\n\t" \
  "ds_read_b64_tr_b8 v[106:107], v86 offset:1024\n\t" \
  "v_mfma_f32_16x16x128_f8f6f4 v[200:203], v[112:119], a[16:23], v[200:203]\n\t" \
  "ds_read_b64_tr_b8 v[108:109], v94 offset:32768\n\t" \
  "s_add_u32 s53, s68, 0x1c1000\n\t" \
  "buffer_load_dwordx4 a[148:151], v4, s[12:15], s53 offen offset:1024 sc0 sc1\n\t" \
  "v_mfma_f32_16x16x128_f8f6f4 v[204:207], v[112:119], a[24:31], v[204:207]\n\t" \
  "ds_read_b64_tr_b8 v[110:111], v94 offset:33792\n\t" \
  "s_waitcnt lgkmcnt(4)\n\t" \
  "v_mfma_f32_16x16x128_f8f6f4 v[220:223], v[120:127], a[24:31], v[220:223]\n\t" \
  "ds_read_b64_tr_b8 v[112:113], v87 offset:0\n\t" \
  "s_add_u32 s54, s68, 0x1c1000\n\t" \
  "buffer_load_dwordx4 a[152:155], v4, s[12:15], s54 offen offset:2048 sc0 sc1\n\t" \
  "v_mfma_f32_16x16x128_f8f6f4 v[216:219], v[120:127], a[16:23], v[216:219]\n\t" \
  "ds_read_b64_tr_b8 v[114:115], v87 offset:1024\n\t" \
  "v_mfma_f32_16x16x128_f8f6f4 v[212:215], v[120:127], a[8:15], v[212:215]\n\t" \
  "ds_read_b64_tr_b8 v[116:117], v95 offset:32768\n\t" \
  "s_add_u32 s55, s68, 0x1c1000\n\t" \
  "buffer_load_dwordx4 a[156:159], v4, s[12:15], s55 offen offset:3072 sc0 sc1\n\t" \
  "v_mfma_f32_16x16x128_f8f6f4 v[208:211], v[120:127], a[0:7], v[208:211]\n\t" \
  "ds_read_b64_tr_b8 v[118:119], v95 offset:33792\n\t" \
  "s_waitcnt vmcnt(52) lgkmcnt(0)\n\t" \
  "s_barrier\n\t" \
  "v_mfma_f32_16x16x128_f8f6f4 v[224:227], v[104:111], a[0:7], v[224:227]\n\t" \
  "ds_read_b64_tr_b8 v[120:121], v80 offset:8192\n\t" \
  "s_add_u32 s48, s68, 0x1d0000\n\t" \
  "buffer_load_dwordx4 a[160:163], v4, s[12:15], s48 offen offset:0 sc0 sc1\n\t" \
  "v_mfma_f32_16x16x128_f8f6f4 v[228:231], v[104:111], a[8:15], v[228:231]\n\t" \
  "ds_read_b64_tr_b8 v[122:123], v80 offset:9216\n\t" \
  "v_mfma_f32_16x16x128_f8f6f4 v[232:235], v[104:111], a[16:23], v[232:235]\n\t" \
  "ds_read_b64_tr_b8 v[124:125], v88 offset:40960\n\t" \
  "s_add_u32 s49, s68, 0x1d0000\n\t" \
  "buffer_load_dwordx4 a[164:167], v4, s[12:15], s49 offen offset:1024 sc0 sc1\n\t" \
  "v_mfma_f32_16x16x128_f8f6f4 v[236:239], v[104:111], a[24:31], v[236:239]\n\t" \
  "ds_read_b64_tr_b8 v[126:127], v88 offset:41984\n\t" \
  "v_mfma_f32_16x16x128_f8f6f4 v[252:255], v[112:119], a[24:31], v[252:255]\n\t" \
  "ds_read_b64_tr_b8 v[104:105], v81 offset:8192\n\t" \
  "s_add_u32 s50, s68, 0x1d0000\n\t" \
  "buffer_load_dwordx4 a[168:171], v4, s[12:15], s50 offen offset:2048 sc0 sc1\n\t" \
  "v_mfma_f32_16x16x128_f8f6f4 v[248:251], v[112:119], a[16:23], v[248:251]\n\t" \
  "ds_read_b64_tr_b8 v[106:107], v81 offset:9216\n\t" \
  "v_mfma_f32_16x16x128_f8f6f4 v[244:247], v[112:119], a[8:15], v[244:247]\n\t" \
  "ds_read_b64_tr_b8 v[108:109], v89 offset:40960\n\t" \
  "s_add_u32 s51, s68, 0x1d0000\n\t" \
  "buffer_load_dwordx4 a[172:175], v4, s[12:15], s51 offen offset:3072 sc0 sc1\n\t" \
  "v_mfma_f32_16x16x128_f8f6f4 v[240:243], v[112:119], a[0:7], v[240:243]\n\t" \
  "ds_read_b64_tr_b8 v[110:111], v89 offset:41984\n\t" \
  "s_waitcnt vmcnt(48) lgkmcnt(4)\n\t" \
  "v_mfma_f32_16x16x128_f8f6f4 v[128:131], v[120:127], a[32:39], v[128:131]\n\t" \
  "ds_read_b64_tr_b8 v[112:113], v82 offset:8192\n\t" \
  "s_add_u32 s52, s68, 0x1d1000\n\t" \
  "buffer_load_dwordx4 a[176:179], v4, s[12:15], s52 offen offset:0 sc0 sc1\n\t" \
  "v_mfma_f32_16x16x128_f8f6f4 v[132:135], v[120:127], a[40:47], v[132:135]\n\t" \
  "ds_read_b64_tr_b8 v[114:115], v82 offset:9216\n\t" \
  "v_mfma_f32_16x16x128_f8f6f4 v[136:139], v[120:127], a[48:55], v[136:139]\n\t" \
  "ds_read_b64_tr_b8 v[116:117], v90 offset:40960\n\t" \
  "s_add_u32 s53, s68, 0x1d1000\n\t" \
  "buffer_load_dwordx4 a[180:183], v4, s[12:15], s53 offen offset:1024 sc0 sc1\n\t" \
  "v_mfma_f32_16x16x128_f8f6f4 v[140:143], v[120:127], a[56:63], v[140:143]\n\t" \
  "ds_read_b64_tr_b8 v[118:119], v90 offset:41984\n\t" \
  "s_waitcnt lgkmcnt(4)\n\t" \
  "v_mfma_f32_16x16x128_f8f6f4 v[156:159], v[104:111], a[56:63], v[156:159]\n\t" \
  "ds_read_b64_tr_b8 v[120:121], v83 offset:8192\n\t" \
  "s_add_u32 s54, s68, 0x1d1000\n\t" \
  "buffer_load_dwordx4 a[184:187], v4, s[12:15], s54 offen offset:2048 sc0 sc1\n\t" \
  "v_mfma_f32_16x16x128_f8f6f4 v[152:155], v[104:111], a[48:55], v[152:155]\n\t" \
  "ds_read_b64_tr_b8 v[122:123], v83 offset:9216\n\t" \
  "v_mfma_f32_16x16x128_f8f6f4 v[148:151], v[104:111], a[40:47], v[148:151]\n\t" \
  "ds_read_b64_tr_b8 v[124:125], v91 offset:40960\n\t" \
  "s_add_u32 s55, s68, 0x1d1000\n\t" \
  "buffer_load_dwordx4 a[188:191], v4, s[12:15], s55 offen offset:3072 sc0 sc1\n\t" \
  "v_mfma_f32_16x16x128_f8f6f4 v[144:147], v[104:111], a[32:39], v[144:147]\n\t" \
  "ds_read_b64_tr_b8 v[126:127], v91 offset:41984\n\t" \
  "s_waitcnt lgkmcnt(4)\n\t" \
  "v_mfma_f32_16x16x128_f8f6f4 v[160:163], v[112:119], a[32:39], v[160:163]\n\t" \
  "ds_read_b64_tr_b8 v[104:105], v84 offset:8192\n\t" \
  "s_add_u32 s48, s68, 0x1e0000\n\t" \
  "buffer_load_dwordx4 a[192:195], v4, s[12:15], s48 offen offset:0 sc0 sc1\n\t" \
  "v_mfma_f32_16x16x128_f8f6f4 v[164:167], v[112:119], a[40:47], v[164:167]\n\t" \
  "ds_read_b64_tr_b8 v[106:107], v84 offset:9216\n\t" \
  "v_mfma_f32_16x16x128_f8f6f4 v[168:171], v[112:119], a[48:55], v[168:171]\n\t" \
  "ds_read_b64_tr_b8 v[108:109], v92 offset:40960\n\t" \
  "s_add_u32 s49, s68, 0x1e0000\n\t" \
  "buffer_load_dwordx4 a[196:199], v4, s[12:15], s49 offen offset:1024 sc0 sc1\n\t" \
  "v_mfma_f32_16x16x128_f8f6f4 v[172:175], v[112:119], a[56:63], v[172:175]\n\t" \
  "ds_read_b64_tr_b8 v[110:111], v92 offset:41984\n\t" \
  "s_waitcnt lgkmcnt(4)\n\t" \
  "v_mfma_f32_16x16x128_f8f6f4 v[188:191], v[120:127], a[56:63], v[188:191]\n\t" \
  "ds_read_b64_tr_b8 v[112:113], v85 offset:8192\n\t" \
  "s_add_u32 s50, s68, 0x1e0000\n\t" \
  "buffer_load_dwordx4 a[200:203], v4, s[12:15], s50 offen offset:2048 sc0 sc1\n\t" \
  "v_mfma_f32_16x16x128_f8f6f4 v[184:187], v[120:127], a[48:55], v[184:187]\n\t" \
  "ds_read_b64_tr_b8 v[114:115], v85 offset:9216\n\t" \
  "v_mfma_f32_16x16x128_f8f6f4 v[180:183], v[120:127], a[40:47], v[180:183]\n\t" \
  "ds_read_b64_tr_b8 v[116:117], v93 offset:40960\n\t" \
  "s_add_u32 s51, s68, 0x1e0000\n\t" \
  "buffer_load_dwordx4 a[204:207], v4, s[12:15], s51 offen offset:3072 sc0 sc1\n\t" \
  "v_mfma_f32_16x16x128_f8f6f4 v[176:179], v[120:127], a[32:39], v[176:179]\n\t" \
  "ds_read_b64_tr_b8 v[118:119], v93 offset:41984\n\t" \
  "s_waitcnt lgkmcnt(4)\n\t" \
  "v_mfma_f32_16x16x128_f8f6f4 v[192:195], v[104:111], a[32:39], v[192:195]\n\t" \
  "ds_read_b64_tr_b8 v[120:121], v86 offset:8192\n\t" \
  "s_add_u32 s52, s68, 0x1e1000\n\t" \
  "buffer_load_dwordx4 a[208:211], v4, s[12:15], s52 offen offset:0 sc0 sc1\n\t" \
  "v_mfma_f32_16x16x128_f8f6f4 v[196:199], v[104:111], a[40:47], v[196:199]\n\t" \
  "ds_read_b64_tr_b8 v[122:123], v86 offset:9216\n\t" \
  "v_mfma_f32_16x16x128_f8f6f4 v[200:203], v[104:111], a[48:55], v[200:203]\n\t" \
  "ds_read_b64_tr_b8 v[124:125], v94 offset:40960\n\t" \
  "s_add_u32 s53, s68, 0x1e1000\n\t" \
  "buffer_load_dwordx4 a[212:215], v4, s[12:15], s53 offen offset:1024 sc0 sc1\n\t" \
  "v_mfma_f32_16x16x128_f8f6f4 v[204:207], v[104:111], a[56:63], v[204:207]\n\t" \
  "ds_read_b64_tr_b8 v[126:127], v94 offset:41984\n\t" \
  "s_waitcnt lgkmcnt(4)\n\t" \
  "v_mfma_f32_16x16x128_f8f6f4 v[220:223], v[112:119], a[56:63], v[220:223]\n\t" \
  "ds_read_b64_tr_b8 v[104:105], v87 offset:8192\n\t" \
  "s_add_u32 s54, s68, 0x1e1000\n\t" \
  "buffer_load_dwordx4 a[216:219], v4, s[12:15], s54 offen offset:2048 sc0 sc1\n\t" \
  "v_mfma_f32_16x16x128_f8f6f4 v[216:219], v[112:119], a[48:55], v[216:219]\n\t" \
  "ds_read_b64_tr_b8 v[106:107], v87 offset:9216\n\t" \
  "v_mfma_f32_16x16x128_f8f6f4 v[212:215], v[112:119], a[40:47], v[212:215]\n\t" \
  "ds_read_b64_tr_b8 v[108:109], v95 offset:40960\n\t" \
  "s_add_u32 s55, s68, 0x1e1000\n\t" \
  "buffer_load_dwordx4 a[220:223], v4, s[12:15], s55 offen offset:3072 sc0 sc1\n\t" \
  "v_mfma_f32_16x16x128_f8f6f4 v[208:211], v[112:119], a[32:39], v[208:211]\n\t" \
  "ds_read_b64_tr_b8 v[110:111], v95 offset:41984\n\t" \
  "s_waitcnt vmcnt(58) lgkmcnt(0)\n\t" \
  "s_barrier\n\t" \
  "v_mfma_f32_16x16x128_f8f6f4 v[224:227], v[120:127], a[32:39], v[224:227]\n\t" \
  "ds_read_b64_tr_b8 v[112:113], v80 offset:16384\n\t" \
  "s_add_u32 s48, s68, 0x1f0000\n\t" \
  "buffer_load_dwordx4 a[224:227], v4, s[12:15], s48 offen offset:0 sc0 sc1\n\t" \
  "v_mfma_f32_16x16x128_f8f6f4 v[228:231], v[120:127], a[40:47], v[228:231]\n\t" \
  "ds_read_b64_tr_b8 v[114:115], v80 offset:17408\n\t" \
  "v_mfma_f32_16x16x128_f8f6f4 v[232:235], v[120:127], a[48:55], v[232:235]\n\t" \
  "ds_read_b64_tr_b8 v[116:117], v88 offset:49152\n\t" \
  "s_add_u32 s49, s68, 0x1f0000\n\t" \
  "buffer_load_dwordx4 a[228:231], v4, s[12:15], s49 offen offset:1024 sc0 sc1\n\t" \
  "v_mfma_f32_16x16x128_f8f6f4 v[236:239], v[120:127], a[56:63], v[236:239]\n\t" \
  "ds_read_b64_tr_b8 v[118:119], v88 offset:50176\n\t" \
  "v_mfma_f32_16x16x128_f8f6f4 v[252:255], v[104:111], a[56:63], v[252:255]\n\t" \
  "ds_read_b64_tr_b8 v[120:121], v81 offset:16384\n\t" \
  "s_add_u32 s50, s68, 0x1f0000\n\t" \
  "buffer_load_dwordx4 a[232:235], v4, s[12:15], s50 offen offset:2048 sc0 sc1\n\t" \
  "v_mfma_f32_16x16x128_f8f6f4 v[248:251], v[104:111], a[48:55], v[248:251]\n\t" \
  "ds_read_b64_tr_b8 v[122:123], v81 offset:17408\n\t" \
  "v_mfma_f32_16x16x128_f8f6f4 v[244:247], v[104:111], a[40:47], v[244:247]\n\t" \
  "ds_read_b64_tr_b8 v[124:125], v89 offset:49152\n\t" \
  "s_add_u32 s51, s68, 0x1f0000\n\t" \
  "buffer_load_dwordx4 a[236:239], v4, s[12:15], s51 offen offset:3072 sc0 sc1\n\t" \
  "v_mfma_f32_16x16x128_f8f6f4 v[240:243], v[104:111], a[32:39], v[240:243]\n\t" \
  "ds_read_b64_tr_b8 v[126:127], v89 offset:50176\n\t" \
  "s_waitcnt vmcnt(38) lgkmcnt(4)\n\t" \
  "v_mfma_f32_16x16x128_f8f6f4 v[128:131], v[112:119], a[64:71], v[128:131]\n\t" \
  "ds_read_b64_tr_b8 v[104:105], v82 offset:16384\n\t" \
  "s_add_u32 s52, s68, 0x1f1000\n\t" \
  "buffer_load_dwordx4 a[240:243], v4, s[12:15], s52 offen offset:0 sc0 sc1\n\t" \
  "v_mfma_f32_16x16x128_f8f6f4 v[132:135], v[112:119], a[72:79], v[132:135]\n\t" \
  "ds_read_b64_tr_b8 v[106:107], v82 offset:17408\n\t" \
  "v_mfma_f32_16x16x128_f8f6f4 v[136:139], v[112:119], a[80:87], v[136:139]\n\t" \
  "ds_read_b64_tr_b8 v[108:109], v90 offset:49152\n\t" \
  "s_add_u32 s53, s68, 0x1f1000\n\t" \
  "buffer_load_dwordx4 a[244:247], v4, s[12:15], s53 offen offset:1024 sc0 sc1\n\t" \
  "v_mfma_f32_16x16x128_f8f6f4 v[140:143], v[112:119], a[88:95], v[140:143]\n\t" \
  "ds_read_b64_tr_b8 v[110:111], v90 offset:50176\n\t" \
  "s_waitcnt lgkmcnt(4)\n\t" \
  "v_mfma_f32_16x16x128_f8f6f4 v[156:159], v[120:127], a[88:95], v[156:159]\n\t" \
  "ds_read_b64_tr_b8 v[112:113], v83 offset:16384\n\t" \
  "s_add_u32 s54, s68, 0x1f1000\n\t" \
  "buffer_load_dwordx4 a[248:251], v4, s[12:15], s54 offen offset:2048 sc0 sc1\n\t" \
  "v_mfma_f32_16x16x128_f8f6f4 v[152:155], v[120:127], a[80:87], v[152:155]\n\t" \
  "ds_read_b64_tr_b8 v[114:115], v83 offset:17408\n\t" \
  "v_mfma_f32_16x16x128_f8f6f4 v[148:151], v[120:127], a[72:79], v[148:151]\n\t" \
  "ds_read_b64_tr_b8 v[116:117], v91 offset:49152\n\t" \
  "s_add_u32 s55, s68, 0x1f1000\n\t" \
  "buffer_load_dwordx4 a[252:255], v4, s[12:15], s55 offen offset:3072 sc0 sc1\n\t" \
  "v_mfma_f32_16x16x128_f8f6f4 v[144:147], v[120:127], a[64:71], v[144:147]\n\t" \
  "ds_read_b64_tr_b8 v[118:119], v91 offset:50176\n\t" \
  "s_waitcnt lgkmcnt(4)\n\t" \
  "v_mfma_f32_16x16x128_f8f6f4 v[160:163], v[104:111], a[64:71], v[160:163]\n\t" \
  "ds_read_b64_tr_b8 v[120:121], v84 offset:16384\n\t" \
  "v_mfma_f32_16x16x128_f8f6f4 v[164:167], v[104:111], a[72:79], v[164:167]\n\t" \
  "ds_read_b64_tr_b8 v[122:123], v84 offset:17408\n\t" \
  "v_mfma_f32_16x16x128_f8f6f4 v[168:171], v[104:111], a[80:87], v[168:171]\n\t" \
  "ds_read_b64_tr_b8 v[124:125], v92 offset:49152\n\t" \
  "v_mfma_f32_16x16x128_f8f6f4 v[172:175], v[104:111], a[88:95], v[172:175]\n\t" \
  "ds_read_b64_tr_b8 v[126:127], v92 offset:50176\n\t" \
  "s_waitcnt lgkmcnt(4)\n\t" \
  "v_mfma_f32_16x16x128_f8f6f4 v[188:191], v[112:119], a[88:95], v[188:191]\n\t" \
  "ds_read_b64_tr_b8 v[104:105], v85 offset:16384\n\t" \
  "v_mfma_f32_16x16x128_f8f6f4 v[184:187], v[112:119], a[80:87], v[184:187]\n\t" \
  "ds_read_b64_tr_b8 v[106:107], v85 offset:17408\n\t" \
  "v_mfma_f32_16x16x128_f8f6f4 v[180:183], v[112:119], a[72:79], v[180:183]\n\t" \
  "ds_read_b64_tr_b8 v[108:109], v93 offset:49152\n\t" \
  "v_mfma_f32_16x16x128_f8f6f4 v[176:179], v[112:119], a[64:71], v[176:179]\n\t" \
  "ds_read_b64_tr_b8 v[110:111], v93 offset:50176\n\t" \
  "s_waitcnt lgkmcnt(4)\n\t" \
  "v_mfma_f32_16x16x128_f8f6f4 v[192:195], v[120:127], a[64:71], v[192:195]\n\t" \
  "ds_read_b64_tr_b8 v[112:113], v86 offset:16384\n\t" \
  "v_mfma_f32_16x16x128_f8f6f4 v[196:199], v[120:127], a[72:79], v[196:199]\n\t" \
  "ds_read_b64_tr_b8 v[114:115], v86 offset:17408\n\t" \
  "v_mfma_f32_16x16x128_f8f6f4 v[200:203], v[120:127], a[80:87], v[200:203]\n\t" \
  "ds_read_b64_tr_b8 v[116:117], v94 offset:49152\n\t" \
  "v_mfma_f32_16x16x128_f8f6f4 v[204:207], v[120:127], a[88:95], v[204:207]\n\t" \
  "ds_read_b64_tr_b8 v[118:119], v94 offset:50176\n\t" \
  "s_waitcnt lgkmcnt(4)\n\t" \
  "v_mfma_f32_16x16x128_f8f6f4 v[220:223], v[104:111], a[88:95], v[220:223]\n\t" \
  "ds_read_b64_tr_b8 v[120:121], v87 offset:16384\n\t" \
  "v_mfma_f32_16x16x128_f8f6f4 v[216:219], v[104:111], a[80:87], v[216:219]\n\t" \
  "ds_read_b64_tr_b8 v[122:123], v87 offset:17408\n\t" \
  "v_mfma_f32_16x16x128_f8f6f4 v[212:215], v[104:111], a[72:79], v[212:215]\n\t" \
  "ds_read_b64_tr_b8 v[124:125], v95 offset:49152\n\t" \
  "v_mfma_f32_16x16x128_f8f6f4 v[208:211], v[104:111], a[64:71], v[208:211]\n\t" \
  "ds_read_b64_tr_b8 v[126:127], v95 offset:50176\n\t" \
  "s_waitcnt vmcnt(32) lgkmcnt(0)\n\t" \
  "s_barrier\n\t" \
  "v_mfma_f32_16x16x128_f8f6f4 v[224:227], v[112:119], a[64:71], v[224:227]\n\t" \
  "ds_read_b64_tr_b8 v[104:105], v80 offset:24576\n\t" \
  "v_mov_b32 v66, s66\n\t" \
  "v_mul_f32 v66, s67, v66\n\t" \
  "v_or_b32 v66, s63, v66\n\t" \
  "v_mov_b32 v67, v66\n\t" \
  "v_mfma_f32_16x16x128_f8f6f4 v[228:231], v[112:119], a[72:79], v[228:231]\n\t" \
  "ds_read_b64_tr_b8 v[106:107], v80 offset:25600\n\t" \
  "v_mfma_f32_16x16x128_f8f6f4 v[232:235], v[112:119], a[80:87], v[232:235]\n\t" \
  "ds_read_b64_tr_b8 v[108:109], v88 offset:57344\n\t" \
  "v_mfma_f32_16x16x128_f8f6f4 v[236:239], v[112:119], a[88:95], v[236:239]\n\t" \
  "ds_read_b64_tr_b8 v[110:111], v88 offset:58368\n\t" \
  "v_mfma_f32_16x16x128_f8f6f4 v[252:255], v[120:127], a[88:95], v[252:255]\n\t" \
  "ds_read_b64_tr_b8 v[112:113], v80 offset:32768\n\t" \
  "v_mfma_f32_16x16x128_f8f6f4 v[248:251], v[120:127], a[80:87], v[248:251]\n\t" \
  "ds_read_b64_tr_b8 v[114:115], v80 offset:33792\n\t" \
  "v_mfma_f32_16x16x128_f8f6f4 v[244:247], v[120:127], a[72:79], v[244:247]\n\t" \
  "ds_read_b64_tr_b8 v[116:117], v96 offset:0\n\t" \
  "v_mfma_f32_16x16x128_f8f6f4 v[240:243], v[120:127], a[64:71], v[240:243]\n\t" \
  "ds_read_b64_tr_b8 v[118:119], v96 offset:1024\n\t" \
  "s_waitcnt lgkmcnt(4)\n\t" \
  "v_mfma_f32_16x16x128_f8f6f4 v[128:131], v[104:111], a[96:103], v[128:131]\n\t" \
  "ds_read_b64_tr_b8 v[120:121], v80 offset:40960\n\t" \
  "v_mfma_f32_16x16x128_f8f6f4 v[132:135], v[104:111], a[104:111], v[132:135]\n\t" \
  "ds_read_b64_tr_b8 v[122:123], v80 offset:41984\n\t" \
  "v_mfma_f32_16x16x128_f8f6f4 v[136:139], v[104:111], a[112:119], v[136:139]\n\t" \
  "ds_read_b64_tr_b8 v[124:125], v96 offset:8192\n\t" \
  "v_mfma_f32_16x16x128_f8f6f4 v[140:143], v[104:111], a[120:127], v[140:143]\n\t" \
  "ds_read_b64_tr_b8 v[126:127], v96 offset:9216\n\t" \
  "s_waitcnt vmcnt(24) lgkmcnt(4)\n\t" \
  "v_mfma_f32_16x16x128_f8f6f4 v[140:143], v[112:119], a[152:159], v[140:143]\n\t" \
  "ds_read_b64_tr_b8 v[104:105], v80 offset:49152\n\t" \
  "v_mfma_f32_16x16x128_f8f6f4 v[136:139], v[112:119], a[144:151], v[136:139]\n\t" \
  "ds_read_b64_tr_b8 v[106:107], v80 offset:50176\n\t" \
  "v_mfma_f32_16x16x128_f8f6f4 v[132:135], v[112:119], a[136:143], v[132:135]\n\t" \
  "ds_read_b64_tr_b8 v[108:109], v96 offset:16384\n\t" \
  "v_mfma_f32_16x16x128_f8f6f4 v[128:131], v[112:119], a[128:135], v[128:131]\n\t" \
  "ds_read_b64_tr_b8 v[110:111], v96 offset:17408\n\t" \
  "s_waitcnt vmcnt(16) lgkmcnt(4)\n\t" \
  "v_mfma_f32_16x16x128_f8f6f4 v[128:131], v[120:127], a[160:167], v[128:131]\n\t" \
  "ds_read_b64_tr_b8 v[112:113], v80 offset:57344\n\t" \
  "v_mfma_f32_16x16x128_f8f6f4 v[132:135], v[120:127], a[168:175], v[132:135]\n\t" \
  "ds_read_b64_tr_b8 v[114:115], v80 offset:58368\n\t" \
  "v_mfma_f32_16x16x128_f8f6f4 v[136:139], v[120:127], a[176:183], v[136:139]\n\t" \
  "ds_read_b64_tr_b8 v[116:117], v96 offset:24576\n\t" \
  "v_mfma_f32_16x16x128_f8f6f4 v[140:143], v[120:127], a[184:191], v[140:143]\n\t" \
  "ds_read_b64_tr_b8 v[118:119], v96 offset:25600\n\t" \
  "s_waitcnt vmcnt(8) lgkmcnt(4)\n\t" \
  "v_mfma_f32_16x16x128_f8f6f4 v[140:143], v[104:111], a[216:223], v[140:143]\n\t" \
  "ds_read_b64_tr_b8 v[120:121], v81 offset:24576\n\t" \
  "v_mfma_f32_16x16x128_f8f6f4 v[136:139], v[104:111], a[208:215], v[136:139]\n\t" \
  "ds_read_b64_tr_b8 v[122:123], v81 offset:25600\n\t" \
  "v_mfma_f32_16x16x128_f8f6f4 v[132:135], v[104:111], a[200:207], v[132:135]\n\t" \
  "ds_read_b64_tr_b8 v[124:125], v89 offset:57344\n\t" \
  "v_mfma_f32_16x16x128_f8f6f4 v[128:131], v[104:111], a[192:199], v[128:131]\n\t" \
  "ds_read_b64_tr_b8 v[126:127], v89 offset:58368\n\t" \
  "s_waitcnt vmcnt(0) lgkmcnt(4)\n\t" \
  "v_mfma_f32_16x16x128_f8f6f4 v[128:131], v[112:119], a[224:231], v[128:131]\n\t" \
  "ds_read_b64_tr_b8 v[104:105], v81 offset:32768\n\t" \
  "v_mfma_f32_16x16x128_f8f6f4 v[132:135], v[112:119], a[232:239], v[132:135]\n\t" \
  "ds_read_b64_tr_b8 v[106:107], v81 offset:33792\n\t" \
  "v_mfma_f32_16x16x128_f8f6f4 v[136:139], v[112:119], a[240:247], v[136:139]\n\t" \
  "ds_read_b64_tr_b8 v[108:109], v97 offset:0\n\t" \
  "v_mfma_f32_16x16x128_f8f6f4 v[140:143], v[112:119], a[248:255], v[140:143]\n\t" \
  "ds_read_b64_tr_b8 v[110:111], v97 offset:1024\n\t" \
  "s_waitcnt lgkmcnt(4)\n\t" \
  "v_mfma_f32_16x16x128_f8f6f4 v[156:159], v[120:127], a[120:127], v[156:159]\n\t" \
  "ds_read_b64_tr_b8 v[112:113], v81 offset:40960\n\t" \
  "v_mfma_f32_16x16x128_f8f6f4 v[152:155], v[120:127], a[112:119], v[152:155]\n\t" \
  "ds_read_b64_tr_b8 v[114:115], v81 offset:41984\n\t" \
  "v_mfma_f32_16x16x128_f8f6f4 v[148:151], v[120:127], a[104:111], v[148:151]\n\t" \
  "ds_read_b64_tr_b8 v[116:117], v97 offset:8192\n\t" \
  "v_mfma_f32_16x16x128_f8f6f4 v[144:147], v[120:127], a[96:103], v[144:147]\n\t" \
  "ds_read_b64_tr_b8 v[118:119], v97 offset:9216\n\t" \
  "s_waitcnt lgkmcnt(4)\n\t" \
  "v_mfma_f32_16x16x128_f8f6f4 v[144:147], v[104:111], a[128:135], v[144:147]\n\t" \
  "ds_read_b64_tr_b8 v[120:121], v81 offset:49152\n\t" \
  "v_pk_mul_f32 v[128:129], v[128:129], v[66:67]\n\t" \
  "v_pk_mul_f32 v[130:131], v[130:131], v[66:67]\n\t" \
  "s_mov_b32 s48, 0x0\n\t" \
  "buffer_store_dwordx4 v[128:131], v13, s[16:19], s48 offen offset:0 sc1\n\t" \
  "v_mfma_f32_16x16x128_f8f6f4 v[148:151], v[104:111], a[136:143], v[148:151]\n\t" \
  "ds_read_b64_tr_b8 v[122:123], v81 offset:50176\n\t" \
  "v_mfma_f32_16x16x128_f8f6f4 v[152:155], v[104:111], a[144:151], v[152:155]\n\t" \
  "ds_read_b64_tr_b8 v[124:125], v97 offset:16384\n\t" \
  "v_mfma_f32_16x16x128_f8f6f4 v[156:159], v[104:111], a[152:159], v[156:159]\n\t" \
  "ds_read_b64_tr_b8 v[126:127], v97 offset:17408\n\t" \
  "s_waitcnt lgkmcnt(4)\n\t" \
  "v_mfma_f32_16x16x128_f8f6f4 v[156:159], v[112:119], a[184:191], v[156:159]\n\t" \
  "ds_read_b64_tr_b8 v[104:105], v81 offset:57344\n\t" \
  "v_pk_mul_f32 v[132:133], v[132:133], v[66:67]\n\t" \
  "v_pk_mul_f32 v[134:135], v[134:135], v[66:67]\n\t" \
  "s_mov_b32 s49, 0x100000\n\t" \
  "buffer_store_dwordx4 v[132:135], v13, s[16:19], s49 offen offset:0 sc1\n\t" \
  "v_mfma_f32_16x16x128_f8f6f4 v[152:155], v[112:119], a[176:183], v[152:155]\n\t" \
  "ds_read_b64_tr_b8 v[106:107], v81 offset:58368\n\t" \
  "v_mfma_f32_16x16x128_f8f6f4 v[148:151], v[112:119], a[168:175], v[148:151]\n\t" \
  "ds_read_b64_tr_b8 v[108:109], v97 offset:24576\n\t" \
  "v_mfma_f32_16x16x128_f8f6f4 v[144:147], v[112:119], a[160:167], v[144:147]\n\t" \
  "ds_read_b64_tr_b8 v[110:111], v97 offset:25600\n\t" \
  "s_waitcnt lgkmcnt(4)\n\t" \
  "v_mfma_f32_16x16x128_f8f6f4 v[144:147], v[120:127], a[192:199], v[144:147]\n\t" \
  "ds_read_b64_tr_b8 v[112:113], v82 offset:24576\n\t" \
  "v_pk_mul_f32 v[136:137], v[136:137], v[66:67]\n\t" \
  "v_pk_mul_f32 v[138:139], v[138:139], v[66:67]\n\t" \
  "s_mov_b32 s50, 0x200000\n\t" \
  "buffer_store_dwordx4 v[136:139], v13, s[16:19], s50 offen offset:0 sc1\n\t" \
  "v_mfma_f32_16x16x128_f8f6f4 v[148:151], v[120:127], a[200:207], v[148:151]\n\t" \
  "ds_read_b64_tr_b8 v[114:115], v82 offset:25600\n\t" \
  "v_mfma_f32_16x16x128_f8f6f4 v[152:155], v[120:127], a[208:215], v[152:155]\n\t" \
  "ds_read_b64_tr_b8 v[116:117], v90 offset:57344\n\t" \
  "v_mfma_f32_16x16x128_f8f6f4 v[156:159], v[120:127], a[216:223], v[156:159]\n\t" \
  "ds_read_b64_tr_b8 v[118:119], v90 offset:58368\n\t" \
  "s_waitcnt lgkmcnt(4)\n\t" \
  "v_mfma_f32_16x16x128_f8f6f4 v[156:159], v[104:111], a[248:255], v[156:159]\n\t" \
  "ds_read_b64_tr_b8 v[120:121], v82 offset:32768\n\t" \
  "v_pk_mul_f32 v[140:141], v[140:141], v[66:67]\n\t" \
  "v_pk_mul_f32 v[142:143], v[142:143], v[66:67]\n\t" \
  "s_mov_b32 s51, 0x300000\n\t" \
  "buffer_store_dwordx4 v[140:143], v13, s[16:19], s51 offen offset:0 sc1\n\t" \
  "v_mfma_f32_16x16x128_f8f6f4 v[152:155], v[104:111], a[240:247], v[152:155]\n\t" \
  "ds_read_b64_tr_b8 v[122:123], v82 offset:33792\n\t" \
  "v_mfma_f32_16x16x128_f8f6f4 v[148:151], v[104:111], a[232:239], v[148:151]\n\t" \
  "ds_read_b64_tr_b8 v[124:125], v98 offset:0\n\t" \
  "v_mfma_f32_16x16x128_f8f6f4 v[144:147], v[104:111], a[224:231], v[144:147]\n\t" \
  "ds_read_b64_tr_b8 v[126:127], v98 offset:1024\n\t" \
  "s_waitcnt lgkmcnt(4)\n\t" \
  "v_mfma_f32_16x16x128_f8f6f4 v[160:163], v[112:119], a[96:103], v[160:163]\n\t" \
  "ds_read_b64_tr_b8 v[104:105], v82 offset:40960\n\t" \
  "v_mfma_f32_16x16x128_f8f6f4 v[164:167], v[112:119], a[104:111], v[164:167]\n\t" \
  "ds_read_b64_tr_b8 v[106:107], v82 offset:41984\n\t" \
  "v_mfma_f32_16x16x128_f8f6f4 v[168:171], v[112:119], a[112:119], v[168:171]\n\t" \
  "ds_read_b64_tr_b8 v[108:109], v98 offset:8192\n\t" \
  "v_mfma_f32_16x16x128_f8f6f4 v[172:175], v[112:119], a[120:127], v[172:175]\n\t" \
  "ds_read_b64_tr_b8 v[110:111], v98 offset:9216\n\t" \
  "s_waitcnt lgkmcnt(4)\n\t" \
  "v_mfma_f32_16x16x128_f8f6f4 v[172:175], v[120:127], a[152:159], v[172:175]\n\t" \
  "ds_read_b64_tr_b8 v[112:113], v82 offset:49152\n\t" \
  "v_pk_mul_f32 v[144:145], v[144:145], v[66:67]\n\t" \
  "v_pk_mul_f32 v[146:147], v[146:147], v[66:67]\n\t" \
  "s_mov_b32 s52, 0x0\n\t" \
  "buffer_store_dwordx4 v[144:147], v13, s[16:19], s52 offen offset:64 sc1\n\t" \
  "v_mfma_f32_16x16x128_f8f6f4 v[168:171], v[120:127], a[144:151], v[168:171]\n\t" \
  "ds_read_b64_tr_b8 v[114:115], v82 offset:50176\n\t" \
  "v_mfma_f32_16x16x128_f8f6f4 v[164:167], v[120:127], a[136:143], v[164:167]\n\t" \
  "ds_read_b64_tr_b8 v[116:117], v98 offset:16384\n\t" \
  "v_mfma_f32_16x16x128_f8f6f4 v[160:163], v[120:127], a[128:135], v[160:163]\n\t" \
  "ds_read_b64_tr_b8 v[118:119], v98 offset:17408\n\t" \
  "s_waitcnt lgkmcnt(4)\n\t" \
  "v_mfma_f32_16x16x128_f8f6f4 v[160:163], v[104:111], a[160:167], v[160:163]\n\t" \
  "ds_read_b64_tr_b8 v[120:121], v82 offset:57344\n\t" \
  "v_pk_mul_f32 v[148:149], v[148:149], v[66:67]\n\t" \
  "v_pk_mul_f32 v[150:151], v[150:151], v[66:67]\n\t" \
  "s_mov_b32 s53, 0x100000\n\t" \
  "buffer_store_dwordx4 v[148:151], v13, s[16:19], s53 offen offset:64 sc1\n\t" \
  "v_mfma_f32_16x16x128_f8f6f4 v[164:167], v[104:111], a[168:175], v[164:167]\n\t" \
  "ds_read_b64_tr_b8 v[122:123], v82 offset:58368\n\t" \
  "v_mfma_f32_16x16x128_f8f6f4 v[168:171], v[104:111], a[176:183], v[168:171]\n\t" \
  "ds_read_b64_tr_b8 v[124:125], v98 offset:24576\n\t" \
  "v_mfma_f32_16x16x128_f8f6f4 v[172:175], v[104:111], a[184:191], v[172:175]\n\t" \
  "ds_read_b64_tr_b8 v[126:127], v98 offset:25600\n\t" \
  "s_waitcnt lgkmcnt(4)\n\t" \
  "v_mfma_f32_16x16x128_f8f6f4 v[172:175], v[112:119], a[216:223], v[172:175]\n\t" \
  "ds_read_b64_tr_b8 v[104:105], v83 offset:24576\n\t" \
  "v_pk_mul_f32 v[152:153], v[152:153], v[66:67]\n\t" \
  "v_pk_mul_f32 v[154:155], v[154:155], v[66:67]\n\t" \
  "s_mov_b32 s54, 0x200000\n\t" \
  "buffer_store_dwordx4 v[152:155], v13, s[16:19], s54 offen offset:64 sc1\n\t" \
  "v_mfma_f32_16x16x128_f8f6f4 v[168:171], v[112:119], a[208:215], v[168:171]\n\t" \
  "ds_read_b64_tr_b8 v[106:107], v83 offset:25600\n\t" \
  "v_mfma_f32_16x16x128_f8f6f4 v[164:167], v[112:119], a[200:207], v[164:167]\n\t" \
  "ds_read_b64_tr_b8 v[108:109], v91 offset:57344\n\t" \
  "v_mfma_f32_16x16x128_f8f6f4 v[160:163], v[112:119], a[192:199], v[160:163]\n\t" \
  "ds_read_b64_tr_b8 v[110:111], v91 offset:58368\n\t" \
  "s_waitcnt lgkmcnt(4)\n\t" \
  "v_mfma_f32_16x16x128_f8f6f4 v[160:163], v[120:127], a[224:231], v[160:163]\n\t" \
  "ds_read_b64_tr_b8 v[112:113], v83 offset:32768\n\t" \
  "v_pk_mul_f32 v[156:157], v[156:157], v[66:67]\n\t" \
  "v_pk_mul_f32 v[158:159], v[158:159], v[66:67]\n\t" \
  "s_mov_b32 s55, 0x300000\n\t" \
  "buffer_store_dwordx4 v[156:159], v13, s[16:19], s55 offen offset:64 sc1\n\t" \
  "v_mfma_f32_16x16x128_f8f6f4 v[164:167], v[120:127], a[232:239], v[164:167]\n\t" \
  "ds_read_b64_tr_b8 v[114:115], v83 offset:33792\n\t" \
  "v_mfma_f32_16x16x128_f8f6f4 v[168:171], v[120:127], a[240:247], v[168:171]\n\t" \
  "ds_read_b64_tr_b8 v[116:117], v99 offset:0\n\t" \
  "v_mfma_f32_16x16x128_f8f6f4 v[172:175], v[120:127], a[248:255], v[172:175]\n\t" \
  "ds_read_b64_tr_b8 v[118:119], v99 offset:1024\n\t" \
  "s_waitcnt lgkmcnt(4)\n\t" \
  "v_mfma_f32_16x16x128_f8f6f4 v[188:191], v[104:111], a[120:127], v[188:191]\n\t" \
  "ds_read_b64_tr_b8 v[120:121], v83 offset:40960\n\t" \
  "v_mfma_f32_16x16x128_f8f6f4 v[184:187], v[104:111], a[112:119], v[184:187]\n\t" \
  "ds_read_b64_tr_b8 v[122:123], v83 offset:41984\n\t" \
  "v_mfma_f32_16x16x128_f8f6f4 v[180:183], v[104:111], a[104:111], v[180:183]\n\t" \
  "ds_read_b64_tr_b8 v[124:125], v99 offset:8192\n\t" \
  "v_mfma_f32_16x16x128_f8f6f4 v[176:179], v[104:111], a[96:103], v[176:179]\n\t" \
  "ds_read_b64_tr_b8 v[126:127], v99 offset:9216\n\t" \
  "s_waitcnt lgkmcnt(4)\n\t" \
  "v_mfma_f32_16x16x128_f8f6f4 v[176:179], v[112:119], a[128:135], v[176:179]\n\t" \
  "ds_read_b64_tr_b8 v[104:105], v83 offset:49152\n\t" \
  "v_pk_mul_f32 v[160:161], v[160:161], v[66:67]\n\t" \
  "v_pk_mul_f32 v[162:163], v[162:163], v[66:67]\n\t" \
  "s_mov_b32 s48, 0x0\n\t" \
  "buffer_store_dwordx4 v[160:163], v13, s[16:19], s48 offen offset:128 sc1\n\t" \
  "v_mfma_f32_16x16x128_f8f6f4 v[180:183], v[112:119], a[136:143], v[180:183]\n\t" \
  "ds_read_b64_tr_b8 v[106:107], v83 offset:50176\n\t" \
  "v_mfma_f32_16x16x128_f8f6f4 v[184:187], v[112:119], a[144:151], v[184:187]\n\t" \
  "ds_read_b64_tr_b8 v[108:109], v99 offset:16384\n\t" \
  "v_mfma_f32_16x16x128_f8f6f4 v[188:191], v[112:119], a[152:159], v[188:191]\n\t" \
  "ds_read_b64_tr_b8 v[110:111], v99 offset:17408\n\t" \
  "s_waitcnt lgkmcnt(4)\n\t" \
  "v_mfma_f32_16x16x128_f8f6f4 v[188:191], v[120:127], a[184:191], v[188:191]\n\t" \
  "ds_read_b64_tr_b8 v[112:113], v83 offset:57344\n\t" \
  "v_pk_mul_f32 v[164:165], v[164:165], v[66:67]\n\t" \
  "v_pk_mul_f32 v[166:167], v[166:167], v[66:67]\n\t" \
  "s_mov_b32 s49, 0x100000\n\t" \
  "buffer_store_dwordx4 v[164:167], v13, s[16:19], s49 offen offset:128 sc1\n\t" \
  "v_mfma_f32_16x16x128_f8f6f4 v[184:187], v[120:127], a[176:183], v[184:187]\n\t" \
  "ds_read_b64_tr_b8 v[114:115], v83 offset:58368\n\t" \
  "v_mfma_f32_16x16x128_f8f6f4 v[180:183], v[120:127], a[168:175], v[180:183]\n\t" \
  "ds_read_b64_tr_b8 v[116:117], v99 offset:24576\n\t" \
  "v_mfma_f32_16x16x128_f8f6f4 v[176:179], v[120:127], a[160:167], v[176:179]\n\t" \
  "ds_read_b64_tr_b8 v[118:119], v99 offset:25600\n\t" \
  "s_waitcnt lgkmcnt(4)\n\t" \
  "v_mfma_f32_16x16x128_f8f6f4 v[176:179], v[104:111], a[192:199], v[176:179]\n\t" \
  "ds_read_b64_tr_b8 v[120:121], v84 offset:24576\n\t" \
  "v_pk_mul_f32 v[168:169], v[168:169], v[66:67]\n\t" \
  "v_pk_mul_f32 v[170:171], v[170:171], v[66:67]\n\t" \
  "s_mov_b32 s50, 0x200000\n\t" \
  "buffer_store_dwordx4 v[168:171], v13, s[16:19], s50 offen offset:128 sc1\n\t" \
  "v_mfma_f32_16x16x128_f8f6f4 v[180:183], v[104:111], a[200:207], v[180:183]\n\t" \
  "ds_read_b64_tr_b8 v[122:123], v84 offset:25600\n\t" \
  "v_mfma_f32_16x16x128_f8f6f4 v[184:187], v[104:111], a[208:215], v[184:187]\n\t" \
  "ds_read_b64_tr_b8 v[124:125], v92 offset:57344\n\t" \
  "v_mfma_f32_16x16x128_f8f6f4 v[188:191], v[104:111], a[216:223], v[188:191]\n\t" \
  "ds_read_b64_tr_b8 v[126:127], v92 offset:58368\n\t" \
  "s_waitcnt lgkmcnt(4)\n\t" \
  "v_mfma_f32_16x16x128_f8f6f4 v[188:191], v[112:119], a[248:255], v[188:191]\n\t" \
  "ds_read_b64_tr_b8 v[104:105], v84 offset:32768\n\t" \
  "v_pk_mul_f32 v[172:173], v[172:173], v[66:67]\n\t" \
  "v_pk_mul_f32 v[174:175], v[174:175], v[66:67]\n\t" \
  "s_mov_b32 s51, 0x300000\n\t" \
  "buffer_store_dwordx4 v[172:175], v13, s[16:19], s51 offen offset:128 sc1\n\t" \
  "v_mfma_f32_16x16x128_f8f6f4 v[184:187], v[112:119], a[240:247], v[184:187]\n\t" \
  "ds_read_b64_tr_b8 v[106:107], v84 offset:33792\n\t" \
  "v_mfma_f32_16x16x128_f8f6f4 v[180:183], v[112:119], a[232:239], v[180:183]\n\t" \
  "ds_read_b64_tr_b8 v[108:109], v100 offset:0\n\t" \
  "v_mfma_f32_16x16x128_f8f6f4 v[176:179], v[112:119], a[224:231], v[176:179]\n\t" \
  "ds_read_b64_tr_b8 v[110:111], v100 offset:1024\n\t" \
  "s_waitcnt lgkmcnt(4)\n\t" \
  "v_mfma_f32_16x16x128_f8f6f4 v[192:195], v[120:127], a[96:103], v[192:195]\n\t" \
  "ds_read_b64_tr_b8 v[112:113], v84 offset:40960\n\t" \
  "v_mfma_f32_16x16x128_f8f6f4 v[196:199], v[120:127], a[104:111], v[196:199]\n\t" \
  "ds_read_b64_tr_b8 v[114:115], v84 offset:41984\n\t" \
  "v_mfma_f32_16x16x128_f8f6f4 v[200:203], v[120:127], a[112:119], v[200:203]\n\t" \
  "ds_read_b64_tr_b8 v[116:117], v100 offset:8192\n\t" \
  "v_mfma_f32_16x16x128_f8f6f4 v[204:207], v[120:127], a[120:127], v[204:207]\n\t" \
  "ds_read_b64_tr_b8 v[118:119], v100 offset:9216\n\t" \
  "s_waitcnt lgkmcnt(4)\n\t" \
  "v_mfma_f32_16x16x128_f8f6f4 v[204:207], v[104:111], a[152:159], v[204:207]\n\t" \
  "ds_read_b64_tr_b8 v[120:121], v84 offset:49152\n\t" \
  "v_pk_mul_f32 v[176:177], v[176:177], v[66:67]\n\t" \
  "v_pk_mul_f32 v[178:179], v[178:179], v[66:67]\n\t" \
  "s_mov_b32 s52, 0x0\n\t" \
  "buffer_store_dwordx4 v[176:179], v13, s[16:19], s52 offen offset:192 sc1\n\t" \
  "v_mfma_f32_16x16x128_f8f6f4 v[200:203], v[104:111], a[144:151], v[200:203]\n\t" \
  "ds_read_b64_tr_b8 v[122:123], v84 offset:50176\n\t" \
  "v_mfma_f32_16x16x128_f8f6f4 v[196:199], v[104:111], a[136:143], v[196:199]\n\t" \
  "ds_read_b64_tr_b8 v[124:125], v100 offset:16384\n\t" \
  "v_mfma_f32_16x16x128_f8f6f4 v[192:195], v[104:111], a[128:135], v[192:195]\n\t" \
  "ds_read_b64_tr_b8 v[126:127], v100 offset:17408\n\t" \
  "s_waitcnt lgkmcnt(4)\n\t" \
  "v_mfma_f32_16x16x128_f8f6f4 v[192:195], v[112:119], a[160:167], v[192:195]\n\t" \
  "ds_read_b64_tr_b8 v[104:105], v84 offset:57344\n\t" \
  "v_pk_mul_f32 v[180:181], v[180:181], v[66:67]\n\t" \
  "v_pk_mul_f32 v[182:183], v[182:183], v[66:67]\n\t" \
  "s_mov_b32 s53, 0x100000\n\t" \
  "buffer_store_dwordx4 v[180:183], v13, s[16:19], s53 offen offset:192 sc1\n\t" \
  "v_mfma_f32_16x16x128_f8f6f4 v[196:199], v[112:119], a[168:175], v[196:199]\n\t" \
  "ds_read_b64_tr_b8 v[106:107], v84 offset:58368\n\t" \
  "v_mfma_f32_16x16x128_f8f6f4 v[200:203], v[112:119], a[176:183], v[200:203]\n\t" \
  "ds_read_b64_tr_b8 v[108:109], v100 offset:24576\n\t" \
  "v_mfma_f32_16x16x128_f8f6f4 v[204:207], v[112:119], a[184:191], v[204:207]\n\t" \
  "ds_read_b64_tr_b8 v[110:111], v100 offset:25600\n\t" \
  "s_waitcnt lgkmcnt(4)\n\t" \
  "v_mfma_f32_16x16x128_f8f6f4 v[204:207], v[120:127], a[216:223], v[204:207]\n\t" \
  "ds_read_b64_tr_b8 v[112:113], v85 offset:24576\n\t" \
  "v_pk_mul_f32 v[184:185], v[184:185], v[66:67]\n\t" \
  "v_pk_mul_f32 v[186:187], v[186:187], v[66:67]\n\t" \
  "s_mov_b32 s54, 0x200000\n\t" \
  "buffer_store_dwordx4 v[184:187], v13, s[16:19], s54 offen offset:192 sc1\n\t" \
  "v_mfma_f32_16x16x128_f8f6f4 v[200:203], v[120:127], a[208:215], v[200:203]\n\t" \
  "ds_read_b64_tr_b8 v[114:115], v85 offset:25600\n\t" \
  "v_mfma_f32_16x16x128_f8f6f4 v[196:199], v[120:127], a[200:207], v[196:199]\n\t" \
  "ds_read_b64_tr_b8 v[116:117], v93 offset:57344\n\t" \
  "v_mfma_f32_16x16x128_f8f6f4 v[192:195], v[120:127], a[192:199], v[192:195]\n\t" \
  "ds_read_b64_tr_b8 v[118:119], v93 offset:58368\n\t" \
  "s_waitcnt lgkmcnt(4)\n\t" \
  "v_mfma_f32_16x16x128_f8f6f4 v[192:195], v[104:111], a[224:231], v[192:195]\n\t" \
  "ds_read_b64_tr_b8 v[120:121], v85 offset:32768\n\t" \
  "v_pk_mul_f32 v[188:189], v[188:189], v[66:67]\n\t" \
  "v_pk_mul_f32 v[190:191], v[190:191], v[66:67]\n\t" \
  "s_mov_b32 s55, 0x300000\n\t" \
  "buffer_store_dwordx4 v[188:191], v13, s[16:19], s55 offen offset:192 sc1\n\t" \
  "v_mfma_f32_16x16x128_f8f6f4 v[196:199], v[104:111], a[232:239], v[196:199]\n\t" \
  "ds_read_b64_tr_b8 v[122:123], v85 offset:33792\n\t" \
  "v_mfma_f32_16x16x128_f8f6f4 v[200:203], v[104:111], a[240:247], v[200:203]\n\t" \
  "ds_read_b64_tr_b8 v[124:125], v101 offset:0\n\t" \
  "v_mfma_f32_16x16x128_f8f6f4 v[204:207], v[104:111], a[248:255], v[204:207]\n\t" \
  "ds_read_b64_tr_b8 v[126:127], v101 offset:1024\n\t" \
  "s_waitcnt lgkmcnt(4)\n\t" \
  "v_mfma_f32_16x16x128_f8f6f4 v[220:223], v[112:119], a[120:127], v[220:223]\n\t" \
  "ds_read_b64_tr_b8 v[104:105], v85 offset:40960\n\t" \
  "v_mfma_f32_16x16x128_f8f6f4 v[216:219], v[112:119], a[112:119], v[216:219]\n\t" \
  "ds_read_b64_tr_b8 v[106:107], v85 offset:41984\n\t" \
  "v_mfma_f32_16x16x128_f8f6f4 v[212:215], v[112:119], a[104:111], v[212:215]\n\t" \
  "ds_read_b64_tr_b8 v[108:109], v101 offset:8192\n\t" \
  "v_mfma_f32_16x16x128_f8f6f4 v[208:211], v[112:119], a[96:103], v[208:211]\n\t" \
  "ds_read_b64_tr_b8 v[110:111], v101 offset:9216\n\t" \
  "s_waitcnt lgkmcnt(4)\n\t" \
  "v_mfma_f32_16x16x128_f8f6f4 v[208:211], v[120:127], a[128:135], v[208:211]\n\t" \
  "ds_read_b64_tr_b8 v[112:113], v85 offset:49152\n\t" \
  "v_pk_mul_f32 v[192:193], v[192:193], v[66:67]\n\t" \
  "v_pk_mul_f32 v[194:195], v[194:195], v[66:67]\n\t" \
  "s_mov_b32 s48, 0x0\n\t" \
  "buffer_store_dwordx4 v[192:195], v13, s[16:19], s48 offen offset:256 sc1\n\t" \
  "v_mfma_f32_16x16x128_f8f6f4 v[212:215], v[120:127], a[136:143], v[212:215]\n\t" \
  "ds_read_b64_tr_b8 v[114:115], v85 offset:50176\n\t" \
  "v_mfma_f32_16x16x128_f8f6f4 v[216:219], v[120:127], a[144:151], v[216:219]\n\t" \
  "ds_read_b64_tr_b8 v[116:117], v101 offset:16384\n\t" \
  "v_mfma_f32_16x16x128_f8f6f4 v[220:223], v[120:127], a[152:159], v[220:223]\n\t" \
  "ds_read_b64_tr_b8 v[118:119], v101 offset:17408\n\t" \
  "s_waitcnt lgkmcnt(4)\n\t" \
  "v_mfma_f32_16x16x128_f8f6f4 v[220:223], v[104:111], a[184:191], v[220:223]\n\t" \
  "ds_read_b64_tr_b8 v[120:121], v85 offset:57344\n\t" \
  "v_pk_mul_f32 v[196:197], v[196:197], v[66:67]\n\t" \
  "v_pk_mul_f32 v[198:199], v[198:199], v[66:67]\n\t" \
  "s_mov_b32 s49, 0x100000\n\t" \
  "buffer_store_dwordx4 v[196:199], v13, s[16:19], s49 offen offset:256 sc1\n\t" \
  "v_mfma_f32_16x16x128_f8f6f4 v[216:219], v[104:111], a[176:183], v[216:219]\n\t" \
  "ds_read_b64_tr_b8 v[122:123], v85 offset:58368\n\t" \
  "v_mfma_f32_16x16x128_f8f6f4 v[212:215], v[104:111], a[168:175], v[212:215]\n\t" \
  "ds_read_b64_tr_b8 v[124:125], v101 offset:24576\n\t" \
  "v_mfma_f32_16x16x128_f8f6f4 v[208:211], v[104:111], a[160:167], v[208:211]\n\t" \
  "ds_read_b64_tr_b8 v[126:127], v101 offset:25600\n\t" \
  "s_waitcnt lgkmcnt(4)\n\t" \
  "v_mfma_f32_16x16x128_f8f6f4 v[208:211], v[112:119], a[192:199], v[208:211]\n\t" \
  "ds_read_b64_tr_b8 v[104:105], v86 offset:24576\n\t" \
  "v_pk_mul_f32 v[200:201], v[200:201], v[66:67]\n\t" \
  "v_pk_mul_f32 v[202:203], v[202:203], v[66:67]\n\t" \
  "s_mov_b32 s50, 0x200000\n\t" \
  "buffer_store_dwordx4 v[200:203], v13, s[16:19], s50 offen offset:256 sc1\n\t" \
  "v_mfma_f32_16x16x128_f8f6f4 v[212:215], v[112:119], a[200:207], v[212:215]\n\t" \
  "ds_read_b64_tr_b8 v[106:107], v86 offset:25600\n\t" \
  "v_mfma_f32_16x16x128_f8f6f4 v[216:219], v[112:119], a[208:215], v[216:219]\n\t" \
  "ds_read_b64_tr_b8 v[108:109], v94 offset:57344\n\t" \
  "v_mfma_f32_16x16x128_f8f6f4 v[220:223], v[112:119], a[216:223], v[220:223]\n\t" \
  "ds_read_b64_tr_b8 v[110:111], v94 offset:58368\n\t" \
  "s_waitcnt lgkmcnt(4)\n\t" \
  "v_mfma_f32_16x16x128_f8f6f4 v[220:223], v[120:127], a[248:255], v[220:223]\n\t" \
  "ds_read_b64_tr_b8 v[112:113], v86 offset:32768\n\t" \
  "v_pk_mul_f32 v[204:205], v[204:205], v[66:67]\n\t" \
  "v_pk_mul_f32 v[206:207], v[206:207], v[66:67]\n\t" \
  "s_mov_b32 s51, 0x300000\n\t" \
  "buffer_store_dwordx4 v[204:207], v13, s[16:19], s51 offen offset:256 sc1\n\t" \
  "v_mfma_f32_16x16x128_f8f6f4 v[216:219], v[120:127], a[240:247], v[216:219]\n\t" \
  "ds_read_b64_tr_b8 v[114:115], v86 offset:33792\n\t" \
  "v_mfma_f32_16x16x128_f8f6f4 v[212:215], v[120:127], a[232:239], v[212:215]\n\t" \
  "ds_read_b64_tr_b8 v[116:117], v102 offset:0\n\t" \
  "v_mfma_f32_16x16x128_f8f6f4 v[208:211], v[120:127], a[224:231], v[208:211]\n\t" \
  "ds_read_b64_tr_b8 v[118:119], v102 offset:1024\n\t" \
  "s_waitcnt lgkmcnt(4)\n\t" \
  "v_mfma_f32_16x16x128_f8f6f4 v[224:227], v[104:111], a[96:103], v[224:227]\n\t" \
  "ds_read_b64_tr_b8 v[120:121], v86 offset:40960\n\t" \
  "v_mfma_f32_16x16x128_f8f6f4 v[228:231], v[104:111], a[104:111], v[228:231]\n\t" \
  "ds_read_b64_tr_b8 v[122:123], v86 offset:41984\n\t" \
  "v_mfma_f32_16x16x128_f8f6f4 v[232:235], v[104:111], a[112:119], v[232:235]\n\t" \
  "ds_read_b64_tr_b8 v[124:125], v102 offset:8192\n\t" \
  "v_mfma_f32_16x16x128_f8f6f4 v[236:239], v[104:111], a[120:127], v[236:239]\n\t" \
  "ds_read_b64_tr_b8 v[126:127], v102 offset:9216\n\t" \
  "s_waitcnt lgkmcnt(4)\n\t" \
  "v_mfma_f32_16x16x128_f8f6f4 v[236:239], v[112:119], a[152:159], v[236:239]\n\t" \
  "ds_read_b64_tr_b8 v[104:105], v86 offset:49152\n\t" \
  "v_pk_mul_f32 v[208:209], v[208:209], v[66:67]\n\t" \
  "v_pk_mul_f32 v[210:211], v[210:211], v[66:67]\n\t" \
  "s_mov_b32 s52, 0x0\n\t" \
  "buffer_store_dwordx4 v[208:211], v13, s[16:19], s52 offen offset:320 sc1\n\t" \
  "v_mfma_f32_16x16x128_f8f6f4 v[232:235], v[112:119], a[144:151], v[232:235]\n\t" \
  "ds_read_b64_tr_b8 v[106:107], v86 offset:50176\n\t" \
  "v_mfma_f32_16x16x128_f8f6f4 v[228:231], v[112:119], a[136:143], v[228:231]\n\t" \
  "ds_read_b64_tr_b8 v[108:109], v102 offset:16384\n\t" \
  "v_mfma_f32_16x16x128_f8f6f4 v[224:227], v[112:119], a[128:135], v[224:227]\n\t" \
  "ds_read_b64_tr_b8 v[110:111], v102 offset:17408\n\t" \
  "s_waitcnt lgkmcnt(4)\n\t" \
  "v_mfma_f32_16x16x128_f8f6f4 v[224:227], v[120:127], a[160:167], v[224:227]\n\t" \
  "ds_read_b64_tr_b8 v[112:113], v86 offset:57344\n\t" \
  "v_pk_mul_f32 v[212:213], v[212:213], v[66:67]\n\t" \
  "v_pk_mul_f32 v[214:215], v[214:215], v[66:67]\n\t" \
  "s_mov_b32 s53, 0x100000\n\t" \
  "buffer_store_dwordx4 v[212:215], v13, s[16:19], s53 offen offset:320 sc1\n\t" \
  "v_mfma_f32_16x16x128_f8f6f4 v[228:231], v[120:127], a[168:175], v[228:231]\n\t" \
  "ds_read_b64_tr_b8 v[114:115], v86 offset:58368\n\t" \
  "v_mfma_f32_16x16x128_f8f6f4 v[232:235], v[120:127], a[176:183], v[232:235]\n\t" \
  "ds_read_b64_tr_b8 v[116:117], v102 offset:24576\n\t" \
  "v_mfma_f32_16x16x128_f8f6f4 v[236:239], v[120:127], a[184:191], v[236:239]\n\t" \
  "ds_read_b64_tr_b8 v[118:119], v102 offset:25600\n\t" \
  "s_waitcnt lgkmcnt(4)\n\t" \
  "v_mfma_f32_16x16x128_f8f6f4 v[236:239], v[104:111], a[216:223], v[236:239]\n\t" \
  "ds_read_b64_tr_b8 v[120:121], v87 offset:24576\n\t" \
  "v_pk_mul_f32 v[216:217], v[216:217], v[66:67]\n\t" \
  "v_pk_mul_f32 v[218:219], v[218:219], v[66:67]\n\t" \
  "s_mov_b32 s54, 0x200000\n\t" \
  "buffer_store_dwordx4 v[216:219], v13, s[16:19], s54 offen offset:320 sc1\n\t" \
  "v_mfma_f32_16x16x128_f8f6f4 v[232:235], v[104:111], a[208:215], v[232:235]\n\t" \
  "ds_read_b64_tr_b8 v[122:123], v87 offset:25600\n\t" \
  "v_mfma_f32_16x16x128_f8f6f4 v[228:231], v[104:111], a[200:207], v[228:231]\n\t" \
  "ds_read_b64_tr_b8 v[124:125], v95 offset:57344\n\t" \
  "v_mfma_f32_16x16x128_f8f6f4 v[224:227], v[104:111], a[192:199], v[224:227]\n\t" \
  "ds_read_b64_tr_b8 v[126:127], v95 offset:58368\n\t" \
  "s_waitcnt lgkmcnt(4)\n\t" \
  "v_mfma_f32_16x16x128_f8f6f4 v[224:227], v[112:119], a[224:231], v[224:227]\n\t" \
  "ds_read_b64_tr_b8 v[104:105], v87 offset:32768\n\t" \
  "v_pk_mul_f32 v[220:221], v[220:221], v[66:67]\n\t" \
  "v_pk_mul_f32 v[222:223], v[222:223], v[66:67]\n\t" \
  "s_mov_b32 s55, 0x300000\n\t" \
  "buffer_store_dwordx4 v[220:223], v13, s[16:19], s55 offen offset:320 sc1\n\t" \
  "v_mfma_f32_16x16x128_f8f6f4 v[228:231], v[112:119], a[232:239], v[228:231]\n\t" \
  "ds_read_b64_tr_b8 v[106:107], v87 offset:33792\n\t" \
  "v_mfma_f32_16x16x128_f8f6f4 v[232:235], v[112:119], a[240:247], v[232:235]\n\t" \
  "ds_read_b64_tr_b8 v[108:109], v103 offset:0\n\t" \
  "v_mfma_f32_16x16x128_f8f6f4 v[236:239], v[112:119], a[248:255], v[236:239]\n\t" \
  "ds_read_b64_tr_b8 v[110:111], v103 offset:1024\n\t" \
  "s_waitcnt lgkmcnt(4)\n\t" \
  "v_mfma_f32_16x16x128_f8f6f4 v[252:255], v[120:127], a[120:127], v[252:255]\n\t" \
  "ds_read_b64_tr_b8 v[112:113], v87 offset:40960\n\t" \
  "v_mfma_f32_16x16x128_f8f6f4 v[248:251], v[120:127], a[112:119], v[248:251]\n\t" \
  "ds_read_b64_tr_b8 v[114:115], v87 offset:41984\n\t" \
  "v_mfma_f32_16x16x128_f8f6f4 v[244:247], v[120:127], a[104:111], v[244:247]\n\t" \
  "ds_read_b64_tr_b8 v[116:117], v103 offset:8192\n\t" \
  "v_mfma_f32_16x16x128_f8f6f4 v[240:243], v[120:127], a[96:103], v[240:243]\n\t" \
  "ds_read_b64_tr_b8 v[118:119], v103 offset:9216\n\t" \
  "s_waitcnt lgkmcnt(4)\n\t" \
  "v_mfma_f32_16x16x128_f8f6f4 v[240:243], v[104:111], a[128:135], v[240:243]\n\t" \
  "ds_read_b64_tr_b8 v[120:121], v87 offset:49152\n\t" \
  "v_pk_mul_f32 v[224:225], v[224:225], v[66:67]\n\t" \
  "v_pk_mul_f32 v[226:227], v[226:227], v[66:67]\n\t" \
  "s_mov_b32 s48, 0x0\n\t" \
  "buffer_store_dwordx4 v[224:227], v13, s[16:19], s48 offen offset:384 sc1\n\t" \
  "v_mfma_f32_16x16x128_f8f6f4 v[244:247], v[104:111], a[136:143], v[244:247]\n\t" \
  "ds_read_b64_tr_b8 v[122:123], v87 offset:50176\n\t" \
  "v_mfma_f32_16x16x128_f8f6f4 v[248:251], v[104:111], a[144:151], v[248:251]\n\t" \
  "ds_read_b64_tr_b8 v[124:125], v103 offset:16384\n\t" \
  "v_mfma_f32_16x16x128_f8f6f4 v[252:255], v[104:111], a[152:159], v[252:255]\n\t" \
  "ds_read_b64_tr_b8 v[126:127], v103 offset:17408\n\t" \
  "s_waitcnt lgkmcnt(4)\n\t" \
  "v_mfma_f32_16x16x128_f8f6f4 v[252:255], v[112:119], a[184:191], v[252:255]\n\t" \
  "ds_read_b64_tr_b8 v[104:105], v87 offset:57344\n\t" \
  "v_pk_mul_f32 v[228:229], v[228:229], v[66:67]\n\t" \
  "v_pk_mul_f32 v[230:231], v[230:231], v[66:67]\n\t" \
  "s_mov_b32 s49, 0x100000\n\t" \
  "buffer_store_dwordx4 v[228:231], v13, s[16:19], s49 offen offset:384 sc1\n\t" \
  "v_mfma_f32_16x16x128_f8f6f4 v[248:251], v[112:119], a[176:183], v[248:251]\n\t" \
  "ds_read_b64_tr_b8 v[106:107], v87 offset:58368\n\t" \
  "v_mfma_f32_16x16x128_f8f6f4 v[244:247], v[112:119], a[168:175], v[244:247]\n\t" \
  "ds_read_b64_tr_b8 v[108:109], v103 offset:24576\n\t" \
  "v_mfma_f32_16x16x128_f8f6f4 v[240:243], v[112:119], a[160:167], v[240:243]\n\t" \
  "ds_read_b64_tr_b8 v[110:111], v103 offset:25600\n\t" \
  "s_waitcnt lgkmcnt(4)\n\t" \
  "v_mfma_f32_16x16x128_f8f6f4 v[240:243], v[120:127], a[192:199], v[240:243]\n\t" \
  "v_pk_mul_f32 v[232:233], v[232:233], v[66:67]\n\t" \
  "v_pk_mul_f32 v[234:235], v[234:235], v[66:67]\n\t" \
  "s_mov_b32 s50, 0x200000\n\t" \
  "buffer_store_dwordx4 v[232:235], v13, s[16:19], s50 offen offset:384 sc1\n\t" \
  "v_mfma_f32_16x16x128_f8f6f4 v[244:247], v[120:127], a[200:207], v[244:247]\n\t" \
  "v_mfma_f32_16x16x128_f8f6f4 v[248:251], v[120:127], a[208:215], v[248:251]\n\t" \
  "v_mfma_f32_16x16x128_f8f6f4 v[252:255], v[120:127], a[216:223], v[252:255]\n\t" \
  "s_waitcnt lgkmcnt(0)\n\t" \
  "v_mfma_f32_16x16x128_f8f6f4 v[252:255], v[104:111], a[248:255], v[252:255]\n\t" \
  "v_pk_mul_f32 v[236:237], v[236:237], v[66:67]\n\t" \
  "v_pk_mul_f32 v[238:239], v[238:239], v[66:67]\n\t" \
  "s_mov_b32 s51, 0x300000\n\t" \
  "buffer_store_dwordx4 v[236:239], v13, s[16:19], s51 offen offset:384 sc1\n\t" \
  "v_mfma_f32_16x16x128_f8f6f4 v[248:251], v[104:111], a[240:247], v[248:251]\n\t" \
  "v_mfma_f32_16x16x128_f8f6f4 v[244:247], v[104:111], a[232:239], v[244:247]\n\t" \
  "v_mfma_f32_16x16x128_f8f6f4 v[240:243], v[104:111], a[224:231], v[240:243]\n\t" \
  "s_nop 15\n\t" \
  "s_nop 7\n\t" \
  "v_pk_mul_f32 v[240:241], v[240:241], v[66:67]\n\t" \
  "v_pk_mul_f32 v[242:243], v[242:243], v[66:67]\n\t" \
  "s_mov_b32 s52, 0x0\n\t" \
  "buffer_store_dwordx4 v[240:243], v13, s[16:19], s52 offen offset:448 sc1\n\t" \
  "v_pk_mul_f32 v[244:245], v[244:245], v[66:67]\n\t" \
  "v_pk_mul_f32 v[246:247], v[246:247], v[66:67]\n\t" \
  "s_mov_b32 s53, 0x100000\n\t" \
  "buffer_store_dwordx4 v[244:247], v13, s[16:19], s53 offen offset:448 sc1\n\t" \
  "v_pk_mul_f32 v[248:249], v[248:249], v[66:67]\n\t" \
  "v_pk_mul_f32 v[250:251], v[250:251], v[66:67]\n\t" \
  "s_mov_b32 s54, 0x200000\n\t" \
  "buffer_store_dwordx4 v[248:251], v13, s[16:19], s54 offen offset:448 sc1\n\t" \
  "v_pk_mul_f32 v[252:253], v[252:253], v[66:67]\n\t" \
  "v_pk_mul_f32 v[254:255], v[254:255], v[66:67]\n\t" \
  "s_mov_b32 s55, 0x300000\n\t" \
  "buffer_store_dwordx4 v[252:255], v13, s[16:19], s55 offen offset:448 sc1\n\t" \
  "s_endpgm\n\t" \
  "Lpoll_3_%=:\n\t" \
  "s_mov_b32 exec_lo, s69\n\t" \
  "s_mov_b32 exec_hi, 1\n\t" \
  "global_load_dwordx4 v[68:71], v10, s[6:7] sc0 sc1\n\t" \
  "s_mov_b64 exec, -1\n\t" \
  "s_waitcnt vmcnt(0)\n\t" \
  "v_min_u32 v68, v68, v69\n\t" \
  "v_min3_u32 v68, v68, v70, v71\n\t" \
  "v_cmp_le_u32 vcc, s57, v68\n\t" \
  "s_and_b32 s64, vcc_lo, s69\n\t" \
  "s_or_b32 s56, s56, s64\n\t" \
  "s_and_b32 s64, s56, s62\n\t" \
  "s_cmp_eq_u32 s64, s62\n\t" \
  "s_cbranch_scc1 Lret_3_%=\n\t" \
  "s_bitcmp1_b32 vcc_hi, 0\n\t" \
  "s_cbranch_scc1 Lfail_3_%=\n\t" \
  "s_memrealtime s[64:65]\n\t" \
  "s_waitcnt lgkmcnt(0)\n\t" \
  "s_sub_u32 s64, s64, s58\n\t" \
  "s_subb_u32 s65, s65, s59\n\t" \
  "s_cmp_lg_u32 s65, 0\n\t" \
  "s_cbranch_scc1 Lfail_3_%=\n\t" \
  "s_cmp_gt_u32 s64, 50000000\n\t" \
  "s_cbranch_scc1 Lfail_3_%=\n\t" \
  "s_sleep 4\n\t" \
  "s_branch Lpoll_3_%=\n\t" \
  "Lfail_3_%=:\n\t" \
  "v_mov_b32 v68, s57\n\t" \
  "v_mov_b32 v69, s57\n\t" \
  "v_mov_b32 v70, s57\n\t" \
  "v_mov_b32 v71, s57\n\t" \
  "s_mov_b32 exec_lo, 0\n\t" \
  "s_mov_b32 exec_hi, 1\n\t" \
  "global_store_dwordx4 v10, v[68:71], s[6:7] sc0 sc1\n\t" \
  "s_mov_b64 exec, -1\n\t" \
  "s_waitcnt vmcnt(0)\n\t" \
  "s_mov_b32 s63, 0x7fc00000\n\t" \
  "s_mov_b32 s56, -1\n\t" \
  "Lret_3_%=:\n\t" \
  "s_setpc_b64 s[60:61]\n\t" \
  "Lend_3_%=:\n\t" \
  "Lfin_%=:\n\t"
#define GEMM_CLOB "v1", "v2", "v3", "v4", "v5", "v6", "v7", "v8", "v9", "v10", "v11", "v12", "v13", "v14", "v15", "v16", "v17", "v18", "v19", "v20", "v21", "v22", "v23", "v62", "v63", "v64", "v65", "v66", "v67", "v68", "v69", "v70", "v71", "v72", "v73", "v74", "v75", "v76", "v77", "v78", "v79", "v80", "v81", "v82", "v83", "v84", "v85", "v86", "v87", "v88", "v89", "v90", "v91", "v92", "v93", "v94", "v95", "v96", "v97", "v98", "v99", "v100", "v101", "v102", "v103", "v104", "v105", "v106", "v107", "v108", "v109", "v110", "v111", "v112", "v113", "v114", "v115", "v116", "v117", "v118", "v119", "v120", "v121", "v122", "v123", "v124", "v125", "v126", "v127", "v128", "v129", "v130", "v131", "v132", "v133", "v134", "v135", "v136", "v137", "v138", "v139", "v140", "v141", "v142", "v143", "v144", "v145", "v146", "v147", "v148", "v149", "v150", "v151", "v152", "v153", "v154", "v155", "v156", "v157", "v158", "v159", "v160", "v161", "v162", "v163", "v164", "v165", "v166", "v167", "v168", "v169", "v170", "v171", "v172", "v173", "v174", "v175", "v176", "v177", "v178", "v179", "v180", "v181", "v182", "v183", "v184", "v185", "v186", "v187", "v188", "v189", "v190", "v191", "v192", "v193", "v194", "v195", "v196", "v197", "v198", "v199", "v200", "v201", "v202", "v203", "v204", "v205", "v206", "v207", "v208", "v209", "v210", "v211", "v212", "v213", "v214", "v215", "v216", "v217", "v218", "v219", "v220", "v221", "v222", "v223", "v224", "v225", "v226", "v227", "v228", "v229", "v230", "v231", "v232", "v233", "v234", "v235", "v236", "v237", "v238", "v239", "v240", "v241", "v242", "v243", "v244", "v245", "v246", "v247", "v248", "v249", "v250", "v251", "v252", "v253", "v254", "v255", "a0", "a1", "a2", "a3", "a4", "a5", "a6", "a7", "a8", "a9", "a10", "a11", "a12", "a13", "a14", "a15", "a16", "a17", "a18", "a19", "a20", "a21", "a22", "a23", "a24", "a25", "a26", "a27", "a28", "a29", "a30", "a31", "a32", "a33", "a34", "a35", "a36", "a37", "a38", "a39", "a40", "a41", "a42", "a43", "a44", "a45", "a46", "a47", "a48", "a49", "a50", "a51", "a52", "a53", "a54", "a55", "a56", "a57", "a58", "a59", "a60", "a61", "a62", "a63", "a64", "a65", "a66", "a67", "a68", "a69", "a70", "a71", "a72", "a73", "a74", "a75", "a76", "a77", "a78", "a79", "a80", "a81", "a82", "a83", "a84", "a85", "a86", "a87", "a88", "a89", "a90", "a91", "a92", "a93", "a94", "a95", "a96", "a97", "a98", "a99", "a100", "a101", "a102", "a103", "a104", "a105", "a106", "a107", "a108", "a109", "a110", "a111", "a112", "a113", "a114", "a115", "a116", "a117", "a118", "a119", "a120", "a121", "a122", "a123", "a124", "a125", "a126", "a127", "a128", "a129", "a130", "a131", "a132", "a133", "a134", "a135", "a136", "a137", "a138", "a139", "a140", "a141", "a142", "a143", "a144", "a145", "a146", "a147", "a148", "a149", "a150", "a151", "a152", "a153", "a154", "a155", "a156", "a157", "a158", "a159", "a160", "a161", "a162", "a163", "a164", "a165", "a166", "a167", "a168", "a169", "a170", "a171", "a172", "a173", "a174", "a175", "a176", "a177", "a178", "a179", "a180", "a181", "a182", "a183", "a184", "a185", "a186", "a187", "a188", "a189", "a190", "a191", "a192", "a193", "a194", "a195", "a196", "a197", "a198", "a199", "a200", "a201", "a202", "a203", "a204", "a205", "a206", "a207", "a208", "a209", "a210", "a211", "a212", "a213", "a214", "a215", "a216", "a217", "a218", "a219", "a220", "a221", "a222", "a223", "a224", "a225", "a226", "a227", "a228", "a229", "a230", "a231", "a232", "a233", "a234", "a235", "a236", "a237", "a238", "a239", "a240", "a241", "a242", "a243", "a244", "a245", "a246", "a247", "a248", "a249", "a250", "a251", "a252", "a253", "a254", "a255", "s4", "s5", "s6", "s7", "s8", "s9", "s10", "s11", "s12", "s13", "s14", "s15", "s16", "s17", "s18", "s19", "s20", "s21", "s22", "s23", "s24", "s25", "s26", "s27", "s28", "s29", "s30", "s31", "s34", "s35", "s36", "s37", "s38", "s39", "s40", "s41", "s42", "s43", "s44", "s45", "s46", "s47", "s48", "s49", "s50", "s51", "s52", "s53", "s54", "s55", "s56", "s57", "s58", "s59", "s60", "s61", "s62", "s63", "s64", "s65", "s66", "s67", "s68", "s69", "s70", "s71", "s72", "s73", "s74", "s75", "s76", "s77", "s78", "s79", "s80", "s81", "s82", "s83", "s84", "s85", "s86", "s87", "s88", "s89", "s90", "s91", "s92", "s93", "s94", "s95", "vcc", "scc", "memory"

#define HIP_CHECK(expr) do { hipError_t _e = (expr); if (_e != hipSuccess) { \
    fprintf(stderr, "HIP error %s at %s:%d\n", hipGetErrorString(_e), __FILE__, __LINE__); \
    return; } } while (0)
typedef unsigned u32x4 __attribute__((ext_vector_type(4)));
constexpr int N = 16384, K = 4096;
constexpr size_t LDS_BYTES = GEMM_LDS;
constexpr unsigned STAGE_BYTES = 32 * 65536;
constexpr unsigned FLAG_OFF = 2 * STAGE_BYTES;
constexpr unsigned TMO_OFF = FLAG_OFF + 8 * 512;
constexpr unsigned CNT_OFF = TMO_OFF + 128;
constexpr unsigned DBG_OFF = CNT_OFF + 256 * 128;
constexpr unsigned WS_NEED = DBG_OFF + 256 * 256;

struct Params {
  const unsigned char* x; const unsigned char* w; const float* sx; const float* sw; float* out;
  unsigned char* self;
  unsigned char* wsr[8];
  int rank; int pad;
  unsigned short duty[1024];
};
static_assert(sizeof(Params) == 0x78 + 2048, "kernarg layout is hard-coded in the asm");

__global__ __launch_bounds__(256, 1) void a2a_gemm(Params P) {
  unsigned stamps = 0;
  asm volatile(GEMM_ASM
    : [stamps] "+v"(stamps)
    : [ka] "s"(__builtin_amdgcn_kernarg_segment_ptr()), [bid] "s"(blockIdx.x), [tid] "v"(threadIdx.x)
    : GEMM_CLOB);
}

extern "C" void kernel_launch(int rank, int nranks, ncclComm_t comm, hipStream_t stream,
                              void* const* d_in, const int* in_sizes, int n_in,
                              void* d_out, int out_size, const int* in_dtypes, int out_dtype,
                              void* const* d_out_all, void* const* d_ws_all, size_t ws_bytes) {
  if (nranks != 8 || (unsigned)rank >= 8u || ws_bytes < WS_NEED || n_in != 4) { fprintf(stderr, "unexpected configuration\n"); return; }
  static bool init = false;
  if (!init) {
    HIP_CHECK(hipFuncSetAttribute((const void*)a2a_gemm, hipFuncAttributeMaxDynamicSharedMemorySize, (int)LDS_BYTES));
    init = true;
  }
  Params P;
  P.x = (const unsigned char*)d_in[0]; P.w = (const unsigned char*)d_in[1];
  P.sx = (const float*)d_in[2]; P.sw = (const float*)d_in[3]; P.out = (float*)d_out;
  P.self = (unsigned char*)d_ws_all[rank];
  for (int i = 0; i < 8; ++i) P.wsr[i] = (unsigned char*)d_ws_all[(rank + 1 + i) & 7];
  P.rank = rank; P.pad = 0;
  for (int b = 0; b < 256; ++b) for (int w = 0; w < 4; ++w) {
    const int xcd = b & 7, q = b >> 3, mh = q & 1, pr = q >> 1, cls = pr & 3;
    int d, r16, sj, sub;
    if (cls == 0 && xcd < 7) { d = xcd; r16 = (mh + 2 * (pr >> 2)) * 4 + w; sj = 0; sub = 0; }
    else if ((pr & 7) == 3) { d = 7; r16 = xcd * 4 + (pr >> 3) * 2 + mh; sj = w; sub = GEMM_SP ? 0 : 15; }
    else {
      const int wi = cls == 0 ? 160 + (pr >> 2) * 2 + mh : cls == 3 ? 128 + xcd * 4 + (pr >> 3) * 2 + mh : (cls - 1) * 64 + xcd * 8 + (pr >> 2) * 2 + mh;
      const int n = w * 168 + wi, idx = n % 224;
      sj = 1 + n / 224; d = idx % 7; r16 = idx / 7; sub = cls == 0 ? 0 : sj;
    }
    P.duty[b * 4 + w] = (unsigned short)(d | r16 << 3 | sj << 8 | sub << 10);
  }
  a2a_gemm<<<256, 256, LDS_BYTES, stream>>>(P);
  HIP_CHECK(hipGetLastError());
}
```
